```python
import numpy as np

kernel_name = "gpu_gemm_ar_m4096_k4096_n16384_e4m3e5m2_dqsilu"


ABSMAX_RTOL = 0.05

COMPUTE_DTYPE = "float32"
EPILOGUE = "dequant_silu"
IN_QUANT = None
OUT_QUANT = None
IN_FP8 = "e4m3e5m2"
IN_DTYPES = {"x": "e4m3", "w_mat": "e5m2",
             "scale_x": "float32", "scale_w": "float32"}
WORLD_SIZE = 8

def _snap_fp8(x, mant_bits, max_abs, min_normal):
    a = np.abs(x.astype(np.float64))
    _m, _e = np.frexp(a)
    q = np.where(a >= min_normal, np.ldexp(1.0, _e - 1 - mant_bits),
                 min_normal * 2.0 ** -mant_bits)
    snapped = np.minimum(np.round(a / q) * q, max_abs)
    return (np.sign(x) * snapped).astype(np.float32)

def setup_inputs():

    x = _snap_fp8(np.random.randn(4096, 4096).astype(np.float32), 3, 448.0, 2.0 ** -6)
    w_mat = _snap_fp8(np.random.randn(4096, 16384).astype(np.float32), 2, 57344.0, 2.0 ** -14)
    scale_x = (0.02 * (1.0 + np.random.rand(1))).astype(np.float32)
    scale_w = (0.02 * (1.0 + np.random.rand(1))).astype(np.float32)
    return {"x": x, "w_mat": w_mat, "scale_x": scale_x, "scale_w": scale_w}

def reference(x, w_mat, scale_x, scale_w):
    acc = (x @ w_mat).astype(np.float32)
    y = (acc * (scale_x * scale_w)).astype(np.float32)
    return (y / (1.0 + np.exp(-np.clip(y, -60.0, 60.0)))).astype(np.float32)
```

```cpp
#include <hip/hip_runtime.h>
#include <rccl/rccl.h>
#include <cstdio>

#define GEMM_CLOBBERS "v0","v1","v2","v3","v4","v5","v6","v7","v8","v9","v10","v11","v12","v13","v14","v15","v16","v17","v18","v19","v20","v21","v22","v23","v24","v25","v26","v27","v28","v29","v30","v31","v32","v33","v34","v35","v36","v37","v38","v39","v40","v41","v42","v43","v44","v45","v46","v47","v48","v49","v50","v51","v52","v53","v54","v55","v56","v57","v58","v59","v60","v61","v62","v63","v64","v65","v66","v67","v68","v69","v70","v71","v72","v73","v74","v75","v76","v77","v78","v79","v80","v81","v82","v83","v84","v85","v86","v87","v88","v89","v90","v91","v92","v93","v94","v95","v96","v97","v98","v99","v100","v101","v102","v103","v104","v105","v106","v107","v108","v109","v110","v111","v112","v113","v114","v115","v116","v117","v118","v119","v120","v121","v122","v123","v124","v125","v126","v127","v128","v129","v130","v131","v132","v133","v134","v135","v136","v137","v138","v139","v140","v141","v142","v143","v144","v145","v146","v147","v148","v149","v150","v151","v152","v153","v154","v155","v156","v157","v158","v159","v160","v161","v162","v163","v164","v165","v166","v167","v168","v169","v170","v171","v172","v173","v174","v175","v176","v177","v178","v179","v180","v181","v182","v183","v184","v185","v186","v187","v188","v189","v190","v191","v192","v193","v194","v195","v196","v197","v198","v199","v200","v201","v202","v203","v204","v205","v206","v207","v208","v209","v210","v211","v212","v213","v214","v215","v216","v217","v218","v219","v220","v221","v222","v223","v224","v225","v226","v227","v228","v229","v230","v231","v232","v233","v234","v235","v236","v237","v238","v239","v240","v241","v242","v243","v244","v245","v246","v247","v248","v249","v250","v251","v252","v253","v254","v255","a0","a1","a2","a3","a4","a5","a6","a7","a8","a9","a10","a11","a12","a13","a14","a15","a16","a17","a18","a19","a20","a21","a22","a23","a24","a25","a26","a27","a28","a29","a30","a31","a32","a33","a34","a35","a36","a37","a38","a39","a40","a41","a42","a43","a44","a45","a46","a47","a48","a49","a50","a51","a52","a53","a54","a55","a56","a57","a58","a59","a60","a61","a62","a63","a64","a65","a66","a67","a68","a69","a70","a71","a72","a73","a74","a75","a76","a77","a78","a79","a80","a81","a82","a83","a84","a85","a86","a87","a88","a89","a90","a91","a92","a93","a94","a95","a96","a97","a98","a99","a100","a101","a102","a103","a104","a105","a106","a107","a108","a109","a110","a111","a112","a113","a114","a115","a116","a117","a118","a119","a120","a121","a122","a123","a124","a125","a126","a127","a128","a129","a130","a131","a132","a133","a134","a135","a136","a137","a138","a139","a140","a141","a142","a143","a144","a145","a146","a147","a148","a149","a150","a151","a152","a153","a154","a155","a156","a157","a158","a159","a160","a161","a162","a163","a164","a165","a166","a167","a168","a169","a170","a171","a172","a173","a174","a175","a176","a177","a178","a179","a180","a181","a182","a183","a184","a185","a186","a187","a188","a189","a190","a191","a192","a193","a194","a195","a196","a197","a198","a199","a200","a201","a202","a203","a204","a205","a206","a207","a208","a209","a210","a211","a212","a213","a214","a215","a216","a217","a218","a219","a220","a221","a222","a223","a224","a225","a226","a227","a228","a229","a230","a231","a232","a233","a234","a235","a236","a237","a238","a239","a240","a241","a242","a243","a244","a245","a246","a247","a248","a249","a250","a251","a252","a253","a254","a255","s30","s31","s33","s34","s35","s36","s37","s38","s39","s40","s41","s42","s43","s44","s45","s46","s47","s48","s49","s50","s51","s52","s53","s54","s55","s56","s57","s58","s59","s60","s61","s62","s63","s64","s65","s66","s67","s68","s69","s70","s71","s72","s73","s74","s75","s76","s77","s78","s79","s80","s81","s82","s83","s84","s85","s86","s87","s88","s89","s90","s91","s92","s93","s94","s95","s96","s97","s98","s99","vcc","scc","memory"
#define COMM_CLOBBERS "v0","v1","v2","v3","v4","v5","v6","v7","v8","v9","v10","v11","v12","v13","v14","v15","v16","v17","v18","v19","v20","v21","v22","v23","v24","v25","v26","v27","v28","v29","v30","v31","v32","v33","v34","v35","v36","v37","v38","v39","v40","v41","v42","v43","v44","v45","v46","v47","v48","v49","v50","v51","v52","v53","v54","v55","v56","v57","v58","v59","v60","v61","v62","v63","v64","v65","v66","v67","v68","v69","v70","v71","v72","v73","v74","v75","v76","v77","v78","v79","v80","v81","v82","v83","v84","v85","v86","v87","v88","v89","v90","v91","v92","v93","v94","v95","v96","v97","v98","v99","v100","v101","v102","v103","v104","v105","v106","v107","v108","v109","v110","v111","v112","v113","v114","v115","v116","v117","v118","v119","v120","v121","v122","v123","v124","v125","v126","v127","v128","v129","v130","v131","v132","v133","v134","v135","v136","v137","v138","v139","v140","v141","v142","v143","v144","v145","v146","v147","v148","v149","v150","v151","v152","v153","v154","v155","v156","v157","v158","v159","v160","v161","v162","v163","v164","v165","v166","v167","v168","v169","v170","v171","v172","v173","v174","v175","v176","v177","v178","v179","v180","v181","v182","v183","v184","v185","v186","v187","v188","v189","v190","v191","v192","v193","v194","v195","v196","v197","v198","v199","v200","v201","v202","v203","v204","v205","v206","v207","v208","v209","v210","v211","v212","v213","v214","v215","v216","v217","v218","v219","v220","v221","v222","v223","v224","v225","v226","v227","s21","s22","s23","s24","s25","s26","s27","s28","s29","s30","s31","s33","s34","s35","s36","s37","s38","s39","s40","s41","s42","s43","s44","s45","s46","s47","s48","s49","s50","s51","s52","s53","s54","s55","s56","s57","s58","s59","s60","s61","s62","s63","s64","s65","s66","s67","s68","s69","s70","s71","s72","s73","s74","s75","s76","s77","s78","s79","s80","s81","s82","s83","s84","s85","s86","s87","s88","s89","s90","s91","s92","s93","s94","s95","s96","s97","s98","s99","vcc","scc","memory"
#define GEMM_ASM \
  "s_load_dwordx2 s[40:41], %[ka], 0x78\n\t" \
  "s_load_dwordx2 s[44:45], %[ka], 0x20\n\t" \
  "s_load_dwordx2 s[88:89], %[ka], 0x28\n\t" \
  "s_load_dword s70, %[tt], 0x4\n\t" \
  "s_load_dwordx2 s[72:73], %[tt], 0xc\n\t" \
  "s_load_dword s74, %[tt], 0x14\n\t" \
  "s_mov_b32 s71, %[t1]\n\t" \
  "s_mov_b32 s42, 0x5028000\n\t" \
  "s_mov_b32 s43, 0x20000\n\t" \
  "s_mov_b32 s46, 0\n\t" \
  "s_mov_b32 s47, 0x20000\n\t" \
  "s_mov_b32 s90, 0x100000\n\t" \
  "s_mov_b32 s91, 0x20000\n\t" \
  "s_mov_b32 s55, %[ntiles]\n\t" \
  "s_mov_b32 s33, %[blk]\n\t" \
  "s_and_b32 s81, %[blk], 255\n\t" \
  "s_lshl_b32 s81, s81, 7\n\t" \
  "s_add_u32 s81, s81, 0x8000\n\t" \
  "s_mov_b32 s80, 0\n\t" \
  "s_mov_b32 s57, 0\n\t" \
  "s_and_b32 s52, %[call], 1\n\t" \
  "s_mul_i32 s52, s52, 0x6800000\n\t" \
  "s_waitcnt lgkmcnt(0)\n\t" \
  "s_add_u32 s40, s40, s52\n\t" \
  "s_addc_u32 s41, s41, 0\n\t" \
  "s_mov_b32 s58, %[cneg]\n\t" \
  "s_mov_b32 s59, %[nls]\n\t" \
  "s_mov_b32 s60, %[invs]\n\t" \
  "s_mov_b32 s61, %[kis]\n\t" \
  "s_mov_b32 s62, 0x42ad1f97\n\t" \
  "s_mov_b32 s63, %[wave]\n\t" \
  "s_mov_b32 s75, %[call]\n\t" \
  "s_lshl_b32 s86, s75, 12\n\t" \
  "s_mov_b32 s69, 0\n\t" \
  "s_mov_b32 s66, 0\n\t" \
  "s_mov_b32 s68, 0\n\t" \
  "s_lshl_b32 s53, s63, 12\n\t" \
  "s_mov_b64 s[92:93], %[nt]\n\t" \
  "s_sub_u32 s92, s92, s53\n\t" \
  "s_subb_u32 s93, s93, 0\n\t" \
  "s_add_u32 s94, s53, 10240\n\t" \
  "s_mov_b32 s95, 0x20000\n\t" \
  "s_mov_b64 s[38:39], %[ka]\n\t" \
  "s_mov_b32 s36, 0\n\t" \
  "s_getpc_b64 s[78:79]\n\t" \
  "L_pc_%=:\n\t" \
  "s_add_u32 s34, s78, L_tile_%=-L_pc_%=\n\t" \
  "s_addc_u32 s35, s79, 0\n\t" \
  "s_add_u32 s78, s78, L_slow_%=-L_pc_%=\n\t" \
  "s_addc_u32 s79, s79, 0\n\t" \
  "v_mbcnt_lo_u32_b32 v240, -1, 0\n\t" \
  "v_mbcnt_hi_u32_b32 v240, -1, v240\n\t" \
  "v_lshlrev_b32 v241, 4, v240\n\t" \
  "v_add_u32 v252, s53, v241\n\t" \
  "v_lshrrev_b32 v242, 4, v240\n\t" \
  "v_and_b32 v243, 15, v240\n\t" \
  "v_lshlrev_b32 v243, 4, v243\n\t" \
  "v_lshlrev_b32 v244, 11, v242\n\t" \
  "v_add_u32 v244, v244, v243\n\t" \
  "s_lshr_b32 s52, s63, 1\n\t" \
  "s_lshl_b32 s52, s52, 10\n\t" \
  "s_add_u32 s52, s52, 32768\n\t" \
  "v_add_u32 v250, s52, v244\n\t" \
  "s_and_b32 s52, s63, 1\n\t" \
  "s_lshl_b32 s52, s52, 10\n\t" \
  "s_add_u32 s52, s52, 98304\n\t" \
  "v_add_u32 v251, s52, v244\n\t" \
  "v_lshlrev_b32 v242, 18, v242\n\t" \
  "v_add_u32 v253, v242, v243\n\t" \
  "s_and_b32 s52, s63, 1\n\t" \
  "s_lshl_b32 s52, s52, 9\n\t" \
  "v_add_u32 v253, s52, v253\n\t" \
  "s_lshr_b32 s52, s63, 1\n\t" \
  "s_lshl_b32 s52, s52, 23\n\t" \
  "v_add_u32 v253, s52, v253\n\t" \
  "v_lshlrev_b32 v0, 7, v240\n\t" \
  "s_lshl_b32 s52, s63, 13\n\t" \
  "v_add_u32 v0, s52, v0\n\t" \
  "v_min_u32 v1, L_end_%=-L_tile_%=-4, v0\n\t" \
  "global_load_dword v2, v1, s[34:35]\n\t" \
  "v_add_u32 v0, 0x8000, v0\n\t" \
  "v_min_u32 v1, L_end_%=-L_tile_%=-4, v0\n\t" \
  "global_load_dword v3, v1, s[34:35]\n\t" \
  "v_add_u32 v0, 0x8000, v0\n\t" \
  "v_min_u32 v1, L_end_%=-L_tile_%=-4, v0\n\t" \
  "global_load_dword v4, v1, s[34:35]\n\t" \
  "v_add_u32 v0, 0x8000, v0\n\t" \
  "s_and_b32 s99, s70, 15\n\t" \
  "s_mul_i32 s48, s99, 0x100800\n\t" \
  "s_lshl_b32 s56, s99, 24\n\t" \
  "s_lshr_b32 s96, s99, 2\n\t" \
  "s_lshl_b32 s96, s96, 9\n\t" \
  "s_and_b32 s99, s99, 3\n\t" \
  "s_lshl2_add_u32 s96, s99, s96\n\t" \
  "s_bfe_u32 s99, s70, 0x60004\n\t" \
  "s_mul_i32 s49, s99, 0x100800\n\t" \
  "s_add_u32 s49, s49, 0x1008000\n\t" \
  "s_lshl_b32 s99, s99, 10\n\t" \
  "s_add_u32 s56, s56, s99\n\t" \
  "s_bfe_u32 s97, s70, 0x40006\n\t" \
  "s_add_u32 s97, s97, 4\n\t" \
  "s_lshl_b32 s97, s97, 9\n\t" \
  "s_bfe_u32 s99, s70, 0x20004\n\t" \
  "s_lshl2_add_u32 s97, s99, s97\n\t" \
  "buffer_load_dword v248, v252, s[92:95], s96 offen\n\t" \
  "buffer_load_dword v249, v252, s[92:95], s97 offen\n\t" \
  "s_bfe_u32 s99, s70, 0x10011\n\t" \
  "s_lshl_b32 s52, s99, 14\n\t" \
  "s_add_u32 s49, s49, s52\n\t" \
  "s_lshl_b32 s52, s99, 8\n\t" \
  "s_add_u32 s56, s56, s52\n\t" \
  "s_bfe_u32 s99, s70, 0x10012\n\t" \
  "s_lshl_b32 s52, s99, 14\n\t" \
  "s_add_u32 s48, s48, s52\n\t" \
  "s_lshl_b32 s52, s99, 22\n\t" \
  "s_add_u32 s56, s56, s52\n\t" \
  "s_waitcnt vmcnt(0)\n\t" \
  "v_max_u32 v255, v248, v249\n\t" \
  "v_add_u32 v255, s86, v255\n\t" \
  "s_mov_b32 s67, 0\n\t" \
  "s_nop 1\n\t" \
  "v_readlane_b32 s65, v255, s67\n\t" \
  "s_nop 3\n\t" \
  "s_add_u32 s67, s67, 1\n\t" \
  "s_cmp_ge_u32 s69, s65\n\t" \
  "s_cbranch_scc0 L_w37_%=\n\t" \
  "L_ok37_%=:\n\t" \
  "s_add_i32 s52, s48, 0\n\t" \
  "s_add_u32 m0, s53, 32768\n\t" \
  "s_nop 0\n\t" \
  "buffer_load_dwordx4 v252, s[40:43], s52 offen offset:0 sc0 sc1 lds\n\t" \
  "buffer_load_dwordx4 v252, s[40:43], s52 offen offset:1024 sc0 sc1 lds\n\t" \
  "buffer_load_dwordx4 v252, s[40:43], s52 offen offset:2048 sc0 sc1 lds\n\t" \
  "buffer_load_dwordx4 v252, s[40:43], s52 offen offset:3072 sc0 sc1 lds\n\t" \
  "s_add_i32 s52, s49, 0\n\t" \
  "s_add_u32 m0, s53, 98304\n\t" \
  "s_nop 0\n\t" \
  "buffer_load_dwordx4 v252, s[40:43], s52 offen offset:0 sc0 sc1 lds\n\t" \
  "buffer_load_dwordx4 v252, s[40:43], s52 offen offset:1024 sc0 sc1 lds\n\t" \
  "buffer_load_dwordx4 v252, s[40:43], s52 offen offset:2048 sc0 sc1 lds\n\t" \
  "buffer_load_dwordx4 v252, s[40:43], s52 offen offset:3072 sc0 sc1 lds\n\t" \
  "s_add_i32 s52, s49, 16384\n\t" \
  "s_add_u32 m0, s53, 114688\n\t" \
  "s_nop 0\n\t" \
  "buffer_load_dwordx4 v252, s[40:43], s52 offen offset:0 sc0 sc1 lds\n\t" \
  "buffer_load_dwordx4 v252, s[40:43], s52 offen offset:1024 sc0 sc1 lds\n\t" \
  "buffer_load_dwordx4 v252, s[40:43], s52 offen offset:2048 sc0 sc1 lds\n\t" \
  "buffer_load_dwordx4 v252, s[40:43], s52 offen offset:3072 sc0 sc1 lds\n\t" \
  "s_add_i32 s52, s48, 16384\n\t" \
  "s_add_u32 m0, s53, 49152\n\t" \
  "s_nop 0\n\t" \
  "buffer_load_dwordx4 v252, s[40:43], s52 offen offset:0 sc0 sc1 lds\n\t" \
  "buffer_load_dwordx4 v252, s[40:43], s52 offen offset:1024 sc0 sc1 lds\n\t" \
  "buffer_load_dwordx4 v252, s[40:43], s52 offen offset:2048 sc0 sc1 lds\n\t" \
  "buffer_load_dwordx4 v252, s[40:43], s52 offen offset:3072 sc0 sc1 lds\n\t" \
  "s_nop 1\n\t" \
  "v_readlane_b32 s65, v255, s67\n\t" \
  "s_nop 3\n\t" \
  "s_add_u32 s67, s67, 1\n\t" \
  "s_cmp_ge_u32 s69, s65\n\t" \
  "s_cbranch_scc0 L_w38_%=\n\t" \
  "L_ok38_%=:\n\t" \
  "s_add_i32 s52, s48, 32768\n\t" \
  "s_add_u32 m0, s53, 65536\n\t" \
  "s_nop 0\n\t" \
  "buffer_load_dwordx4 v252, s[40:43], s52 offen offset:0 sc0 sc1 lds\n\t" \
  "buffer_load_dwordx4 v252, s[40:43], s52 offen offset:1024 sc0 sc1 lds\n\t" \
  "buffer_load_dwordx4 v252, s[40:43], s52 offen offset:2048 sc0 sc1 lds\n\t" \
  "buffer_load_dwordx4 v252, s[40:43], s52 offen offset:3072 sc0 sc1 lds\n\t" \
  "s_add_i32 s52, s49, 49152\n\t" \
  "s_add_u32 m0, s53, 147456\n\t" \
  "s_nop 0\n\t" \
  "buffer_load_dwordx4 v252, s[40:43], s52 offen offset:0 sc0 sc1 lds\n\t" \
  "buffer_load_dwordx4 v252, s[40:43], s52 offen offset:1024 sc0 sc1 lds\n\t" \
  "buffer_load_dwordx4 v252, s[40:43], s52 offen offset:2048 sc0 sc1 lds\n\t" \
  "buffer_load_dwordx4 v252, s[40:43], s52 offen offset:3072 sc0 sc1 lds\n\t" \
  "s_add_i32 s52, s49, 32768\n\t" \
  "s_add_u32 m0, s53, 131072\n\t" \
  "s_nop 0\n\t" \
  "buffer_load_dwordx4 v252, s[40:43], s52 offen offset:0 sc0 sc1 lds\n\t" \
  "buffer_load_dwordx4 v252, s[40:43], s52 offen offset:1024 sc0 sc1 lds\n\t" \
  "buffer_load_dwordx4 v252, s[40:43], s52 offen offset:2048 sc0 sc1 lds\n\t" \
  "buffer_load_dwordx4 v252, s[40:43], s52 offen offset:3072 sc0 sc1 lds\n\t" \
  "s_waitcnt vmcnt(0)\n\t" \
  "s_barrier\n\t" \
  "ds_read_b128 v[160:163], v251 offset:0\n\t" \
  "ds_read_b128 v[164:167], v251 offset:8192\n\t" \
  "ds_read_b128 v[168:171], v251 offset:256\n\t" \
  "ds_read_b128 v[172:175], v251 offset:8448\n\t" \
  "ds_read_b128 v[176:179], v251 offset:512\n\t" \
  "ds_read_b128 v[180:183], v251 offset:8704\n\t" \
  "ds_read_b128 v[184:187], v251 offset:768\n\t" \
  "ds_read_b128 v[188:191], v251 offset:8960\n\t" \
  "ds_read_b128 v[128:131], v250 offset:0\n\t" \
  "ds_read_b128 v[132:135], v250 offset:8192\n\t" \
  "ds_read_b128 v[136:139], v250 offset:256\n\t" \
  "ds_read_b128 v[140:143], v250 offset:8448\n\t" \
  "ds_read_b128 v[144:147], v250 offset:512\n\t" \
  "ds_read_b128 v[148:151], v250 offset:8704\n\t" \
  "ds_read_b128 v[152:155], v250 offset:768\n\t" \
  "ds_read_b128 v[156:159], v250 offset:8960\n\t" \
  "s_add_u32 s48, s48, 0x10000\n\t" \
  "s_add_u32 s49, s49, 0x10000\n\t" \
  "s_bitcmp1_b32 s70, 16\n\t" \
  "s_cbranch_scc1 L_half_%=\n\t" \
  "L_first_%=:\n\t" \
  "s_waitcnt lgkmcnt(2)\n\t" \
  "v_mfma_f32_16x16x128_f8f6f4 a[0:3], v[128:135], v[160:167], 0 blgp:1\n\t" \
  "ds_read_b128 v[192:195], v251 offset:16384\n\t" \
  "v_mfma_f32_16x16x128_f8f6f4 a[4:7], v[128:135], v[168:175], 0 blgp:1\n\t" \
  "ds_read_b128 v[196:199], v251 offset:24576\n\t" \
  "v_mfma_f32_16x16x128_f8f6f4 a[8:11], v[128:135], v[176:183], 0 blgp:1\n\t" \
  "ds_read_b128 v[200:203], v251 offset:16640\n\t" \
  "v_mfma_f32_16x16x128_f8f6f4 a[12:15], v[128:135], v[184:191], 0 blgp:1\n\t" \
  "ds_read_b128 v[204:207], v251 offset:24832\n\t" \
  "v_mfma_f32_16x16x128_f8f6f4 a[28:31], v[136:143], v[184:191], 0 blgp:1\n\t" \
  "ds_read_b128 v[208:211], v251 offset:16896\n\t" \
  "v_mfma_f32_16x16x128_f8f6f4 a[24:27], v[136:143], v[176:183], 0 blgp:1\n\t" \
  "ds_read_b128 v[212:215], v251 offset:25088\n\t" \
  "v_mfma_f32_16x16x128_f8f6f4 a[20:23], v[136:143], v[168:175], 0 blgp:1\n\t" \
  "ds_read_b128 v[216:219], v251 offset:17152\n\t" \
  "v_mfma_f32_16x16x128_f8f6f4 a[16:19], v[136:143], v[160:167], 0 blgp:1\n\t" \
  "ds_read_b128 v[220:223], v251 offset:25344\n\t" \
  "s_add_i32 s52, s48, -16384\n\t" \
  "s_add_u32 m0, s53, 81920\n\t" \
  "v_mfma_f32_16x16x128_f8f6f4 a[32:35], v[144:151], v[160:167], 0 blgp:1\n\t" \
  "buffer_load_dwordx4 v252, s[40:43], s52 offen offset:0 sc0 sc1 lds\n\t" \
  "v_mfma_f32_16x16x128_f8f6f4 a[36:39], v[144:151], v[168:175], 0 blgp:1\n\t" \
  "v_mfma_f32_16x16x128_f8f6f4 a[40:43], v[144:151], v[176:183], 0 blgp:1\n\t" \
  "buffer_load_dwordx4 v252, s[40:43], s52 offen offset:1024 sc0 sc1 lds\n\t" \
  "v_mfma_f32_16x16x128_f8f6f4 a[44:47], v[144:151], v[184:191], 0 blgp:1\n\t" \
  "s_waitcnt lgkmcnt(0)\n\t" \
  "v_mfma_f32_16x16x128_f8f6f4 a[60:63], v[152:159], v[184:191], 0 blgp:1\n\t" \
  "buffer_load_dwordx4 v252, s[40:43], s52 offen offset:2048 sc0 sc1 lds\n\t" \
  "v_mfma_f32_16x16x128_f8f6f4 a[56:59], v[152:159], v[176:183], 0 blgp:1\n\t" \
  "v_mfma_f32_16x16x128_f8f6f4 a[52:55], v[152:159], v[168:175], 0 blgp:1\n\t" \
  "buffer_load_dwordx4 v252, s[40:43], s52 offen offset:3072 sc0 sc1 lds\n\t" \
  "buffer_load_dword v254, off, s[88:91], s81 sc1\n\t" \
  "v_mfma_f32_16x16x128_f8f6f4 a[48:51], v[152:159], v[160:167], 0 blgp:1\n\t" \
  "s_waitcnt vmcnt(17) lgkmcnt(0)\n\t" \
  "s_barrier\n\t" \
  "v_mfma_f32_16x16x128_f8f6f4 v[0:3], v[128:135], v[192:199], 0 blgp:1\n\t" \
  "v_readlane_b32 s65, v255, s67\n\t" \
  "v_mfma_f32_16x16x128_f8f6f4 v[4:7], v[128:135], v[200:207], 0 blgp:1\n\t" \
  "v_mfma_f32_16x16x128_f8f6f4 v[8:11], v[128:135], v[208:215], 0 blgp:1\n\t" \
  "s_add_u32 s67, s67, 1\n\t" \
  "s_cmp_ge_u32 s69, s65\n\t" \
  "s_cbranch_scc0 L_w29_%=\n\t" \
  "L_ok29_%=:\n\t" \
  "v_mfma_f32_16x16x128_f8f6f4 v[12:15], v[128:135], v[216:223], 0 blgp:1\n\t" \
  "ds_read_b128 v[128:131], v250 offset:16384\n\t" \
  "v_mfma_f32_16x16x128_f8f6f4 v[28:31], v[136:143], v[216:223], 0 blgp:1\n\t" \
  "s_add_i32 s52, s48, 0\n\t" \
  "s_add_u32 m0, s53, 32768\n\t" \
  "ds_read_b128 v[132:135], v250 offset:24576\n\t" \
  "v_mfma_f32_16x16x128_f8f6f4 v[24:27], v[136:143], v[208:215], 0 blgp:1\n\t" \
  "buffer_load_dwordx4 v252, s[40:43], s52 offen offset:0 sc0 sc1 lds\n\t" \
  "v_mfma_f32_16x16x128_f8f6f4 v[20:23], v[136:143], v[200:207], 0 blgp:1\n\t" \
  "buffer_load_dwordx4 v252, s[40:43], s52 offen offset:1024 sc0 sc1 lds\n\t" \
  "v_mfma_f32_16x16x128_f8f6f4 v[16:19], v[136:143], v[192:199], 0 blgp:1\n\t" \
  "ds_read_b128 v[136:139], v250 offset:16640\n\t" \
  "v_mfma_f32_16x16x128_f8f6f4 v[32:35], v[144:151], v[192:199], 0 blgp:1\n\t" \
  "ds_read_b128 v[140:143], v250 offset:24832\n\t" \
  "v_mfma_f32_16x16x128_f8f6f4 v[36:39], v[144:151], v[200:207], 0 blgp:1\n\t" \
  "buffer_load_dwordx4 v252, s[40:43], s52 offen offset:2048 sc0 sc1 lds\n\t" \
  "v_mfma_f32_16x16x128_f8f6f4 v[40:43], v[144:151], v[208:215], 0 blgp:1\n\t" \
  "buffer_load_dwordx4 v252, s[40:43], s52 offen offset:3072 sc0 sc1 lds\n\t" \
  "v_mfma_f32_16x16x128_f8f6f4 v[44:47], v[144:151], v[216:223], 0 blgp:1\n\t" \
  "ds_read_b128 v[144:147], v250 offset:16896\n\t" \
  "v_mfma_f32_16x16x128_f8f6f4 v[60:63], v[152:159], v[216:223], 0 blgp:1\n\t" \
  "ds_read_b128 v[148:151], v250 offset:25088\n\t" \
  "v_mfma_f32_16x16x128_f8f6f4 v[56:59], v[152:159], v[208:215], 0 blgp:1\n\t" \
  "v_mfma_f32_16x16x128_f8f6f4 v[52:55], v[152:159], v[200:207], 0 blgp:1\n\t" \
  "v_mfma_f32_16x16x128_f8f6f4 v[48:51], v[152:159], v[192:199], 0 blgp:1\n\t" \
  "ds_read_b128 v[152:155], v250 offset:17152\n\t" \
  "ds_read_b128 v[156:159], v250 offset:25344\n\t" \
  "s_waitcnt lgkmcnt(2)\n\t" \
  "v_mfma_f32_16x16x128_f8f6f4 a[64:67], v[128:135], v[192:199], 0 blgp:1\n\t" \
  "s_add_i32 s52, s49, 0\n\t" \
  "s_add_u32 m0, s53, 98304\n\t" \
  "v_mfma_f32_16x16x128_f8f6f4 a[68:71], v[128:135], v[200:207], 0 blgp:1\n\t" \
  "buffer_load_dwordx4 v252, s[40:43], s52 offen offset:0 sc0 sc1 lds\n\t" \
  "v_mfma_f32_16x16x128_f8f6f4 a[72:75], v[128:135], v[208:215], 0 blgp:1\n\t" \
  "buffer_load_dwordx4 v252, s[40:43], s52 offen offset:1024 sc0 sc1 lds\n\t" \
  "v_mfma_f32_16x16x128_f8f6f4 a[76:79], v[128:135], v[216:223], 0 blgp:1\n\t" \
  "v_mfma_f32_16x16x128_f8f6f4 a[92:95], v[136:143], v[216:223], 0 blgp:1\n\t" \
  "buffer_load_dwordx4 v252, s[40:43], s52 offen offset:2048 sc0 sc1 lds\n\t" \
  "v_mfma_f32_16x16x128_f8f6f4 a[88:91], v[136:143], v[208:215], 0 blgp:1\n\t" \
  "buffer_load_dwordx4 v252, s[40:43], s52 offen offset:3072 sc0 sc1 lds\n\t" \
  "v_mfma_f32_16x16x128_f8f6f4 a[84:87], v[136:143], v[200:207], 0 blgp:1\n\t" \
  "s_add_i32 s52, s49, 16384\n\t" \
  "s_add_u32 m0, s53, 114688\n\t" \
  "v_mfma_f32_16x16x128_f8f6f4 a[80:83], v[136:143], v[192:199], 0 blgp:1\n\t" \
  "buffer_load_dwordx4 v252, s[40:43], s52 offen offset:0 sc0 sc1 lds\n\t" \
  "v_mfma_f32_16x16x128_f8f6f4 a[96:99], v[144:151], v[192:199], 0 blgp:1\n\t" \
  "buffer_load_dwordx4 v252, s[40:43], s52 offen offset:1024 sc0 sc1 lds\n\t" \
  "v_mfma_f32_16x16x128_f8f6f4 a[100:103], v[144:151], v[200:207], 0 blgp:1\n\t" \
  "v_mfma_f32_16x16x128_f8f6f4 a[104:107], v[144:151], v[208:215], 0 blgp:1\n\t" \
  "buffer_load_dwordx4 v252, s[40:43], s52 offen offset:2048 sc0 sc1 lds\n\t" \
  "v_mfma_f32_16x16x128_f8f6f4 a[108:111], v[144:151], v[216:223], 0 blgp:1\n\t" \
  "buffer_load_dwordx4 v252, s[40:43], s52 offen offset:3072 sc0 sc1 lds\n\t" \
  "s_waitcnt lgkmcnt(0)\n\t" \
  "v_mfma_f32_16x16x128_f8f6f4 a[124:127], v[152:159], v[216:223], 0 blgp:1\n\t" \
  "v_mfma_f32_16x16x128_f8f6f4 a[120:123], v[152:159], v[208:215], 0 blgp:1\n\t" \
  "v_mfma_f32_16x16x128_f8f6f4 a[116:119], v[152:159], v[200:207], 0 blgp:1\n\t" \
  "v_mfma_f32_16x16x128_f8f6f4 a[112:115], v[152:159], v[192:199], 0 blgp:1\n\t" \
  "s_waitcnt vmcnt(17) lgkmcnt(0)\n\t" \
  "s_barrier\n\t" \
  "v_mfma_f32_16x16x128_f8f6f4 v[64:67], v[128:135], v[160:167], 0 blgp:1\n\t" \
  "ds_read_b128 v[192:195], v251 offset:49152\n\t" \
  "s_mov_b32 s98, s70\n\t" \
  "v_mfma_f32_16x16x128_f8f6f4 v[68:71], v[128:135], v[168:175], 0 blgp:1\n\t" \
  "ds_read_b128 v[196:199], v251 offset:57344\n\t" \
  "s_cmp_gt_u32 s55, 1\n\t" \
  "s_cselect_b32 s70, s71, s70\n\t" \
  "v_mfma_f32_16x16x128_f8f6f4 v[72:75], v[128:135], v[176:183], 0 blgp:1\n\t" \
  "ds_read_b128 v[200:203], v251 offset:49408\n\t" \
  "s_mov_b32 s71, s72\n\t" \
  "v_mfma_f32_16x16x128_f8f6f4 v[76:79], v[128:135], v[184:191], 0 blgp:1\n\t" \
  "s_mov_b32 s72, s73\n\t" \
  "ds_read_b128 v[128:131], v250 offset:32768\n\t" \
  "ds_read_b128 v[132:135], v250 offset:40960\n\t" \
  "v_mfma_f32_16x16x128_f8f6f4 v[92:95], v[136:143], v[184:191], 0 blgp:1\n\t" \
  "ds_read_b128 v[204:207], v251 offset:57600\n\t" \
  "s_mov_b32 s73, s74\n\t" \
  "v_mfma_f32_16x16x128_f8f6f4 v[88:91], v[136:143], v[176:183], 0 blgp:1\n\t" \
  "ds_read_b128 v[208:211], v251 offset:49664\n\t" \
  "s_and_b32 s99, s70, 15\n\t" \
  "v_mfma_f32_16x16x128_f8f6f4 v[84:87], v[136:143], v[168:175], 0 blgp:1\n\t" \
  "ds_read_b128 v[212:215], v251 offset:57856\n\t" \
  "s_mul_i32 s50, s99, 0x100800\n\t" \
  "v_mfma_f32_16x16x128_f8f6f4 v[80:83], v[136:143], v[160:167], 0 blgp:1\n\t" \
  "s_lshl_b32 s64, s99, 24\n\t" \
  "ds_read_b128 v[136:139], v250 offset:33024\n\t" \
  "ds_read_b128 v[140:143], v250 offset:41216\n\t" \
  "v_mfma_f32_16x16x128_f8f6f4 v[96:99], v[144:151], v[160:167], 0 blgp:1\n\t" \
  "ds_read_b128 v[216:219], v251 offset:49920\n\t" \
  "s_lshr_b32 s96, s99, 2\n\t" \
  "v_mfma_f32_16x16x128_f8f6f4 v[100:103], v[144:151], v[168:175], 0 blgp:1\n\t" \
  "ds_read_b128 v[220:223], v251 offset:58112\n\t" \
  "s_lshl_b32 s96, s96, 9\n\t" \
  "v_mfma_f32_16x16x128_f8f6f4 v[104:107], v[144:151], v[176:183], 0 blgp:1\n\t" \
  "s_and_b32 s99, s99, 3\n\t" \
  "v_mfma_f32_16x16x128_f8f6f4 v[108:111], v[144:151], v[184:191], 0 blgp:1\n\t" \
  "s_lshl2_add_u32 s96, s99, s96\n\t" \
  "ds_read_b128 v[144:147], v250 offset:33280\n\t" \
  "v_mfma_f32_16x16x128_f8f6f4 v[124:127], v[152:159], v[184:191], 0 blgp:1\n\t" \
  "s_bfe_u32 s99, s70, 0x60004\n\t" \
  "ds_read_b128 v[148:151], v250 offset:41472\n\t" \
  "v_mfma_f32_16x16x128_f8f6f4 v[120:123], v[152:159], v[176:183], 0 blgp:1\n\t" \
  "s_mul_i32 s51, s99, 0x100800\n\t" \
  "v_mfma_f32_16x16x128_f8f6f4 v[116:119], v[152:159], v[168:175], 0 blgp:1\n\t" \
  "v_mfma_f32_16x16x128_f8f6f4 v[112:115], v[152:159], v[160:167], 0 blgp:1\n\t" \
  "ds_read_b128 v[152:155], v250 offset:33536\n\t" \
  "ds_read_b128 v[156:159], v250 offset:41728\n\t" \
  "s_waitcnt lgkmcnt(2)\n\t" \
  "v_mfma_f32_16x16x128_f8f6f4 v[0:3], v[128:135], v[192:199], v[0:3] blgp:1\n\t" \
  "ds_read_b128 v[160:163], v251 offset:32768\n\t" \
  "s_add_u32 s51, s51, 0x1008000\n\t" \
  "v_mfma_f32_16x16x128_f8f6f4 v[4:7], v[128:135], v[200:207], v[4:7] blgp:1\n\t" \
  "ds_read_b128 v[164:167], v251 offset:40960\n\t" \
  "s_lshl_b32 s99, s99, 10\n\t" \
  "v_mfma_f32_16x16x128_f8f6f4 v[8:11], v[128:135], v[208:215], v[8:11] blgp:1\n\t" \
  "ds_read_b128 v[168:171], v251 offset:33024\n\t" \
  "s_add_u32 s64, s64, s99\n\t" \
  "v_mfma_f32_16x16x128_f8f6f4 v[12:15], v[128:135], v[216:223], v[12:15] blgp:1\n\t" \
  "ds_read_b128 v[172:175], v251 offset:41216\n\t" \
  "s_bfe_u32 s97, s70, 0x40006\n\t" \
  "v_mfma_f32_16x16x128_f8f6f4 v[28:31], v[136:143], v[216:223], v[28:31] blgp:1\n\t" \
  "ds_read_b128 v[176:179], v251 offset:33280\n\t" \
  "s_add_u32 s97, s97, 4\n\t" \
  "v_mfma_f32_16x16x128_f8f6f4 v[24:27], v[136:143], v[208:215], v[24:27] blgp:1\n\t" \
  "ds_read_b128 v[180:183], v251 offset:41472\n\t" \
  "s_lshl_b32 s97, s97, 9\n\t" \
  "v_mfma_f32_16x16x128_f8f6f4 v[20:23], v[136:143], v[200:207], v[20:23] blgp:1\n\t" \
  "ds_read_b128 v[184:187], v251 offset:33536\n\t" \
  "s_bfe_u32 s99, s70, 0x20004\n\t" \
  "v_mfma_f32_16x16x128_f8f6f4 v[16:19], v[136:143], v[192:199], v[16:19] blgp:1\n\t" \
  "ds_read_b128 v[188:191], v251 offset:41728\n\t" \
  "s_add_i32 s52, s48, 16384\n\t" \
  "s_add_u32 m0, s53, 49152\n\t" \
  "s_lshl2_add_u32 s97, s99, s97\n\t" \
  "v_mfma_f32_16x16x128_f8f6f4 v[32:35], v[144:151], v[192:199], v[32:35] blgp:1\n\t" \
  "buffer_load_dwordx4 v252, s[40:43], s52 offen offset:0 sc0 sc1 lds\n\t" \
  "s_mov_b32 s54, 14\n\t" \
  "v_mfma_f32_16x16x128_f8f6f4 v[36:39], v[144:151], v[200:207], v[36:39] blgp:1\n\t" \
  "v_mfma_f32_16x16x128_f8f6f4 v[40:43], v[144:151], v[208:215], v[40:43] blgp:1\n\t" \
  "buffer_load_dwordx4 v252, s[40:43], s52 offen offset:1024 sc0 sc1 lds\n\t" \
  "v_mfma_f32_16x16x128_f8f6f4 v[44:47], v[144:151], v[216:223], v[44:47] blgp:1\n\t" \
  "s_waitcnt lgkmcnt(0)\n\t" \
  "v_mfma_f32_16x16x128_f8f6f4 v[60:63], v[152:159], v[216:223], v[60:63] blgp:1\n\t" \
  "buffer_load_dwordx4 v252, s[40:43], s52 offen offset:2048 sc0 sc1 lds\n\t" \
  "v_mfma_f32_16x16x128_f8f6f4 v[56:59], v[152:159], v[208:215], v[56:59] blgp:1\n\t" \
  "v_mfma_f32_16x16x128_f8f6f4 v[52:55], v[152:159], v[200:207], v[52:55] blgp:1\n\t" \
  "buffer_load_dwordx4 v252, s[40:43], s52 offen offset:3072 sc0 sc1 lds\n\t" \
  "v_mfma_f32_16x16x128_f8f6f4 v[48:51], v[152:159], v[192:199], v[48:51] blgp:1\n\t" \
  "s_waitcnt vmcnt(16) lgkmcnt(0)\n\t" \
  "s_barrier\n\t" \
  "v_mfma_f32_16x16x128_f8f6f4 a[0:3], v[128:135], v[160:167], a[0:3] blgp:1\n\t" \
  "v_readfirstlane_b32 s87, v254\n\t" \
  "v_readlane_b32 s65, v255, s67\n\t" \
  "v_mfma_f32_16x16x128_f8f6f4 a[4:7], v[128:135], v[168:175], a[4:7] blgp:1\n\t" \
  "v_mfma_f32_16x16x128_f8f6f4 a[8:11], v[128:135], v[176:183], a[8:11] blgp:1\n\t" \
  "s_max_u32 s69, s69, s87\n\t" \
  "s_add_u32 s67, s67, 1\n\t" \
  "s_cmp_ge_u32 s69, s65\n\t" \
  "s_cbranch_scc0 L_w30_%=\n\t" \
  "L_ok30_%=:\n\t" \
  "v_mfma_f32_16x16x128_f8f6f4 a[12:15], v[128:135], v[184:191], a[12:15] blgp:1\n\t" \
  "ds_read_b128 v[128:131], v250 offset:49152\n\t" \
  "v_mfma_f32_16x16x128_f8f6f4 a[28:31], v[136:143], v[184:191], a[28:31] blgp:1\n\t" \
  "s_add_i32 s52, s48, 32768\n\t" \
  "s_add_u32 m0, s53, 65536\n\t" \
  "ds_read_b128 v[132:135], v250 offset:57344\n\t" \
  "v_mfma_f32_16x16x128_f8f6f4 a[24:27], v[136:143], v[176:183], a[24:27] blgp:1\n\t" \
  "buffer_load_dwordx4 v252, s[40:43], s52 offen offset:0 sc0 sc1 lds\n\t" \
  "v_mfma_f32_16x16x128_f8f6f4 a[20:23], v[136:143], v[168:175], a[20:23] blgp:1\n\t" \
  "buffer_load_dwordx4 v252, s[40:43], s52 offen offset:1024 sc0 sc1 lds\n\t" \
  "v_mfma_f32_16x16x128_f8f6f4 a[16:19], v[136:143], v[160:167], a[16:19] blgp:1\n\t" \
  "ds_read_b128 v[136:139], v250 offset:49408\n\t" \
  "v_mfma_f32_16x16x128_f8f6f4 a[32:35], v[144:151], v[160:167], a[32:35] blgp:1\n\t" \
  "ds_read_b128 v[140:143], v250 offset:57600\n\t" \
  "v_mfma_f32_16x16x128_f8f6f4 a[36:39], v[144:151], v[168:175], a[36:39] blgp:1\n\t" \
  "buffer_load_dwordx4 v252, s[40:43], s52 offen offset:2048 sc0 sc1 lds\n\t" \
  "v_mfma_f32_16x16x128_f8f6f4 a[40:43], v[144:151], v[176:183], a[40:43] blgp:1\n\t" \
  "buffer_load_dwordx4 v252, s[40:43], s52 offen offset:3072 sc0 sc1 lds\n\t" \
  "v_mfma_f32_16x16x128_f8f6f4 a[44:47], v[144:151], v[184:191], a[44:47] blgp:1\n\t" \
  "ds_read_b128 v[144:147], v250 offset:49664\n\t" \
  "v_mfma_f32_16x16x128_f8f6f4 a[60:63], v[152:159], v[184:191], a[60:63] blgp:1\n\t" \
  "ds_read_b128 v[148:151], v250 offset:57856\n\t" \
  "v_mfma_f32_16x16x128_f8f6f4 a[56:59], v[152:159], v[176:183], a[56:59] blgp:1\n\t" \
  "v_mfma_f32_16x16x128_f8f6f4 a[52:55], v[152:159], v[168:175], a[52:55] blgp:1\n\t" \
  "v_mfma_f32_16x16x128_f8f6f4 a[48:51], v[152:159], v[160:167], a[48:51] blgp:1\n\t" \
  "ds_read_b128 v[152:155], v250 offset:49920\n\t" \
  "ds_read_b128 v[156:159], v250 offset:58112\n\t" \
  "s_waitcnt lgkmcnt(2)\n\t" \
  "v_mfma_f32_16x16x128_f8f6f4 v[64:67], v[128:135], v[160:167], v[64:67] blgp:1\n\t" \
  "s_add_i32 s52, s49, 49152\n\t" \
  "s_add_u32 m0, s53, 147456\n\t" \
  "v_mfma_f32_16x16x128_f8f6f4 v[68:71], v[128:135], v[168:175], v[68:71] blgp:1\n\t" \
  "buffer_load_dwordx4 v252, s[40:43], s52 offen offset:0 sc0 sc1 lds\n\t" \
  "v_mfma_f32_16x16x128_f8f6f4 v[72:75], v[128:135], v[176:183], v[72:75] blgp:1\n\t" \
  "buffer_load_dwordx4 v252, s[40:43], s52 offen offset:1024 sc0 sc1 lds\n\t" \
  "v_mfma_f32_16x16x128_f8f6f4 v[76:79], v[128:135], v[184:191], v[76:79] blgp:1\n\t" \
  "buffer_load_dword v248, v252, s[92:95], s96 offen\n\t" \
  "buffer_load_dword v249, v252, s[92:95], s97 offen\n\t" \
  "v_mfma_f32_16x16x128_f8f6f4 v[92:95], v[136:143], v[184:191], v[92:95] blgp:1\n\t" \
  "buffer_load_dwordx4 v252, s[40:43], s52 offen offset:2048 sc0 sc1 lds\n\t" \
  "v_mfma_f32_16x16x128_f8f6f4 v[88:91], v[136:143], v[176:183], v[88:91] blgp:1\n\t" \
  "buffer_load_dwordx4 v252, s[40:43], s52 offen offset:3072 sc0 sc1 lds\n\t" \
  "v_mfma_f32_16x16x128_f8f6f4 v[84:87], v[136:143], v[168:175], v[84:87] blgp:1\n\t" \
  "s_add_i32 s52, s49, 32768\n\t" \
  "s_add_u32 m0, s53, 131072\n\t" \
  "v_mfma_f32_16x16x128_f8f6f4 v[80:83], v[136:143], v[160:167], v[80:83] blgp:1\n\t" \
  "buffer_load_dwordx4 v252, s[40:43], s52 offen offset:0 sc0 sc1 lds\n\t" \
  "v_mfma_f32_16x16x128_f8f6f4 v[96:99], v[144:151], v[160:167], v[96:99] blgp:1\n\t" \
  "buffer_load_dwordx4 v252, s[40:43], s52 offen offset:1024 sc0 sc1 lds\n\t" \
  "v_mfma_f32_16x16x128_f8f6f4 v[100:103], v[144:151], v[168:175], v[100:103] blgp:1\n\t" \
  "v_mfma_f32_16x16x128_f8f6f4 v[104:107], v[144:151], v[176:183], v[104:107] blgp:1\n\t" \
  "buffer_load_dwordx4 v252, s[40:43], s52 offen offset:2048 sc0 sc1 lds\n\t" \
  "v_mfma_f32_16x16x128_f8f6f4 v[108:111], v[144:151], v[184:191], v[108:111] blgp:1\n\t" \
  "buffer_load_dwordx4 v252, s[40:43], s52 offen offset:3072 sc0 sc1 lds\n\t" \
  "s_waitcnt lgkmcnt(0)\n\t" \
  "v_mfma_f32_16x16x128_f8f6f4 v[124:127], v[152:159], v[184:191], v[124:127] blgp:1\n\t" \
  "v_mfma_f32_16x16x128_f8f6f4 v[120:123], v[152:159], v[176:183], v[120:123] blgp:1\n\t" \
  "v_mfma_f32_16x16x128_f8f6f4 v[116:119], v[152:159], v[168:175], v[116:119] blgp:1\n\t" \
  "v_mfma_f32_16x16x128_f8f6f4 v[112:115], v[152:159], v[160:167], v[112:115] blgp:1\n\t" \
  "s_waitcnt vmcnt(18) lgkmcnt(0)\n\t" \
  "s_barrier\n\t" \
  "v_mfma_f32_16x16x128_f8f6f4 a[64:67], v[128:135], v[192:199], a[64:67] blgp:1\n\t" \
  "ds_read_b128 v[160:163], v251 offset:0\n\t" \
  "v_mfma_f32_16x16x128_f8f6f4 a[68:71], v[128:135], v[200:207], a[68:71] blgp:1\n\t" \
  "ds_read_b128 v[164:167], v251 offset:8192\n\t" \
  "v_mfma_f32_16x16x128_f8f6f4 a[72:75], v[128:135], v[208:215], a[72:75] blgp:1\n\t" \
  "ds_read_b128 v[168:171], v251 offset:256\n\t" \
  "v_mfma_f32_16x16x128_f8f6f4 a[76:79], v[128:135], v[216:223], a[76:79] blgp:1\n\t" \
  "ds_read_b128 v[128:131], v250 offset:0\n\t" \
  "ds_read_b128 v[132:135], v250 offset:8192\n\t" \
  "v_mfma_f32_16x16x128_f8f6f4 a[92:95], v[136:143], v[216:223], a[92:95] blgp:1\n\t" \
  "ds_read_b128 v[172:175], v251 offset:8448\n\t" \
  "v_mfma_f32_16x16x128_f8f6f4 a[88:91], v[136:143], v[208:215], a[88:91] blgp:1\n\t" \
  "ds_read_b128 v[176:179], v251 offset:512\n\t" \
  "v_mfma_f32_16x16x128_f8f6f4 a[84:87], v[136:143], v[200:207], a[84:87] blgp:1\n\t" \
  "ds_read_b128 v[180:183], v251 offset:8704\n\t" \
  "v_mfma_f32_16x16x128_f8f6f4 a[80:83], v[136:143], v[192:199], a[80:83] blgp:1\n\t" \
  "ds_read_b128 v[136:139], v250 offset:256\n\t" \
  "ds_read_b128 v[140:143], v250 offset:8448\n\t" \
  "v_mfma_f32_16x16x128_f8f6f4 a[96:99], v[144:151], v[192:199], a[96:99] blgp:1\n\t" \
  "ds_read_b128 v[184:187], v251 offset:768\n\t" \
  "v_mfma_f32_16x16x128_f8f6f4 a[100:103], v[144:151], v[200:207], a[100:103] blgp:1\n\t" \
  "ds_read_b128 v[188:191], v251 offset:8960\n\t" \
  "v_mfma_f32_16x16x128_f8f6f4 a[104:107], v[144:151], v[208:215], a[104:107] blgp:1\n\t" \
  "s_add_u32 s48, s48, 0x10000\n\t" \
  "s_add_u32 s49, s49, 0x10000\n\t" \
  "v_mfma_f32_16x16x128_f8f6f4 a[108:111], v[144:151], v[216:223], a[108:111] blgp:1\n\t" \
  "ds_read_b128 v[144:147], v250 offset:512\n\t" \
  "v_mfma_f32_16x16x128_f8f6f4 a[124:127], v[152:159], v[216:223], a[124:127] blgp:1\n\t" \
  "ds_read_b128 v[148:151], v250 offset:8704\n\t" \
  "v_mfma_f32_16x16x128_f8f6f4 a[120:123], v[152:159], v[208:215], a[120:123] blgp:1\n\t" \
  "v_mfma_f32_16x16x128_f8f6f4 a[116:119], v[152:159], v[200:207], a[116:119] blgp:1\n\t" \
  "v_mfma_f32_16x16x128_f8f6f4 a[112:115], v[152:159], v[192:199], a[112:115] blgp:1\n\t" \
  "ds_read_b128 v[152:155], v250 offset:768\n\t" \
  "ds_read_b128 v[156:159], v250 offset:8960\n\t" \
  "s_branch L_k_%=\n\t" \
  "L_tile_%=:\n\t" \
  "v_pk_fma_f32 v[200:201], v[32:33], s[58:59], s[58:59] op_sel:[0,0,1] op_sel_hi:[1,0,1]\n\t" \
  "v_pk_fma_f32 v[202:203], v[36:37], s[58:59], s[58:59] op_sel:[0,0,1] op_sel_hi:[1,0,1]\n\t" \
  "v_pk_fma_f32 v[204:205], v[40:41], s[58:59], s[58:59] op_sel:[0,0,1] op_sel_hi:[1,0,1]\n\t" \
  "v_pk_fma_f32 v[206:207], v[44:45], s[58:59], s[58:59] op_sel:[0,0,1] op_sel_hi:[1,0,1]\n\t" \
  "v_exp_f32_e64 v192, v200 clamp\n\t" \
  "v_exp_f32_e64 v193, v202 clamp\n\t" \
  "v_exp_f32_e64 v194, v204 clamp\n\t" \
  "v_exp_f32_e64 v195, v206 clamp\n\t" \
  "v_exp_f32_e64 v196, v201 clamp\n\t" \
  "v_exp_f32_e64 v197, v203 clamp\n\t" \
  "v_exp_f32_e64 v198, v205 clamp\n\t" \
  "v_exp_f32_e64 v199, v207 clamp\n\t" \
  "v_pk_fma_f32 v[192:193], v[192:193], s[60:61], s[60:61] op_sel:[0,1,0] op_sel_hi:[1,1,0]\n\t" \
  "v_pk_fma_f32 v[194:195], v[194:195], s[60:61], s[60:61] op_sel:[0,1,0] op_sel_hi:[1,1,0]\n\t" \
  "v_pk_fma_f32 v[196:197], v[196:197], s[60:61], s[60:61] op_sel:[0,1,0] op_sel_hi:[1,1,0]\n\t" \
  "v_pk_fma_f32 v[198:199], v[198:199], s[60:61], s[60:61] op_sel:[0,1,0] op_sel_hi:[1,1,0]\n\t" \
  "v_rcp_f32 v192, v192\n\t" \
  "v_rcp_f32 v193, v193\n\t" \
  "v_rcp_f32 v194, v194\n\t" \
  "v_rcp_f32 v195, v195\n\t" \
  "v_rcp_f32 v196, v196\n\t" \
  "v_rcp_f32 v197, v197\n\t" \
  "v_rcp_f32 v198, v198\n\t" \
  "v_rcp_f32 v199, v199\n\t" \
  "v_mul_f32 v192, v32, v192\n\t" \
  "v_mul_f32 v193, v36, v193\n\t" \
  "v_mul_f32 v194, v40, v194\n\t" \
  "v_mul_f32 v195, v44, v195\n\t" \
  "s_add_u32 s57, s80, 2097152\n\t" \
  "buffer_store_dwordx4 v[192:195], v253, s[44:47], s57 offen offset:256 nt\n\t" \
  "v_mul_f32 v196, v33, v196\n\t" \
  "v_mul_f32 v197, v37, v197\n\t" \
  "v_mul_f32 v198, v41, v198\n\t" \
  "v_mul_f32 v199, v45, v199\n\t" \
  "s_add_u32 s57, s80, 2162688\n\t" \
  "buffer_store_dwordx4 v[196:199], v253, s[44:47], s57 offen offset:256 nt\n\t" \
  "v_pk_fma_f32 v[216:217], v[34:35], s[58:59], s[58:59] op_sel:[0,0,1] op_sel_hi:[1,0,1]\n\t" \
  "v_pk_fma_f32 v[218:219], v[38:39], s[58:59], s[58:59] op_sel:[0,0,1] op_sel_hi:[1,0,1]\n\t" \
  "v_pk_fma_f32 v[220:221], v[42:43], s[58:59], s[58:59] op_sel:[0,0,1] op_sel_hi:[1,0,1]\n\t" \
  "v_pk_fma_f32 v[222:223], v[46:47], s[58:59], s[58:59] op_sel:[0,0,1] op_sel_hi:[1,0,1]\n\t" \
  "v_exp_f32_e64 v208, v216 clamp\n\t" \
  "v_exp_f32_e64 v209, v218 clamp\n\t" \
  "v_exp_f32_e64 v210, v220 clamp\n\t" \
  "v_exp_f32_e64 v211, v222 clamp\n\t" \
  "v_exp_f32_e64 v212, v217 clamp\n\t" \
  "v_exp_f32_e64 v213, v219 clamp\n\t" \
  "v_exp_f32_e64 v214, v221 clamp\n\t" \
  "v_exp_f32_e64 v215, v223 clamp\n\t" \
  "v_pk_fma_f32 v[208:209], v[208:209], s[60:61], s[60:61] op_sel:[0,1,0] op_sel_hi:[1,1,0]\n\t" \
  "v_pk_fma_f32 v[210:211], v[210:211], s[60:61], s[60:61] op_sel:[0,1,0] op_sel_hi:[1,1,0]\n\t" \
  "v_pk_fma_f32 v[212:213], v[212:213], s[60:61], s[60:61] op_sel:[0,1,0] op_sel_hi:[1,1,0]\n\t" \
  "v_pk_fma_f32 v[214:215], v[214:215], s[60:61], s[60:61] op_sel:[0,1,0] op_sel_hi:[1,1,0]\n\t" \
  "v_rcp_f32 v208, v208\n\t" \
  "v_rcp_f32 v209, v209\n\t" \
  "v_rcp_f32 v210, v210\n\t" \
  "v_rcp_f32 v211, v211\n\t" \
  "v_rcp_f32 v212, v212\n\t" \
  "v_rcp_f32 v213, v213\n\t" \
  "v_rcp_f32 v214, v214\n\t" \
  "v_rcp_f32 v215, v215\n\t" \
  "v_mul_f32 v208, v34, v208\n\t" \
  "v_mul_f32 v209, v38, v209\n\t" \
  "v_mul_f32 v210, v42, v210\n\t" \
  "v_mul_f32 v211, v46, v211\n\t" \
  "s_add_u32 s57, s80, 2228224\n\t" \
  "buffer_store_dwordx4 v[208:211], v253, s[44:47], s57 offen offset:256 nt\n\t" \
  "v_mul_f32 v212, v35, v212\n\t" \
  "v_mul_f32 v213, v39, v213\n\t" \
  "v_mul_f32 v214, v43, v214\n\t" \
  "v_mul_f32 v215, v47, v215\n\t" \
  "s_add_u32 s57, s80, 2293760\n\t" \
  "buffer_store_dwordx4 v[212:215], v253, s[44:47], s57 offen offset:256 nt\n\t" \
  "s_waitcnt lgkmcnt(2)\n\t" \
  "v_mfma_f32_16x16x128_f8f6f4 a[0:3], v[128:135], v[160:167], 0 blgp:1\n\t" \
  "ds_read_b128 v[192:195], v251 offset:16384\n\t" \
  "v_fma_f32 v240, v64, s58, v248\n\t" \
  "v_fma_f32 v241, v68, s58, v248\n\t" \
  "v_fma_f32 v242, v72, s58, v248\n\t" \
  "v_fma_f32 v243, v76, s58, v248\n\t" \
  "s_add_u32 s57, s80, 1245184\n\t" \
  "v_mfma_f32_16x16x128_f8f6f4 a[4:7], v[128:135], v[168:175], 0 blgp:1\n\t" \
  "ds_read_b128 v[196:199], v251 offset:24576\n\t" \
  "buffer_store_dwordx4 v[244:247], v253, s[44:47], s57 offen offset:256 nt\n\t" \
  "v_exp_f32_e64 v240, v240 clamp\n\t" \
  "v_mfma_f32_16x16x128_f8f6f4 a[8:11], v[128:135], v[176:183], 0 blgp:1\n\t" \
  "ds_read_b128 v[200:203], v251 offset:16640\n\t" \
  "v_exp_f32_e64 v241, v241 clamp\n\t" \
  "v_exp_f32_e64 v242, v242 clamp\n\t" \
  "v_exp_f32_e64 v243, v243 clamp\n\t" \
  "v_mfma_f32_16x16x128_f8f6f4 a[12:15], v[128:135], v[184:191], 0 blgp:1\n\t" \
  "ds_read_b128 v[204:207], v251 offset:24832\n\t" \
  "v_fma_f32 v240, v240, v249, s60\n\t" \
  "v_fma_f32 v241, v241, v249, s60\n\t" \
  "v_fma_f32 v242, v242, v249, s60\n\t" \
  "v_fma_f32 v243, v243, v249, s60\n\t" \
  "v_rcp_f32 v240, v240\n\t" \
  "v_mfma_f32_16x16x128_f8f6f4 a[28:31], v[136:143], v[184:191], 0 blgp:1\n\t" \
  "ds_read_b128 v[208:211], v251 offset:16896\n\t" \
  "v_rcp_f32 v241, v241\n\t" \
  "v_rcp_f32 v242, v242\n\t" \
  "v_mfma_f32_16x16x128_f8f6f4 a[24:27], v[136:143], v[176:183], 0 blgp:1\n\t" \
  "ds_read_b128 v[212:215], v251 offset:25088\n\t" \
  "v_rcp_f32 v243, v243\n\t" \
  "v_mul_f32 v240, v64, v240\n\t" \
  "v_mul_f32 v241, v68, v241\n\t" \
  "v_mul_f32 v242, v72, v242\n\t" \
  "v_mul_f32 v243, v76, v243\n\t" \
  "v_mfma_f32_16x16x128_f8f6f4 a[20:23], v[136:143], v[168:175], 0 blgp:1\n\t" \
  "ds_read_b128 v[216:219], v251 offset:17152\n\t" \
  "v_fma_f32 v244, v65, s58, v248\n\t" \
  "v_fma_f32 v245, v69, s58, v248\n\t" \
  "v_fma_f32 v246, v73, s58, v248\n\t" \
  "v_fma_f32 v247, v77, s58, v248\n\t" \
  "s_add_u32 s57, s80, 4194304\n\t" \
  "v_mfma_f32_16x16x128_f8f6f4 a[16:19], v[136:143], v[160:167], 0 blgp:1\n\t" \
  "ds_read_b128 v[220:223], v251 offset:25344\n\t" \
  "s_add_i32 s52, s48, -16384\n\t" \
  "s_add_u32 m0, s53, 81920\n\t" \
  "buffer_store_dwordx4 v[240:243], v253, s[44:47], s57 offen offset:0 nt\n\t" \
  "v_exp_f32_e64 v244, v244 clamp\n\t" \
  "v_mfma_f32_16x16x128_f8f6f4 a[32:35], v[144:151], v[160:167], 0 blgp:1\n\t" \
  "buffer_load_dwordx4 v252, s[40:43], s52 offen offset:0 sc0 sc1 lds\n\t" \
  "v_exp_f32_e64 v245, v245 clamp\n\t" \
  "v_exp_f32_e64 v246, v246 clamp\n\t" \
  "v_mfma_f32_16x16x128_f8f6f4 a[36:39], v[144:151], v[168:175], 0 blgp:1\n\t" \
  "v_exp_f32_e64 v247, v247 clamp\n\t" \
  "v_fma_f32 v244, v244, v249, s60\n\t" \
  "v_fma_f32 v245, v245, v249, s60\n\t" \
  "v_fma_f32 v246, v246, v249, s60\n\t" \
  "v_fma_f32 v247, v247, v249, s60\n\t" \
  "v_mfma_f32_16x16x128_f8f6f4 a[40:43], v[144:151], v[176:183], 0 blgp:1\n\t" \
  "buffer_load_dwordx4 v252, s[40:43], s52 offen offset:1024 sc0 sc1 lds\n\t" \
  "v_rcp_f32 v244, v244\n\t" \
  "v_rcp_f32 v245, v245\n\t" \
  "v_mfma_f32_16x16x128_f8f6f4 a[44:47], v[144:151], v[184:191], 0 blgp:1\n\t" \
  "v_rcp_f32 v246, v246\n\t" \
  "v_rcp_f32 v247, v247\n\t" \
  "v_mul_f32 v244, v65, v244\n\t" \
  "v_mul_f32 v245, v69, v245\n\t" \
  "v_mul_f32 v246, v73, v246\n\t" \
  "s_waitcnt lgkmcnt(0)\n\t" \
  "v_mfma_f32_16x16x128_f8f6f4 a[60:63], v[152:159], v[184:191], 0 blgp:1\n\t" \
  "buffer_load_dwordx4 v252, s[40:43], s52 offen offset:2048 sc0 sc1 lds\n\t" \
  "v_mul_f32 v247, v77, v247\n\t" \
  "v_fma_f32 v240, v66, s58, v248\n\t" \
  "v_fma_f32 v241, v70, s58, v248\n\t" \
  "v_fma_f32 v242, v74, s58, v248\n\t" \
  "v_fma_f32 v243, v78, s58, v248\n\t" \
  "v_mfma_f32_16x16x128_f8f6f4 a[56:59], v[152:159], v[176:183], 0 blgp:1\n\t" \
  "s_add_u32 s57, s80, 4259840\n\t" \
  "buffer_store_dwordx4 v[244:247], v253, s[44:47], s57 offen offset:0 nt\n\t" \
  "v_exp_f32_e64 v240, v240 clamp\n\t" \
  "v_mfma_f32_16x16x128_f8f6f4 a[52:55], v[152:159], v[168:175], 0 blgp:1\n\t" \
  "buffer_load_dwordx4 v252, s[40:43], s52 offen offset:3072 sc0 sc1 lds\n\t" \
  "buffer_load_dword v254, off, s[88:91], s81 sc1\n\t" \
  "v_exp_f32_e64 v241, v241 clamp\n\t" \
  "v_mfma_f32_16x16x128_f8f6f4 a[48:51], v[152:159], v[160:167], 0 blgp:1\n\t" \
  "v_exp_f32_e64 v242, v242 clamp\n\t" \
  "v_exp_f32_e64 v243, v243 clamp\n\t" \
  "v_fma_f32 v240, v240, v249, s60\n\t" \
  "s_waitcnt vmcnt(27) lgkmcnt(0)\n\t" \
  "s_barrier\n\t" \
  "v_mfma_f32_16x16x128_f8f6f4 v[0:3], v[128:135], v[192:199], 0 blgp:1\n\t" \
  "v_readlane_b32 s65, v255, s67\n\t" \
  "v_fma_f32 v241, v241, v249, s60\n\t" \
  "v_fma_f32 v242, v242, v249, s60\n\t" \
  "v_fma_f32 v243, v243, v249, s60\n\t" \
  "v_rcp_f32 v240, v240\n\t" \
  "v_rcp_f32 v241, v241\n\t" \
  "v_mfma_f32_16x16x128_f8f6f4 v[4:7], v[128:135], v[200:207], 0 blgp:1\n\t" \
  "ds_write_b128 v252, v[96:99] offset:16384\n\t" \
  "v_rcp_f32 v242, v242\n\t" \
  "v_rcp_f32 v243, v243\n\t" \
  "v_mul_f32 v240, v66, v240\n\t" \
  "v_mfma_f32_16x16x128_f8f6f4 v[8:11], v[128:135], v[208:215], 0 blgp:1\n\t" \
  "s_add_u32 s67, s67, 1\n\t" \
  "s_cmp_ge_u32 s69, s65\n\t" \
  "s_cbranch_scc0 L_w1_%=\n\t" \
  "L_ok1_%=:\n\t" \
  "v_mul_f32 v241, v70, v241\n\t" \
  "v_mul_f32 v242, v74, v242\n\t" \
  "v_mul_f32 v243, v78, v243\n\t" \
  "v_fma_f32 v244, v67, s58, v248\n\t" \
  "v_fma_f32 v245, v71, s58, v248\n\t" \
  "v_mfma_f32_16x16x128_f8f6f4 v[12:15], v[128:135], v[216:223], 0 blgp:1\n\t" \
  "ds_read_b128 v[128:131], v250 offset:16384\n\t" \
  "v_fma_f32 v246, v75, s58, v248\n\t" \
  "ds_write_b128 v252, v[100:103] offset:17408\n\t" \
  "v_fma_f32 v247, v79, s58, v248\n\t" \
  "s_add_u32 s57, s80, 4325376\n\t" \
  "v_mfma_f32_16x16x128_f8f6f4 v[28:31], v[136:143], v[216:223], 0 blgp:1\n\t" \
  "s_add_i32 s52, s48, 0\n\t" \
  "s_add_u32 m0, s53, 32768\n\t" \
  "ds_read_b128 v[132:135], v250 offset:24576\n\t" \
  "buffer_store_dwordx4 v[240:243], v253, s[44:47], s57 offen offset:0 nt\n\t" \
  "v_exp_f32_e64 v244, v244 clamp\n\t" \
  "v_mfma_f32_16x16x128_f8f6f4 v[24:27], v[136:143], v[208:215], 0 blgp:1\n\t" \
  "buffer_load_dwordx4 v252, s[40:43], s52 offen offset:0 sc0 sc1 lds\n\t" \
  "v_exp_f32_e64 v245, v245 clamp\n\t" \
  "v_exp_f32_e64 v246, v246 clamp\n\t" \
  "v_mfma_f32_16x16x128_f8f6f4 v[20:23], v[136:143], v[200:207], 0 blgp:1\n\t" \
  "buffer_load_dwordx4 v252, s[40:43], s52 offen offset:1024 sc0 sc1 lds\n\t" \
  "v_exp_f32_e64 v247, v247 clamp\n\t" \
  "v_fma_f32 v244, v244, v249, s60\n\t" \
  "v_fma_f32 v245, v245, v249, s60\n\t" \
  "ds_write_b128 v252, v[104:107] offset:18432\n\t" \
  "v_mfma_f32_16x16x128_f8f6f4 v[16:19], v[136:143], v[192:199], 0 blgp:1\n\t" \
  "ds_read_b128 v[136:139], v250 offset:16640\n\t" \
  "v_fma_f32 v246, v246, v249, s60\n\t" \
  "v_fma_f32 v247, v247, v249, s60\n\t" \
  "v_rcp_f32 v244, v244\n\t" \
  "v_rcp_f32 v245, v245\n\t" \
  "v_mfma_f32_16x16x128_f8f6f4 v[32:35], v[144:151], v[192:199], 0 blgp:1\n\t" \
  "ds_read_b128 v[140:143], v250 offset:24832\n\t" \
  "v_rcp_f32 v246, v246\n\t" \
  "v_rcp_f32 v247, v247\n\t" \
  "v_mul_f32 v244, v67, v244\n\t" \
  "v_mfma_f32_16x16x128_f8f6f4 v[36:39], v[144:151], v[200:207], 0 blgp:1\n\t" \
  "buffer_load_dwordx4 v252, s[40:43], s52 offen offset:2048 sc0 sc1 lds\n\t" \
  "v_mul_f32 v245, v71, v245\n\t" \
  "ds_write_b128 v252, v[108:111] offset:19456\n\t" \
  "v_mul_f32 v246, v75, v246\n\t" \
  "v_mfma_f32_16x16x128_f8f6f4 v[40:43], v[144:151], v[208:215], 0 blgp:1\n\t" \
  "buffer_load_dwordx4 v252, s[40:43], s52 offen offset:3072 sc0 sc1 lds\n\t" \
  "v_mul_f32 v247, v79, v247\n\t" \
  "v_fma_f32 v240, v80, s58, v248\n\t" \
  "v_fma_f32 v241, v84, s58, v248\n\t" \
  "v_fma_f32 v242, v88, s58, v248\n\t" \
  "v_fma_f32 v243, v92, s58, v248\n\t" \
  "s_add_u32 s57, s80, 4390912\n\t" \
  "v_mfma_f32_16x16x128_f8f6f4 v[44:47], v[144:151], v[216:223], 0 blgp:1\n\t" \
  "ds_read_b128 v[144:147], v250 offset:16896\n\t" \
  "buffer_store_dwordx4 v[244:247], v253, s[44:47], s57 offen offset:0 nt\n\t" \
  "v_exp_f32_e64 v240, v240 clamp\n\t" \
  "v_mfma_f32_16x16x128_f8f6f4 v[60:63], v[152:159], v[216:223], 0 blgp:1\n\t" \
  "ds_read_b128 v[148:151], v250 offset:25088\n\t" \
  "v_exp_f32_e64 v241, v241 clamp\n\t" \
  "v_exp_f32_e64 v242, v242 clamp\n\t" \
  "v_exp_f32_e64 v243, v243 clamp\n\t" \
  "v_fma_f32 v240, v240, v249, s60\n\t" \
  "v_mfma_f32_16x16x128_f8f6f4 v[56:59], v[152:159], v[208:215], 0 blgp:1\n\t" \
  "v_fma_f32 v241, v241, v249, s60\n\t" \
  "v_fma_f32 v242, v242, v249, s60\n\t" \
  "v_fma_f32 v243, v243, v249, s60\n\t" \
  "v_rcp_f32 v240, v240\n\t" \
  "v_rcp_f32 v241, v241\n\t" \
  "v_mfma_f32_16x16x128_f8f6f4 v[52:55], v[152:159], v[200:207], 0 blgp:1\n\t" \
  "v_rcp_f32 v242, v242\n\t" \
  "v_rcp_f32 v243, v243\n\t" \
  "v_mul_f32 v240, v80, v240\n\t" \
  "v_mul_f32 v241, v84, v241\n\t" \
  "v_mul_f32 v242, v88, v242\n\t" \
  "v_mul_f32 v243, v92, v243\n\t" \
  "v_mfma_f32_16x16x128_f8f6f4 v[48:51], v[152:159], v[192:199], 0 blgp:1\n\t" \
  "v_fma_f32 v244, v81, s58, v248\n\t" \
  "v_fma_f32 v245, v85, s58, v248\n\t" \
  "v_fma_f32 v246, v89, s58, v248\n\t" \
  "ds_read_b128 v[152:155], v250 offset:17152\n\t" \
  "ds_read_b128 v[156:159], v250 offset:25344\n\t" \
  "s_waitcnt lgkmcnt(2)\n\t" \
  "v_mfma_f32_16x16x128_f8f6f4 a[64:67], v[128:135], v[192:199], 0 blgp:1\n\t" \
  "s_add_i32 s52, s49, 0\n\t" \
  "s_add_u32 m0, s53, 98304\n\t" \
  "v_fma_f32 v247, v93, s58, v248\n\t" \
  "s_add_u32 s57, s80, 5242880\n\t" \
  "buffer_store_dwordx4 v[240:243], v253, s[44:47], s57 offen offset:0 nt\n\t" \
  "v_mfma_f32_16x16x128_f8f6f4 a[68:71], v[128:135], v[200:207], 0 blgp:1\n\t" \
  "buffer_load_dwordx4 v252, s[40:43], s52 offen offset:0 sc0 sc1 lds\n\t" \
  "v_exp_f32_e64 v244, v244 clamp\n\t" \
  "v_exp_f32_e64 v245, v245 clamp\n\t" \
  "v_exp_f32_e64 v246, v246 clamp\n\t" \
  "v_mfma_f32_16x16x128_f8f6f4 a[72:75], v[128:135], v[208:215], 0 blgp:1\n\t" \
  "buffer_load_dwordx4 v252, s[40:43], s52 offen offset:1024 sc0 sc1 lds\n\t" \
  "v_exp_f32_e64 v247, v247 clamp\n\t" \
  "v_fma_f32 v244, v244, v249, s60\n\t" \
  "v_fma_f32 v245, v245, v249, s60\n\t" \
  "v_fma_f32 v246, v246, v249, s60\n\t" \
  "v_mfma_f32_16x16x128_f8f6f4 a[76:79], v[128:135], v[216:223], 0 blgp:1\n\t" \
  "v_fma_f32 v247, v247, v249, s60\n\t" \
  "v_rcp_f32 v244, v244\n\t" \
  "v_rcp_f32 v245, v245\n\t" \
  "v_rcp_f32 v246, v246\n\t" \
  "v_rcp_f32 v247, v247\n\t" \
  "v_mfma_f32_16x16x128_f8f6f4 a[92:95], v[136:143], v[216:223], 0 blgp:1\n\t" \
  "buffer_load_dwordx4 v252, s[40:43], s52 offen offset:2048 sc0 sc1 lds\n\t" \
  "v_mul_f32 v244, v81, v244\n\t" \
  "v_mul_f32 v245, v85, v245\n\t" \
  "v_mul_f32 v246, v89, v246\n\t" \
  "v_mul_f32 v247, v93, v247\n\t" \
  "v_mfma_f32_16x16x128_f8f6f4 a[88:91], v[136:143], v[208:215], 0 blgp:1\n\t" \
  "buffer_load_dwordx4 v252, s[40:43], s52 offen offset:3072 sc0 sc1 lds\n\t" \
  "v_fma_f32 v240, v82, s58, v248\n\t" \
  "v_fma_f32 v241, v86, s58, v248\n\t" \
  "v_fma_f32 v242, v90, s58, v248\n\t" \
  "v_fma_f32 v243, v94, s58, v248\n\t" \
  "s_add_u32 s57, s80, 5308416\n\t" \
  "v_mfma_f32_16x16x128_f8f6f4 a[84:87], v[136:143], v[200:207], 0 blgp:1\n\t" \
  "s_add_i32 s52, s49, 16384\n\t" \
  "s_add_u32 m0, s53, 114688\n\t" \
  "buffer_store_dwordx4 v[244:247], v253, s[44:47], s57 offen offset:0 nt\n\t" \
  "v_exp_f32_e64 v240, v240 clamp\n\t" \
  "v_exp_f32_e64 v241, v241 clamp\n\t" \
  "v_mfma_f32_16x16x128_f8f6f4 a[80:83], v[136:143], v[192:199], 0 blgp:1\n\t" \
  "buffer_load_dwordx4 v252, s[40:43], s52 offen offset:0 sc0 sc1 lds\n\t" \
  "v_exp_f32_e64 v242, v242 clamp\n\t" \
  "v_exp_f32_e64 v243, v243 clamp\n\t" \
  "v_fma_f32 v240, v240, v249, s60\n\t" \
  "v_mfma_f32_16x16x128_f8f6f4 a[96:99], v[144:151], v[192:199], 0 blgp:1\n\t" \
  "buffer_load_dwordx4 v252, s[40:43], s52 offen offset:1024 sc0 sc1 lds\n\t" \
  "v_fma_f32 v241, v241, v249, s60\n\t" \
  "v_fma_f32 v242, v242, v249, s60\n\t" \
  "v_fma_f32 v243, v243, v249, s60\n\t" \
  "v_rcp_f32 v240, v240\n\t" \
  "v_mfma_f32_16x16x128_f8f6f4 a[100:103], v[144:151], v[200:207], 0 blgp:1\n\t" \
  "v_rcp_f32 v241, v241\n\t" \
  "v_rcp_f32 v242, v242\n\t" \
  "v_rcp_f32 v243, v243\n\t" \
  "v_mul_f32 v240, v82, v240\n\t" \
  "v_mul_f32 v241, v86, v241\n\t" \
  "v_mfma_f32_16x16x128_f8f6f4 a[104:107], v[144:151], v[208:215], 0 blgp:1\n\t" \
  "buffer_load_dwordx4 v252, s[40:43], s52 offen offset:2048 sc0 sc1 lds\n\t" \
  "v_mul_f32 v242, v90, v242\n\t" \
  "v_mul_f32 v243, v94, v243\n\t" \
  "v_fma_f32 v244, v83, s58, v248\n\t" \
  "v_fma_f32 v245, v87, s58, v248\n\t" \
  "v_fma_f32 v246, v91, s58, v248\n\t" \
  "v_mfma_f32_16x16x128_f8f6f4 a[108:111], v[144:151], v[216:223], 0 blgp:1\n\t" \
  "buffer_load_dwordx4 v252, s[40:43], s52 offen offset:3072 sc0 sc1 lds\n\t" \
  "v_fma_f32 v247, v95, s58, v248\n\t" \
  "s_add_u32 s57, s80, 5373952\n\t" \
  "buffer_store_dwordx4 v[240:243], v253, s[44:47], s57 offen offset:0 nt\n\t" \
  "s_waitcnt lgkmcnt(0)\n\t" \
  "v_mfma_f32_16x16x128_f8f6f4 a[124:127], v[152:159], v[216:223], 0 blgp:1\n\t" \
  "v_exp_f32_e64 v244, v244 clamp\n\t" \
  "v_exp_f32_e64 v245, v245 clamp\n\t" \
  "v_exp_f32_e64 v246, v246 clamp\n\t" \
  "v_exp_f32_e64 v247, v247 clamp\n\t" \
  "v_mfma_f32_16x16x128_f8f6f4 a[120:123], v[152:159], v[208:215], 0 blgp:1\n\t" \
  "v_fma_f32 v244, v244, v249, s60\n\t" \
  "v_fma_f32 v245, v245, v249, s60\n\t" \
  "v_fma_f32 v246, v246, v249, s60\n\t" \
  "v_fma_f32 v247, v247, v249, s60\n\t" \
  "v_rcp_f32 v244, v244\n\t" \
  "v_mfma_f32_16x16x128_f8f6f4 a[116:119], v[152:159], v[200:207], 0 blgp:1\n\t" \
  "v_rcp_f32 v245, v245\n\t" \
  "v_rcp_f32 v246, v246\n\t" \
  "v_rcp_f32 v247, v247\n\t" \
  "v_mul_f32 v244, v83, v244\n\t" \
  "v_mul_f32 v245, v87, v245\n\t" \
  "v_mul_f32 v246, v91, v246\n\t" \
  "v_mfma_f32_16x16x128_f8f6f4 a[112:115], v[152:159], v[192:199], 0 blgp:1\n\t" \
  "v_mul_f32 v247, v95, v247\n\t" \
  "s_add_u32 s57, s80, 5439488\n\t" \
  "buffer_store_dwordx4 v[244:247], v253, s[44:47], s57 offen offset:0 nt\n\t" \
  "s_waitcnt vmcnt(29) lgkmcnt(0)\n\t" \
  "s_barrier\n\t" \
  "v_mfma_f32_16x16x128_f8f6f4 v[64:67], v[128:135], v[160:167], 0 blgp:1\n\t" \
  "ds_read_b128 v[192:195], v251 offset:49152\n\t" \
  "s_mov_b32 s98, s70\n\t" \
  "v_mfma_f32_16x16x128_f8f6f4 v[68:71], v[128:135], v[168:175], 0 blgp:1\n\t" \
  "ds_read_b128 v[196:199], v251 offset:57344\n\t" \
  "s_cmp_gt_u32 s55, 1\n\t" \
  "s_cselect_b32 s70, s71, s70\n\t" \
  "v_mfma_f32_16x16x128_f8f6f4 v[72:75], v[128:135], v[176:183], 0 blgp:1\n\t" \
  "ds_read_b128 v[200:203], v251 offset:49408\n\t" \
  "s_mov_b32 s71, s72\n\t" \
  "v_mfma_f32_16x16x128_f8f6f4 v[76:79], v[128:135], v[184:191], 0 blgp:1\n\t" \
  "s_mov_b32 s72, s73\n\t" \
  "ds_read_b128 v[128:131], v250 offset:32768\n\t" \
  "ds_read_b128 v[132:135], v250 offset:40960\n\t" \
  "v_mfma_f32_16x16x128_f8f6f4 v[92:95], v[136:143], v[184:191], 0 blgp:1\n\t" \
  "ds_read_b128 v[204:207], v251 offset:57600\n\t" \
  "s_mov_b32 s73, s74\n\t" \
  "v_mul_f32 v244, s58, v224\n\t" \
  "v_mfma_f32_16x16x128_f8f6f4 v[88:91], v[136:143], v[176:183], 0 blgp:1\n\t" \
  "ds_read_b128 v[208:211], v251 offset:49664\n\t" \
  "s_and_b32 s99, s70, 15\n\t" \
  "v_mul_f32 v245, s58, v228\n\t" \
  "v_mfma_f32_16x16x128_f8f6f4 v[84:87], v[136:143], v[168:175], 0 blgp:1\n\t" \
  "ds_read_b128 v[212:215], v251 offset:57856\n\t" \
  "s_mul_i32 s50, s99, 0x100800\n\t" \
  "v_mul_f32 v246, s58, v232\n\t" \
  "v_mfma_f32_16x16x128_f8f6f4 v[80:83], v[136:143], v[160:167], 0 blgp:1\n\t" \
  "s_lshl_b32 s64, s99, 24\n\t" \
  "ds_read_b128 v[136:139], v250 offset:33024\n\t" \
  "ds_read_b128 v[140:143], v250 offset:41216\n\t" \
  "v_mul_f32 v247, s58, v236\n\t" \
  "v_mfma_f32_16x16x128_f8f6f4 v[96:99], v[144:151], v[160:167], 0 blgp:1\n\t" \
  "ds_read_b128 v[216:219], v251 offset:49920\n\t" \
  "s_lshr_b32 s96, s99, 2\n\t" \
  "v_med3_f32 v244, v244, -s62, s62\n\t" \
  "v_mfma_f32_16x16x128_f8f6f4 v[100:103], v[144:151], v[168:175], 0 blgp:1\n\t" \
  "ds_read_b128 v[220:223], v251 offset:58112\n\t" \
  "s_lshl_b32 s96, s96, 9\n\t" \
  "v_med3_f32 v245, v245, -s62, s62\n\t" \
  "v_mfma_f32_16x16x128_f8f6f4 v[104:107], v[144:151], v[176:183], 0 blgp:1\n\t" \
  "s_and_b32 s99, s99, 3\n\t" \
  "v_med3_f32 v246, v246, -s62, s62\n\t" \
  "v_mfma_f32_16x16x128_f8f6f4 v[108:111], v[144:151], v[184:191], 0 blgp:1\n\t" \
  "s_lshl2_add_u32 s96, s99, s96\n\t" \
  "ds_read_b128 v[144:147], v250 offset:33280\n\t" \
  "v_med3_f32 v247, v247, -s62, s62\n\t" \
  "v_mfma_f32_16x16x128_f8f6f4 v[124:127], v[152:159], v[184:191], 0 blgp:1\n\t" \
  "s_bfe_u32 s99, s70, 0x60004\n\t" \
  "ds_read_b128 v[148:151], v250 offset:41472\n\t" \
  "v_exp_f32 v244, v244\n\t" \
  "v_mfma_f32_16x16x128_f8f6f4 v[120:123], v[152:159], v[176:183], 0 blgp:1\n\t" \
  "s_mul_i32 s51, s99, 0x100800\n\t" \
  "v_exp_f32 v245, v245\n\t" \
  "v_mfma_f32_16x16x128_f8f6f4 v[116:119], v[152:159], v[168:175], 0 blgp:1\n\t" \
  "v_exp_f32 v246, v246\n\t" \
  "v_mfma_f32_16x16x128_f8f6f4 v[112:115], v[152:159], v[160:167], 0 blgp:1\n\t" \
  "ds_read_b128 v[152:155], v250 offset:33536\n\t" \
  "ds_read_b128 v[156:159], v250 offset:41728\n\t" \
  "s_waitcnt lgkmcnt(2)\n\t" \
  "v_mfma_f32_16x16x128_f8f6f4 v[0:3], v[128:135], v[192:199], v[0:3] blgp:1\n\t" \
  "ds_read_b128 v[160:163], v251 offset:32768\n\t" \
  "s_add_u32 s51, s51, 0x1008000\n\t" \
  "v_exp_f32 v247, v247\n\t" \
  "v_mfma_f32_16x16x128_f8f6f4 v[4:7], v[128:135], v[200:207], v[4:7] blgp:1\n\t" \
  "ds_read_b128 v[164:167], v251 offset:40960\n\t" \
  "s_lshl_b32 s99, s99, 10\n\t" \
  "v_mfma_f32_16x16x128_f8f6f4 v[8:11], v[128:135], v[208:215], v[8:11] blgp:1\n\t" \
  "ds_read_b128 v[168:171], v251 offset:33024\n\t" \
  "s_add_u32 s64, s64, s99\n\t" \
  "v_fma_f32 v244, v244, s60, s60\n\t" \
  "v_mfma_f32_16x16x128_f8f6f4 v[12:15], v[128:135], v[216:223], v[12:15] blgp:1\n\t" \
  "ds_read_b128 v[172:175], v251 offset:41216\n\t" \
  "s_bfe_u32 s97, s70, 0x40006\n\t" \
  "v_fma_f32 v245, v245, s60, s60\n\t" \
  "v_mfma_f32_16x16x128_f8f6f4 v[28:31], v[136:143], v[216:223], v[28:31] blgp:1\n\t" \
  "ds_read_b128 v[176:179], v251 offset:33280\n\t" \
  "s_add_u32 s97, s97, 4\n\t" \
  "v_fma_f32 v246, v246, s60, s60\n\t" \
  "v_mfma_f32_16x16x128_f8f6f4 v[24:27], v[136:143], v[208:215], v[24:27] blgp:1\n\t" \
  "ds_read_b128 v[180:183], v251 offset:41472\n\t" \
  "s_lshl_b32 s97, s97, 9\n\t" \
  "v_fma_f32 v247, v247, s60, s60\n\t" \
  "v_mfma_f32_16x16x128_f8f6f4 v[20:23], v[136:143], v[200:207], v[20:23] blgp:1\n\t" \
  "ds_read_b128 v[184:187], v251 offset:33536\n\t" \
  "s_bfe_u32 s99, s70, 0x20004\n\t" \
  "v_rcp_f32 v244, v244\n\t" \
  "v_mfma_f32_16x16x128_f8f6f4 v[16:19], v[136:143], v[192:199], v[16:19] blgp:1\n\t" \
  "ds_read_b128 v[188:191], v251 offset:41728\n\t" \
  "s_add_i32 s52, s48, 16384\n\t" \
  "s_add_u32 m0, s53, 49152\n\t" \
  "s_lshl2_add_u32 s97, s99, s97\n\t" \
  "v_rcp_f32 v245, v245\n\t" \
  "v_mfma_f32_16x16x128_f8f6f4 v[32:35], v[144:151], v[192:199], v[32:35] blgp:1\n\t" \
  "buffer_load_dwordx4 v252, s[40:43], s52 offen offset:0 sc0 sc1 lds\n\t" \
  "s_mov_b32 s54, 3\n\t" \
  "v_rcp_f32 v246, v246\n\t" \
  "v_mfma_f32_16x16x128_f8f6f4 v[36:39], v[144:151], v[200:207], v[36:39] blgp:1\n\t" \
  "v_rcp_f32 v247, v247\n\t" \
  "v_mfma_f32_16x16x128_f8f6f4 v[40:43], v[144:151], v[208:215], v[40:43] blgp:1\n\t" \
  "buffer_load_dwordx4 v252, s[40:43], s52 offen offset:1024 sc0 sc1 lds\n\t" \
  "v_mul_f32 v244, v224, v244\n\t" \
  "v_mfma_f32_16x16x128_f8f6f4 v[44:47], v[144:151], v[216:223], v[44:47] blgp:1\n\t" \
  "v_mul_f32 v245, v228, v245\n\t" \
  "s_waitcnt lgkmcnt(0)\n\t" \
  "v_mfma_f32_16x16x128_f8f6f4 v[60:63], v[152:159], v[216:223], v[60:63] blgp:1\n\t" \
  "buffer_load_dwordx4 v252, s[40:43], s52 offen offset:2048 sc0 sc1 lds\n\t" \
  "v_mul_f32 v246, v232, v246\n\t" \
  "v_mfma_f32_16x16x128_f8f6f4 v[56:59], v[152:159], v[208:215], v[56:59] blgp:1\n\t" \
  "v_mul_f32 v247, v236, v247\n\t" \
  "v_mfma_f32_16x16x128_f8f6f4 v[52:55], v[152:159], v[200:207], v[52:55] blgp:1\n\t" \
  "buffer_load_dwordx4 v252, s[40:43], s52 offen offset:3072 sc0 sc1 lds\n\t" \
  "v_mfma_f32_16x16x128_f8f6f4 v[48:51], v[152:159], v[192:199], v[48:51] blgp:1\n\t" \
  "s_waitcnt vmcnt(22) lgkmcnt(0)\n\t" \
  "s_barrier\n\t" \
  "v_mfma_f32_16x16x128_f8f6f4 a[0:3], v[128:135], v[160:167], a[0:3] blgp:1\n\t" \
  "v_readfirstlane_b32 s87, v254\n\t" \
  "v_readlane_b32 s65, v255, s67\n\t" \
  "v_mfma_f32_16x16x128_f8f6f4 a[4:7], v[128:135], v[168:175], a[4:7] blgp:1\n\t" \
  "s_add_u32 s57, s80, 7340032\n\t" \
  "v_mfma_f32_16x16x128_f8f6f4 a[8:11], v[128:135], v[176:183], a[8:11] blgp:1\n\t" \
  "buffer_store_dwordx4 v[244:247], v253, s[44:47], s57 offen offset:0 nt\n\t" \
  "s_max_u32 s69, s69, s87\n\t" \
  "s_add_u32 s67, s67, 1\n\t" \
  "s_cmp_ge_u32 s69, s65\n\t" \
  "s_cbranch_scc0 L_w2_%=\n\t" \
  "L_ok2_%=:\n\t" \
  "v_mfma_f32_16x16x128_f8f6f4 a[12:15], v[128:135], v[184:191], a[12:15] blgp:1\n\t" \
  "ds_read_b128 v[128:131], v250 offset:49152\n\t" \
  "v_mfma_f32_16x16x128_f8f6f4 a[28:31], v[136:143], v[184:191], a[28:31] blgp:1\n\t" \
  "s_add_i32 s52, s48, 32768\n\t" \
  "s_add_u32 m0, s53, 65536\n\t" \
  "ds_read_b128 v[132:135], v250 offset:57344\n\t" \
  "v_mul_f32 v244, s58, v225\n\t" \
  "v_mfma_f32_16x16x128_f8f6f4 a[24:27], v[136:143], v[176:183], a[24:27] blgp:1\n\t" \
  "buffer_load_dwordx4 v252, s[40:43], s52 offen offset:0 sc0 sc1 lds\n\t" \
  "v_mul_f32 v245, s58, v229\n\t" \
  "v_mfma_f32_16x16x128_f8f6f4 a[20:23], v[136:143], v[168:175], a[20:23] blgp:1\n\t" \
  "buffer_load_dwordx4 v252, s[40:43], s52 offen offset:1024 sc0 sc1 lds\n\t" \
  "v_mul_f32 v246, s58, v233\n\t" \
  "v_mfma_f32_16x16x128_f8f6f4 a[16:19], v[136:143], v[160:167], a[16:19] blgp:1\n\t" \
  "ds_read_b128 v[136:139], v250 offset:49408\n\t" \
  "v_mul_f32 v247, s58, v237\n\t" \
  "v_mfma_f32_16x16x128_f8f6f4 a[32:35], v[144:151], v[160:167], a[32:35] blgp:1\n\t" \
  "ds_read_b128 v[140:143], v250 offset:57600\n\t" \
  "v_med3_f32 v244, v244, -s62, s62\n\t" \
  "v_mfma_f32_16x16x128_f8f6f4 a[36:39], v[144:151], v[168:175], a[36:39] blgp:1\n\t" \
  "buffer_load_dwordx4 v252, s[40:43], s52 offen offset:2048 sc0 sc1 lds\n\t" \
  "v_med3_f32 v245, v245, -s62, s62\n\t" \
  "v_mfma_f32_16x16x128_f8f6f4 a[40:43], v[144:151], v[176:183], a[40:43] blgp:1\n\t" \
  "buffer_load_dwordx4 v252, s[40:43], s52 offen offset:3072 sc0 sc1 lds\n\t" \
  "v_med3_f32 v246, v246, -s62, s62\n\t" \
  "v_mfma_f32_16x16x128_f8f6f4 a[44:47], v[144:151], v[184:191], a[44:47] blgp:1\n\t" \
  "ds_read_b128 v[144:147], v250 offset:49664\n\t" \
  "v_med3_f32 v247, v247, -s62, s62\n\t" \
  "v_mfma_f32_16x16x128_f8f6f4 a[60:63], v[152:159], v[184:191], a[60:63] blgp:1\n\t" \
  "ds_read_b128 v[148:151], v250 offset:57856\n\t" \
  "v_exp_f32 v244, v244\n\t" \
  "v_mfma_f32_16x16x128_f8f6f4 a[56:59], v[152:159], v[176:183], a[56:59] blgp:1\n\t" \
  "s_mov_b32 s36, 1\n\t" \
  "v_exp_f32 v245, v245\n\t" \
  "v_mfma_f32_16x16x128_f8f6f4 a[52:55], v[152:159], v[168:175], a[52:55] blgp:1\n\t" \
  "v_exp_f32 v246, v246\n\t" \
  "v_mfma_f32_16x16x128_f8f6f4 a[48:51], v[152:159], v[160:167], a[48:51] blgp:1\n\t" \
  "ds_read_b128 v[152:155], v250 offset:49920\n\t" \
  "ds_read_b128 v[156:159], v250 offset:58112\n\t" \
  "s_waitcnt lgkmcnt(2)\n\t" \
  "v_mfma_f32_16x16x128_f8f6f4 v[64:67], v[128:135], v[160:167], v[64:67] blgp:1\n\t" \
  "s_add_i32 s52, s49, 49152\n\t" \
  "s_add_u32 m0, s53, 147456\n\t" \
  "v_exp_f32 v247, v247\n\t" \
  "v_mfma_f32_16x16x128_f8f6f4 v[68:71], v[128:135], v[168:175], v[68:71] blgp:1\n\t" \
  "buffer_load_dwordx4 v252, s[40:43], s52 offen offset:0 sc0 sc1 lds\n\t" \
  "v_mfma_f32_16x16x128_f8f6f4 v[72:75], v[128:135], v[176:183], v[72:75] blgp:1\n\t" \
  "buffer_load_dwordx4 v252, s[40:43], s52 offen offset:1024 sc0 sc1 lds\n\t" \
  "v_fma_f32 v244, v244, s60, s60\n\t" \
  "v_mfma_f32_16x16x128_f8f6f4 v[76:79], v[128:135], v[184:191], v[76:79] blgp:1\n\t" \
  "buffer_load_dword v248, v252, s[92:95], s96 offen\n\t" \
  "buffer_load_dword v249, v252, s[92:95], s97 offen\n\t" \
  "v_fma_f32 v245, v245, s60, s60\n\t" \
  "v_mfma_f32_16x16x128_f8f6f4 v[92:95], v[136:143], v[184:191], v[92:95] blgp:1\n\t" \
  "buffer_load_dwordx4 v252, s[40:43], s52 offen offset:2048 sc0 sc1 lds\n\t" \
  "v_fma_f32 v246, v246, s60, s60\n\t" \
  "v_mfma_f32_16x16x128_f8f6f4 v[88:91], v[136:143], v[176:183], v[88:91] blgp:1\n\t" \
  "buffer_load_dwordx4 v252, s[40:43], s52 offen offset:3072 sc0 sc1 lds\n\t" \
  "v_fma_f32 v247, v247, s60, s60\n\t" \
  "v_mfma_f32_16x16x128_f8f6f4 v[84:87], v[136:143], v[168:175], v[84:87] blgp:1\n\t" \
  "s_add_i32 s52, s49, 32768\n\t" \
  "s_add_u32 m0, s53, 131072\n\t" \
  "v_rcp_f32 v244, v244\n\t" \
  "v_mfma_f32_16x16x128_f8f6f4 v[80:83], v[136:143], v[160:167], v[80:83] blgp:1\n\t" \
  "buffer_load_dwordx4 v252, s[40:43], s52 offen offset:0 sc0 sc1 lds\n\t" \
  "v_rcp_f32 v245, v245\n\t" \
  "v_mfma_f32_16x16x128_f8f6f4 v[96:99], v[144:151], v[160:167], v[96:99] blgp:1\n\t" \
  "buffer_load_dwordx4 v252, s[40:43], s52 offen offset:1024 sc0 sc1 lds\n\t" \
  "v_rcp_f32 v246, v246\n\t" \
  "v_mfma_f32_16x16x128_f8f6f4 v[100:103], v[144:151], v[168:175], v[100:103] blgp:1\n\t" \
  "v_rcp_f32 v247, v247\n\t" \
  "v_mfma_f32_16x16x128_f8f6f4 v[104:107], v[144:151], v[176:183], v[104:107] blgp:1\n\t" \
  "buffer_load_dwordx4 v252, s[40:43], s52 offen offset:2048 sc0 sc1 lds\n\t" \
  "v_mul_f32 v244, v225, v244\n\t" \
  "v_mfma_f32_16x16x128_f8f6f4 v[108:111], v[144:151], v[184:191], v[108:111] blgp:1\n\t" \
  "buffer_load_dwordx4 v252, s[40:43], s52 offen offset:3072 sc0 sc1 lds\n\t" \
  "v_mul_f32 v245, v229, v245\n\t" \
  "s_waitcnt lgkmcnt(0)\n\t" \
  "v_mfma_f32_16x16x128_f8f6f4 v[124:127], v[152:159], v[184:191], v[124:127] blgp:1\n\t" \
  "v_mul_f32 v246, v233, v246\n\t" \
  "v_mfma_f32_16x16x128_f8f6f4 v[120:123], v[152:159], v[176:183], v[120:123] blgp:1\n\t" \
  "v_mul_f32 v247, v237, v247\n\t" \
  "v_mfma_f32_16x16x128_f8f6f4 v[116:119], v[152:159], v[168:175], v[116:119] blgp:1\n\t" \
  "v_mfma_f32_16x16x128_f8f6f4 v[112:115], v[152:159], v[160:167], v[112:115] blgp:1\n\t" \
  "s_waitcnt vmcnt(21) lgkmcnt(0)\n\t" \
  "s_barrier\n\t" \
  "v_mfma_f32_16x16x128_f8f6f4 a[64:67], v[128:135], v[192:199], a[64:67] blgp:1\n\t" \
  "ds_read_b128 v[160:163], v251 offset:0\n\t" \
  "v_mfma_f32_16x16x128_f8f6f4 a[68:71], v[128:135], v[200:207], a[68:71] blgp:1\n\t" \
  "ds_read_b128 v[164:167], v251 offset:8192\n\t" \
  "s_add_u32 s57, s80, 7405568\n\t" \
  "v_mfma_f32_16x16x128_f8f6f4 a[72:75], v[128:135], v[208:215], a[72:75] blgp:1\n\t" \
  "buffer_store_dwordx4 v[244:247], v253, s[44:47], s57 offen offset:0 nt\n\t" \
  "ds_read_b128 v[168:171], v251 offset:256\n\t" \
  "v_mfma_f32_16x16x128_f8f6f4 a[76:79], v[128:135], v[216:223], a[76:79] blgp:1\n\t" \
  "ds_read_b128 v[128:131], v250 offset:0\n\t" \
  "ds_read_b128 v[132:135], v250 offset:8192\n\t" \
  "v_mfma_f32_16x16x128_f8f6f4 a[92:95], v[136:143], v[216:223], a[92:95] blgp:1\n\t" \
  "ds_read_b128 v[172:175], v251 offset:8448\n\t" \
  "v_mul_f32 v244, s58, v226\n\t" \
  "v_mfma_f32_16x16x128_f8f6f4 a[88:91], v[136:143], v[208:215], a[88:91] blgp:1\n\t" \
  "ds_read_b128 v[176:179], v251 offset:512\n\t" \
  "v_mul_f32 v245, s58, v230\n\t" \
  "v_mfma_f32_16x16x128_f8f6f4 a[84:87], v[136:143], v[200:207], a[84:87] blgp:1\n\t" \
  "ds_read_b128 v[180:183], v251 offset:8704\n\t" \
  "v_mul_f32 v246, s58, v234\n\t" \
  "v_mfma_f32_16x16x128_f8f6f4 a[80:83], v[136:143], v[192:199], a[80:83] blgp:1\n\t" \
  "ds_read_b128 v[136:139], v250 offset:256\n\t" \
  "ds_read_b128 v[140:143], v250 offset:8448\n\t" \
  "v_mul_f32 v247, s58, v238\n\t" \
  "v_mfma_f32_16x16x128_f8f6f4 a[96:99], v[144:151], v[192:199], a[96:99] blgp:1\n\t" \
  "ds_read_b128 v[184:187], v251 offset:768\n\t" \
  "v_med3_f32 v244, v244, -s62, s62\n\t" \
  "v_mfma_f32_16x16x128_f8f6f4 a[100:103], v[144:151], v[200:207], a[100:103] blgp:1\n\t" \
  "ds_read_b128 v[188:191], v251 offset:8960\n\t" \
  "v_med3_f32 v245, v245, -s62, s62\n\t" \
  "v_mfma_f32_16x16x128_f8f6f4 a[104:107], v[144:151], v[208:215], a[104:107] blgp:1\n\t" \
  "s_add_u32 s48, s48, 0x10000\n\t" \
  "s_add_u32 s49, s49, 0x10000\n\t" \
  "v_med3_f32 v246, v246, -s62, s62\n\t" \
  "v_mfma_f32_16x16x128_f8f6f4 a[108:111], v[144:151], v[216:223], a[108:111] blgp:1\n\t" \
  "ds_read_b128 v[144:147], v250 offset:512\n\t" \
  "v_med3_f32 v247, v247, -s62, s62\n\t" \
  "v_mfma_f32_16x16x128_f8f6f4 a[124:127], v[152:159], v[216:223], a[124:127] blgp:1\n\t" \
  "ds_read_b128 v[148:151], v250 offset:8704\n\t" \
  "v_exp_f32 v244, v244\n\t" \
  "v_mfma_f32_16x16x128_f8f6f4 a[120:123], v[152:159], v[208:215], a[120:123] blgp:1\n\t" \
  "v_exp_f32 v245, v245\n\t" \
  "v_mfma_f32_16x16x128_f8f6f4 a[116:119], v[152:159], v[200:207], a[116:119] blgp:1\n\t" \
  "v_exp_f32 v246, v246\n\t" \
  "v_mfma_f32_16x16x128_f8f6f4 a[112:115], v[152:159], v[192:199], a[112:115] blgp:1\n\t" \
  "ds_read_b128 v[152:155], v250 offset:768\n\t" \
  "ds_read_b128 v[156:159], v250 offset:8960\n\t" \
  "s_waitcnt lgkmcnt(2)\n\t" \
  "v_mfma_f32_16x16x128_f8f6f4 a[0:3], v[128:135], v[160:167], a[0:3] blgp:1\n\t" \
  "ds_read_b128 v[192:195], v251 offset:16384\n\t" \
  "v_exp_f32 v247, v247\n\t" \
  "v_mfma_f32_16x16x128_f8f6f4 a[4:7], v[128:135], v[168:175], a[4:7] blgp:1\n\t" \
  "ds_read_b128 v[196:199], v251 offset:24576\n\t" \
  "v_mfma_f32_16x16x128_f8f6f4 a[8:11], v[128:135], v[176:183], a[8:11] blgp:1\n\t" \
  "ds_read_b128 v[200:203], v251 offset:16640\n\t" \
  "v_fma_f32 v244, v244, s60, s60\n\t" \
  "v_mfma_f32_16x16x128_f8f6f4 a[12:15], v[128:135], v[184:191], a[12:15] blgp:1\n\t" \
  "ds_read_b128 v[204:207], v251 offset:24832\n\t" \
  "v_fma_f32 v245, v245, s60, s60\n\t" \
  "v_mfma_f32_16x16x128_f8f6f4 a[28:31], v[136:143], v[184:191], a[28:31] blgp:1\n\t" \
  "ds_read_b128 v[208:211], v251 offset:16896\n\t" \
  "v_fma_f32 v246, v246, s60, s60\n\t" \
  "v_mfma_f32_16x16x128_f8f6f4 a[24:27], v[136:143], v[176:183], a[24:27] blgp:1\n\t" \
  "ds_read_b128 v[212:215], v251 offset:25088\n\t" \
  "v_fma_f32 v247, v247, s60, s60\n\t" \
  "v_mfma_f32_16x16x128_f8f6f4 a[20:23], v[136:143], v[168:175], a[20:23] blgp:1\n\t" \
  "ds_read_b128 v[216:219], v251 offset:17152\n\t" \
  "v_rcp_f32 v244, v244\n\t" \
  "v_mfma_f32_16x16x128_f8f6f4 a[16:19], v[136:143], v[160:167], a[16:19] blgp:1\n\t" \
  "ds_read_b128 v[220:223], v251 offset:25344\n\t" \
  "s_add_i32 s52, s48, -16384\n\t" \
  "s_add_u32 m0, s53, 81920\n\t" \
  "v_rcp_f32 v245, v245\n\t" \
  "v_mfma_f32_16x16x128_f8f6f4 a[32:35], v[144:151], v[160:167], a[32:35] blgp:1\n\t" \
  "buffer_load_dwordx4 v252, s[40:43], s52 offen offset:0 sc0 sc1 lds\n\t" \
  "v_rcp_f32 v246, v246\n\t" \
  "v_mfma_f32_16x16x128_f8f6f4 a[36:39], v[144:151], v[168:175], a[36:39] blgp:1\n\t" \
  "v_rcp_f32 v247, v247\n\t" \
  "v_mfma_f32_16x16x128_f8f6f4 a[40:43], v[144:151], v[176:183], a[40:43] blgp:1\n\t" \
  "buffer_load_dwordx4 v252, s[40:43], s52 offen offset:1024 sc0 sc1 lds\n\t" \
  "v_mul_f32 v244, v226, v244\n\t" \
  "v_mfma_f32_16x16x128_f8f6f4 a[44:47], v[144:151], v[184:191], a[44:47] blgp:1\n\t" \
  "v_mul_f32 v245, v230, v245\n\t" \
  "s_waitcnt lgkmcnt(0)\n\t" \
  "v_mfma_f32_16x16x128_f8f6f4 a[60:63], v[152:159], v[184:191], a[60:63] blgp:1\n\t" \
  "buffer_load_dwordx4 v252, s[40:43], s52 offen offset:2048 sc0 sc1 lds\n\t" \
  "v_mul_f32 v246, v234, v246\n\t" \
  "v_mfma_f32_16x16x128_f8f6f4 a[56:59], v[152:159], v[176:183], a[56:59] blgp:1\n\t" \
  "v_mul_f32 v247, v238, v247\n\t" \
  "v_mfma_f32_16x16x128_f8f6f4 a[52:55], v[152:159], v[168:175], a[52:55] blgp:1\n\t" \
  "buffer_load_dwordx4 v252, s[40:43], s52 offen offset:3072 sc0 sc1 lds\n\t" \
  "buffer_load_dword v254, off, s[88:91], s81 sc1\n\t" \
  "v_mfma_f32_16x16x128_f8f6f4 a[48:51], v[152:159], v[160:167], a[48:51] blgp:1\n\t" \
  "s_waitcnt vmcnt(21) lgkmcnt(0)\n\t" \
  "s_barrier\n\t" \
  "v_mfma_f32_16x16x128_f8f6f4 v[0:3], v[128:135], v[192:199], v[0:3] blgp:1\n\t" \
  "v_readlane_b32 s65, v255, s67\n\t" \
  "v_mfma_f32_16x16x128_f8f6f4 v[4:7], v[128:135], v[200:207], v[4:7] blgp:1\n\t" \
  "s_add_u32 s57, s80, 7471104\n\t" \
  "v_mfma_f32_16x16x128_f8f6f4 v[8:11], v[128:135], v[208:215], v[8:11] blgp:1\n\t" \
  "buffer_store_dwordx4 v[244:247], v253, s[44:47], s57 offen offset:0 nt\n\t" \
  "s_add_u32 s67, s67, 1\n\t" \
  "s_cmp_ge_u32 s69, s65\n\t" \
  "s_cbranch_scc0 L_w3_%=\n\t" \
  "L_ok3_%=:\n\t" \
  "v_mfma_f32_16x16x128_f8f6f4 v[12:15], v[128:135], v[216:223], v[12:15] blgp:1\n\t" \
  "ds_read_b128 v[128:131], v250 offset:16384\n\t" \
  "v_mfma_f32_16x16x128_f8f6f4 v[28:31], v[136:143], v[216:223], v[28:31] blgp:1\n\t" \
  "s_add_i32 s52, s48, 0\n\t" \
  "s_add_u32 m0, s53, 32768\n\t" \
  "ds_read_b128 v[132:135], v250 offset:24576\n\t" \
  "v_mul_f32 v244, s58, v227\n\t" \
  "v_mfma_f32_16x16x128_f8f6f4 v[24:27], v[136:143], v[208:215], v[24:27] blgp:1\n\t" \
  "buffer_load_dwordx4 v252, s[40:43], s52 offen offset:0 sc0 sc1 lds\n\t" \
  "v_mul_f32 v245, s58, v231\n\t" \
  "v_mfma_f32_16x16x128_f8f6f4 v[20:23], v[136:143], v[200:207], v[20:23] blgp:1\n\t" \
  "buffer_load_dwordx4 v252, s[40:43], s52 offen offset:1024 sc0 sc1 lds\n\t" \
  "v_mul_f32 v246, s58, v235\n\t" \
  "v_mfma_f32_16x16x128_f8f6f4 v[16:19], v[136:143], v[192:199], v[16:19] blgp:1\n\t" \
  "ds_read_b128 v[136:139], v250 offset:16640\n\t" \
  "v_mul_f32 v247, s58, v239\n\t" \
  "v_mfma_f32_16x16x128_f8f6f4 v[32:35], v[144:151], v[192:199], v[32:35] blgp:1\n\t" \
  "ds_read_b128 v[140:143], v250 offset:24832\n\t" \
  "v_med3_f32 v244, v244, -s62, s62\n\t" \
  "v_mfma_f32_16x16x128_f8f6f4 v[36:39], v[144:151], v[200:207], v[36:39] blgp:1\n\t" \
  "buffer_load_dwordx4 v252, s[40:43], s52 offen offset:2048 sc0 sc1 lds\n\t" \
  "v_med3_f32 v245, v245, -s62, s62\n\t" \
  "v_mfma_f32_16x16x128_f8f6f4 v[40:43], v[144:151], v[208:215], v[40:43] blgp:1\n\t" \
  "buffer_load_dwordx4 v252, s[40:43], s52 offen offset:3072 sc0 sc1 lds\n\t" \
  "v_med3_f32 v246, v246, -s62, s62\n\t" \
  "v_mfma_f32_16x16x128_f8f6f4 v[44:47], v[144:151], v[216:223], v[44:47] blgp:1\n\t" \
  "ds_read_b128 v[144:147], v250 offset:16896\n\t" \
  "v_med3_f32 v247, v247, -s62, s62\n\t" \
  "v_mfma_f32_16x16x128_f8f6f4 v[60:63], v[152:159], v[216:223], v[60:63] blgp:1\n\t" \
  "ds_read_b128 v[148:151], v250 offset:25088\n\t" \
  "v_exp_f32 v244, v244\n\t" \
  "v_mfma_f32_16x16x128_f8f6f4 v[56:59], v[152:159], v[208:215], v[56:59] blgp:1\n\t" \
  "v_exp_f32 v245, v245\n\t" \
  "v_mfma_f32_16x16x128_f8f6f4 v[52:55], v[152:159], v[200:207], v[52:55] blgp:1\n\t" \
  "v_exp_f32 v246, v246\n\t" \
  "v_mfma_f32_16x16x128_f8f6f4 v[48:51], v[152:159], v[192:199], v[48:51] blgp:1\n\t" \
  "ds_read_b128 v[152:155], v250 offset:17152\n\t" \
  "ds_read_b128 v[156:159], v250 offset:25344\n\t" \
  "s_waitcnt lgkmcnt(2)\n\t" \
  "v_mfma_f32_16x16x128_f8f6f4 a[64:67], v[128:135], v[192:199], a[64:67] blgp:1\n\t" \
  "s_add_i32 s52, s49, 0\n\t" \
  "s_add_u32 m0, s53, 98304\n\t" \
  "v_exp_f32 v247, v247\n\t" \
  "v_mfma_f32_16x16x128_f8f6f4 a[68:71], v[128:135], v[200:207], a[68:71] blgp:1\n\t" \
  "buffer_load_dwordx4 v252, s[40:43], s52 offen offset:0 sc0 sc1 lds\n\t" \
  "v_mfma_f32_16x16x128_f8f6f4 a[72:75], v[128:135], v[208:215], a[72:75] blgp:1\n\t" \
  "buffer_load_dwordx4 v252, s[40:43], s52 offen offset:1024 sc0 sc1 lds\n\t" \
  "v_fma_f32 v244, v244, s60, s60\n\t" \
  "v_mfma_f32_16x16x128_f8f6f4 a[76:79], v[128:135], v[216:223], a[76:79] blgp:1\n\t" \
  "v_fma_f32 v245, v245, s60, s60\n\t" \
  "v_mfma_f32_16x16x128_f8f6f4 a[92:95], v[136:143], v[216:223], a[92:95] blgp:1\n\t" \
  "buffer_load_dwordx4 v252, s[40:43], s52 offen offset:2048 sc0 sc1 lds\n\t" \
  "v_fma_f32 v246, v246, s60, s60\n\t" \
  "v_mfma_f32_16x16x128_f8f6f4 a[88:91], v[136:143], v[208:215], a[88:91] blgp:1\n\t" \
  "buffer_load_dwordx4 v252, s[40:43], s52 offen offset:3072 sc0 sc1 lds\n\t" \
  "v_fma_f32 v247, v247, s60, s60\n\t" \
  "v_mfma_f32_16x16x128_f8f6f4 a[84:87], v[136:143], v[200:207], a[84:87] blgp:1\n\t" \
  "s_add_i32 s52, s49, 16384\n\t" \
  "s_add_u32 m0, s53, 114688\n\t" \
  "v_rcp_f32 v244, v244\n\t" \
  "v_mfma_f32_16x16x128_f8f6f4 a[80:83], v[136:143], v[192:199], a[80:83] blgp:1\n\t" \
  "buffer_load_dwordx4 v252, s[40:43], s52 offen offset:0 sc0 sc1 lds\n\t" \
  "v_rcp_f32 v245, v245\n\t" \
  "v_mfma_f32_16x16x128_f8f6f4 a[96:99], v[144:151], v[192:199], a[96:99] blgp:1\n\t" \
  "buffer_load_dwordx4 v252, s[40:43], s52 offen offset:1024 sc0 sc1 lds\n\t" \
  "v_rcp_f32 v246, v246\n\t" \
  "v_mfma_f32_16x16x128_f8f6f4 a[100:103], v[144:151], v[200:207], a[100:103] blgp:1\n\t" \
  "v_rcp_f32 v247, v247\n\t" \
  "v_mfma_f32_16x16x128_f8f6f4 a[104:107], v[144:151], v[208:215], a[104:107] blgp:1\n\t" \
  "buffer_load_dwordx4 v252, s[40:43], s52 offen offset:2048 sc0 sc1 lds\n\t" \
  "v_mul_f32 v244, v227, v244\n\t" \
  "v_mfma_f32_16x16x128_f8f6f4 a[108:111], v[144:151], v[216:223], a[108:111] blgp:1\n\t" \
  "buffer_load_dwordx4 v252, s[40:43], s52 offen offset:3072 sc0 sc1 lds\n\t" \
  "v_mul_f32 v245, v231, v245\n\t" \
  "s_waitcnt lgkmcnt(0)\n\t" \
  "v_mfma_f32_16x16x128_f8f6f4 a[124:127], v[152:159], v[216:223], a[124:127] blgp:1\n\t" \
  "v_mul_f32 v246, v235, v246\n\t" \
  "v_mfma_f32_16x16x128_f8f6f4 a[120:123], v[152:159], v[208:215], a[120:123] blgp:1\n\t" \
  "v_mul_f32 v247, v239, v247\n\t" \
  "v_mfma_f32_16x16x128_f8f6f4 a[116:119], v[152:159], v[200:207], a[116:119] blgp:1\n\t" \
  "v_mfma_f32_16x16x128_f8f6f4 a[112:115], v[152:159], v[192:199], a[112:115] blgp:1\n\t" \
  "s_waitcnt vmcnt(19) lgkmcnt(0)\n\t" \
  "s_barrier\n\t" \
  "v_mfma_f32_16x16x128_f8f6f4 v[64:67], v[128:135], v[160:167], v[64:67] blgp:1\n\t" \
  "ds_read_b128 v[192:195], v251 offset:49152\n\t" \
  "v_accvgpr_read_b32 v240, a128\n\t" \
  "v_mfma_f32_16x16x128_f8f6f4 v[68:71], v[128:135], v[168:175], v[68:71] blgp:1\n\t" \
  "ds_read_b128 v[196:199], v251 offset:57344\n\t" \
  "s_add_u32 s57, s80, 7536640\n\t" \
  "v_accvgpr_read_b32 v241, a132\n\t" \
  "v_mfma_f32_16x16x128_f8f6f4 v[72:75], v[128:135], v[176:183], v[72:75] blgp:1\n\t" \
  "buffer_store_dwordx4 v[244:247], v253, s[44:47], s57 offen offset:0 nt\n\t" \
  "ds_read_b128 v[200:203], v251 offset:49408\n\t" \
  "v_accvgpr_read_b32 v242, a136\n\t" \
  "v_mfma_f32_16x16x128_f8f6f4 v[76:79], v[128:135], v[184:191], v[76:79] blgp:1\n\t" \
  "ds_read_b128 v[128:131], v250 offset:32768\n\t" \
  "ds_read_b128 v[132:135], v250 offset:40960\n\t" \
  "v_accvgpr_read_b32 v243, a140\n\t" \
  "v_mfma_f32_16x16x128_f8f6f4 v[92:95], v[136:143], v[184:191], v[92:95] blgp:1\n\t" \
  "ds_read_b128 v[204:207], v251 offset:57600\n\t" \
  "v_mul_f32 v244, s58, v240\n\t" \
  "v_mfma_f32_16x16x128_f8f6f4 v[88:91], v[136:143], v[176:183], v[88:91] blgp:1\n\t" \
  "ds_read_b128 v[208:211], v251 offset:49664\n\t" \
  "v_mul_f32 v245, s58, v241\n\t" \
  "v_mfma_f32_16x16x128_f8f6f4 v[84:87], v[136:143], v[168:175], v[84:87] blgp:1\n\t" \
  "ds_read_b128 v[212:215], v251 offset:57856\n\t" \
  "v_mul_f32 v246, s58, v242\n\t" \
  "v_mfma_f32_16x16x128_f8f6f4 v[80:83], v[136:143], v[160:167], v[80:83] blgp:1\n\t" \
  "ds_read_b128 v[136:139], v250 offset:33024\n\t" \
  "ds_read_b128 v[140:143], v250 offset:41216\n\t" \
  "v_mul_f32 v247, s58, v243\n\t" \
  "v_mfma_f32_16x16x128_f8f6f4 v[96:99], v[144:151], v[160:167], v[96:99] blgp:1\n\t" \
  "ds_read_b128 v[216:219], v251 offset:49920\n\t" \
  "v_med3_f32 v244, v244, -s62, s62\n\t" \
  "v_mfma_f32_16x16x128_f8f6f4 v[100:103], v[144:151], v[168:175], v[100:103] blgp:1\n\t" \
  "ds_read_b128 v[220:223], v251 offset:58112\n\t" \
  "v_med3_f32 v245, v245, -s62, s62\n\t" \
  "v_mfma_f32_16x16x128_f8f6f4 v[104:107], v[144:151], v[176:183], v[104:107] blgp:1\n\t" \
  "v_med3_f32 v246, v246, -s62, s62\n\t" \
  "v_mfma_f32_16x16x128_f8f6f4 v[108:111], v[144:151], v[184:191], v[108:111] blgp:1\n\t" \
  "ds_read_b128 v[144:147], v250 offset:33280\n\t" \
  "v_med3_f32 v247, v247, -s62, s62\n\t" \
  "v_mfma_f32_16x16x128_f8f6f4 v[124:127], v[152:159], v[184:191], v[124:127] blgp:1\n\t" \
  "ds_read_b128 v[148:151], v250 offset:41472\n\t" \
  "v_exp_f32 v244, v244\n\t" \
  "v_mfma_f32_16x16x128_f8f6f4 v[120:123], v[152:159], v[176:183], v[120:123] blgp:1\n\t" \
  "v_exp_f32 v245, v245\n\t" \
  "v_mfma_f32_16x16x128_f8f6f4 v[116:119], v[152:159], v[168:175], v[116:119] blgp:1\n\t" \
  "v_exp_f32 v246, v246\n\t" \
  "v_mfma_f32_16x16x128_f8f6f4 v[112:115], v[152:159], v[160:167], v[112:115] blgp:1\n\t" \
  "ds_read_b128 v[152:155], v250 offset:33536\n\t" \
  "ds_read_b128 v[156:159], v250 offset:41728\n\t" \
  "s_waitcnt lgkmcnt(2)\n\t" \
  "v_mfma_f32_16x16x128_f8f6f4 v[0:3], v[128:135], v[192:199], v[0:3] blgp:1\n\t" \
  "ds_read_b128 v[160:163], v251 offset:32768\n\t" \
  "v_exp_f32 v247, v247\n\t" \
  "v_mfma_f32_16x16x128_f8f6f4 v[4:7], v[128:135], v[200:207], v[4:7] blgp:1\n\t" \
  "ds_read_b128 v[164:167], v251 offset:40960\n\t" \
  "v_mfma_f32_16x16x128_f8f6f4 v[8:11], v[128:135], v[208:215], v[8:11] blgp:1\n\t" \
  "ds_read_b128 v[168:171], v251 offset:33024\n\t" \
  "v_fma_f32 v244, v244, s60, s60\n\t" \
  "v_mfma_f32_16x16x128_f8f6f4 v[12:15], v[128:135], v[216:223], v[12:15] blgp:1\n\t" \
  "ds_read_b128 v[172:175], v251 offset:41216\n\t" \
  "v_fma_f32 v245, v245, s60, s60\n\t" \
  "v_mfma_f32_16x16x128_f8f6f4 v[28:31], v[136:143], v[216:223], v[28:31] blgp:1\n\t" \
  "ds_read_b128 v[176:179], v251 offset:33280\n\t" \
  "v_fma_f32 v246, v246, s60, s60\n\t" \
  "v_mfma_f32_16x16x128_f8f6f4 v[24:27], v[136:143], v[208:215], v[24:27] blgp:1\n\t" \
  "ds_read_b128 v[180:183], v251 offset:41472\n\t" \
  "v_fma_f32 v247, v247, s60, s60\n\t" \
  "v_mfma_f32_16x16x128_f8f6f4 v[20:23], v[136:143], v[200:207], v[20:23] blgp:1\n\t" \
  "ds_read_b128 v[184:187], v251 offset:33536\n\t" \
  "v_rcp_f32 v244, v244\n\t" \
  "v_mfma_f32_16x16x128_f8f6f4 v[16:19], v[136:143], v[192:199], v[16:19] blgp:1\n\t" \
  "ds_read_b128 v[188:191], v251 offset:41728\n\t" \
  "s_add_i32 s52, s48, 16384\n\t" \
  "s_add_u32 m0, s53, 49152\n\t" \
  "v_rcp_f32 v245, v245\n\t" \
  "v_mfma_f32_16x16x128_f8f6f4 v[32:35], v[144:151], v[192:199], v[32:35] blgp:1\n\t" \
  "buffer_load_dwordx4 v252, s[40:43], s52 offen offset:0 sc0 sc1 lds\n\t" \
  "v_rcp_f32 v246, v246\n\t" \
  "v_mfma_f32_16x16x128_f8f6f4 v[36:39], v[144:151], v[200:207], v[36:39] blgp:1\n\t" \
  "v_rcp_f32 v247, v247\n\t" \
  "v_mfma_f32_16x16x128_f8f6f4 v[40:43], v[144:151], v[208:215], v[40:43] blgp:1\n\t" \
  "buffer_load_dwordx4 v252, s[40:43], s52 offen offset:1024 sc0 sc1 lds\n\t" \
  "v_mul_f32 v244, v240, v244\n\t" \
  "v_mfma_f32_16x16x128_f8f6f4 v[44:47], v[144:151], v[216:223], v[44:47] blgp:1\n\t" \
  "v_mul_f32 v245, v241, v245\n\t" \
  "s_waitcnt lgkmcnt(0)\n\t" \
  "v_mfma_f32_16x16x128_f8f6f4 v[60:63], v[152:159], v[216:223], v[60:63] blgp:1\n\t" \
  "buffer_load_dwordx4 v252, s[40:43], s52 offen offset:2048 sc0 sc1 lds\n\t" \
  "v_mul_f32 v246, v242, v246\n\t" \
  "v_mfma_f32_16x16x128_f8f6f4 v[56:59], v[152:159], v[208:215], v[56:59] blgp:1\n\t" \
  "v_mul_f32 v247, v243, v247\n\t" \
  "v_mfma_f32_16x16x128_f8f6f4 v[52:55], v[152:159], v[200:207], v[52:55] blgp:1\n\t" \
  "buffer_load_dwordx4 v252, s[40:43], s52 offen offset:3072 sc0 sc1 lds\n\t" \
  "v_mfma_f32_16x16x128_f8f6f4 v[48:51], v[152:159], v[192:199], v[48:51] blgp:1\n\t" \
  "s_waitcnt vmcnt(18) lgkmcnt(0)\n\t" \
  "s_barrier\n\t" \
  "v_mfma_f32_16x16x128_f8f6f4 a[0:3], v[128:135], v[160:167], a[0:3] blgp:1\n\t" \
  "v_readfirstlane_b32 s87, v254\n\t" \
  "v_readlane_b32 s65, v255, s67\n\t" \
  "v_accvgpr_read_b32 v240, a129\n\t" \
  "v_mfma_f32_16x16x128_f8f6f4 a[4:7], v[128:135], v[168:175], a[4:7] blgp:1\n\t" \
  "s_add_u32 s57, s80, 0\n\t" \
  "v_accvgpr_read_b32 v241, a133\n\t" \
  "v_mfma_f32_16x16x128_f8f6f4 a[8:11], v[128:135], v[176:183], a[8:11] blgp:1\n\t" \
  "buffer_store_dwordx4 v[244:247], v253, s[44:47], s57 offen offset:0 nt\n\t" \
  "s_max_u32 s69, s69, s87\n\t" \
  "s_add_u32 s67, s67, 1\n\t" \
  "s_cmp_ge_u32 s69, s65\n\t" \
  "s_cbranch_scc0 L_w4_%=\n\t" \
  "L_ok4_%=:\n\t" \
  "v_accvgpr_read_b32 v242, a137\n\t" \
  "v_mfma_f32_16x16x128_f8f6f4 a[12:15], v[128:135], v[184:191], a[12:15] blgp:1\n\t" \
  "ds_read_b128 v[128:131], v250 offset:49152\n\t" \
  "v_accvgpr_read_b32 v243, a141\n\t" \
  "v_mfma_f32_16x16x128_f8f6f4 a[28:31], v[136:143], v[184:191], a[28:31] blgp:1\n\t" \
  "s_add_i32 s52, s48, 32768\n\t" \
  "s_add_u32 m0, s53, 65536\n\t" \
  "ds_read_b128 v[132:135], v250 offset:57344\n\t" \
  "v_mul_f32 v244, s58, v240\n\t" \
  "v_mfma_f32_16x16x128_f8f6f4 a[24:27], v[136:143], v[176:183], a[24:27] blgp:1\n\t" \
  "buffer_load_dwordx4 v252, s[40:43], s52 offen offset:0 sc0 sc1 lds\n\t" \
  "v_mul_f32 v245, s58, v241\n\t" \
  "v_mfma_f32_16x16x128_f8f6f4 a[20:23], v[136:143], v[168:175], a[20:23] blgp:1\n\t" \
  "buffer_load_dwordx4 v252, s[40:43], s52 offen offset:1024 sc0 sc1 lds\n\t" \
  "v_mul_f32 v246, s58, v242\n\t" \
  "v_mfma_f32_16x16x128_f8f6f4 a[16:19], v[136:143], v[160:167], a[16:19] blgp:1\n\t" \
  "ds_read_b128 v[136:139], v250 offset:49408\n\t" \
  "v_mul_f32 v247, s58, v243\n\t" \
  "v_mfma_f32_16x16x128_f8f6f4 a[32:35], v[144:151], v[160:167], a[32:35] blgp:1\n\t" \
  "ds_read_b128 v[140:143], v250 offset:57600\n\t" \
  "v_med3_f32 v244, v244, -s62, s62\n\t" \
  "v_mfma_f32_16x16x128_f8f6f4 a[36:39], v[144:151], v[168:175], a[36:39] blgp:1\n\t" \
  "buffer_load_dwordx4 v252, s[40:43], s52 offen offset:2048 sc0 sc1 lds\n\t" \
  "v_med3_f32 v245, v245, -s62, s62\n\t" \
  "v_mfma_f32_16x16x128_f8f6f4 a[40:43], v[144:151], v[176:183], a[40:43] blgp:1\n\t" \
  "buffer_load_dwordx4 v252, s[40:43], s52 offen offset:3072 sc0 sc1 lds\n\t" \
  "v_med3_f32 v246, v246, -s62, s62\n\t" \
  "v_mfma_f32_16x16x128_f8f6f4 a[44:47], v[144:151], v[184:191], a[44:47] blgp:1\n\t" \
  "ds_read_b128 v[144:147], v250 offset:49664\n\t" \
  "v_med3_f32 v247, v247, -s62, s62\n\t" \
  "v_mfma_f32_16x16x128_f8f6f4 a[60:63], v[152:159], v[184:191], a[60:63] blgp:1\n\t" \
  "ds_read_b128 v[148:151], v250 offset:57856\n\t" \
  "v_exp_f32 v244, v244\n\t" \
  "v_mfma_f32_16x16x128_f8f6f4 a[56:59], v[152:159], v[176:183], a[56:59] blgp:1\n\t" \
  "v_exp_f32 v245, v245\n\t" \
  "v_mfma_f32_16x16x128_f8f6f4 a[52:55], v[152:159], v[168:175], a[52:55] blgp:1\n\t" \
  "v_exp_f32 v246, v246\n\t" \
  "v_mfma_f32_16x16x128_f8f6f4 a[48:51], v[152:159], v[160:167], a[48:51] blgp:1\n\t" \
  "ds_read_b128 v[152:155], v250 offset:49920\n\t" \
  "ds_read_b128 v[156:159], v250 offset:58112\n\t" \
  "s_waitcnt lgkmcnt(2)\n\t" \
  "v_mfma_f32_16x16x128_f8f6f4 v[64:67], v[128:135], v[160:167], v[64:67] blgp:1\n\t" \
  "s_add_i32 s52, s49, 49152\n\t" \
  "s_add_u32 m0, s53, 147456\n\t" \
  "v_exp_f32 v247, v247\n\t" \
  "v_mfma_f32_16x16x128_f8f6f4 v[68:71], v[128:135], v[168:175], v[68:71] blgp:1\n\t" \
  "buffer_load_dwordx4 v252, s[40:43], s52 offen offset:0 sc0 sc1 lds\n\t" \
  "v_mfma_f32_16x16x128_f8f6f4 v[72:75], v[128:135], v[176:183], v[72:75] blgp:1\n\t" \
  "buffer_load_dwordx4 v252, s[40:43], s52 offen offset:1024 sc0 sc1 lds\n\t" \
  "v_fma_f32 v244, v244, s60, s60\n\t" \
  "v_mfma_f32_16x16x128_f8f6f4 v[76:79], v[128:135], v[184:191], v[76:79] blgp:1\n\t" \
  "v_fma_f32 v245, v245, s60, s60\n\t" \
  "v_mfma_f32_16x16x128_f8f6f4 v[92:95], v[136:143], v[184:191], v[92:95] blgp:1\n\t" \
  "buffer_load_dwordx4 v252, s[40:43], s52 offen offset:2048 sc0 sc1 lds\n\t" \
  "v_fma_f32 v246, v246, s60, s60\n\t" \
  "v_mfma_f32_16x16x128_f8f6f4 v[88:91], v[136:143], v[176:183], v[88:91] blgp:1\n\t" \
  "buffer_load_dwordx4 v252, s[40:43], s52 offen offset:3072 sc0 sc1 lds\n\t" \
  "v_fma_f32 v247, v247, s60, s60\n\t" \
  "v_mfma_f32_16x16x128_f8f6f4 v[84:87], v[136:143], v[168:175], v[84:87] blgp:1\n\t" \
  "s_add_i32 s52, s49, 32768\n\t" \
  "s_add_u32 m0, s53, 131072\n\t" \
  "v_rcp_f32 v244, v244\n\t" \
  "v_mfma_f32_16x16x128_f8f6f4 v[80:83], v[136:143], v[160:167], v[80:83] blgp:1\n\t" \
  "buffer_load_dwordx4 v252, s[40:43], s52 offen offset:0 sc0 sc1 lds\n\t" \
  "v_rcp_f32 v245, v245\n\t" \
  "v_mfma_f32_16x16x128_f8f6f4 v[96:99], v[144:151], v[160:167], v[96:99] blgp:1\n\t" \
  "buffer_load_dwordx4 v252, s[40:43], s52 offen offset:1024 sc0 sc1 lds\n\t" \
  "v_rcp_f32 v246, v246\n\t" \
  "v_mfma_f32_16x16x128_f8f6f4 v[100:103], v[144:151], v[168:175], v[100:103] blgp:1\n\t" \
  "v_rcp_f32 v247, v247\n\t" \
  "v_mfma_f32_16x16x128_f8f6f4 v[104:107], v[144:151], v[176:183], v[104:107] blgp:1\n\t" \
  "buffer_load_dwordx4 v252, s[40:43], s52 offen offset:2048 sc0 sc1 lds\n\t" \
  "v_mul_f32 v244, v240, v244\n\t" \
  "v_mfma_f32_16x16x128_f8f6f4 v[108:111], v[144:151], v[184:191], v[108:111] blgp:1\n\t" \
  "buffer_load_dwordx4 v252, s[40:43], s52 offen offset:3072 sc0 sc1 lds\n\t" \
  "v_mul_f32 v245, v241, v245\n\t" \
  "s_waitcnt lgkmcnt(0)\n\t" \
  "v_mfma_f32_16x16x128_f8f6f4 v[124:127], v[152:159], v[184:191], v[124:127] blgp:1\n\t" \
  "v_mul_f32 v246, v242, v246\n\t" \
  "v_mfma_f32_16x16x128_f8f6f4 v[120:123], v[152:159], v[176:183], v[120:123] blgp:1\n\t" \
  "v_mul_f32 v247, v243, v247\n\t" \
  "v_mfma_f32_16x16x128_f8f6f4 v[116:119], v[152:159], v[168:175], v[116:119] blgp:1\n\t" \
  "v_mfma_f32_16x16x128_f8f6f4 v[112:115], v[152:159], v[160:167], v[112:115] blgp:1\n\t" \
  "s_waitcnt vmcnt(18) lgkmcnt(0)\n\t" \
  "s_barrier\n\t" \
  "v_mfma_f32_16x16x128_f8f6f4 a[64:67], v[128:135], v[192:199], a[64:67] blgp:1\n\t" \
  "ds_read_b128 v[160:163], v251 offset:0\n\t" \
  "v_accvgpr_read_b32 v240, a130\n\t" \
  "v_mfma_f32_16x16x128_f8f6f4 a[68:71], v[128:135], v[200:207], a[68:71] blgp:1\n\t" \
  "ds_read_b128 v[164:167], v251 offset:8192\n\t" \
  "s_add_u32 s57, s80, 65536\n\t" \
  "v_accvgpr_read_b32 v241, a134\n\t" \
  "v_mfma_f32_16x16x128_f8f6f4 a[72:75], v[128:135], v[208:215], a[72:75] blgp:1\n\t" \
  "buffer_store_dwordx4 v[244:247], v253, s[44:47], s57 offen offset:0 nt\n\t" \
  "ds_read_b128 v[168:171], v251 offset:256\n\t" \
  "v_accvgpr_read_b32 v242, a138\n\t" \
  "v_mfma_f32_16x16x128_f8f6f4 a[76:79], v[128:135], v[216:223], a[76:79] blgp:1\n\t" \
  "ds_read_b128 v[128:131], v250 offset:0\n\t" \
  "ds_read_b128 v[132:135], v250 offset:8192\n\t" \
  "v_accvgpr_read_b32 v243, a142\n\t" \
  "v_mfma_f32_16x16x128_f8f6f4 a[92:95], v[136:143], v[216:223], a[92:95] blgp:1\n\t" \
  "ds_read_b128 v[172:175], v251 offset:8448\n\t" \
  "v_mul_f32 v244, s58, v240\n\t" \
  "v_mfma_f32_16x16x128_f8f6f4 a[88:91], v[136:143], v[208:215], a[88:91] blgp:1\n\t" \
  "ds_read_b128 v[176:179], v251 offset:512\n\t" \
  "v_mul_f32 v245, s58, v241\n\t" \
  "v_mfma_f32_16x16x128_f8f6f4 a[84:87], v[136:143], v[200:207], a[84:87] blgp:1\n\t" \
  "ds_read_b128 v[180:183], v251 offset:8704\n\t" \
  "v_mul_f32 v246, s58, v242\n\t" \
  "v_mfma_f32_16x16x128_f8f6f4 a[80:83], v[136:143], v[192:199], a[80:83] blgp:1\n\t" \
  "ds_read_b128 v[136:139], v250 offset:256\n\t" \
  "ds_read_b128 v[140:143], v250 offset:8448\n\t" \
  "v_mul_f32 v247, s58, v243\n\t" \
  "v_mfma_f32_16x16x128_f8f6f4 a[96:99], v[144:151], v[192:199], a[96:99] blgp:1\n\t" \
  "ds_read_b128 v[184:187], v251 offset:768\n\t" \
  "v_med3_f32 v244, v244, -s62, s62\n\t" \
  "v_mfma_f32_16x16x128_f8f6f4 a[100:103], v[144:151], v[200:207], a[100:103] blgp:1\n\t" \
  "ds_read_b128 v[188:191], v251 offset:8960\n\t" \
  "v_med3_f32 v245, v245, -s62, s62\n\t" \
  "v_mfma_f32_16x16x128_f8f6f4 a[104:107], v[144:151], v[208:215], a[104:107] blgp:1\n\t" \
  "s_add_u32 s48, s48, 0x10000\n\t" \
  "s_add_u32 s49, s49, 0x10000\n\t" \
  "v_med3_f32 v246, v246, -s62, s62\n\t" \
  "v_mfma_f32_16x16x128_f8f6f4 a[108:111], v[144:151], v[216:223], a[108:111] blgp:1\n\t" \
  "ds_read_b128 v[144:147], v250 offset:512\n\t" \
  "v_med3_f32 v247, v247, -s62, s62\n\t" \
  "v_mfma_f32_16x16x128_f8f6f4 a[124:127], v[152:159], v[216:223], a[124:127] blgp:1\n\t" \
  "ds_read_b128 v[148:151], v250 offset:8704\n\t" \
  "v_exp_f32 v244, v244\n\t" \
  "v_mfma_f32_16x16x128_f8f6f4 a[120:123], v[152:159], v[208:215], a[120:123] blgp:1\n\t" \
  "v_exp_f32 v245, v245\n\t" \
  "v_mfma_f32_16x16x128_f8f6f4 a[116:119], v[152:159], v[200:207], a[116:119] blgp:1\n\t" \
  "v_exp_f32 v246, v246\n\t" \
  "v_mfma_f32_16x16x128_f8f6f4 a[112:115], v[152:159], v[192:199], a[112:115] blgp:1\n\t" \
  "ds_read_b128 v[152:155], v250 offset:768\n\t" \
  "ds_read_b128 v[156:159], v250 offset:8960\n\t" \
  "s_waitcnt lgkmcnt(2)\n\t" \
  "v_mfma_f32_16x16x128_f8f6f4 a[0:3], v[128:135], v[160:167], a[0:3] blgp:1\n\t" \
  "ds_read_b128 v[192:195], v251 offset:16384\n\t" \
  "v_exp_f32 v247, v247\n\t" \
  "v_mfma_f32_16x16x128_f8f6f4 a[4:7], v[128:135], v[168:175], a[4:7] blgp:1\n\t" \
  "ds_read_b128 v[196:199], v251 offset:24576\n\t" \
  "v_mfma_f32_16x16x128_f8f6f4 a[8:11], v[128:135], v[176:183], a[8:11] blgp:1\n\t" \
  "ds_read_b128 v[200:203], v251 offset:16640\n\t" \
  "v_fma_f32 v244, v244, s60, s60\n\t" \
  "v_mfma_f32_16x16x128_f8f6f4 a[12:15], v[128:135], v[184:191], a[12:15] blgp:1\n\t" \
  "ds_read_b128 v[204:207], v251 offset:24832\n\t" \
  "v_fma_f32 v245, v245, s60, s60\n\t" \
  "v_mfma_f32_16x16x128_f8f6f4 a[28:31], v[136:143], v[184:191], a[28:31] blgp:1\n\t" \
  "ds_read_b128 v[208:211], v251 offset:16896\n\t" \
  "v_fma_f32 v246, v246, s60, s60\n\t" \
  "v_mfma_f32_16x16x128_f8f6f4 a[24:27], v[136:143], v[176:183], a[24:27] blgp:1\n\t" \
  "ds_read_b128 v[212:215], v251 offset:25088\n\t" \
  "v_fma_f32 v247, v247, s60, s60\n\t" \
  "v_mfma_f32_16x16x128_f8f6f4 a[20:23], v[136:143], v[168:175], a[20:23] blgp:1\n\t" \
  "ds_read_b128 v[216:219], v251 offset:17152\n\t" \
  "v_rcp_f32 v244, v244\n\t" \
  "v_mfma_f32_16x16x128_f8f6f4 a[16:19], v[136:143], v[160:167], a[16:19] blgp:1\n\t" \
  "ds_read_b128 v[220:223], v251 offset:25344\n\t" \
  "s_add_i32 s52, s48, -16384\n\t" \
  "s_add_u32 m0, s53, 81920\n\t" \
  "v_rcp_f32 v245, v245\n\t" \
  "v_mfma_f32_16x16x128_f8f6f4 a[32:35], v[144:151], v[160:167], a[32:35] blgp:1\n\t" \
  "buffer_load_dwordx4 v252, s[40:43], s52 offen offset:0 sc0 sc1 lds\n\t" \
  "v_rcp_f32 v246, v246\n\t" \
  "v_mfma_f32_16x16x128_f8f6f4 a[36:39], v[144:151], v[168:175], a[36:39] blgp:1\n\t" \
  "v_rcp_f32 v247, v247\n\t" \
  "v_mfma_f32_16x16x128_f8f6f4 a[40:43], v[144:151], v[176:183], a[40:43] blgp:1\n\t" \
  "buffer_load_dwordx4 v252, s[40:43], s52 offen offset:1024 sc0 sc1 lds\n\t" \
  "v_mul_f32 v244, v240, v244\n\t" \
  "v_mfma_f32_16x16x128_f8f6f4 a[44:47], v[144:151], v[184:191], a[44:47] blgp:1\n\t" \
  "v_mul_f32 v245, v241, v245\n\t" \
  "s_waitcnt lgkmcnt(0)\n\t" \
  "v_mfma_f32_16x16x128_f8f6f4 a[60:63], v[152:159], v[184:191], a[60:63] blgp:1\n\t" \
  "buffer_load_dwordx4 v252, s[40:43], s52 offen offset:2048 sc0 sc1 lds\n\t" \
  "v_mul_f32 v246, v242, v246\n\t" \
  "v_mfma_f32_16x16x128_f8f6f4 a[56:59], v[152:159], v[176:183], a[56:59] blgp:1\n\t" \
  "v_mul_f32 v247, v243, v247\n\t" \
  "v_mfma_f32_16x16x128_f8f6f4 a[52:55], v[152:159], v[168:175], a[52:55] blgp:1\n\t" \
  "buffer_load_dwordx4 v252, s[40:43], s52 offen offset:3072 sc0 sc1 lds\n\t" \
  "buffer_load_dword v254, off, s[88:91], s81 sc1\n\t" \
  "v_mfma_f32_16x16x128_f8f6f4 a[48:51], v[152:159], v[160:167], a[48:51] blgp:1\n\t" \
  "s_waitcnt vmcnt(19) lgkmcnt(0)\n\t" \
  "s_barrier\n\t" \
  "v_mfma_f32_16x16x128_f8f6f4 v[0:3], v[128:135], v[192:199], v[0:3] blgp:1\n\t" \
  "v_readlane_b32 s65, v255, s67\n\t" \
  "v_accvgpr_read_b32 v240, a131\n\t" \
  "v_mfma_f32_16x16x128_f8f6f4 v[4:7], v[128:135], v[200:207], v[4:7] blgp:1\n\t" \
  "s_add_u32 s57, s80, 131072\n\t" \
  "v_accvgpr_read_b32 v241, a135\n\t" \
  "v_mfma_f32_16x16x128_f8f6f4 v[8:11], v[128:135], v[208:215], v[8:11] blgp:1\n\t" \
  "buffer_store_dwordx4 v[244:247], v253, s[44:47], s57 offen offset:0 nt\n\t" \
  "s_add_u32 s67, s67, 1\n\t" \
  "s_cmp_ge_u32 s69, s65\n\t" \
  "s_cbranch_scc0 L_w5_%=\n\t" \
  "L_ok5_%=:\n\t" \
  "v_accvgpr_read_b32 v242, a139\n\t" \
  "v_mfma_f32_16x16x128_f8f6f4 v[12:15], v[128:135], v[216:223], v[12:15] blgp:1\n\t" \
  "ds_read_b128 v[128:131], v250 offset:16384\n\t" \
  "v_accvgpr_read_b32 v243, a143\n\t" \
  "v_mfma_f32_16x16x128_f8f6f4 v[28:31], v[136:143], v[216:223], v[28:31] blgp:1\n\t" \
  "s_add_i32 s52, s48, 0\n\t" \
  "s_add_u32 m0, s53, 32768\n\t" \
  "ds_read_b128 v[132:135], v250 offset:24576\n\t" \
  "v_mul_f32 v244, s58, v240\n\t" \
  "v_mfma_f32_16x16x128_f8f6f4 v[24:27], v[136:143], v[208:215], v[24:27] blgp:1\n\t" \
  "buffer_load_dwordx4 v252, s[40:43], s52 offen offset:0 sc0 sc1 lds\n\t" \
  "v_mul_f32 v245, s58, v241\n\t" \
  "v_mfma_f32_16x16x128_f8f6f4 v[20:23], v[136:143], v[200:207], v[20:23] blgp:1\n\t" \
  "buffer_load_dwordx4 v252, s[40:43], s52 offen offset:1024 sc0 sc1 lds\n\t" \
  "v_mul_f32 v246, s58, v242\n\t" \
  "v_mfma_f32_16x16x128_f8f6f4 v[16:19], v[136:143], v[192:199], v[16:19] blgp:1\n\t" \
  "ds_read_b128 v[136:139], v250 offset:16640\n\t" \
  "v_mul_f32 v247, s58, v243\n\t" \
  "v_mfma_f32_16x16x128_f8f6f4 v[32:35], v[144:151], v[192:199], v[32:35] blgp:1\n\t" \
  "ds_read_b128 v[140:143], v250 offset:24832\n\t" \
  "v_med3_f32 v244, v244, -s62, s62\n\t" \
  "v_mfma_f32_16x16x128_f8f6f4 v[36:39], v[144:151], v[200:207], v[36:39] blgp:1\n\t" \
  "buffer_load_dwordx4 v252, s[40:43], s52 offen offset:2048 sc0 sc1 lds\n\t" \
  "v_med3_f32 v245, v245, -s62, s62\n\t" \
  "v_mfma_f32_16x16x128_f8f6f4 v[40:43], v[144:151], v[208:215], v[40:43] blgp:1\n\t" \
  "buffer_load_dwordx4 v252, s[40:43], s52 offen offset:3072 sc0 sc1 lds\n\t" \
  "v_med3_f32 v246, v246, -s62, s62\n\t" \
  "v_mfma_f32_16x16x128_f8f6f4 v[44:47], v[144:151], v[216:223], v[44:47] blgp:1\n\t" \
  "ds_read_b128 v[144:147], v250 offset:16896\n\t" \
  "v_med3_f32 v247, v247, -s62, s62\n\t" \
  "v_mfma_f32_16x16x128_f8f6f4 v[60:63], v[152:159], v[216:223], v[60:63] blgp:1\n\t" \
  "ds_read_b128 v[148:151], v250 offset:25088\n\t" \
  "v_exp_f32 v244, v244\n\t" \
  "v_mfma_f32_16x16x128_f8f6f4 v[56:59], v[152:159], v[208:215], v[56:59] blgp:1\n\t" \
  "v_exp_f32 v245, v245\n\t" \
  "v_mfma_f32_16x16x128_f8f6f4 v[52:55], v[152:159], v[200:207], v[52:55] blgp:1\n\t" \
  "v_exp_f32 v246, v246\n\t" \
  "v_mfma_f32_16x16x128_f8f6f4 v[48:51], v[152:159], v[192:199], v[48:51] blgp:1\n\t" \
  "ds_read_b128 v[152:155], v250 offset:17152\n\t" \
  "ds_read_b128 v[156:159], v250 offset:25344\n\t" \
  "s_waitcnt lgkmcnt(2)\n\t" \
  "v_mfma_f32_16x16x128_f8f6f4 a[64:67], v[128:135], v[192:199], a[64:67] blgp:1\n\t" \
  "s_add_i32 s52, s49, 0\n\t" \
  "s_add_u32 m0, s53, 98304\n\t" \
  "ds_read_b128 v[224:227], v252 offset:0\n\t" \
  "v_exp_f32 v247, v247\n\t" \
  "v_mfma_f32_16x16x128_f8f6f4 a[68:71], v[128:135], v[200:207], a[68:71] blgp:1\n\t" \
  "buffer_load_dwordx4 v252, s[40:43], s52 offen offset:0 sc0 sc1 lds\n\t" \
  "v_mfma_f32_16x16x128_f8f6f4 a[72:75], v[128:135], v[208:215], a[72:75] blgp:1\n\t" \
  "buffer_load_dwordx4 v252, s[40:43], s52 offen offset:1024 sc0 sc1 lds\n\t" \
  "v_fma_f32 v244, v244, s60, s60\n\t" \
  "v_mfma_f32_16x16x128_f8f6f4 a[76:79], v[128:135], v[216:223], a[76:79] blgp:1\n\t" \
  "ds_read_b128 v[228:231], v252 offset:1024\n\t" \
  "v_fma_f32 v245, v245, s60, s60\n\t" \
  "v_mfma_f32_16x16x128_f8f6f4 a[92:95], v[136:143], v[216:223], a[92:95] blgp:1\n\t" \
  "buffer_load_dwordx4 v252, s[40:43], s52 offen offset:2048 sc0 sc1 lds\n\t" \
  "ds_read_b128 v[232:235], v252 offset:2048\n\t" \
  "v_fma_f32 v246, v246, s60, s60\n\t" \
  "v_mfma_f32_16x16x128_f8f6f4 a[88:91], v[136:143], v[208:215], a[88:91] blgp:1\n\t" \
  "buffer_load_dwordx4 v252, s[40:43], s52 offen offset:3072 sc0 sc1 lds\n\t" \
  "v_fma_f32 v247, v247, s60, s60\n\t" \
  "v_mfma_f32_16x16x128_f8f6f4 a[84:87], v[136:143], v[200:207], a[84:87] blgp:1\n\t" \
  "s_add_i32 s52, s49, 16384\n\t" \
  "s_add_u32 m0, s53, 114688\n\t" \
  "ds_read_b128 v[236:239], v252 offset:3072\n\t" \
  "v_rcp_f32 v244, v244\n\t" \
  "v_mfma_f32_16x16x128_f8f6f4 a[80:83], v[136:143], v[192:199], a[80:83] blgp:1\n\t" \
  "buffer_load_dwordx4 v252, s[40:43], s52 offen offset:0 sc0 sc1 lds\n\t" \
  "v_rcp_f32 v245, v245\n\t" \
  "v_mfma_f32_16x16x128_f8f6f4 a[96:99], v[144:151], v[192:199], a[96:99] blgp:1\n\t" \
  "buffer_load_dwordx4 v252, s[40:43], s52 offen offset:1024 sc0 sc1 lds\n\t" \
  "v_rcp_f32 v246, v246\n\t" \
  "v_mfma_f32_16x16x128_f8f6f4 a[100:103], v[144:151], v[200:207], a[100:103] blgp:1\n\t" \
  "v_rcp_f32 v247, v247\n\t" \
  "v_mfma_f32_16x16x128_f8f6f4 a[104:107], v[144:151], v[208:215], a[104:107] blgp:1\n\t" \
  "buffer_load_dwordx4 v252, s[40:43], s52 offen offset:2048 sc0 sc1 lds\n\t" \
  "v_mul_f32 v244, v240, v244\n\t" \
  "v_mfma_f32_16x16x128_f8f6f4 a[108:111], v[144:151], v[216:223], a[108:111] blgp:1\n\t" \
  "buffer_load_dwordx4 v252, s[40:43], s52 offen offset:3072 sc0 sc1 lds\n\t" \
  "v_mul_f32 v245, v241, v245\n\t" \
  "s_waitcnt lgkmcnt(0)\n\t" \
  "v_mfma_f32_16x16x128_f8f6f4 a[124:127], v[152:159], v[216:223], a[124:127] blgp:1\n\t" \
  "v_mul_f32 v246, v242, v246\n\t" \
  "v_mfma_f32_16x16x128_f8f6f4 a[120:123], v[152:159], v[208:215], a[120:123] blgp:1\n\t" \
  "v_mul_f32 v247, v243, v247\n\t" \
  "v_mfma_f32_16x16x128_f8f6f4 a[116:119], v[152:159], v[200:207], a[116:119] blgp:1\n\t" \
  "v_mfma_f32_16x16x128_f8f6f4 a[112:115], v[152:159], v[192:199], a[112:115] blgp:1\n\t" \
  "s_waitcnt vmcnt(19) lgkmcnt(0)\n\t" \
  "s_barrier\n\t" \
  "v_mfma_f32_16x16x128_f8f6f4 v[64:67], v[128:135], v[160:167], v[64:67] blgp:1\n\t" \
  "ds_read_b128 v[192:195], v251 offset:49152\n\t" \
  "v_accvgpr_read_b32 v240, a144\n\t" \
  "v_mfma_f32_16x16x128_f8f6f4 v[68:71], v[128:135], v[168:175], v[68:71] blgp:1\n\t" \
  "ds_read_b128 v[196:199], v251 offset:57344\n\t" \
  "s_add_u32 s57, s80, 196608\n\t" \
  "v_accvgpr_read_b32 v241, a148\n\t" \
  "v_mfma_f32_16x16x128_f8f6f4 v[72:75], v[128:135], v[176:183], v[72:75] blgp:1\n\t" \
  "buffer_store_dwordx4 v[244:247], v253, s[44:47], s57 offen offset:0 nt\n\t" \
  "ds_read_b128 v[200:203], v251 offset:49408\n\t" \
  "v_accvgpr_read_b32 v242, a152\n\t" \
  "v_mfma_f32_16x16x128_f8f6f4 v[76:79], v[128:135], v[184:191], v[76:79] blgp:1\n\t" \
  "ds_read_b128 v[128:131], v250 offset:32768\n\t" \
  "ds_read_b128 v[132:135], v250 offset:40960\n\t" \
  "v_accvgpr_read_b32 v243, a156\n\t" \
  "v_mfma_f32_16x16x128_f8f6f4 v[92:95], v[136:143], v[184:191], v[92:95] blgp:1\n\t" \
  "ds_read_b128 v[204:207], v251 offset:57600\n\t" \
  "v_mul_f32 v244, s58, v240\n\t" \
  "v_mfma_f32_16x16x128_f8f6f4 v[88:91], v[136:143], v[176:183], v[88:91] blgp:1\n\t" \
  "ds_read_b128 v[208:211], v251 offset:49664\n\t" \
  "v_mul_f32 v245, s58, v241\n\t" \
  "v_mfma_f32_16x16x128_f8f6f4 v[84:87], v[136:143], v[168:175], v[84:87] blgp:1\n\t" \
  "ds_read_b128 v[212:215], v251 offset:57856\n\t" \
  "v_mul_f32 v246, s58, v242\n\t" \
  "v_mfma_f32_16x16x128_f8f6f4 v[80:83], v[136:143], v[160:167], v[80:83] blgp:1\n\t" \
  "ds_read_b128 v[136:139], v250 offset:33024\n\t" \
  "ds_read_b128 v[140:143], v250 offset:41216\n\t" \
  "v_mul_f32 v247, s58, v243\n\t" \
  "v_mfma_f32_16x16x128_f8f6f4 v[96:99], v[144:151], v[160:167], v[96:99] blgp:1\n\t" \
  "ds_read_b128 v[216:219], v251 offset:49920\n\t" \
  "v_med3_f32 v244, v244, -s62, s62\n\t" \
  "v_mfma_f32_16x16x128_f8f6f4 v[100:103], v[144:151], v[168:175], v[100:103] blgp:1\n\t" \
  "ds_read_b128 v[220:223], v251 offset:58112\n\t" \
  "v_med3_f32 v245, v245, -s62, s62\n\t" \
  "v_mfma_f32_16x16x128_f8f6f4 v[104:107], v[144:151], v[176:183], v[104:107] blgp:1\n\t" \
  "v_med3_f32 v246, v246, -s62, s62\n\t" \
  "v_mfma_f32_16x16x128_f8f6f4 v[108:111], v[144:151], v[184:191], v[108:111] blgp:1\n\t" \
  "ds_read_b128 v[144:147], v250 offset:33280\n\t" \
  "v_med3_f32 v247, v247, -s62, s62\n\t" \
  "v_mfma_f32_16x16x128_f8f6f4 v[124:127], v[152:159], v[184:191], v[124:127] blgp:1\n\t" \
  "ds_read_b128 v[148:151], v250 offset:41472\n\t" \
  "v_exp_f32 v244, v244\n\t" \
  "v_mfma_f32_16x16x128_f8f6f4 v[120:123], v[152:159], v[176:183], v[120:123] blgp:1\n\t" \
  "v_exp_f32 v245, v245\n\t" \
  "v_mfma_f32_16x16x128_f8f6f4 v[116:119], v[152:159], v[168:175], v[116:119] blgp:1\n\t" \
  "v_exp_f32 v246, v246\n\t" \
  "v_mfma_f32_16x16x128_f8f6f4 v[112:115], v[152:159], v[160:167], v[112:115] blgp:1\n\t" \
  "ds_read_b128 v[152:155], v250 offset:33536\n\t" \
  "ds_read_b128 v[156:159], v250 offset:41728\n\t" \
  "s_waitcnt lgkmcnt(2)\n\t" \
  "v_mfma_f32_16x16x128_f8f6f4 v[0:3], v[128:135], v[192:199], v[0:3] blgp:1\n\t" \
  "ds_read_b128 v[160:163], v251 offset:32768\n\t" \
  "v_exp_f32 v247, v247\n\t" \
  "v_mfma_f32_16x16x128_f8f6f4 v[4:7], v[128:135], v[200:207], v[4:7] blgp:1\n\t" \
  "ds_read_b128 v[164:167], v251 offset:40960\n\t" \
  "v_mfma_f32_16x16x128_f8f6f4 v[8:11], v[128:135], v[208:215], v[8:11] blgp:1\n\t" \
  "ds_read_b128 v[168:171], v251 offset:33024\n\t" \
  "v_fma_f32 v244, v244, s60, s60\n\t" \
  "v_mfma_f32_16x16x128_f8f6f4 v[12:15], v[128:135], v[216:223], v[12:15] blgp:1\n\t" \
  "ds_read_b128 v[172:175], v251 offset:41216\n\t" \
  "v_fma_f32 v245, v245, s60, s60\n\t" \
  "v_mfma_f32_16x16x128_f8f6f4 v[28:31], v[136:143], v[216:223], v[28:31] blgp:1\n\t" \
  "ds_read_b128 v[176:179], v251 offset:33280\n\t" \
  "v_fma_f32 v246, v246, s60, s60\n\t" \
  "v_mfma_f32_16x16x128_f8f6f4 v[24:27], v[136:143], v[208:215], v[24:27] blgp:1\n\t" \
  "ds_read_b128 v[180:183], v251 offset:41472\n\t" \
  "v_fma_f32 v247, v247, s60, s60\n\t" \
  "v_mfma_f32_16x16x128_f8f6f4 v[20:23], v[136:143], v[200:207], v[20:23] blgp:1\n\t" \
  "ds_read_b128 v[184:187], v251 offset:33536\n\t" \
  "v_rcp_f32 v244, v244\n\t" \
  "v_mfma_f32_16x16x128_f8f6f4 v[16:19], v[136:143], v[192:199], v[16:19] blgp:1\n\t" \
  "ds_read_b128 v[188:191], v251 offset:41728\n\t" \
  "s_add_i32 s52, s48, 16384\n\t" \
  "s_add_u32 m0, s53, 49152\n\t" \
  "v_rcp_f32 v245, v245\n\t" \
  "v_mfma_f32_16x16x128_f8f6f4 v[32:35], v[144:151], v[192:199], v[32:35] blgp:1\n\t" \
  "buffer_load_dwordx4 v252, s[40:43], s52 offen offset:0 sc0 sc1 lds\n\t" \
  "v_rcp_f32 v246, v246\n\t" \
  "v_mfma_f32_16x16x128_f8f6f4 v[36:39], v[144:151], v[200:207], v[36:39] blgp:1\n\t" \
  "v_rcp_f32 v247, v247\n\t" \
  "v_mfma_f32_16x16x128_f8f6f4 v[40:43], v[144:151], v[208:215], v[40:43] blgp:1\n\t" \
  "buffer_load_dwordx4 v252, s[40:43], s52 offen offset:1024 sc0 sc1 lds\n\t" \
  "v_mul_f32 v244, v240, v244\n\t" \
  "v_mfma_f32_16x16x128_f8f6f4 v[44:47], v[144:151], v[216:223], v[44:47] blgp:1\n\t" \
  "v_mul_f32 v245, v241, v245\n\t" \
  "s_waitcnt lgkmcnt(0)\n\t" \
  "v_mfma_f32_16x16x128_f8f6f4 v[60:63], v[152:159], v[216:223], v[60:63] blgp:1\n\t" \
  "buffer_load_dwordx4 v252, s[40:43], s52 offen offset:2048 sc0 sc1 lds\n\t" \
  "v_mul_f32 v246, v242, v246\n\t" \
  "v_mfma_f32_16x16x128_f8f6f4 v[56:59], v[152:159], v[208:215], v[56:59] blgp:1\n\t" \
  "v_mul_f32 v247, v243, v247\n\t" \
  "v_mfma_f32_16x16x128_f8f6f4 v[52:55], v[152:159], v[200:207], v[52:55] blgp:1\n\t" \
  "buffer_load_dwordx4 v252, s[40:43], s52 offen offset:3072 sc0 sc1 lds\n\t" \
  "v_mfma_f32_16x16x128_f8f6f4 v[48:51], v[152:159], v[192:199], v[48:51] blgp:1\n\t" \
  "s_waitcnt vmcnt(18) lgkmcnt(0)\n\t" \
  "s_barrier\n\t" \
  "v_mfma_f32_16x16x128_f8f6f4 a[0:3], v[128:135], v[160:167], a[0:3] blgp:1\n\t" \
  "v_readfirstlane_b32 s87, v254\n\t" \
  "v_readlane_b32 s65, v255, s67\n\t" \
  "v_accvgpr_read_b32 v240, a145\n\t" \
  "v_mfma_f32_16x16x128_f8f6f4 a[4:7], v[128:135], v[168:175], a[4:7] blgp:1\n\t" \
  "s_add_u32 s57, s80, 1048576\n\t" \
  "v_accvgpr_read_b32 v241, a149\n\t" \
  "v_mfma_f32_16x16x128_f8f6f4 a[8:11], v[128:135], v[176:183], a[8:11] blgp:1\n\t" \
  "buffer_store_dwordx4 v[244:247], v253, s[44:47], s57 offen offset:0 nt\n\t" \
  "s_max_u32 s69, s69, s87\n\t" \
  "s_add_u32 s67, s67, 1\n\t" \
  "s_cmp_ge_u32 s69, s65\n\t" \
  "s_cbranch_scc0 L_w6_%=\n\t" \
  "L_ok6_%=:\n\t" \
  "v_accvgpr_read_b32 v242, a153\n\t" \
  "v_mfma_f32_16x16x128_f8f6f4 a[12:15], v[128:135], v[184:191], a[12:15] blgp:1\n\t" \
  "ds_read_b128 v[128:131], v250 offset:49152\n\t" \
  "v_accvgpr_read_b32 v243, a157\n\t" \
  "v_mfma_f32_16x16x128_f8f6f4 a[28:31], v[136:143], v[184:191], a[28:31] blgp:1\n\t" \
  "s_add_i32 s52, s48, 32768\n\t" \
  "s_add_u32 m0, s53, 65536\n\t" \
  "ds_read_b128 v[132:135], v250 offset:57344\n\t" \
  "v_mul_f32 v244, s58, v240\n\t" \
  "v_mfma_f32_16x16x128_f8f6f4 a[24:27], v[136:143], v[176:183], a[24:27] blgp:1\n\t" \
  "buffer_load_dwordx4 v252, s[40:43], s52 offen offset:0 sc0 sc1 lds\n\t" \
  "v_mul_f32 v245, s58, v241\n\t" \
  "v_mfma_f32_16x16x128_f8f6f4 a[20:23], v[136:143], v[168:175], a[20:23] blgp:1\n\t" \
  "buffer_load_dwordx4 v252, s[40:43], s52 offen offset:1024 sc0 sc1 lds\n\t" \
  "v_mul_f32 v246, s58, v242\n\t" \
  "v_mfma_f32_16x16x128_f8f6f4 a[16:19], v[136:143], v[160:167], a[16:19] blgp:1\n\t" \
  "ds_read_b128 v[136:139], v250 offset:49408\n\t" \
  "v_mul_f32 v247, s58, v243\n\t" \
  "v_mfma_f32_16x16x128_f8f6f4 a[32:35], v[144:151], v[160:167], a[32:35] blgp:1\n\t" \
  "ds_read_b128 v[140:143], v250 offset:57600\n\t" \
  "v_med3_f32 v244, v244, -s62, s62\n\t" \
  "v_mfma_f32_16x16x128_f8f6f4 a[36:39], v[144:151], v[168:175], a[36:39] blgp:1\n\t" \
  "buffer_load_dwordx4 v252, s[40:43], s52 offen offset:2048 sc0 sc1 lds\n\t" \
  "v_med3_f32 v245, v245, -s62, s62\n\t" \
  "v_mfma_f32_16x16x128_f8f6f4 a[40:43], v[144:151], v[176:183], a[40:43] blgp:1\n\t" \
  "buffer_load_dwordx4 v252, s[40:43], s52 offen offset:3072 sc0 sc1 lds\n\t" \
  "v_med3_f32 v246, v246, -s62, s62\n\t" \
  "v_mfma_f32_16x16x128_f8f6f4 a[44:47], v[144:151], v[184:191], a[44:47] blgp:1\n\t" \
  "ds_read_b128 v[144:147], v250 offset:49664\n\t" \
  "v_med3_f32 v247, v247, -s62, s62\n\t" \
  "v_mfma_f32_16x16x128_f8f6f4 a[60:63], v[152:159], v[184:191], a[60:63] blgp:1\n\t" \
  "ds_read_b128 v[148:151], v250 offset:57856\n\t" \
  "v_exp_f32 v244, v244\n\t" \
  "v_mfma_f32_16x16x128_f8f6f4 a[56:59], v[152:159], v[176:183], a[56:59] blgp:1\n\t" \
  "v_exp_f32 v245, v245\n\t" \
  "v_mfma_f32_16x16x128_f8f6f4 a[52:55], v[152:159], v[168:175], a[52:55] blgp:1\n\t" \
  "v_exp_f32 v246, v246\n\t" \
  "v_mfma_f32_16x16x128_f8f6f4 a[48:51], v[152:159], v[160:167], a[48:51] blgp:1\n\t" \
  "ds_read_b128 v[152:155], v250 offset:49920\n\t" \
  "ds_read_b128 v[156:159], v250 offset:58112\n\t" \
  "s_waitcnt lgkmcnt(2)\n\t" \
  "v_mfma_f32_16x16x128_f8f6f4 v[64:67], v[128:135], v[160:167], v[64:67] blgp:1\n\t" \
  "s_add_i32 s52, s49, 49152\n\t" \
  "s_add_u32 m0, s53, 147456\n\t" \
  "v_exp_f32 v247, v247\n\t" \
  "v_mfma_f32_16x16x128_f8f6f4 v[68:71], v[128:135], v[168:175], v[68:71] blgp:1\n\t" \
  "buffer_load_dwordx4 v252, s[40:43], s52 offen offset:0 sc0 sc1 lds\n\t" \
  "v_mfma_f32_16x16x128_f8f6f4 v[72:75], v[128:135], v[176:183], v[72:75] blgp:1\n\t" \
  "buffer_load_dwordx4 v252, s[40:43], s52 offen offset:1024 sc0 sc1 lds\n\t" \
  "v_fma_f32 v244, v244, s60, s60\n\t" \
  "v_mfma_f32_16x16x128_f8f6f4 v[76:79], v[128:135], v[184:191], v[76:79] blgp:1\n\t" \
  "v_fma_f32 v245, v245, s60, s60\n\t" \
  "v_mfma_f32_16x16x128_f8f6f4 v[92:95], v[136:143], v[184:191], v[92:95] blgp:1\n\t" \
  "buffer_load_dwordx4 v252, s[40:43], s52 offen offset:2048 sc0 sc1 lds\n\t" \
  "v_fma_f32 v246, v246, s60, s60\n\t" \
  "v_mfma_f32_16x16x128_f8f6f4 v[88:91], v[136:143], v[176:183], v[88:91] blgp:1\n\t" \
  "buffer_load_dwordx4 v252, s[40:43], s52 offen offset:3072 sc0 sc1 lds\n\t" \
  "v_fma_f32 v247, v247, s60, s60\n\t" \
  "v_mfma_f32_16x16x128_f8f6f4 v[84:87], v[136:143], v[168:175], v[84:87] blgp:1\n\t" \
  "s_add_i32 s52, s49, 32768\n\t" \
  "s_add_u32 m0, s53, 131072\n\t" \
  "v_rcp_f32 v244, v244\n\t" \
  "v_mfma_f32_16x16x128_f8f6f4 v[80:83], v[136:143], v[160:167], v[80:83] blgp:1\n\t" \
  "buffer_load_dwordx4 v252, s[40:43], s52 offen offset:0 sc0 sc1 lds\n\t" \
  "v_rcp_f32 v245, v245\n\t" \
  "v_mfma_f32_16x16x128_f8f6f4 v[96:99], v[144:151], v[160:167], v[96:99] blgp:1\n\t" \
  "buffer_load_dwordx4 v252, s[40:43], s52 offen offset:1024 sc0 sc1 lds\n\t" \
  "v_rcp_f32 v246, v246\n\t" \
  "v_mfma_f32_16x16x128_f8f6f4 v[100:103], v[144:151], v[168:175], v[100:103] blgp:1\n\t" \
  "v_rcp_f32 v247, v247\n\t" \
  "v_mfma_f32_16x16x128_f8f6f4 v[104:107], v[144:151], v[176:183], v[104:107] blgp:1\n\t" \
  "buffer_load_dwordx4 v252, s[40:43], s52 offen offset:2048 sc0 sc1 lds\n\t" \
  "v_mul_f32 v244, v240, v244\n\t" \
  "v_mfma_f32_16x16x128_f8f6f4 v[108:111], v[144:151], v[184:191], v[108:111] blgp:1\n\t" \
  "buffer_load_dwordx4 v252, s[40:43], s52 offen offset:3072 sc0 sc1 lds\n\t" \
  "v_mul_f32 v245, v241, v245\n\t" \
  "s_waitcnt lgkmcnt(0)\n\t" \
  "v_mfma_f32_16x16x128_f8f6f4 v[124:127], v[152:159], v[184:191], v[124:127] blgp:1\n\t" \
  "v_mul_f32 v246, v242, v246\n\t" \
  "v_mfma_f32_16x16x128_f8f6f4 v[120:123], v[152:159], v[176:183], v[120:123] blgp:1\n\t" \
  "v_mul_f32 v247, v243, v247\n\t" \
  "v_mfma_f32_16x16x128_f8f6f4 v[116:119], v[152:159], v[168:175], v[116:119] blgp:1\n\t" \
  "v_mfma_f32_16x16x128_f8f6f4 v[112:115], v[152:159], v[160:167], v[112:115] blgp:1\n\t" \
  "s_waitcnt vmcnt(18) lgkmcnt(0)\n\t" \
  "s_barrier\n\t" \
  "v_mfma_f32_16x16x128_f8f6f4 a[64:67], v[128:135], v[192:199], a[64:67] blgp:1\n\t" \
  "ds_read_b128 v[160:163], v251 offset:0\n\t" \
  "v_accvgpr_read_b32 v240, a146\n\t" \
  "v_mfma_f32_16x16x128_f8f6f4 a[68:71], v[128:135], v[200:207], a[68:71] blgp:1\n\t" \
  "ds_read_b128 v[164:167], v251 offset:8192\n\t" \
  "s_add_u32 s57, s80, 1114112\n\t" \
  "v_accvgpr_read_b32 v241, a150\n\t" \
  "v_mfma_f32_16x16x128_f8f6f4 a[72:75], v[128:135], v[208:215], a[72:75] blgp:1\n\t" \
  "buffer_store_dwordx4 v[244:247], v253, s[44:47], s57 offen offset:0 nt\n\t" \
  "ds_read_b128 v[168:171], v251 offset:256\n\t" \
  "v_accvgpr_read_b32 v242, a154\n\t" \
  "v_mfma_f32_16x16x128_f8f6f4 a[76:79], v[128:135], v[216:223], a[76:79] blgp:1\n\t" \
  "ds_read_b128 v[128:131], v250 offset:0\n\t" \
  "ds_read_b128 v[132:135], v250 offset:8192\n\t" \
  "v_accvgpr_read_b32 v243, a158\n\t" \
  "v_mfma_f32_16x16x128_f8f6f4 a[92:95], v[136:143], v[216:223], a[92:95] blgp:1\n\t" \
  "ds_read_b128 v[172:175], v251 offset:8448\n\t" \
  "v_mul_f32 v244, s58, v240\n\t" \
  "v_mfma_f32_16x16x128_f8f6f4 a[88:91], v[136:143], v[208:215], a[88:91] blgp:1\n\t" \
  "ds_read_b128 v[176:179], v251 offset:512\n\t" \
  "v_mul_f32 v245, s58, v241\n\t" \
  "v_mfma_f32_16x16x128_f8f6f4 a[84:87], v[136:143], v[200:207], a[84:87] blgp:1\n\t" \
  "ds_read_b128 v[180:183], v251 offset:8704\n\t" \
  "v_mul_f32 v246, s58, v242\n\t" \
  "v_mfma_f32_16x16x128_f8f6f4 a[80:83], v[136:143], v[192:199], a[80:83] blgp:1\n\t" \
  "ds_read_b128 v[136:139], v250 offset:256\n\t" \
  "ds_read_b128 v[140:143], v250 offset:8448\n\t" \
  "v_mul_f32 v247, s58, v243\n\t" \
  "v_mfma_f32_16x16x128_f8f6f4 a[96:99], v[144:151], v[192:199], a[96:99] blgp:1\n\t" \
  "ds_read_b128 v[184:187], v251 offset:768\n\t" \
  "v_med3_f32 v244, v244, -s62, s62\n\t" \
  "v_mfma_f32_16x16x128_f8f6f4 a[100:103], v[144:151], v[200:207], a[100:103] blgp:1\n\t" \
  "ds_read_b128 v[188:191], v251 offset:8960\n\t" \
  "v_med3_f32 v245, v245, -s62, s62\n\t" \
  "v_mfma_f32_16x16x128_f8f6f4 a[104:107], v[144:151], v[208:215], a[104:107] blgp:1\n\t" \
  "s_add_u32 s48, s48, 0x10000\n\t" \
  "s_add_u32 s49, s49, 0x10000\n\t" \
  "v_med3_f32 v246, v246, -s62, s62\n\t" \
  "v_mfma_f32_16x16x128_f8f6f4 a[108:111], v[144:151], v[216:223], a[108:111] blgp:1\n\t" \
  "ds_read_b128 v[144:147], v250 offset:512\n\t" \
  "v_med3_f32 v247, v247, -s62, s62\n\t" \
  "v_mfma_f32_16x16x128_f8f6f4 a[124:127], v[152:159], v[216:223], a[124:127] blgp:1\n\t" \
  "ds_read_b128 v[148:151], v250 offset:8704\n\t" \
  "v_exp_f32 v244, v244\n\t" \
  "v_mfma_f32_16x16x128_f8f6f4 a[120:123], v[152:159], v[208:215], a[120:123] blgp:1\n\t" \
  "v_exp_f32 v245, v245\n\t" \
  "v_mfma_f32_16x16x128_f8f6f4 a[116:119], v[152:159], v[200:207], a[116:119] blgp:1\n\t" \
  "v_exp_f32 v246, v246\n\t" \
  "v_mfma_f32_16x16x128_f8f6f4 a[112:115], v[152:159], v[192:199], a[112:115] blgp:1\n\t" \
  "ds_read_b128 v[152:155], v250 offset:768\n\t" \
  "ds_read_b128 v[156:159], v250 offset:8960\n\t" \
  "s_waitcnt lgkmcnt(2)\n\t" \
  "v_mfma_f32_16x16x128_f8f6f4 a[0:3], v[128:135], v[160:167], a[0:3] blgp:1\n\t" \
  "ds_read_b128 v[192:195], v251 offset:16384\n\t" \
  "v_exp_f32 v247, v247\n\t" \
  "v_mfma_f32_16x16x128_f8f6f4 a[4:7], v[128:135], v[168:175], a[4:7] blgp:1\n\t" \
  "ds_read_b128 v[196:199], v251 offset:24576\n\t" \
  "v_mfma_f32_16x16x128_f8f6f4 a[8:11], v[128:135], v[176:183], a[8:11] blgp:1\n\t" \
  "ds_read_b128 v[200:203], v251 offset:16640\n\t" \
  "v_fma_f32 v244, v244, s60, s60\n\t" \
  "v_mfma_f32_16x16x128_f8f6f4 a[12:15], v[128:135], v[184:191], a[12:15] blgp:1\n\t" \
  "ds_read_b128 v[204:207], v251 offset:24832\n\t" \
  "v_fma_f32 v245, v245, s60, s60\n\t" \
  "v_mfma_f32_16x16x128_f8f6f4 a[28:31], v[136:143], v[184:191], a[28:31] blgp:1\n\t" \
  "ds_read_b128 v[208:211], v251 offset:16896\n\t" \
  "v_fma_f32 v246, v246, s60, s60\n\t" \
  "v_mfma_f32_16x16x128_f8f6f4 a[24:27], v[136:143], v[176:183], a[24:27] blgp:1\n\t" \
  "ds_read_b128 v[212:215], v251 offset:25088\n\t" \
  "v_fma_f32 v247, v247, s60, s60\n\t" \
  "v_mfma_f32_16x16x128_f8f6f4 a[20:23], v[136:143], v[168:175], a[20:23] blgp:1\n\t" \
  "ds_read_b128 v[216:219], v251 offset:17152\n\t" \
  "v_rcp_f32 v244, v244\n\t" \
  "v_mfma_f32_16x16x128_f8f6f4 a[16:19], v[136:143], v[160:167], a[16:19] blgp:1\n\t" \
  "ds_read_b128 v[220:223], v251 offset:25344\n\t" \
  "s_add_i32 s52, s48, -16384\n\t" \
  "s_add_u32 m0, s53, 81920\n\t" \
  "v_rcp_f32 v245, v245\n\t" \
  "v_mfma_f32_16x16x128_f8f6f4 a[32:35], v[144:151], v[160:167], a[32:35] blgp:1\n\t" \
  "buffer_load_dwordx4 v252, s[40:43], s52 offen offset:0 sc0 sc1 lds\n\t" \
  "v_rcp_f32 v246, v246\n\t" \
  "v_mfma_f32_16x16x128_f8f6f4 a[36:39], v[144:151], v[168:175], a[36:39] blgp:1\n\t" \
  "v_rcp_f32 v247, v247\n\t" \
  "v_mfma_f32_16x16x128_f8f6f4 a[40:43], v[144:151], v[176:183], a[40:43] blgp:1\n\t" \
  "buffer_load_dwordx4 v252, s[40:43], s52 offen offset:1024 sc0 sc1 lds\n\t" \
  "v_mul_f32 v244, v240, v244\n\t" \
  "v_mfma_f32_16x16x128_f8f6f4 a[44:47], v[144:151], v[184:191], a[44:47] blgp:1\n\t" \
  "v_mul_f32 v245, v241, v245\n\t" \
  "s_waitcnt lgkmcnt(0)\n\t" \
  "v_mfma_f32_16x16x128_f8f6f4 a[60:63], v[152:159], v[184:191], a[60:63] blgp:1\n\t" \
  "buffer_load_dwordx4 v252, s[40:43], s52 offen offset:2048 sc0 sc1 lds\n\t" \
  "v_mul_f32 v246, v242, v246\n\t" \
  "v_mfma_f32_16x16x128_f8f6f4 a[56:59], v[152:159], v[176:183], a[56:59] blgp:1\n\t" \
  "v_mul_f32 v247, v243, v247\n\t" \
  "v_mfma_f32_16x16x128_f8f6f4 a[52:55], v[152:159], v[168:175], a[52:55] blgp:1\n\t" \
  "buffer_load_dwordx4 v252, s[40:43], s52 offen offset:3072 sc0 sc1 lds\n\t" \
  "buffer_load_dword v254, off, s[88:91], s81 sc1\n\t" \
  "v_mfma_f32_16x16x128_f8f6f4 a[48:51], v[152:159], v[160:167], a[48:51] blgp:1\n\t" \
  "s_waitcnt vmcnt(19) lgkmcnt(0)\n\t" \
  "s_barrier\n\t" \
  "v_mfma_f32_16x16x128_f8f6f4 v[0:3], v[128:135], v[192:199], v[0:3] blgp:1\n\t" \
  "v_readlane_b32 s65, v255, s67\n\t" \
  "v_accvgpr_read_b32 v240, a147\n\t" \
  "v_mfma_f32_16x16x128_f8f6f4 v[4:7], v[128:135], v[200:207], v[4:7] blgp:1\n\t" \
  "s_add_u32 s57, s80, 1179648\n\t" \
  "v_accvgpr_read_b32 v241, a151\n\t" \
  "v_mfma_f32_16x16x128_f8f6f4 v[8:11], v[128:135], v[208:215], v[8:11] blgp:1\n\t" \
  "buffer_store_dwordx4 v[244:247], v253, s[44:47], s57 offen offset:0 nt\n\t" \
  "s_add_u32 s67, s67, 1\n\t" \
  "s_cmp_ge_u32 s69, s65\n\t" \
  "s_cbranch_scc0 L_w7_%=\n\t" \
  "L_ok7_%=:\n\t" \
  "v_accvgpr_read_b32 v242, a155\n\t" \
  "v_mfma_f32_16x16x128_f8f6f4 v[12:15], v[128:135], v[216:223], v[12:15] blgp:1\n\t" \
  "ds_read_b128 v[128:131], v250 offset:16384\n\t" \
  "v_accvgpr_read_b32 v243, a159\n\t" \
  "v_mfma_f32_16x16x128_f8f6f4 v[28:31], v[136:143], v[216:223], v[28:31] blgp:1\n\t" \
  "s_add_i32 s52, s48, 0\n\t" \
  "s_add_u32 m0, s53, 32768\n\t" \
  "ds_read_b128 v[132:135], v250 offset:24576\n\t" \
  "v_mul_f32 v244, s58, v240\n\t" \
  "v_mfma_f32_16x16x128_f8f6f4 v[24:27], v[136:143], v[208:215], v[24:27] blgp:1\n\t" \
  "buffer_load_dwordx4 v252, s[40:43], s52 offen offset:0 sc0 sc1 lds\n\t" \
  "v_mul_f32 v245, s58, v241\n\t" \
  "v_mfma_f32_16x16x128_f8f6f4 v[20:23], v[136:143], v[200:207], v[20:23] blgp:1\n\t" \
  "buffer_load_dwordx4 v252, s[40:43], s52 offen offset:1024 sc0 sc1 lds\n\t" \
  "v_mul_f32 v246, s58, v242\n\t" \
  "v_mfma_f32_16x16x128_f8f6f4 v[16:19], v[136:143], v[192:199], v[16:19] blgp:1\n\t" \
  "ds_read_b128 v[136:139], v250 offset:16640\n\t" \
  "v_mul_f32 v247, s58, v243\n\t" \
  "v_mfma_f32_16x16x128_f8f6f4 v[32:35], v[144:151], v[192:199], v[32:35] blgp:1\n\t" \
  "ds_read_b128 v[140:143], v250 offset:24832\n\t" \
  "v_med3_f32 v244, v244, -s62, s62\n\t" \
  "v_mfma_f32_16x16x128_f8f6f4 v[36:39], v[144:151], v[200:207], v[36:39] blgp:1\n\t" \
  "buffer_load_dwordx4 v252, s[40:43], s52 offen offset:2048 sc0 sc1 lds\n\t" \
  "v_med3_f32 v245, v245, -s62, s62\n\t" \
  "v_mfma_f32_16x16x128_f8f6f4 v[40:43], v[144:151], v[208:215], v[40:43] blgp:1\n\t" \
  "buffer_load_dwordx4 v252, s[40:43], s52 offen offset:3072 sc0 sc1 lds\n\t" \
  "v_med3_f32 v246, v246, -s62, s62\n\t" \
  "v_mfma_f32_16x16x128_f8f6f4 v[44:47], v[144:151], v[216:223], v[44:47] blgp:1\n\t" \
  "ds_read_b128 v[144:147], v250 offset:16896\n\t" \
  "v_med3_f32 v247, v247, -s62, s62\n\t" \
  "v_mfma_f32_16x16x128_f8f6f4 v[60:63], v[152:159], v[216:223], v[60:63] blgp:1\n\t" \
  "ds_read_b128 v[148:151], v250 offset:25088\n\t" \
  "v_exp_f32 v244, v244\n\t" \
  "v_mfma_f32_16x16x128_f8f6f4 v[56:59], v[152:159], v[208:215], v[56:59] blgp:1\n\t" \
  "v_exp_f32 v245, v245\n\t" \
  "v_mfma_f32_16x16x128_f8f6f4 v[52:55], v[152:159], v[200:207], v[52:55] blgp:1\n\t" \
  "v_exp_f32 v246, v246\n\t" \
  "v_mfma_f32_16x16x128_f8f6f4 v[48:51], v[152:159], v[192:199], v[48:51] blgp:1\n\t" \
  "ds_read_b128 v[152:155], v250 offset:17152\n\t" \
  "ds_read_b128 v[156:159], v250 offset:25344\n\t" \
  "s_waitcnt lgkmcnt(2)\n\t" \
  "v_mfma_f32_16x16x128_f8f6f4 a[64:67], v[128:135], v[192:199], a[64:67] blgp:1\n\t" \
  "s_add_i32 s52, s49, 0\n\t" \
  "s_add_u32 m0, s53, 98304\n\t" \
  "v_exp_f32 v247, v247\n\t" \
  "v_mfma_f32_16x16x128_f8f6f4 a[68:71], v[128:135], v[200:207], a[68:71] blgp:1\n\t" \
  "buffer_load_dwordx4 v252, s[40:43], s52 offen offset:0 sc0 sc1 lds\n\t" \
  "v_mfma_f32_16x16x128_f8f6f4 a[72:75], v[128:135], v[208:215], a[72:75] blgp:1\n\t" \
  "buffer_load_dwordx4 v252, s[40:43], s52 offen offset:1024 sc0 sc1 lds\n\t" \
  "v_fma_f32 v244, v244, s60, s60\n\t" \
  "v_mfma_f32_16x16x128_f8f6f4 a[76:79], v[128:135], v[216:223], a[76:79] blgp:1\n\t" \
  "v_fma_f32 v245, v245, s60, s60\n\t" \
  "v_mfma_f32_16x16x128_f8f6f4 a[92:95], v[136:143], v[216:223], a[92:95] blgp:1\n\t" \
  "buffer_load_dwordx4 v252, s[40:43], s52 offen offset:2048 sc0 sc1 lds\n\t" \
  "v_fma_f32 v246, v246, s60, s60\n\t" \
  "v_mfma_f32_16x16x128_f8f6f4 a[88:91], v[136:143], v[208:215], a[88:91] blgp:1\n\t" \
  "buffer_load_dwordx4 v252, s[40:43], s52 offen offset:3072 sc0 sc1 lds\n\t" \
  "v_fma_f32 v247, v247, s60, s60\n\t" \
  "v_mfma_f32_16x16x128_f8f6f4 a[84:87], v[136:143], v[200:207], a[84:87] blgp:1\n\t" \
  "s_add_i32 s52, s49, 16384\n\t" \
  "s_add_u32 m0, s53, 114688\n\t" \
  "v_rcp_f32 v244, v244\n\t" \
  "v_mfma_f32_16x16x128_f8f6f4 a[80:83], v[136:143], v[192:199], a[80:83] blgp:1\n\t" \
  "buffer_load_dwordx4 v252, s[40:43], s52 offen offset:0 sc0 sc1 lds\n\t" \
  "v_rcp_f32 v245, v245\n\t" \
  "v_mfma_f32_16x16x128_f8f6f4 a[96:99], v[144:151], v[192:199], a[96:99] blgp:1\n\t" \
  "buffer_load_dwordx4 v252, s[40:43], s52 offen offset:1024 sc0 sc1 lds\n\t" \
  "v_rcp_f32 v246, v246\n\t" \
  "v_mfma_f32_16x16x128_f8f6f4 a[100:103], v[144:151], v[200:207], a[100:103] blgp:1\n\t" \
  "v_rcp_f32 v247, v247\n\t" \
  "v_mfma_f32_16x16x128_f8f6f4 a[104:107], v[144:151], v[208:215], a[104:107] blgp:1\n\t" \
  "buffer_load_dwordx4 v252, s[40:43], s52 offen offset:2048 sc0 sc1 lds\n\t" \
  "v_mul_f32 v244, v240, v244\n\t" \
  "v_mfma_f32_16x16x128_f8f6f4 a[108:111], v[144:151], v[216:223], a[108:111] blgp:1\n\t" \
  "buffer_load_dwordx4 v252, s[40:43], s52 offen offset:3072 sc0 sc1 lds\n\t" \
  "v_mul_f32 v245, v241, v245\n\t" \
  "s_waitcnt lgkmcnt(0)\n\t" \
  "v_mfma_f32_16x16x128_f8f6f4 a[124:127], v[152:159], v[216:223], a[124:127] blgp:1\n\t" \
  "v_mul_f32 v246, v242, v246\n\t" \
  "v_mfma_f32_16x16x128_f8f6f4 a[120:123], v[152:159], v[208:215], a[120:123] blgp:1\n\t" \
  "v_mul_f32 v247, v243, v247\n\t" \
  "v_mfma_f32_16x16x128_f8f6f4 a[116:119], v[152:159], v[200:207], a[116:119] blgp:1\n\t" \
  "v_mfma_f32_16x16x128_f8f6f4 a[112:115], v[152:159], v[192:199], a[112:115] blgp:1\n\t" \
  "s_waitcnt vmcnt(19) lgkmcnt(0)\n\t" \
  "s_barrier\n\t" \
  "v_mfma_f32_16x16x128_f8f6f4 v[64:67], v[128:135], v[160:167], v[64:67] blgp:1\n\t" \
  "ds_read_b128 v[192:195], v251 offset:49152\n\t" \
  "v_mfma_f32_16x16x128_f8f6f4 v[68:71], v[128:135], v[168:175], v[68:71] blgp:1\n\t" \
  "ds_read_b128 v[196:199], v251 offset:57344\n\t" \
  "s_add_u32 s57, s80, 1245184\n\t" \
  "v_mfma_f32_16x16x128_f8f6f4 v[72:75], v[128:135], v[176:183], v[72:75] blgp:1\n\t" \
  "buffer_store_dwordx4 v[244:247], v253, s[44:47], s57 offen offset:0 nt\n\t" \
  "ds_read_b128 v[200:203], v251 offset:49408\n\t" \
  "v_mfma_f32_16x16x128_f8f6f4 v[76:79], v[128:135], v[184:191], v[76:79] blgp:1\n\t" \
  "ds_read_b128 v[128:131], v250 offset:32768\n\t" \
  "ds_read_b128 v[132:135], v250 offset:40960\n\t" \
  "v_mfma_f32_16x16x128_f8f6f4 v[92:95], v[136:143], v[184:191], v[92:95] blgp:1\n\t" \
  "ds_read_b128 v[204:207], v251 offset:57600\n\t" \
  "v_mul_f32 v244, s58, v224\n\t" \
  "v_mfma_f32_16x16x128_f8f6f4 v[88:91], v[136:143], v[176:183], v[88:91] blgp:1\n\t" \
  "ds_read_b128 v[208:211], v251 offset:49664\n\t" \
  "v_mul_f32 v245, s58, v228\n\t" \
  "v_mfma_f32_16x16x128_f8f6f4 v[84:87], v[136:143], v[168:175], v[84:87] blgp:1\n\t" \
  "ds_read_b128 v[212:215], v251 offset:57856\n\t" \
  "v_mul_f32 v246, s58, v232\n\t" \
  "v_mfma_f32_16x16x128_f8f6f4 v[80:83], v[136:143], v[160:167], v[80:83] blgp:1\n\t" \
  "ds_read_b128 v[136:139], v250 offset:33024\n\t" \
  "ds_read_b128 v[140:143], v250 offset:41216\n\t" \
  "v_mul_f32 v247, s58, v236\n\t" \
  "v_mfma_f32_16x16x128_f8f6f4 v[96:99], v[144:151], v[160:167], v[96:99] blgp:1\n\t" \
  "ds_read_b128 v[216:219], v251 offset:49920\n\t" \
  "v_med3_f32 v244, v244, -s62, s62\n\t" \
  "v_mfma_f32_16x16x128_f8f6f4 v[100:103], v[144:151], v[168:175], v[100:103] blgp:1\n\t" \
  "ds_read_b128 v[220:223], v251 offset:58112\n\t" \
  "v_med3_f32 v245, v245, -s62, s62\n\t" \
  "v_mfma_f32_16x16x128_f8f6f4 v[104:107], v[144:151], v[176:183], v[104:107] blgp:1\n\t" \
  "v_med3_f32 v246, v246, -s62, s62\n\t" \
  "v_mfma_f32_16x16x128_f8f6f4 v[108:111], v[144:151], v[184:191], v[108:111] blgp:1\n\t" \
  "ds_read_b128 v[144:147], v250 offset:33280\n\t" \
  "v_med3_f32 v247, v247, -s62, s62\n\t" \
  "v_mfma_f32_16x16x128_f8f6f4 v[124:127], v[152:159], v[184:191], v[124:127] blgp:1\n\t" \
  "ds_read_b128 v[148:151], v250 offset:41472\n\t" \
  "v_exp_f32 v244, v244\n\t" \
  "v_mfma_f32_16x16x128_f8f6f4 v[120:123], v[152:159], v[176:183], v[120:123] blgp:1\n\t" \
  "v_exp_f32 v245, v245\n\t" \
  "v_mfma_f32_16x16x128_f8f6f4 v[116:119], v[152:159], v[168:175], v[116:119] blgp:1\n\t" \
  "v_exp_f32 v246, v246\n\t" \
  "v_mfma_f32_16x16x128_f8f6f4 v[112:115], v[152:159], v[160:167], v[112:115] blgp:1\n\t" \
  "ds_read_b128 v[152:155], v250 offset:33536\n\t" \
  "ds_read_b128 v[156:159], v250 offset:41728\n\t" \
  "s_waitcnt lgkmcnt(2)\n\t" \
  "v_mfma_f32_16x16x128_f8f6f4 v[0:3], v[128:135], v[192:199], v[0:3] blgp:1\n\t" \
  "ds_read_b128 v[160:163], v251 offset:32768\n\t" \
  "v_exp_f32 v247, v247\n\t" \
  "v_mfma_f32_16x16x128_f8f6f4 v[4:7], v[128:135], v[200:207], v[4:7] blgp:1\n\t" \
  "ds_read_b128 v[164:167], v251 offset:40960\n\t" \
  "v_mfma_f32_16x16x128_f8f6f4 v[8:11], v[128:135], v[208:215], v[8:11] blgp:1\n\t" \
  "ds_read_b128 v[168:171], v251 offset:33024\n\t" \
  "v_fma_f32 v244, v244, s60, s60\n\t" \
  "v_mfma_f32_16x16x128_f8f6f4 v[12:15], v[128:135], v[216:223], v[12:15] blgp:1\n\t" \
  "ds_read_b128 v[172:175], v251 offset:41216\n\t" \
  "v_fma_f32 v245, v245, s60, s60\n\t" \
  "v_mfma_f32_16x16x128_f8f6f4 v[28:31], v[136:143], v[216:223], v[28:31] blgp:1\n\t" \
  "ds_read_b128 v[176:179], v251 offset:33280\n\t" \
  "v_fma_f32 v246, v246, s60, s60\n\t" \
  "v_mfma_f32_16x16x128_f8f6f4 v[24:27], v[136:143], v[208:215], v[24:27] blgp:1\n\t" \
  "ds_read_b128 v[180:183], v251 offset:41472\n\t" \
  "v_fma_f32 v247, v247, s60, s60\n\t" \
  "v_mfma_f32_16x16x128_f8f6f4 v[20:23], v[136:143], v[200:207], v[20:23] blgp:1\n\t" \
  "ds_read_b128 v[184:187], v251 offset:33536\n\t" \
  "v_rcp_f32 v244, v244\n\t" \
  "v_mfma_f32_16x16x128_f8f6f4 v[16:19], v[136:143], v[192:199], v[16:19] blgp:1\n\t" \
  "ds_read_b128 v[188:191], v251 offset:41728\n\t" \
  "s_add_i32 s52, s48, 16384\n\t" \
  "s_add_u32 m0, s53, 49152\n\t" \
  "v_rcp_f32 v245, v245\n\t" \
  "v_mfma_f32_16x16x128_f8f6f4 v[32:35], v[144:151], v[192:199], v[32:35] blgp:1\n\t" \
  "buffer_load_dwordx4 v252, s[40:43], s52 offen offset:0 sc0 sc1 lds\n\t" \
  "v_rcp_f32 v246, v246\n\t" \
  "v_mfma_f32_16x16x128_f8f6f4 v[36:39], v[144:151], v[200:207], v[36:39] blgp:1\n\t" \
  "v_rcp_f32 v247, v247\n\t" \
  "v_mfma_f32_16x16x128_f8f6f4 v[40:43], v[144:151], v[208:215], v[40:43] blgp:1\n\t" \
  "buffer_load_dwordx4 v252, s[40:43], s52 offen offset:1024 sc0 sc1 lds\n\t" \
  "v_mul_f32 v244, v224, v244\n\t" \
  "v_mfma_f32_16x16x128_f8f6f4 v[44:47], v[144:151], v[216:223], v[44:47] blgp:1\n\t" \
  "v_mul_f32 v245, v228, v245\n\t" \
  "s_waitcnt lgkmcnt(0)\n\t" \
  "v_mfma_f32_16x16x128_f8f6f4 v[60:63], v[152:159], v[216:223], v[60:63] blgp:1\n\t" \
  "buffer_load_dwordx4 v252, s[40:43], s52 offen offset:2048 sc0 sc1 lds\n\t" \
  "v_mul_f32 v246, v232, v246\n\t" \
  "v_mfma_f32_16x16x128_f8f6f4 v[56:59], v[152:159], v[208:215], v[56:59] blgp:1\n\t" \
  "v_mul_f32 v247, v236, v247\n\t" \
  "v_mfma_f32_16x16x128_f8f6f4 v[52:55], v[152:159], v[200:207], v[52:55] blgp:1\n\t" \
  "buffer_load_dwordx4 v252, s[40:43], s52 offen offset:3072 sc0 sc1 lds\n\t" \
  "v_mfma_f32_16x16x128_f8f6f4 v[48:51], v[152:159], v[192:199], v[48:51] blgp:1\n\t" \
  "s_waitcnt vmcnt(18) lgkmcnt(0)\n\t" \
  "s_barrier\n\t" \
  "v_mfma_f32_16x16x128_f8f6f4 a[0:3], v[128:135], v[160:167], a[0:3] blgp:1\n\t" \
  "v_readfirstlane_b32 s87, v254\n\t" \
  "v_readlane_b32 s65, v255, s67\n\t" \
  "v_mfma_f32_16x16x128_f8f6f4 a[4:7], v[128:135], v[168:175], a[4:7] blgp:1\n\t" \
  "s_add_u32 s57, s80, 3145728\n\t" \
  "v_mfma_f32_16x16x128_f8f6f4 a[8:11], v[128:135], v[176:183], a[8:11] blgp:1\n\t" \
  "buffer_store_dwordx4 v[244:247], v253, s[44:47], s57 offen offset:256 nt\n\t" \
  "s_max_u32 s69, s69, s87\n\t" \
  "s_add_u32 s67, s67, 1\n\t" \
  "s_cmp_ge_u32 s69, s65\n\t" \
  "s_cbranch_scc0 L_w8_%=\n\t" \
  "L_ok8_%=:\n\t" \
  "v_mfma_f32_16x16x128_f8f6f4 a[12:15], v[128:135], v[184:191], a[12:15] blgp:1\n\t" \
  "ds_read_b128 v[128:131], v250 offset:49152\n\t" \
  "v_mfma_f32_16x16x128_f8f6f4 a[28:31], v[136:143], v[184:191], a[28:31] blgp:1\n\t" \
  "s_add_i32 s52, s48, 32768\n\t" \
  "s_add_u32 m0, s53, 65536\n\t" \
  "ds_read_b128 v[132:135], v250 offset:57344\n\t" \
  "v_mul_f32 v244, s58, v225\n\t" \
  "v_mfma_f32_16x16x128_f8f6f4 a[24:27], v[136:143], v[176:183], a[24:27] blgp:1\n\t" \
  "buffer_load_dwordx4 v252, s[40:43], s52 offen offset:0 sc0 sc1 lds\n\t" \
  "v_mul_f32 v245, s58, v229\n\t" \
  "v_mfma_f32_16x16x128_f8f6f4 a[20:23], v[136:143], v[168:175], a[20:23] blgp:1\n\t" \
  "buffer_load_dwordx4 v252, s[40:43], s52 offen offset:1024 sc0 sc1 lds\n\t" \
  "v_mul_f32 v246, s58, v233\n\t" \
  "v_mfma_f32_16x16x128_f8f6f4 a[16:19], v[136:143], v[160:167], a[16:19] blgp:1\n\t" \
  "ds_read_b128 v[136:139], v250 offset:49408\n\t" \
  "v_mul_f32 v247, s58, v237\n\t" \
  "v_mfma_f32_16x16x128_f8f6f4 a[32:35], v[144:151], v[160:167], a[32:35] blgp:1\n\t" \
  "ds_read_b128 v[140:143], v250 offset:57600\n\t" \
  "v_med3_f32 v244, v244, -s62, s62\n\t" \
  "v_mfma_f32_16x16x128_f8f6f4 a[36:39], v[144:151], v[168:175], a[36:39] blgp:1\n\t" \
  "buffer_load_dwordx4 v252, s[40:43], s52 offen offset:2048 sc0 sc1 lds\n\t" \
  "v_med3_f32 v245, v245, -s62, s62\n\t" \
  "v_mfma_f32_16x16x128_f8f6f4 a[40:43], v[144:151], v[176:183], a[40:43] blgp:1\n\t" \
  "buffer_load_dwordx4 v252, s[40:43], s52 offen offset:3072 sc0 sc1 lds\n\t" \
  "v_med3_f32 v246, v246, -s62, s62\n\t" \
  "v_mfma_f32_16x16x128_f8f6f4 a[44:47], v[144:151], v[184:191], a[44:47] blgp:1\n\t" \
  "ds_read_b128 v[144:147], v250 offset:49664\n\t" \
  "v_med3_f32 v247, v247, -s62, s62\n\t" \
  "v_mfma_f32_16x16x128_f8f6f4 a[60:63], v[152:159], v[184:191], a[60:63] blgp:1\n\t" \
  "ds_read_b128 v[148:151], v250 offset:57856\n\t" \
  "v_exp_f32 v244, v244\n\t" \
  "v_mfma_f32_16x16x128_f8f6f4 a[56:59], v[152:159], v[176:183], a[56:59] blgp:1\n\t" \
  "v_exp_f32 v245, v245\n\t" \
  "v_mfma_f32_16x16x128_f8f6f4 a[52:55], v[152:159], v[168:175], a[52:55] blgp:1\n\t" \
  "v_exp_f32 v246, v246\n\t" \
  "v_mfma_f32_16x16x128_f8f6f4 a[48:51], v[152:159], v[160:167], a[48:51] blgp:1\n\t" \
  "ds_read_b128 v[152:155], v250 offset:49920\n\t" \
  "ds_read_b128 v[156:159], v250 offset:58112\n\t" \
  "s_waitcnt lgkmcnt(2)\n\t" \
  "v_mfma_f32_16x16x128_f8f6f4 v[64:67], v[128:135], v[160:167], v[64:67] blgp:1\n\t" \
  "s_add_i32 s52, s49, 49152\n\t" \
  "s_add_u32 m0, s53, 147456\n\t" \
  "v_exp_f32 v247, v247\n\t" \
  "v_mfma_f32_16x16x128_f8f6f4 v[68:71], v[128:135], v[168:175], v[68:71] blgp:1\n\t" \
  "buffer_load_dwordx4 v252, s[40:43], s52 offen offset:0 sc0 sc1 lds\n\t" \
  "v_mfma_f32_16x16x128_f8f6f4 v[72:75], v[128:135], v[176:183], v[72:75] blgp:1\n\t" \
  "buffer_load_dwordx4 v252, s[40:43], s52 offen offset:1024 sc0 sc1 lds\n\t" \
  "v_fma_f32 v244, v244, s60, s60\n\t" \
  "v_mfma_f32_16x16x128_f8f6f4 v[76:79], v[128:135], v[184:191], v[76:79] blgp:1\n\t" \
  "v_fma_f32 v245, v245, s60, s60\n\t" \
  "v_mfma_f32_16x16x128_f8f6f4 v[92:95], v[136:143], v[184:191], v[92:95] blgp:1\n\t" \
  "buffer_load_dwordx4 v252, s[40:43], s52 offen offset:2048 sc0 sc1 lds\n\t" \
  "v_fma_f32 v246, v246, s60, s60\n\t" \
  "v_mfma_f32_16x16x128_f8f6f4 v[88:91], v[136:143], v[176:183], v[88:91] blgp:1\n\t" \
  "buffer_load_dwordx4 v252, s[40:43], s52 offen offset:3072 sc0 sc1 lds\n\t" \
  "v_fma_f32 v247, v247, s60, s60\n\t" \
  "v_mfma_f32_16x16x128_f8f6f4 v[84:87], v[136:143], v[168:175], v[84:87] blgp:1\n\t" \
  "s_add_i32 s52, s49, 32768\n\t" \
  "s_add_u32 m0, s53, 131072\n\t" \
  "v_rcp_f32 v244, v244\n\t" \
  "v_mfma_f32_16x16x128_f8f6f4 v[80:83], v[136:143], v[160:167], v[80:83] blgp:1\n\t" \
  "buffer_load_dwordx4 v252, s[40:43], s52 offen offset:0 sc0 sc1 lds\n\t" \
  "v_rcp_f32 v245, v245\n\t" \
  "v_mfma_f32_16x16x128_f8f6f4 v[96:99], v[144:151], v[160:167], v[96:99] blgp:1\n\t" \
  "buffer_load_dwordx4 v252, s[40:43], s52 offen offset:1024 sc0 sc1 lds\n\t" \
  "v_rcp_f32 v246, v246\n\t" \
  "v_mfma_f32_16x16x128_f8f6f4 v[100:103], v[144:151], v[168:175], v[100:103] blgp:1\n\t" \
  "v_rcp_f32 v247, v247\n\t" \
  "v_mfma_f32_16x16x128_f8f6f4 v[104:107], v[144:151], v[176:183], v[104:107] blgp:1\n\t" \
  "buffer_load_dwordx4 v252, s[40:43], s52 offen offset:2048 sc0 sc1 lds\n\t" \
  "v_mul_f32 v244, v225, v244\n\t" \
  "v_mfma_f32_16x16x128_f8f6f4 v[108:111], v[144:151], v[184:191], v[108:111] blgp:1\n\t" \
  "buffer_load_dwordx4 v252, s[40:43], s52 offen offset:3072 sc0 sc1 lds\n\t" \
  "v_mul_f32 v245, v229, v245\n\t" \
  "s_waitcnt lgkmcnt(0)\n\t" \
  "v_mfma_f32_16x16x128_f8f6f4 v[124:127], v[152:159], v[184:191], v[124:127] blgp:1\n\t" \
  "v_mul_f32 v246, v233, v246\n\t" \
  "v_mfma_f32_16x16x128_f8f6f4 v[120:123], v[152:159], v[176:183], v[120:123] blgp:1\n\t" \
  "v_mul_f32 v247, v237, v247\n\t" \
  "v_mfma_f32_16x16x128_f8f6f4 v[116:119], v[152:159], v[168:175], v[116:119] blgp:1\n\t" \
  "v_mfma_f32_16x16x128_f8f6f4 v[112:115], v[152:159], v[160:167], v[112:115] blgp:1\n\t" \
  "s_waitcnt vmcnt(18) lgkmcnt(0)\n\t" \
  "s_barrier\n\t" \
  "v_mfma_f32_16x16x128_f8f6f4 a[64:67], v[128:135], v[192:199], a[64:67] blgp:1\n\t" \
  "ds_read_b128 v[160:163], v251 offset:0\n\t" \
  "v_mfma_f32_16x16x128_f8f6f4 a[68:71], v[128:135], v[200:207], a[68:71] blgp:1\n\t" \
  "ds_read_b128 v[164:167], v251 offset:8192\n\t" \
  "s_add_u32 s57, s80, 3211264\n\t" \
  "v_mfma_f32_16x16x128_f8f6f4 a[72:75], v[128:135], v[208:215], a[72:75] blgp:1\n\t" \
  "buffer_store_dwordx4 v[244:247], v253, s[44:47], s57 offen offset:256 nt\n\t" \
  "ds_read_b128 v[168:171], v251 offset:256\n\t" \
  "v_mfma_f32_16x16x128_f8f6f4 a[76:79], v[128:135], v[216:223], a[76:79] blgp:1\n\t" \
  "ds_read_b128 v[128:131], v250 offset:0\n\t" \
  "ds_read_b128 v[132:135], v250 offset:8192\n\t" \
  "v_mfma_f32_16x16x128_f8f6f4 a[92:95], v[136:143], v[216:223], a[92:95] blgp:1\n\t" \
  "ds_read_b128 v[172:175], v251 offset:8448\n\t" \
  "v_mul_f32 v244, s58, v226\n\t" \
  "v_mfma_f32_16x16x128_f8f6f4 a[88:91], v[136:143], v[208:215], a[88:91] blgp:1\n\t" \
  "ds_read_b128 v[176:179], v251 offset:512\n\t" \
  "v_mul_f32 v245, s58, v230\n\t" \
  "v_mfma_f32_16x16x128_f8f6f4 a[84:87], v[136:143], v[200:207], a[84:87] blgp:1\n\t" \
  "ds_read_b128 v[180:183], v251 offset:8704\n\t" \
  "v_mul_f32 v246, s58, v234\n\t" \
  "v_mfma_f32_16x16x128_f8f6f4 a[80:83], v[136:143], v[192:199], a[80:83] blgp:1\n\t" \
  "ds_read_b128 v[136:139], v250 offset:256\n\t" \
  "ds_read_b128 v[140:143], v250 offset:8448\n\t" \
  "v_mul_f32 v247, s58, v238\n\t" \
  "v_mfma_f32_16x16x128_f8f6f4 a[96:99], v[144:151], v[192:199], a[96:99] blgp:1\n\t" \
  "ds_read_b128 v[184:187], v251 offset:768\n\t" \
  "v_med3_f32 v244, v244, -s62, s62\n\t" \
  "v_mfma_f32_16x16x128_f8f6f4 a[100:103], v[144:151], v[200:207], a[100:103] blgp:1\n\t" \
  "ds_read_b128 v[188:191], v251 offset:8960\n\t" \
  "v_med3_f32 v245, v245, -s62, s62\n\t" \
  "v_mfma_f32_16x16x128_f8f6f4 a[104:107], v[144:151], v[208:215], a[104:107] blgp:1\n\t" \
  "s_add_u32 s48, s48, 0x10000\n\t" \
  "s_add_u32 s49, s49, 0x10000\n\t" \
  "v_med3_f32 v246, v246, -s62, s62\n\t" \
  "v_mfma_f32_16x16x128_f8f6f4 a[108:111], v[144:151], v[216:223], a[108:111] blgp:1\n\t" \
  "ds_read_b128 v[144:147], v250 offset:512\n\t" \
  "v_med3_f32 v247, v247, -s62, s62\n\t" \
  "v_mfma_f32_16x16x128_f8f6f4 a[124:127], v[152:159], v[216:223], a[124:127] blgp:1\n\t" \
  "ds_read_b128 v[148:151], v250 offset:8704\n\t" \
  "v_exp_f32 v244, v244\n\t" \
  "v_mfma_f32_16x16x128_f8f6f4 a[120:123], v[152:159], v[208:215], a[120:123] blgp:1\n\t" \
  "v_exp_f32 v245, v245\n\t" \
  "v_mfma_f32_16x16x128_f8f6f4 a[116:119], v[152:159], v[200:207], a[116:119] blgp:1\n\t" \
  "v_exp_f32 v246, v246\n\t" \
  "v_mfma_f32_16x16x128_f8f6f4 a[112:115], v[152:159], v[192:199], a[112:115] blgp:1\n\t" \
  "ds_read_b128 v[152:155], v250 offset:768\n\t" \
  "ds_read_b128 v[156:159], v250 offset:8960\n\t" \
  "s_waitcnt lgkmcnt(2)\n\t" \
  "v_mfma_f32_16x16x128_f8f6f4 a[0:3], v[128:135], v[160:167], a[0:3] blgp:1\n\t" \
  "ds_read_b128 v[192:195], v251 offset:16384\n\t" \
  "v_exp_f32 v247, v247\n\t" \
  "v_mfma_f32_16x16x128_f8f6f4 a[4:7], v[128:135], v[168:175], a[4:7] blgp:1\n\t" \
  "ds_read_b128 v[196:199], v251 offset:24576\n\t" \
  "v_mfma_f32_16x16x128_f8f6f4 a[8:11], v[128:135], v[176:183], a[8:11] blgp:1\n\t" \
  "ds_read_b128 v[200:203], v251 offset:16640\n\t" \
  "v_fma_f32 v244, v244, s60, s60\n\t" \
  "v_mfma_f32_16x16x128_f8f6f4 a[12:15], v[128:135], v[184:191], a[12:15] blgp:1\n\t" \
  "ds_read_b128 v[204:207], v251 offset:24832\n\t" \
  "v_fma_f32 v245, v245, s60, s60\n\t" \
  "v_mfma_f32_16x16x128_f8f6f4 a[28:31], v[136:143], v[184:191], a[28:31] blgp:1\n\t" \
  "ds_read_b128 v[208:211], v251 offset:16896\n\t" \
  "v_fma_f32 v246, v246, s60, s60\n\t" \
  "v_mfma_f32_16x16x128_f8f6f4 a[24:27], v[136:143], v[176:183], a[24:27] blgp:1\n\t" \
  "ds_read_b128 v[212:215], v251 offset:25088\n\t" \
  "v_fma_f32 v247, v247, s60, s60\n\t" \
  "v_mfma_f32_16x16x128_f8f6f4 a[20:23], v[136:143], v[168:175], a[20:23] blgp:1\n\t" \
  "ds_read_b128 v[216:219], v251 offset:17152\n\t" \
  "v_rcp_f32 v244, v244\n\t" \
  "v_mfma_f32_16x16x128_f8f6f4 a[16:19], v[136:143], v[160:167], a[16:19] blgp:1\n\t" \
  "ds_read_b128 v[220:223], v251 offset:25344\n\t" \
  "s_add_i32 s52, s48, -16384\n\t" \
  "s_add_u32 m0, s53, 81920\n\t" \
  "v_rcp_f32 v245, v245\n\t" \
  "v_mfma_f32_16x16x128_f8f6f4 a[32:35], v[144:151], v[160:167], a[32:35] blgp:1\n\t" \
  "buffer_load_dwordx4 v252, s[40:43], s52 offen offset:0 sc0 sc1 lds\n\t" \
  "v_rcp_f32 v246, v246\n\t" \
  "v_mfma_f32_16x16x128_f8f6f4 a[36:39], v[144:151], v[168:175], a[36:39] blgp:1\n\t" \
  "v_rcp_f32 v247, v247\n\t" \
  "v_mfma_f32_16x16x128_f8f6f4 a[40:43], v[144:151], v[176:183], a[40:43] blgp:1\n\t" \
  "buffer_load_dwordx4 v252, s[40:43], s52 offen offset:1024 sc0 sc1 lds\n\t" \
  "v_mul_f32 v244, v226, v244\n\t" \
  "v_mfma_f32_16x16x128_f8f6f4 a[44:47], v[144:151], v[184:191], a[44:47] blgp:1\n\t" \
  "v_mul_f32 v245, v230, v245\n\t" \
  "s_waitcnt lgkmcnt(0)\n\t" \
  "v_mfma_f32_16x16x128_f8f6f4 a[60:63], v[152:159], v[184:191], a[60:63] blgp:1\n\t" \
  "buffer_load_dwordx4 v252, s[40:43], s52 offen offset:2048 sc0 sc1 lds\n\t" \
  "v_mul_f32 v246, v234, v246\n\t" \
  "v_mfma_f32_16x16x128_f8f6f4 a[56:59], v[152:159], v[176:183], a[56:59] blgp:1\n\t" \
  "v_mul_f32 v247, v238, v247\n\t" \
  "v_mfma_f32_16x16x128_f8f6f4 a[52:55], v[152:159], v[168:175], a[52:55] blgp:1\n\t" \
  "buffer_load_dwordx4 v252, s[40:43], s52 offen offset:3072 sc0 sc1 lds\n\t" \
  "buffer_load_dword v254, off, s[88:91], s81 sc1\n\t" \
  "v_mfma_f32_16x16x128_f8f6f4 a[48:51], v[152:159], v[160:167], a[48:51] blgp:1\n\t" \
  "s_waitcnt vmcnt(19) lgkmcnt(0)\n\t" \
  "s_barrier\n\t" \
  "v_mfma_f32_16x16x128_f8f6f4 v[0:3], v[128:135], v[192:199], v[0:3] blgp:1\n\t" \
  "v_readlane_b32 s65, v255, s67\n\t" \
  "v_mfma_f32_16x16x128_f8f6f4 v[4:7], v[128:135], v[200:207], v[4:7] blgp:1\n\t" \
  "s_add_u32 s57, s80, 3276800\n\t" \
  "v_mfma_f32_16x16x128_f8f6f4 v[8:11], v[128:135], v[208:215], v[8:11] blgp:1\n\t" \
  "buffer_store_dwordx4 v[244:247], v253, s[44:47], s57 offen offset:256 nt\n\t" \
  "s_add_u32 s67, s67, 1\n\t" \
  "s_cmp_ge_u32 s69, s65\n\t" \
  "s_cbranch_scc0 L_w9_%=\n\t" \
  "L_ok9_%=:\n\t" \
  "v_mfma_f32_16x16x128_f8f6f4 v[12:15], v[128:135], v[216:223], v[12:15] blgp:1\n\t" \
  "ds_read_b128 v[128:131], v250 offset:16384\n\t" \
  "v_mfma_f32_16x16x128_f8f6f4 v[28:31], v[136:143], v[216:223], v[28:31] blgp:1\n\t" \
  "s_add_i32 s52, s48, 0\n\t" \
  "s_add_u32 m0, s53, 32768\n\t" \
  "ds_read_b128 v[132:135], v250 offset:24576\n\t" \
  "v_mul_f32 v244, s58, v227\n\t" \
  "v_mfma_f32_16x16x128_f8f6f4 v[24:27], v[136:143], v[208:215], v[24:27] blgp:1\n\t" \
  "buffer_load_dwordx4 v252, s[40:43], s52 offen offset:0 sc0 sc1 lds\n\t" \
  "v_mul_f32 v245, s58, v231\n\t" \
  "v_mfma_f32_16x16x128_f8f6f4 v[20:23], v[136:143], v[200:207], v[20:23] blgp:1\n\t" \
  "buffer_load_dwordx4 v252, s[40:43], s52 offen offset:1024 sc0 sc1 lds\n\t" \
  "v_mul_f32 v246, s58, v235\n\t" \
  "v_mfma_f32_16x16x128_f8f6f4 v[16:19], v[136:143], v[192:199], v[16:19] blgp:1\n\t" \
  "ds_read_b128 v[136:139], v250 offset:16640\n\t" \
  "v_mul_f32 v247, s58, v239\n\t" \
  "v_mfma_f32_16x16x128_f8f6f4 v[32:35], v[144:151], v[192:199], v[32:35] blgp:1\n\t" \
  "ds_read_b128 v[140:143], v250 offset:24832\n\t" \
  "v_med3_f32 v244, v244, -s62, s62\n\t" \
  "v_mfma_f32_16x16x128_f8f6f4 v[36:39], v[144:151], v[200:207], v[36:39] blgp:1\n\t" \
  "buffer_load_dwordx4 v252, s[40:43], s52 offen offset:2048 sc0 sc1 lds\n\t" \
  "v_med3_f32 v245, v245, -s62, s62\n\t" \
  "v_mfma_f32_16x16x128_f8f6f4 v[40:43], v[144:151], v[208:215], v[40:43] blgp:1\n\t" \
  "buffer_load_dwordx4 v252, s[40:43], s52 offen offset:3072 sc0 sc1 lds\n\t" \
  "v_med3_f32 v246, v246, -s62, s62\n\t" \
  "v_mfma_f32_16x16x128_f8f6f4 v[44:47], v[144:151], v[216:223], v[44:47] blgp:1\n\t" \
  "ds_read_b128 v[144:147], v250 offset:16896\n\t" \
  "v_med3_f32 v247, v247, -s62, s62\n\t" \
  "v_mfma_f32_16x16x128_f8f6f4 v[60:63], v[152:159], v[216:223], v[60:63] blgp:1\n\t" \
  "ds_read_b128 v[148:151], v250 offset:25088\n\t" \
  "v_exp_f32 v244, v244\n\t" \
  "v_mfma_f32_16x16x128_f8f6f4 v[56:59], v[152:159], v[208:215], v[56:59] blgp:1\n\t" \
  "v_exp_f32 v245, v245\n\t" \
  "v_mfma_f32_16x16x128_f8f6f4 v[52:55], v[152:159], v[200:207], v[52:55] blgp:1\n\t" \
  "v_exp_f32 v246, v246\n\t" \
  "v_mfma_f32_16x16x128_f8f6f4 v[48:51], v[152:159], v[192:199], v[48:51] blgp:1\n\t" \
  "ds_read_b128 v[152:155], v250 offset:17152\n\t" \
  "ds_read_b128 v[156:159], v250 offset:25344\n\t" \
  "s_waitcnt lgkmcnt(2)\n\t" \
  "v_mfma_f32_16x16x128_f8f6f4 a[64:67], v[128:135], v[192:199], a[64:67] blgp:1\n\t" \
  "s_add_i32 s52, s49, 0\n\t" \
  "s_add_u32 m0, s53, 98304\n\t" \
  "v_exp_f32 v247, v247\n\t" \
  "v_mfma_f32_16x16x128_f8f6f4 a[68:71], v[128:135], v[200:207], a[68:71] blgp:1\n\t" \
  "buffer_load_dwordx4 v252, s[40:43], s52 offen offset:0 sc0 sc1 lds\n\t" \
  "v_mfma_f32_16x16x128_f8f6f4 a[72:75], v[128:135], v[208:215], a[72:75] blgp:1\n\t" \
  "buffer_load_dwordx4 v252, s[40:43], s52 offen offset:1024 sc0 sc1 lds\n\t" \
  "v_fma_f32 v244, v244, s60, s60\n\t" \
  "v_mfma_f32_16x16x128_f8f6f4 a[76:79], v[128:135], v[216:223], a[76:79] blgp:1\n\t" \
  "v_fma_f32 v245, v245, s60, s60\n\t" \
  "v_mfma_f32_16x16x128_f8f6f4 a[92:95], v[136:143], v[216:223], a[92:95] blgp:1\n\t" \
  "buffer_load_dwordx4 v252, s[40:43], s52 offen offset:2048 sc0 sc1 lds\n\t" \
  "v_fma_f32 v246, v246, s60, s60\n\t" \
  "v_mfma_f32_16x16x128_f8f6f4 a[88:91], v[136:143], v[208:215], a[88:91] blgp:1\n\t" \
  "buffer_load_dwordx4 v252, s[40:43], s52 offen offset:3072 sc0 sc1 lds\n\t" \
  "v_fma_f32 v247, v247, s60, s60\n\t" \
  "v_mfma_f32_16x16x128_f8f6f4 a[84:87], v[136:143], v[200:207], a[84:87] blgp:1\n\t" \
  "s_add_i32 s52, s49, 16384\n\t" \
  "s_add_u32 m0, s53, 114688\n\t" \
  "v_rcp_f32 v244, v244\n\t" \
  "v_mfma_f32_16x16x128_f8f6f4 a[80:83], v[136:143], v[192:199], a[80:83] blgp:1\n\t" \
  "buffer_load_dwordx4 v252, s[40:43], s52 offen offset:0 sc0 sc1 lds\n\t" \
  "v_rcp_f32 v245, v245\n\t" \
  "v_mfma_f32_16x16x128_f8f6f4 a[96:99], v[144:151], v[192:199], a[96:99] blgp:1\n\t" \
  "buffer_load_dwordx4 v252, s[40:43], s52 offen offset:1024 sc0 sc1 lds\n\t" \
  "v_rcp_f32 v246, v246\n\t" \
  "v_mfma_f32_16x16x128_f8f6f4 a[100:103], v[144:151], v[200:207], a[100:103] blgp:1\n\t" \
  "v_rcp_f32 v247, v247\n\t" \
  "v_mfma_f32_16x16x128_f8f6f4 a[104:107], v[144:151], v[208:215], a[104:107] blgp:1\n\t" \
  "buffer_load_dwordx4 v252, s[40:43], s52 offen offset:2048 sc0 sc1 lds\n\t" \
  "v_mul_f32 v244, v227, v244\n\t" \
  "v_mfma_f32_16x16x128_f8f6f4 a[108:111], v[144:151], v[216:223], a[108:111] blgp:1\n\t" \
  "buffer_load_dwordx4 v252, s[40:43], s52 offen offset:3072 sc0 sc1 lds\n\t" \
  "v_mul_f32 v245, v231, v245\n\t" \
  "s_waitcnt lgkmcnt(0)\n\t" \
  "v_mfma_f32_16x16x128_f8f6f4 a[124:127], v[152:159], v[216:223], a[124:127] blgp:1\n\t" \
  "v_mul_f32 v246, v235, v246\n\t" \
  "v_mfma_f32_16x16x128_f8f6f4 a[120:123], v[152:159], v[208:215], a[120:123] blgp:1\n\t" \
  "v_mul_f32 v247, v239, v247\n\t" \
  "v_mfma_f32_16x16x128_f8f6f4 a[116:119], v[152:159], v[200:207], a[116:119] blgp:1\n\t" \
  "v_mfma_f32_16x16x128_f8f6f4 a[112:115], v[152:159], v[192:199], a[112:115] blgp:1\n\t" \
  "s_waitcnt vmcnt(19) lgkmcnt(0)\n\t" \
  "s_barrier\n\t" \
  "v_mfma_f32_16x16x128_f8f6f4 v[64:67], v[128:135], v[160:167], v[64:67] blgp:1\n\t" \
  "ds_read_b128 v[192:195], v251 offset:49152\n\t" \
  "v_accvgpr_read_b32 v240, a160\n\t" \
  "v_mfma_f32_16x16x128_f8f6f4 v[68:71], v[128:135], v[168:175], v[68:71] blgp:1\n\t" \
  "ds_read_b128 v[196:199], v251 offset:57344\n\t" \
  "s_add_u32 s57, s80, 3342336\n\t" \
  "v_accvgpr_read_b32 v241, a164\n\t" \
  "v_mfma_f32_16x16x128_f8f6f4 v[72:75], v[128:135], v[176:183], v[72:75] blgp:1\n\t" \
  "buffer_store_dwordx4 v[244:247], v253, s[44:47], s57 offen offset:256 nt\n\t" \
  "ds_read_b128 v[200:203], v251 offset:49408\n\t" \
  "v_accvgpr_read_b32 v242, a168\n\t" \
  "v_mfma_f32_16x16x128_f8f6f4 v[76:79], v[128:135], v[184:191], v[76:79] blgp:1\n\t" \
  "ds_read_b128 v[128:131], v250 offset:32768\n\t" \
  "ds_read_b128 v[132:135], v250 offset:40960\n\t" \
  "v_accvgpr_read_b32 v243, a172\n\t" \
  "v_mfma_f32_16x16x128_f8f6f4 v[92:95], v[136:143], v[184:191], v[92:95] blgp:1\n\t" \
  "ds_read_b128 v[204:207], v251 offset:57600\n\t" \
  "v_mul_f32 v244, s58, v240\n\t" \
  "v_mfma_f32_16x16x128_f8f6f4 v[88:91], v[136:143], v[176:183], v[88:91] blgp:1\n\t" \
  "ds_read_b128 v[208:211], v251 offset:49664\n\t" \
  "v_mul_f32 v245, s58, v241\n\t" \
  "v_mfma_f32_16x16x128_f8f6f4 v[84:87], v[136:143], v[168:175], v[84:87] blgp:1\n\t" \
  "ds_read_b128 v[212:215], v251 offset:57856\n\t" \
  "v_mul_f32 v246, s58, v242\n\t" \
  "v_mfma_f32_16x16x128_f8f6f4 v[80:83], v[136:143], v[160:167], v[80:83] blgp:1\n\t" \
  "ds_read_b128 v[136:139], v250 offset:33024\n\t" \
  "ds_read_b128 v[140:143], v250 offset:41216\n\t" \
  "v_mul_f32 v247, s58, v243\n\t" \
  "v_mfma_f32_16x16x128_f8f6f4 v[96:99], v[144:151], v[160:167], v[96:99] blgp:1\n\t" \
  "ds_read_b128 v[216:219], v251 offset:49920\n\t" \
  "v_med3_f32 v244, v244, -s62, s62\n\t" \
  "v_mfma_f32_16x16x128_f8f6f4 v[100:103], v[144:151], v[168:175], v[100:103] blgp:1\n\t" \
  "ds_read_b128 v[220:223], v251 offset:58112\n\t" \
  "v_med3_f32 v245, v245, -s62, s62\n\t" \
  "v_mfma_f32_16x16x128_f8f6f4 v[104:107], v[144:151], v[176:183], v[104:107] blgp:1\n\t" \
  "v_med3_f32 v246, v246, -s62, s62\n\t" \
  "v_mfma_f32_16x16x128_f8f6f4 v[108:111], v[144:151], v[184:191], v[108:111] blgp:1\n\t" \
  "ds_read_b128 v[144:147], v250 offset:33280\n\t" \
  "v_med3_f32 v247, v247, -s62, s62\n\t" \
  "v_mfma_f32_16x16x128_f8f6f4 v[124:127], v[152:159], v[184:191], v[124:127] blgp:1\n\t" \
  "ds_read_b128 v[148:151], v250 offset:41472\n\t" \
  "v_exp_f32 v244, v244\n\t" \
  "v_mfma_f32_16x16x128_f8f6f4 v[120:123], v[152:159], v[176:183], v[120:123] blgp:1\n\t" \
  "v_exp_f32 v245, v245\n\t" \
  "v_mfma_f32_16x16x128_f8f6f4 v[116:119], v[152:159], v[168:175], v[116:119] blgp:1\n\t" \
  "v_exp_f32 v246, v246\n\t" \
  "v_mfma_f32_16x16x128_f8f6f4 v[112:115], v[152:159], v[160:167], v[112:115] blgp:1\n\t" \
  "ds_read_b128 v[152:155], v250 offset:33536\n\t" \
  "ds_read_b128 v[156:159], v250 offset:41728\n\t" \
  "s_waitcnt lgkmcnt(2)\n\t" \
  "v_mfma_f32_16x16x128_f8f6f4 v[0:3], v[128:135], v[192:199], v[0:3] blgp:1\n\t" \
  "ds_read_b128 v[160:163], v251 offset:32768\n\t" \
  "v_exp_f32 v247, v247\n\t" \
  "v_mfma_f32_16x16x128_f8f6f4 v[4:7], v[128:135], v[200:207], v[4:7] blgp:1\n\t" \
  "ds_read_b128 v[164:167], v251 offset:40960\n\t" \
  "v_mfma_f32_16x16x128_f8f6f4 v[8:11], v[128:135], v[208:215], v[8:11] blgp:1\n\t" \
  "ds_read_b128 v[168:171], v251 offset:33024\n\t" \
  "v_fma_f32 v244, v244, s60, s60\n\t" \
  "v_mfma_f32_16x16x128_f8f6f4 v[12:15], v[128:135], v[216:223], v[12:15] blgp:1\n\t" \
  "ds_read_b128 v[172:175], v251 offset:41216\n\t" \
  "v_fma_f32 v245, v245, s60, s60\n\t" \
  "v_mfma_f32_16x16x128_f8f6f4 v[28:31], v[136:143], v[216:223], v[28:31] blgp:1\n\t" \
  "ds_read_b128 v[176:179], v251 offset:33280\n\t" \
  "v_fma_f32 v246, v246, s60, s60\n\t" \
  "v_mfma_f32_16x16x128_f8f6f4 v[24:27], v[136:143], v[208:215], v[24:27] blgp:1\n\t" \
  "ds_read_b128 v[180:183], v251 offset:41472\n\t" \
  "v_fma_f32 v247, v247, s60, s60\n\t" \
  "v_mfma_f32_16x16x128_f8f6f4 v[20:23], v[136:143], v[200:207], v[20:23] blgp:1\n\t" \
  "ds_read_b128 v[184:187], v251 offset:33536\n\t" \
  "v_rcp_f32 v244, v244\n\t" \
  "v_mfma_f32_16x16x128_f8f6f4 v[16:19], v[136:143], v[192:199], v[16:19] blgp:1\n\t" \
  "ds_read_b128 v[188:191], v251 offset:41728\n\t" \
  "s_add_i32 s52, s48, 16384\n\t" \
  "s_add_u32 m0, s53, 49152\n\t" \
  "v_rcp_f32 v245, v245\n\t" \
  "v_mfma_f32_16x16x128_f8f6f4 v[32:35], v[144:151], v[192:199], v[32:35] blgp:1\n\t" \
  "buffer_load_dwordx4 v252, s[40:43], s52 offen offset:0 sc0 sc1 lds\n\t" \
  "v_rcp_f32 v246, v246\n\t" \
  "v_mfma_f32_16x16x128_f8f6f4 v[36:39], v[144:151], v[200:207], v[36:39] blgp:1\n\t" \
  "v_rcp_f32 v247, v247\n\t" \
  "v_mfma_f32_16x16x128_f8f6f4 v[40:43], v[144:151], v[208:215], v[40:43] blgp:1\n\t" \
  "buffer_load_dwordx4 v252, s[40:43], s52 offen offset:1024 sc0 sc1 lds\n\t" \
  "v_mul_f32 v244, v240, v244\n\t" \
  "v_mfma_f32_16x16x128_f8f6f4 v[44:47], v[144:151], v[216:223], v[44:47] blgp:1\n\t" \
  "v_mul_f32 v245, v241, v245\n\t" \
  "s_waitcnt lgkmcnt(0)\n\t" \
  "v_mfma_f32_16x16x128_f8f6f4 v[60:63], v[152:159], v[216:223], v[60:63] blgp:1\n\t" \
  "buffer_load_dwordx4 v252, s[40:43], s52 offen offset:2048 sc0 sc1 lds\n\t" \
  "v_mul_f32 v246, v242, v246\n\t" \
  "v_mfma_f32_16x16x128_f8f6f4 v[56:59], v[152:159], v[208:215], v[56:59] blgp:1\n\t" \
  "v_mul_f32 v247, v243, v247\n\t" \
  "v_mfma_f32_16x16x128_f8f6f4 v[52:55], v[152:159], v[200:207], v[52:55] blgp:1\n\t" \
  "buffer_load_dwordx4 v252, s[40:43], s52 offen offset:3072 sc0 sc1 lds\n\t" \
  "v_mfma_f32_16x16x128_f8f6f4 v[48:51], v[152:159], v[192:199], v[48:51] blgp:1\n\t" \
  "s_waitcnt vmcnt(18) lgkmcnt(0)\n\t" \
  "s_barrier\n\t" \
  "v_mfma_f32_16x16x128_f8f6f4 a[0:3], v[128:135], v[160:167], a[0:3] blgp:1\n\t" \
  "v_readfirstlane_b32 s87, v254\n\t" \
  "v_readlane_b32 s65, v255, s67\n\t" \
  "v_accvgpr_read_b32 v240, a161\n\t" \
  "v_mfma_f32_16x16x128_f8f6f4 a[4:7], v[128:135], v[168:175], a[4:7] blgp:1\n\t" \
  "s_add_u32 s57, s80, 2097152\n\t" \
  "v_accvgpr_read_b32 v241, a165\n\t" \
  "v_mfma_f32_16x16x128_f8f6f4 a[8:11], v[128:135], v[176:183], a[8:11] blgp:1\n\t" \
  "buffer_store_dwordx4 v[244:247], v253, s[44:47], s57 offen offset:0 nt\n\t" \
  "s_max_u32 s69, s69, s87\n\t" \
  "s_add_u32 s67, s67, 1\n\t" \
  "s_cmp_ge_u32 s69, s65\n\t" \
  "s_cbranch_scc0 L_w10_%=\n\t" \
  "L_ok10_%=:\n\t" \
  "v_accvgpr_read_b32 v242, a169\n\t" \
  "v_mfma_f32_16x16x128_f8f6f4 a[12:15], v[128:135], v[184:191], a[12:15] blgp:1\n\t" \
  "ds_read_b128 v[128:131], v250 offset:49152\n\t" \
  "v_accvgpr_read_b32 v243, a173\n\t" \
  "v_mfma_f32_16x16x128_f8f6f4 a[28:31], v[136:143], v[184:191], a[28:31] blgp:1\n\t" \
  "s_add_i32 s52, s48, 32768\n\t" \
  "s_add_u32 m0, s53, 65536\n\t" \
  "ds_read_b128 v[132:135], v250 offset:57344\n\t" \
  "v_mul_f32 v244, s58, v240\n\t" \
  "v_mfma_f32_16x16x128_f8f6f4 a[24:27], v[136:143], v[176:183], a[24:27] blgp:1\n\t" \
  "buffer_load_dwordx4 v252, s[40:43], s52 offen offset:0 sc0 sc1 lds\n\t" \
  "v_mul_f32 v245, s58, v241\n\t" \
  "v_mfma_f32_16x16x128_f8f6f4 a[20:23], v[136:143], v[168:175], a[20:23] blgp:1\n\t" \
  "buffer_load_dwordx4 v252, s[40:43], s52 offen offset:1024 sc0 sc1 lds\n\t" \
  "v_mul_f32 v246, s58, v242\n\t" \
  "v_mfma_f32_16x16x128_f8f6f4 a[16:19], v[136:143], v[160:167], a[16:19] blgp:1\n\t" \
  "ds_read_b128 v[136:139], v250 offset:49408\n\t" \
  "v_mul_f32 v247, s58, v243\n\t" \
  "v_mfma_f32_16x16x128_f8f6f4 a[32:35], v[144:151], v[160:167], a[32:35] blgp:1\n\t" \
  "ds_read_b128 v[140:143], v250 offset:57600\n\t" \
  "v_med3_f32 v244, v244, -s62, s62\n\t" \
  "v_mfma_f32_16x16x128_f8f6f4 a[36:39], v[144:151], v[168:175], a[36:39] blgp:1\n\t" \
  "buffer_load_dwordx4 v252, s[40:43], s52 offen offset:2048 sc0 sc1 lds\n\t" \
  "v_med3_f32 v245, v245, -s62, s62\n\t" \
  "v_mfma_f32_16x16x128_f8f6f4 a[40:43], v[144:151], v[176:183], a[40:43] blgp:1\n\t" \
  "buffer_load_dwordx4 v252, s[40:43], s52 offen offset:3072 sc0 sc1 lds\n\t" \
  "v_med3_f32 v246, v246, -s62, s62\n\t" \
  "v_mfma_f32_16x16x128_f8f6f4 a[44:47], v[144:151], v[184:191], a[44:47] blgp:1\n\t" \
  "ds_read_b128 v[144:147], v250 offset:49664\n\t" \
  "v_med3_f32 v247, v247, -s62, s62\n\t" \
  "v_mfma_f32_16x16x128_f8f6f4 a[60:63], v[152:159], v[184:191], a[60:63] blgp:1\n\t" \
  "ds_read_b128 v[148:151], v250 offset:57856\n\t" \
  "v_exp_f32 v244, v244\n\t" \
  "v_mfma_f32_16x16x128_f8f6f4 a[56:59], v[152:159], v[176:183], a[56:59] blgp:1\n\t" \
  "v_exp_f32 v245, v245\n\t" \
  "v_mfma_f32_16x16x128_f8f6f4 a[52:55], v[152:159], v[168:175], a[52:55] blgp:1\n\t" \
  "v_exp_f32 v246, v246\n\t" \
  "v_mfma_f32_16x16x128_f8f6f4 a[48:51], v[152:159], v[160:167], a[48:51] blgp:1\n\t" \
  "ds_read_b128 v[152:155], v250 offset:49920\n\t" \
  "ds_read_b128 v[156:159], v250 offset:58112\n\t" \
  "s_waitcnt lgkmcnt(2)\n\t" \
  "v_mfma_f32_16x16x128_f8f6f4 v[64:67], v[128:135], v[160:167], v[64:67] blgp:1\n\t" \
  "s_add_i32 s52, s49, 49152\n\t" \
  "s_add_u32 m0, s53, 147456\n\t" \
  "v_exp_f32 v247, v247\n\t" \
  "v_mfma_f32_16x16x128_f8f6f4 v[68:71], v[128:135], v[168:175], v[68:71] blgp:1\n\t" \
  "buffer_load_dwordx4 v252, s[40:43], s52 offen offset:0 sc0 sc1 lds\n\t" \
  "v_mfma_f32_16x16x128_f8f6f4 v[72:75], v[128:135], v[176:183], v[72:75] blgp:1\n\t" \
  "buffer_load_dwordx4 v252, s[40:43], s52 offen offset:1024 sc0 sc1 lds\n\t" \
  "v_fma_f32 v244, v244, s60, s60\n\t" \
  "v_mfma_f32_16x16x128_f8f6f4 v[76:79], v[128:135], v[184:191], v[76:79] blgp:1\n\t" \
  "v_fma_f32 v245, v245, s60, s60\n\t" \
  "v_mfma_f32_16x16x128_f8f6f4 v[92:95], v[136:143], v[184:191], v[92:95] blgp:1\n\t" \
  "buffer_load_dwordx4 v252, s[40:43], s52 offen offset:2048 sc0 sc1 lds\n\t" \
  "v_fma_f32 v246, v246, s60, s60\n\t" \
  "v_mfma_f32_16x16x128_f8f6f4 v[88:91], v[136:143], v[176:183], v[88:91] blgp:1\n\t" \
  "buffer_load_dwordx4 v252, s[40:43], s52 offen offset:3072 sc0 sc1 lds\n\t" \
  "v_fma_f32 v247, v247, s60, s60\n\t" \
  "v_mfma_f32_16x16x128_f8f6f4 v[84:87], v[136:143], v[168:175], v[84:87] blgp:1\n\t" \
  "s_add_i32 s52, s49, 32768\n\t" \
  "s_add_u32 m0, s53, 131072\n\t" \
  "v_rcp_f32 v244, v244\n\t" \
  "v_mfma_f32_16x16x128_f8f6f4 v[80:83], v[136:143], v[160:167], v[80:83] blgp:1\n\t" \
  "buffer_load_dwordx4 v252, s[40:43], s52 offen offset:0 sc0 sc1 lds\n\t" \
  "v_rcp_f32 v245, v245\n\t" \
  "v_mfma_f32_16x16x128_f8f6f4 v[96:99], v[144:151], v[160:167], v[96:99] blgp:1\n\t" \
  "buffer_load_dwordx4 v252, s[40:43], s52 offen offset:1024 sc0 sc1 lds\n\t" \
  "v_rcp_f32 v246, v246\n\t" \
  "v_mfma_f32_16x16x128_f8f6f4 v[100:103], v[144:151], v[168:175], v[100:103] blgp:1\n\t" \
  "v_rcp_f32 v247, v247\n\t" \
  "v_mfma_f32_16x16x128_f8f6f4 v[104:107], v[144:151], v[176:183], v[104:107] blgp:1\n\t" \
  "buffer_load_dwordx4 v252, s[40:43], s52 offen offset:2048 sc0 sc1 lds\n\t" \
  "v_mul_f32 v244, v240, v244\n\t" \
  "v_mfma_f32_16x16x128_f8f6f4 v[108:111], v[144:151], v[184:191], v[108:111] blgp:1\n\t" \
  "buffer_load_dwordx4 v252, s[40:43], s52 offen offset:3072 sc0 sc1 lds\n\t" \
  "v_mul_f32 v245, v241, v245\n\t" \
  "s_waitcnt lgkmcnt(0)\n\t" \
  "v_mfma_f32_16x16x128_f8f6f4 v[124:127], v[152:159], v[184:191], v[124:127] blgp:1\n\t" \
  "v_mul_f32 v246, v242, v246\n\t" \
  "v_mfma_f32_16x16x128_f8f6f4 v[120:123], v[152:159], v[176:183], v[120:123] blgp:1\n\t" \
  "v_mul_f32 v247, v243, v247\n\t" \
  "v_mfma_f32_16x16x128_f8f6f4 v[116:119], v[152:159], v[168:175], v[116:119] blgp:1\n\t" \
  "v_mfma_f32_16x16x128_f8f6f4 v[112:115], v[152:159], v[160:167], v[112:115] blgp:1\n\t" \
  "s_waitcnt vmcnt(18) lgkmcnt(0)\n\t" \
  "s_barrier\n\t" \
  "v_mfma_f32_16x16x128_f8f6f4 a[64:67], v[128:135], v[192:199], a[64:67] blgp:1\n\t" \
  "ds_read_b128 v[160:163], v251 offset:0\n\t" \
  "v_accvgpr_read_b32 v240, a162\n\t" \
  "v_mfma_f32_16x16x128_f8f6f4 a[68:71], v[128:135], v[200:207], a[68:71] blgp:1\n\t" \
  "ds_read_b128 v[164:167], v251 offset:8192\n\t" \
  "s_add_u32 s57, s80, 2162688\n\t" \
  "v_accvgpr_read_b32 v241, a166\n\t" \
  "v_mfma_f32_16x16x128_f8f6f4 a[72:75], v[128:135], v[208:215], a[72:75] blgp:1\n\t" \
  "buffer_store_dwordx4 v[244:247], v253, s[44:47], s57 offen offset:0 nt\n\t" \
  "ds_read_b128 v[168:171], v251 offset:256\n\t" \
  "v_accvgpr_read_b32 v242, a170\n\t" \
  "v_mfma_f32_16x16x128_f8f6f4 a[76:79], v[128:135], v[216:223], a[76:79] blgp:1\n\t" \
  "ds_read_b128 v[128:131], v250 offset:0\n\t" \
  "ds_read_b128 v[132:135], v250 offset:8192\n\t" \
  "v_accvgpr_read_b32 v243, a174\n\t" \
  "v_mfma_f32_16x16x128_f8f6f4 a[92:95], v[136:143], v[216:223], a[92:95] blgp:1\n\t" \
  "ds_read_b128 v[172:175], v251 offset:8448\n\t" \
  "v_mul_f32 v244, s58, v240\n\t" \
  "v_mfma_f32_16x16x128_f8f6f4 a[88:91], v[136:143], v[208:215], a[88:91] blgp:1\n\t" \
  "ds_read_b128 v[176:179], v251 offset:512\n\t" \
  "v_mul_f32 v245, s58, v241\n\t" \
  "v_mfma_f32_16x16x128_f8f6f4 a[84:87], v[136:143], v[200:207], a[84:87] blgp:1\n\t" \
  "ds_read_b128 v[180:183], v251 offset:8704\n\t" \
  "v_mul_f32 v246, s58, v242\n\t" \
  "v_mfma_f32_16x16x128_f8f6f4 a[80:83], v[136:143], v[192:199], a[80:83] blgp:1\n\t" \
  "ds_read_b128 v[136:139], v250 offset:256\n\t" \
  "ds_read_b128 v[140:143], v250 offset:8448\n\t" \
  "v_mul_f32 v247, s58, v243\n\t" \
  "v_mfma_f32_16x16x128_f8f6f4 a[96:99], v[144:151], v[192:199], a[96:99] blgp:1\n\t" \
  "ds_read_b128 v[184:187], v251 offset:768\n\t" \
  "v_med3_f32 v244, v244, -s62, s62\n\t" \
  "v_mfma_f32_16x16x128_f8f6f4 a[100:103], v[144:151], v[200:207], a[100:103] blgp:1\n\t" \
  "ds_read_b128 v[188:191], v251 offset:8960\n\t" \
  "v_med3_f32 v245, v245, -s62, s62\n\t" \
  "v_mfma_f32_16x16x128_f8f6f4 a[104:107], v[144:151], v[208:215], a[104:107] blgp:1\n\t" \
  "s_add_u32 s48, s48, 0x10000\n\t" \
  "s_add_u32 s49, s49, 0x10000\n\t" \
  "v_med3_f32 v246, v246, -s62, s62\n\t" \
  "v_mfma_f32_16x16x128_f8f6f4 a[108:111], v[144:151], v[216:223], a[108:111] blgp:1\n\t" \
  "ds_read_b128 v[144:147], v250 offset:512\n\t" \
  "v_med3_f32 v247, v247, -s62, s62\n\t" \
  "v_mfma_f32_16x16x128_f8f6f4 a[124:127], v[152:159], v[216:223], a[124:127] blgp:1\n\t" \
  "ds_read_b128 v[148:151], v250 offset:8704\n\t" \
  "v_exp_f32 v244, v244\n\t" \
  "v_mfma_f32_16x16x128_f8f6f4 a[120:123], v[152:159], v[208:215], a[120:123] blgp:1\n\t" \
  "v_exp_f32 v245, v245\n\t" \
  "v_mfma_f32_16x16x128_f8f6f4 a[116:119], v[152:159], v[200:207], a[116:119] blgp:1\n\t" \
  "v_exp_f32 v246, v246\n\t" \
  "v_mfma_f32_16x16x128_f8f6f4 a[112:115], v[152:159], v[192:199], a[112:115] blgp:1\n\t" \
  "ds_read_b128 v[152:155], v250 offset:768\n\t" \
  "ds_read_b128 v[156:159], v250 offset:8960\n\t" \
  "s_waitcnt lgkmcnt(2)\n\t" \
  "v_mfma_f32_16x16x128_f8f6f4 a[0:3], v[128:135], v[160:167], a[0:3] blgp:1\n\t" \
  "ds_read_b128 v[192:195], v251 offset:16384\n\t" \
  "v_exp_f32 v247, v247\n\t" \
  "v_mfma_f32_16x16x128_f8f6f4 a[4:7], v[128:135], v[168:175], a[4:7] blgp:1\n\t" \
  "ds_read_b128 v[196:199], v251 offset:24576\n\t" \
  "v_mfma_f32_16x16x128_f8f6f4 a[8:11], v[128:135], v[176:183], a[8:11] blgp:1\n\t" \
  "ds_read_b128 v[200:203], v251 offset:16640\n\t" \
  "v_fma_f32 v244, v244, s60, s60\n\t" \
  "v_mfma_f32_16x16x128_f8f6f4 a[12:15], v[128:135], v[184:191], a[12:15] blgp:1\n\t" \
  "ds_read_b128 v[204:207], v251 offset:24832\n\t" \
  "v_fma_f32 v245, v245, s60, s60\n\t" \
  "v_mfma_f32_16x16x128_f8f6f4 a[28:31], v[136:143], v[184:191], a[28:31] blgp:1\n\t" \
  "ds_read_b128 v[208:211], v251 offset:16896\n\t" \
  "v_fma_f32 v246, v246, s60, s60\n\t" \
  "v_mfma_f32_16x16x128_f8f6f4 a[24:27], v[136:143], v[176:183], a[24:27] blgp:1\n\t" \
  "ds_read_b128 v[212:215], v251 offset:25088\n\t" \
  "v_fma_f32 v247, v247, s60, s60\n\t" \
  "v_mfma_f32_16x16x128_f8f6f4 a[20:23], v[136:143], v[168:175], a[20:23] blgp:1\n\t" \
  "ds_read_b128 v[216:219], v251 offset:17152\n\t" \
  "v_rcp_f32 v244, v244\n\t" \
  "v_mfma_f32_16x16x128_f8f6f4 a[16:19], v[136:143], v[160:167], a[16:19] blgp:1\n\t" \
  "ds_read_b128 v[220:223], v251 offset:25344\n\t" \
  "s_add_i32 s52, s48, -16384\n\t" \
  "s_add_u32 m0, s53, 81920\n\t" \
  "v_rcp_f32 v245, v245\n\t" \
  "v_mfma_f32_16x16x128_f8f6f4 a[32:35], v[144:151], v[160:167], a[32:35] blgp:1\n\t" \
  "buffer_load_dwordx4 v252, s[40:43], s52 offen offset:0 sc0 sc1 lds\n\t" \
  "v_rcp_f32 v246, v246\n\t" \
  "v_mfma_f32_16x16x128_f8f6f4 a[36:39], v[144:151], v[168:175], a[36:39] blgp:1\n\t" \
  "v_rcp_f32 v247, v247\n\t" \
  "v_mfma_f32_16x16x128_f8f6f4 a[40:43], v[144:151], v[176:183], a[40:43] blgp:1\n\t" \
  "buffer_load_dwordx4 v252, s[40:43], s52 offen offset:1024 sc0 sc1 lds\n\t" \
  "v_mul_f32 v244, v240, v244\n\t" \
  "v_mfma_f32_16x16x128_f8f6f4 a[44:47], v[144:151], v[184:191], a[44:47] blgp:1\n\t" \
  "v_mul_f32 v245, v241, v245\n\t" \
  "s_waitcnt lgkmcnt(0)\n\t" \
  "v_mfma_f32_16x16x128_f8f6f4 a[60:63], v[152:159], v[184:191], a[60:63] blgp:1\n\t" \
  "buffer_load_dwordx4 v252, s[40:43], s52 offen offset:2048 sc0 sc1 lds\n\t" \
  "v_mul_f32 v246, v242, v246\n\t" \
  "v_mfma_f32_16x16x128_f8f6f4 a[56:59], v[152:159], v[176:183], a[56:59] blgp:1\n\t" \
  "v_mul_f32 v247, v243, v247\n\t" \
  "v_mfma_f32_16x16x128_f8f6f4 a[52:55], v[152:159], v[168:175], a[52:55] blgp:1\n\t" \
  "buffer_load_dwordx4 v252, s[40:43], s52 offen offset:3072 sc0 sc1 lds\n\t" \
  "buffer_load_dword v254, off, s[88:91], s81 sc1\n\t" \
  "v_mfma_f32_16x16x128_f8f6f4 a[48:51], v[152:159], v[160:167], a[48:51] blgp:1\n\t" \
  "s_waitcnt vmcnt(19) lgkmcnt(0)\n\t" \
  "s_barrier\n\t" \
  "v_mfma_f32_16x16x128_f8f6f4 v[0:3], v[128:135], v[192:199], v[0:3] blgp:1\n\t" \
  "v_readlane_b32 s65, v255, s67\n\t" \
  "v_accvgpr_read_b32 v240, a163\n\t" \
  "v_mfma_f32_16x16x128_f8f6f4 v[4:7], v[128:135], v[200:207], v[4:7] blgp:1\n\t" \
  "s_add_u32 s57, s80, 2228224\n\t" \
  "v_accvgpr_read_b32 v241, a167\n\t" \
  "v_mfma_f32_16x16x128_f8f6f4 v[8:11], v[128:135], v[208:215], v[8:11] blgp:1\n\t" \
  "buffer_store_dwordx4 v[244:247], v253, s[44:47], s57 offen offset:0 nt\n\t" \
  "s_add_u32 s67, s67, 1\n\t" \
  "s_cmp_ge_u32 s69, s65\n\t" \
  "s_cbranch_scc0 L_w11_%=\n\t" \
  "L_ok11_%=:\n\t" \
  "v_accvgpr_read_b32 v242, a171\n\t" \
  "v_mfma_f32_16x16x128_f8f6f4 v[12:15], v[128:135], v[216:223], v[12:15] blgp:1\n\t" \
  "ds_read_b128 v[128:131], v250 offset:16384\n\t" \
  "v_accvgpr_read_b32 v243, a175\n\t" \
  "v_mfma_f32_16x16x128_f8f6f4 v[28:31], v[136:143], v[216:223], v[28:31] blgp:1\n\t" \
  "s_add_i32 s52, s48, 0\n\t" \
  "s_add_u32 m0, s53, 32768\n\t" \
  "ds_read_b128 v[132:135], v250 offset:24576\n\t" \
  "v_mul_f32 v244, s58, v240\n\t" \
  "v_mfma_f32_16x16x128_f8f6f4 v[24:27], v[136:143], v[208:215], v[24:27] blgp:1\n\t" \
  "buffer_load_dwordx4 v252, s[40:43], s52 offen offset:0 sc0 sc1 lds\n\t" \
  "v_mul_f32 v245, s58, v241\n\t" \
  "v_mfma_f32_16x16x128_f8f6f4 v[20:23], v[136:143], v[200:207], v[20:23] blgp:1\n\t" \
  "buffer_load_dwordx4 v252, s[40:43], s52 offen offset:1024 sc0 sc1 lds\n\t" \
  "v_mul_f32 v246, s58, v242\n\t" \
  "v_mfma_f32_16x16x128_f8f6f4 v[16:19], v[136:143], v[192:199], v[16:19] blgp:1\n\t" \
  "ds_read_b128 v[136:139], v250 offset:16640\n\t" \
  "v_mul_f32 v247, s58, v243\n\t" \
  "v_mfma_f32_16x16x128_f8f6f4 v[32:35], v[144:151], v[192:199], v[32:35] blgp:1\n\t" \
  "ds_read_b128 v[140:143], v250 offset:24832\n\t" \
  "v_med3_f32 v244, v244, -s62, s62\n\t" \
  "v_mfma_f32_16x16x128_f8f6f4 v[36:39], v[144:151], v[200:207], v[36:39] blgp:1\n\t" \
  "buffer_load_dwordx4 v252, s[40:43], s52 offen offset:2048 sc0 sc1 lds\n\t" \
  "v_med3_f32 v245, v245, -s62, s62\n\t" \
  "v_mfma_f32_16x16x128_f8f6f4 v[40:43], v[144:151], v[208:215], v[40:43] blgp:1\n\t" \
  "buffer_load_dwordx4 v252, s[40:43], s52 offen offset:3072 sc0 sc1 lds\n\t" \
  "v_med3_f32 v246, v246, -s62, s62\n\t" \
  "v_mfma_f32_16x16x128_f8f6f4 v[44:47], v[144:151], v[216:223], v[44:47] blgp:1\n\t" \
  "ds_read_b128 v[144:147], v250 offset:16896\n\t" \
  "v_med3_f32 v247, v247, -s62, s62\n\t" \
  "v_mfma_f32_16x16x128_f8f6f4 v[60:63], v[152:159], v[216:223], v[60:63] blgp:1\n\t" \
  "ds_read_b128 v[148:151], v250 offset:25088\n\t" \
  "v_exp_f32 v244, v244\n\t" \
  "v_mfma_f32_16x16x128_f8f6f4 v[56:59], v[152:159], v[208:215], v[56:59] blgp:1\n\t" \
  "v_exp_f32 v245, v245\n\t" \
  "v_mfma_f32_16x16x128_f8f6f4 v[52:55], v[152:159], v[200:207], v[52:55] blgp:1\n\t" \
  "v_exp_f32 v246, v246\n\t" \
  "v_mfma_f32_16x16x128_f8f6f4 v[48:51], v[152:159], v[192:199], v[48:51] blgp:1\n\t" \
  "ds_read_b128 v[152:155], v250 offset:17152\n\t" \
  "ds_read_b128 v[156:159], v250 offset:25344\n\t" \
  "s_waitcnt lgkmcnt(2)\n\t" \
  "v_mfma_f32_16x16x128_f8f6f4 a[64:67], v[128:135], v[192:199], a[64:67] blgp:1\n\t" \
  "s_add_i32 s52, s49, 0\n\t" \
  "s_add_u32 m0, s53, 98304\n\t" \
  "ds_read_b128 v[224:227], v252 offset:16384\n\t" \
  "v_exp_f32 v247, v247\n\t" \
  "v_mfma_f32_16x16x128_f8f6f4 a[68:71], v[128:135], v[200:207], a[68:71] blgp:1\n\t" \
  "buffer_load_dwordx4 v252, s[40:43], s52 offen offset:0 sc0 sc1 lds\n\t" \
  "v_mfma_f32_16x16x128_f8f6f4 a[72:75], v[128:135], v[208:215], a[72:75] blgp:1\n\t" \
  "buffer_load_dwordx4 v252, s[40:43], s52 offen offset:1024 sc0 sc1 lds\n\t" \
  "v_fma_f32 v244, v244, s60, s60\n\t" \
  "v_mfma_f32_16x16x128_f8f6f4 a[76:79], v[128:135], v[216:223], a[76:79] blgp:1\n\t" \
  "ds_read_b128 v[228:231], v252 offset:17408\n\t" \
  "v_fma_f32 v245, v245, s60, s60\n\t" \
  "v_mfma_f32_16x16x128_f8f6f4 a[92:95], v[136:143], v[216:223], a[92:95] blgp:1\n\t" \
  "buffer_load_dwordx4 v252, s[40:43], s52 offen offset:2048 sc0 sc1 lds\n\t" \
  "ds_read_b128 v[232:235], v252 offset:18432\n\t" \
  "v_fma_f32 v246, v246, s60, s60\n\t" \
  "v_mfma_f32_16x16x128_f8f6f4 a[88:91], v[136:143], v[208:215], a[88:91] blgp:1\n\t" \
  "buffer_load_dwordx4 v252, s[40:43], s52 offen offset:3072 sc0 sc1 lds\n\t" \
  "v_fma_f32 v247, v247, s60, s60\n\t" \
  "v_mfma_f32_16x16x128_f8f6f4 a[84:87], v[136:143], v[200:207], a[84:87] blgp:1\n\t" \
  "s_add_i32 s52, s49, 16384\n\t" \
  "s_add_u32 m0, s53, 114688\n\t" \
  "ds_read_b128 v[236:239], v252 offset:19456\n\t" \
  "v_rcp_f32 v244, v244\n\t" \
  "v_mfma_f32_16x16x128_f8f6f4 a[80:83], v[136:143], v[192:199], a[80:83] blgp:1\n\t" \
  "buffer_load_dwordx4 v252, s[40:43], s52 offen offset:0 sc0 sc1 lds\n\t" \
  "v_rcp_f32 v245, v245\n\t" \
  "v_mfma_f32_16x16x128_f8f6f4 a[96:99], v[144:151], v[192:199], a[96:99] blgp:1\n\t" \
  "buffer_load_dwordx4 v252, s[40:43], s52 offen offset:1024 sc0 sc1 lds\n\t" \
  "v_rcp_f32 v246, v246\n\t" \
  "v_mfma_f32_16x16x128_f8f6f4 a[100:103], v[144:151], v[200:207], a[100:103] blgp:1\n\t" \
  "v_rcp_f32 v247, v247\n\t" \
  "v_mfma_f32_16x16x128_f8f6f4 a[104:107], v[144:151], v[208:215], a[104:107] blgp:1\n\t" \
  "buffer_load_dwordx4 v252, s[40:43], s52 offen offset:2048 sc0 sc1 lds\n\t" \
  "v_mul_f32 v244, v240, v244\n\t" \
  "v_mfma_f32_16x16x128_f8f6f4 a[108:111], v[144:151], v[216:223], a[108:111] blgp:1\n\t" \
  "buffer_load_dwordx4 v252, s[40:43], s52 offen offset:3072 sc0 sc1 lds\n\t" \
  "v_mul_f32 v245, v241, v245\n\t" \
  "s_waitcnt lgkmcnt(0)\n\t" \
  "v_mfma_f32_16x16x128_f8f6f4 a[124:127], v[152:159], v[216:223], a[124:127] blgp:1\n\t" \
  "v_mul_f32 v246, v242, v246\n\t" \
  "v_mfma_f32_16x16x128_f8f6f4 a[120:123], v[152:159], v[208:215], a[120:123] blgp:1\n\t" \
  "v_mul_f32 v247, v243, v247\n\t" \
  "v_mfma_f32_16x16x128_f8f6f4 a[116:119], v[152:159], v[200:207], a[116:119] blgp:1\n\t" \
  "v_mfma_f32_16x16x128_f8f6f4 a[112:115], v[152:159], v[192:199], a[112:115] blgp:1\n\t" \
  "s_waitcnt vmcnt(19) lgkmcnt(0)\n\t" \
  "s_barrier\n\t" \
  "v_mfma_f32_16x16x128_f8f6f4 v[64:67], v[128:135], v[160:167], v[64:67] blgp:1\n\t" \
  "ds_read_b128 v[192:195], v251 offset:49152\n\t" \
  "v_accvgpr_read_b32 v240, a176\n\t" \
  "v_mfma_f32_16x16x128_f8f6f4 v[68:71], v[128:135], v[168:175], v[68:71] blgp:1\n\t" \
  "ds_read_b128 v[196:199], v251 offset:57344\n\t" \
  "s_add_u32 s57, s80, 2293760\n\t" \
  "v_accvgpr_read_b32 v241, a180\n\t" \
  "v_mfma_f32_16x16x128_f8f6f4 v[72:75], v[128:135], v[176:183], v[72:75] blgp:1\n\t" \
  "buffer_store_dwordx4 v[244:247], v253, s[44:47], s57 offen offset:0 nt\n\t" \
  "ds_read_b128 v[200:203], v251 offset:49408\n\t" \
  "v_accvgpr_read_b32 v242, a184\n\t" \
  "v_mfma_f32_16x16x128_f8f6f4 v[76:79], v[128:135], v[184:191], v[76:79] blgp:1\n\t" \
  "ds_read_b128 v[128:131], v250 offset:32768\n\t" \
  "ds_read_b128 v[132:135], v250 offset:40960\n\t" \
  "v_accvgpr_read_b32 v243, a188\n\t" \
  "v_mfma_f32_16x16x128_f8f6f4 v[92:95], v[136:143], v[184:191], v[92:95] blgp:1\n\t" \
  "ds_read_b128 v[204:207], v251 offset:57600\n\t" \
  "v_mul_f32 v244, s58, v240\n\t" \
  "v_mfma_f32_16x16x128_f8f6f4 v[88:91], v[136:143], v[176:183], v[88:91] blgp:1\n\t" \
  "ds_read_b128 v[208:211], v251 offset:49664\n\t" \
  "v_mul_f32 v245, s58, v241\n\t" \
  "v_mfma_f32_16x16x128_f8f6f4 v[84:87], v[136:143], v[168:175], v[84:87] blgp:1\n\t" \
  "ds_read_b128 v[212:215], v251 offset:57856\n\t" \
  "v_mul_f32 v246, s58, v242\n\t" \
  "v_mfma_f32_16x16x128_f8f6f4 v[80:83], v[136:143], v[160:167], v[80:83] blgp:1\n\t" \
  "ds_read_b128 v[136:139], v250 offset:33024\n\t" \
  "ds_read_b128 v[140:143], v250 offset:41216\n\t" \
  "v_mul_f32 v247, s58, v243\n\t" \
  "v_mfma_f32_16x16x128_f8f6f4 v[96:99], v[144:151], v[160:167], v[96:99] blgp:1\n\t" \
  "ds_read_b128 v[216:219], v251 offset:49920\n\t" \
  "v_med3_f32 v244, v244, -s62, s62\n\t" \
  "v_mfma_f32_16x16x128_f8f6f4 v[100:103], v[144:151], v[168:175], v[100:103] blgp:1\n\t" \
  "ds_read_b128 v[220:223], v251 offset:58112\n\t" \
  "v_med3_f32 v245, v245, -s62, s62\n\t" \
  "v_mfma_f32_16x16x128_f8f6f4 v[104:107], v[144:151], v[176:183], v[104:107] blgp:1\n\t" \
  "v_med3_f32 v246, v246, -s62, s62\n\t" \
  "v_mfma_f32_16x16x128_f8f6f4 v[108:111], v[144:151], v[184:191], v[108:111] blgp:1\n\t" \
  "ds_read_b128 v[144:147], v250 offset:33280\n\t" \
  "v_med3_f32 v247, v247, -s62, s62\n\t" \
  "v_mfma_f32_16x16x128_f8f6f4 v[124:127], v[152:159], v[184:191], v[124:127] blgp:1\n\t" \
  "ds_read_b128 v[148:151], v250 offset:41472\n\t" \
  "v_exp_f32 v244, v244\n\t" \
  "v_mfma_f32_16x16x128_f8f6f4 v[120:123], v[152:159], v[176:183], v[120:123] blgp:1\n\t" \
  "v_exp_f32 v245, v245\n\t" \
  "v_mfma_f32_16x16x128_f8f6f4 v[116:119], v[152:159], v[168:175], v[116:119] blgp:1\n\t" \
  "v_exp_f32 v246, v246\n\t" \
  "v_mfma_f32_16x16x128_f8f6f4 v[112:115], v[152:159], v[160:167], v[112:115] blgp:1\n\t" \
  "ds_read_b128 v[152:155], v250 offset:33536\n\t" \
  "ds_read_b128 v[156:159], v250 offset:41728\n\t" \
  "s_waitcnt lgkmcnt(2)\n\t" \
  "v_mfma_f32_16x16x128_f8f6f4 v[0:3], v[128:135], v[192:199], v[0:3] blgp:1\n\t" \
  "ds_read_b128 v[160:163], v251 offset:32768\n\t" \
  "v_exp_f32 v247, v247\n\t" \
  "v_mfma_f32_16x16x128_f8f6f4 v[4:7], v[128:135], v[200:207], v[4:7] blgp:1\n\t" \
  "ds_read_b128 v[164:167], v251 offset:40960\n\t" \
  "v_mfma_f32_16x16x128_f8f6f4 v[8:11], v[128:135], v[208:215], v[8:11] blgp:1\n\t" \
  "ds_read_b128 v[168:171], v251 offset:33024\n\t" \
  "v_fma_f32 v244, v244, s60, s60\n\t" \
  "v_mfma_f32_16x16x128_f8f6f4 v[12:15], v[128:135], v[216:223], v[12:15] blgp:1\n\t" \
  "ds_read_b128 v[172:175], v251 offset:41216\n\t" \
  "v_fma_f32 v245, v245, s60, s60\n\t" \
  "v_mfma_f32_16x16x128_f8f6f4 v[28:31], v[136:143], v[216:223], v[28:31] blgp:1\n\t" \
  "ds_read_b128 v[176:179], v251 offset:33280\n\t" \
  "v_fma_f32 v246, v246, s60, s60\n\t" \
  "v_mfma_f32_16x16x128_f8f6f4 v[24:27], v[136:143], v[208:215], v[24:27] blgp:1\n\t" \
  "ds_read_b128 v[180:183], v251 offset:41472\n\t" \
  "v_fma_f32 v247, v247, s60, s60\n\t" \
  "v_mfma_f32_16x16x128_f8f6f4 v[20:23], v[136:143], v[200:207], v[20:23] blgp:1\n\t" \
  "ds_read_b128 v[184:187], v251 offset:33536\n\t" \
  "v_rcp_f32 v244, v244\n\t" \
  "v_mfma_f32_16x16x128_f8f6f4 v[16:19], v[136:143], v[192:199], v[16:19] blgp:1\n\t" \
  "ds_read_b128 v[188:191], v251 offset:41728\n\t" \
  "s_add_i32 s52, s48, 16384\n\t" \
  "s_add_u32 m0, s53, 49152\n\t" \
  "v_rcp_f32 v245, v245\n\t" \
  "v_mfma_f32_16x16x128_f8f6f4 v[32:35], v[144:151], v[192:199], v[32:35] blgp:1\n\t" \
  "buffer_load_dwordx4 v252, s[40:43], s52 offen offset:0 sc0 sc1 lds\n\t" \
  "v_rcp_f32 v246, v246\n\t" \
  "v_mfma_f32_16x16x128_f8f6f4 v[36:39], v[144:151], v[200:207], v[36:39] blgp:1\n\t" \
  "v_rcp_f32 v247, v247\n\t" \
  "v_mfma_f32_16x16x128_f8f6f4 v[40:43], v[144:151], v[208:215], v[40:43] blgp:1\n\t" \
  "buffer_load_dwordx4 v252, s[40:43], s52 offen offset:1024 sc0 sc1 lds\n\t" \
  "v_mul_f32 v244, v240, v244\n\t" \
  "v_mfma_f32_16x16x128_f8f6f4 v[44:47], v[144:151], v[216:223], v[44:47] blgp:1\n\t" \
  "v_mul_f32 v245, v241, v245\n\t" \
  "s_waitcnt lgkmcnt(0)\n\t" \
  "v_mfma_f32_16x16x128_f8f6f4 v[60:63], v[152:159], v[216:223], v[60:63] blgp:1\n\t" \
  "buffer_load_dwordx4 v252, s[40:43], s52 offen offset:2048 sc0 sc1 lds\n\t" \
  "v_mul_f32 v246, v242, v246\n\t" \
  "v_mfma_f32_16x16x128_f8f6f4 v[56:59], v[152:159], v[208:215], v[56:59] blgp:1\n\t" \
  "v_mul_f32 v247, v243, v247\n\t" \
  "v_mfma_f32_16x16x128_f8f6f4 v[52:55], v[152:159], v[200:207], v[52:55] blgp:1\n\t" \
  "buffer_load_dwordx4 v252, s[40:43], s52 offen offset:3072 sc0 sc1 lds\n\t" \
  "v_mfma_f32_16x16x128_f8f6f4 v[48:51], v[152:159], v[192:199], v[48:51] blgp:1\n\t" \
  "s_waitcnt vmcnt(18) lgkmcnt(0)\n\t" \
  "s_barrier\n\t" \
  "v_mfma_f32_16x16x128_f8f6f4 a[0:3], v[128:135], v[160:167], a[0:3] blgp:1\n\t" \
  "v_readfirstlane_b32 s87, v254\n\t" \
  "v_readlane_b32 s65, v255, s67\n\t" \
  "v_accvgpr_read_b32 v240, a177\n\t" \
  "v_mfma_f32_16x16x128_f8f6f4 a[4:7], v[128:135], v[168:175], a[4:7] blgp:1\n\t" \
  "s_add_u32 s57, s80, 3145728\n\t" \
  "v_accvgpr_read_b32 v241, a181\n\t" \
  "v_mfma_f32_16x16x128_f8f6f4 a[8:11], v[128:135], v[176:183], a[8:11] blgp:1\n\t" \
  "buffer_store_dwordx4 v[244:247], v253, s[44:47], s57 offen offset:0 nt\n\t" \
  "s_max_u32 s69, s69, s87\n\t" \
  "s_add_u32 s67, s67, 1\n\t" \
  "s_cmp_ge_u32 s69, s65\n\t" \
  "s_cbranch_scc0 L_w12_%=\n\t" \
  "L_ok12_%=:\n\t" \
  "v_accvgpr_read_b32 v242, a185\n\t" \
  "v_mfma_f32_16x16x128_f8f6f4 a[12:15], v[128:135], v[184:191], a[12:15] blgp:1\n\t" \
  "ds_read_b128 v[128:131], v250 offset:49152\n\t" \
  "v_accvgpr_read_b32 v243, a189\n\t" \
  "v_mfma_f32_16x16x128_f8f6f4 a[28:31], v[136:143], v[184:191], a[28:31] blgp:1\n\t" \
  "s_add_i32 s52, s48, 32768\n\t" \
  "s_add_u32 m0, s53, 65536\n\t" \
  "ds_read_b128 v[132:135], v250 offset:57344\n\t" \
  "v_mul_f32 v244, s58, v240\n\t" \
  "v_mfma_f32_16x16x128_f8f6f4 a[24:27], v[136:143], v[176:183], a[24:27] blgp:1\n\t" \
  "buffer_load_dwordx4 v252, s[40:43], s52 offen offset:0 sc0 sc1 lds\n\t" \
  "v_mul_f32 v245, s58, v241\n\t" \
  "v_mfma_f32_16x16x128_f8f6f4 a[20:23], v[136:143], v[168:175], a[20:23] blgp:1\n\t" \
  "buffer_load_dwordx4 v252, s[40:43], s52 offen offset:1024 sc0 sc1 lds\n\t" \
  "v_mul_f32 v246, s58, v242\n\t" \
  "v_mfma_f32_16x16x128_f8f6f4 a[16:19], v[136:143], v[160:167], a[16:19] blgp:1\n\t" \
  "ds_read_b128 v[136:139], v250 offset:49408\n\t" \
  "v_mul_f32 v247, s58, v243\n\t" \
  "v_mfma_f32_16x16x128_f8f6f4 a[32:35], v[144:151], v[160:167], a[32:35] blgp:1\n\t" \
  "ds_read_b128 v[140:143], v250 offset:57600\n\t" \
  "v_med3_f32 v244, v244, -s62, s62\n\t" \
  "v_mfma_f32_16x16x128_f8f6f4 a[36:39], v[144:151], v[168:175], a[36:39] blgp:1\n\t" \
  "buffer_load_dwordx4 v252, s[40:43], s52 offen offset:2048 sc0 sc1 lds\n\t" \
  "v_med3_f32 v245, v245, -s62, s62\n\t" \
  "v_mfma_f32_16x16x128_f8f6f4 a[40:43], v[144:151], v[176:183], a[40:43] blgp:1\n\t" \
  "buffer_load_dwordx4 v252, s[40:43], s52 offen offset:3072 sc0 sc1 lds\n\t" \
  "v_med3_f32 v246, v246, -s62, s62\n\t" \
  "v_mfma_f32_16x16x128_f8f6f4 a[44:47], v[144:151], v[184:191], a[44:47] blgp:1\n\t" \
  "ds_read_b128 v[144:147], v250 offset:49664\n\t" \
  "v_med3_f32 v247, v247, -s62, s62\n\t" \
  "v_mfma_f32_16x16x128_f8f6f4 a[60:63], v[152:159], v[184:191], a[60:63] blgp:1\n\t" \
  "ds_read_b128 v[148:151], v250 offset:57856\n\t" \
  "v_exp_f32 v244, v244\n\t" \
  "v_mfma_f32_16x16x128_f8f6f4 a[56:59], v[152:159], v[176:183], a[56:59] blgp:1\n\t" \
  "v_exp_f32 v245, v245\n\t" \
  "v_mfma_f32_16x16x128_f8f6f4 a[52:55], v[152:159], v[168:175], a[52:55] blgp:1\n\t" \
  "v_exp_f32 v246, v246\n\t" \
  "v_mfma_f32_16x16x128_f8f6f4 a[48:51], v[152:159], v[160:167], a[48:51] blgp:1\n\t" \
  "ds_read_b128 v[152:155], v250 offset:49920\n\t" \
  "ds_read_b128 v[156:159], v250 offset:58112\n\t" \
  "s_waitcnt lgkmcnt(2)\n\t" \
  "v_mfma_f32_16x16x128_f8f6f4 v[64:67], v[128:135], v[160:167], v[64:67] blgp:1\n\t" \
  "s_add_i32 s52, s49, 49152\n\t" \
  "s_add_u32 m0, s53, 147456\n\t" \
  "v_exp_f32 v247, v247\n\t" \
  "v_mfma_f32_16x16x128_f8f6f4 v[68:71], v[128:135], v[168:175], v[68:71] blgp:1\n\t" \
  "buffer_load_dwordx4 v252, s[40:43], s52 offen offset:0 sc0 sc1 lds\n\t" \
  "v_mfma_f32_16x16x128_f8f6f4 v[72:75], v[128:135], v[176:183], v[72:75] blgp:1\n\t" \
  "buffer_load_dwordx4 v252, s[40:43], s52 offen offset:1024 sc0 sc1 lds\n\t" \
  "v_fma_f32 v244, v244, s60, s60\n\t" \
  "v_mfma_f32_16x16x128_f8f6f4 v[76:79], v[128:135], v[184:191], v[76:79] blgp:1\n\t" \
  "v_fma_f32 v245, v245, s60, s60\n\t" \
  "v_mfma_f32_16x16x128_f8f6f4 v[92:95], v[136:143], v[184:191], v[92:95] blgp:1\n\t" \
  "buffer_load_dwordx4 v252, s[40:43], s52 offen offset:2048 sc0 sc1 lds\n\t" \
  "v_fma_f32 v246, v246, s60, s60\n\t" \
  "v_mfma_f32_16x16x128_f8f6f4 v[88:91], v[136:143], v[176:183], v[88:91] blgp:1\n\t" \
  "buffer_load_dwordx4 v252, s[40:43], s52 offen offset:3072 sc0 sc1 lds\n\t" \
  "v_fma_f32 v247, v247, s60, s60\n\t" \
  "v_mfma_f32_16x16x128_f8f6f4 v[84:87], v[136:143], v[168:175], v[84:87] blgp:1\n\t" \
  "s_add_i32 s52, s49, 32768\n\t" \
  "s_add_u32 m0, s53, 131072\n\t" \
  "v_rcp_f32 v244, v244\n\t" \
  "v_mfma_f32_16x16x128_f8f6f4 v[80:83], v[136:143], v[160:167], v[80:83] blgp:1\n\t" \
  "buffer_load_dwordx4 v252, s[40:43], s52 offen offset:0 sc0 sc1 lds\n\t" \
  "v_rcp_f32 v245, v245\n\t" \
  "v_mfma_f32_16x16x128_f8f6f4 v[96:99], v[144:151], v[160:167], v[96:99] blgp:1\n\t" \
  "buffer_load_dwordx4 v252, s[40:43], s52 offen offset:1024 sc0 sc1 lds\n\t" \
  "v_rcp_f32 v246, v246\n\t" \
  "v_mfma_f32_16x16x128_f8f6f4 v[100:103], v[144:151], v[168:175], v[100:103] blgp:1\n\t" \
  "v_rcp_f32 v247, v247\n\t" \
  "v_mfma_f32_16x16x128_f8f6f4 v[104:107], v[144:151], v[176:183], v[104:107] blgp:1\n\t" \
  "buffer_load_dwordx4 v252, s[40:43], s52 offen offset:2048 sc0 sc1 lds\n\t" \
  "v_mul_f32 v244, v240, v244\n\t" \
  "v_mfma_f32_16x16x128_f8f6f4 v[108:111], v[144:151], v[184:191], v[108:111] blgp:1\n\t" \
  "buffer_load_dwordx4 v252, s[40:43], s52 offen offset:3072 sc0 sc1 lds\n\t" \
  "v_mul_f32 v245, v241, v245\n\t" \
  "s_waitcnt lgkmcnt(0)\n\t" \
  "v_mfma_f32_16x16x128_f8f6f4 v[124:127], v[152:159], v[184:191], v[124:127] blgp:1\n\t" \
  "v_mul_f32 v246, v242, v246\n\t" \
  "v_mfma_f32_16x16x128_f8f6f4 v[120:123], v[152:159], v[176:183], v[120:123] blgp:1\n\t" \
  "v_mul_f32 v247, v243, v247\n\t" \
  "v_mfma_f32_16x16x128_f8f6f4 v[116:119], v[152:159], v[168:175], v[116:119] blgp:1\n\t" \
  "v_mfma_f32_16x16x128_f8f6f4 v[112:115], v[152:159], v[160:167], v[112:115] blgp:1\n\t" \
  "s_waitcnt vmcnt(18) lgkmcnt(0)\n\t" \
  "s_barrier\n\t" \
  "v_mfma_f32_16x16x128_f8f6f4 a[64:67], v[128:135], v[192:199], a[64:67] blgp:1\n\t" \
  "ds_read_b128 v[160:163], v251 offset:0\n\t" \
  "v_accvgpr_read_b32 v240, a178\n\t" \
  "v_mfma_f32_16x16x128_f8f6f4 a[68:71], v[128:135], v[200:207], a[68:71] blgp:1\n\t" \
  "ds_read_b128 v[164:167], v251 offset:8192\n\t" \
  "s_add_u32 s57, s80, 3211264\n\t" \
  "v_accvgpr_read_b32 v241, a182\n\t" \
  "v_mfma_f32_16x16x128_f8f6f4 a[72:75], v[128:135], v[208:215], a[72:75] blgp:1\n\t" \
  "buffer_store_dwordx4 v[244:247], v253, s[44:47], s57 offen offset:0 nt\n\t" \
  "ds_read_b128 v[168:171], v251 offset:256\n\t" \
  "v_accvgpr_read_b32 v242, a186\n\t" \
  "v_mfma_f32_16x16x128_f8f6f4 a[76:79], v[128:135], v[216:223], a[76:79] blgp:1\n\t" \
  "ds_read_b128 v[128:131], v250 offset:0\n\t" \
  "ds_read_b128 v[132:135], v250 offset:8192\n\t" \
  "v_accvgpr_read_b32 v243, a190\n\t" \
  "v_mfma_f32_16x16x128_f8f6f4 a[92:95], v[136:143], v[216:223], a[92:95] blgp:1\n\t" \
  "ds_read_b128 v[172:175], v251 offset:8448\n\t" \
  "v_mul_f32 v244, s58, v240\n\t" \
  "v_mfma_f32_16x16x128_f8f6f4 a[88:91], v[136:143], v[208:215], a[88:91] blgp:1\n\t" \
  "ds_read_b128 v[176:179], v251 offset:512\n\t" \
  "v_mul_f32 v245, s58, v241\n\t" \
  "v_mfma_f32_16x16x128_f8f6f4 a[84:87], v[136:143], v[200:207], a[84:87] blgp:1\n\t" \
  "ds_read_b128 v[180:183], v251 offset:8704\n\t" \
  "v_mul_f32 v246, s58, v242\n\t" \
  "v_mfma_f32_16x16x128_f8f6f4 a[80:83], v[136:143], v[192:199], a[80:83] blgp:1\n\t" \
  "ds_read_b128 v[136:139], v250 offset:256\n\t" \
  "ds_read_b128 v[140:143], v250 offset:8448\n\t" \
  "v_mul_f32 v247, s58, v243\n\t" \
  "v_mfma_f32_16x16x128_f8f6f4 a[96:99], v[144:151], v[192:199], a[96:99] blgp:1\n\t" \
  "ds_read_b128 v[184:187], v251 offset:768\n\t" \
  "v_med3_f32 v244, v244, -s62, s62\n\t" \
  "v_mfma_f32_16x16x128_f8f6f4 a[100:103], v[144:151], v[200:207], a[100:103] blgp:1\n\t" \
  "ds_read_b128 v[188:191], v251 offset:8960\n\t" \
  "v_med3_f32 v245, v245, -s62, s62\n\t" \
  "v_mfma_f32_16x16x128_f8f6f4 a[104:107], v[144:151], v[208:215], a[104:107] blgp:1\n\t" \
  "s_add_u32 s48, s48, 0x10000\n\t" \
  "s_add_u32 s49, s49, 0x10000\n\t" \
  "v_med3_f32 v246, v246, -s62, s62\n\t" \
  "v_mfma_f32_16x16x128_f8f6f4 a[108:111], v[144:151], v[216:223], a[108:111] blgp:1\n\t" \
  "ds_read_b128 v[144:147], v250 offset:512\n\t" \
  "v_med3_f32 v247, v247, -s62, s62\n\t" \
  "v_mfma_f32_16x16x128_f8f6f4 a[124:127], v[152:159], v[216:223], a[124:127] blgp:1\n\t" \
  "ds_read_b128 v[148:151], v250 offset:8704\n\t" \
  "v_exp_f32 v244, v244\n\t" \
  "v_mfma_f32_16x16x128_f8f6f4 a[120:123], v[152:159], v[208:215], a[120:123] blgp:1\n\t" \
  "v_exp_f32 v245, v245\n\t" \
  "v_mfma_f32_16x16x128_f8f6f4 a[116:119], v[152:159], v[200:207], a[116:119] blgp:1\n\t" \
  "v_exp_f32 v246, v246\n\t" \
  "v_mfma_f32_16x16x128_f8f6f4 a[112:115], v[152:159], v[192:199], a[112:115] blgp:1\n\t" \
  "ds_read_b128 v[152:155], v250 offset:768\n\t" \
  "ds_read_b128 v[156:159], v250 offset:8960\n\t" \
  "s_waitcnt lgkmcnt(2)\n\t" \
  "v_mfma_f32_16x16x128_f8f6f4 a[0:3], v[128:135], v[160:167], a[0:3] blgp:1\n\t" \
  "ds_read_b128 v[192:195], v251 offset:16384\n\t" \
  "v_exp_f32 v247, v247\n\t" \
  "v_mfma_f32_16x16x128_f8f6f4 a[4:7], v[128:135], v[168:175], a[4:7] blgp:1\n\t" \
  "ds_read_b128 v[196:199], v251 offset:24576\n\t" \
  "v_mfma_f32_16x16x128_f8f6f4 a[8:11], v[128:135], v[176:183], a[8:11] blgp:1\n\t" \
  "ds_read_b128 v[200:203], v251 offset:16640\n\t" \
  "v_fma_f32 v244, v244, s60, s60\n\t" \
  "v_mfma_f32_16x16x128_f8f6f4 a[12:15], v[128:135], v[184:191], a[12:15] blgp:1\n\t" \
  "ds_read_b128 v[204:207], v251 offset:24832\n\t" \
  "v_fma_f32 v245, v245, s60, s60\n\t" \
  "v_mfma_f32_16x16x128_f8f6f4 a[28:31], v[136:143], v[184:191], a[28:31] blgp:1\n\t" \
  "ds_read_b128 v[208:211], v251 offset:16896\n\t" \
  "v_fma_f32 v246, v246, s60, s60\n\t" \
  "v_mfma_f32_16x16x128_f8f6f4 a[24:27], v[136:143], v[176:183], a[24:27] blgp:1\n\t" \
  "ds_read_b128 v[212:215], v251 offset:25088\n\t" \
  "v_fma_f32 v247, v247, s60, s60\n\t" \
  "v_mfma_f32_16x16x128_f8f6f4 a[20:23], v[136:143], v[168:175], a[20:23] blgp:1\n\t" \
  "ds_read_b128 v[216:219], v251 offset:17152\n\t" \
  "v_rcp_f32 v244, v244\n\t" \
  "v_mfma_f32_16x16x128_f8f6f4 a[16:19], v[136:143], v[160:167], a[16:19] blgp:1\n\t" \
  "ds_read_b128 v[220:223], v251 offset:25344\n\t" \
  "s_add_i32 s52, s48, -16384\n\t" \
  "s_add_u32 m0, s53, 81920\n\t" \
  "v_rcp_f32 v245, v245\n\t" \
  "v_mfma_f32_16x16x128_f8f6f4 a[32:35], v[144:151], v[160:167], a[32:35] blgp:1\n\t" \
  "buffer_load_dwordx4 v252, s[40:43], s52 offen offset:0 sc0 sc1 lds\n\t" \
  "v_rcp_f32 v246, v246\n\t" \
  "v_mfma_f32_16x16x128_f8f6f4 a[36:39], v[144:151], v[168:175], a[36:39] blgp:1\n\t" \
  "v_rcp_f32 v247, v247\n\t" \
  "v_mfma_f32_16x16x128_f8f6f4 a[40:43], v[144:151], v[176:183], a[40:43] blgp:1\n\t" \
  "buffer_load_dwordx4 v252, s[40:43], s52 offen offset:1024 sc0 sc1 lds\n\t" \
  "v_mul_f32 v244, v240, v244\n\t" \
  "v_mfma_f32_16x16x128_f8f6f4 a[44:47], v[144:151], v[184:191], a[44:47] blgp:1\n\t" \
  "v_mul_f32 v245, v241, v245\n\t" \
  "s_waitcnt lgkmcnt(0)\n\t" \
  "v_mfma_f32_16x16x128_f8f6f4 a[60:63], v[152:159], v[184:191], a[60:63] blgp:1\n\t" \
  "buffer_load_dwordx4 v252, s[40:43], s52 offen offset:2048 sc0 sc1 lds\n\t" \
  "v_mul_f32 v246, v242, v246\n\t" \
  "v_mfma_f32_16x16x128_f8f6f4 a[56:59], v[152:159], v[176:183], a[56:59] blgp:1\n\t" \
  "v_mul_f32 v247, v243, v247\n\t" \
  "v_mfma_f32_16x16x128_f8f6f4 a[52:55], v[152:159], v[168:175], a[52:55] blgp:1\n\t" \
  "buffer_load_dwordx4 v252, s[40:43], s52 offen offset:3072 sc0 sc1 lds\n\t" \
  "buffer_load_dword v254, off, s[88:91], s81 sc1\n\t" \
  "v_mfma_f32_16x16x128_f8f6f4 a[48:51], v[152:159], v[160:167], a[48:51] blgp:1\n\t" \
  "s_waitcnt vmcnt(19) lgkmcnt(0)\n\t" \
  "s_barrier\n\t" \
  "v_mfma_f32_16x16x128_f8f6f4 v[0:3], v[128:135], v[192:199], v[0:3] blgp:1\n\t" \
  "v_readlane_b32 s65, v255, s67\n\t" \
  "v_accvgpr_read_b32 v240, a179\n\t" \
  "v_mfma_f32_16x16x128_f8f6f4 v[4:7], v[128:135], v[200:207], v[4:7] blgp:1\n\t" \
  "s_add_u32 s57, s80, 3276800\n\t" \
  "v_accvgpr_read_b32 v241, a183\n\t" \
  "v_mfma_f32_16x16x128_f8f6f4 v[8:11], v[128:135], v[208:215], v[8:11] blgp:1\n\t" \
  "buffer_store_dwordx4 v[244:247], v253, s[44:47], s57 offen offset:0 nt\n\t" \
  "s_add_u32 s67, s67, 1\n\t" \
  "s_cmp_ge_u32 s69, s65\n\t" \
  "s_cbranch_scc0 L_w13_%=\n\t" \
  "L_ok13_%=:\n\t" \
  "v_accvgpr_read_b32 v242, a187\n\t" \
  "v_mfma_f32_16x16x128_f8f6f4 v[12:15], v[128:135], v[216:223], v[12:15] blgp:1\n\t" \
  "ds_read_b128 v[128:131], v250 offset:16384\n\t" \
  "v_accvgpr_read_b32 v243, a191\n\t" \
  "v_mfma_f32_16x16x128_f8f6f4 v[28:31], v[136:143], v[216:223], v[28:31] blgp:1\n\t" \
  "s_add_i32 s52, s48, 0\n\t" \
  "s_add_u32 m0, s53, 32768\n\t" \
  "ds_read_b128 v[132:135], v250 offset:24576\n\t" \
  "v_mul_f32 v244, s58, v240\n\t" \
  "v_mfma_f32_16x16x128_f8f6f4 v[24:27], v[136:143], v[208:215], v[24:27] blgp:1\n\t" \
  "buffer_load_dwordx4 v252, s[40:43], s52 offen offset:0 sc0 sc1 lds\n\t" \
  "v_mul_f32 v245, s58, v241\n\t" \
  "v_mfma_f32_16x16x128_f8f6f4 v[20:23], v[136:143], v[200:207], v[20:23] blgp:1\n\t" \
  "buffer_load_dwordx4 v252, s[40:43], s52 offen offset:1024 sc0 sc1 lds\n\t" \
  "v_mul_f32 v246, s58, v242\n\t" \
  "v_mfma_f32_16x16x128_f8f6f4 v[16:19], v[136:143], v[192:199], v[16:19] blgp:1\n\t" \
  "ds_read_b128 v[136:139], v250 offset:16640\n\t" \
  "v_mul_f32 v247, s58, v243\n\t" \
  "v_mfma_f32_16x16x128_f8f6f4 v[32:35], v[144:151], v[192:199], v[32:35] blgp:1\n\t" \
  "ds_read_b128 v[140:143], v250 offset:24832\n\t" \
  "v_med3_f32 v244, v244, -s62, s62\n\t" \
  "v_mfma_f32_16x16x128_f8f6f4 v[36:39], v[144:151], v[200:207], v[36:39] blgp:1\n\t" \
  "buffer_load_dwordx4 v252, s[40:43], s52 offen offset:2048 sc0 sc1 lds\n\t" \
  "v_med3_f32 v245, v245, -s62, s62\n\t" \
  "v_mfma_f32_16x16x128_f8f6f4 v[40:43], v[144:151], v[208:215], v[40:43] blgp:1\n\t" \
  "buffer_load_dwordx4 v252, s[40:43], s52 offen offset:3072 sc0 sc1 lds\n\t" \
  "v_med3_f32 v246, v246, -s62, s62\n\t" \
  "v_mfma_f32_16x16x128_f8f6f4 v[44:47], v[144:151], v[216:223], v[44:47] blgp:1\n\t" \
  "ds_read_b128 v[144:147], v250 offset:16896\n\t" \
  "v_med3_f32 v247, v247, -s62, s62\n\t" \
  "v_mfma_f32_16x16x128_f8f6f4 v[60:63], v[152:159], v[216:223], v[60:63] blgp:1\n\t" \
  "ds_read_b128 v[148:151], v250 offset:25088\n\t" \
  "v_exp_f32 v244, v244\n\t" \
  "v_mfma_f32_16x16x128_f8f6f4 v[56:59], v[152:159], v[208:215], v[56:59] blgp:1\n\t" \
  "v_exp_f32 v245, v245\n\t" \
  "v_mfma_f32_16x16x128_f8f6f4 v[52:55], v[152:159], v[200:207], v[52:55] blgp:1\n\t" \
  "v_exp_f32 v246, v246\n\t" \
  "v_mfma_f32_16x16x128_f8f6f4 v[48:51], v[152:159], v[192:199], v[48:51] blgp:1\n\t" \
  "ds_read_b128 v[152:155], v250 offset:17152\n\t" \
  "ds_read_b128 v[156:159], v250 offset:25344\n\t" \
  "s_waitcnt lgkmcnt(2)\n\t" \
  "v_mfma_f32_16x16x128_f8f6f4 a[64:67], v[128:135], v[192:199], a[64:67] blgp:1\n\t" \
  "s_add_i32 s52, s49, 0\n\t" \
  "s_add_u32 m0, s53, 98304\n\t" \
  "v_exp_f32 v247, v247\n\t" \
  "v_mfma_f32_16x16x128_f8f6f4 a[68:71], v[128:135], v[200:207], a[68:71] blgp:1\n\t" \
  "buffer_load_dwordx4 v252, s[40:43], s52 offen offset:0 sc0 sc1 lds\n\t" \
  "v_mfma_f32_16x16x128_f8f6f4 a[72:75], v[128:135], v[208:215], a[72:75] blgp:1\n\t" \
  "buffer_load_dwordx4 v252, s[40:43], s52 offen offset:1024 sc0 sc1 lds\n\t" \
  "v_fma_f32 v244, v244, s60, s60\n\t" \
  "v_mfma_f32_16x16x128_f8f6f4 a[76:79], v[128:135], v[216:223], a[76:79] blgp:1\n\t" \
  "v_fma_f32 v245, v245, s60, s60\n\t" \
  "v_mfma_f32_16x16x128_f8f6f4 a[92:95], v[136:143], v[216:223], a[92:95] blgp:1\n\t" \
  "buffer_load_dwordx4 v252, s[40:43], s52 offen offset:2048 sc0 sc1 lds\n\t" \
  "v_fma_f32 v246, v246, s60, s60\n\t" \
  "v_mfma_f32_16x16x128_f8f6f4 a[88:91], v[136:143], v[208:215], a[88:91] blgp:1\n\t" \
  "buffer_load_dwordx4 v252, s[40:43], s52 offen offset:3072 sc0 sc1 lds\n\t" \
  "v_fma_f32 v247, v247, s60, s60\n\t" \
  "v_mfma_f32_16x16x128_f8f6f4 a[84:87], v[136:143], v[200:207], a[84:87] blgp:1\n\t" \
  "s_add_i32 s52, s49, 16384\n\t" \
  "s_add_u32 m0, s53, 114688\n\t" \
  "v_rcp_f32 v244, v244\n\t" \
  "v_mfma_f32_16x16x128_f8f6f4 a[80:83], v[136:143], v[192:199], a[80:83] blgp:1\n\t" \
  "buffer_load_dwordx4 v252, s[40:43], s52 offen offset:0 sc0 sc1 lds\n\t" \
  "v_rcp_f32 v245, v245\n\t" \
  "v_mfma_f32_16x16x128_f8f6f4 a[96:99], v[144:151], v[192:199], a[96:99] blgp:1\n\t" \
  "buffer_load_dwordx4 v252, s[40:43], s52 offen offset:1024 sc0 sc1 lds\n\t" \
  "v_rcp_f32 v246, v246\n\t" \
  "v_mfma_f32_16x16x128_f8f6f4 a[100:103], v[144:151], v[200:207], a[100:103] blgp:1\n\t" \
  "v_rcp_f32 v247, v247\n\t" \
  "v_mfma_f32_16x16x128_f8f6f4 a[104:107], v[144:151], v[208:215], a[104:107] blgp:1\n\t" \
  "buffer_load_dwordx4 v252, s[40:43], s52 offen offset:2048 sc0 sc1 lds\n\t" \
  "v_mul_f32 v244, v240, v244\n\t" \
  "v_mfma_f32_16x16x128_f8f6f4 a[108:111], v[144:151], v[216:223], a[108:111] blgp:1\n\t" \
  "buffer_load_dwordx4 v252, s[40:43], s52 offen offset:3072 sc0 sc1 lds\n\t" \
  "v_mul_f32 v245, v241, v245\n\t" \
  "s_waitcnt lgkmcnt(0)\n\t" \
  "v_mfma_f32_16x16x128_f8f6f4 a[124:127], v[152:159], v[216:223], a[124:127] blgp:1\n\t" \
  "v_mul_f32 v246, v242, v246\n\t" \
  "v_mfma_f32_16x16x128_f8f6f4 a[120:123], v[152:159], v[208:215], a[120:123] blgp:1\n\t" \
  "v_mul_f32 v247, v243, v247\n\t" \
  "v_mfma_f32_16x16x128_f8f6f4 a[116:119], v[152:159], v[200:207], a[116:119] blgp:1\n\t" \
  "v_mfma_f32_16x16x128_f8f6f4 a[112:115], v[152:159], v[192:199], a[112:115] blgp:1\n\t" \
  "s_waitcnt vmcnt(19) lgkmcnt(0)\n\t" \
  "s_barrier\n\t" \
  "v_mfma_f32_16x16x128_f8f6f4 v[64:67], v[128:135], v[160:167], v[64:67] blgp:1\n\t" \
  "ds_read_b128 v[192:195], v251 offset:49152\n\t" \
  "v_mfma_f32_16x16x128_f8f6f4 v[68:71], v[128:135], v[168:175], v[68:71] blgp:1\n\t" \
  "ds_read_b128 v[196:199], v251 offset:57344\n\t" \
  "s_add_u32 s57, s80, 3342336\n\t" \
  "v_mfma_f32_16x16x128_f8f6f4 v[72:75], v[128:135], v[176:183], v[72:75] blgp:1\n\t" \
  "buffer_store_dwordx4 v[244:247], v253, s[44:47], s57 offen offset:0 nt\n\t" \
  "ds_read_b128 v[200:203], v251 offset:49408\n\t" \
  "v_mfma_f32_16x16x128_f8f6f4 v[76:79], v[128:135], v[184:191], v[76:79] blgp:1\n\t" \
  "ds_read_b128 v[128:131], v250 offset:32768\n\t" \
  "ds_read_b128 v[132:135], v250 offset:40960\n\t" \
  "v_mfma_f32_16x16x128_f8f6f4 v[92:95], v[136:143], v[184:191], v[92:95] blgp:1\n\t" \
  "ds_read_b128 v[204:207], v251 offset:57600\n\t" \
  "v_mul_f32 v244, s58, v224\n\t" \
  "v_mfma_f32_16x16x128_f8f6f4 v[88:91], v[136:143], v[176:183], v[88:91] blgp:1\n\t" \
  "ds_read_b128 v[208:211], v251 offset:49664\n\t" \
  "v_mul_f32 v245, s58, v228\n\t" \
  "v_mfma_f32_16x16x128_f8f6f4 v[84:87], v[136:143], v[168:175], v[84:87] blgp:1\n\t" \
  "ds_read_b128 v[212:215], v251 offset:57856\n\t" \
  "v_mul_f32 v246, s58, v232\n\t" \
  "v_mfma_f32_16x16x128_f8f6f4 v[80:83], v[136:143], v[160:167], v[80:83] blgp:1\n\t" \
  "ds_read_b128 v[136:139], v250 offset:33024\n\t" \
  "ds_read_b128 v[140:143], v250 offset:41216\n\t" \
  "v_mul_f32 v247, s58, v236\n\t" \
  "v_mfma_f32_16x16x128_f8f6f4 v[96:99], v[144:151], v[160:167], v[96:99] blgp:1\n\t" \
  "ds_read_b128 v[216:219], v251 offset:49920\n\t" \
  "v_med3_f32 v244, v244, -s62, s62\n\t" \
  "v_mfma_f32_16x16x128_f8f6f4 v[100:103], v[144:151], v[168:175], v[100:103] blgp:1\n\t" \
  "ds_read_b128 v[220:223], v251 offset:58112\n\t" \
  "v_med3_f32 v245, v245, -s62, s62\n\t" \
  "v_mfma_f32_16x16x128_f8f6f4 v[104:107], v[144:151], v[176:183], v[104:107] blgp:1\n\t" \
  "v_med3_f32 v246, v246, -s62, s62\n\t" \
  "v_mfma_f32_16x16x128_f8f6f4 v[108:111], v[144:151], v[184:191], v[108:111] blgp:1\n\t" \
  "ds_read_b128 v[144:147], v250 offset:33280\n\t" \
  "v_med3_f32 v247, v247, -s62, s62\n\t" \
  "v_mfma_f32_16x16x128_f8f6f4 v[124:127], v[152:159], v[184:191], v[124:127] blgp:1\n\t" \
  "ds_read_b128 v[148:151], v250 offset:41472\n\t" \
  "v_exp_f32 v244, v244\n\t" \
  "v_mfma_f32_16x16x128_f8f6f4 v[120:123], v[152:159], v[176:183], v[120:123] blgp:1\n\t" \
  "v_exp_f32 v245, v245\n\t" \
  "v_mfma_f32_16x16x128_f8f6f4 v[116:119], v[152:159], v[168:175], v[116:119] blgp:1\n\t" \
  "v_exp_f32 v246, v246\n\t" \
  "v_mfma_f32_16x16x128_f8f6f4 v[112:115], v[152:159], v[160:167], v[112:115] blgp:1\n\t" \
  "ds_read_b128 v[152:155], v250 offset:33536\n\t" \
  "ds_read_b128 v[156:159], v250 offset:41728\n\t" \
  "s_waitcnt lgkmcnt(2)\n\t" \
  "v_mfma_f32_16x16x128_f8f6f4 v[0:3], v[128:135], v[192:199], v[0:3] blgp:1\n\t" \
  "ds_read_b128 v[160:163], v251 offset:32768\n\t" \
  "v_exp_f32 v247, v247\n\t" \
  "v_mfma_f32_16x16x128_f8f6f4 v[4:7], v[128:135], v[200:207], v[4:7] blgp:1\n\t" \
  "ds_read_b128 v[164:167], v251 offset:40960\n\t" \
  "v_mfma_f32_16x16x128_f8f6f4 v[8:11], v[128:135], v[208:215], v[8:11] blgp:1\n\t" \
  "ds_read_b128 v[168:171], v251 offset:33024\n\t" \
  "v_fma_f32 v244, v244, s60, s60\n\t" \
  "v_mfma_f32_16x16x128_f8f6f4 v[12:15], v[128:135], v[216:223], v[12:15] blgp:1\n\t" \
  "ds_read_b128 v[172:175], v251 offset:41216\n\t" \
  "v_fma_f32 v245, v245, s60, s60\n\t" \
  "v_mfma_f32_16x16x128_f8f6f4 v[28:31], v[136:143], v[216:223], v[28:31] blgp:1\n\t" \
  "ds_read_b128 v[176:179], v251 offset:33280\n\t" \
  "v_fma_f32 v246, v246, s60, s60\n\t" \
  "v_mfma_f32_16x16x128_f8f6f4 v[24:27], v[136:143], v[208:215], v[24:27] blgp:1\n\t" \
  "ds_read_b128 v[180:183], v251 offset:41472\n\t" \
  "v_fma_f32 v247, v247, s60, s60\n\t" \
  "v_mfma_f32_16x16x128_f8f6f4 v[20:23], v[136:143], v[200:207], v[20:23] blgp:1\n\t" \
  "ds_read_b128 v[184:187], v251 offset:33536\n\t" \
  "v_rcp_f32 v244, v244\n\t" \
  "v_mfma_f32_16x16x128_f8f6f4 v[16:19], v[136:143], v[192:199], v[16:19] blgp:1\n\t" \
  "ds_read_b128 v[188:191], v251 offset:41728\n\t" \
  "s_add_i32 s52, s48, 16384\n\t" \
  "s_add_u32 m0, s53, 49152\n\t" \
  "v_rcp_f32 v245, v245\n\t" \
  "v_mfma_f32_16x16x128_f8f6f4 v[32:35], v[144:151], v[192:199], v[32:35] blgp:1\n\t" \
  "buffer_load_dwordx4 v252, s[40:43], s52 offen offset:0 sc0 sc1 lds\n\t" \
  "v_rcp_f32 v246, v246\n\t" \
  "v_mfma_f32_16x16x128_f8f6f4 v[36:39], v[144:151], v[200:207], v[36:39] blgp:1\n\t" \
  "v_rcp_f32 v247, v247\n\t" \
  "v_mfma_f32_16x16x128_f8f6f4 v[40:43], v[144:151], v[208:215], v[40:43] blgp:1\n\t" \
  "buffer_load_dwordx4 v252, s[40:43], s52 offen offset:1024 sc0 sc1 lds\n\t" \
  "v_mul_f32 v244, v224, v244\n\t" \
  "v_mfma_f32_16x16x128_f8f6f4 v[44:47], v[144:151], v[216:223], v[44:47] blgp:1\n\t" \
  "v_mul_f32 v245, v228, v245\n\t" \
  "s_waitcnt lgkmcnt(0)\n\t" \
  "v_mfma_f32_16x16x128_f8f6f4 v[60:63], v[152:159], v[216:223], v[60:63] blgp:1\n\t" \
  "buffer_load_dwordx4 v252, s[40:43], s52 offen offset:2048 sc0 sc1 lds\n\t" \
  "v_mul_f32 v246, v232, v246\n\t" \
  "v_mfma_f32_16x16x128_f8f6f4 v[56:59], v[152:159], v[208:215], v[56:59] blgp:1\n\t" \
  "v_mul_f32 v247, v236, v247\n\t" \
  "v_mfma_f32_16x16x128_f8f6f4 v[52:55], v[152:159], v[200:207], v[52:55] blgp:1\n\t" \
  "buffer_load_dwordx4 v252, s[40:43], s52 offen offset:3072 sc0 sc1 lds\n\t" \
  "v_mfma_f32_16x16x128_f8f6f4 v[48:51], v[152:159], v[192:199], v[48:51] blgp:1\n\t" \
  "s_waitcnt vmcnt(18) lgkmcnt(0)\n\t" \
  "s_barrier\n\t" \
  "v_mfma_f32_16x16x128_f8f6f4 a[0:3], v[128:135], v[160:167], a[0:3] blgp:1\n\t" \
  "v_readfirstlane_b32 s87, v254\n\t" \
  "v_readlane_b32 s65, v255, s67\n\t" \
  "v_mfma_f32_16x16x128_f8f6f4 a[4:7], v[128:135], v[168:175], a[4:7] blgp:1\n\t" \
  "s_add_u32 s57, s80, 6291456\n\t" \
  "v_mfma_f32_16x16x128_f8f6f4 a[8:11], v[128:135], v[176:183], a[8:11] blgp:1\n\t" \
  "buffer_store_dwordx4 v[244:247], v253, s[44:47], s57 offen offset:0 nt\n\t" \
  "s_max_u32 s69, s69, s87\n\t" \
  "s_add_u32 s67, s67, 1\n\t" \
  "s_cmp_ge_u32 s69, s65\n\t" \
  "s_cbranch_scc0 L_w14_%=\n\t" \
  "L_ok14_%=:\n\t" \
  "v_mfma_f32_16x16x128_f8f6f4 a[12:15], v[128:135], v[184:191], a[12:15] blgp:1\n\t" \
  "ds_read_b128 v[128:131], v250 offset:49152\n\t" \
  "v_mfma_f32_16x16x128_f8f6f4 a[28:31], v[136:143], v[184:191], a[28:31] blgp:1\n\t" \
  "s_add_i32 s52, s48, 32768\n\t" \
  "s_add_u32 m0, s53, 65536\n\t" \
  "ds_read_b128 v[132:135], v250 offset:57344\n\t" \
  "v_mul_f32 v244, s58, v225\n\t" \
  "v_mfma_f32_16x16x128_f8f6f4 a[24:27], v[136:143], v[176:183], a[24:27] blgp:1\n\t" \
  "buffer_load_dwordx4 v252, s[40:43], s52 offen offset:0 sc0 sc1 lds\n\t" \
  "v_mul_f32 v245, s58, v229\n\t" \
  "v_mfma_f32_16x16x128_f8f6f4 a[20:23], v[136:143], v[168:175], a[20:23] blgp:1\n\t" \
  "buffer_load_dwordx4 v252, s[40:43], s52 offen offset:1024 sc0 sc1 lds\n\t" \
  "v_mul_f32 v246, s58, v233\n\t" \
  "v_mfma_f32_16x16x128_f8f6f4 a[16:19], v[136:143], v[160:167], a[16:19] blgp:1\n\t" \
  "ds_read_b128 v[136:139], v250 offset:49408\n\t" \
  "v_mul_f32 v247, s58, v237\n\t" \
  "v_mfma_f32_16x16x128_f8f6f4 a[32:35], v[144:151], v[160:167], a[32:35] blgp:1\n\t" \
  "ds_read_b128 v[140:143], v250 offset:57600\n\t" \
  "v_med3_f32 v244, v244, -s62, s62\n\t" \
  "v_mfma_f32_16x16x128_f8f6f4 a[36:39], v[144:151], v[168:175], a[36:39] blgp:1\n\t" \
  "buffer_load_dwordx4 v252, s[40:43], s52 offen offset:2048 sc0 sc1 lds\n\t" \
  "v_med3_f32 v245, v245, -s62, s62\n\t" \
  "v_mfma_f32_16x16x128_f8f6f4 a[40:43], v[144:151], v[176:183], a[40:43] blgp:1\n\t" \
  "buffer_load_dwordx4 v252, s[40:43], s52 offen offset:3072 sc0 sc1 lds\n\t" \
  "v_med3_f32 v246, v246, -s62, s62\n\t" \
  "v_mfma_f32_16x16x128_f8f6f4 a[44:47], v[144:151], v[184:191], a[44:47] blgp:1\n\t" \
  "ds_read_b128 v[144:147], v250 offset:49664\n\t" \
  "v_med3_f32 v247, v247, -s62, s62\n\t" \
  "v_mfma_f32_16x16x128_f8f6f4 a[60:63], v[152:159], v[184:191], a[60:63] blgp:1\n\t" \
  "ds_read_b128 v[148:151], v250 offset:57856\n\t" \
  "v_exp_f32 v244, v244\n\t" \
  "v_mfma_f32_16x16x128_f8f6f4 a[56:59], v[152:159], v[176:183], a[56:59] blgp:1\n\t" \
  "v_exp_f32 v245, v245\n\t" \
  "v_mfma_f32_16x16x128_f8f6f4 a[52:55], v[152:159], v[168:175], a[52:55] blgp:1\n\t" \
  "v_exp_f32 v246, v246\n\t" \
  "v_mfma_f32_16x16x128_f8f6f4 a[48:51], v[152:159], v[160:167], a[48:51] blgp:1\n\t" \
  "ds_read_b128 v[152:155], v250 offset:49920\n\t" \
  "ds_read_b128 v[156:159], v250 offset:58112\n\t" \
  "s_waitcnt lgkmcnt(2)\n\t" \
  "v_mfma_f32_16x16x128_f8f6f4 v[64:67], v[128:135], v[160:167], v[64:67] blgp:1\n\t" \
  "s_add_i32 s52, s49, 49152\n\t" \
  "s_add_u32 m0, s53, 147456\n\t" \
  "v_exp_f32 v247, v247\n\t" \
  "v_mfma_f32_16x16x128_f8f6f4 v[68:71], v[128:135], v[168:175], v[68:71] blgp:1\n\t" \
  "buffer_load_dwordx4 v252, s[40:43], s52 offen offset:0 sc0 sc1 lds\n\t" \
  "v_mfma_f32_16x16x128_f8f6f4 v[72:75], v[128:135], v[176:183], v[72:75] blgp:1\n\t" \
  "buffer_load_dwordx4 v252, s[40:43], s52 offen offset:1024 sc0 sc1 lds\n\t" \
  "v_fma_f32 v244, v244, s60, s60\n\t" \
  "v_mfma_f32_16x16x128_f8f6f4 v[76:79], v[128:135], v[184:191], v[76:79] blgp:1\n\t" \
  "v_fma_f32 v245, v245, s60, s60\n\t" \
  "v_mfma_f32_16x16x128_f8f6f4 v[92:95], v[136:143], v[184:191], v[92:95] blgp:1\n\t" \
  "buffer_load_dwordx4 v252, s[40:43], s52 offen offset:2048 sc0 sc1 lds\n\t" \
  "v_fma_f32 v246, v246, s60, s60\n\t" \
  "v_mfma_f32_16x16x128_f8f6f4 v[88:91], v[136:143], v[176:183], v[88:91] blgp:1\n\t" \
  "buffer_load_dwordx4 v252, s[40:43], s52 offen offset:3072 sc0 sc1 lds\n\t" \
  "v_fma_f32 v247, v247, s60, s60\n\t" \
  "v_mfma_f32_16x16x128_f8f6f4 v[84:87], v[136:143], v[168:175], v[84:87] blgp:1\n\t" \
  "s_add_i32 s52, s49, 32768\n\t" \
  "s_add_u32 m0, s53, 131072\n\t" \
  "v_rcp_f32 v244, v244\n\t" \
  "v_mfma_f32_16x16x128_f8f6f4 v[80:83], v[136:143], v[160:167], v[80:83] blgp:1\n\t" \
  "buffer_load_dwordx4 v252, s[40:43], s52 offen offset:0 sc0 sc1 lds\n\t" \
  "v_rcp_f32 v245, v245\n\t" \
  "v_mfma_f32_16x16x128_f8f6f4 v[96:99], v[144:151], v[160:167], v[96:99] blgp:1\n\t" \
  "buffer_load_dwordx4 v252, s[40:43], s52 offen offset:1024 sc0 sc1 lds\n\t" \
  "v_rcp_f32 v246, v246\n\t" \
  "v_mfma_f32_16x16x128_f8f6f4 v[100:103], v[144:151], v[168:175], v[100:103] blgp:1\n\t" \
  "v_rcp_f32 v247, v247\n\t" \
  "v_mfma_f32_16x16x128_f8f6f4 v[104:107], v[144:151], v[176:183], v[104:107] blgp:1\n\t" \
  "buffer_load_dwordx4 v252, s[40:43], s52 offen offset:2048 sc0 sc1 lds\n\t" \
  "v_mul_f32 v244, v225, v244\n\t" \
  "v_mfma_f32_16x16x128_f8f6f4 v[108:111], v[144:151], v[184:191], v[108:111] blgp:1\n\t" \
  "buffer_load_dwordx4 v252, s[40:43], s52 offen offset:3072 sc0 sc1 lds\n\t" \
  "v_mul_f32 v245, v229, v245\n\t" \
  "s_waitcnt lgkmcnt(0)\n\t" \
  "v_mfma_f32_16x16x128_f8f6f4 v[124:127], v[152:159], v[184:191], v[124:127] blgp:1\n\t" \
  "v_mul_f32 v246, v233, v246\n\t" \
  "v_mfma_f32_16x16x128_f8f6f4 v[120:123], v[152:159], v[176:183], v[120:123] blgp:1\n\t" \
  "v_mul_f32 v247, v237, v247\n\t" \
  "v_mfma_f32_16x16x128_f8f6f4 v[116:119], v[152:159], v[168:175], v[116:119] blgp:1\n\t" \
  "v_mfma_f32_16x16x128_f8f6f4 v[112:115], v[152:159], v[160:167], v[112:115] blgp:1\n\t" \
  "s_waitcnt vmcnt(18) lgkmcnt(0)\n\t" \
  "s_barrier\n\t" \
  "v_mfma_f32_16x16x128_f8f6f4 a[64:67], v[128:135], v[192:199], a[64:67] blgp:1\n\t" \
  "ds_read_b128 v[160:163], v251 offset:0\n\t" \
  "v_mfma_f32_16x16x128_f8f6f4 a[68:71], v[128:135], v[200:207], a[68:71] blgp:1\n\t" \
  "ds_read_b128 v[164:167], v251 offset:8192\n\t" \
  "s_add_u32 s57, s80, 6356992\n\t" \
  "v_mfma_f32_16x16x128_f8f6f4 a[72:75], v[128:135], v[208:215], a[72:75] blgp:1\n\t" \
  "buffer_store_dwordx4 v[244:247], v253, s[44:47], s57 offen offset:0 nt\n\t" \
  "ds_read_b128 v[168:171], v251 offset:256\n\t" \
  "v_mfma_f32_16x16x128_f8f6f4 a[76:79], v[128:135], v[216:223], a[76:79] blgp:1\n\t" \
  "ds_read_b128 v[128:131], v250 offset:0\n\t" \
  "ds_read_b128 v[132:135], v250 offset:8192\n\t" \
  "v_mfma_f32_16x16x128_f8f6f4 a[92:95], v[136:143], v[216:223], a[92:95] blgp:1\n\t" \
  "ds_read_b128 v[172:175], v251 offset:8448\n\t" \
  "v_mul_f32 v244, s58, v226\n\t" \
  "v_mfma_f32_16x16x128_f8f6f4 a[88:91], v[136:143], v[208:215], a[88:91] blgp:1\n\t" \
  "ds_read_b128 v[176:179], v251 offset:512\n\t" \
  "v_mul_f32 v245, s58, v230\n\t" \
  "v_mfma_f32_16x16x128_f8f6f4 a[84:87], v[136:143], v[200:207], a[84:87] blgp:1\n\t" \
  "ds_read_b128 v[180:183], v251 offset:8704\n\t" \
  "v_mul_f32 v246, s58, v234\n\t" \
  "v_mfma_f32_16x16x128_f8f6f4 a[80:83], v[136:143], v[192:199], a[80:83] blgp:1\n\t" \
  "ds_read_b128 v[136:139], v250 offset:256\n\t" \
  "ds_read_b128 v[140:143], v250 offset:8448\n\t" \
  "v_mul_f32 v247, s58, v238\n\t" \
  "v_mfma_f32_16x16x128_f8f6f4 a[96:99], v[144:151], v[192:199], a[96:99] blgp:1\n\t" \
  "ds_read_b128 v[184:187], v251 offset:768\n\t" \
  "v_med3_f32 v244, v244, -s62, s62\n\t" \
  "v_mfma_f32_16x16x128_f8f6f4 a[100:103], v[144:151], v[200:207], a[100:103] blgp:1\n\t" \
  "ds_read_b128 v[188:191], v251 offset:8960\n\t" \
  "v_med3_f32 v245, v245, -s62, s62\n\t" \
  "v_mfma_f32_16x16x128_f8f6f4 a[104:107], v[144:151], v[208:215], a[104:107] blgp:1\n\t" \
  "s_add_u32 s48, s48, 0x10000\n\t" \
  "s_add_u32 s49, s49, 0x10000\n\t" \
  "v_med3_f32 v246, v246, -s62, s62\n\t" \
  "v_mfma_f32_16x16x128_f8f6f4 a[108:111], v[144:151], v[216:223], a[108:111] blgp:1\n\t" \
  "ds_read_b128 v[144:147], v250 offset:512\n\t" \
  "v_med3_f32 v247, v247, -s62, s62\n\t" \
  "v_mfma_f32_16x16x128_f8f6f4 a[124:127], v[152:159], v[216:223], a[124:127] blgp:1\n\t" \
  "ds_read_b128 v[148:151], v250 offset:8704\n\t" \
  "v_exp_f32 v244, v244\n\t" \
  "v_mfma_f32_16x16x128_f8f6f4 a[120:123], v[152:159], v[208:215], a[120:123] blgp:1\n\t" \
  "v_exp_f32 v245, v245\n\t" \
  "v_mfma_f32_16x16x128_f8f6f4 a[116:119], v[152:159], v[200:207], a[116:119] blgp:1\n\t" \
  "v_exp_f32 v246, v246\n\t" \
  "v_mfma_f32_16x16x128_f8f6f4 a[112:115], v[152:159], v[192:199], a[112:115] blgp:1\n\t" \
  "ds_read_b128 v[152:155], v250 offset:768\n\t" \
  "ds_read_b128 v[156:159], v250 offset:8960\n\t" \
  "s_waitcnt lgkmcnt(2)\n\t" \
  "v_mfma_f32_16x16x128_f8f6f4 a[0:3], v[128:135], v[160:167], a[0:3] blgp:1\n\t" \
  "ds_read_b128 v[192:195], v251 offset:16384\n\t" \
  "v_exp_f32 v247, v247\n\t" \
  "v_mfma_f32_16x16x128_f8f6f4 a[4:7], v[128:135], v[168:175], a[4:7] blgp:1\n\t" \
  "ds_read_b128 v[196:199], v251 offset:24576\n\t" \
  "v_mfma_f32_16x16x128_f8f6f4 a[8:11], v[128:135], v[176:183], a[8:11] blgp:1\n\t" \
  "ds_read_b128 v[200:203], v251 offset:16640\n\t" \
  "v_fma_f32 v244, v244, s60, s60\n\t" \
  "v_mfma_f32_16x16x128_f8f6f4 a[12:15], v[128:135], v[184:191], a[12:15] blgp:1\n\t" \
  "ds_read_b128 v[204:207], v251 offset:24832\n\t" \
  "v_fma_f32 v245, v245, s60, s60\n\t" \
  "v_mfma_f32_16x16x128_f8f6f4 a[28:31], v[136:143], v[184:191], a[28:31] blgp:1\n\t" \
  "ds_read_b128 v[208:211], v251 offset:16896\n\t" \
  "v_fma_f32 v246, v246, s60, s60\n\t" \
  "v_mfma_f32_16x16x128_f8f6f4 a[24:27], v[136:143], v[176:183], a[24:27] blgp:1\n\t" \
  "ds_read_b128 v[212:215], v251 offset:25088\n\t" \
  "v_fma_f32 v247, v247, s60, s60\n\t" \
  "v_mfma_f32_16x16x128_f8f6f4 a[20:23], v[136:143], v[168:175], a[20:23] blgp:1\n\t" \
  "ds_read_b128 v[216:219], v251 offset:17152\n\t" \
  "v_rcp_f32 v244, v244\n\t" \
  "v_mfma_f32_16x16x128_f8f6f4 a[16:19], v[136:143], v[160:167], a[16:19] blgp:1\n\t" \
  "ds_read_b128 v[220:223], v251 offset:25344\n\t" \
  "s_add_i32 s52, s48, -16384\n\t" \
  "s_add_u32 m0, s53, 81920\n\t" \
  "v_rcp_f32 v245, v245\n\t" \
  "v_mfma_f32_16x16x128_f8f6f4 a[32:35], v[144:151], v[160:167], a[32:35] blgp:1\n\t" \
  "buffer_load_dwordx4 v252, s[40:43], s52 offen offset:0 sc0 sc1 lds\n\t" \
  "v_rcp_f32 v246, v246\n\t" \
  "v_mfma_f32_16x16x128_f8f6f4 a[36:39], v[144:151], v[168:175], a[36:39] blgp:1\n\t" \
  "v_rcp_f32 v247, v247\n\t" \
  "v_mfma_f32_16x16x128_f8f6f4 a[40:43], v[144:151], v[176:183], a[40:43] blgp:1\n\t" \
  "buffer_load_dwordx4 v252, s[40:43], s52 offen offset:1024 sc0 sc1 lds\n\t" \
  "v_mul_f32 v244, v226, v244\n\t" \
  "v_mfma_f32_16x16x128_f8f6f4 a[44:47], v[144:151], v[184:191], a[44:47] blgp:1\n\t" \
  "v_mul_f32 v245, v230, v245\n\t" \
  "s_waitcnt lgkmcnt(0)\n\t" \
  "v_mfma_f32_16x16x128_f8f6f4 a[60:63], v[152:159], v[184:191], a[60:63] blgp:1\n\t" \
  "buffer_load_dwordx4 v252, s[40:43], s52 offen offset:2048 sc0 sc1 lds\n\t" \
  "v_mul_f32 v246, v234, v246\n\t" \
  "v_mfma_f32_16x16x128_f8f6f4 a[56:59], v[152:159], v[176:183], a[56:59] blgp:1\n\t" \
  "v_mul_f32 v247, v238, v247\n\t" \
  "v_mfma_f32_16x16x128_f8f6f4 a[52:55], v[152:159], v[168:175], a[52:55] blgp:1\n\t" \
  "buffer_load_dwordx4 v252, s[40:43], s52 offen offset:3072 sc0 sc1 lds\n\t" \
  "buffer_load_dword v254, off, s[88:91], s81 sc1\n\t" \
  "v_mfma_f32_16x16x128_f8f6f4 a[48:51], v[152:159], v[160:167], a[48:51] blgp:1\n\t" \
  "s_waitcnt vmcnt(19) lgkmcnt(0)\n\t" \
  "s_barrier\n\t" \
  "v_mfma_f32_16x16x128_f8f6f4 v[0:3], v[128:135], v[192:199], v[0:3] blgp:1\n\t" \
  "v_readlane_b32 s65, v255, s67\n\t" \
  "v_mfma_f32_16x16x128_f8f6f4 v[4:7], v[128:135], v[200:207], v[4:7] blgp:1\n\t" \
  "s_add_u32 s57, s80, 6422528\n\t" \
  "v_mfma_f32_16x16x128_f8f6f4 v[8:11], v[128:135], v[208:215], v[8:11] blgp:1\n\t" \
  "buffer_store_dwordx4 v[244:247], v253, s[44:47], s57 offen offset:0 nt\n\t" \
  "s_add_u32 s67, s67, 1\n\t" \
  "s_cmp_ge_u32 s69, s65\n\t" \
  "s_cbranch_scc0 L_w15_%=\n\t" \
  "L_ok15_%=:\n\t" \
  "v_mfma_f32_16x16x128_f8f6f4 v[12:15], v[128:135], v[216:223], v[12:15] blgp:1\n\t" \
  "ds_read_b128 v[128:131], v250 offset:16384\n\t" \
  "v_mfma_f32_16x16x128_f8f6f4 v[28:31], v[136:143], v[216:223], v[28:31] blgp:1\n\t" \
  "s_add_i32 s52, s48, 0\n\t" \
  "s_add_u32 m0, s53, 32768\n\t" \
  "ds_read_b128 v[132:135], v250 offset:24576\n\t" \
  "v_mul_f32 v244, s58, v227\n\t" \
  "v_mfma_f32_16x16x128_f8f6f4 v[24:27], v[136:143], v[208:215], v[24:27] blgp:1\n\t" \
  "buffer_load_dwordx4 v252, s[40:43], s52 offen offset:0 sc0 sc1 lds\n\t" \
  "v_mul_f32 v245, s58, v231\n\t" \
  "v_mfma_f32_16x16x128_f8f6f4 v[20:23], v[136:143], v[200:207], v[20:23] blgp:1\n\t" \
  "buffer_load_dwordx4 v252, s[40:43], s52 offen offset:1024 sc0 sc1 lds\n\t" \
  "v_mul_f32 v246, s58, v235\n\t" \
  "v_mfma_f32_16x16x128_f8f6f4 v[16:19], v[136:143], v[192:199], v[16:19] blgp:1\n\t" \
  "ds_read_b128 v[136:139], v250 offset:16640\n\t" \
  "v_mul_f32 v247, s58, v239\n\t" \
  "v_mfma_f32_16x16x128_f8f6f4 v[32:35], v[144:151], v[192:199], v[32:35] blgp:1\n\t" \
  "ds_read_b128 v[140:143], v250 offset:24832\n\t" \
  "v_med3_f32 v244, v244, -s62, s62\n\t" \
  "v_mfma_f32_16x16x128_f8f6f4 v[36:39], v[144:151], v[200:207], v[36:39] blgp:1\n\t" \
  "buffer_load_dwordx4 v252, s[40:43], s52 offen offset:2048 sc0 sc1 lds\n\t" \
  "v_med3_f32 v245, v245, -s62, s62\n\t" \
  "v_mfma_f32_16x16x128_f8f6f4 v[40:43], v[144:151], v[208:215], v[40:43] blgp:1\n\t" \
  "buffer_load_dwordx4 v252, s[40:43], s52 offen offset:3072 sc0 sc1 lds\n\t" \
  "v_med3_f32 v246, v246, -s62, s62\n\t" \
  "v_mfma_f32_16x16x128_f8f6f4 v[44:47], v[144:151], v[216:223], v[44:47] blgp:1\n\t" \
  "ds_read_b128 v[144:147], v250 offset:16896\n\t" \
  "v_med3_f32 v247, v247, -s62, s62\n\t" \
  "v_mfma_f32_16x16x128_f8f6f4 v[60:63], v[152:159], v[216:223], v[60:63] blgp:1\n\t" \
  "ds_read_b128 v[148:151], v250 offset:25088\n\t" \
  "v_exp_f32 v244, v244\n\t" \
  "v_mfma_f32_16x16x128_f8f6f4 v[56:59], v[152:159], v[208:215], v[56:59] blgp:1\n\t" \
  "v_exp_f32 v245, v245\n\t" \
  "v_mfma_f32_16x16x128_f8f6f4 v[52:55], v[152:159], v[200:207], v[52:55] blgp:1\n\t" \
  "v_exp_f32 v246, v246\n\t" \
  "v_mfma_f32_16x16x128_f8f6f4 v[48:51], v[152:159], v[192:199], v[48:51] blgp:1\n\t" \
  "ds_read_b128 v[152:155], v250 offset:17152\n\t" \
  "ds_read_b128 v[156:159], v250 offset:25344\n\t" \
  "s_waitcnt lgkmcnt(2)\n\t" \
  "v_mfma_f32_16x16x128_f8f6f4 a[64:67], v[128:135], v[192:199], a[64:67] blgp:1\n\t" \
  "s_add_i32 s52, s49, 0\n\t" \
  "s_add_u32 m0, s53, 98304\n\t" \
  "v_exp_f32 v247, v247\n\t" \
  "v_mfma_f32_16x16x128_f8f6f4 a[68:71], v[128:135], v[200:207], a[68:71] blgp:1\n\t" \
  "buffer_load_dwordx4 v252, s[40:43], s52 offen offset:0 sc0 sc1 lds\n\t" \
  "v_mfma_f32_16x16x128_f8f6f4 a[72:75], v[128:135], v[208:215], a[72:75] blgp:1\n\t" \
  "buffer_load_dwordx4 v252, s[40:43], s52 offen offset:1024 sc0 sc1 lds\n\t" \
  "v_fma_f32 v244, v244, s60, s60\n\t" \
  "v_mfma_f32_16x16x128_f8f6f4 a[76:79], v[128:135], v[216:223], a[76:79] blgp:1\n\t" \
  "v_fma_f32 v245, v245, s60, s60\n\t" \
  "v_mfma_f32_16x16x128_f8f6f4 a[92:95], v[136:143], v[216:223], a[92:95] blgp:1\n\t" \
  "buffer_load_dwordx4 v252, s[40:43], s52 offen offset:2048 sc0 sc1 lds\n\t" \
  "v_fma_f32 v246, v246, s60, s60\n\t" \
  "v_mfma_f32_16x16x128_f8f6f4 a[88:91], v[136:143], v[208:215], a[88:91] blgp:1\n\t" \
  "buffer_load_dwordx4 v252, s[40:43], s52 offen offset:3072 sc0 sc1 lds\n\t" \
  "v_fma_f32 v247, v247, s60, s60\n\t" \
  "v_mfma_f32_16x16x128_f8f6f4 a[84:87], v[136:143], v[200:207], a[84:87] blgp:1\n\t" \
  "s_add_i32 s52, s49, 16384\n\t" \
  "s_add_u32 m0, s53, 114688\n\t" \
  "v_rcp_f32 v244, v244\n\t" \
  "v_mfma_f32_16x16x128_f8f6f4 a[80:83], v[136:143], v[192:199], a[80:83] blgp:1\n\t" \
  "buffer_load_dwordx4 v252, s[40:43], s52 offen offset:0 sc0 sc1 lds\n\t" \
  "v_rcp_f32 v245, v245\n\t" \
  "v_mfma_f32_16x16x128_f8f6f4 a[96:99], v[144:151], v[192:199], a[96:99] blgp:1\n\t" \
  "buffer_load_dwordx4 v252, s[40:43], s52 offen offset:1024 sc0 sc1 lds\n\t" \
  "v_rcp_f32 v246, v246\n\t" \
  "v_mfma_f32_16x16x128_f8f6f4 a[100:103], v[144:151], v[200:207], a[100:103] blgp:1\n\t" \
  "v_rcp_f32 v247, v247\n\t" \
  "v_mfma_f32_16x16x128_f8f6f4 a[104:107], v[144:151], v[208:215], a[104:107] blgp:1\n\t" \
  "buffer_load_dwordx4 v252, s[40:43], s52 offen offset:2048 sc0 sc1 lds\n\t" \
  "v_mul_f32 v244, v227, v244\n\t" \
  "v_mfma_f32_16x16x128_f8f6f4 a[108:111], v[144:151], v[216:223], a[108:111] blgp:1\n\t" \
  "buffer_load_dwordx4 v252, s[40:43], s52 offen offset:3072 sc0 sc1 lds\n\t" \
  "v_mul_f32 v245, v231, v245\n\t" \
  "s_waitcnt lgkmcnt(0)\n\t" \
  "v_mfma_f32_16x16x128_f8f6f4 a[124:127], v[152:159], v[216:223], a[124:127] blgp:1\n\t" \
  "v_mul_f32 v246, v235, v246\n\t" \
  "v_mfma_f32_16x16x128_f8f6f4 a[120:123], v[152:159], v[208:215], a[120:123] blgp:1\n\t" \
  "v_mul_f32 v247, v239, v247\n\t" \
  "v_mfma_f32_16x16x128_f8f6f4 a[116:119], v[152:159], v[200:207], a[116:119] blgp:1\n\t" \
  "v_mfma_f32_16x16x128_f8f6f4 a[112:115], v[152:159], v[192:199], a[112:115] blgp:1\n\t" \
  "s_waitcnt vmcnt(19) lgkmcnt(0)\n\t" \
  "s_barrier\n\t" \
  "v_mfma_f32_16x16x128_f8f6f4 v[64:67], v[128:135], v[160:167], v[64:67] blgp:1\n\t" \
  "ds_read_b128 v[192:195], v251 offset:49152\n\t" \
  "v_accvgpr_read_b32 v240, a192\n\t" \
  "v_mfma_f32_16x16x128_f8f6f4 v[68:71], v[128:135], v[168:175], v[68:71] blgp:1\n\t" \
  "ds_read_b128 v[196:199], v251 offset:57344\n\t" \
  "s_add_u32 s57, s80, 6488064\n\t" \
  "v_accvgpr_read_b32 v241, a196\n\t" \
  "v_mfma_f32_16x16x128_f8f6f4 v[72:75], v[128:135], v[176:183], v[72:75] blgp:1\n\t" \
  "buffer_store_dwordx4 v[244:247], v253, s[44:47], s57 offen offset:0 nt\n\t" \
  "ds_read_b128 v[200:203], v251 offset:49408\n\t" \
  "v_accvgpr_read_b32 v242, a200\n\t" \
  "v_mfma_f32_16x16x128_f8f6f4 v[76:79], v[128:135], v[184:191], v[76:79] blgp:1\n\t" \
  "ds_read_b128 v[128:131], v250 offset:32768\n\t" \
  "ds_read_b128 v[132:135], v250 offset:40960\n\t" \
  "v_accvgpr_read_b32 v243, a204\n\t" \
  "v_mfma_f32_16x16x128_f8f6f4 v[92:95], v[136:143], v[184:191], v[92:95] blgp:1\n\t" \
  "ds_read_b128 v[204:207], v251 offset:57600\n\t" \
  "v_mul_f32 v244, s58, v240\n\t" \
  "v_mfma_f32_16x16x128_f8f6f4 v[88:91], v[136:143], v[176:183], v[88:91] blgp:1\n\t" \
  "ds_read_b128 v[208:211], v251 offset:49664\n\t" \
  "v_mul_f32 v245, s58, v241\n\t" \
  "v_mfma_f32_16x16x128_f8f6f4 v[84:87], v[136:143], v[168:175], v[84:87] blgp:1\n\t" \
  "ds_read_b128 v[212:215], v251 offset:57856\n\t" \
  "v_mul_f32 v246, s58, v242\n\t" \
  "v_mfma_f32_16x16x128_f8f6f4 v[80:83], v[136:143], v[160:167], v[80:83] blgp:1\n\t" \
  "ds_read_b128 v[136:139], v250 offset:33024\n\t" \
  "ds_read_b128 v[140:143], v250 offset:41216\n\t" \
  "v_mul_f32 v247, s58, v243\n\t" \
  "v_mfma_f32_16x16x128_f8f6f4 v[96:99], v[144:151], v[160:167], v[96:99] blgp:1\n\t" \
  "ds_read_b128 v[216:219], v251 offset:49920\n\t" \
  "v_med3_f32 v244, v244, -s62, s62\n\t" \
  "v_mfma_f32_16x16x128_f8f6f4 v[100:103], v[144:151], v[168:175], v[100:103] blgp:1\n\t" \
  "ds_read_b128 v[220:223], v251 offset:58112\n\t" \
  "v_med3_f32 v245, v245, -s62, s62\n\t" \
  "v_mfma_f32_16x16x128_f8f6f4 v[104:107], v[144:151], v[176:183], v[104:107] blgp:1\n\t" \
  "v_med3_f32 v246, v246, -s62, s62\n\t" \
  "v_mfma_f32_16x16x128_f8f6f4 v[108:111], v[144:151], v[184:191], v[108:111] blgp:1\n\t" \
  "ds_read_b128 v[144:147], v250 offset:33280\n\t" \
  "v_med3_f32 v247, v247, -s62, s62\n\t" \
  "v_mfma_f32_16x16x128_f8f6f4 v[124:127], v[152:159], v[184:191], v[124:127] blgp:1\n\t" \
  "ds_read_b128 v[148:151], v250 offset:41472\n\t" \
  "v_exp_f32 v244, v244\n\t" \
  "v_mfma_f32_16x16x128_f8f6f4 v[120:123], v[152:159], v[176:183], v[120:123] blgp:1\n\t" \
  "v_exp_f32 v245, v245\n\t" \
  "v_mfma_f32_16x16x128_f8f6f4 v[116:119], v[152:159], v[168:175], v[116:119] blgp:1\n\t" \
  "v_exp_f32 v246, v246\n\t" \
  "v_mfma_f32_16x16x128_f8f6f4 v[112:115], v[152:159], v[160:167], v[112:115] blgp:1\n\t" \
  "ds_read_b128 v[152:155], v250 offset:33536\n\t" \
  "ds_read_b128 v[156:159], v250 offset:41728\n\t" \
  "s_waitcnt lgkmcnt(2)\n\t" \
  "v_mfma_f32_16x16x128_f8f6f4 v[0:3], v[128:135], v[192:199], v[0:3] blgp:1\n\t" \
  "ds_read_b128 v[160:163], v251 offset:32768\n\t" \
  "v_exp_f32 v247, v247\n\t" \
  "v_mfma_f32_16x16x128_f8f6f4 v[4:7], v[128:135], v[200:207], v[4:7] blgp:1\n\t" \
  "ds_read_b128 v[164:167], v251 offset:40960\n\t" \
  "v_mfma_f32_16x16x128_f8f6f4 v[8:11], v[128:135], v[208:215], v[8:11] blgp:1\n\t" \
  "ds_read_b128 v[168:171], v251 offset:33024\n\t" \
  "v_fma_f32 v244, v244, s60, s60\n\t" \
  "v_mfma_f32_16x16x128_f8f6f4 v[12:15], v[128:135], v[216:223], v[12:15] blgp:1\n\t" \
  "ds_read_b128 v[172:175], v251 offset:41216\n\t" \
  "v_fma_f32 v245, v245, s60, s60\n\t" \
  "v_mfma_f32_16x16x128_f8f6f4 v[28:31], v[136:143], v[216:223], v[28:31] blgp:1\n\t" \
  "ds_read_b128 v[176:179], v251 offset:33280\n\t" \
  "v_fma_f32 v246, v246, s60, s60\n\t" \
  "v_mfma_f32_16x16x128_f8f6f4 v[24:27], v[136:143], v[208:215], v[24:27] blgp:1\n\t" \
  "ds_read_b128 v[180:183], v251 offset:41472\n\t" \
  "v_fma_f32 v247, v247, s60, s60\n\t" \
  "v_mfma_f32_16x16x128_f8f6f4 v[20:23], v[136:143], v[200:207], v[20:23] blgp:1\n\t" \
  "ds_read_b128 v[184:187], v251 offset:33536\n\t" \
  "v_rcp_f32 v244, v244\n\t" \
  "v_mfma_f32_16x16x128_f8f6f4 v[16:19], v[136:143], v[192:199], v[16:19] blgp:1\n\t" \
  "ds_read_b128 v[188:191], v251 offset:41728\n\t" \
  "s_add_i32 s52, s48, 16384\n\t" \
  "s_add_u32 m0, s53, 49152\n\t" \
  "v_rcp_f32 v245, v245\n\t" \
  "v_mfma_f32_16x16x128_f8f6f4 v[32:35], v[144:151], v[192:199], v[32:35] blgp:1\n\t" \
  "buffer_load_dwordx4 v252, s[40:43], s52 offen offset:0 sc0 sc1 lds\n\t" \
  "v_rcp_f32 v246, v246\n\t" \
  "v_mfma_f32_16x16x128_f8f6f4 v[36:39], v[144:151], v[200:207], v[36:39] blgp:1\n\t" \
  "v_rcp_f32 v247, v247\n\t" \
  "v_mfma_f32_16x16x128_f8f6f4 v[40:43], v[144:151], v[208:215], v[40:43] blgp:1\n\t" \
  "buffer_load_dwordx4 v252, s[40:43], s52 offen offset:1024 sc0 sc1 lds\n\t" \
  "v_mul_f32 v244, v240, v244\n\t" \
  "v_mfma_f32_16x16x128_f8f6f4 v[44:47], v[144:151], v[216:223], v[44:47] blgp:1\n\t" \
  "v_mul_f32 v245, v241, v245\n\t" \
  "s_waitcnt lgkmcnt(0)\n\t" \
  "v_mfma_f32_16x16x128_f8f6f4 v[60:63], v[152:159], v[216:223], v[60:63] blgp:1\n\t" \
  "buffer_load_dwordx4 v252, s[40:43], s52 offen offset:2048 sc0 sc1 lds\n\t" \
  "v_mul_f32 v246, v242, v246\n\t" \
  "v_mfma_f32_16x16x128_f8f6f4 v[56:59], v[152:159], v[208:215], v[56:59] blgp:1\n\t" \
  "v_mul_f32 v247, v243, v247\n\t" \
  "v_mfma_f32_16x16x128_f8f6f4 v[52:55], v[152:159], v[200:207], v[52:55] blgp:1\n\t" \
  "buffer_load_dwordx4 v252, s[40:43], s52 offen offset:3072 sc0 sc1 lds\n\t" \
  "v_mfma_f32_16x16x128_f8f6f4 v[48:51], v[152:159], v[192:199], v[48:51] blgp:1\n\t" \
  "s_waitcnt vmcnt(18) lgkmcnt(0)\n\t" \
  "s_barrier\n\t" \
  "v_mfma_f32_16x16x128_f8f6f4 a[0:3], v[128:135], v[160:167], a[0:3] blgp:1\n\t" \
  "v_readfirstlane_b32 s87, v254\n\t" \
  "v_readlane_b32 s65, v255, s67\n\t" \
  "v_accvgpr_read_b32 v240, a193\n\t" \
  "v_mfma_f32_16x16x128_f8f6f4 a[4:7], v[128:135], v[168:175], a[4:7] blgp:1\n\t" \
  "s_add_u32 s57, s80, 4194304\n\t" \
  "v_accvgpr_read_b32 v241, a197\n\t" \
  "v_mfma_f32_16x16x128_f8f6f4 a[8:11], v[128:135], v[176:183], a[8:11] blgp:1\n\t" \
  "buffer_store_dwordx4 v[244:247], v253, s[44:47], s57 offen offset:256 nt\n\t" \
  "s_max_u32 s69, s69, s87\n\t" \
  "s_add_u32 s67, s67, 1\n\t" \
  "s_cmp_ge_u32 s69, s65\n\t" \
  "s_cbranch_scc0 L_w16_%=\n\t" \
  "L_ok16_%=:\n\t" \
  "v_accvgpr_read_b32 v242, a201\n\t" \
  "v_mfma_f32_16x16x128_f8f6f4 a[12:15], v[128:135], v[184:191], a[12:15] blgp:1\n\t" \
  "ds_read_b128 v[128:131], v250 offset:49152\n\t" \
  "v_accvgpr_read_b32 v243, a205\n\t" \
  "v_mfma_f32_16x16x128_f8f6f4 a[28:31], v[136:143], v[184:191], a[28:31] blgp:1\n\t" \
  "s_add_i32 s52, s48, 32768\n\t" \
  "s_add_u32 m0, s53, 65536\n\t" \
  "ds_read_b128 v[132:135], v250 offset:57344\n\t" \
  "v_mul_f32 v244, s58, v240\n\t" \
  "v_mfma_f32_16x16x128_f8f6f4 a[24:27], v[136:143], v[176:183], a[24:27] blgp:1\n\t" \
  "buffer_load_dwordx4 v252, s[40:43], s52 offen offset:0 sc0 sc1 lds\n\t" \
  "v_mul_f32 v245, s58, v241\n\t" \
  "v_mfma_f32_16x16x128_f8f6f4 a[20:23], v[136:143], v[168:175], a[20:23] blgp:1\n\t" \
  "buffer_load_dwordx4 v252, s[40:43], s52 offen offset:1024 sc0 sc1 lds\n\t" \
  "v_mul_f32 v246, s58, v242\n\t" \
  "v_mfma_f32_16x16x128_f8f6f4 a[16:19], v[136:143], v[160:167], a[16:19] blgp:1\n\t" \
  "ds_read_b128 v[136:139], v250 offset:49408\n\t" \
  "v_mul_f32 v247, s58, v243\n\t" \
  "v_mfma_f32_16x16x128_f8f6f4 a[32:35], v[144:151], v[160:167], a[32:35] blgp:1\n\t" \
  "ds_read_b128 v[140:143], v250 offset:57600\n\t" \
  "v_med3_f32 v244, v244, -s62, s62\n\t" \
  "v_mfma_f32_16x16x128_f8f6f4 a[36:39], v[144:151], v[168:175], a[36:39] blgp:1\n\t" \
  "buffer_load_dwordx4 v252, s[40:43], s52 offen offset:2048 sc0 sc1 lds\n\t" \
  "v_med3_f32 v245, v245, -s62, s62\n\t" \
  "v_mfma_f32_16x16x128_f8f6f4 a[40:43], v[144:151], v[176:183], a[40:43] blgp:1\n\t" \
  "buffer_load_dwordx4 v252, s[40:43], s52 offen offset:3072 sc0 sc1 lds\n\t" \
  "v_med3_f32 v246, v246, -s62, s62\n\t" \
  "v_mfma_f32_16x16x128_f8f6f4 a[44:47], v[144:151], v[184:191], a[44:47] blgp:1\n\t" \
  "ds_read_b128 v[144:147], v250 offset:49664\n\t" \
  "v_med3_f32 v247, v247, -s62, s62\n\t" \
  "v_mfma_f32_16x16x128_f8f6f4 a[60:63], v[152:159], v[184:191], a[60:63] blgp:1\n\t" \
  "ds_read_b128 v[148:151], v250 offset:57856\n\t" \
  "v_exp_f32 v244, v244\n\t" \
  "v_mfma_f32_16x16x128_f8f6f4 a[56:59], v[152:159], v[176:183], a[56:59] blgp:1\n\t" \
  "v_exp_f32 v245, v245\n\t" \
  "v_mfma_f32_16x16x128_f8f6f4 a[52:55], v[152:159], v[168:175], a[52:55] blgp:1\n\t" \
  "v_exp_f32 v246, v246\n\t" \
  "v_mfma_f32_16x16x128_f8f6f4 a[48:51], v[152:159], v[160:167], a[48:51] blgp:1\n\t" \
  "ds_read_b128 v[152:155], v250 offset:49920\n\t" \
  "ds_read_b128 v[156:159], v250 offset:58112\n\t" \
  "s_waitcnt lgkmcnt(2)\n\t" \
  "v_mfma_f32_16x16x128_f8f6f4 v[64:67], v[128:135], v[160:167], v[64:67] blgp:1\n\t" \
  "s_add_i32 s52, s49, 49152\n\t" \
  "s_add_u32 m0, s53, 147456\n\t" \
  "v_exp_f32 v247, v247\n\t" \
  "v_mfma_f32_16x16x128_f8f6f4 v[68:71], v[128:135], v[168:175], v[68:71] blgp:1\n\t" \
  "buffer_load_dwordx4 v252, s[40:43], s52 offen offset:0 sc0 sc1 lds\n\t" \
  "v_mfma_f32_16x16x128_f8f6f4 v[72:75], v[128:135], v[176:183], v[72:75] blgp:1\n\t" \
  "buffer_load_dwordx4 v252, s[40:43], s52 offen offset:1024 sc0 sc1 lds\n\t" \
  "v_fma_f32 v244, v244, s60, s60\n\t" \
  "v_mfma_f32_16x16x128_f8f6f4 v[76:79], v[128:135], v[184:191], v[76:79] blgp:1\n\t" \
  "v_fma_f32 v245, v245, s60, s60\n\t" \
  "v_mfma_f32_16x16x128_f8f6f4 v[92:95], v[136:143], v[184:191], v[92:95] blgp:1\n\t" \
  "buffer_load_dwordx4 v252, s[40:43], s52 offen offset:2048 sc0 sc1 lds\n\t" \
  "v_fma_f32 v246, v246, s60, s60\n\t" \
  "v_mfma_f32_16x16x128_f8f6f4 v[88:91], v[136:143], v[176:183], v[88:91] blgp:1\n\t" \
  "buffer_load_dwordx4 v252, s[40:43], s52 offen offset:3072 sc0 sc1 lds\n\t" \
  "v_fma_f32 v247, v247, s60, s60\n\t" \
  "v_mfma_f32_16x16x128_f8f6f4 v[84:87], v[136:143], v[168:175], v[84:87] blgp:1\n\t" \
  "s_add_i32 s52, s49, 32768\n\t" \
  "s_add_u32 m0, s53, 131072\n\t" \
  "v_rcp_f32 v244, v244\n\t" \
  "v_mfma_f32_16x16x128_f8f6f4 v[80:83], v[136:143], v[160:167], v[80:83] blgp:1\n\t" \
  "buffer_load_dwordx4 v252, s[40:43], s52 offen offset:0 sc0 sc1 lds\n\t" \
  "v_rcp_f32 v245, v245\n\t" \
  "v_mfma_f32_16x16x128_f8f6f4 v[96:99], v[144:151], v[160:167], v[96:99] blgp:1\n\t" \
  "buffer_load_dwordx4 v252, s[40:43], s52 offen offset:1024 sc0 sc1 lds\n\t" \
  "v_rcp_f32 v246, v246\n\t" \
  "v_mfma_f32_16x16x128_f8f6f4 v[100:103], v[144:151], v[168:175], v[100:103] blgp:1\n\t" \
  "v_rcp_f32 v247, v247\n\t" \
  "v_mfma_f32_16x16x128_f8f6f4 v[104:107], v[144:151], v[176:183], v[104:107] blgp:1\n\t" \
  "buffer_load_dwordx4 v252, s[40:43], s52 offen offset:2048 sc0 sc1 lds\n\t" \
  "v_mul_f32 v244, v240, v244\n\t" \
  "v_mfma_f32_16x16x128_f8f6f4 v[108:111], v[144:151], v[184:191], v[108:111] blgp:1\n\t" \
  "buffer_load_dwordx4 v252, s[40:43], s52 offen offset:3072 sc0 sc1 lds\n\t" \
  "v_mul_f32 v245, v241, v245\n\t" \
  "s_waitcnt lgkmcnt(0)\n\t" \
  "v_mfma_f32_16x16x128_f8f6f4 v[124:127], v[152:159], v[184:191], v[124:127] blgp:1\n\t" \
  "v_mul_f32 v246, v242, v246\n\t" \
  "v_mfma_f32_16x16x128_f8f6f4 v[120:123], v[152:159], v[176:183], v[120:123] blgp:1\n\t" \
  "v_mul_f32 v247, v243, v247\n\t" \
  "v_mfma_f32_16x16x128_f8f6f4 v[116:119], v[152:159], v[168:175], v[116:119] blgp:1\n\t" \
  "v_mfma_f32_16x16x128_f8f6f4 v[112:115], v[152:159], v[160:167], v[112:115] blgp:1\n\t" \
  "s_waitcnt vmcnt(18) lgkmcnt(0)\n\t" \
  "s_barrier\n\t" \
  "v_mfma_f32_16x16x128_f8f6f4 a[64:67], v[128:135], v[192:199], a[64:67] blgp:1\n\t" \
  "ds_read_b128 v[160:163], v251 offset:0\n\t" \
  "v_accvgpr_read_b32 v240, a194\n\t" \
  "v_mfma_f32_16x16x128_f8f6f4 a[68:71], v[128:135], v[200:207], a[68:71] blgp:1\n\t" \
  "ds_read_b128 v[164:167], v251 offset:8192\n\t" \
  "s_add_u32 s57, s80, 4259840\n\t" \
  "v_accvgpr_read_b32 v241, a198\n\t" \
  "v_mfma_f32_16x16x128_f8f6f4 a[72:75], v[128:135], v[208:215], a[72:75] blgp:1\n\t" \
  "buffer_store_dwordx4 v[244:247], v253, s[44:47], s57 offen offset:256 nt\n\t" \
  "ds_read_b128 v[168:171], v251 offset:256\n\t" \
  "v_accvgpr_read_b32 v242, a202\n\t" \
  "v_mfma_f32_16x16x128_f8f6f4 a[76:79], v[128:135], v[216:223], a[76:79] blgp:1\n\t" \
  "ds_read_b128 v[128:131], v250 offset:0\n\t" \
  "ds_read_b128 v[132:135], v250 offset:8192\n\t" \
  "v_accvgpr_read_b32 v243, a206\n\t" \
  "v_mfma_f32_16x16x128_f8f6f4 a[92:95], v[136:143], v[216:223], a[92:95] blgp:1\n\t" \
  "ds_read_b128 v[172:175], v251 offset:8448\n\t" \
  "v_mul_f32 v244, s58, v240\n\t" \
  "v_mfma_f32_16x16x128_f8f6f4 a[88:91], v[136:143], v[208:215], a[88:91] blgp:1\n\t" \
  "ds_read_b128 v[176:179], v251 offset:512\n\t" \
  "v_mul_f32 v245, s58, v241\n\t" \
  "v_mfma_f32_16x16x128_f8f6f4 a[84:87], v[136:143], v[200:207], a[84:87] blgp:1\n\t" \
  "ds_read_b128 v[180:183], v251 offset:8704\n\t" \
  "v_mul_f32 v246, s58, v242\n\t" \
  "v_mfma_f32_16x16x128_f8f6f4 a[80:83], v[136:143], v[192:199], a[80:83] blgp:1\n\t" \
  "ds_read_b128 v[136:139], v250 offset:256\n\t" \
  "ds_read_b128 v[140:143], v250 offset:8448\n\t" \
  "v_mul_f32 v247, s58, v243\n\t" \
  "v_mfma_f32_16x16x128_f8f6f4 a[96:99], v[144:151], v[192:199], a[96:99] blgp:1\n\t" \
  "ds_read_b128 v[184:187], v251 offset:768\n\t" \
  "v_med3_f32 v244, v244, -s62, s62\n\t" \
  "v_mfma_f32_16x16x128_f8f6f4 a[100:103], v[144:151], v[200:207], a[100:103] blgp:1\n\t" \
  "ds_read_b128 v[188:191], v251 offset:8960\n\t" \
  "v_med3_f32 v245, v245, -s62, s62\n\t" \
  "v_mfma_f32_16x16x128_f8f6f4 a[104:107], v[144:151], v[208:215], a[104:107] blgp:1\n\t" \
  "s_add_u32 s48, s48, 0x10000\n\t" \
  "s_add_u32 s49, s49, 0x10000\n\t" \
  "v_med3_f32 v246, v246, -s62, s62\n\t" \
  "v_mfma_f32_16x16x128_f8f6f4 a[108:111], v[144:151], v[216:223], a[108:111] blgp:1\n\t" \
  "ds_read_b128 v[144:147], v250 offset:512\n\t" \
  "v_med3_f32 v247, v247, -s62, s62\n\t" \
  "v_mfma_f32_16x16x128_f8f6f4 a[124:127], v[152:159], v[216:223], a[124:127] blgp:1\n\t" \
  "ds_read_b128 v[148:151], v250 offset:8704\n\t" \
  "v_exp_f32 v244, v244\n\t" \
  "v_mfma_f32_16x16x128_f8f6f4 a[120:123], v[152:159], v[208:215], a[120:123] blgp:1\n\t" \
  "v_exp_f32 v245, v245\n\t" \
  "v_mfma_f32_16x16x128_f8f6f4 a[116:119], v[152:159], v[200:207], a[116:119] blgp:1\n\t" \
  "v_exp_f32 v246, v246\n\t" \
  "v_mfma_f32_16x16x128_f8f6f4 a[112:115], v[152:159], v[192:199], a[112:115] blgp:1\n\t" \
  "ds_read_b128 v[152:155], v250 offset:768\n\t" \
  "ds_read_b128 v[156:159], v250 offset:8960\n\t" \
  "s_waitcnt lgkmcnt(2)\n\t" \
  "v_mfma_f32_16x16x128_f8f6f4 a[0:3], v[128:135], v[160:167], a[0:3] blgp:1\n\t" \
  "ds_read_b128 v[192:195], v251 offset:16384\n\t" \
  "v_exp_f32 v247, v247\n\t" \
  "v_mfma_f32_16x16x128_f8f6f4 a[4:7], v[128:135], v[168:175], a[4:7] blgp:1\n\t" \
  "ds_read_b128 v[196:199], v251 offset:24576\n\t" \
  "v_mfma_f32_16x16x128_f8f6f4 a[8:11], v[128:135], v[176:183], a[8:11] blgp:1\n\t" \
  "ds_read_b128 v[200:203], v251 offset:16640\n\t" \
  "v_fma_f32 v244, v244, s60, s60\n\t" \
  "v_mfma_f32_16x16x128_f8f6f4 a[12:15], v[128:135], v[184:191], a[12:15] blgp:1\n\t" \
  "ds_read_b128 v[204:207], v251 offset:24832\n\t" \
  "v_fma_f32 v245, v245, s60, s60\n\t" \
  "v_mfma_f32_16x16x128_f8f6f4 a[28:31], v[136:143], v[184:191], a[28:31] blgp:1\n\t" \
  "ds_read_b128 v[208:211], v251 offset:16896\n\t" \
  "v_fma_f32 v246, v246, s60, s60\n\t" \
  "v_mfma_f32_16x16x128_f8f6f4 a[24:27], v[136:143], v[176:183], a[24:27] blgp:1\n\t" \
  "ds_read_b128 v[212:215], v251 offset:25088\n\t" \
  "v_fma_f32 v247, v247, s60, s60\n\t" \
  "v_mfma_f32_16x16x128_f8f6f4 a[20:23], v[136:143], v[168:175], a[20:23] blgp:1\n\t" \
  "ds_read_b128 v[216:219], v251 offset:17152\n\t" \
  "v_rcp_f32 v244, v244\n\t" \
  "v_mfma_f32_16x16x128_f8f6f4 a[16:19], v[136:143], v[160:167], a[16:19] blgp:1\n\t" \
  "ds_read_b128 v[220:223], v251 offset:25344\n\t" \
  "s_add_i32 s52, s48, -16384\n\t" \
  "s_add_u32 m0, s53, 81920\n\t" \
  "v_rcp_f32 v245, v245\n\t" \
  "v_mfma_f32_16x16x128_f8f6f4 a[32:35], v[144:151], v[160:167], a[32:35] blgp:1\n\t" \
  "buffer_load_dwordx4 v252, s[40:43], s52 offen offset:0 sc0 sc1 lds\n\t" \
  "v_rcp_f32 v246, v246\n\t" \
  "v_mfma_f32_16x16x128_f8f6f4 a[36:39], v[144:151], v[168:175], a[36:39] blgp:1\n\t" \
  "v_rcp_f32 v247, v247\n\t" \
  "v_mfma_f32_16x16x128_f8f6f4 a[40:43], v[144:151], v[176:183], a[40:43] blgp:1\n\t" \
  "buffer_load_dwordx4 v252, s[40:43], s52 offen offset:1024 sc0 sc1 lds\n\t" \
  "v_mul_f32 v244, v240, v244\n\t" \
  "v_mfma_f32_16x16x128_f8f6f4 a[44:47], v[144:151], v[184:191], a[44:47] blgp:1\n\t" \
  "v_mul_f32 v245, v241, v245\n\t" \
  "s_waitcnt lgkmcnt(0)\n\t" \
  "v_mfma_f32_16x16x128_f8f6f4 a[60:63], v[152:159], v[184:191], a[60:63] blgp:1\n\t" \
  "buffer_load_dwordx4 v252, s[40:43], s52 offen offset:2048 sc0 sc1 lds\n\t" \
  "v_mul_f32 v246, v242, v246\n\t" \
  "v_mfma_f32_16x16x128_f8f6f4 a[56:59], v[152:159], v[176:183], a[56:59] blgp:1\n\t" \
  "v_mul_f32 v247, v243, v247\n\t" \
  "v_mfma_f32_16x16x128_f8f6f4 a[52:55], v[152:159], v[168:175], a[52:55] blgp:1\n\t" \
  "buffer_load_dwordx4 v252, s[40:43], s52 offen offset:3072 sc0 sc1 lds\n\t" \
  "buffer_load_dword v254, off, s[88:91], s81 sc1\n\t" \
  "v_mfma_f32_16x16x128_f8f6f4 a[48:51], v[152:159], v[160:167], a[48:51] blgp:1\n\t" \
  "s_waitcnt vmcnt(19) lgkmcnt(0)\n\t" \
  "s_barrier\n\t" \
  "v_mfma_f32_16x16x128_f8f6f4 v[0:3], v[128:135], v[192:199], v[0:3] blgp:1\n\t" \
  "v_readlane_b32 s65, v255, s67\n\t" \
  "v_accvgpr_read_b32 v240, a195\n\t" \
  "v_mfma_f32_16x16x128_f8f6f4 v[4:7], v[128:135], v[200:207], v[4:7] blgp:1\n\t" \
  "s_add_u32 s57, s80, 4325376\n\t" \
  "v_accvgpr_read_b32 v241, a199\n\t" \
  "v_mfma_f32_16x16x128_f8f6f4 v[8:11], v[128:135], v[208:215], v[8:11] blgp:1\n\t" \
  "buffer_store_dwordx4 v[244:247], v253, s[44:47], s57 offen offset:256 nt\n\t" \
  "s_add_u32 s67, s67, 1\n\t" \
  "s_cmp_ge_u32 s69, s65\n\t" \
  "s_cbranch_scc0 L_w17_%=\n\t" \
  "L_ok17_%=:\n\t" \
  "v_accvgpr_read_b32 v242, a203\n\t" \
  "v_mfma_f32_16x16x128_f8f6f4 v[12:15], v[128:135], v[216:223], v[12:15] blgp:1\n\t" \
  "ds_read_b128 v[128:131], v250 offset:16384\n\t" \
  "v_accvgpr_read_b32 v243, a207\n\t" \
  "v_mfma_f32_16x16x128_f8f6f4 v[28:31], v[136:143], v[216:223], v[28:31] blgp:1\n\t" \
  "s_add_i32 s52, s48, 0\n\t" \
  "s_add_u32 m0, s53, 32768\n\t" \
  "ds_read_b128 v[132:135], v250 offset:24576\n\t" \
  "v_mul_f32 v244, s58, v240\n\t" \
  "v_mfma_f32_16x16x128_f8f6f4 v[24:27], v[136:143], v[208:215], v[24:27] blgp:1\n\t" \
  "buffer_load_dwordx4 v252, s[40:43], s52 offen offset:0 sc0 sc1 lds\n\t" \
  "v_mul_f32 v245, s58, v241\n\t" \
  "v_mfma_f32_16x16x128_f8f6f4 v[20:23], v[136:143], v[200:207], v[20:23] blgp:1\n\t" \
  "buffer_load_dwordx4 v252, s[40:43], s52 offen offset:1024 sc0 sc1 lds\n\t" \
  "v_mul_f32 v246, s58, v242\n\t" \
  "v_mfma_f32_16x16x128_f8f6f4 v[16:19], v[136:143], v[192:199], v[16:19] blgp:1\n\t" \
  "ds_read_b128 v[136:139], v250 offset:16640\n\t" \
  "v_mul_f32 v247, s58, v243\n\t" \
  "v_mfma_f32_16x16x128_f8f6f4 v[32:35], v[144:151], v[192:199], v[32:35] blgp:1\n\t" \
  "ds_read_b128 v[140:143], v250 offset:24832\n\t" \
  "v_med3_f32 v244, v244, -s62, s62\n\t" \
  "v_mfma_f32_16x16x128_f8f6f4 v[36:39], v[144:151], v[200:207], v[36:39] blgp:1\n\t" \
  "buffer_load_dwordx4 v252, s[40:43], s52 offen offset:2048 sc0 sc1 lds\n\t" \
  "v_med3_f32 v245, v245, -s62, s62\n\t" \
  "v_mfma_f32_16x16x128_f8f6f4 v[40:43], v[144:151], v[208:215], v[40:43] blgp:1\n\t" \
  "buffer_load_dwordx4 v252, s[40:43], s52 offen offset:3072 sc0 sc1 lds\n\t" \
  "v_med3_f32 v246, v246, -s62, s62\n\t" \
  "v_mfma_f32_16x16x128_f8f6f4 v[44:47], v[144:151], v[216:223], v[44:47] blgp:1\n\t" \
  "ds_read_b128 v[144:147], v250 offset:16896\n\t" \
  "v_med3_f32 v247, v247, -s62, s62\n\t" \
  "v_mfma_f32_16x16x128_f8f6f4 v[60:63], v[152:159], v[216:223], v[60:63] blgp:1\n\t" \
  "ds_read_b128 v[148:151], v250 offset:25088\n\t" \
  "v_exp_f32 v244, v244\n\t" \
  "v_mfma_f32_16x16x128_f8f6f4 v[56:59], v[152:159], v[208:215], v[56:59] blgp:1\n\t" \
  "v_exp_f32 v245, v245\n\t" \
  "v_mfma_f32_16x16x128_f8f6f4 v[52:55], v[152:159], v[200:207], v[52:55] blgp:1\n\t" \
  "v_exp_f32 v246, v246\n\t" \
  "v_mfma_f32_16x16x128_f8f6f4 v[48:51], v[152:159], v[192:199], v[48:51] blgp:1\n\t" \
  "ds_read_b128 v[152:155], v250 offset:17152\n\t" \
  "ds_read_b128 v[156:159], v250 offset:25344\n\t" \
  "s_waitcnt lgkmcnt(2)\n\t" \
  "v_mfma_f32_16x16x128_f8f6f4 a[64:67], v[128:135], v[192:199], a[64:67] blgp:1\n\t" \
  "s_add_i32 s52, s49, 0\n\t" \
  "s_add_u32 m0, s53, 98304\n\t" \
  "v_exp_f32 v247, v247\n\t" \
  "v_mfma_f32_16x16x128_f8f6f4 a[68:71], v[128:135], v[200:207], a[68:71] blgp:1\n\t" \
  "buffer_load_dwordx4 v252, s[40:43], s52 offen offset:0 sc0 sc1 lds\n\t" \
  "v_mfma_f32_16x16x128_f8f6f4 a[72:75], v[128:135], v[208:215], a[72:75] blgp:1\n\t" \
  "buffer_load_dwordx4 v252, s[40:43], s52 offen offset:1024 sc0 sc1 lds\n\t" \
  "v_fma_f32 v244, v244, s60, s60\n\t" \
  "v_mfma_f32_16x16x128_f8f6f4 a[76:79], v[128:135], v[216:223], a[76:79] blgp:1\n\t" \
  "v_fma_f32 v245, v245, s60, s60\n\t" \
  "v_mfma_f32_16x16x128_f8f6f4 a[92:95], v[136:143], v[216:223], a[92:95] blgp:1\n\t" \
  "buffer_load_dwordx4 v252, s[40:43], s52 offen offset:2048 sc0 sc1 lds\n\t" \
  "v_fma_f32 v246, v246, s60, s60\n\t" \
  "v_mfma_f32_16x16x128_f8f6f4 a[88:91], v[136:143], v[208:215], a[88:91] blgp:1\n\t" \
  "buffer_load_dwordx4 v252, s[40:43], s52 offen offset:3072 sc0 sc1 lds\n\t" \
  "v_fma_f32 v247, v247, s60, s60\n\t" \
  "v_mfma_f32_16x16x128_f8f6f4 a[84:87], v[136:143], v[200:207], a[84:87] blgp:1\n\t" \
  "s_add_i32 s52, s49, 16384\n\t" \
  "s_add_u32 m0, s53, 114688\n\t" \
  "v_rcp_f32 v244, v244\n\t" \
  "v_mfma_f32_16x16x128_f8f6f4 a[80:83], v[136:143], v[192:199], a[80:83] blgp:1\n\t" \
  "buffer_load_dwordx4 v252, s[40:43], s52 offen offset:0 sc0 sc1 lds\n\t" \
  "v_rcp_f32 v245, v245\n\t" \
  "v_mfma_f32_16x16x128_f8f6f4 a[96:99], v[144:151], v[192:199], a[96:99] blgp:1\n\t" \
  "buffer_load_dwordx4 v252, s[40:43], s52 offen offset:1024 sc0 sc1 lds\n\t" \
  "v_rcp_f32 v246, v246\n\t" \
  "v_mfma_f32_16x16x128_f8f6f4 a[100:103], v[144:151], v[200:207], a[100:103] blgp:1\n\t" \
  "v_rcp_f32 v247, v247\n\t" \
  "v_mfma_f32_16x16x128_f8f6f4 a[104:107], v[144:151], v[208:215], a[104:107] blgp:1\n\t" \
  "buffer_load_dwordx4 v252, s[40:43], s52 offen offset:2048 sc0 sc1 lds\n\t" \
  "v_mul_f32 v244, v240, v244\n\t" \
  "v_mfma_f32_16x16x128_f8f6f4 a[108:111], v[144:151], v[216:223], a[108:111] blgp:1\n\t" \
  "buffer_load_dwordx4 v252, s[40:43], s52 offen offset:3072 sc0 sc1 lds\n\t" \
  "v_mul_f32 v245, v241, v245\n\t" \
  "s_waitcnt lgkmcnt(0)\n\t" \
  "v_mfma_f32_16x16x128_f8f6f4 a[124:127], v[152:159], v[216:223], a[124:127] blgp:1\n\t" \
  "v_mul_f32 v246, v242, v246\n\t" \
  "v_mfma_f32_16x16x128_f8f6f4 a[120:123], v[152:159], v[208:215], a[120:123] blgp:1\n\t" \
  "v_mul_f32 v247, v243, v247\n\t" \
  "v_mfma_f32_16x16x128_f8f6f4 a[116:119], v[152:159], v[200:207], a[116:119] blgp:1\n\t" \
  "v_mfma_f32_16x16x128_f8f6f4 a[112:115], v[152:159], v[192:199], a[112:115] blgp:1\n\t" \
  "s_waitcnt vmcnt(19) lgkmcnt(0)\n\t" \
  "s_barrier\n\t" \
  "v_mfma_f32_16x16x128_f8f6f4 v[64:67], v[128:135], v[160:167], v[64:67] blgp:1\n\t" \
  "ds_read_b128 v[192:195], v251 offset:49152\n\t" \
  "v_accvgpr_read_b32 v240, a208\n\t" \
  "v_mfma_f32_16x16x128_f8f6f4 v[68:71], v[128:135], v[168:175], v[68:71] blgp:1\n\t" \
  "ds_read_b128 v[196:199], v251 offset:57344\n\t" \
  "s_add_u32 s57, s80, 4390912\n\t" \
  "v_accvgpr_read_b32 v241, a212\n\t" \
  "v_mfma_f32_16x16x128_f8f6f4 v[72:75], v[128:135], v[176:183], v[72:75] blgp:1\n\t" \
  "buffer_store_dwordx4 v[244:247], v253, s[44:47], s57 offen offset:256 nt\n\t" \
  "ds_read_b128 v[200:203], v251 offset:49408\n\t" \
  "v_accvgpr_read_b32 v242, a216\n\t" \
  "v_mfma_f32_16x16x128_f8f6f4 v[76:79], v[128:135], v[184:191], v[76:79] blgp:1\n\t" \
  "ds_read_b128 v[128:131], v250 offset:32768\n\t" \
  "ds_read_b128 v[132:135], v250 offset:40960\n\t" \
  "v_accvgpr_read_b32 v243, a220\n\t" \
  "v_mfma_f32_16x16x128_f8f6f4 v[92:95], v[136:143], v[184:191], v[92:95] blgp:1\n\t" \
  "ds_read_b128 v[204:207], v251 offset:57600\n\t" \
  "v_mul_f32 v244, s58, v240\n\t" \
  "v_mfma_f32_16x16x128_f8f6f4 v[88:91], v[136:143], v[176:183], v[88:91] blgp:1\n\t" \
  "ds_read_b128 v[208:211], v251 offset:49664\n\t" \
  "v_mul_f32 v245, s58, v241\n\t" \
  "v_mfma_f32_16x16x128_f8f6f4 v[84:87], v[136:143], v[168:175], v[84:87] blgp:1\n\t" \
  "ds_read_b128 v[212:215], v251 offset:57856\n\t" \
  "v_mul_f32 v246, s58, v242\n\t" \
  "v_mfma_f32_16x16x128_f8f6f4 v[80:83], v[136:143], v[160:167], v[80:83] blgp:1\n\t" \
  "ds_read_b128 v[136:139], v250 offset:33024\n\t" \
  "ds_read_b128 v[140:143], v250 offset:41216\n\t" \
  "v_mul_f32 v247, s58, v243\n\t" \
  "v_mfma_f32_16x16x128_f8f6f4 v[96:99], v[144:151], v[160:167], v[96:99] blgp:1\n\t" \
  "ds_read_b128 v[216:219], v251 offset:49920\n\t" \
  "v_med3_f32 v244, v244, -s62, s62\n\t" \
  "v_mfma_f32_16x16x128_f8f6f4 v[100:103], v[144:151], v[168:175], v[100:103] blgp:1\n\t" \
  "ds_read_b128 v[220:223], v251 offset:58112\n\t" \
  "v_med3_f32 v245, v245, -s62, s62\n\t" \
  "v_mfma_f32_16x16x128_f8f6f4 v[104:107], v[144:151], v[176:183], v[104:107] blgp:1\n\t" \
  "v_med3_f32 v246, v246, -s62, s62\n\t" \
  "v_mfma_f32_16x16x128_f8f6f4 v[108:111], v[144:151], v[184:191], v[108:111] blgp:1\n\t" \
  "ds_read_b128 v[144:147], v250 offset:33280\n\t" \
  "v_med3_f32 v247, v247, -s62, s62\n\t" \
  "v_mfma_f32_16x16x128_f8f6f4 v[124:127], v[152:159], v[184:191], v[124:127] blgp:1\n\t" \
  "ds_read_b128 v[148:151], v250 offset:41472\n\t" \
  "v_exp_f32 v244, v244\n\t" \
  "v_mfma_f32_16x16x128_f8f6f4 v[120:123], v[152:159], v[176:183], v[120:123] blgp:1\n\t" \
  "v_exp_f32 v245, v245\n\t" \
  "v_mfma_f32_16x16x128_f8f6f4 v[116:119], v[152:159], v[168:175], v[116:119] blgp:1\n\t" \
  "v_exp_f32 v246, v246\n\t" \
  "v_mfma_f32_16x16x128_f8f6f4 v[112:115], v[152:159], v[160:167], v[112:115] blgp:1\n\t" \
  "ds_read_b128 v[152:155], v250 offset:33536\n\t" \
  "ds_read_b128 v[156:159], v250 offset:41728\n\t" \
  "s_waitcnt lgkmcnt(2)\n\t" \
  "v_mfma_f32_16x16x128_f8f6f4 v[0:3], v[128:135], v[192:199], v[0:3] blgp:1\n\t" \
  "ds_read_b128 v[160:163], v251 offset:32768\n\t" \
  "v_exp_f32 v247, v247\n\t" \
  "v_mfma_f32_16x16x128_f8f6f4 v[4:7], v[128:135], v[200:207], v[4:7] blgp:1\n\t" \
  "ds_read_b128 v[164:167], v251 offset:40960\n\t" \
  "v_mfma_f32_16x16x128_f8f6f4 v[8:11], v[128:135], v[208:215], v[8:11] blgp:1\n\t" \
  "ds_read_b128 v[168:171], v251 offset:33024\n\t" \
  "v_fma_f32 v244, v244, s60, s60\n\t" \
  "v_mfma_f32_16x16x128_f8f6f4 v[12:15], v[128:135], v[216:223], v[12:15] blgp:1\n\t" \
  "ds_read_b128 v[172:175], v251 offset:41216\n\t" \
  "v_fma_f32 v245, v245, s60, s60\n\t" \
  "v_mfma_f32_16x16x128_f8f6f4 v[28:31], v[136:143], v[216:223], v[28:31] blgp:1\n\t" \
  "ds_read_b128 v[176:179], v251 offset:33280\n\t" \
  "v_fma_f32 v246, v246, s60, s60\n\t" \
  "v_mfma_f32_16x16x128_f8f6f4 v[24:27], v[136:143], v[208:215], v[24:27] blgp:1\n\t" \
  "ds_read_b128 v[180:183], v251 offset:41472\n\t" \
  "v_fma_f32 v247, v247, s60, s60\n\t" \
  "v_mfma_f32_16x16x128_f8f6f4 v[20:23], v[136:143], v[200:207], v[20:23] blgp:1\n\t" \
  "ds_read_b128 v[184:187], v251 offset:33536\n\t" \
  "v_rcp_f32 v244, v244\n\t" \
  "v_mfma_f32_16x16x128_f8f6f4 v[16:19], v[136:143], v[192:199], v[16:19] blgp:1\n\t" \
  "ds_read_b128 v[188:191], v251 offset:41728\n\t" \
  "s_add_i32 s52, s48, 16384\n\t" \
  "s_add_u32 m0, s53, 49152\n\t" \
  "v_rcp_f32 v245, v245\n\t" \
  "v_mfma_f32_16x16x128_f8f6f4 v[32:35], v[144:151], v[192:199], v[32:35] blgp:1\n\t" \
  "buffer_load_dwordx4 v252, s[40:43], s52 offen offset:0 sc0 sc1 lds\n\t" \
  "v_rcp_f32 v246, v246\n\t" \
  "v_mfma_f32_16x16x128_f8f6f4 v[36:39], v[144:151], v[200:207], v[36:39] blgp:1\n\t" \
  "v_rcp_f32 v247, v247\n\t" \
  "v_mfma_f32_16x16x128_f8f6f4 v[40:43], v[144:151], v[208:215], v[40:43] blgp:1\n\t" \
  "buffer_load_dwordx4 v252, s[40:43], s52 offen offset:1024 sc0 sc1 lds\n\t" \
  "v_mul_f32 v244, v240, v244\n\t" \
  "v_mfma_f32_16x16x128_f8f6f4 v[44:47], v[144:151], v[216:223], v[44:47] blgp:1\n\t" \
  "v_mul_f32 v245, v241, v245\n\t" \
  "s_waitcnt lgkmcnt(0)\n\t" \
  "v_mfma_f32_16x16x128_f8f6f4 v[60:63], v[152:159], v[216:223], v[60:63] blgp:1\n\t" \
  "buffer_load_dwordx4 v252, s[40:43], s52 offen offset:2048 sc0 sc1 lds\n\t" \
  "v_mul_f32 v246, v242, v246\n\t" \
  "v_mfma_f32_16x16x128_f8f6f4 v[56:59], v[152:159], v[208:215], v[56:59] blgp:1\n\t" \
  "v_mul_f32 v247, v243, v247\n\t" \
  "v_mfma_f32_16x16x128_f8f6f4 v[52:55], v[152:159], v[200:207], v[52:55] blgp:1\n\t" \
  "buffer_load_dwordx4 v252, s[40:43], s52 offen offset:3072 sc0 sc1 lds\n\t" \
  "v_mfma_f32_16x16x128_f8f6f4 v[48:51], v[152:159], v[192:199], v[48:51] blgp:1\n\t" \
  "s_waitcnt vmcnt(18) lgkmcnt(0)\n\t" \
  "s_barrier\n\t" \
  "v_mfma_f32_16x16x128_f8f6f4 a[0:3], v[128:135], v[160:167], a[0:3] blgp:1\n\t" \
  "v_readfirstlane_b32 s87, v254\n\t" \
  "v_readlane_b32 s65, v255, s67\n\t" \
  "v_accvgpr_read_b32 v240, a209\n\t" \
  "v_mfma_f32_16x16x128_f8f6f4 a[4:7], v[128:135], v[168:175], a[4:7] blgp:1\n\t" \
  "s_add_u32 s57, s80, 5242880\n\t" \
  "v_accvgpr_read_b32 v241, a213\n\t" \
  "v_mfma_f32_16x16x128_f8f6f4 a[8:11], v[128:135], v[176:183], a[8:11] blgp:1\n\t" \
  "buffer_store_dwordx4 v[244:247], v253, s[44:47], s57 offen offset:256 nt\n\t" \
  "s_max_u32 s69, s69, s87\n\t" \
  "s_add_u32 s67, s67, 1\n\t" \
  "s_cmp_ge_u32 s69, s65\n\t" \
  "s_cbranch_scc0 L_w18_%=\n\t" \
  "L_ok18_%=:\n\t" \
  "v_accvgpr_read_b32 v242, a217\n\t" \
  "v_mfma_f32_16x16x128_f8f6f4 a[12:15], v[128:135], v[184:191], a[12:15] blgp:1\n\t" \
  "ds_read_b128 v[128:131], v250 offset:49152\n\t" \
  "v_accvgpr_read_b32 v243, a221\n\t" \
  "v_mfma_f32_16x16x128_f8f6f4 a[28:31], v[136:143], v[184:191], a[28:31] blgp:1\n\t" \
  "s_add_i32 s52, s48, 32768\n\t" \
  "s_add_u32 m0, s53, 65536\n\t" \
  "ds_read_b128 v[132:135], v250 offset:57344\n\t" \
  "v_mul_f32 v244, s58, v240\n\t" \
  "v_mfma_f32_16x16x128_f8f6f4 a[24:27], v[136:143], v[176:183], a[24:27] blgp:1\n\t" \
  "buffer_load_dwordx4 v252, s[40:43], s52 offen offset:0 sc0 sc1 lds\n\t" \
  "v_mul_f32 v245, s58, v241\n\t" \
  "v_mfma_f32_16x16x128_f8f6f4 a[20:23], v[136:143], v[168:175], a[20:23] blgp:1\n\t" \
  "buffer_load_dwordx4 v252, s[40:43], s52 offen offset:1024 sc0 sc1 lds\n\t" \
  "v_mul_f32 v246, s58, v242\n\t" \
  "v_mfma_f32_16x16x128_f8f6f4 a[16:19], v[136:143], v[160:167], a[16:19] blgp:1\n\t" \
  "ds_read_b128 v[136:139], v250 offset:49408\n\t" \
  "v_mul_f32 v247, s58, v243\n\t" \
  "v_mfma_f32_16x16x128_f8f6f4 a[32:35], v[144:151], v[160:167], a[32:35] blgp:1\n\t" \
  "ds_read_b128 v[140:143], v250 offset:57600\n\t" \
  "v_med3_f32 v244, v244, -s62, s62\n\t" \
  "v_mfma_f32_16x16x128_f8f6f4 a[36:39], v[144:151], v[168:175], a[36:39] blgp:1\n\t" \
  "buffer_load_dwordx4 v252, s[40:43], s52 offen offset:2048 sc0 sc1 lds\n\t" \
  "v_med3_f32 v245, v245, -s62, s62\n\t" \
  "v_mfma_f32_16x16x128_f8f6f4 a[40:43], v[144:151], v[176:183], a[40:43] blgp:1\n\t" \
  "buffer_load_dwordx4 v252, s[40:43], s52 offen offset:3072 sc0 sc1 lds\n\t" \
  "v_med3_f32 v246, v246, -s62, s62\n\t" \
  "v_mfma_f32_16x16x128_f8f6f4 a[44:47], v[144:151], v[184:191], a[44:47] blgp:1\n\t" \
  "ds_read_b128 v[144:147], v250 offset:49664\n\t" \
  "v_med3_f32 v247, v247, -s62, s62\n\t" \
  "v_mfma_f32_16x16x128_f8f6f4 a[60:63], v[152:159], v[184:191], a[60:63] blgp:1\n\t" \
  "ds_read_b128 v[148:151], v250 offset:57856\n\t" \
  "v_exp_f32 v244, v244\n\t" \
  "v_mfma_f32_16x16x128_f8f6f4 a[56:59], v[152:159], v[176:183], a[56:59] blgp:1\n\t" \
  "v_exp_f32 v245, v245\n\t" \
  "v_mfma_f32_16x16x128_f8f6f4 a[52:55], v[152:159], v[168:175], a[52:55] blgp:1\n\t" \
  "v_exp_f32 v246, v246\n\t" \
  "v_mfma_f32_16x16x128_f8f6f4 a[48:51], v[152:159], v[160:167], a[48:51] blgp:1\n\t" \
  "ds_read_b128 v[152:155], v250 offset:49920\n\t" \
  "ds_read_b128 v[156:159], v250 offset:58112\n\t" \
  "s_waitcnt lgkmcnt(2)\n\t" \
  "v_mfma_f32_16x16x128_f8f6f4 v[64:67], v[128:135], v[160:167], v[64:67] blgp:1\n\t" \
  "s_add_i32 s52, s49, 49152\n\t" \
  "s_add_u32 m0, s53, 147456\n\t" \
  "v_exp_f32 v247, v247\n\t" \
  "v_mfma_f32_16x16x128_f8f6f4 v[68:71], v[128:135], v[168:175], v[68:71] blgp:1\n\t" \
  "buffer_load_dwordx4 v252, s[40:43], s52 offen offset:0 sc0 sc1 lds\n\t" \
  "v_mfma_f32_16x16x128_f8f6f4 v[72:75], v[128:135], v[176:183], v[72:75] blgp:1\n\t" \
  "buffer_load_dwordx4 v252, s[40:43], s52 offen offset:1024 sc0 sc1 lds\n\t" \
  "v_fma_f32 v244, v244, s60, s60\n\t" \
  "v_mfma_f32_16x16x128_f8f6f4 v[76:79], v[128:135], v[184:191], v[76:79] blgp:1\n\t" \
  "v_fma_f32 v245, v245, s60, s60\n\t" \
  "v_mfma_f32_16x16x128_f8f6f4 v[92:95], v[136:143], v[184:191], v[92:95] blgp:1\n\t" \
  "buffer_load_dwordx4 v252, s[40:43], s52 offen offset:2048 sc0 sc1 lds\n\t" \
  "v_fma_f32 v246, v246, s60, s60\n\t" \
  "v_mfma_f32_16x16x128_f8f6f4 v[88:91], v[136:143], v[176:183], v[88:91] blgp:1\n\t" \
  "buffer_load_dwordx4 v252, s[40:43], s52 offen offset:3072 sc0 sc1 lds\n\t" \
  "v_fma_f32 v247, v247, s60, s60\n\t" \
  "v_mfma_f32_16x16x128_f8f6f4 v[84:87], v[136:143], v[168:175], v[84:87] blgp:1\n\t" \
  "s_add_i32 s52, s49, 32768\n\t" \
  "s_add_u32 m0, s53, 131072\n\t" \
  "v_rcp_f32 v244, v244\n\t" \
  "v_mfma_f32_16x16x128_f8f6f4 v[80:83], v[136:143], v[160:167], v[80:83] blgp:1\n\t" \
  "buffer_load_dwordx4 v252, s[40:43], s52 offen offset:0 sc0 sc1 lds\n\t" \
  "v_rcp_f32 v245, v245\n\t" \
  "v_mfma_f32_16x16x128_f8f6f4 v[96:99], v[144:151], v[160:167], v[96:99] blgp:1\n\t" \
  "buffer_load_dwordx4 v252, s[40:43], s52 offen offset:1024 sc0 sc1 lds\n\t" \
  "v_rcp_f32 v246, v246\n\t" \
  "v_mfma_f32_16x16x128_f8f6f4 v[100:103], v[144:151], v[168:175], v[100:103] blgp:1\n\t" \
  "v_rcp_f32 v247, v247\n\t" \
  "v_mfma_f32_16x16x128_f8f6f4 v[104:107], v[144:151], v[176:183], v[104:107] blgp:1\n\t" \
  "buffer_load_dwordx4 v252, s[40:43], s52 offen offset:2048 sc0 sc1 lds\n\t" \
  "v_mul_f32 v244, v240, v244\n\t" \
  "v_mfma_f32_16x16x128_f8f6f4 v[108:111], v[144:151], v[184:191], v[108:111] blgp:1\n\t" \
  "buffer_load_dwordx4 v252, s[40:43], s52 offen offset:3072 sc0 sc1 lds\n\t" \
  "v_mul_f32 v245, v241, v245\n\t" \
  "s_waitcnt lgkmcnt(0)\n\t" \
  "v_mfma_f32_16x16x128_f8f6f4 v[124:127], v[152:159], v[184:191], v[124:127] blgp:1\n\t" \
  "v_mul_f32 v246, v242, v246\n\t" \
  "v_mfma_f32_16x16x128_f8f6f4 v[120:123], v[152:159], v[176:183], v[120:123] blgp:1\n\t" \
  "v_mul_f32 v247, v243, v247\n\t" \
  "v_mfma_f32_16x16x128_f8f6f4 v[116:119], v[152:159], v[168:175], v[116:119] blgp:1\n\t" \
  "v_mfma_f32_16x16x128_f8f6f4 v[112:115], v[152:159], v[160:167], v[112:115] blgp:1\n\t" \
  "s_waitcnt vmcnt(18) lgkmcnt(0)\n\t" \
  "s_barrier\n\t" \
  "v_mfma_f32_16x16x128_f8f6f4 a[64:67], v[128:135], v[192:199], a[64:67] blgp:1\n\t" \
  "ds_read_b128 v[160:163], v251 offset:0\n\t" \
  "v_accvgpr_read_b32 v240, a210\n\t" \
  "v_mfma_f32_16x16x128_f8f6f4 a[68:71], v[128:135], v[200:207], a[68:71] blgp:1\n\t" \
  "ds_read_b128 v[164:167], v251 offset:8192\n\t" \
  "s_add_u32 s57, s80, 5308416\n\t" \
  "v_accvgpr_read_b32 v241, a214\n\t" \
  "v_mfma_f32_16x16x128_f8f6f4 a[72:75], v[128:135], v[208:215], a[72:75] blgp:1\n\t" \
  "buffer_store_dwordx4 v[244:247], v253, s[44:47], s57 offen offset:256 nt\n\t" \
  "ds_read_b128 v[168:171], v251 offset:256\n\t" \
  "v_accvgpr_read_b32 v242, a218\n\t" \
  "v_mfma_f32_16x16x128_f8f6f4 a[76:79], v[128:135], v[216:223], a[76:79] blgp:1\n\t" \
  "ds_read_b128 v[128:131], v250 offset:0\n\t" \
  "ds_read_b128 v[132:135], v250 offset:8192\n\t" \
  "v_accvgpr_read_b32 v243, a222\n\t" \
  "v_mfma_f32_16x16x128_f8f6f4 a[92:95], v[136:143], v[216:223], a[92:95] blgp:1\n\t" \
  "ds_read_b128 v[172:175], v251 offset:8448\n\t" \
  "v_mul_f32 v244, s58, v240\n\t" \
  "v_mfma_f32_16x16x128_f8f6f4 a[88:91], v[136:143], v[208:215], a[88:91] blgp:1\n\t" \
  "ds_read_b128 v[176:179], v251 offset:512\n\t" \
  "v_mul_f32 v245, s58, v241\n\t" \
  "v_mfma_f32_16x16x128_f8f6f4 a[84:87], v[136:143], v[200:207], a[84:87] blgp:1\n\t" \
  "ds_read_b128 v[180:183], v251 offset:8704\n\t" \
  "v_mul_f32 v246, s58, v242\n\t" \
  "v_mfma_f32_16x16x128_f8f6f4 a[80:83], v[136:143], v[192:199], a[80:83] blgp:1\n\t" \
  "ds_read_b128 v[136:139], v250 offset:256\n\t" \
  "ds_read_b128 v[140:143], v250 offset:8448\n\t" \
  "v_mul_f32 v247, s58, v243\n\t" \
  "v_mfma_f32_16x16x128_f8f6f4 a[96:99], v[144:151], v[192:199], a[96:99] blgp:1\n\t" \
  "ds_read_b128 v[184:187], v251 offset:768\n\t" \
  "v_med3_f32 v244, v244, -s62, s62\n\t" \
  "v_mfma_f32_16x16x128_f8f6f4 a[100:103], v[144:151], v[200:207], a[100:103] blgp:1\n\t" \
  "ds_read_b128 v[188:191], v251 offset:8960\n\t" \
  "v_med3_f32 v245, v245, -s62, s62\n\t" \
  "v_mfma_f32_16x16x128_f8f6f4 a[104:107], v[144:151], v[208:215], a[104:107] blgp:1\n\t" \
  "s_add_u32 s48, s48, 0x10000\n\t" \
  "s_add_u32 s49, s49, 0x10000\n\t" \
  "v_med3_f32 v246, v246, -s62, s62\n\t" \
  "v_mfma_f32_16x16x128_f8f6f4 a[108:111], v[144:151], v[216:223], a[108:111] blgp:1\n\t" \
  "ds_read_b128 v[144:147], v250 offset:512\n\t" \
  "v_med3_f32 v247, v247, -s62, s62\n\t" \
  "v_mfma_f32_16x16x128_f8f6f4 a[124:127], v[152:159], v[216:223], a[124:127] blgp:1\n\t" \
  "ds_read_b128 v[148:151], v250 offset:8704\n\t" \
  "v_exp_f32 v244, v244\n\t" \
  "v_mfma_f32_16x16x128_f8f6f4 a[120:123], v[152:159], v[208:215], a[120:123] blgp:1\n\t" \
  "v_exp_f32 v245, v245\n\t" \
  "v_mfma_f32_16x16x128_f8f6f4 a[116:119], v[152:159], v[200:207], a[116:119] blgp:1\n\t" \
  "v_exp_f32 v246, v246\n\t" \
  "v_mfma_f32_16x16x128_f8f6f4 a[112:115], v[152:159], v[192:199], a[112:115] blgp:1\n\t" \
  "ds_read_b128 v[152:155], v250 offset:768\n\t" \
  "ds_read_b128 v[156:159], v250 offset:8960\n\t" \
  "s_waitcnt lgkmcnt(2)\n\t" \
  "v_mfma_f32_16x16x128_f8f6f4 a[0:3], v[128:135], v[160:167], a[0:3] blgp:1\n\t" \
  "ds_read_b128 v[192:195], v251 offset:16384\n\t" \
  "v_exp_f32 v247, v247\n\t" \
  "v_mfma_f32_16x16x128_f8f6f4 a[4:7], v[128:135], v[168:175], a[4:7] blgp:1\n\t" \
  "ds_read_b128 v[196:199], v251 offset:24576\n\t" \
  "v_mfma_f32_16x16x128_f8f6f4 a[8:11], v[128:135], v[176:183], a[8:11] blgp:1\n\t" \
  "ds_read_b128 v[200:203], v251 offset:16640\n\t" \
  "v_fma_f32 v244, v244, s60, s60\n\t" \
  "v_mfma_f32_16x16x128_f8f6f4 a[12:15], v[128:135], v[184:191], a[12:15] blgp:1\n\t" \
  "ds_read_b128 v[204:207], v251 offset:24832\n\t" \
  "v_fma_f32 v245, v245, s60, s60\n\t" \
  "v_mfma_f32_16x16x128_f8f6f4 a[28:31], v[136:143], v[184:191], a[28:31] blgp:1\n\t" \
  "ds_read_b128 v[208:211], v251 offset:16896\n\t" \
  "v_fma_f32 v246, v246, s60, s60\n\t" \
  "v_mfma_f32_16x16x128_f8f6f4 a[24:27], v[136:143], v[176:183], a[24:27] blgp:1\n\t" \
  "ds_read_b128 v[212:215], v251 offset:25088\n\t" \
  "v_fma_f32 v247, v247, s60, s60\n\t" \
  "v_mfma_f32_16x16x128_f8f6f4 a[20:23], v[136:143], v[168:175], a[20:23] blgp:1\n\t" \
  "ds_read_b128 v[216:219], v251 offset:17152\n\t" \
  "v_rcp_f32 v244, v244\n\t" \
  "v_mfma_f32_16x16x128_f8f6f4 a[16:19], v[136:143], v[160:167], a[16:19] blgp:1\n\t" \
  "ds_read_b128 v[220:223], v251 offset:25344\n\t" \
  "s_add_i32 s52, s48, -16384\n\t" \
  "s_add_u32 m0, s53, 81920\n\t" \
  "v_rcp_f32 v245, v245\n\t" \
  "v_mfma_f32_16x16x128_f8f6f4 a[32:35], v[144:151], v[160:167], a[32:35] blgp:1\n\t" \
  "buffer_load_dwordx4 v252, s[40:43], s52 offen offset:0 sc0 sc1 lds\n\t" \
  "v_rcp_f32 v246, v246\n\t" \
  "v_mfma_f32_16x16x128_f8f6f4 a[36:39], v[144:151], v[168:175], a[36:39] blgp:1\n\t" \
  "v_rcp_f32 v247, v247\n\t" \
  "v_mfma_f32_16x16x128_f8f6f4 a[40:43], v[144:151], v[176:183], a[40:43] blgp:1\n\t" \
  "buffer_load_dwordx4 v252, s[40:43], s52 offen offset:1024 sc0 sc1 lds\n\t" \
  "v_mul_f32 v244, v240, v244\n\t" \
  "v_mfma_f32_16x16x128_f8f6f4 a[44:47], v[144:151], v[184:191], a[44:47] blgp:1\n\t" \
  "v_mul_f32 v245, v241, v245\n\t" \
  "s_waitcnt lgkmcnt(0)\n\t" \
  "v_mfma_f32_16x16x128_f8f6f4 a[60:63], v[152:159], v[184:191], a[60:63] blgp:1\n\t" \
  "buffer_load_dwordx4 v252, s[40:43], s52 offen offset:2048 sc0 sc1 lds\n\t" \
  "v_mul_f32 v246, v242, v246\n\t" \
  "v_mfma_f32_16x16x128_f8f6f4 a[56:59], v[152:159], v[176:183], a[56:59] blgp:1\n\t" \
  "v_mul_f32 v247, v243, v247\n\t" \
  "v_mfma_f32_16x16x128_f8f6f4 a[52:55], v[152:159], v[168:175], a[52:55] blgp:1\n\t" \
  "buffer_load_dwordx4 v252, s[40:43], s52 offen offset:3072 sc0 sc1 lds\n\t" \
  "buffer_load_dword v254, off, s[88:91], s81 sc1\n\t" \
  "v_mfma_f32_16x16x128_f8f6f4 a[48:51], v[152:159], v[160:167], a[48:51] blgp:1\n\t" \
  "s_waitcnt vmcnt(19) lgkmcnt(0)\n\t" \
  "s_barrier\n\t" \
  "v_mfma_f32_16x16x128_f8f6f4 v[0:3], v[128:135], v[192:199], v[0:3] blgp:1\n\t" \
  "v_readlane_b32 s65, v255, s67\n\t" \
  "v_accvgpr_read_b32 v240, a211\n\t" \
  "v_mfma_f32_16x16x128_f8f6f4 v[4:7], v[128:135], v[200:207], v[4:7] blgp:1\n\t" \
  "s_add_u32 s57, s80, 5373952\n\t" \
  "v_accvgpr_read_b32 v241, a215\n\t" \
  "v_mfma_f32_16x16x128_f8f6f4 v[8:11], v[128:135], v[208:215], v[8:11] blgp:1\n\t" \
  "buffer_store_dwordx4 v[244:247], v253, s[44:47], s57 offen offset:256 nt\n\t" \
  "s_add_u32 s67, s67, 1\n\t" \
  "s_cmp_ge_u32 s69, s65\n\t" \
  "s_cbranch_scc0 L_w19_%=\n\t" \
  "L_ok19_%=:\n\t" \
  "v_accvgpr_read_b32 v242, a219\n\t" \
  "v_mfma_f32_16x16x128_f8f6f4 v[12:15], v[128:135], v[216:223], v[12:15] blgp:1\n\t" \
  "ds_read_b128 v[128:131], v250 offset:16384\n\t" \
  "v_accvgpr_read_b32 v243, a223\n\t" \
  "v_mfma_f32_16x16x128_f8f6f4 v[28:31], v[136:143], v[216:223], v[28:31] blgp:1\n\t" \
  "s_add_i32 s52, s48, 0\n\t" \
  "s_add_u32 m0, s53, 32768\n\t" \
  "ds_read_b128 v[132:135], v250 offset:24576\n\t" \
  "v_mul_f32 v244, s58, v240\n\t" \
  "v_mfma_f32_16x16x128_f8f6f4 v[24:27], v[136:143], v[208:215], v[24:27] blgp:1\n\t" \
  "buffer_load_dwordx4 v252, s[40:43], s52 offen offset:0 sc0 sc1 lds\n\t" \
  "v_mul_f32 v245, s58, v241\n\t" \
  "v_mfma_f32_16x16x128_f8f6f4 v[20:23], v[136:143], v[200:207], v[20:23] blgp:1\n\t" \
  "buffer_load_dwordx4 v252, s[40:43], s52 offen offset:1024 sc0 sc1 lds\n\t" \
  "v_mul_f32 v246, s58, v242\n\t" \
  "v_mfma_f32_16x16x128_f8f6f4 v[16:19], v[136:143], v[192:199], v[16:19] blgp:1\n\t" \
  "ds_read_b128 v[136:139], v250 offset:16640\n\t" \
  "v_mul_f32 v247, s58, v243\n\t" \
  "v_mfma_f32_16x16x128_f8f6f4 v[32:35], v[144:151], v[192:199], v[32:35] blgp:1\n\t" \
  "ds_read_b128 v[140:143], v250 offset:24832\n\t" \
  "v_med3_f32 v244, v244, -s62, s62\n\t" \
  "v_mfma_f32_16x16x128_f8f6f4 v[36:39], v[144:151], v[200:207], v[36:39] blgp:1\n\t" \
  "buffer_load_dwordx4 v252, s[40:43], s52 offen offset:2048 sc0 sc1 lds\n\t" \
  "v_med3_f32 v245, v245, -s62, s62\n\t" \
  "v_mfma_f32_16x16x128_f8f6f4 v[40:43], v[144:151], v[208:215], v[40:43] blgp:1\n\t" \
  "buffer_load_dwordx4 v252, s[40:43], s52 offen offset:3072 sc0 sc1 lds\n\t" \
  "v_med3_f32 v246, v246, -s62, s62\n\t" \
  "v_mfma_f32_16x16x128_f8f6f4 v[44:47], v[144:151], v[216:223], v[44:47] blgp:1\n\t" \
  "ds_read_b128 v[144:147], v250 offset:16896\n\t" \
  "v_med3_f32 v247, v247, -s62, s62\n\t" \
  "v_mfma_f32_16x16x128_f8f6f4 v[60:63], v[152:159], v[216:223], v[60:63] blgp:1\n\t" \
  "ds_read_b128 v[148:151], v250 offset:25088\n\t" \
  "v_exp_f32 v244, v244\n\t" \
  "v_mfma_f32_16x16x128_f8f6f4 v[56:59], v[152:159], v[208:215], v[56:59] blgp:1\n\t" \
  "v_exp_f32 v245, v245\n\t" \
  "v_mfma_f32_16x16x128_f8f6f4 v[52:55], v[152:159], v[200:207], v[52:55] blgp:1\n\t" \
  "v_exp_f32 v246, v246\n\t" \
  "v_mfma_f32_16x16x128_f8f6f4 v[48:51], v[152:159], v[192:199], v[48:51] blgp:1\n\t" \
  "ds_read_b128 v[152:155], v250 offset:17152\n\t" \
  "ds_read_b128 v[156:159], v250 offset:25344\n\t" \
  "s_waitcnt lgkmcnt(2)\n\t" \
  "v_mfma_f32_16x16x128_f8f6f4 a[64:67], v[128:135], v[192:199], a[64:67] blgp:1\n\t" \
  "s_add_i32 s52, s49, 0\n\t" \
  "s_add_u32 m0, s53, 98304\n\t" \
  "v_exp_f32 v247, v247\n\t" \
  "v_mfma_f32_16x16x128_f8f6f4 a[68:71], v[128:135], v[200:207], a[68:71] blgp:1\n\t" \
  "buffer_load_dwordx4 v252, s[40:43], s52 offen offset:0 sc0 sc1 lds\n\t" \
  "v_mfma_f32_16x16x128_f8f6f4 a[72:75], v[128:135], v[208:215], a[72:75] blgp:1\n\t" \
  "buffer_load_dwordx4 v252, s[40:43], s52 offen offset:1024 sc0 sc1 lds\n\t" \
  "v_fma_f32 v244, v244, s60, s60\n\t" \
  "v_mfma_f32_16x16x128_f8f6f4 a[76:79], v[128:135], v[216:223], a[76:79] blgp:1\n\t" \
  "v_fma_f32 v245, v245, s60, s60\n\t" \
  "v_mfma_f32_16x16x128_f8f6f4 a[92:95], v[136:143], v[216:223], a[92:95] blgp:1\n\t" \
  "buffer_load_dwordx4 v252, s[40:43], s52 offen offset:2048 sc0 sc1 lds\n\t" \
  "v_fma_f32 v246, v246, s60, s60\n\t" \
  "v_mfma_f32_16x16x128_f8f6f4 a[88:91], v[136:143], v[208:215], a[88:91] blgp:1\n\t" \
  "buffer_load_dwordx4 v252, s[40:43], s52 offen offset:3072 sc0 sc1 lds\n\t" \
  "v_fma_f32 v247, v247, s60, s60\n\t" \
  "v_mfma_f32_16x16x128_f8f6f4 a[84:87], v[136:143], v[200:207], a[84:87] blgp:1\n\t" \
  "s_add_i32 s52, s49, 16384\n\t" \
  "s_add_u32 m0, s53, 114688\n\t" \
  "v_rcp_f32 v244, v244\n\t" \
  "v_mfma_f32_16x16x128_f8f6f4 a[80:83], v[136:143], v[192:199], a[80:83] blgp:1\n\t" \
  "buffer_load_dwordx4 v252, s[40:43], s52 offen offset:0 sc0 sc1 lds\n\t" \
  "v_rcp_f32 v245, v245\n\t" \
  "v_mfma_f32_16x16x128_f8f6f4 a[96:99], v[144:151], v[192:199], a[96:99] blgp:1\n\t" \
  "buffer_load_dwordx4 v252, s[40:43], s52 offen offset:1024 sc0 sc1 lds\n\t" \
  "v_rcp_f32 v246, v246\n\t" \
  "v_mfma_f32_16x16x128_f8f6f4 a[100:103], v[144:151], v[200:207], a[100:103] blgp:1\n\t" \
  "v_rcp_f32 v247, v247\n\t" \
  "v_mfma_f32_16x16x128_f8f6f4 a[104:107], v[144:151], v[208:215], a[104:107] blgp:1\n\t" \
  "buffer_load_dwordx4 v252, s[40:43], s52 offen offset:2048 sc0 sc1 lds\n\t" \
  "v_mul_f32 v244, v240, v244\n\t" \
  "v_mfma_f32_16x16x128_f8f6f4 a[108:111], v[144:151], v[216:223], a[108:111] blgp:1\n\t" \
  "buffer_load_dwordx4 v252, s[40:43], s52 offen offset:3072 sc0 sc1 lds\n\t" \
  "v_mul_f32 v245, v241, v245\n\t" \
  "s_waitcnt lgkmcnt(0)\n\t" \
  "v_mfma_f32_16x16x128_f8f6f4 a[124:127], v[152:159], v[216:223], a[124:127] blgp:1\n\t" \
  "v_mul_f32 v246, v242, v246\n\t" \
  "v_mfma_f32_16x16x128_f8f6f4 a[120:123], v[152:159], v[208:215], a[120:123] blgp:1\n\t" \
  "v_mul_f32 v247, v243, v247\n\t" \
  "v_mfma_f32_16x16x128_f8f6f4 a[116:119], v[152:159], v[200:207], a[116:119] blgp:1\n\t" \
  "v_mfma_f32_16x16x128_f8f6f4 a[112:115], v[152:159], v[192:199], a[112:115] blgp:1\n\t" \
  "s_waitcnt vmcnt(19) lgkmcnt(0)\n\t" \
  "s_barrier\n\t" \
  "v_mfma_f32_16x16x128_f8f6f4 v[64:67], v[128:135], v[160:167], v[64:67] blgp:1\n\t" \
  "ds_read_b128 v[192:195], v251 offset:49152\n\t" \
  "v_accvgpr_read_b32 v240, a224\n\t" \
  "v_mfma_f32_16x16x128_f8f6f4 v[68:71], v[128:135], v[168:175], v[68:71] blgp:1\n\t" \
  "ds_read_b128 v[196:199], v251 offset:57344\n\t" \
  "s_add_u32 s57, s80, 5439488\n\t" \
  "v_accvgpr_read_b32 v241, a228\n\t" \
  "v_mfma_f32_16x16x128_f8f6f4 v[72:75], v[128:135], v[176:183], v[72:75] blgp:1\n\t" \
  "buffer_store_dwordx4 v[244:247], v253, s[44:47], s57 offen offset:256 nt\n\t" \
  "ds_read_b128 v[200:203], v251 offset:49408\n\t" \
  "v_accvgpr_read_b32 v242, a232\n\t" \
  "v_mfma_f32_16x16x128_f8f6f4 v[76:79], v[128:135], v[184:191], v[76:79] blgp:1\n\t" \
  "ds_read_b128 v[128:131], v250 offset:32768\n\t" \
  "ds_read_b128 v[132:135], v250 offset:40960\n\t" \
  "v_accvgpr_read_b32 v243, a236\n\t" \
  "v_mfma_f32_16x16x128_f8f6f4 v[92:95], v[136:143], v[184:191], v[92:95] blgp:1\n\t" \
  "ds_read_b128 v[204:207], v251 offset:57600\n\t" \
  "v_mul_f32 v244, s58, v240\n\t" \
  "v_mfma_f32_16x16x128_f8f6f4 v[88:91], v[136:143], v[176:183], v[88:91] blgp:1\n\t" \
  "ds_read_b128 v[208:211], v251 offset:49664\n\t" \
  "v_mul_f32 v245, s58, v241\n\t" \
  "v_mfma_f32_16x16x128_f8f6f4 v[84:87], v[136:143], v[168:175], v[84:87] blgp:1\n\t" \
  "ds_read_b128 v[212:215], v251 offset:57856\n\t" \
  "v_mul_f32 v246, s58, v242\n\t" \
  "v_mfma_f32_16x16x128_f8f6f4 v[80:83], v[136:143], v[160:167], v[80:83] blgp:1\n\t" \
  "ds_read_b128 v[136:139], v250 offset:33024\n\t" \
  "ds_read_b128 v[140:143], v250 offset:41216\n\t" \
  "v_mul_f32 v247, s58, v243\n\t" \
  "v_mfma_f32_16x16x128_f8f6f4 v[96:99], v[144:151], v[160:167], v[96:99] blgp:1\n\t" \
  "ds_read_b128 v[216:219], v251 offset:49920\n\t" \
  "v_med3_f32 v244, v244, -s62, s62\n\t" \
  "v_mfma_f32_16x16x128_f8f6f4 v[100:103], v[144:151], v[168:175], v[100:103] blgp:1\n\t" \
  "ds_read_b128 v[220:223], v251 offset:58112\n\t" \
  "v_med3_f32 v245, v245, -s62, s62\n\t" \
  "v_mfma_f32_16x16x128_f8f6f4 v[104:107], v[144:151], v[176:183], v[104:107] blgp:1\n\t" \
  "v_med3_f32 v246, v246, -s62, s62\n\t" \
  "v_mfma_f32_16x16x128_f8f6f4 v[108:111], v[144:151], v[184:191], v[108:111] blgp:1\n\t" \
  "ds_read_b128 v[144:147], v250 offset:33280\n\t" \
  "v_med3_f32 v247, v247, -s62, s62\n\t" \
  "v_mfma_f32_16x16x128_f8f6f4 v[124:127], v[152:159], v[184:191], v[124:127] blgp:1\n\t" \
  "ds_read_b128 v[148:151], v250 offset:41472\n\t" \
  "v_exp_f32 v244, v244\n\t" \
  "v_mfma_f32_16x16x128_f8f6f4 v[120:123], v[152:159], v[176:183], v[120:123] blgp:1\n\t" \
  "v_exp_f32 v245, v245\n\t" \
  "v_mfma_f32_16x16x128_f8f6f4 v[116:119], v[152:159], v[168:175], v[116:119] blgp:1\n\t" \
  "v_exp_f32 v246, v246\n\t" \
  "v_mfma_f32_16x16x128_f8f6f4 v[112:115], v[152:159], v[160:167], v[112:115] blgp:1\n\t" \
  "ds_read_b128 v[152:155], v250 offset:33536\n\t" \
  "ds_read_b128 v[156:159], v250 offset:41728\n\t" \
  "s_waitcnt lgkmcnt(2)\n\t" \
  "v_mfma_f32_16x16x128_f8f6f4 v[0:3], v[128:135], v[192:199], v[0:3] blgp:1\n\t" \
  "ds_read_b128 v[160:163], v251 offset:32768\n\t" \
  "v_exp_f32 v247, v247\n\t" \
  "v_mfma_f32_16x16x128_f8f6f4 v[4:7], v[128:135], v[200:207], v[4:7] blgp:1\n\t" \
  "ds_read_b128 v[164:167], v251 offset:40960\n\t" \
  "v_mfma_f32_16x16x128_f8f6f4 v[8:11], v[128:135], v[208:215], v[8:11] blgp:1\n\t" \
  "ds_read_b128 v[168:171], v251 offset:33024\n\t" \
  "v_fma_f32 v244, v244, s60, s60\n\t" \
  "v_mfma_f32_16x16x128_f8f6f4 v[12:15], v[128:135], v[216:223], v[12:15] blgp:1\n\t" \
  "ds_read_b128 v[172:175], v251 offset:41216\n\t" \
  "v_fma_f32 v245, v245, s60, s60\n\t" \
  "v_mfma_f32_16x16x128_f8f6f4 v[28:31], v[136:143], v[216:223], v[28:31] blgp:1\n\t" \
  "ds_read_b128 v[176:179], v251 offset:33280\n\t" \
  "v_fma_f32 v246, v246, s60, s60\n\t" \
  "v_mfma_f32_16x16x128_f8f6f4 v[24:27], v[136:143], v[208:215], v[24:27] blgp:1\n\t" \
  "ds_read_b128 v[180:183], v251 offset:41472\n\t" \
  "v_fma_f32 v247, v247, s60, s60\n\t" \
  "v_mfma_f32_16x16x128_f8f6f4 v[20:23], v[136:143], v[200:207], v[20:23] blgp:1\n\t" \
  "ds_read_b128 v[184:187], v251 offset:33536\n\t" \
  "v_rcp_f32 v244, v244\n\t" \
  "v_mfma_f32_16x16x128_f8f6f4 v[16:19], v[136:143], v[192:199], v[16:19] blgp:1\n\t" \
  "ds_read_b128 v[188:191], v251 offset:41728\n\t" \
  "s_add_i32 s52, s48, 16384\n\t" \
  "s_add_u32 m0, s53, 49152\n\t" \
  "v_rcp_f32 v245, v245\n\t" \
  "v_mfma_f32_16x16x128_f8f6f4 v[32:35], v[144:151], v[192:199], v[32:35] blgp:1\n\t" \
  "buffer_load_dwordx4 v252, s[40:43], s52 offen offset:0 sc0 sc1 lds\n\t" \
  "v_rcp_f32 v246, v246\n\t" \
  "v_mfma_f32_16x16x128_f8f6f4 v[36:39], v[144:151], v[200:207], v[36:39] blgp:1\n\t" \
  "v_rcp_f32 v247, v247\n\t" \
  "v_mfma_f32_16x16x128_f8f6f4 v[40:43], v[144:151], v[208:215], v[40:43] blgp:1\n\t" \
  "buffer_load_dwordx4 v252, s[40:43], s52 offen offset:1024 sc0 sc1 lds\n\t" \
  "v_mul_f32 v244, v240, v244\n\t" \
  "v_mfma_f32_16x16x128_f8f6f4 v[44:47], v[144:151], v[216:223], v[44:47] blgp:1\n\t" \
  "v_mul_f32 v245, v241, v245\n\t" \
  "s_waitcnt lgkmcnt(0)\n\t" \
  "v_mfma_f32_16x16x128_f8f6f4 v[60:63], v[152:159], v[216:223], v[60:63] blgp:1\n\t" \
  "buffer_load_dwordx4 v252, s[40:43], s52 offen offset:2048 sc0 sc1 lds\n\t" \
  "v_mul_f32 v246, v242, v246\n\t" \
  "v_mfma_f32_16x16x128_f8f6f4 v[56:59], v[152:159], v[208:215], v[56:59] blgp:1\n\t" \
  "v_mul_f32 v247, v243, v247\n\t" \
  "v_mfma_f32_16x16x128_f8f6f4 v[52:55], v[152:159], v[200:207], v[52:55] blgp:1\n\t" \
  "buffer_load_dwordx4 v252, s[40:43], s52 offen offset:3072 sc0 sc1 lds\n\t" \
  "v_mfma_f32_16x16x128_f8f6f4 v[48:51], v[152:159], v[192:199], v[48:51] blgp:1\n\t" \
  "s_waitcnt vmcnt(18) lgkmcnt(0)\n\t" \
  "s_barrier\n\t" \
  "v_mfma_f32_16x16x128_f8f6f4 a[0:3], v[128:135], v[160:167], a[0:3] blgp:1\n\t" \
  "v_readfirstlane_b32 s87, v254\n\t" \
  "v_readlane_b32 s65, v255, s67\n\t" \
  "v_accvgpr_read_b32 v240, a225\n\t" \
  "v_mfma_f32_16x16x128_f8f6f4 a[4:7], v[128:135], v[168:175], a[4:7] blgp:1\n\t" \
  "s_add_u32 s57, s80, 6291456\n\t" \
  "v_accvgpr_read_b32 v241, a229\n\t" \
  "v_mfma_f32_16x16x128_f8f6f4 a[8:11], v[128:135], v[176:183], a[8:11] blgp:1\n\t" \
  "buffer_store_dwordx4 v[244:247], v253, s[44:47], s57 offen offset:256 nt\n\t" \
  "s_max_u32 s69, s69, s87\n\t" \
  "s_add_u32 s67, s67, 1\n\t" \
  "s_cmp_ge_u32 s69, s65\n\t" \
  "s_cbranch_scc0 L_w20_%=\n\t" \
  "L_ok20_%=:\n\t" \
  "v_accvgpr_read_b32 v242, a233\n\t" \
  "v_mfma_f32_16x16x128_f8f6f4 a[12:15], v[128:135], v[184:191], a[12:15] blgp:1\n\t" \
  "ds_read_b128 v[128:131], v250 offset:49152\n\t" \
  "v_accvgpr_read_b32 v243, a237\n\t" \
  "v_mfma_f32_16x16x128_f8f6f4 a[28:31], v[136:143], v[184:191], a[28:31] blgp:1\n\t" \
  "s_add_i32 s52, s48, 32768\n\t" \
  "s_add_u32 m0, s53, 65536\n\t" \
  "ds_read_b128 v[132:135], v250 offset:57344\n\t" \
  "v_mul_f32 v244, s58, v240\n\t" \
  "v_mfma_f32_16x16x128_f8f6f4 a[24:27], v[136:143], v[176:183], a[24:27] blgp:1\n\t" \
  "buffer_load_dwordx4 v252, s[40:43], s52 offen offset:0 sc0 sc1 lds\n\t" \
  "v_mul_f32 v245, s58, v241\n\t" \
  "v_mfma_f32_16x16x128_f8f6f4 a[20:23], v[136:143], v[168:175], a[20:23] blgp:1\n\t" \
  "buffer_load_dwordx4 v252, s[40:43], s52 offen offset:1024 sc0 sc1 lds\n\t" \
  "v_mul_f32 v246, s58, v242\n\t" \
  "v_mfma_f32_16x16x128_f8f6f4 a[16:19], v[136:143], v[160:167], a[16:19] blgp:1\n\t" \
  "ds_read_b128 v[136:139], v250 offset:49408\n\t" \
  "v_mul_f32 v247, s58, v243\n\t" \
  "v_mfma_f32_16x16x128_f8f6f4 a[32:35], v[144:151], v[160:167], a[32:35] blgp:1\n\t" \
  "ds_read_b128 v[140:143], v250 offset:57600\n\t" \
  "v_med3_f32 v244, v244, -s62, s62\n\t" \
  "v_mfma_f32_16x16x128_f8f6f4 a[36:39], v[144:151], v[168:175], a[36:39] blgp:1\n\t" \
  "buffer_load_dwordx4 v252, s[40:43], s52 offen offset:2048 sc0 sc1 lds\n\t" \
  "v_med3_f32 v245, v245, -s62, s62\n\t" \
  "v_mfma_f32_16x16x128_f8f6f4 a[40:43], v[144:151], v[176:183], a[40:43] blgp:1\n\t" \
  "buffer_load_dwordx4 v252, s[40:43], s52 offen offset:3072 sc0 sc1 lds\n\t" \
  "v_med3_f32 v246, v246, -s62, s62\n\t" \
  "v_mfma_f32_16x16x128_f8f6f4 a[44:47], v[144:151], v[184:191], a[44:47] blgp:1\n\t" \
  "ds_read_b128 v[144:147], v250 offset:49664\n\t" \
  "v_med3_f32 v247, v247, -s62, s62\n\t" \
  "v_mfma_f32_16x16x128_f8f6f4 a[60:63], v[152:159], v[184:191], a[60:63] blgp:1\n\t" \
  "ds_read_b128 v[148:151], v250 offset:57856\n\t" \
  "v_exp_f32 v244, v244\n\t" \
  "v_mfma_f32_16x16x128_f8f6f4 a[56:59], v[152:159], v[176:183], a[56:59] blgp:1\n\t" \
  "v_exp_f32 v245, v245\n\t" \
  "v_mfma_f32_16x16x128_f8f6f4 a[52:55], v[152:159], v[168:175], a[52:55] blgp:1\n\t" \
  "v_exp_f32 v246, v246\n\t" \
  "v_mfma_f32_16x16x128_f8f6f4 a[48:51], v[152:159], v[160:167], a[48:51] blgp:1\n\t" \
  "ds_read_b128 v[152:155], v250 offset:49920\n\t" \
  "ds_read_b128 v[156:159], v250 offset:58112\n\t" \
  "s_waitcnt lgkmcnt(2)\n\t" \
  "v_mfma_f32_16x16x128_f8f6f4 v[64:67], v[128:135], v[160:167], v[64:67] blgp:1\n\t" \
  "s_add_i32 s52, s49, 49152\n\t" \
  "s_add_u32 m0, s53, 147456\n\t" \
  "v_exp_f32 v247, v247\n\t" \
  "v_mfma_f32_16x16x128_f8f6f4 v[68:71], v[128:135], v[168:175], v[68:71] blgp:1\n\t" \
  "buffer_load_dwordx4 v252, s[40:43], s52 offen offset:0 sc0 sc1 lds\n\t" \
  "v_mfma_f32_16x16x128_f8f6f4 v[72:75], v[128:135], v[176:183], v[72:75] blgp:1\n\t" \
  "buffer_load_dwordx4 v252, s[40:43], s52 offen offset:1024 sc0 sc1 lds\n\t" \
  "v_fma_f32 v244, v244, s60, s60\n\t" \
  "v_mfma_f32_16x16x128_f8f6f4 v[76:79], v[128:135], v[184:191], v[76:79] blgp:1\n\t" \
  "v_fma_f32 v245, v245, s60, s60\n\t" \
  "v_mfma_f32_16x16x128_f8f6f4 v[92:95], v[136:143], v[184:191], v[92:95] blgp:1\n\t" \
  "buffer_load_dwordx4 v252, s[40:43], s52 offen offset:2048 sc0 sc1 lds\n\t" \
  "v_fma_f32 v246, v246, s60, s60\n\t" \
  "v_mfma_f32_16x16x128_f8f6f4 v[88:91], v[136:143], v[176:183], v[88:91] blgp:1\n\t" \
  "buffer_load_dwordx4 v252, s[40:43], s52 offen offset:3072 sc0 sc1 lds\n\t" \
  "v_fma_f32 v247, v247, s60, s60\n\t" \
  "v_mfma_f32_16x16x128_f8f6f4 v[84:87], v[136:143], v[168:175], v[84:87] blgp:1\n\t" \
  "s_add_i32 s52, s49, 32768\n\t" \
  "s_add_u32 m0, s53, 131072\n\t" \
  "v_rcp_f32 v244, v244\n\t" \
  "v_mfma_f32_16x16x128_f8f6f4 v[80:83], v[136:143], v[160:167], v[80:83] blgp:1\n\t" \
  "buffer_load_dwordx4 v252, s[40:43], s52 offen offset:0 sc0 sc1 lds\n\t" \
  "v_rcp_f32 v245, v245\n\t" \
  "v_mfma_f32_16x16x128_f8f6f4 v[96:99], v[144:151], v[160:167], v[96:99] blgp:1\n\t" \
  "buffer_load_dwordx4 v252, s[40:43], s52 offen offset:1024 sc0 sc1 lds\n\t" \
  "v_rcp_f32 v246, v246\n\t" \
  "v_mfma_f32_16x16x128_f8f6f4 v[100:103], v[144:151], v[168:175], v[100:103] blgp:1\n\t" \
  "v_rcp_f32 v247, v247\n\t" \
  "v_mfma_f32_16x16x128_f8f6f4 v[104:107], v[144:151], v[176:183], v[104:107] blgp:1\n\t" \
  "buffer_load_dwordx4 v252, s[40:43], s52 offen offset:2048 sc0 sc1 lds\n\t" \
  "v_mul_f32 v244, v240, v244\n\t" \
  "v_mfma_f32_16x16x128_f8f6f4 v[108:111], v[144:151], v[184:191], v[108:111] blgp:1\n\t" \
  "buffer_load_dwordx4 v252, s[40:43], s52 offen offset:3072 sc0 sc1 lds\n\t" \
  "v_mul_f32 v245, v241, v245\n\t" \
  "s_waitcnt lgkmcnt(0)\n\t" \
  "v_mfma_f32_16x16x128_f8f6f4 v[124:127], v[152:159], v[184:191], v[124:127] blgp:1\n\t" \
  "v_mul_f32 v246, v242, v246\n\t" \
  "v_mfma_f32_16x16x128_f8f6f4 v[120:123], v[152:159], v[176:183], v[120:123] blgp:1\n\t" \
  "v_mul_f32 v247, v243, v247\n\t" \
  "v_mfma_f32_16x16x128_f8f6f4 v[116:119], v[152:159], v[168:175], v[116:119] blgp:1\n\t" \
  "v_mfma_f32_16x16x128_f8f6f4 v[112:115], v[152:159], v[160:167], v[112:115] blgp:1\n\t" \
  "s_waitcnt vmcnt(18) lgkmcnt(0)\n\t" \
  "s_barrier\n\t" \
  "v_mfma_f32_16x16x128_f8f6f4 a[64:67], v[128:135], v[192:199], a[64:67] blgp:1\n\t" \
  "ds_read_b128 v[160:163], v251 offset:0\n\t" \
  "v_accvgpr_read_b32 v240, a226\n\t" \
  "v_mfma_f32_16x16x128_f8f6f4 a[68:71], v[128:135], v[200:207], a[68:71] blgp:1\n\t" \
  "ds_read_b128 v[164:167], v251 offset:8192\n\t" \
  "s_add_u32 s57, s80, 6356992\n\t" \
  "v_accvgpr_read_b32 v241, a230\n\t" \
  "v_mfma_f32_16x16x128_f8f6f4 a[72:75], v[128:135], v[208:215], a[72:75] blgp:1\n\t" \
  "buffer_store_dwordx4 v[244:247], v253, s[44:47], s57 offen offset:256 nt\n\t" \
  "ds_read_b128 v[168:171], v251 offset:256\n\t" \
  "v_accvgpr_read_b32 v242, a234\n\t" \
  "v_mfma_f32_16x16x128_f8f6f4 a[76:79], v[128:135], v[216:223], a[76:79] blgp:1\n\t" \
  "ds_read_b128 v[128:131], v250 offset:0\n\t" \
  "ds_read_b128 v[132:135], v250 offset:8192\n\t" \
  "v_accvgpr_read_b32 v243, a238\n\t" \
  "v_mfma_f32_16x16x128_f8f6f4 a[92:95], v[136:143], v[216:223], a[92:95] blgp:1\n\t" \
  "ds_read_b128 v[172:175], v251 offset:8448\n\t" \
  "v_mul_f32 v244, s58, v240\n\t" \
  "v_mfma_f32_16x16x128_f8f6f4 a[88:91], v[136:143], v[208:215], a[88:91] blgp:1\n\t" \
  "ds_read_b128 v[176:179], v251 offset:512\n\t" \
  "v_mul_f32 v245, s58, v241\n\t" \
  "v_mfma_f32_16x16x128_f8f6f4 a[84:87], v[136:143], v[200:207], a[84:87] blgp:1\n\t" \
  "ds_read_b128 v[180:183], v251 offset:8704\n\t" \
  "v_mul_f32 v246, s58, v242\n\t" \
  "v_mfma_f32_16x16x128_f8f6f4 a[80:83], v[136:143], v[192:199], a[80:83] blgp:1\n\t" \
  "ds_read_b128 v[136:139], v250 offset:256\n\t" \
  "ds_read_b128 v[140:143], v250 offset:8448\n\t" \
  "v_mul_f32 v247, s58, v243\n\t" \
  "v_mfma_f32_16x16x128_f8f6f4 a[96:99], v[144:151], v[192:199], a[96:99] blgp:1\n\t" \
  "ds_read_b128 v[184:187], v251 offset:768\n\t" \
  "v_med3_f32 v244, v244, -s62, s62\n\t" \
  "v_mfma_f32_16x16x128_f8f6f4 a[100:103], v[144:151], v[200:207], a[100:103] blgp:1\n\t" \
  "ds_read_b128 v[188:191], v251 offset:8960\n\t" \
  "v_med3_f32 v245, v245, -s62, s62\n\t" \
  "v_mfma_f32_16x16x128_f8f6f4 a[104:107], v[144:151], v[208:215], a[104:107] blgp:1\n\t" \
  "s_add_u32 s48, s48, 0x10000\n\t" \
  "s_add_u32 s49, s49, 0x10000\n\t" \
  "v_med3_f32 v246, v246, -s62, s62\n\t" \
  "v_mfma_f32_16x16x128_f8f6f4 a[108:111], v[144:151], v[216:223], a[108:111] blgp:1\n\t" \
  "ds_read_b128 v[144:147], v250 offset:512\n\t" \
  "v_med3_f32 v247, v247, -s62, s62\n\t" \
  "v_mfma_f32_16x16x128_f8f6f4 a[124:127], v[152:159], v[216:223], a[124:127] blgp:1\n\t" \
  "ds_read_b128 v[148:151], v250 offset:8704\n\t" \
  "v_exp_f32 v244, v244\n\t" \
  "v_mfma_f32_16x16x128_f8f6f4 a[120:123], v[152:159], v[208:215], a[120:123] blgp:1\n\t" \
  "v_exp_f32 v245, v245\n\t" \
  "v_mfma_f32_16x16x128_f8f6f4 a[116:119], v[152:159], v[200:207], a[116:119] blgp:1\n\t" \
  "v_exp_f32 v246, v246\n\t" \
  "v_mfma_f32_16x16x128_f8f6f4 a[112:115], v[152:159], v[192:199], a[112:115] blgp:1\n\t" \
  "ds_read_b128 v[152:155], v250 offset:768\n\t" \
  "ds_read_b128 v[156:159], v250 offset:8960\n\t" \
  "s_waitcnt lgkmcnt(2)\n\t" \
  "v_mfma_f32_16x16x128_f8f6f4 a[0:3], v[128:135], v[160:167], a[0:3] blgp:1\n\t" \
  "ds_read_b128 v[192:195], v251 offset:16384\n\t" \
  "v_exp_f32 v247, v247\n\t" \
  "v_mfma_f32_16x16x128_f8f6f4 a[4:7], v[128:135], v[168:175], a[4:7] blgp:1\n\t" \
  "ds_read_b128 v[196:199], v251 offset:24576\n\t" \
  "v_mfma_f32_16x16x128_f8f6f4 a[8:11], v[128:135], v[176:183], a[8:11] blgp:1\n\t" \
  "ds_read_b128 v[200:203], v251 offset:16640\n\t" \
  "v_fma_f32 v244, v244, s60, s60\n\t" \
  "v_mfma_f32_16x16x128_f8f6f4 a[12:15], v[128:135], v[184:191], a[12:15] blgp:1\n\t" \
  "ds_read_b128 v[204:207], v251 offset:24832\n\t" \
  "v_fma_f32 v245, v245, s60, s60\n\t" \
  "v_mfma_f32_16x16x128_f8f6f4 a[28:31], v[136:143], v[184:191], a[28:31] blgp:1\n\t" \
  "ds_read_b128 v[208:211], v251 offset:16896\n\t" \
  "v_fma_f32 v246, v246, s60, s60\n\t" \
  "v_mfma_f32_16x16x128_f8f6f4 a[24:27], v[136:143], v[176:183], a[24:27] blgp:1\n\t" \
  "ds_read_b128 v[212:215], v251 offset:25088\n\t" \
  "v_fma_f32 v247, v247, s60, s60\n\t" \
  "v_mfma_f32_16x16x128_f8f6f4 a[20:23], v[136:143], v[168:175], a[20:23] blgp:1\n\t" \
  "ds_read_b128 v[216:219], v251 offset:17152\n\t" \
  "v_rcp_f32 v244, v244\n\t" \
  "v_mfma_f32_16x16x128_f8f6f4 a[16:19], v[136:143], v[160:167], a[16:19] blgp:1\n\t" \
  "ds_read_b128 v[220:223], v251 offset:25344\n\t" \
  "s_add_i32 s52, s48, -16384\n\t" \
  "s_add_u32 m0, s53, 81920\n\t" \
  "v_rcp_f32 v245, v245\n\t" \
  "v_mfma_f32_16x16x128_f8f6f4 a[32:35], v[144:151], v[160:167], a[32:35] blgp:1\n\t" \
  "buffer_load_dwordx4 v252, s[40:43], s52 offen offset:0 sc0 sc1 lds\n\t" \
  "v_rcp_f32 v246, v246\n\t" \
  "v_mfma_f32_16x16x128_f8f6f4 a[36:39], v[144:151], v[168:175], a[36:39] blgp:1\n\t" \
  "v_rcp_f32 v247, v247\n\t" \
  "v_mfma_f32_16x16x128_f8f6f4 a[40:43], v[144:151], v[176:183], a[40:43] blgp:1\n\t" \
  "buffer_load_dwordx4 v252, s[40:43], s52 offen offset:1024 sc0 sc1 lds\n\t" \
  "v_mul_f32 v244, v240, v244\n\t" \
  "v_mfma_f32_16x16x128_f8f6f4 a[44:47], v[144:151], v[184:191], a[44:47] blgp:1\n\t" \
  "v_mul_f32 v245, v241, v245\n\t" \
  "s_waitcnt lgkmcnt(0)\n\t" \
  "v_mfma_f32_16x16x128_f8f6f4 a[60:63], v[152:159], v[184:191], a[60:63] blgp:1\n\t" \
  "buffer_load_dwordx4 v252, s[40:43], s52 offen offset:2048 sc0 sc1 lds\n\t" \
  "v_mul_f32 v246, v242, v246\n\t" \
  "v_mfma_f32_16x16x128_f8f6f4 a[56:59], v[152:159], v[176:183], a[56:59] blgp:1\n\t" \
  "v_mul_f32 v247, v243, v247\n\t" \
  "v_mfma_f32_16x16x128_f8f6f4 a[52:55], v[152:159], v[168:175], a[52:55] blgp:1\n\t" \
  "buffer_load_dwordx4 v252, s[40:43], s52 offen offset:3072 sc0 sc1 lds\n\t" \
  "buffer_load_dword v254, off, s[88:91], s81 sc1\n\t" \
  "v_mfma_f32_16x16x128_f8f6f4 a[48:51], v[152:159], v[160:167], a[48:51] blgp:1\n\t" \
  "s_waitcnt vmcnt(19) lgkmcnt(0)\n\t" \
  "s_barrier\n\t" \
  "v_mfma_f32_16x16x128_f8f6f4 v[0:3], v[128:135], v[192:199], v[0:3] blgp:1\n\t" \
  "v_readlane_b32 s65, v255, s67\n\t" \
  "v_accvgpr_read_b32 v240, a227\n\t" \
  "v_mfma_f32_16x16x128_f8f6f4 v[4:7], v[128:135], v[200:207], v[4:7] blgp:1\n\t" \
  "s_add_u32 s57, s80, 6422528\n\t" \
  "v_accvgpr_read_b32 v241, a231\n\t" \
  "v_mfma_f32_16x16x128_f8f6f4 v[8:11], v[128:135], v[208:215], v[8:11] blgp:1\n\t" \
  "buffer_store_dwordx4 v[244:247], v253, s[44:47], s57 offen offset:256 nt\n\t" \
  "s_add_u32 s67, s67, 1\n\t" \
  "s_cmp_ge_u32 s69, s65\n\t" \
  "s_cbranch_scc0 L_w21_%=\n\t" \
  "L_ok21_%=:\n\t" \
  "v_accvgpr_read_b32 v242, a235\n\t" \
  "v_mfma_f32_16x16x128_f8f6f4 v[12:15], v[128:135], v[216:223], v[12:15] blgp:1\n\t" \
  "ds_read_b128 v[128:131], v250 offset:16384\n\t" \
  "v_accvgpr_read_b32 v243, a239\n\t" \
  "v_mfma_f32_16x16x128_f8f6f4 v[28:31], v[136:143], v[216:223], v[28:31] blgp:1\n\t" \
  "s_add_i32 s52, s48, 0\n\t" \
  "s_add_u32 m0, s53, 32768\n\t" \
  "ds_read_b128 v[132:135], v250 offset:24576\n\t" \
  "v_mul_f32 v244, s58, v240\n\t" \
  "v_mfma_f32_16x16x128_f8f6f4 v[24:27], v[136:143], v[208:215], v[24:27] blgp:1\n\t" \
  "buffer_load_dwordx4 v252, s[40:43], s52 offen offset:0 sc0 sc1 lds\n\t" \
  "v_mul_f32 v245, s58, v241\n\t" \
  "v_mfma_f32_16x16x128_f8f6f4 v[20:23], v[136:143], v[200:207], v[20:23] blgp:1\n\t" \
  "buffer_load_dwordx4 v252, s[40:43], s52 offen offset:1024 sc0 sc1 lds\n\t" \
  "v_mul_f32 v246, s58, v242\n\t" \
  "v_mfma_f32_16x16x128_f8f6f4 v[16:19], v[136:143], v[192:199], v[16:19] blgp:1\n\t" \
  "ds_read_b128 v[136:139], v250 offset:16640\n\t" \
  "v_mul_f32 v247, s58, v243\n\t" \
  "v_mfma_f32_16x16x128_f8f6f4 v[32:35], v[144:151], v[192:199], v[32:35] blgp:1\n\t" \
  "ds_read_b128 v[140:143], v250 offset:24832\n\t" \
  "v_med3_f32 v244, v244, -s62, s62\n\t" \
  "v_mfma_f32_16x16x128_f8f6f4 v[36:39], v[144:151], v[200:207], v[36:39] blgp:1\n\t" \
  "buffer_load_dwordx4 v252, s[40:43], s52 offen offset:2048 sc0 sc1 lds\n\t" \
  "v_med3_f32 v245, v245, -s62, s62\n\t" \
  "v_mfma_f32_16x16x128_f8f6f4 v[40:43], v[144:151], v[208:215], v[40:43] blgp:1\n\t" \
  "buffer_load_dwordx4 v252, s[40:43], s52 offen offset:3072 sc0 sc1 lds\n\t" \
  "v_med3_f32 v246, v246, -s62, s62\n\t" \
  "v_mfma_f32_16x16x128_f8f6f4 v[44:47], v[144:151], v[216:223], v[44:47] blgp:1\n\t" \
  "ds_read_b128 v[144:147], v250 offset:16896\n\t" \
  "v_med3_f32 v247, v247, -s62, s62\n\t" \
  "v_mfma_f32_16x16x128_f8f6f4 v[60:63], v[152:159], v[216:223], v[60:63] blgp:1\n\t" \
  "ds_read_b128 v[148:151], v250 offset:25088\n\t" \
  "v_exp_f32 v244, v244\n\t" \
  "v_mfma_f32_16x16x128_f8f6f4 v[56:59], v[152:159], v[208:215], v[56:59] blgp:1\n\t" \
  "v_exp_f32 v245, v245\n\t" \
  "v_mfma_f32_16x16x128_f8f6f4 v[52:55], v[152:159], v[200:207], v[52:55] blgp:1\n\t" \
  "v_exp_f32 v246, v246\n\t" \
  "v_mfma_f32_16x16x128_f8f6f4 v[48:51], v[152:159], v[192:199], v[48:51] blgp:1\n\t" \
  "ds_read_b128 v[152:155], v250 offset:17152\n\t" \
  "ds_read_b128 v[156:159], v250 offset:25344\n\t" \
  "s_waitcnt lgkmcnt(2)\n\t" \
  "v_mfma_f32_16x16x128_f8f6f4 a[64:67], v[128:135], v[192:199], a[64:67] blgp:1\n\t" \
  "s_add_i32 s52, s49, 0\n\t" \
  "s_add_u32 m0, s53, 98304\n\t" \
  "v_exp_f32 v247, v247\n\t" \
  "v_mfma_f32_16x16x128_f8f6f4 a[68:71], v[128:135], v[200:207], a[68:71] blgp:1\n\t" \
  "buffer_load_dwordx4 v252, s[40:43], s52 offen offset:0 sc0 sc1 lds\n\t" \
  "v_mfma_f32_16x16x128_f8f6f4 a[72:75], v[128:135], v[208:215], a[72:75] blgp:1\n\t" \
  "buffer_load_dwordx4 v252, s[40:43], s52 offen offset:1024 sc0 sc1 lds\n\t" \
  "v_fma_f32 v244, v244, s60, s60\n\t" \
  "v_mfma_f32_16x16x128_f8f6f4 a[76:79], v[128:135], v[216:223], a[76:79] blgp:1\n\t" \
  "v_fma_f32 v245, v245, s60, s60\n\t" \
  "v_mfma_f32_16x16x128_f8f6f4 a[92:95], v[136:143], v[216:223], a[92:95] blgp:1\n\t" \
  "buffer_load_dwordx4 v252, s[40:43], s52 offen offset:2048 sc0 sc1 lds\n\t" \
  "v_fma_f32 v246, v246, s60, s60\n\t" \
  "v_mfma_f32_16x16x128_f8f6f4 a[88:91], v[136:143], v[208:215], a[88:91] blgp:1\n\t" \
  "buffer_load_dwordx4 v252, s[40:43], s52 offen offset:3072 sc0 sc1 lds\n\t" \
  "v_fma_f32 v247, v247, s60, s60\n\t" \
  "v_mfma_f32_16x16x128_f8f6f4 a[84:87], v[136:143], v[200:207], a[84:87] blgp:1\n\t" \
  "s_add_i32 s52, s49, 16384\n\t" \
  "s_add_u32 m0, s53, 114688\n\t" \
  "v_rcp_f32 v244, v244\n\t" \
  "v_mfma_f32_16x16x128_f8f6f4 a[80:83], v[136:143], v[192:199], a[80:83] blgp:1\n\t" \
  "buffer_load_dwordx4 v252, s[40:43], s52 offen offset:0 sc0 sc1 lds\n\t" \
  "v_rcp_f32 v245, v245\n\t" \
  "v_mfma_f32_16x16x128_f8f6f4 a[96:99], v[144:151], v[192:199], a[96:99] blgp:1\n\t" \
  "buffer_load_dwordx4 v252, s[40:43], s52 offen offset:1024 sc0 sc1 lds\n\t" \
  "v_rcp_f32 v246, v246\n\t" \
  "v_mfma_f32_16x16x128_f8f6f4 a[100:103], v[144:151], v[200:207], a[100:103] blgp:1\n\t" \
  "v_rcp_f32 v247, v247\n\t" \
  "v_mfma_f32_16x16x128_f8f6f4 a[104:107], v[144:151], v[208:215], a[104:107] blgp:1\n\t" \
  "buffer_load_dwordx4 v252, s[40:43], s52 offen offset:2048 sc0 sc1 lds\n\t" \
  "v_mul_f32 v244, v240, v244\n\t" \
  "v_mfma_f32_16x16x128_f8f6f4 a[108:111], v[144:151], v[216:223], a[108:111] blgp:1\n\t" \
  "buffer_load_dwordx4 v252, s[40:43], s52 offen offset:3072 sc0 sc1 lds\n\t" \
  "v_mul_f32 v245, v241, v245\n\t" \
  "s_waitcnt lgkmcnt(0)\n\t" \
  "v_mfma_f32_16x16x128_f8f6f4 a[124:127], v[152:159], v[216:223], a[124:127] blgp:1\n\t" \
  "v_mul_f32 v246, v242, v246\n\t" \
  "v_mfma_f32_16x16x128_f8f6f4 a[120:123], v[152:159], v[208:215], a[120:123] blgp:1\n\t" \
  "v_mul_f32 v247, v243, v247\n\t" \
  "v_mfma_f32_16x16x128_f8f6f4 a[116:119], v[152:159], v[200:207], a[116:119] blgp:1\n\t" \
  "v_mfma_f32_16x16x128_f8f6f4 a[112:115], v[152:159], v[192:199], a[112:115] blgp:1\n\t" \
  "s_waitcnt vmcnt(19) lgkmcnt(0)\n\t" \
  "s_barrier\n\t" \
  "v_mfma_f32_16x16x128_f8f6f4 v[64:67], v[128:135], v[160:167], v[64:67] blgp:1\n\t" \
  "ds_read_b128 v[192:195], v251 offset:49152\n\t" \
  "v_accvgpr_read_b32 v240, a240\n\t" \
  "v_mfma_f32_16x16x128_f8f6f4 v[68:71], v[128:135], v[168:175], v[68:71] blgp:1\n\t" \
  "ds_read_b128 v[196:199], v251 offset:57344\n\t" \
  "s_add_u32 s57, s80, 6488064\n\t" \
  "v_accvgpr_read_b32 v241, a244\n\t" \
  "v_mfma_f32_16x16x128_f8f6f4 v[72:75], v[128:135], v[176:183], v[72:75] blgp:1\n\t" \
  "buffer_store_dwordx4 v[244:247], v253, s[44:47], s57 offen offset:256 nt\n\t" \
  "ds_read_b128 v[200:203], v251 offset:49408\n\t" \
  "v_accvgpr_read_b32 v242, a248\n\t" \
  "v_mfma_f32_16x16x128_f8f6f4 v[76:79], v[128:135], v[184:191], v[76:79] blgp:1\n\t" \
  "ds_read_b128 v[128:131], v250 offset:32768\n\t" \
  "ds_read_b128 v[132:135], v250 offset:40960\n\t" \
  "v_accvgpr_read_b32 v243, a252\n\t" \
  "v_mfma_f32_16x16x128_f8f6f4 v[92:95], v[136:143], v[184:191], v[92:95] blgp:1\n\t" \
  "ds_read_b128 v[204:207], v251 offset:57600\n\t" \
  "v_mul_f32 v244, s58, v240\n\t" \
  "v_mfma_f32_16x16x128_f8f6f4 v[88:91], v[136:143], v[176:183], v[88:91] blgp:1\n\t" \
  "ds_read_b128 v[208:211], v251 offset:49664\n\t" \
  "v_mul_f32 v245, s58, v241\n\t" \
  "v_mfma_f32_16x16x128_f8f6f4 v[84:87], v[136:143], v[168:175], v[84:87] blgp:1\n\t" \
  "ds_read_b128 v[212:215], v251 offset:57856\n\t" \
  "v_mul_f32 v246, s58, v242\n\t" \
  "v_mfma_f32_16x16x128_f8f6f4 v[80:83], v[136:143], v[160:167], v[80:83] blgp:1\n\t" \
  "ds_read_b128 v[136:139], v250 offset:33024\n\t" \
  "ds_read_b128 v[140:143], v250 offset:41216\n\t" \
  "v_mul_f32 v247, s58, v243\n\t" \
  "v_mfma_f32_16x16x128_f8f6f4 v[96:99], v[144:151], v[160:167], v[96:99] blgp:1\n\t" \
  "ds_read_b128 v[216:219], v251 offset:49920\n\t" \
  "v_med3_f32 v244, v244, -s62, s62\n\t" \
  "v_mfma_f32_16x16x128_f8f6f4 v[100:103], v[144:151], v[168:175], v[100:103] blgp:1\n\t" \
  "ds_read_b128 v[220:223], v251 offset:58112\n\t" \
  "v_med3_f32 v245, v245, -s62, s62\n\t" \
  "v_mfma_f32_16x16x128_f8f6f4 v[104:107], v[144:151], v[176:183], v[104:107] blgp:1\n\t" \
  "v_med3_f32 v246, v246, -s62, s62\n\t" \
  "v_mfma_f32_16x16x128_f8f6f4 v[108:111], v[144:151], v[184:191], v[108:111] blgp:1\n\t" \
  "ds_read_b128 v[144:147], v250 offset:33280\n\t" \
  "v_med3_f32 v247, v247, -s62, s62\n\t" \
  "v_mfma_f32_16x16x128_f8f6f4 v[124:127], v[152:159], v[184:191], v[124:127] blgp:1\n\t" \
  "ds_read_b128 v[148:151], v250 offset:41472\n\t" \
  "v_exp_f32 v244, v244\n\t" \
  "v_mfma_f32_16x16x128_f8f6f4 v[120:123], v[152:159], v[176:183], v[120:123] blgp:1\n\t" \
  "v_exp_f32 v245, v245\n\t" \
  "v_mfma_f32_16x16x128_f8f6f4 v[116:119], v[152:159], v[168:175], v[116:119] blgp:1\n\t" \
  "v_exp_f32 v246, v246\n\t" \
  "v_mfma_f32_16x16x128_f8f6f4 v[112:115], v[152:159], v[160:167], v[112:115] blgp:1\n\t" \
  "ds_read_b128 v[152:155], v250 offset:33536\n\t" \
  "ds_read_b128 v[156:159], v250 offset:41728\n\t" \
  "s_waitcnt lgkmcnt(2)\n\t" \
  "v_mfma_f32_16x16x128_f8f6f4 v[0:3], v[128:135], v[192:199], v[0:3] blgp:1\n\t" \
  "ds_read_b128 v[160:163], v251 offset:32768\n\t" \
  "v_exp_f32 v247, v247\n\t" \
  "v_mfma_f32_16x16x128_f8f6f4 v[4:7], v[128:135], v[200:207], v[4:7] blgp:1\n\t" \
  "ds_read_b128 v[164:167], v251 offset:40960\n\t" \
  "v_mfma_f32_16x16x128_f8f6f4 v[8:11], v[128:135], v[208:215], v[8:11] blgp:1\n\t" \
  "ds_read_b128 v[168:171], v251 offset:33024\n\t" \
  "v_fma_f32 v244, v244, s60, s60\n\t" \
  "v_mfma_f32_16x16x128_f8f6f4 v[12:15], v[128:135], v[216:223], v[12:15] blgp:1\n\t" \
  "ds_read_b128 v[172:175], v251 offset:41216\n\t" \
  "v_fma_f32 v245, v245, s60, s60\n\t" \
  "v_mfma_f32_16x16x128_f8f6f4 v[28:31], v[136:143], v[216:223], v[28:31] blgp:1\n\t" \
  "ds_read_b128 v[176:179], v251 offset:33280\n\t" \
  "v_fma_f32 v246, v246, s60, s60\n\t" \
  "v_mfma_f32_16x16x128_f8f6f4 v[24:27], v[136:143], v[208:215], v[24:27] blgp:1\n\t" \
  "ds_read_b128 v[180:183], v251 offset:41472\n\t" \
  "v_fma_f32 v247, v247, s60, s60\n\t" \
  "v_mfma_f32_16x16x128_f8f6f4 v[20:23], v[136:143], v[200:207], v[20:23] blgp:1\n\t" \
  "ds_read_b128 v[184:187], v251 offset:33536\n\t" \
  "v_rcp_f32 v244, v244\n\t" \
  "v_mfma_f32_16x16x128_f8f6f4 v[16:19], v[136:143], v[192:199], v[16:19] blgp:1\n\t" \
  "ds_read_b128 v[188:191], v251 offset:41728\n\t" \
  "s_add_i32 s52, s48, 16384\n\t" \
  "s_add_u32 m0, s53, 49152\n\t" \
  "v_rcp_f32 v245, v245\n\t" \
  "v_mfma_f32_16x16x128_f8f6f4 v[32:35], v[144:151], v[192:199], v[32:35] blgp:1\n\t" \
  "buffer_load_dwordx4 v252, s[40:43], s52 offen offset:0 sc0 sc1 lds\n\t" \
  "v_rcp_f32 v246, v246\n\t" \
  "v_mfma_f32_16x16x128_f8f6f4 v[36:39], v[144:151], v[200:207], v[36:39] blgp:1\n\t" \
  "v_rcp_f32 v247, v247\n\t" \
  "v_mfma_f32_16x16x128_f8f6f4 v[40:43], v[144:151], v[208:215], v[40:43] blgp:1\n\t" \
  "buffer_load_dwordx4 v252, s[40:43], s52 offen offset:1024 sc0 sc1 lds\n\t" \
  "v_mul_f32 v244, v240, v244\n\t" \
  "v_mfma_f32_16x16x128_f8f6f4 v[44:47], v[144:151], v[216:223], v[44:47] blgp:1\n\t" \
  "v_mul_f32 v245, v241, v245\n\t" \
  "s_waitcnt lgkmcnt(0)\n\t" \
  "v_mfma_f32_16x16x128_f8f6f4 v[60:63], v[152:159], v[216:223], v[60:63] blgp:1\n\t" \
  "buffer_load_dwordx4 v252, s[40:43], s52 offen offset:2048 sc0 sc1 lds\n\t" \
  "v_mul_f32 v246, v242, v246\n\t" \
  "v_mfma_f32_16x16x128_f8f6f4 v[56:59], v[152:159], v[208:215], v[56:59] blgp:1\n\t" \
  "v_mul_f32 v247, v243, v247\n\t" \
  "v_mfma_f32_16x16x128_f8f6f4 v[52:55], v[152:159], v[200:207], v[52:55] blgp:1\n\t" \
  "buffer_load_dwordx4 v252, s[40:43], s52 offen offset:3072 sc0 sc1 lds\n\t" \
  "v_mfma_f32_16x16x128_f8f6f4 v[48:51], v[152:159], v[192:199], v[48:51] blgp:1\n\t" \
  "s_waitcnt vmcnt(18) lgkmcnt(0)\n\t" \
  "s_barrier\n\t" \
  "v_mfma_f32_16x16x128_f8f6f4 a[0:3], v[128:135], v[160:167], a[0:3] blgp:1\n\t" \
  "v_readfirstlane_b32 s87, v254\n\t" \
  "v_readlane_b32 s65, v255, s67\n\t" \
  "v_accvgpr_read_b32 v240, a241\n\t" \
  "v_mfma_f32_16x16x128_f8f6f4 a[4:7], v[128:135], v[168:175], a[4:7] blgp:1\n\t" \
  "s_add_u32 s57, s80, 7340032\n\t" \
  "v_accvgpr_read_b32 v241, a245\n\t" \
  "v_mfma_f32_16x16x128_f8f6f4 a[8:11], v[128:135], v[176:183], a[8:11] blgp:1\n\t" \
  "buffer_store_dwordx4 v[244:247], v253, s[44:47], s57 offen offset:256 nt\n\t" \
  "s_max_u32 s69, s69, s87\n\t" \
  "s_add_u32 s67, s67, 1\n\t" \
  "s_cmp_ge_u32 s69, s65\n\t" \
  "s_cbranch_scc0 L_w22_%=\n\t" \
  "L_ok22_%=:\n\t" \
  "v_accvgpr_read_b32 v242, a249\n\t" \
  "v_mfma_f32_16x16x128_f8f6f4 a[12:15], v[128:135], v[184:191], a[12:15] blgp:1\n\t" \
  "ds_read_b128 v[128:131], v250 offset:49152\n\t" \
  "v_accvgpr_read_b32 v243, a253\n\t" \
  "v_mfma_f32_16x16x128_f8f6f4 a[28:31], v[136:143], v[184:191], a[28:31] blgp:1\n\t" \
  "s_add_i32 s52, s48, 32768\n\t" \
  "s_add_u32 m0, s53, 65536\n\t" \
  "ds_read_b128 v[132:135], v250 offset:57344\n\t" \
  "v_mul_f32 v244, s58, v240\n\t" \
  "v_mfma_f32_16x16x128_f8f6f4 a[24:27], v[136:143], v[176:183], a[24:27] blgp:1\n\t" \
  "buffer_load_dwordx4 v252, s[40:43], s52 offen offset:0 sc0 sc1 lds\n\t" \
  "v_mul_f32 v245, s58, v241\n\t" \
  "v_mfma_f32_16x16x128_f8f6f4 a[20:23], v[136:143], v[168:175], a[20:23] blgp:1\n\t" \
  "buffer_load_dwordx4 v252, s[40:43], s52 offen offset:1024 sc0 sc1 lds\n\t" \
  "v_mul_f32 v246, s58, v242\n\t" \
  "v_mfma_f32_16x16x128_f8f6f4 a[16:19], v[136:143], v[160:167], a[16:19] blgp:1\n\t" \
  "ds_read_b128 v[136:139], v250 offset:49408\n\t" \
  "v_mul_f32 v247, s58, v243\n\t" \
  "v_mfma_f32_16x16x128_f8f6f4 a[32:35], v[144:151], v[160:167], a[32:35] blgp:1\n\t" \
  "ds_read_b128 v[140:143], v250 offset:57600\n\t" \
  "v_med3_f32 v244, v244, -s62, s62\n\t" \
  "v_mfma_f32_16x16x128_f8f6f4 a[36:39], v[144:151], v[168:175], a[36:39] blgp:1\n\t" \
  "buffer_load_dwordx4 v252, s[40:43], s52 offen offset:2048 sc0 sc1 lds\n\t" \
  "v_med3_f32 v245, v245, -s62, s62\n\t" \
  "v_mfma_f32_16x16x128_f8f6f4 a[40:43], v[144:151], v[176:183], a[40:43] blgp:1\n\t" \
  "buffer_load_dwordx4 v252, s[40:43], s52 offen offset:3072 sc0 sc1 lds\n\t" \
  "v_med3_f32 v246, v246, -s62, s62\n\t" \
  "v_mfma_f32_16x16x128_f8f6f4 a[44:47], v[144:151], v[184:191], a[44:47] blgp:1\n\t" \
  "ds_read_b128 v[144:147], v250 offset:49664\n\t" \
  "v_med3_f32 v247, v247, -s62, s62\n\t" \
  "v_mfma_f32_16x16x128_f8f6f4 a[60:63], v[152:159], v[184:191], a[60:63] blgp:1\n\t" \
  "ds_read_b128 v[148:151], v250 offset:57856\n\t" \
  "v_exp_f32 v244, v244\n\t" \
  "v_mfma_f32_16x16x128_f8f6f4 a[56:59], v[152:159], v[176:183], a[56:59] blgp:1\n\t" \
  "v_exp_f32 v245, v245\n\t" \
  "v_mfma_f32_16x16x128_f8f6f4 a[52:55], v[152:159], v[168:175], a[52:55] blgp:1\n\t" \
  "v_exp_f32 v246, v246\n\t" \
  "v_mfma_f32_16x16x128_f8f6f4 a[48:51], v[152:159], v[160:167], a[48:51] blgp:1\n\t" \
  "ds_read_b128 v[152:155], v250 offset:49920\n\t" \
  "ds_read_b128 v[156:159], v250 offset:58112\n\t" \
  "s_waitcnt lgkmcnt(2)\n\t" \
  "v_mfma_f32_16x16x128_f8f6f4 v[64:67], v[128:135], v[160:167], v[64:67] blgp:1\n\t" \
  "s_add_i32 s52, s49, 49152\n\t" \
  "s_add_u32 m0, s53, 147456\n\t" \
  "v_exp_f32 v247, v247\n\t" \
  "v_mfma_f32_16x16x128_f8f6f4 v[68:71], v[128:135], v[168:175], v[68:71] blgp:1\n\t" \
  "buffer_load_dwordx4 v252, s[40:43], s52 offen offset:0 sc0 sc1 lds\n\t" \
  "v_mfma_f32_16x16x128_f8f6f4 v[72:75], v[128:135], v[176:183], v[72:75] blgp:1\n\t" \
  "buffer_load_dwordx4 v252, s[40:43], s52 offen offset:1024 sc0 sc1 lds\n\t" \
  "v_fma_f32 v244, v244, s60, s60\n\t" \
  "v_mfma_f32_16x16x128_f8f6f4 v[76:79], v[128:135], v[184:191], v[76:79] blgp:1\n\t" \
  "v_fma_f32 v245, v245, s60, s60\n\t" \
  "v_mfma_f32_16x16x128_f8f6f4 v[92:95], v[136:143], v[184:191], v[92:95] blgp:1\n\t" \
  "buffer_load_dwordx4 v252, s[40:43], s52 offen offset:2048 sc0 sc1 lds\n\t" \
  "v_fma_f32 v246, v246, s60, s60\n\t" \
  "v_mfma_f32_16x16x128_f8f6f4 v[88:91], v[136:143], v[176:183], v[88:91] blgp:1\n\t" \
  "buffer_load_dwordx4 v252, s[40:43], s52 offen offset:3072 sc0 sc1 lds\n\t" \
  "v_fma_f32 v247, v247, s60, s60\n\t" \
  "v_mfma_f32_16x16x128_f8f6f4 v[84:87], v[136:143], v[168:175], v[84:87] blgp:1\n\t" \
  "s_add_i32 s52, s49, 32768\n\t" \
  "s_add_u32 m0, s53, 131072\n\t" \
  "v_rcp_f32 v244, v244\n\t" \
  "v_mfma_f32_16x16x128_f8f6f4 v[80:83], v[136:143], v[160:167], v[80:83] blgp:1\n\t" \
  "buffer_load_dwordx4 v252, s[40:43], s52 offen offset:0 sc0 sc1 lds\n\t" \
  "v_rcp_f32 v245, v245\n\t" \
  "v_mfma_f32_16x16x128_f8f6f4 v[96:99], v[144:151], v[160:167], v[96:99] blgp:1\n\t" \
  "buffer_load_dwordx4 v252, s[40:43], s52 offen offset:1024 sc0 sc1 lds\n\t" \
  "v_rcp_f32 v246, v246\n\t" \
  "v_mfma_f32_16x16x128_f8f6f4 v[100:103], v[144:151], v[168:175], v[100:103] blgp:1\n\t" \
  "v_rcp_f32 v247, v247\n\t" \
  "v_mfma_f32_16x16x128_f8f6f4 v[104:107], v[144:151], v[176:183], v[104:107] blgp:1\n\t" \
  "buffer_load_dwordx4 v252, s[40:43], s52 offen offset:2048 sc0 sc1 lds\n\t" \
  "v_mul_f32 v244, v240, v244\n\t" \
  "v_mfma_f32_16x16x128_f8f6f4 v[108:111], v[144:151], v[184:191], v[108:111] blgp:1\n\t" \
  "buffer_load_dwordx4 v252, s[40:43], s52 offen offset:3072 sc0 sc1 lds\n\t" \
  "v_mul_f32 v245, v241, v245\n\t" \
  "s_waitcnt lgkmcnt(0)\n\t" \
  "v_mfma_f32_16x16x128_f8f6f4 v[124:127], v[152:159], v[184:191], v[124:127] blgp:1\n\t" \
  "v_mul_f32 v246, v242, v246\n\t" \
  "v_mfma_f32_16x16x128_f8f6f4 v[120:123], v[152:159], v[176:183], v[120:123] blgp:1\n\t" \
  "v_mul_f32 v247, v243, v247\n\t" \
  "v_mfma_f32_16x16x128_f8f6f4 v[116:119], v[152:159], v[168:175], v[116:119] blgp:1\n\t" \
  "v_mfma_f32_16x16x128_f8f6f4 v[112:115], v[152:159], v[160:167], v[112:115] blgp:1\n\t" \
  "s_waitcnt vmcnt(18) lgkmcnt(0)\n\t" \
  "s_barrier\n\t" \
  "v_mfma_f32_16x16x128_f8f6f4 a[64:67], v[128:135], v[192:199], a[64:67] blgp:1\n\t" \
  "ds_read_b128 v[160:163], v251 offset:0\n\t" \
  "v_accvgpr_read_b32 v240, a242\n\t" \
  "v_mfma_f32_16x16x128_f8f6f4 a[68:71], v[128:135], v[200:207], a[68:71] blgp:1\n\t" \
  "ds_read_b128 v[164:167], v251 offset:8192\n\t" \
  "s_add_u32 s57, s80, 7405568\n\t" \
  "v_accvgpr_read_b32 v241, a246\n\t" \
  "v_mfma_f32_16x16x128_f8f6f4 a[72:75], v[128:135], v[208:215], a[72:75] blgp:1\n\t" \
  "buffer_store_dwordx4 v[244:247], v253, s[44:47], s57 offen offset:256 nt\n\t" \
  "ds_read_b128 v[168:171], v251 offset:256\n\t" \
  "v_accvgpr_read_b32 v242, a250\n\t" \
  "v_mfma_f32_16x16x128_f8f6f4 a[76:79], v[128:135], v[216:223], a[76:79] blgp:1\n\t" \
  "ds_read_b128 v[128:131], v250 offset:0\n\t" \
  "ds_read_b128 v[132:135], v250 offset:8192\n\t" \
  "v_accvgpr_read_b32 v243, a254\n\t" \
  "v_mfma_f32_16x16x128_f8f6f4 a[92:95], v[136:143], v[216:223], a[92:95] blgp:1\n\t" \
  "ds_read_b128 v[172:175], v251 offset:8448\n\t" \
  "v_mul_f32 v244, s58, v240\n\t" \
  "v_mfma_f32_16x16x128_f8f6f4 a[88:91], v[136:143], v[208:215], a[88:91] blgp:1\n\t" \
  "ds_read_b128 v[176:179], v251 offset:512\n\t" \
  "v_mul_f32 v245, s58, v241\n\t" \
  "v_mfma_f32_16x16x128_f8f6f4 a[84:87], v[136:143], v[200:207], a[84:87] blgp:1\n\t" \
  "ds_read_b128 v[180:183], v251 offset:8704\n\t" \
  "v_mul_f32 v246, s58, v242\n\t" \
  "v_mfma_f32_16x16x128_f8f6f4 a[80:83], v[136:143], v[192:199], a[80:83] blgp:1\n\t" \
  "ds_read_b128 v[136:139], v250 offset:256\n\t" \
  "ds_read_b128 v[140:143], v250 offset:8448\n\t" \
  "v_mul_f32 v247, s58, v243\n\t" \
  "v_mfma_f32_16x16x128_f8f6f4 a[96:99], v[144:151], v[192:199], a[96:99] blgp:1\n\t" \
  "ds_read_b128 v[184:187], v251 offset:768\n\t" \
  "v_med3_f32 v244, v244, -s62, s62\n\t" \
  "v_mfma_f32_16x16x128_f8f6f4 a[100:103], v[144:151], v[200:207], a[100:103] blgp:1\n\t" \
  "ds_read_b128 v[188:191], v251 offset:8960\n\t" \
  "v_med3_f32 v245, v245, -s62, s62\n\t" \
  "v_mfma_f32_16x16x128_f8f6f4 a[104:107], v[144:151], v[208:215], a[104:107] blgp:1\n\t" \
  "s_add_u32 s48, s48, 0x10000\n\t" \
  "s_add_u32 s49, s49, 0x10000\n\t" \
  "v_med3_f32 v246, v246, -s62, s62\n\t" \
  "v_mfma_f32_16x16x128_f8f6f4 a[108:111], v[144:151], v[216:223], a[108:111] blgp:1\n\t" \
  "ds_read_b128 v[144:147], v250 offset:512\n\t" \
  "v_med3_f32 v247, v247, -s62, s62\n\t" \
  "v_mfma_f32_16x16x128_f8f6f4 a[124:127], v[152:159], v[216:223], a[124:127] blgp:1\n\t" \
  "ds_read_b128 v[148:151], v250 offset:8704\n\t" \
  "v_exp_f32 v244, v244\n\t" \
  "v_mfma_f32_16x16x128_f8f6f4 a[120:123], v[152:159], v[208:215], a[120:123] blgp:1\n\t" \
  "v_exp_f32 v245, v245\n\t" \
  "v_mfma_f32_16x16x128_f8f6f4 a[116:119], v[152:159], v[200:207], a[116:119] blgp:1\n\t" \
  "v_exp_f32 v246, v246\n\t" \
  "v_mfma_f32_16x16x128_f8f6f4 a[112:115], v[152:159], v[192:199], a[112:115] blgp:1\n\t" \
  "ds_read_b128 v[152:155], v250 offset:768\n\t" \
  "ds_read_b128 v[156:159], v250 offset:8960\n\t" \
  "s_waitcnt lgkmcnt(2)\n\t" \
  "v_mfma_f32_16x16x128_f8f6f4 a[0:3], v[128:135], v[160:167], a[0:3] blgp:1\n\t" \
  "ds_read_b128 v[192:195], v251 offset:16384\n\t" \
  "v_exp_f32 v247, v247\n\t" \
  "v_mfma_f32_16x16x128_f8f6f4 a[4:7], v[128:135], v[168:175], a[4:7] blgp:1\n\t" \
  "ds_read_b128 v[196:199], v251 offset:24576\n\t" \
  "v_mfma_f32_16x16x128_f8f6f4 a[8:11], v[128:135], v[176:183], a[8:11] blgp:1\n\t" \
  "ds_read_b128 v[200:203], v251 offset:16640\n\t" \
  "v_fma_f32 v244, v244, s60, s60\n\t" \
  "v_mfma_f32_16x16x128_f8f6f4 a[12:15], v[128:135], v[184:191], a[12:15] blgp:1\n\t" \
  "ds_read_b128 v[204:207], v251 offset:24832\n\t" \
  "v_fma_f32 v245, v245, s60, s60\n\t" \
  "v_mfma_f32_16x16x128_f8f6f4 a[28:31], v[136:143], v[184:191], a[28:31] blgp:1\n\t" \
  "ds_read_b128 v[208:211], v251 offset:16896\n\t" \
  "v_fma_f32 v246, v246, s60, s60\n\t" \
  "v_mfma_f32_16x16x128_f8f6f4 a[24:27], v[136:143], v[176:183], a[24:27] blgp:1\n\t" \
  "ds_read_b128 v[212:215], v251 offset:25088\n\t" \
  "v_fma_f32 v247, v247, s60, s60\n\t" \
  "v_mfma_f32_16x16x128_f8f6f4 a[20:23], v[136:143], v[168:175], a[20:23] blgp:1\n\t" \
  "ds_read_b128 v[216:219], v251 offset:17152\n\t" \
  "v_rcp_f32 v244, v244\n\t" \
  "v_mfma_f32_16x16x128_f8f6f4 a[16:19], v[136:143], v[160:167], a[16:19] blgp:1\n\t" \
  "ds_read_b128 v[220:223], v251 offset:25344\n\t" \
  "s_add_i32 s52, s48, -16384\n\t" \
  "s_add_u32 m0, s53, 81920\n\t" \
  "v_rcp_f32 v245, v245\n\t" \
  "v_mfma_f32_16x16x128_f8f6f4 a[32:35], v[144:151], v[160:167], a[32:35] blgp:1\n\t" \
  "buffer_load_dwordx4 v252, s[40:43], s52 offen offset:0 sc0 sc1 lds\n\t" \
  "v_rcp_f32 v246, v246\n\t" \
  "v_mfma_f32_16x16x128_f8f6f4 a[36:39], v[144:151], v[168:175], a[36:39] blgp:1\n\t" \
  "v_rcp_f32 v247, v247\n\t" \
  "v_mfma_f32_16x16x128_f8f6f4 a[40:43], v[144:151], v[176:183], a[40:43] blgp:1\n\t" \
  "buffer_load_dwordx4 v252, s[40:43], s52 offen offset:1024 sc0 sc1 lds\n\t" \
  "v_mul_f32 v244, v240, v244\n\t" \
  "v_mfma_f32_16x16x128_f8f6f4 a[44:47], v[144:151], v[184:191], a[44:47] blgp:1\n\t" \
  "v_mul_f32 v245, v241, v245\n\t" \
  "s_waitcnt lgkmcnt(0)\n\t" \
  "v_mfma_f32_16x16x128_f8f6f4 a[60:63], v[152:159], v[184:191], a[60:63] blgp:1\n\t" \
  "buffer_load_dwordx4 v252, s[40:43], s52 offen offset:2048 sc0 sc1 lds\n\t" \
  "v_mul_f32 v246, v242, v246\n\t" \
  "v_mfma_f32_16x16x128_f8f6f4 a[56:59], v[152:159], v[176:183], a[56:59] blgp:1\n\t" \
  "v_mul_f32 v247, v243, v247\n\t" \
  "v_mfma_f32_16x16x128_f8f6f4 a[52:55], v[152:159], v[168:175], a[52:55] blgp:1\n\t" \
  "buffer_load_dwordx4 v252, s[40:43], s52 offen offset:3072 sc0 sc1 lds\n\t" \
  "buffer_load_dword v254, off, s[88:91], s81 sc1\n\t" \
  "v_mfma_f32_16x16x128_f8f6f4 a[48:51], v[152:159], v[160:167], a[48:51] blgp:1\n\t" \
  "s_waitcnt vmcnt(19) lgkmcnt(0)\n\t" \
  "s_barrier\n\t" \
  "v_mfma_f32_16x16x128_f8f6f4 v[0:3], v[128:135], v[192:199], v[0:3] blgp:1\n\t" \
  "v_readlane_b32 s65, v255, s67\n\t" \
  "v_accvgpr_read_b32 v240, a243\n\t" \
  "v_mfma_f32_16x16x128_f8f6f4 v[4:7], v[128:135], v[200:207], v[4:7] blgp:1\n\t" \
  "s_add_u32 s57, s80, 7471104\n\t" \
  "v_accvgpr_read_b32 v241, a247\n\t" \
  "v_mfma_f32_16x16x128_f8f6f4 v[8:11], v[128:135], v[208:215], v[8:11] blgp:1\n\t" \
  "buffer_store_dwordx4 v[244:247], v253, s[44:47], s57 offen offset:256 nt\n\t" \
  "s_add_u32 s67, s67, 1\n\t" \
  "s_cmp_ge_u32 s69, s65\n\t" \
  "s_cbranch_scc0 L_w23_%=\n\t" \
  "L_ok23_%=:\n\t" \
  "v_accvgpr_read_b32 v242, a251\n\t" \
  "v_mfma_f32_16x16x128_f8f6f4 v[12:15], v[128:135], v[216:223], v[12:15] blgp:1\n\t" \
  "ds_read_b128 v[128:131], v250 offset:16384\n\t" \
  "v_accvgpr_read_b32 v243, a255\n\t" \
  "v_mfma_f32_16x16x128_f8f6f4 v[28:31], v[136:143], v[216:223], v[28:31] blgp:1\n\t" \
  "s_add_i32 s52, s48, 0\n\t" \
  "s_add_u32 m0, s53, 32768\n\t" \
  "ds_read_b128 v[132:135], v250 offset:24576\n\t" \
  "v_mul_f32 v244, s58, v240\n\t" \
  "v_mfma_f32_16x16x128_f8f6f4 v[24:27], v[136:143], v[208:215], v[24:27] blgp:1\n\t" \
  "buffer_load_dwordx4 v252, s[40:43], s52 offen offset:0 sc0 sc1 lds\n\t" \
  "v_mul_f32 v245, s58, v241\n\t" \
  "v_mfma_f32_16x16x128_f8f6f4 v[20:23], v[136:143], v[200:207], v[20:23] blgp:1\n\t" \
  "buffer_load_dwordx4 v252, s[40:43], s52 offen offset:1024 sc0 sc1 lds\n\t" \
  "v_mul_f32 v246, s58, v242\n\t" \
  "v_mfma_f32_16x16x128_f8f6f4 v[16:19], v[136:143], v[192:199], v[16:19] blgp:1\n\t" \
  "ds_read_b128 v[136:139], v250 offset:16640\n\t" \
  "v_mul_f32 v247, s58, v243\n\t" \
  "v_mfma_f32_16x16x128_f8f6f4 v[32:35], v[144:151], v[192:199], v[32:35] blgp:1\n\t" \
  "ds_read_b128 v[140:143], v250 offset:24832\n\t" \
  "v_med3_f32 v244, v244, -s62, s62\n\t" \
  "v_mfma_f32_16x16x128_f8f6f4 v[36:39], v[144:151], v[200:207], v[36:39] blgp:1\n\t" \
  "buffer_load_dwordx4 v252, s[40:43], s52 offen offset:2048 sc0 sc1 lds\n\t" \
  "v_med3_f32 v245, v245, -s62, s62\n\t" \
  "v_mfma_f32_16x16x128_f8f6f4 v[40:43], v[144:151], v[208:215], v[40:43] blgp:1\n\t" \
  "buffer_load_dwordx4 v252, s[40:43], s52 offen offset:3072 sc0 sc1 lds\n\t" \
  "v_med3_f32 v246, v246, -s62, s62\n\t" \
  "v_mfma_f32_16x16x128_f8f6f4 v[44:47], v[144:151], v[216:223], v[44:47] blgp:1\n\t" \
  "ds_read_b128 v[144:147], v250 offset:16896\n\t" \
  "v_med3_f32 v247, v247, -s62, s62\n\t" \
  "v_mfma_f32_16x16x128_f8f6f4 v[60:63], v[152:159], v[216:223], v[60:63] blgp:1\n\t" \
  "ds_read_b128 v[148:151], v250 offset:25088\n\t" \
  "v_exp_f32 v244, v244\n\t" \
  "v_mfma_f32_16x16x128_f8f6f4 v[56:59], v[152:159], v[208:215], v[56:59] blgp:1\n\t" \
  "v_exp_f32 v245, v245\n\t" \
  "v_mfma_f32_16x16x128_f8f6f4 v[52:55], v[152:159], v[200:207], v[52:55] blgp:1\n\t" \
  "v_exp_f32 v246, v246\n\t" \
  "v_mfma_f32_16x16x128_f8f6f4 v[48:51], v[152:159], v[192:199], v[48:51] blgp:1\n\t" \
  "ds_read_b128 v[152:155], v250 offset:17152\n\t" \
  "ds_read_b128 v[156:159], v250 offset:25344\n\t" \
  "s_waitcnt lgkmcnt(2)\n\t" \
  "v_mfma_f32_16x16x128_f8f6f4 a[64:67], v[128:135], v[192:199], a[64:67] blgp:1\n\t" \
  "s_add_i32 s52, s49, 0\n\t" \
  "s_add_u32 m0, s53, 98304\n\t" \
  "v_exp_f32 v247, v247\n\t" \
  "v_mfma_f32_16x16x128_f8f6f4 a[68:71], v[128:135], v[200:207], a[68:71] blgp:1\n\t" \
  "buffer_load_dwordx4 v252, s[40:43], s52 offen offset:0 sc0 sc1 lds\n\t" \
  "v_mfma_f32_16x16x128_f8f6f4 a[72:75], v[128:135], v[208:215], a[72:75] blgp:1\n\t" \
  "buffer_load_dwordx4 v252, s[40:43], s52 offen offset:1024 sc0 sc1 lds\n\t" \
  "v_fma_f32 v244, v244, s60, s60\n\t" \
  "v_mfma_f32_16x16x128_f8f6f4 a[76:79], v[128:135], v[216:223], a[76:79] blgp:1\n\t" \
  "v_fma_f32 v245, v245, s60, s60\n\t" \
  "v_mfma_f32_16x16x128_f8f6f4 a[92:95], v[136:143], v[216:223], a[92:95] blgp:1\n\t" \
  "buffer_load_dwordx4 v252, s[40:43], s52 offen offset:2048 sc0 sc1 lds\n\t" \
  "v_fma_f32 v246, v246, s60, s60\n\t" \
  "v_mfma_f32_16x16x128_f8f6f4 a[88:91], v[136:143], v[208:215], a[88:91] blgp:1\n\t" \
  "buffer_load_dwordx4 v252, s[40:43], s52 offen offset:3072 sc0 sc1 lds\n\t" \
  "v_fma_f32 v247, v247, s60, s60\n\t" \
  "v_mfma_f32_16x16x128_f8f6f4 a[84:87], v[136:143], v[200:207], a[84:87] blgp:1\n\t" \
  "s_add_i32 s52, s49, 16384\n\t" \
  "s_add_u32 m0, s53, 114688\n\t" \
  "v_rcp_f32 v244, v244\n\t" \
  "v_mfma_f32_16x16x128_f8f6f4 a[80:83], v[136:143], v[192:199], a[80:83] blgp:1\n\t" \
  "buffer_load_dwordx4 v252, s[40:43], s52 offen offset:0 sc0 sc1 lds\n\t" \
  "v_rcp_f32 v245, v245\n\t" \
  "v_mfma_f32_16x16x128_f8f6f4 a[96:99], v[144:151], v[192:199], a[96:99] blgp:1\n\t" \
  "buffer_load_dwordx4 v252, s[40:43], s52 offen offset:1024 sc0 sc1 lds\n\t" \
  "v_rcp_f32 v246, v246\n\t" \
  "v_mfma_f32_16x16x128_f8f6f4 a[100:103], v[144:151], v[200:207], a[100:103] blgp:1\n\t" \
  "v_rcp_f32 v247, v247\n\t" \
  "v_mfma_f32_16x16x128_f8f6f4 a[104:107], v[144:151], v[208:215], a[104:107] blgp:1\n\t" \
  "buffer_load_dwordx4 v252, s[40:43], s52 offen offset:2048 sc0 sc1 lds\n\t" \
  "v_mul_f32 v244, v240, v244\n\t" \
  "v_mfma_f32_16x16x128_f8f6f4 a[108:111], v[144:151], v[216:223], a[108:111] blgp:1\n\t" \
  "buffer_load_dwordx4 v252, s[40:43], s52 offen offset:3072 sc0 sc1 lds\n\t" \
  "v_mul_f32 v245, v241, v245\n\t" \
  "s_waitcnt lgkmcnt(0)\n\t" \
  "v_mfma_f32_16x16x128_f8f6f4 a[124:127], v[152:159], v[216:223], a[124:127] blgp:1\n\t" \
  "v_mul_f32 v246, v242, v246\n\t" \
  "v_mfma_f32_16x16x128_f8f6f4 a[120:123], v[152:159], v[208:215], a[120:123] blgp:1\n\t" \
  "v_mul_f32 v247, v243, v247\n\t" \
  "v_mfma_f32_16x16x128_f8f6f4 a[116:119], v[152:159], v[200:207], a[116:119] blgp:1\n\t" \
  "v_mfma_f32_16x16x128_f8f6f4 a[112:115], v[152:159], v[192:199], a[112:115] blgp:1\n\t" \
  "s_waitcnt vmcnt(19) lgkmcnt(0)\n\t" \
  "s_barrier\n\t" \
  "v_mfma_f32_16x16x128_f8f6f4 v[64:67], v[128:135], v[160:167], v[64:67] blgp:1\n\t" \
  "ds_read_b128 v[192:195], v251 offset:49152\n\t" \
  "v_mfma_f32_16x16x128_f8f6f4 v[68:71], v[128:135], v[168:175], v[68:71] blgp:1\n\t" \
  "ds_read_b128 v[196:199], v251 offset:57344\n\t" \
  "s_add_u32 s57, s80, 7536640\n\t" \
  "v_mfma_f32_16x16x128_f8f6f4 v[72:75], v[128:135], v[176:183], v[72:75] blgp:1\n\t" \
  "buffer_store_dwordx4 v[244:247], v253, s[44:47], s57 offen offset:256 nt\n\t" \
  "ds_read_b128 v[200:203], v251 offset:49408\n\t" \
  "v_mfma_f32_16x16x128_f8f6f4 v[76:79], v[128:135], v[184:191], v[76:79] blgp:1\n\t" \
  "ds_read_b128 v[128:131], v250 offset:32768\n\t" \
  "ds_read_b128 v[132:135], v250 offset:40960\n\t" \
  "v_mfma_f32_16x16x128_f8f6f4 v[92:95], v[136:143], v[184:191], v[92:95] blgp:1\n\t" \
  "ds_read_b128 v[204:207], v251 offset:57600\n\t" \
  "v_mfma_f32_16x16x128_f8f6f4 v[88:91], v[136:143], v[176:183], v[88:91] blgp:1\n\t" \
  "ds_read_b128 v[208:211], v251 offset:49664\n\t" \
  "v_mfma_f32_16x16x128_f8f6f4 v[84:87], v[136:143], v[168:175], v[84:87] blgp:1\n\t" \
  "ds_read_b128 v[212:215], v251 offset:57856\n\t" \
  "v_mfma_f32_16x16x128_f8f6f4 v[80:83], v[136:143], v[160:167], v[80:83] blgp:1\n\t" \
  "ds_read_b128 v[136:139], v250 offset:33024\n\t" \
  "ds_read_b128 v[140:143], v250 offset:41216\n\t" \
  "v_mfma_f32_16x16x128_f8f6f4 v[96:99], v[144:151], v[160:167], v[96:99] blgp:1\n\t" \
  "ds_read_b128 v[216:219], v251 offset:49920\n\t" \
  "v_mfma_f32_16x16x128_f8f6f4 v[100:103], v[144:151], v[168:175], v[100:103] blgp:1\n\t" \
  "ds_read_b128 v[220:223], v251 offset:58112\n\t" \
  "v_mfma_f32_16x16x128_f8f6f4 v[104:107], v[144:151], v[176:183], v[104:107] blgp:1\n\t" \
  "v_mfma_f32_16x16x128_f8f6f4 v[108:111], v[144:151], v[184:191], v[108:111] blgp:1\n\t" \
  "ds_read_b128 v[144:147], v250 offset:33280\n\t" \
  "v_mfma_f32_16x16x128_f8f6f4 v[124:127], v[152:159], v[184:191], v[124:127] blgp:1\n\t" \
  "ds_read_b128 v[148:151], v250 offset:41472\n\t" \
  "v_mfma_f32_16x16x128_f8f6f4 v[120:123], v[152:159], v[176:183], v[120:123] blgp:1\n\t" \
  "v_mfma_f32_16x16x128_f8f6f4 v[116:119], v[152:159], v[168:175], v[116:119] blgp:1\n\t" \
  "v_mfma_f32_16x16x128_f8f6f4 v[112:115], v[152:159], v[160:167], v[112:115] blgp:1\n\t" \
  "ds_read_b128 v[152:155], v250 offset:33536\n\t" \
  "ds_read_b128 v[156:159], v250 offset:41728\n\t" \
  "s_waitcnt lgkmcnt(2)\n\t" \
  "v_mfma_f32_16x16x128_f8f6f4 v[0:3], v[128:135], v[192:199], v[0:3] blgp:1\n\t" \
  "ds_read_b128 v[160:163], v251 offset:32768\n\t" \
  "v_mfma_f32_16x16x128_f8f6f4 v[4:7], v[128:135], v[200:207], v[4:7] blgp:1\n\t" \
  "ds_read_b128 v[164:167], v251 offset:40960\n\t" \
  "v_mfma_f32_16x16x128_f8f6f4 v[8:11], v[128:135], v[208:215], v[8:11] blgp:1\n\t" \
  "ds_read_b128 v[168:171], v251 offset:33024\n\t" \
  "v_mfma_f32_16x16x128_f8f6f4 v[12:15], v[128:135], v[216:223], v[12:15] blgp:1\n\t" \
  "ds_read_b128 v[172:175], v251 offset:41216\n\t" \
  "v_mfma_f32_16x16x128_f8f6f4 v[28:31], v[136:143], v[216:223], v[28:31] blgp:1\n\t" \
  "ds_read_b128 v[176:179], v251 offset:33280\n\t" \
  "v_mfma_f32_16x16x128_f8f6f4 v[24:27], v[136:143], v[208:215], v[24:27] blgp:1\n\t" \
  "ds_read_b128 v[180:183], v251 offset:41472\n\t" \
  "v_mfma_f32_16x16x128_f8f6f4 v[20:23], v[136:143], v[200:207], v[20:23] blgp:1\n\t" \
  "ds_read_b128 v[184:187], v251 offset:33536\n\t" \
  "v_mfma_f32_16x16x128_f8f6f4 v[16:19], v[136:143], v[192:199], v[16:19] blgp:1\n\t" \
  "ds_read_b128 v[188:191], v251 offset:41728\n\t" \
  "s_add_i32 s52, s48, 16384\n\t" \
  "s_add_u32 m0, s53, 49152\n\t" \
  "v_mfma_f32_16x16x128_f8f6f4 v[32:35], v[144:151], v[192:199], v[32:35] blgp:1\n\t" \
  "buffer_load_dwordx4 v252, s[40:43], s52 offen offset:0 sc0 sc1 lds\n\t" \
  "v_mfma_f32_16x16x128_f8f6f4 v[36:39], v[144:151], v[200:207], v[36:39] blgp:1\n\t" \
  "v_mfma_f32_16x16x128_f8f6f4 v[40:43], v[144:151], v[208:215], v[40:43] blgp:1\n\t" \
  "buffer_load_dwordx4 v252, s[40:43], s52 offen offset:1024 sc0 sc1 lds\n\t" \
  "v_mfma_f32_16x16x128_f8f6f4 v[44:47], v[144:151], v[216:223], v[44:47] blgp:1\n\t" \
  "s_waitcnt lgkmcnt(0)\n\t" \
  "v_mfma_f32_16x16x128_f8f6f4 v[60:63], v[152:159], v[216:223], v[60:63] blgp:1\n\t" \
  "buffer_load_dwordx4 v252, s[40:43], s52 offen offset:2048 sc0 sc1 lds\n\t" \
  "v_mfma_f32_16x16x128_f8f6f4 v[56:59], v[152:159], v[208:215], v[56:59] blgp:1\n\t" \
  "v_mfma_f32_16x16x128_f8f6f4 v[52:55], v[152:159], v[200:207], v[52:55] blgp:1\n\t" \
  "buffer_load_dwordx4 v252, s[40:43], s52 offen offset:3072 sc0 sc1 lds\n\t" \
  "v_mfma_f32_16x16x128_f8f6f4 v[48:51], v[152:159], v[192:199], v[48:51] blgp:1\n\t" \
  "s_waitcnt vmcnt(18) lgkmcnt(0)\n\t" \
  "s_barrier\n\t" \
  "v_mfma_f32_16x16x128_f8f6f4 a[0:3], v[128:135], v[160:167], a[0:3] blgp:1\n\t" \
  "v_readfirstlane_b32 s87, v254\n\t" \
  "v_readlane_b32 s65, v255, s67\n\t" \
  "v_mfma_f32_16x16x128_f8f6f4 a[4:7], v[128:135], v[168:175], a[4:7] blgp:1\n\t" \
  "v_mfma_f32_16x16x128_f8f6f4 a[8:11], v[128:135], v[176:183], a[8:11] blgp:1\n\t" \
  "s_max_u32 s69, s69, s87\n\t" \
  "s_add_u32 s67, s67, 1\n\t" \
  "s_cmp_ge_u32 s69, s65\n\t" \
  "s_cbranch_scc0 L_w24_%=\n\t" \
  "L_ok24_%=:\n\t" \
  "v_mfma_f32_16x16x128_f8f6f4 a[12:15], v[128:135], v[184:191], a[12:15] blgp:1\n\t" \
  "ds_read_b128 v[128:131], v250 offset:49152\n\t" \
  "v_mfma_f32_16x16x128_f8f6f4 a[28:31], v[136:143], v[184:191], a[28:31] blgp:1\n\t" \
  "s_add_i32 s52, s48, 32768\n\t" \
  "s_add_u32 m0, s53, 65536\n\t" \
  "ds_read_b128 v[132:135], v250 offset:57344\n\t" \
  "v_mfma_f32_16x16x128_f8f6f4 a[24:27], v[136:143], v[176:183], a[24:27] blgp:1\n\t" \
  "buffer_load_dwordx4 v252, s[40:43], s52 offen offset:0 sc0 sc1 lds\n\t" \
  "v_mfma_f32_16x16x128_f8f6f4 a[20:23], v[136:143], v[168:175], a[20:23] blgp:1\n\t" \
  "buffer_load_dwordx4 v252, s[40:43], s52 offen offset:1024 sc0 sc1 lds\n\t" \
  "v_mfma_f32_16x16x128_f8f6f4 a[16:19], v[136:143], v[160:167], a[16:19] blgp:1\n\t" \
  "ds_read_b128 v[136:139], v250 offset:49408\n\t" \
  "v_mfma_f32_16x16x128_f8f6f4 a[32:35], v[144:151], v[160:167], a[32:35] blgp:1\n\t" \
  "ds_read_b128 v[140:143], v250 offset:57600\n\t" \
  "v_mfma_f32_16x16x128_f8f6f4 a[36:39], v[144:151], v[168:175], a[36:39] blgp:1\n\t" \
  "buffer_load_dwordx4 v252, s[40:43], s52 offen offset:2048 sc0 sc1 lds\n\t" \
  "v_mfma_f32_16x16x128_f8f6f4 a[40:43], v[144:151], v[176:183], a[40:43] blgp:1\n\t" \
  "buffer_load_dwordx4 v252, s[40:43], s52 offen offset:3072 sc0 sc1 lds\n\t" \
  "v_mfma_f32_16x16x128_f8f6f4 a[44:47], v[144:151], v[184:191], a[44:47] blgp:1\n\t" \
  "ds_read_b128 v[144:147], v250 offset:49664\n\t" \
  "v_mfma_f32_16x16x128_f8f6f4 a[60:63], v[152:159], v[184:191], a[60:63] blgp:1\n\t" \
  "ds_read_b128 v[148:151], v250 offset:57856\n\t" \
  "v_mfma_f32_16x16x128_f8f6f4 a[56:59], v[152:159], v[176:183], a[56:59] blgp:1\n\t" \
  "v_mfma_f32_16x16x128_f8f6f4 a[52:55], v[152:159], v[168:175], a[52:55] blgp:1\n\t" \
  "v_mfma_f32_16x16x128_f8f6f4 a[48:51], v[152:159], v[160:167], a[48:51] blgp:1\n\t" \
  "ds_read_b128 v[152:155], v250 offset:49920\n\t" \
  "ds_read_b128 v[156:159], v250 offset:58112\n\t" \
  "s_waitcnt lgkmcnt(2)\n\t" \
  "v_mfma_f32_16x16x128_f8f6f4 v[64:67], v[128:135], v[160:167], v[64:67] blgp:1\n\t" \
  "s_add_i32 s52, s49, 49152\n\t" \
  "s_add_u32 m0, s53, 147456\n\t" \
  "v_mfma_f32_16x16x128_f8f6f4 v[68:71], v[128:135], v[168:175], v[68:71] blgp:1\n\t" \
  "buffer_load_dwordx4 v252, s[40:43], s52 offen offset:0 sc0 sc1 lds\n\t" \
  "v_mfma_f32_16x16x128_f8f6f4 v[72:75], v[128:135], v[176:183], v[72:75] blgp:1\n\t" \
  "buffer_load_dwordx4 v252, s[40:43], s52 offen offset:1024 sc0 sc1 lds\n\t" \
  "v_mfma_f32_16x16x128_f8f6f4 v[76:79], v[128:135], v[184:191], v[76:79] blgp:1\n\t" \
  "v_mfma_f32_16x16x128_f8f6f4 v[92:95], v[136:143], v[184:191], v[92:95] blgp:1\n\t" \
  "buffer_load_dwordx4 v252, s[40:43], s52 offen offset:2048 sc0 sc1 lds\n\t" \
  "v_mfma_f32_16x16x128_f8f6f4 v[88:91], v[136:143], v[176:183], v[88:91] blgp:1\n\t" \
  "buffer_load_dwordx4 v252, s[40:43], s52 offen offset:3072 sc0 sc1 lds\n\t" \
  "v_mfma_f32_16x16x128_f8f6f4 v[84:87], v[136:143], v[168:175], v[84:87] blgp:1\n\t" \
  "s_add_i32 s52, s49, 32768\n\t" \
  "s_add_u32 m0, s53, 131072\n\t" \
  "v_mfma_f32_16x16x128_f8f6f4 v[80:83], v[136:143], v[160:167], v[80:83] blgp:1\n\t" \
  "buffer_load_dwordx4 v252, s[40:43], s52 offen offset:0 sc0 sc1 lds\n\t" \
  "v_mfma_f32_16x16x128_f8f6f4 v[96:99], v[144:151], v[160:167], v[96:99] blgp:1\n\t" \
  "buffer_load_dwordx4 v252, s[40:43], s52 offen offset:1024 sc0 sc1 lds\n\t" \
  "v_mfma_f32_16x16x128_f8f6f4 v[100:103], v[144:151], v[168:175], v[100:103] blgp:1\n\t" \
  "v_mfma_f32_16x16x128_f8f6f4 v[104:107], v[144:151], v[176:183], v[104:107] blgp:1\n\t" \
  "buffer_load_dwordx4 v252, s[40:43], s52 offen offset:2048 sc0 sc1 lds\n\t" \
  "v_mfma_f32_16x16x128_f8f6f4 v[108:111], v[144:151], v[184:191], v[108:111] blgp:1\n\t" \
  "buffer_load_dwordx4 v252, s[40:43], s52 offen offset:3072 sc0 sc1 lds\n\t" \
  "s_waitcnt lgkmcnt(0)\n\t" \
  "v_mfma_f32_16x16x128_f8f6f4 v[124:127], v[152:159], v[184:191], v[124:127] blgp:1\n\t" \
  "v_mfma_f32_16x16x128_f8f6f4 v[120:123], v[152:159], v[176:183], v[120:123] blgp:1\n\t" \
  "v_mfma_f32_16x16x128_f8f6f4 v[116:119], v[152:159], v[168:175], v[116:119] blgp:1\n\t" \
  "v_mfma_f32_16x16x128_f8f6f4 v[112:115], v[152:159], v[160:167], v[112:115] blgp:1\n\t" \
  "s_waitcnt vmcnt(17) lgkmcnt(0)\n\t" \
  "s_barrier\n\t" \
  "v_mfma_f32_16x16x128_f8f6f4 a[64:67], v[128:135], v[192:199], a[64:67] blgp:1\n\t" \
  "ds_read_b128 v[160:163], v251 offset:0\n\t" \
  "v_mfma_f32_16x16x128_f8f6f4 a[68:71], v[128:135], v[200:207], a[68:71] blgp:1\n\t" \
  "ds_read_b128 v[164:167], v251 offset:8192\n\t" \
  "v_mfma_f32_16x16x128_f8f6f4 a[72:75], v[128:135], v[208:215], a[72:75] blgp:1\n\t" \
  "ds_read_b128 v[168:171], v251 offset:256\n\t" \
  "v_mfma_f32_16x16x128_f8f6f4 a[76:79], v[128:135], v[216:223], a[76:79] blgp:1\n\t" \
  "ds_read_b128 v[128:131], v250 offset:0\n\t" \
  "ds_read_b128 v[132:135], v250 offset:8192\n\t" \
  "v_mfma_f32_16x16x128_f8f6f4 a[92:95], v[136:143], v[216:223], a[92:95] blgp:1\n\t" \
  "ds_read_b128 v[172:175], v251 offset:8448\n\t" \
  "v_mfma_f32_16x16x128_f8f6f4 a[88:91], v[136:143], v[208:215], a[88:91] blgp:1\n\t" \
  "ds_read_b128 v[176:179], v251 offset:512\n\t" \
  "v_mfma_f32_16x16x128_f8f6f4 a[84:87], v[136:143], v[200:207], a[84:87] blgp:1\n\t" \
  "ds_read_b128 v[180:183], v251 offset:8704\n\t" \
  "v_mfma_f32_16x16x128_f8f6f4 a[80:83], v[136:143], v[192:199], a[80:83] blgp:1\n\t" \
  "ds_read_b128 v[136:139], v250 offset:256\n\t" \
  "ds_read_b128 v[140:143], v250 offset:8448\n\t" \
  "v_mfma_f32_16x16x128_f8f6f4 a[96:99], v[144:151], v[192:199], a[96:99] blgp:1\n\t" \
  "ds_read_b128 v[184:187], v251 offset:768\n\t" \
  "v_mfma_f32_16x16x128_f8f6f4 a[100:103], v[144:151], v[200:207], a[100:103] blgp:1\n\t" \
  "ds_read_b128 v[188:191], v251 offset:8960\n\t" \
  "v_mfma_f32_16x16x128_f8f6f4 a[104:107], v[144:151], v[208:215], a[104:107] blgp:1\n\t" \
  "s_add_u32 s48, s48, 0x10000\n\t" \
  "s_add_u32 s49, s49, 0x10000\n\t" \
  "v_mfma_f32_16x16x128_f8f6f4 a[108:111], v[144:151], v[216:223], a[108:111] blgp:1\n\t" \
  "ds_read_b128 v[144:147], v250 offset:512\n\t" \
  "v_mfma_f32_16x16x128_f8f6f4 a[124:127], v[152:159], v[216:223], a[124:127] blgp:1\n\t" \
  "ds_read_b128 v[148:151], v250 offset:8704\n\t" \
  "v_mfma_f32_16x16x128_f8f6f4 a[120:123], v[152:159], v[208:215], a[120:123] blgp:1\n\t" \
  "v_mfma_f32_16x16x128_f8f6f4 a[116:119], v[152:159], v[200:207], a[116:119] blgp:1\n\t" \
  "v_mfma_f32_16x16x128_f8f6f4 a[112:115], v[152:159], v[192:199], a[112:115] blgp:1\n\t" \
  "ds_read_b128 v[152:155], v250 offset:768\n\t" \
  "ds_read_b128 v[156:159], v250 offset:8960\n\t" \
  "L_k_%=:\n\t" \
  "s_waitcnt lgkmcnt(2)\n\t" \
  "v_mfma_f32_16x16x128_f8f6f4 a[0:3], v[128:135], v[160:167], a[0:3] blgp:1\n\t" \
  "ds_read_b128 v[192:195], v251 offset:16384\n\t" \
  "v_mfma_f32_16x16x128_f8f6f4 a[4:7], v[128:135], v[168:175], a[4:7] blgp:1\n\t" \
  "ds_read_b128 v[196:199], v251 offset:24576\n\t" \
  "v_mfma_f32_16x16x128_f8f6f4 a[8:11], v[128:135], v[176:183], a[8:11] blgp:1\n\t" \
  "ds_read_b128 v[200:203], v251 offset:16640\n\t" \
  "v_mfma_f32_16x16x128_f8f6f4 a[12:15], v[128:135], v[184:191], a[12:15] blgp:1\n\t" \
  "ds_read_b128 v[204:207], v251 offset:24832\n\t" \
  "v_mfma_f32_16x16x128_f8f6f4 a[28:31], v[136:143], v[184:191], a[28:31] blgp:1\n\t" \
  "ds_read_b128 v[208:211], v251 offset:16896\n\t" \
  "v_mfma_f32_16x16x128_f8f6f4 a[24:27], v[136:143], v[176:183], a[24:27] blgp:1\n\t" \
  "ds_read_b128 v[212:215], v251 offset:25088\n\t" \
  "v_mfma_f32_16x16x128_f8f6f4 a[20:23], v[136:143], v[168:175], a[20:23] blgp:1\n\t" \
  "ds_read_b128 v[216:219], v251 offset:17152\n\t" \
  "v_mfma_f32_16x16x128_f8f6f4 a[16:19], v[136:143], v[160:167], a[16:19] blgp:1\n\t" \
  "ds_read_b128 v[220:223], v251 offset:25344\n\t" \
  "s_add_i32 s52, s48, -16384\n\t" \
  "s_add_u32 m0, s53, 81920\n\t" \
  "v_mfma_f32_16x16x128_f8f6f4 a[32:35], v[144:151], v[160:167], a[32:35] blgp:1\n\t" \
  "buffer_load_dwordx4 v252, s[40:43], s52 offen offset:0 sc0 sc1 lds\n\t" \
  "v_mfma_f32_16x16x128_f8f6f4 a[36:39], v[144:151], v[168:175], a[36:39] blgp:1\n\t" \
  "v_mfma_f32_16x16x128_f8f6f4 a[40:43], v[144:151], v[176:183], a[40:43] blgp:1\n\t" \
  "buffer_load_dwordx4 v252, s[40:43], s52 offen offset:1024 sc0 sc1 lds\n\t" \
  "v_mfma_f32_16x16x128_f8f6f4 a[44:47], v[144:151], v[184:191], a[44:47] blgp:1\n\t" \
  "s_waitcnt lgkmcnt(0)\n\t" \
  "v_mfma_f32_16x16x128_f8f6f4 a[60:63], v[152:159], v[184:191], a[60:63] blgp:1\n\t" \
  "buffer_load_dwordx4 v252, s[40:43], s52 offen offset:2048 sc0 sc1 lds\n\t" \
  "v_mfma_f32_16x16x128_f8f6f4 a[56:59], v[152:159], v[176:183], a[56:59] blgp:1\n\t" \
  "v_mfma_f32_16x16x128_f8f6f4 a[52:55], v[152:159], v[168:175], a[52:55] blgp:1\n\t" \
  "buffer_load_dwordx4 v252, s[40:43], s52 offen offset:3072 sc0 sc1 lds\n\t" \
  "buffer_load_dword v254, off, s[88:91], s81 sc1\n\t" \
  "v_mfma_f32_16x16x128_f8f6f4 a[48:51], v[152:159], v[160:167], a[48:51] blgp:1\n\t" \
  "s_waitcnt vmcnt(17) lgkmcnt(0)\n\t" \
  "s_barrier\n\t" \
  "v_mfma_f32_16x16x128_f8f6f4 v[0:3], v[128:135], v[192:199], v[0:3] blgp:1\n\t" \
  "v_readlane_b32 s65, v255, s67\n\t" \
  "v_mfma_f32_16x16x128_f8f6f4 v[4:7], v[128:135], v[200:207], v[4:7] blgp:1\n\t" \
  "v_mfma_f32_16x16x128_f8f6f4 v[8:11], v[128:135], v[208:215], v[8:11] blgp:1\n\t" \
  "s_add_u32 s67, s67, 1\n\t" \
  "s_cmp_ge_u32 s69, s65\n\t" \
  "s_cbranch_scc0 L_w25_%=\n\t" \
  "L_ok25_%=:\n\t" \
  "v_mfma_f32_16x16x128_f8f6f4 v[12:15], v[128:135], v[216:223], v[12:15] blgp:1\n\t" \
  "ds_read_b128 v[128:131], v250 offset:16384\n\t" \
  "v_mfma_f32_16x16x128_f8f6f4 v[28:31], v[136:143], v[216:223], v[28:31] blgp:1\n\t" \
  "s_add_i32 s52, s48, 0\n\t" \
  "s_add_u32 m0, s53, 32768\n\t" \
  "ds_read_b128 v[132:135], v250 offset:24576\n\t" \
  "v_mfma_f32_16x16x128_f8f6f4 v[24:27], v[136:143], v[208:215], v[24:27] blgp:1\n\t" \
  "buffer_load_dwordx4 v252, s[40:43], s52 offen offset:0 sc0 sc1 lds\n\t" \
  "v_mfma_f32_16x16x128_f8f6f4 v[20:23], v[136:143], v[200:207], v[20:23] blgp:1\n\t" \
  "buffer_load_dwordx4 v252, s[40:43], s52 offen offset:1024 sc0 sc1 lds\n\t" \
  "v_mfma_f32_16x16x128_f8f6f4 v[16:19], v[136:143], v[192:199], v[16:19] blgp:1\n\t" \
  "ds_read_b128 v[136:139], v250 offset:16640\n\t" \
  "v_mfma_f32_16x16x128_f8f6f4 v[32:35], v[144:151], v[192:199], v[32:35] blgp:1\n\t" \
  "ds_read_b128 v[140:143], v250 offset:24832\n\t" \
  "v_mfma_f32_16x16x128_f8f6f4 v[36:39], v[144:151], v[200:207], v[36:39] blgp:1\n\t" \
  "buffer_load_dwordx4 v252, s[40:43], s52 offen offset:2048 sc0 sc1 lds\n\t" \
  "v_mfma_f32_16x16x128_f8f6f4 v[40:43], v[144:151], v[208:215], v[40:43] blgp:1\n\t" \
  "buffer_load_dwordx4 v252, s[40:43], s52 offen offset:3072 sc0 sc1 lds\n\t" \
  "v_mfma_f32_16x16x128_f8f6f4 v[44:47], v[144:151], v[216:223], v[44:47] blgp:1\n\t" \
  "ds_read_b128 v[144:147], v250 offset:16896\n\t" \
  "v_mfma_f32_16x16x128_f8f6f4 v[60:63], v[152:159], v[216:223], v[60:63] blgp:1\n\t" \
  "ds_read_b128 v[148:151], v250 offset:25088\n\t" \
  "v_mfma_f32_16x16x128_f8f6f4 v[56:59], v[152:159], v[208:215], v[56:59] blgp:1\n\t" \
  "v_mfma_f32_16x16x128_f8f6f4 v[52:55], v[152:159], v[200:207], v[52:55] blgp:1\n\t" \
  "v_mfma_f32_16x16x128_f8f6f4 v[48:51], v[152:159], v[192:199], v[48:51] blgp:1\n\t" \
  "ds_read_b128 v[152:155], v250 offset:17152\n\t" \
  "ds_read_b128 v[156:159], v250 offset:25344\n\t" \
  "s_waitcnt lgkmcnt(2)\n\t" \
  "v_mfma_f32_16x16x128_f8f6f4 a[64:67], v[128:135], v[192:199], a[64:67] blgp:1\n\t" \
  "s_add_i32 s52, s49, 0\n\t" \
  "s_add_u32 m0, s53, 98304\n\t" \
  "v_mfma_f32_16x16x128_f8f6f4 a[68:71], v[128:135], v[200:207], a[68:71] blgp:1\n\t" \
  "buffer_load_dwordx4 v252, s[40:43], s52 offen offset:0 sc0 sc1 lds\n\t" \
  "v_mfma_f32_16x16x128_f8f6f4 a[72:75], v[128:135], v[208:215], a[72:75] blgp:1\n\t" \
  "buffer_load_dwordx4 v252, s[40:43], s52 offen offset:1024 sc0 sc1 lds\n\t" \
  "v_mfma_f32_16x16x128_f8f6f4 a[76:79], v[128:135], v[216:223], a[76:79] blgp:1\n\t" \
  "v_mfma_f32_16x16x128_f8f6f4 a[92:95], v[136:143], v[216:223], a[92:95] blgp:1\n\t" \
  "buffer_load_dwordx4 v252, s[40:43], s52 offen offset:2048 sc0 sc1 lds\n\t" \
  "v_mfma_f32_16x16x128_f8f6f4 a[88:91], v[136:143], v[208:215], a[88:91] blgp:1\n\t" \
  "buffer_load_dwordx4 v252, s[40:43], s52 offen offset:3072 sc0 sc1 lds\n\t" \
  "v_mfma_f32_16x16x128_f8f6f4 a[84:87], v[136:143], v[200:207], a[84:87] blgp:1\n\t" \
  "s_add_i32 s52, s49, 16384\n\t" \
  "s_add_u32 m0, s53, 114688\n\t" \
  "v_mfma_f32_16x16x128_f8f6f4 a[80:83], v[136:143], v[192:199], a[80:83] blgp:1\n\t" \
  "buffer_load_dwordx4 v252, s[40:43], s52 offen offset:0 sc0 sc1 lds\n\t" \
  "v_mfma_f32_16x16x128_f8f6f4 a[96:99], v[144:151], v[192:199], a[96:99] blgp:1\n\t" \
  "buffer_load_dwordx4 v252, s[40:43], s52 offen offset:1024 sc0 sc1 lds\n\t" \
  "v_mfma_f32_16x16x128_f8f6f4 a[100:103], v[144:151], v[200:207], a[100:103] blgp:1\n\t" \
  "v_mfma_f32_16x16x128_f8f6f4 a[104:107], v[144:151], v[208:215], a[104:107] blgp:1\n\t" \
  "buffer_load_dwordx4 v252, s[40:43], s52 offen offset:2048 sc0 sc1 lds\n\t" \
  "v_mfma_f32_16x16x128_f8f6f4 a[108:111], v[144:151], v[216:223], a[108:111] blgp:1\n\t" \
  "buffer_load_dwordx4 v252, s[40:43], s52 offen offset:3072 sc0 sc1 lds\n\t" \
  "s_waitcnt lgkmcnt(0)\n\t" \
  "v_mfma_f32_16x16x128_f8f6f4 a[124:127], v[152:159], v[216:223], a[124:127] blgp:1\n\t" \
  "v_mfma_f32_16x16x128_f8f6f4 a[120:123], v[152:159], v[208:215], a[120:123] blgp:1\n\t" \
  "v_mfma_f32_16x16x128_f8f6f4 a[116:119], v[152:159], v[200:207], a[116:119] blgp:1\n\t" \
  "v_mfma_f32_16x16x128_f8f6f4 a[112:115], v[152:159], v[192:199], a[112:115] blgp:1\n\t" \
  "s_waitcnt vmcnt(17) lgkmcnt(0)\n\t" \
  "s_barrier\n\t" \
  "v_mfma_f32_16x16x128_f8f6f4 v[64:67], v[128:135], v[160:167], v[64:67] blgp:1\n\t" \
  "ds_read_b128 v[192:195], v251 offset:49152\n\t" \
  "v_mfma_f32_16x16x128_f8f6f4 v[68:71], v[128:135], v[168:175], v[68:71] blgp:1\n\t" \
  "ds_read_b128 v[196:199], v251 offset:57344\n\t" \
  "v_mfma_f32_16x16x128_f8f6f4 v[72:75], v[128:135], v[176:183], v[72:75] blgp:1\n\t" \
  "ds_read_b128 v[200:203], v251 offset:49408\n\t" \
  "v_mfma_f32_16x16x128_f8f6f4 v[76:79], v[128:135], v[184:191], v[76:79] blgp:1\n\t" \
  "ds_read_b128 v[128:131], v250 offset:32768\n\t" \
  "ds_read_b128 v[132:135], v250 offset:40960\n\t" \
  "v_mfma_f32_16x16x128_f8f6f4 v[92:95], v[136:143], v[184:191], v[92:95] blgp:1\n\t" \
  "ds_read_b128 v[204:207], v251 offset:57600\n\t" \
  "v_mfma_f32_16x16x128_f8f6f4 v[88:91], v[136:143], v[176:183], v[88:91] blgp:1\n\t" \
  "ds_read_b128 v[208:211], v251 offset:49664\n\t" \
  "v_mfma_f32_16x16x128_f8f6f4 v[84:87], v[136:143], v[168:175], v[84:87] blgp:1\n\t" \
  "ds_read_b128 v[212:215], v251 offset:57856\n\t" \
  "v_mfma_f32_16x16x128_f8f6f4 v[80:83], v[136:143], v[160:167], v[80:83] blgp:1\n\t" \
  "ds_read_b128 v[136:139], v250 offset:33024\n\t" \
  "ds_read_b128 v[140:143], v250 offset:41216\n\t" \
  "v_mfma_f32_16x16x128_f8f6f4 v[96:99], v[144:151], v[160:167], v[96:99] blgp:1\n\t" \
  "ds_read_b128 v[216:219], v251 offset:49920\n\t" \
  "v_mfma_f32_16x16x128_f8f6f4 v[100:103], v[144:151], v[168:175], v[100:103] blgp:1\n\t" \
  "ds_read_b128 v[220:223], v251 offset:58112\n\t" \
  "v_mfma_f32_16x16x128_f8f6f4 v[104:107], v[144:151], v[176:183], v[104:107] blgp:1\n\t" \
  "v_mfma_f32_16x16x128_f8f6f4 v[108:111], v[144:151], v[184:191], v[108:111] blgp:1\n\t" \
  "ds_read_b128 v[144:147], v250 offset:33280\n\t" \
  "v_mfma_f32_16x16x128_f8f6f4 v[124:127], v[152:159], v[184:191], v[124:127] blgp:1\n\t" \
  "ds_read_b128 v[148:151], v250 offset:41472\n\t" \
  "v_mfma_f32_16x16x128_f8f6f4 v[120:123], v[152:159], v[176:183], v[120:123] blgp:1\n\t" \
  "v_mfma_f32_16x16x128_f8f6f4 v[116:119], v[152:159], v[168:175], v[116:119] blgp:1\n\t" \
  "v_mfma_f32_16x16x128_f8f6f4 v[112:115], v[152:159], v[160:167], v[112:115] blgp:1\n\t" \
  "ds_read_b128 v[152:155], v250 offset:33536\n\t" \
  "ds_read_b128 v[156:159], v250 offset:41728\n\t" \
  "s_waitcnt lgkmcnt(2)\n\t" \
  "v_mfma_f32_16x16x128_f8f6f4 v[0:3], v[128:135], v[192:199], v[0:3] blgp:1\n\t" \
  "ds_read_b128 v[160:163], v251 offset:32768\n\t" \
  "v_mfma_f32_16x16x128_f8f6f4 v[4:7], v[128:135], v[200:207], v[4:7] blgp:1\n\t" \
  "ds_read_b128 v[164:167], v251 offset:40960\n\t" \
  "v_mfma_f32_16x16x128_f8f6f4 v[8:11], v[128:135], v[208:215], v[8:11] blgp:1\n\t" \
  "ds_read_b128 v[168:171], v251 offset:33024\n\t" \
  "v_mfma_f32_16x16x128_f8f6f4 v[12:15], v[128:135], v[216:223], v[12:15] blgp:1\n\t" \
  "ds_read_b128 v[172:175], v251 offset:41216\n\t" \
  "v_mfma_f32_16x16x128_f8f6f4 v[28:31], v[136:143], v[216:223], v[28:31] blgp:1\n\t" \
  "ds_read_b128 v[176:179], v251 offset:33280\n\t" \
  "v_mfma_f32_16x16x128_f8f6f4 v[24:27], v[136:143], v[208:215], v[24:27] blgp:1\n\t" \
  "ds_read_b128 v[180:183], v251 offset:41472\n\t" \
  "v_mfma_f32_16x16x128_f8f6f4 v[20:23], v[136:143], v[200:207], v[20:23] blgp:1\n\t" \
  "ds_read_b128 v[184:187], v251 offset:33536\n\t" \
  "v_mfma_f32_16x16x128_f8f6f4 v[16:19], v[136:143], v[192:199], v[16:19] blgp:1\n\t" \
  "ds_read_b128 v[188:191], v251 offset:41728\n\t" \
  "s_add_i32 s52, s48, 16384\n\t" \
  "s_add_u32 m0, s53, 49152\n\t" \
  "v_mfma_f32_16x16x128_f8f6f4 v[32:35], v[144:151], v[192:199], v[32:35] blgp:1\n\t" \
  "buffer_load_dwordx4 v252, s[40:43], s52 offen offset:0 sc0 sc1 lds\n\t" \
  "v_mfma_f32_16x16x128_f8f6f4 v[36:39], v[144:151], v[200:207], v[36:39] blgp:1\n\t" \
  "v_mfma_f32_16x16x128_f8f6f4 v[40:43], v[144:151], v[208:215], v[40:43] blgp:1\n\t" \
  "buffer_load_dwordx4 v252, s[40:43], s52 offen offset:1024 sc0 sc1 lds\n\t" \
  "v_mfma_f32_16x16x128_f8f6f4 v[44:47], v[144:151], v[216:223], v[44:47] blgp:1\n\t" \
  "s_waitcnt lgkmcnt(0)\n\t" \
  "v_mfma_f32_16x16x128_f8f6f4 v[60:63], v[152:159], v[216:223], v[60:63] blgp:1\n\t" \
  "buffer_load_dwordx4 v252, s[40:43], s52 offen offset:2048 sc0 sc1 lds\n\t" \
  "v_mfma_f32_16x16x128_f8f6f4 v[56:59], v[152:159], v[208:215], v[56:59] blgp:1\n\t" \
  "v_mfma_f32_16x16x128_f8f6f4 v[52:55], v[152:159], v[200:207], v[52:55] blgp:1\n\t" \
  "buffer_load_dwordx4 v252, s[40:43], s52 offen offset:3072 sc0 sc1 lds\n\t" \
  "v_mfma_f32_16x16x128_f8f6f4 v[48:51], v[152:159], v[192:199], v[48:51] blgp:1\n\t" \
  "s_waitcnt vmcnt(16) lgkmcnt(0)\n\t" \
  "s_barrier\n\t" \
  "v_mfma_f32_16x16x128_f8f6f4 a[0:3], v[128:135], v[160:167], a[0:3] blgp:1\n\t" \
  "v_readfirstlane_b32 s87, v254\n\t" \
  "v_readlane_b32 s65, v255, s67\n\t" \
  "v_mfma_f32_16x16x128_f8f6f4 a[4:7], v[128:135], v[168:175], a[4:7] blgp:1\n\t" \
  "v_mfma_f32_16x16x128_f8f6f4 a[8:11], v[128:135], v[176:183], a[8:11] blgp:1\n\t" \
  "s_max_u32 s69, s69, s87\n\t" \
  "s_add_u32 s67, s67, 1\n\t" \
  "s_cmp_ge_u32 s69, s65\n\t" \
  "s_cbranch_scc0 L_w26_%=\n\t" \
  "L_ok26_%=:\n\t" \
  "v_mfma_f32_16x16x128_f8f6f4 a[12:15], v[128:135], v[184:191], a[12:15] blgp:1\n\t" \
  "ds_read_b128 v[128:131], v250 offset:49152\n\t" \
  "v_mfma_f32_16x16x128_f8f6f4 a[28:31], v[136:143], v[184:191], a[28:31] blgp:1\n\t" \
  "s_add_i32 s52, s48, 32768\n\t" \
  "s_add_u32 m0, s53, 65536\n\t" \
  "ds_read_b128 v[132:135], v250 offset:57344\n\t" \
  "v_mfma_f32_16x16x128_f8f6f4 a[24:27], v[136:143], v[176:183], a[24:27] blgp:1\n\t" \
  "buffer_load_dwordx4 v252, s[40:43], s52 offen offset:0 sc0 sc1 lds\n\t" \
  "v_mfma_f32_16x16x128_f8f6f4 a[20:23], v[136:143], v[168:175], a[20:23] blgp:1\n\t" \
  "buffer_load_dwordx4 v252, s[40:43], s52 offen offset:1024 sc0 sc1 lds\n\t" \
  "v_mfma_f32_16x16x128_f8f6f4 a[16:19], v[136:143], v[160:167], a[16:19] blgp:1\n\t" \
  "ds_read_b128 v[136:139], v250 offset:49408\n\t" \
  "v_mfma_f32_16x16x128_f8f6f4 a[32:35], v[144:151], v[160:167], a[32:35] blgp:1\n\t" \
  "ds_read_b128 v[140:143], v250 offset:57600\n\t" \
  "v_mfma_f32_16x16x128_f8f6f4 a[36:39], v[144:151], v[168:175], a[36:39] blgp:1\n\t" \
  "buffer_load_dwordx4 v252, s[40:43], s52 offen offset:2048 sc0 sc1 lds\n\t" \
  "v_mfma_f32_16x16x128_f8f6f4 a[40:43], v[144:151], v[176:183], a[40:43] blgp:1\n\t" \
  "buffer_load_dwordx4 v252, s[40:43], s52 offen offset:3072 sc0 sc1 lds\n\t" \
  "v_mfma_f32_16x16x128_f8f6f4 a[44:47], v[144:151], v[184:191], a[44:47] blgp:1\n\t" \
  "ds_read_b128 v[144:147], v250 offset:49664\n\t" \
  "v_mfma_f32_16x16x128_f8f6f4 a[60:63], v[152:159], v[184:191], a[60:63] blgp:1\n\t" \
  "ds_read_b128 v[148:151], v250 offset:57856\n\t" \
  "v_mfma_f32_16x16x128_f8f6f4 a[56:59], v[152:159], v[176:183], a[56:59] blgp:1\n\t" \
  "v_mfma_f32_16x16x128_f8f6f4 a[52:55], v[152:159], v[168:175], a[52:55] blgp:1\n\t" \
  "v_mfma_f32_16x16x128_f8f6f4 a[48:51], v[152:159], v[160:167], a[48:51] blgp:1\n\t" \
  "ds_read_b128 v[152:155], v250 offset:49920\n\t" \
  "ds_read_b128 v[156:159], v250 offset:58112\n\t" \
  "s_waitcnt lgkmcnt(2)\n\t" \
  "v_mfma_f32_16x16x128_f8f6f4 v[64:67], v[128:135], v[160:167], v[64:67] blgp:1\n\t" \
  "s_add_i32 s52, s49, 49152\n\t" \
  "s_add_u32 m0, s53, 147456\n\t" \
  "v_mfma_f32_16x16x128_f8f6f4 v[68:71], v[128:135], v[168:175], v[68:71] blgp:1\n\t" \
  "buffer_load_dwordx4 v252, s[40:43], s52 offen offset:0 sc0 sc1 lds\n\t" \
  "v_mfma_f32_16x16x128_f8f6f4 v[72:75], v[128:135], v[176:183], v[72:75] blgp:1\n\t" \
  "buffer_load_dwordx4 v252, s[40:43], s52 offen offset:1024 sc0 sc1 lds\n\t" \
  "v_mfma_f32_16x16x128_f8f6f4 v[76:79], v[128:135], v[184:191], v[76:79] blgp:1\n\t" \
  "v_mfma_f32_16x16x128_f8f6f4 v[92:95], v[136:143], v[184:191], v[92:95] blgp:1\n\t" \
  "buffer_load_dwordx4 v252, s[40:43], s52 offen offset:2048 sc0 sc1 lds\n\t" \
  "v_mfma_f32_16x16x128_f8f6f4 v[88:91], v[136:143], v[176:183], v[88:91] blgp:1\n\t" \
  "buffer_load_dwordx4 v252, s[40:43], s52 offen offset:3072 sc0 sc1 lds\n\t" \
  "v_mfma_f32_16x16x128_f8f6f4 v[84:87], v[136:143], v[168:175], v[84:87] blgp:1\n\t" \
  "s_add_i32 s52, s49, 32768\n\t" \
  "s_add_u32 m0, s53, 131072\n\t" \
  "v_mfma_f32_16x16x128_f8f6f4 v[80:83], v[136:143], v[160:167], v[80:83] blgp:1\n\t" \
  "buffer_load_dwordx4 v252, s[40:43], s52 offen offset:0 sc0 sc1 lds\n\t" \
  "v_mfma_f32_16x16x128_f8f6f4 v[96:99], v[144:151], v[160:167], v[96:99] blgp:1\n\t" \
  "buffer_load_dwordx4 v252, s[40:43], s52 offen offset:1024 sc0 sc1 lds\n\t" \
  "v_mfma_f32_16x16x128_f8f6f4 v[100:103], v[144:151], v[168:175], v[100:103] blgp:1\n\t" \
  "v_mfma_f32_16x16x128_f8f6f4 v[104:107], v[144:151], v[176:183], v[104:107] blgp:1\n\t" \
  "buffer_load_dwordx4 v252, s[40:43], s52 offen offset:2048 sc0 sc1 lds\n\t" \
  "v_mfma_f32_16x16x128_f8f6f4 v[108:111], v[144:151], v[184:191], v[108:111] blgp:1\n\t" \
  "buffer_load_dwordx4 v252, s[40:43], s52 offen offset:3072 sc0 sc1 lds\n\t" \
  "s_waitcnt lgkmcnt(0)\n\t" \
  "v_mfma_f32_16x16x128_f8f6f4 v[124:127], v[152:159], v[184:191], v[124:127] blgp:1\n\t" \
  "v_mfma_f32_16x16x128_f8f6f4 v[120:123], v[152:159], v[176:183], v[120:123] blgp:1\n\t" \
  "v_mfma_f32_16x16x128_f8f6f4 v[116:119], v[152:159], v[168:175], v[116:119] blgp:1\n\t" \
  "v_mfma_f32_16x16x128_f8f6f4 v[112:115], v[152:159], v[160:167], v[112:115] blgp:1\n\t" \
  "s_waitcnt vmcnt(16) lgkmcnt(0)\n\t" \
  "s_barrier\n\t" \
  "v_mfma_f32_16x16x128_f8f6f4 a[64:67], v[128:135], v[192:199], a[64:67] blgp:1\n\t" \
  "ds_read_b128 v[160:163], v251 offset:0\n\t" \
  "v_mfma_f32_16x16x128_f8f6f4 a[68:71], v[128:135], v[200:207], a[68:71] blgp:1\n\t" \
  "ds_read_b128 v[164:167], v251 offset:8192\n\t" \
  "v_mfma_f32_16x16x128_f8f6f4 a[72:75], v[128:135], v[208:215], a[72:75] blgp:1\n\t" \
  "ds_read_b128 v[168:171], v251 offset:256\n\t" \
  "v_mfma_f32_16x16x128_f8f6f4 a[76:79], v[128:135], v[216:223], a[76:79] blgp:1\n\t" \
  "ds_read_b128 v[128:131], v250 offset:0\n\t" \
  "ds_read_b128 v[132:135], v250 offset:8192\n\t" \
  "v_mfma_f32_16x16x128_f8f6f4 a[92:95], v[136:143], v[216:223], a[92:95] blgp:1\n\t" \
  "ds_read_b128 v[172:175], v251 offset:8448\n\t" \
  "v_mfma_f32_16x16x128_f8f6f4 a[88:91], v[136:143], v[208:215], a[88:91] blgp:1\n\t" \
  "ds_read_b128 v[176:179], v251 offset:512\n\t" \
  "v_mfma_f32_16x16x128_f8f6f4 a[84:87], v[136:143], v[200:207], a[84:87] blgp:1\n\t" \
  "ds_read_b128 v[180:183], v251 offset:8704\n\t" \
  "v_mfma_f32_16x16x128_f8f6f4 a[80:83], v[136:143], v[192:199], a[80:83] blgp:1\n\t" \
  "ds_read_b128 v[136:139], v250 offset:256\n\t" \
  "ds_read_b128 v[140:143], v250 offset:8448\n\t" \
  "v_mfma_f32_16x16x128_f8f6f4 a[96:99], v[144:151], v[192:199], a[96:99] blgp:1\n\t" \
  "ds_read_b128 v[184:187], v251 offset:768\n\t" \
  "v_mfma_f32_16x16x128_f8f6f4 a[100:103], v[144:151], v[200:207], a[100:103] blgp:1\n\t" \
  "ds_read_b128 v[188:191], v251 offset:8960\n\t" \
  "v_mfma_f32_16x16x128_f8f6f4 a[104:107], v[144:151], v[208:215], a[104:107] blgp:1\n\t" \
  "s_add_u32 s48, s48, 0x10000\n\t" \
  "s_add_u32 s49, s49, 0x10000\n\t" \
  "v_mfma_f32_16x16x128_f8f6f4 a[108:111], v[144:151], v[216:223], a[108:111] blgp:1\n\t" \
  "ds_read_b128 v[144:147], v250 offset:512\n\t" \
  "v_mfma_f32_16x16x128_f8f6f4 a[124:127], v[152:159], v[216:223], a[124:127] blgp:1\n\t" \
  "ds_read_b128 v[148:151], v250 offset:8704\n\t" \
  "v_mfma_f32_16x16x128_f8f6f4 a[120:123], v[152:159], v[208:215], a[120:123] blgp:1\n\t" \
  "s_sub_u32 s54, s54, 1\n\t" \
  "s_cmp_lg_u32 s54, 0\n\t" \
  "v_mfma_f32_16x16x128_f8f6f4 a[116:119], v[152:159], v[200:207], a[116:119] blgp:1\n\t" \
  "v_mfma_f32_16x16x128_f8f6f4 a[112:115], v[152:159], v[192:199], a[112:115] blgp:1\n\t" \
  "ds_read_b128 v[152:155], v250 offset:768\n\t" \
  "ds_read_b128 v[156:159], v250 offset:8960\n\t" \
  "s_cbranch_scc1 L_k_%=\n\t" \
  "s_waitcnt lgkmcnt(2)\n\t" \
  "v_mfma_f32_16x16x128_f8f6f4 a[0:3], v[128:135], v[160:167], a[0:3] blgp:1\n\t" \
  "ds_read_b128 v[192:195], v251 offset:16384\n\t" \
  "s_and_b32 s99, s98, 0x400\n\t" \
  "s_cselect_b32 s99, 1, s55\n\t" \
  "v_mfma_f32_16x16x128_f8f6f4 a[4:7], v[128:135], v[168:175], a[4:7] blgp:1\n\t" \
  "ds_read_b128 v[196:199], v251 offset:24576\n\t" \
  "s_cmp_eq_u32 s99, 1\n\t" \
  "s_cselect_b32 s99, L_tail_%=-L_tile_%=, 0\n\t" \
  "s_cselect_b32 s96, 0, 1\n\t" \
  "s_cselect_b32 s36, 0, s36\n\t" \
  "v_mfma_f32_16x16x128_f8f6f4 a[8:11], v[128:135], v[176:183], a[8:11] blgp:1\n\t" \
  "ds_read_b128 v[200:203], v251 offset:16640\n\t" \
  "v_and_b32 v240, 0x3ff, v252\n\t" \
  "v_mfma_f32_16x16x128_f8f6f4 a[12:15], v[128:135], v[184:191], a[12:15] blgp:1\n\t" \
  "ds_read_b128 v[204:207], v251 offset:24832\n\t" \
  "v_lshl_add_u32 v240, v240, 2, s53\n\t" \
  "v_mfma_f32_16x16x128_f8f6f4 a[28:31], v[136:143], v[184:191], a[28:31] blgp:1\n\t" \
  "ds_read_b128 v[208:211], v251 offset:16896\n\t" \
  "v_lshlrev_b32 v240, s96, v240\n\t" \
  "v_mfma_f32_16x16x128_f8f6f4 a[24:27], v[136:143], v[176:183], a[24:27] blgp:1\n\t" \
  "ds_read_b128 v[212:215], v251 offset:25088\n\t" \
  "v_add_u32 v240, s99, v240\n\t" \
  "v_mfma_f32_16x16x128_f8f6f4 a[20:23], v[136:143], v[168:175], a[20:23] blgp:1\n\t" \
  "ds_read_b128 v[216:219], v251 offset:17152\n\t" \
  "v_min_u32 v240, L_end_%=-L_tile_%=-4, v240\n\t" \
  "v_mfma_f32_16x16x128_f8f6f4 a[16:19], v[136:143], v[160:167], a[16:19] blgp:1\n\t" \
  "ds_read_b128 v[220:223], v251 offset:25344\n\t" \
  "s_add_i32 s52, s48, -16384\n\t" \
  "s_add_u32 m0, s53, 81920\n\t" \
  "v_mfma_f32_16x16x128_f8f6f4 a[32:35], v[144:151], v[160:167], a[32:35] blgp:1\n\t" \
  "buffer_load_dwordx4 v252, s[40:43], s52 offen offset:0 sc0 sc1 lds\n\t" \
  "v_mfma_f32_16x16x128_f8f6f4 a[36:39], v[144:151], v[168:175], a[36:39] blgp:1\n\t" \
  "v_max_u32 v255, v248, v249\n\t" \
  "v_mfma_f32_16x16x128_f8f6f4 a[40:43], v[144:151], v[176:183], a[40:43] blgp:1\n\t" \
  "buffer_load_dwordx4 v252, s[40:43], s52 offen offset:1024 sc0 sc1 lds\n\t" \
  "v_mfma_f32_16x16x128_f8f6f4 a[44:47], v[144:151], v[184:191], a[44:47] blgp:1\n\t" \
  "v_add_u32 v255, s86, v255\n\t" \
  "s_mov_b32 s67, 0\n\t" \
  "s_waitcnt lgkmcnt(0)\n\t" \
  "v_mfma_f32_16x16x128_f8f6f4 a[60:63], v[152:159], v[184:191], a[60:63] blgp:1\n\t" \
  "buffer_load_dwordx4 v252, s[40:43], s52 offen offset:2048 sc0 sc1 lds\n\t" \
  "v_mfma_f32_16x16x128_f8f6f4 a[56:59], v[152:159], v[176:183], a[56:59] blgp:1\n\t" \
  "v_mfma_f32_16x16x128_f8f6f4 a[52:55], v[152:159], v[168:175], a[52:55] blgp:1\n\t" \
  "buffer_load_dwordx4 v252, s[40:43], s52 offen offset:3072 sc0 sc1 lds\n\t" \
  "buffer_load_dword v254, off, s[88:91], s81 sc1\n\t" \
  "v_mfma_f32_16x16x128_f8f6f4 a[48:51], v[152:159], v[160:167], a[48:51] blgp:1\n\t" \
  "s_mov_b32 s48, s50\n\t" \
  "s_mov_b32 s49, s51\n\t" \
  "s_waitcnt vmcnt(17) lgkmcnt(0)\n\t" \
  "s_barrier\n\t" \
  "v_mfma_f32_16x16x128_f8f6f4 v[0:3], v[128:135], v[192:199], v[0:3] blgp:1\n\t" \
  "v_readlane_b32 s65, v255, s67\n\t" \
  "v_mfma_f32_16x16x128_f8f6f4 v[4:7], v[128:135], v[200:207], v[4:7] blgp:1\n\t" \
  "v_mfma_f32_16x16x128_f8f6f4 v[8:11], v[128:135], v[208:215], v[8:11] blgp:1\n\t" \
  "s_add_u32 s67, s67, 1\n\t" \
  "s_cmp_ge_u32 s69, s65\n\t" \
  "s_cbranch_scc0 L_w27_%=\n\t" \
  "L_ok27_%=:\n\t" \
  "v_mfma_f32_16x16x128_f8f6f4 v[12:15], v[128:135], v[216:223], v[12:15] blgp:1\n\t" \
  "ds_read_b128 v[128:131], v250 offset:16384\n\t" \
  "v_mfma_f32_16x16x128_f8f6f4 v[28:31], v[136:143], v[216:223], v[28:31] blgp:1\n\t" \
  "s_add_i32 s52, s48, 0\n\t" \
  "s_add_u32 m0, s53, 32768\n\t" \
  "ds_read_b128 v[132:135], v250 offset:24576\n\t" \
  "v_mfma_f32_16x16x128_f8f6f4 v[24:27], v[136:143], v[208:215], v[24:27] blgp:1\n\t" \
  "buffer_load_dwordx4 v252, s[40:43], s52 offen offset:0 sc0 sc1 lds\n\t" \
  "v_mfma_f32_16x16x128_f8f6f4 v[20:23], v[136:143], v[200:207], v[20:23] blgp:1\n\t" \
  "buffer_load_dwordx4 v252, s[40:43], s52 offen offset:1024 sc0 sc1 lds\n\t" \
  "v_mfma_f32_16x16x128_f8f6f4 v[16:19], v[136:143], v[192:199], v[16:19] blgp:1\n\t" \
  "ds_read_b128 v[136:139], v250 offset:16640\n\t" \
  "v_mfma_f32_16x16x128_f8f6f4 v[32:35], v[144:151], v[192:199], v[32:35] blgp:1\n\t" \
  "ds_read_b128 v[140:143], v250 offset:24832\n\t" \
  "v_mfma_f32_16x16x128_f8f6f4 v[36:39], v[144:151], v[200:207], v[36:39] blgp:1\n\t" \
  "buffer_load_dwordx4 v252, s[40:43], s52 offen offset:2048 sc0 sc1 lds\n\t" \
  "v_mfma_f32_16x16x128_f8f6f4 v[40:43], v[144:151], v[208:215], v[40:43] blgp:1\n\t" \
  "buffer_load_dwordx4 v252, s[40:43], s52 offen offset:3072 sc0 sc1 lds\n\t" \
  "v_mfma_f32_16x16x128_f8f6f4 v[44:47], v[144:151], v[216:223], v[44:47] blgp:1\n\t" \
  "ds_read_b128 v[144:147], v250 offset:16896\n\t" \
  "v_mfma_f32_16x16x128_f8f6f4 v[60:63], v[152:159], v[216:223], v[60:63] blgp:1\n\t" \
  "ds_read_b128 v[148:151], v250 offset:25088\n\t" \
  "v_mfma_f32_16x16x128_f8f6f4 v[56:59], v[152:159], v[208:215], v[56:59] blgp:1\n\t" \
  "s_cmp_eq_u32 s36, 0\n\t" \
  "s_cbranch_scc1 L_ct_%=\n\t" \
  "L_ctr_%=:\n\t" \
  "v_mfma_f32_16x16x128_f8f6f4 v[52:55], v[152:159], v[200:207], v[52:55] blgp:1\n\t" \
  "v_mfma_f32_16x16x128_f8f6f4 v[48:51], v[152:159], v[192:199], v[48:51] blgp:1\n\t" \
  "ds_read_b128 v[152:155], v250 offset:17152\n\t" \
  "ds_read_b128 v[156:159], v250 offset:25344\n\t" \
  "s_waitcnt lgkmcnt(2)\n\t" \
  "v_mfma_f32_16x16x128_f8f6f4 a[64:67], v[128:135], v[192:199], a[64:67] blgp:1\n\t" \
  "s_add_i32 s52, s49, 0\n\t" \
  "s_add_u32 m0, s53, 98304\n\t" \
  "v_mfma_f32_16x16x128_f8f6f4 a[68:71], v[128:135], v[200:207], a[68:71] blgp:1\n\t" \
  "buffer_load_dwordx4 v252, s[40:43], s52 offen offset:0 sc0 sc1 lds\n\t" \
  "v_mfma_f32_16x16x128_f8f6f4 a[72:75], v[128:135], v[208:215], a[72:75] blgp:1\n\t" \
  "buffer_load_dwordx4 v252, s[40:43], s52 offen offset:1024 sc0 sc1 lds\n\t" \
  "v_mfma_f32_16x16x128_f8f6f4 a[76:79], v[128:135], v[216:223], a[76:79] blgp:1\n\t" \
  "v_mfma_f32_16x16x128_f8f6f4 a[92:95], v[136:143], v[216:223], a[92:95] blgp:1\n\t" \
  "buffer_load_dwordx4 v252, s[40:43], s52 offen offset:2048 sc0 sc1 lds\n\t" \
  "v_mfma_f32_16x16x128_f8f6f4 a[88:91], v[136:143], v[208:215], a[88:91] blgp:1\n\t" \
  "buffer_load_dwordx4 v252, s[40:43], s52 offen offset:3072 sc0 sc1 lds\n\t" \
  "v_mfma_f32_16x16x128_f8f6f4 a[84:87], v[136:143], v[200:207], a[84:87] blgp:1\n\t" \
  "s_add_i32 s52, s49, 16384\n\t" \
  "s_add_u32 m0, s53, 114688\n\t" \
  "v_mfma_f32_16x16x128_f8f6f4 a[80:83], v[136:143], v[192:199], a[80:83] blgp:1\n\t" \
  "buffer_load_dwordx4 v252, s[40:43], s52 offen offset:0 sc0 sc1 lds\n\t" \
  "v_mfma_f32_16x16x128_f8f6f4 a[96:99], v[144:151], v[192:199], a[96:99] blgp:1\n\t" \
  "buffer_load_dwordx4 v252, s[40:43], s52 offen offset:1024 sc0 sc1 lds\n\t" \
  "v_mfma_f32_16x16x128_f8f6f4 a[100:103], v[144:151], v[200:207], a[100:103] blgp:1\n\t" \
  "v_mfma_f32_16x16x128_f8f6f4 a[104:107], v[144:151], v[208:215], a[104:107] blgp:1\n\t" \
  "buffer_load_dwordx4 v252, s[40:43], s52 offen offset:2048 sc0 sc1 lds\n\t" \
  "v_mfma_f32_16x16x128_f8f6f4 a[108:111], v[144:151], v[216:223], a[108:111] blgp:1\n\t" \
  "buffer_load_dwordx4 v252, s[40:43], s52 offen offset:3072 sc0 sc1 lds\n\t" \
  "s_waitcnt lgkmcnt(0)\n\t" \
  "v_mfma_f32_16x16x128_f8f6f4 a[124:127], v[152:159], v[216:223], a[124:127] blgp:1\n\t" \
  "v_mfma_f32_16x16x128_f8f6f4 a[120:123], v[152:159], v[208:215], a[120:123] blgp:1\n\t" \
  "v_mfma_f32_16x16x128_f8f6f4 a[116:119], v[152:159], v[200:207], a[116:119] blgp:1\n\t" \
  "v_mfma_f32_16x16x128_f8f6f4 a[112:115], v[152:159], v[192:199], a[112:115] blgp:1\n\t" \
  "s_waitcnt vmcnt(17) lgkmcnt(0)\n\t" \
  "s_barrier\n\t" \
  "v_mfma_f32_16x16x128_f8f6f4 v[64:67], v[128:135], v[160:167], v[64:67] blgp:1\n\t" \
  "ds_read_b128 v[192:195], v251 offset:49152\n\t" \
  "v_mfma_f32_16x16x128_f8f6f4 v[68:71], v[128:135], v[168:175], v[68:71] blgp:1\n\t" \
  "ds_read_b128 v[196:199], v251 offset:57344\n\t" \
  "v_mfma_f32_16x16x128_f8f6f4 v[72:75], v[128:135], v[176:183], v[72:75] blgp:1\n\t" \
  "ds_read_b128 v[200:203], v251 offset:49408\n\t" \
  "v_mfma_f32_16x16x128_f8f6f4 v[76:79], v[128:135], v[184:191], v[76:79] blgp:1\n\t" \
  "ds_read_b128 v[128:131], v250 offset:32768\n\t" \
  "ds_read_b128 v[132:135], v250 offset:40960\n\t" \
  "v_mfma_f32_16x16x128_f8f6f4 v[92:95], v[136:143], v[184:191], v[92:95] blgp:1\n\t" \
  "ds_read_b128 v[204:207], v251 offset:57600\n\t" \
  "v_mfma_f32_16x16x128_f8f6f4 v[88:91], v[136:143], v[176:183], v[88:91] blgp:1\n\t" \
  "ds_read_b128 v[208:211], v251 offset:49664\n\t" \
  "v_mfma_f32_16x16x128_f8f6f4 v[84:87], v[136:143], v[168:175], v[84:87] blgp:1\n\t" \
  "ds_read_b128 v[212:215], v251 offset:57856\n\t" \
  "v_mfma_f32_16x16x128_f8f6f4 v[80:83], v[136:143], v[160:167], v[80:83] blgp:1\n\t" \
  "ds_read_b128 v[136:139], v250 offset:33024\n\t" \
  "ds_read_b128 v[140:143], v250 offset:41216\n\t" \
  "v_mfma_f32_16x16x128_f8f6f4 v[96:99], v[144:151], v[160:167], v[96:99] blgp:1\n\t" \
  "ds_read_b128 v[216:219], v251 offset:49920\n\t" \
  "v_mfma_f32_16x16x128_f8f6f4 v[100:103], v[144:151], v[168:175], v[100:103] blgp:1\n\t" \
  "ds_read_b128 v[220:223], v251 offset:58112\n\t" \
  "v_mfma_f32_16x16x128_f8f6f4 v[104:107], v[144:151], v[176:183], v[104:107] blgp:1\n\t" \
  "s_mov_b32 s80, s56\n\t" \
  "s_mov_b32 s56, s64\n\t" \
  "v_mfma_f32_16x16x128_f8f6f4 v[108:111], v[144:151], v[184:191], v[108:111] blgp:1\n\t" \
  "ds_read_b128 v[144:147], v250 offset:33280\n\t" \
  "v_mfma_f32_16x16x128_f8f6f4 v[124:127], v[152:159], v[184:191], v[124:127] blgp:1\n\t" \
  "ds_read_b128 v[148:151], v250 offset:41472\n\t" \
  "v_mfma_f32_16x16x128_f8f6f4 v[120:123], v[152:159], v[176:183], v[120:123] blgp:1\n\t" \
  "s_bitcmp1_b32 s98, 10\n\t" \
  "s_cselect_b32 s46, 0, 0x10000000\n\t" \
  "v_mfma_f32_16x16x128_f8f6f4 v[116:119], v[152:159], v[168:175], v[116:119] blgp:1\n\t" \
  "v_mfma_f32_16x16x128_f8f6f4 v[112:115], v[152:159], v[160:167], v[112:115] blgp:1\n\t" \
  "ds_read_b128 v[152:155], v250 offset:33536\n\t" \
  "ds_read_b128 v[156:159], v250 offset:41728\n\t" \
  "s_waitcnt lgkmcnt(2)\n\t" \
  "v_mfma_f32_16x16x128_f8f6f4 v[0:3], v[128:135], v[192:199], v[0:3] blgp:1\n\t" \
  "ds_read_b128 v[160:163], v251 offset:32768\n\t" \
  "v_mfma_f32_16x16x128_f8f6f4 v[4:7], v[128:135], v[200:207], v[4:7] blgp:1\n\t" \
  "ds_read_b128 v[164:167], v251 offset:40960\n\t" \
  "v_mfma_f32_16x16x128_f8f6f4 v[8:11], v[128:135], v[208:215], v[8:11] blgp:1\n\t" \
  "ds_read_b128 v[168:171], v251 offset:33024\n\t" \
  "v_mfma_f32_16x16x128_f8f6f4 v[12:15], v[128:135], v[216:223], v[12:15] blgp:1\n\t" \
  "ds_read_b128 v[172:175], v251 offset:41216\n\t" \
  "v_mfma_f32_16x16x128_f8f6f4 v[28:31], v[136:143], v[216:223], v[28:31] blgp:1\n\t" \
  "ds_read_b128 v[176:179], v251 offset:33280\n\t" \
  "v_mfma_f32_16x16x128_f8f6f4 v[24:27], v[136:143], v[208:215], v[24:27] blgp:1\n\t" \
  "ds_read_b128 v[180:183], v251 offset:41472\n\t" \
  "v_mfma_f32_16x16x128_f8f6f4 v[20:23], v[136:143], v[200:207], v[20:23] blgp:1\n\t" \
  "ds_read_b128 v[184:187], v251 offset:33536\n\t" \
  "v_mfma_f32_16x16x128_f8f6f4 v[16:19], v[136:143], v[192:199], v[16:19] blgp:1\n\t" \
  "ds_read_b128 v[188:191], v251 offset:41728\n\t" \
  "s_add_i32 s52, s48, 16384\n\t" \
  "s_add_u32 m0, s53, 49152\n\t" \
  "v_mfma_f32_16x16x128_f8f6f4 v[32:35], v[144:151], v[192:199], v[32:35] blgp:1\n\t" \
  "buffer_load_dwordx4 v252, s[40:43], s52 offen offset:0 sc0 sc1 lds\n\t" \
  "v_mfma_f32_16x16x128_f8f6f4 v[36:39], v[144:151], v[200:207], v[36:39] blgp:1\n\t" \
  "v_mfma_f32_16x16x128_f8f6f4 v[40:43], v[144:151], v[208:215], v[40:43] blgp:1\n\t" \
  "buffer_load_dwordx4 v252, s[40:43], s52 offen offset:1024 sc0 sc1 lds\n\t" \
  "v_mfma_f32_16x16x128_f8f6f4 v[44:47], v[144:151], v[216:223], v[44:47] blgp:1\n\t" \
  "s_waitcnt lgkmcnt(0)\n\t" \
  "v_mfma_f32_16x16x128_f8f6f4 v[60:63], v[152:159], v[216:223], v[60:63] blgp:1\n\t" \
  "buffer_load_dwordx4 v252, s[40:43], s52 offen offset:2048 sc0 sc1 lds\n\t" \
  "v_mfma_f32_16x16x128_f8f6f4 v[56:59], v[152:159], v[208:215], v[56:59] blgp:1\n\t" \
  "v_mov_b32 v248, s59\n\t" \
  "v_mov_b32 v249, s61\n\t" \
  "v_mfma_f32_16x16x128_f8f6f4 v[52:55], v[152:159], v[200:207], v[52:55] blgp:1\n\t" \
  "buffer_load_dwordx4 v252, s[40:43], s52 offen offset:3072 sc0 sc1 lds\n\t" \
  "v_mfma_f32_16x16x128_f8f6f4 v[48:51], v[152:159], v[192:199], v[48:51] blgp:1\n\t" \
  "s_waitcnt vmcnt(16) lgkmcnt(0)\n\t" \
  "s_barrier\n\t" \
  "v_mfma_f32_16x16x128_f8f6f4 a[128:131], v[128:135], v[160:167], a[0:3] blgp:1\n\t" \
  "v_readfirstlane_b32 s87, v254\n\t" \
  "v_readlane_b32 s65, v255, s67\n\t" \
  "v_fma_f32 v240, v0, s58, v248\n\t" \
  "v_fma_f32 v241, v4, s58, v248\n\t" \
  "v_fma_f32 v242, v8, s58, v248\n\t" \
  "v_fma_f32 v243, v12, s58, v248\n\t" \
  "v_exp_f32_e64 v240, v240 clamp\n\t" \
  "v_mfma_f32_16x16x128_f8f6f4 a[132:135], v[128:135], v[168:175], a[4:7] blgp:1\n\t" \
  "v_exp_f32_e64 v241, v241 clamp\n\t" \
  "v_exp_f32_e64 v242, v242 clamp\n\t" \
  "v_exp_f32_e64 v243, v243 clamp\n\t" \
  "v_fma_f32 v240, v240, v249, s60\n\t" \
  "v_mfma_f32_16x16x128_f8f6f4 a[136:139], v[128:135], v[176:183], a[8:11] blgp:1\n\t" \
  "s_max_u32 s69, s69, s87\n\t" \
  "s_add_u32 s67, s67, 1\n\t" \
  "s_cmp_ge_u32 s69, s65\n\t" \
  "s_cbranch_scc0 L_w28_%=\n\t" \
  "L_ok28_%=:\n\t" \
  "v_fma_f32 v241, v241, v249, s60\n\t" \
  "v_fma_f32 v242, v242, v249, s60\n\t" \
  "v_fma_f32 v243, v243, v249, s60\n\t" \
  "v_mfma_f32_16x16x128_f8f6f4 a[140:143], v[128:135], v[184:191], a[12:15] blgp:1\n\t" \
  "ds_read_b128 v[128:131], v250 offset:49152\n\t" \
  "v_rcp_f32 v240, v240\n\t" \
  "v_rcp_f32 v241, v241\n\t" \
  "v_rcp_f32 v242, v242\n\t" \
  "v_mfma_f32_16x16x128_f8f6f4 a[156:159], v[136:143], v[184:191], a[28:31] blgp:1\n\t" \
  "s_add_i32 s52, s48, 32768\n\t" \
  "s_add_u32 m0, s53, 65536\n\t" \
  "ds_read_b128 v[132:135], v250 offset:57344\n\t" \
  "v_rcp_f32 v243, v243\n\t" \
  "v_mul_f32 v240, v0, v240\n\t" \
  "v_mul_f32 v241, v4, v241\n\t" \
  "v_mfma_f32_16x16x128_f8f6f4 a[152:155], v[136:143], v[176:183], a[24:27] blgp:1\n\t" \
  "buffer_load_dwordx4 v252, s[40:43], s52 offen offset:0 sc0 sc1 lds\n\t" \
  "v_mul_f32 v242, v8, v242\n\t" \
  "v_mul_f32 v243, v12, v243\n\t" \
  "v_fma_f32 v244, v1, s58, v248\n\t" \
  "v_fma_f32 v245, v5, s58, v248\n\t" \
  "v_fma_f32 v246, v9, s58, v248\n\t" \
  "v_mfma_f32_16x16x128_f8f6f4 a[148:151], v[136:143], v[168:175], a[20:23] blgp:1\n\t" \
  "buffer_load_dwordx4 v252, s[40:43], s52 offen offset:1024 sc0 sc1 lds\n\t" \
  "v_fma_f32 v247, v13, s58, v248\n\t" \
  "s_add_u32 s57, s80, 0\n\t" \
  "buffer_store_dwordx4 v[240:243], v253, s[44:47], s57 offen offset:256 nt\n\t" \
  "v_mfma_f32_16x16x128_f8f6f4 a[144:147], v[136:143], v[160:167], a[16:19] blgp:1\n\t" \
  "ds_read_b128 v[136:139], v250 offset:49408\n\t" \
  "v_exp_f32_e64 v244, v244 clamp\n\t" \
  "v_exp_f32_e64 v245, v245 clamp\n\t" \
  "v_mfma_f32_16x16x128_f8f6f4 a[160:163], v[144:151], v[160:167], a[32:35] blgp:1\n\t" \
  "ds_read_b128 v[140:143], v250 offset:57600\n\t" \
  "v_exp_f32_e64 v246, v246 clamp\n\t" \
  "v_exp_f32_e64 v247, v247 clamp\n\t" \
  "v_fma_f32 v244, v244, v249, s60\n\t" \
  "v_fma_f32 v245, v245, v249, s60\n\t" \
  "v_mfma_f32_16x16x128_f8f6f4 a[164:167], v[144:151], v[168:175], a[36:39] blgp:1\n\t" \
  "buffer_load_dwordx4 v252, s[40:43], s52 offen offset:2048 sc0 sc1 lds\n\t" \
  "v_fma_f32 v246, v246, v249, s60\n\t" \
  "v_fma_f32 v247, v247, v249, s60\n\t" \
  "v_rcp_f32 v244, v244\n\t" \
  "v_mfma_f32_16x16x128_f8f6f4 a[168:171], v[144:151], v[176:183], a[40:43] blgp:1\n\t" \
  "buffer_load_dwordx4 v252, s[40:43], s52 offen offset:3072 sc0 sc1 lds\n\t" \
  "v_rcp_f32 v245, v245\n\t" \
  "v_rcp_f32 v246, v246\n\t" \
  "v_rcp_f32 v247, v247\n\t" \
  "v_mfma_f32_16x16x128_f8f6f4 a[172:175], v[144:151], v[184:191], a[44:47] blgp:1\n\t" \
  "ds_read_b128 v[144:147], v250 offset:49664\n\t" \
  "v_mul_f32 v244, v1, v244\n\t" \
  "v_mul_f32 v245, v5, v245\n\t" \
  "v_mul_f32 v246, v9, v246\n\t" \
  "v_mul_f32 v247, v13, v247\n\t" \
  "v_fma_f32 v240, v2, s58, v248\n\t" \
  "v_mfma_f32_16x16x128_f8f6f4 a[188:191], v[152:159], v[184:191], a[60:63] blgp:1\n\t" \
  "ds_read_b128 v[148:151], v250 offset:57856\n\t" \
  "v_fma_f32 v241, v6, s58, v248\n\t" \
  "v_fma_f32 v242, v10, s58, v248\n\t" \
  "v_fma_f32 v243, v14, s58, v248\n\t" \
  "s_add_u32 s57, s80, 65536\n\t" \
  "v_mfma_f32_16x16x128_f8f6f4 a[184:187], v[152:159], v[176:183], a[56:59] blgp:1\n\t" \
  "buffer_store_dwordx4 v[244:247], v253, s[44:47], s57 offen offset:256 nt\n\t" \
  "v_exp_f32_e64 v240, v240 clamp\n\t" \
  "v_exp_f32_e64 v241, v241 clamp\n\t" \
  "v_mfma_f32_16x16x128_f8f6f4 a[180:183], v[152:159], v[168:175], a[52:55] blgp:1\n\t" \
  "v_exp_f32_e64 v242, v242 clamp\n\t" \
  "v_exp_f32_e64 v243, v243 clamp\n\t" \
  "v_fma_f32 v240, v240, v249, s60\n\t" \
  "v_fma_f32 v241, v241, v249, s60\n\t" \
  "v_fma_f32 v242, v242, v249, s60\n\t" \
  "v_fma_f32 v243, v243, v249, s60\n\t" \
  "v_mfma_f32_16x16x128_f8f6f4 a[176:179], v[152:159], v[160:167], a[48:51] blgp:1\n\t" \
  "v_rcp_f32 v240, v240\n\t" \
  "ds_read_b128 v[152:155], v250 offset:49920\n\t" \
  "ds_read_b128 v[156:159], v250 offset:58112\n\t" \
  "s_waitcnt lgkmcnt(2)\n\t" \
  "v_mfma_f32_16x16x128_f8f6f4 v[64:67], v[128:135], v[160:167], v[64:67] blgp:1\n\t" \
  "s_add_i32 s52, s49, 49152\n\t" \
  "s_add_u32 m0, s53, 147456\n\t" \
  "v_rcp_f32 v241, v241\n\t" \
  "v_rcp_f32 v242, v242\n\t" \
  "v_rcp_f32 v243, v243\n\t" \
  "v_mul_f32 v240, v2, v240\n\t" \
  "v_mfma_f32_16x16x128_f8f6f4 v[68:71], v[128:135], v[168:175], v[68:71] blgp:1\n\t" \
  "buffer_load_dwordx4 v252, s[40:43], s52 offen offset:0 sc0 sc1 lds\n\t" \
  "v_mul_f32 v241, v6, v241\n\t" \
  "v_mul_f32 v242, v10, v242\n\t" \
  "ds_write_b128 v252, v[48:51] offset:0\n\t" \
  "v_mul_f32 v243, v14, v243\n\t" \
  "v_mfma_f32_16x16x128_f8f6f4 v[72:75], v[128:135], v[176:183], v[72:75] blgp:1\n\t" \
  "buffer_load_dwordx4 v252, s[40:43], s52 offen offset:1024 sc0 sc1 lds\n\t" \
  "v_fma_f32 v244, v3, s58, v248\n\t" \
  "v_fma_f32 v245, v7, s58, v248\n\t" \
  "v_fma_f32 v246, v11, s58, v248\n\t" \
  "v_fma_f32 v247, v15, s58, v248\n\t" \
  "s_add_u32 s57, s80, 131072\n\t" \
  "v_mfma_f32_16x16x128_f8f6f4 v[76:79], v[128:135], v[184:191], v[76:79] blgp:1\n\t" \
  "buffer_store_dwordx4 v[240:243], v253, s[44:47], s57 offen offset:256 nt\n\t" \
  "v_exp_f32_e64 v244, v244 clamp\n\t" \
  "v_exp_f32_e64 v245, v245 clamp\n\t" \
  "v_exp_f32_e64 v246, v246 clamp\n\t" \
  "v_mfma_f32_16x16x128_f8f6f4 v[92:95], v[136:143], v[184:191], v[92:95] blgp:1\n\t" \
  "buffer_load_dwordx4 v252, s[40:43], s52 offen offset:2048 sc0 sc1 lds\n\t" \
  "ds_write_b128 v252, v[52:55] offset:1024\n\t" \
  "v_exp_f32_e64 v247, v247 clamp\n\t" \
  "v_fma_f32 v244, v244, v249, s60\n\t" \
  "v_mfma_f32_16x16x128_f8f6f4 v[88:91], v[136:143], v[176:183], v[88:91] blgp:1\n\t" \
  "buffer_load_dwordx4 v252, s[40:43], s52 offen offset:3072 sc0 sc1 lds\n\t" \
  "v_fma_f32 v245, v245, v249, s60\n\t" \
  "v_fma_f32 v246, v246, v249, s60\n\t" \
  "v_fma_f32 v247, v247, v249, s60\n\t" \
  "v_rcp_f32 v244, v244\n\t" \
  "v_mfma_f32_16x16x128_f8f6f4 v[84:87], v[136:143], v[168:175], v[84:87] blgp:1\n\t" \
  "s_add_i32 s52, s49, 32768\n\t" \
  "s_add_u32 m0, s53, 131072\n\t" \
  "v_rcp_f32 v245, v245\n\t" \
  "v_rcp_f32 v246, v246\n\t" \
  "v_rcp_f32 v247, v247\n\t" \
  "v_mul_f32 v244, v3, v244\n\t" \
  "v_mfma_f32_16x16x128_f8f6f4 v[80:83], v[136:143], v[160:167], v[80:83] blgp:1\n\t" \
  "buffer_load_dwordx4 v252, s[40:43], s52 offen offset:0 sc0 sc1 lds\n\t" \
  "ds_write_b128 v252, v[56:59] offset:2048\n\t" \
  "v_mul_f32 v245, v7, v245\n\t" \
  "v_mul_f32 v246, v11, v246\n\t" \
  "v_mul_f32 v247, v15, v247\n\t" \
  "v_mfma_f32_16x16x128_f8f6f4 v[96:99], v[144:151], v[160:167], v[96:99] blgp:1\n\t" \
  "buffer_load_dwordx4 v252, s[40:43], s52 offen offset:1024 sc0 sc1 lds\n\t" \
  "v_fma_f32 v240, v16, s58, v248\n\t" \
  "v_fma_f32 v241, v20, s58, v248\n\t" \
  "v_fma_f32 v242, v24, s58, v248\n\t" \
  "v_fma_f32 v243, v28, s58, v248\n\t" \
  "s_add_u32 s57, s80, 196608\n\t" \
  "v_mfma_f32_16x16x128_f8f6f4 v[100:103], v[144:151], v[168:175], v[100:103] blgp:1\n\t" \
  "buffer_store_dwordx4 v[244:247], v253, s[44:47], s57 offen offset:256 nt\n\t" \
  "v_exp_f32_e64 v240, v240 clamp\n\t" \
  "ds_write_b128 v252, v[60:63] offset:3072\n\t" \
  "v_exp_f32_e64 v241, v241 clamp\n\t" \
  "v_mfma_f32_16x16x128_f8f6f4 v[104:107], v[144:151], v[176:183], v[104:107] blgp:1\n\t" \
  "buffer_load_dwordx4 v252, s[40:43], s52 offen offset:2048 sc0 sc1 lds\n\t" \
  "v_exp_f32_e64 v242, v242 clamp\n\t" \
  "v_exp_f32_e64 v243, v243 clamp\n\t" \
  "v_fma_f32 v240, v240, v249, s60\n\t" \
  "v_mfma_f32_16x16x128_f8f6f4 v[108:111], v[144:151], v[184:191], v[108:111] blgp:1\n\t" \
  "buffer_load_dwordx4 v252, s[40:43], s52 offen offset:3072 sc0 sc1 lds\n\t" \
  "v_fma_f32 v241, v241, v249, s60\n\t" \
  "v_fma_f32 v242, v242, v249, s60\n\t" \
  "v_fma_f32 v243, v243, v249, s60\n\t" \
  "v_rcp_f32 v240, v240\n\t" \
  "s_waitcnt lgkmcnt(0)\n\t" \
  "v_mfma_f32_16x16x128_f8f6f4 v[236:239], v[152:159], v[184:191], v[124:127] blgp:1\n\t" \
  "v_rcp_f32 v241, v241\n\t" \
  "v_rcp_f32 v242, v242\n\t" \
  "v_rcp_f32 v243, v243\n\t" \
  "v_mul_f32 v240, v16, v240\n\t" \
  "v_mul_f32 v241, v20, v241\n\t" \
  "v_mul_f32 v242, v24, v242\n\t" \
  "v_mfma_f32_16x16x128_f8f6f4 v[232:235], v[152:159], v[176:183], v[120:123] blgp:1\n\t" \
  "v_mul_f32 v243, v28, v243\n\t" \
  "v_fma_f32 v244, v17, s58, v248\n\t" \
  "v_fma_f32 v245, v21, s58, v248\n\t" \
  "v_fma_f32 v246, v25, s58, v248\n\t" \
  "v_fma_f32 v247, v29, s58, v248\n\t" \
  "s_add_u32 s57, s80, 1048576\n\t" \
  "buffer_store_dwordx4 v[240:243], v253, s[44:47], s57 offen offset:256 nt\n\t" \
  "v_mfma_f32_16x16x128_f8f6f4 v[228:231], v[152:159], v[168:175], v[116:119] blgp:1\n\t" \
  "v_exp_f32_e64 v244, v244 clamp\n\t" \
  "v_exp_f32_e64 v245, v245 clamp\n\t" \
  "v_exp_f32_e64 v246, v246 clamp\n\t" \
  "v_exp_f32_e64 v247, v247 clamp\n\t" \
  "v_mfma_f32_16x16x128_f8f6f4 v[224:227], v[152:159], v[160:167], v[112:115] blgp:1\n\t" \
  "v_fma_f32 v244, v244, v249, s60\n\t" \
  "v_fma_f32 v245, v245, v249, s60\n\t" \
  "v_fma_f32 v246, v246, v249, s60\n\t" \
  "v_fma_f32 v247, v247, v249, s60\n\t" \
  "v_rcp_f32 v244, v244\n\t" \
  "s_waitcnt vmcnt(21) lgkmcnt(0)\n\t" \
  "s_barrier\n\t" \
  "v_mfma_f32_16x16x128_f8f6f4 a[192:195], v[128:135], v[192:199], a[64:67] blgp:1\n\t" \
  "ds_read_b128 v[160:163], v251 offset:0\n\t" \
  "v_rcp_f32 v245, v245\n\t" \
  "v_rcp_f32 v246, v246\n\t" \
  "v_mfma_f32_16x16x128_f8f6f4 a[196:199], v[128:135], v[200:207], a[68:71] blgp:1\n\t" \
  "ds_read_b128 v[164:167], v251 offset:8192\n\t" \
  "v_rcp_f32 v247, v247\n\t" \
  "v_mul_f32 v244, v17, v244\n\t" \
  "v_mul_f32 v245, v21, v245\n\t" \
  "v_mul_f32 v246, v25, v246\n\t" \
  "v_mul_f32 v247, v29, v247\n\t" \
  "v_mfma_f32_16x16x128_f8f6f4 a[200:203], v[128:135], v[208:215], a[72:75] blgp:1\n\t" \
  "ds_read_b128 v[168:171], v251 offset:256\n\t" \
  "v_fma_f32 v240, v18, s58, v248\n\t" \
  "v_fma_f32 v241, v22, s58, v248\n\t" \
  "v_fma_f32 v242, v26, s58, v248\n\t" \
  "v_fma_f32 v243, v30, s58, v248\n\t" \
  "s_add_u32 s57, s80, 1114112\n\t" \
  "v_mfma_f32_16x16x128_f8f6f4 a[204:207], v[128:135], v[216:223], a[76:79] blgp:1\n\t" \
  "ds_read_b128 v[128:131], v250 offset:0\n\t" \
  "ds_read_b128 v[132:135], v250 offset:8192\n\t" \
  "buffer_store_dwordx4 v[244:247], v253, s[44:47], s57 offen offset:256 nt\n\t" \
  "v_mfma_f32_16x16x128_f8f6f4 a[220:223], v[136:143], v[216:223], a[92:95] blgp:1\n\t" \
  "ds_read_b128 v[172:175], v251 offset:8448\n\t" \
  "v_exp_f32_e64 v240, v240 clamp\n\t" \
  "v_exp_f32_e64 v241, v241 clamp\n\t" \
  "v_mfma_f32_16x16x128_f8f6f4 a[216:219], v[136:143], v[208:215], a[88:91] blgp:1\n\t" \
  "ds_read_b128 v[176:179], v251 offset:512\n\t" \
  "v_exp_f32_e64 v242, v242 clamp\n\t" \
  "v_exp_f32_e64 v243, v243 clamp\n\t" \
  "v_fma_f32 v240, v240, v249, s60\n\t" \
  "v_mfma_f32_16x16x128_f8f6f4 a[212:215], v[136:143], v[200:207], a[84:87] blgp:1\n\t" \
  "ds_read_b128 v[180:183], v251 offset:8704\n\t" \
  "v_fma_f32 v241, v241, v249, s60\n\t" \
  "v_fma_f32 v242, v242, v249, s60\n\t" \
  "v_fma_f32 v243, v243, v249, s60\n\t" \
  "v_rcp_f32 v240, v240\n\t" \
  "v_mfma_f32_16x16x128_f8f6f4 a[208:211], v[136:143], v[192:199], a[80:83] blgp:1\n\t" \
  "ds_read_b128 v[136:139], v250 offset:256\n\t" \
  "ds_read_b128 v[140:143], v250 offset:8448\n\t" \
  "v_rcp_f32 v241, v241\n\t" \
  "v_rcp_f32 v242, v242\n\t" \
  "v_mfma_f32_16x16x128_f8f6f4 a[224:227], v[144:151], v[192:199], a[96:99] blgp:1\n\t" \
  "ds_read_b128 v[184:187], v251 offset:768\n\t" \
  "v_rcp_f32 v243, v243\n\t" \
  "v_mul_f32 v240, v18, v240\n\t" \
  "v_mul_f32 v241, v22, v241\n\t" \
  "v_mul_f32 v242, v26, v242\n\t" \
  "v_mfma_f32_16x16x128_f8f6f4 a[228:231], v[144:151], v[200:207], a[100:103] blgp:1\n\t" \
  "ds_read_b128 v[188:191], v251 offset:8960\n\t" \
  "v_mul_f32 v243, v30, v243\n\t" \
  "v_fma_f32 v244, v19, s58, v248\n\t" \
  "v_fma_f32 v245, v23, s58, v248\n\t" \
  "v_fma_f32 v246, v27, s58, v248\n\t" \
  "v_fma_f32 v247, v31, s58, v248\n\t" \
  "v_mfma_f32_16x16x128_f8f6f4 a[232:235], v[144:151], v[208:215], a[104:107] blgp:1\n\t" \
  "s_add_u32 s48, s48, 0x10000\n\t" \
  "s_add_u32 s49, s49, 0x10000\n\t" \
  "s_add_u32 s57, s80, 1179648\n\t" \
  "buffer_store_dwordx4 v[240:243], v253, s[44:47], s57 offen offset:256 nt\n\t" \
  "v_mfma_f32_16x16x128_f8f6f4 a[236:239], v[144:151], v[216:223], a[108:111] blgp:1\n\t" \
  "ds_read_b128 v[144:147], v250 offset:512\n\t" \
  "v_exp_f32_e64 v244, v244 clamp\n\t" \
  "v_exp_f32_e64 v245, v245 clamp\n\t" \
  "v_mfma_f32_16x16x128_f8f6f4 a[252:255], v[152:159], v[216:223], a[124:127] blgp:1\n\t" \
  "ds_read_b128 v[148:151], v250 offset:8704\n\t" \
  "v_exp_f32_e64 v246, v246 clamp\n\t" \
  "v_exp_f32_e64 v247, v247 clamp\n\t" \
  "v_fma_f32 v244, v244, v249, s60\n\t" \
  "v_fma_f32 v245, v245, v249, s60\n\t" \
  "v_mfma_f32_16x16x128_f8f6f4 a[248:251], v[152:159], v[208:215], a[120:123] blgp:1\n\t" \
  "v_fma_f32 v246, v246, v249, s60\n\t" \
  "v_fma_f32 v247, v247, v249, s60\n\t" \
  "v_rcp_f32 v244, v244\n\t" \
  "v_rcp_f32 v245, v245\n\t" \
  "v_mfma_f32_16x16x128_f8f6f4 a[244:247], v[152:159], v[200:207], a[116:119] blgp:1\n\t" \
  "v_rcp_f32 v246, v246\n\t" \
  "v_rcp_f32 v247, v247\n\t" \
  "v_mul_f32 v244, v19, v244\n\t" \
  "v_mul_f32 v245, v23, v245\n\t" \
  "v_mul_f32 v246, v27, v246\n\t" \
  "v_mfma_f32_16x16x128_f8f6f4 a[240:243], v[152:159], v[192:199], a[112:115] blgp:1\n\t" \
  "v_mul_f32 v247, v31, v247\n\t" \
  "ds_read_b128 v[152:155], v250 offset:768\n\t" \
  "ds_read_b128 v[156:159], v250 offset:8960\n\t" \
  "s_bitcmp1_b32 s98, 10\n\t" \
  "s_cbranch_scc1 L_shared_%=\n\t" \
  "s_sub_u32 s55, s55, 1\n\t" \
  "s_cmp_lg_u32 s55, 0\n\t" \
  "s_cbranch_scc1 L_tile_%=\n\t" \
  "L_tail_%=:\n\t" \
  "s_nop 7\n\t" \
  "s_add_u32 s57, s80, 1245184\n\t" \
  "buffer_store_dwordx4 v[244:247], v253, s[44:47], s57 offen offset:256 nt\n\t" \
  "v_pk_fma_f32 v[200:201], v[32:33], s[58:59], s[58:59] op_sel:[0,0,1] op_sel_hi:[1,0,1]\n\t" \
  "v_pk_fma_f32 v[202:203], v[36:37], s[58:59], s[58:59] op_sel:[0,0,1] op_sel_hi:[1,0,1]\n\t" \
  "v_pk_fma_f32 v[204:205], v[40:41], s[58:59], s[58:59] op_sel:[0,0,1] op_sel_hi:[1,0,1]\n\t" \
  "v_pk_fma_f32 v[206:207], v[44:45], s[58:59], s[58:59] op_sel:[0,0,1] op_sel_hi:[1,0,1]\n\t" \
  "v_exp_f32_e64 v192, v200 clamp\n\t" \
  "v_exp_f32_e64 v193, v202 clamp\n\t" \
  "v_exp_f32_e64 v194, v204 clamp\n\t" \
  "v_exp_f32_e64 v195, v206 clamp\n\t" \
  "v_exp_f32_e64 v196, v201 clamp\n\t" \
  "v_exp_f32_e64 v197, v203 clamp\n\t" \
  "v_exp_f32_e64 v198, v205 clamp\n\t" \
  "v_exp_f32_e64 v199, v207 clamp\n\t" \
  "v_pk_fma_f32 v[192:193], v[192:193], s[60:61], s[60:61] op_sel:[0,1,0] op_sel_hi:[1,1,0]\n\t" \
  "v_pk_fma_f32 v[194:195], v[194:195], s[60:61], s[60:61] op_sel:[0,1,0] op_sel_hi:[1,1,0]\n\t" \
  "v_pk_fma_f32 v[196:197], v[196:197], s[60:61], s[60:61] op_sel:[0,1,0] op_sel_hi:[1,1,0]\n\t" \
  "v_pk_fma_f32 v[198:199], v[198:199], s[60:61], s[60:61] op_sel:[0,1,0] op_sel_hi:[1,1,0]\n\t" \
  "v_rcp_f32 v192, v192\n\t" \
  "v_rcp_f32 v193, v193\n\t" \
  "v_rcp_f32 v194, v194\n\t" \
  "v_rcp_f32 v195, v195\n\t" \
  "v_rcp_f32 v196, v196\n\t" \
  "v_rcp_f32 v197, v197\n\t" \
  "v_rcp_f32 v198, v198\n\t" \
  "v_rcp_f32 v199, v199\n\t" \
  "v_mul_f32 v192, v32, v192\n\t" \
  "v_mul_f32 v193, v36, v193\n\t" \
  "v_mul_f32 v194, v40, v194\n\t" \
  "v_mul_f32 v195, v44, v195\n\t" \
  "s_add_u32 s57, s80, 2097152\n\t" \
  "buffer_store_dwordx4 v[192:195], v253, s[44:47], s57 offen offset:256 \n\t" \
  "v_mul_f32 v196, v33, v196\n\t" \
  "v_mul_f32 v197, v37, v197\n\t" \
  "v_mul_f32 v198, v41, v198\n\t" \
  "v_mul_f32 v199, v45, v199\n\t" \
  "s_add_u32 s57, s80, 2162688\n\t" \
  "buffer_store_dwordx4 v[196:199], v253, s[44:47], s57 offen offset:256 \n\t" \
  "v_pk_fma_f32 v[216:217], v[34:35], s[58:59], s[58:59] op_sel:[0,0,1] op_sel_hi:[1,0,1]\n\t" \
  "v_pk_fma_f32 v[218:219], v[38:39], s[58:59], s[58:59] op_sel:[0,0,1] op_sel_hi:[1,0,1]\n\t" \
  "v_pk_fma_f32 v[220:221], v[42:43], s[58:59], s[58:59] op_sel:[0,0,1] op_sel_hi:[1,0,1]\n\t" \
  "v_pk_fma_f32 v[222:223], v[46:47], s[58:59], s[58:59] op_sel:[0,0,1] op_sel_hi:[1,0,1]\n\t" \
  "v_exp_f32_e64 v208, v216 clamp\n\t" \
  "v_exp_f32_e64 v209, v218 clamp\n\t" \
  "v_exp_f32_e64 v210, v220 clamp\n\t" \
  "v_exp_f32_e64 v211, v222 clamp\n\t" \
  "v_exp_f32_e64 v212, v217 clamp\n\t" \
  "v_exp_f32_e64 v213, v219 clamp\n\t" \
  "v_exp_f32_e64 v214, v221 clamp\n\t" \
  "v_exp_f32_e64 v215, v223 clamp\n\t" \
  "v_pk_fma_f32 v[208:209], v[208:209], s[60:61], s[60:61] op_sel:[0,1,0] op_sel_hi:[1,1,0]\n\t" \
  "v_pk_fma_f32 v[210:211], v[210:211], s[60:61], s[60:61] op_sel:[0,1,0] op_sel_hi:[1,1,0]\n\t" \
  "v_pk_fma_f32 v[212:213], v[212:213], s[60:61], s[60:61] op_sel:[0,1,0] op_sel_hi:[1,1,0]\n\t" \
  "v_pk_fma_f32 v[214:215], v[214:215], s[60:61], s[60:61] op_sel:[0,1,0] op_sel_hi:[1,1,0]\n\t" \
  "v_rcp_f32 v208, v208\n\t" \
  "v_rcp_f32 v209, v209\n\t" \
  "v_rcp_f32 v210, v210\n\t" \
  "v_rcp_f32 v211, v211\n\t" \
  "v_rcp_f32 v212, v212\n\t" \
  "v_rcp_f32 v213, v213\n\t" \
  "v_rcp_f32 v214, v214\n\t" \
  "v_rcp_f32 v215, v215\n\t" \
  "v_mul_f32 v208, v34, v208\n\t" \
  "v_mul_f32 v209, v38, v209\n\t" \
  "v_mul_f32 v210, v42, v210\n\t" \
  "v_mul_f32 v211, v46, v211\n\t" \
  "s_add_u32 s57, s80, 2228224\n\t" \
  "buffer_store_dwordx4 v[208:211], v253, s[44:47], s57 offen offset:256 \n\t" \
  "v_mul_f32 v212, v35, v212\n\t" \
  "v_mul_f32 v213, v39, v213\n\t" \
  "v_mul_f32 v214, v43, v214\n\t" \
  "v_mul_f32 v215, v47, v215\n\t" \
  "s_add_u32 s57, s80, 2293760\n\t" \
  "buffer_store_dwordx4 v[212:215], v253, s[44:47], s57 offen offset:256 \n\t" \
  "v_pk_fma_f32 v[200:201], v[48:49], s[58:59], s[58:59] op_sel:[0,0,1] op_sel_hi:[1,0,1]\n\t" \
  "v_pk_fma_f32 v[202:203], v[52:53], s[58:59], s[58:59] op_sel:[0,0,1] op_sel_hi:[1,0,1]\n\t" \
  "v_pk_fma_f32 v[204:205], v[56:57], s[58:59], s[58:59] op_sel:[0,0,1] op_sel_hi:[1,0,1]\n\t" \
  "v_pk_fma_f32 v[206:207], v[60:61], s[58:59], s[58:59] op_sel:[0,0,1] op_sel_hi:[1,0,1]\n\t" \
  "v_exp_f32_e64 v192, v200 clamp\n\t" \
  "v_exp_f32_e64 v193, v202 clamp\n\t" \
  "v_exp_f32_e64 v194, v204 clamp\n\t" \
  "v_exp_f32_e64 v195, v206 clamp\n\t" \
  "v_exp_f32_e64 v196, v201 clamp\n\t" \
  "v_exp_f32_e64 v197, v203 clamp\n\t" \
  "v_exp_f32_e64 v198, v205 clamp\n\t" \
  "v_exp_f32_e64 v199, v207 clamp\n\t" \
  "v_pk_fma_f32 v[192:193], v[192:193], s[60:61], s[60:61] op_sel:[0,1,0] op_sel_hi:[1,1,0]\n\t" \
  "v_pk_fma_f32 v[194:195], v[194:195], s[60:61], s[60:61] op_sel:[0,1,0] op_sel_hi:[1,1,0]\n\t" \
  "v_pk_fma_f32 v[196:197], v[196:197], s[60:61], s[60:61] op_sel:[0,1,0] op_sel_hi:[1,1,0]\n\t" \
  "v_pk_fma_f32 v[198:199], v[198:199], s[60:61], s[60:61] op_sel:[0,1,0] op_sel_hi:[1,1,0]\n\t" \
  "v_rcp_f32 v192, v192\n\t" \
  "v_rcp_f32 v193, v193\n\t" \
  "v_rcp_f32 v194, v194\n\t" \
  "v_rcp_f32 v195, v195\n\t" \
  "v_rcp_f32 v196, v196\n\t" \
  "v_rcp_f32 v197, v197\n\t" \
  "v_rcp_f32 v198, v198\n\t" \
  "v_rcp_f32 v199, v199\n\t" \
  "v_mul_f32 v192, v48, v192\n\t" \
  "v_mul_f32 v193, v52, v193\n\t" \
  "v_mul_f32 v194, v56, v194\n\t" \
  "v_mul_f32 v195, v60, v195\n\t" \
  "s_add_u32 s57, s80, 3145728\n\t" \
  "buffer_store_dwordx4 v[192:195], v253, s[44:47], s57 offen offset:256 nt\n\t" \
  "v_mul_f32 v196, v49, v196\n\t" \
  "v_mul_f32 v197, v53, v197\n\t" \
  "v_mul_f32 v198, v57, v198\n\t" \
  "v_mul_f32 v199, v61, v199\n\t" \
  "s_add_u32 s57, s80, 3211264\n\t" \
  "buffer_store_dwordx4 v[196:199], v253, s[44:47], s57 offen offset:256 nt\n\t" \
  "v_pk_fma_f32 v[216:217], v[50:51], s[58:59], s[58:59] op_sel:[0,0,1] op_sel_hi:[1,0,1]\n\t" \
  "v_pk_fma_f32 v[218:219], v[54:55], s[58:59], s[58:59] op_sel:[0,0,1] op_sel_hi:[1,0,1]\n\t" \
  "v_pk_fma_f32 v[220:221], v[58:59], s[58:59], s[58:59] op_sel:[0,0,1] op_sel_hi:[1,0,1]\n\t" \
  "v_pk_fma_f32 v[222:223], v[62:63], s[58:59], s[58:59] op_sel:[0,0,1] op_sel_hi:[1,0,1]\n\t" \
  "v_exp_f32_e64 v208, v216 clamp\n\t" \
  "v_exp_f32_e64 v209, v218 clamp\n\t" \
  "v_exp_f32_e64 v210, v220 clamp\n\t" \
  "v_exp_f32_e64 v211, v222 clamp\n\t" \
  "v_exp_f32_e64 v212, v217 clamp\n\t" \
  "v_exp_f32_e64 v213, v219 clamp\n\t" \
  "v_exp_f32_e64 v214, v221 clamp\n\t" \
  "v_exp_f32_e64 v215, v223 clamp\n\t" \
  "v_pk_fma_f32 v[208:209], v[208:209], s[60:61], s[60:61] op_sel:[0,1,0] op_sel_hi:[1,1,0]\n\t" \
  "v_pk_fma_f32 v[210:211], v[210:211], s[60:61], s[60:61] op_sel:[0,1,0] op_sel_hi:[1,1,0]\n\t" \
  "v_pk_fma_f32 v[212:213], v[212:213], s[60:61], s[60:61] op_sel:[0,1,0] op_sel_hi:[1,1,0]\n\t" \
  "v_pk_fma_f32 v[214:215], v[214:215], s[60:61], s[60:61] op_sel:[0,1,0] op_sel_hi:[1,1,0]\n\t" \
  "v_rcp_f32 v208, v208\n\t" \
  "v_rcp_f32 v209, v209\n\t" \
  "v_rcp_f32 v210, v210\n\t" \
  "v_rcp_f32 v211, v211\n\t" \
  "v_rcp_f32 v212, v212\n\t" \
  "v_rcp_f32 v213, v213\n\t" \
  "v_rcp_f32 v214, v214\n\t" \
  "v_rcp_f32 v215, v215\n\t" \
  "v_mul_f32 v208, v50, v208\n\t" \
  "v_mul_f32 v209, v54, v209\n\t" \
  "v_mul_f32 v210, v58, v210\n\t" \
  "v_mul_f32 v211, v62, v211\n\t" \
  "s_add_u32 s57, s80, 3276800\n\t" \
  "buffer_store_dwordx4 v[208:211], v253, s[44:47], s57 offen offset:256 nt\n\t" \
  "v_mul_f32 v212, v51, v212\n\t" \
  "v_mul_f32 v213, v55, v213\n\t" \
  "v_mul_f32 v214, v59, v214\n\t" \
  "v_mul_f32 v215, v63, v215\n\t" \
  "s_add_u32 s57, s80, 3342336\n\t" \
  "buffer_store_dwordx4 v[212:215], v253, s[44:47], s57 offen offset:256 nt\n\t" \
  "v_accvgpr_read_b32 v192, a128\n\t" \
  "v_accvgpr_read_b32 v193, a132\n\t" \
  "v_accvgpr_read_b32 v194, a136\n\t" \
  "v_accvgpr_read_b32 v195, a140\n\t" \
  "v_pk_fma_f32 v[196:197], v[192:193], s[58:59], s[58:59] op_sel:[0,0,1] op_sel_hi:[1,0,1]\n\t" \
  "v_pk_fma_f32 v[198:199], v[194:195], s[58:59], s[58:59] op_sel:[0,0,1] op_sel_hi:[1,0,1]\n\t" \
  "v_exp_f32_e64 v196, v196 clamp\n\t" \
  "v_exp_f32_e64 v197, v197 clamp\n\t" \
  "v_exp_f32_e64 v198, v198 clamp\n\t" \
  "v_exp_f32_e64 v199, v199 clamp\n\t" \
  "v_pk_fma_f32 v[196:197], v[196:197], s[60:61], s[60:61] op_sel:[0,1,0] op_sel_hi:[1,1,0]\n\t" \
  "v_pk_fma_f32 v[198:199], v[198:199], s[60:61], s[60:61] op_sel:[0,1,0] op_sel_hi:[1,1,0]\n\t" \
  "v_rcp_f32 v196, v196\n\t" \
  "v_rcp_f32 v197, v197\n\t" \
  "v_rcp_f32 v198, v198\n\t" \
  "v_rcp_f32 v199, v199\n\t" \
  "v_pk_mul_f32 v[192:193], v[192:193], v[196:197]\n\t" \
  "v_pk_mul_f32 v[194:195], v[194:195], v[198:199]\n\t" \
  "s_add_u32 s57, s80, 0\n\t" \
  "buffer_store_dwordx4 v[192:195], v253, s[44:47], s57 offen offset:0 \n\t" \
  "v_accvgpr_read_b32 v200, a129\n\t" \
  "v_accvgpr_read_b32 v201, a133\n\t" \
  "v_accvgpr_read_b32 v202, a137\n\t" \
  "v_accvgpr_read_b32 v203, a141\n\t" \
  "v_pk_fma_f32 v[204:205], v[200:201], s[58:59], s[58:59] op_sel:[0,0,1] op_sel_hi:[1,0,1]\n\t" \
  "v_pk_fma_f32 v[206:207], v[202:203], s[58:59], s[58:59] op_sel:[0,0,1] op_sel_hi:[1,0,1]\n\t" \
  "v_exp_f32_e64 v204, v204 clamp\n\t" \
  "v_exp_f32_e64 v205, v205 clamp\n\t" \
  "v_exp_f32_e64 v206, v206 clamp\n\t" \
  "v_exp_f32_e64 v207, v207 clamp\n\t" \
  "v_pk_fma_f32 v[204:205], v[204:205], s[60:61], s[60:61] op_sel:[0,1,0] op_sel_hi:[1,1,0]\n\t" \
  "v_pk_fma_f32 v[206:207], v[206:207], s[60:61], s[60:61] op_sel:[0,1,0] op_sel_hi:[1,1,0]\n\t" \
  "v_rcp_f32 v204, v204\n\t" \
  "v_rcp_f32 v205, v205\n\t" \
  "v_rcp_f32 v206, v206\n\t" \
  "v_rcp_f32 v207, v207\n\t" \
  "v_pk_mul_f32 v[200:201], v[200:201], v[204:205]\n\t" \
  "v_pk_mul_f32 v[202:203], v[202:203], v[206:207]\n\t" \
  "s_add_u32 s57, s80, 65536\n\t" \
  "buffer_store_dwordx4 v[200:203], v253, s[44:47], s57 offen offset:0 \n\t" \
  "v_accvgpr_read_b32 v208, a130\n\t" \
  "v_accvgpr_read_b32 v209, a134\n\t" \
  "v_accvgpr_read_b32 v210, a138\n\t" \
  "v_accvgpr_read_b32 v211, a142\n\t" \
  "v_pk_fma_f32 v[212:213], v[208:209], s[58:59], s[58:59] op_sel:[0,0,1] op_sel_hi:[1,0,1]\n\t" \
  "v_pk_fma_f32 v[214:215], v[210:211], s[58:59], s[58:59] op_sel:[0,0,1] op_sel_hi:[1,0,1]\n\t" \
  "v_exp_f32_e64 v212, v212 clamp\n\t" \
  "v_exp_f32_e64 v213, v213 clamp\n\t" \
  "v_exp_f32_e64 v214, v214 clamp\n\t" \
  "v_exp_f32_e64 v215, v215 clamp\n\t" \
  "v_pk_fma_f32 v[212:213], v[212:213], s[60:61], s[60:61] op_sel:[0,1,0] op_sel_hi:[1,1,0]\n\t" \
  "v_pk_fma_f32 v[214:215], v[214:215], s[60:61], s[60:61] op_sel:[0,1,0] op_sel_hi:[1,1,0]\n\t" \
  "v_rcp_f32 v212, v212\n\t" \
  "v_rcp_f32 v213, v213\n\t" \
  "v_rcp_f32 v214, v214\n\t" \
  "v_rcp_f32 v215, v215\n\t" \
  "v_pk_mul_f32 v[208:209], v[208:209], v[212:213]\n\t" \
  "v_pk_mul_f32 v[210:211], v[210:211], v[214:215]\n\t" \
  "s_add_u32 s57, s80, 131072\n\t" \
  "buffer_store_dwordx4 v[208:211], v253, s[44:47], s57 offen offset:0 \n\t" \
  "v_accvgpr_read_b32 v216, a131\n\t" \
  "v_accvgpr_read_b32 v217, a135\n\t" \
  "v_accvgpr_read_b32 v218, a139\n\t" \
  "v_accvgpr_read_b32 v219, a143\n\t" \
  "v_pk_fma_f32 v[220:221], v[216:217], s[58:59], s[58:59] op_sel:[0,0,1] op_sel_hi:[1,0,1]\n\t" \
  "v_pk_fma_f32 v[222:223], v[218:219], s[58:59], s[58:59] op_sel:[0,0,1] op_sel_hi:[1,0,1]\n\t" \
  "v_exp_f32_e64 v220, v220 clamp\n\t" \
  "v_exp_f32_e64 v221, v221 clamp\n\t" \
  "v_exp_f32_e64 v222, v222 clamp\n\t" \
  "v_exp_f32_e64 v223, v223 clamp\n\t" \
  "v_pk_fma_f32 v[220:221], v[220:221], s[60:61], s[60:61] op_sel:[0,1,0] op_sel_hi:[1,1,0]\n\t" \
  "v_pk_fma_f32 v[222:223], v[222:223], s[60:61], s[60:61] op_sel:[0,1,0] op_sel_hi:[1,1,0]\n\t" \
  "v_rcp_f32 v220, v220\n\t" \
  "v_rcp_f32 v221, v221\n\t" \
  "v_rcp_f32 v222, v222\n\t" \
  "v_rcp_f32 v223, v223\n\t" \
  "v_pk_mul_f32 v[216:217], v[216:217], v[220:221]\n\t" \
  "v_pk_mul_f32 v[218:219], v[218:219], v[222:223]\n\t" \
  "s_add_u32 s57, s80, 196608\n\t" \
  "buffer_store_dwordx4 v[216:219], v253, s[44:47], s57 offen offset:0 \n\t" \
  "v_accvgpr_read_b32 v192, a144\n\t" \
  "v_accvgpr_read_b32 v193, a148\n\t" \
  "v_accvgpr_read_b32 v194, a152\n\t" \
  "v_accvgpr_read_b32 v195, a156\n\t" \
  "v_pk_fma_f32 v[196:197], v[192:193], s[58:59], s[58:59] op_sel:[0,0,1] op_sel_hi:[1,0,1]\n\t" \
  "v_pk_fma_f32 v[198:199], v[194:195], s[58:59], s[58:59] op_sel:[0,0,1] op_sel_hi:[1,0,1]\n\t" \
  "v_exp_f32_e64 v196, v196 clamp\n\t" \
  "v_exp_f32_e64 v197, v197 clamp\n\t" \
  "v_exp_f32_e64 v198, v198 clamp\n\t" \
  "v_exp_f32_e64 v199, v199 clamp\n\t" \
  "v_pk_fma_f32 v[196:197], v[196:197], s[60:61], s[60:61] op_sel:[0,1,0] op_sel_hi:[1,1,0]\n\t" \
  "v_pk_fma_f32 v[198:199], v[198:199], s[60:61], s[60:61] op_sel:[0,1,0] op_sel_hi:[1,1,0]\n\t" \
  "v_rcp_f32 v196, v196\n\t" \
  "v_rcp_f32 v197, v197\n\t" \
  "v_rcp_f32 v198, v198\n\t" \
  "v_rcp_f32 v199, v199\n\t" \
  "v_pk_mul_f32 v[192:193], v[192:193], v[196:197]\n\t" \
  "v_pk_mul_f32 v[194:195], v[194:195], v[198:199]\n\t" \
  "s_add_u32 s57, s80, 1048576\n\t" \
  "buffer_store_dwordx4 v[192:195], v253, s[44:47], s57 offen offset:0 nt\n\t" \
  "v_accvgpr_read_b32 v200, a145\n\t" \
  "v_accvgpr_read_b32 v201, a149\n\t" \
  "v_accvgpr_read_b32 v202, a153\n\t" \
  "v_accvgpr_read_b32 v203, a157\n\t" \
  "v_pk_fma_f32 v[204:205], v[200:201], s[58:59], s[58:59] op_sel:[0,0,1] op_sel_hi:[1,0,1]\n\t" \
  "v_pk_fma_f32 v[206:207], v[202:203], s[58:59], s[58:59] op_sel:[0,0,1] op_sel_hi:[1,0,1]\n\t" \
  "v_exp_f32_e64 v204, v204 clamp\n\t" \
  "v_exp_f32_e64 v205, v205 clamp\n\t" \
  "v_exp_f32_e64 v206, v206 clamp\n\t" \
  "v_exp_f32_e64 v207, v207 clamp\n\t" \
  "v_pk_fma_f32 v[204:205], v[204:205], s[60:61], s[60:61] op_sel:[0,1,0] op_sel_hi:[1,1,0]\n\t" \
  "v_pk_fma_f32 v[206:207], v[206:207], s[60:61], s[60:61] op_sel:[0,1,0] op_sel_hi:[1,1,0]\n\t" \
  "v_rcp_f32 v204, v204\n\t" \
  "v_rcp_f32 v205, v205\n\t" \
  "v_rcp_f32 v206, v206\n\t" \
  "v_rcp_f32 v207, v207\n\t" \
  "v_pk_mul_f32 v[200:201], v[200:201], v[204:205]\n\t" \
  "v_pk_mul_f32 v[202:203], v[202:203], v[206:207]\n\t" \
  "s_add_u32 s57, s80, 1114112\n\t" \
  "buffer_store_dwordx4 v[200:203], v253, s[44:47], s57 offen offset:0 nt\n\t" \
  "v_accvgpr_read_b32 v208, a146\n\t" \
  "v_accvgpr_read_b32 v209, a150\n\t" \
  "v_accvgpr_read_b32 v210, a154\n\t" \
  "v_accvgpr_read_b32 v211, a158\n\t" \
  "v_pk_fma_f32 v[212:213], v[208:209], s[58:59], s[58:59] op_sel:[0,0,1] op_sel_hi:[1,0,1]\n\t" \
  "v_pk_fma_f32 v[214:215], v[210:211], s[58:59], s[58:59] op_sel:[0,0,1] op_sel_hi:[1,0,1]\n\t" \
  "v_exp_f32_e64 v212, v212 clamp\n\t" \
  "v_exp_f32_e64 v213, v213 clamp\n\t" \
  "v_exp_f32_e64 v214, v214 clamp\n\t" \
  "v_exp_f32_e64 v215, v215 clamp\n\t" \
  "v_pk_fma_f32 v[212:213], v[212:213], s[60:61], s[60:61] op_sel:[0,1,0] op_sel_hi:[1,1,0]\n\t" \
  "v_pk_fma_f32 v[214:215], v[214:215], s[60:61], s[60:61] op_sel:[0,1,0] op_sel_hi:[1,1,0]\n\t" \
  "v_rcp_f32 v212, v212\n\t" \
  "v_rcp_f32 v213, v213\n\t" \
  "v_rcp_f32 v214, v214\n\t" \
  "v_rcp_f32 v215, v215\n\t" \
  "v_pk_mul_f32 v[208:209], v[208:209], v[212:213]\n\t" \
  "v_pk_mul_f32 v[210:211], v[210:211], v[214:215]\n\t" \
  "s_add_u32 s57, s80, 1179648\n\t" \
  "buffer_store_dwordx4 v[208:211], v253, s[44:47], s57 offen offset:0 nt\n\t" \
  "v_accvgpr_read_b32 v216, a147\n\t" \
  "v_accvgpr_read_b32 v217, a151\n\t" \
  "v_accvgpr_read_b32 v218, a155\n\t" \
  "v_accvgpr_read_b32 v219, a159\n\t" \
  "v_pk_fma_f32 v[220:221], v[216:217], s[58:59], s[58:59] op_sel:[0,0,1] op_sel_hi:[1,0,1]\n\t" \
  "v_pk_fma_f32 v[222:223], v[218:219], s[58:59], s[58:59] op_sel:[0,0,1] op_sel_hi:[1,0,1]\n\t" \
  "v_exp_f32_e64 v220, v220 clamp\n\t" \
  "v_exp_f32_e64 v221, v221 clamp\n\t" \
  "v_exp_f32_e64 v222, v222 clamp\n\t" \
  "v_exp_f32_e64 v223, v223 clamp\n\t" \
  "v_pk_fma_f32 v[220:221], v[220:221], s[60:61], s[60:61] op_sel:[0,1,0] op_sel_hi:[1,1,0]\n\t" \
  "v_pk_fma_f32 v[222:223], v[222:223], s[60:61], s[60:61] op_sel:[0,1,0] op_sel_hi:[1,1,0]\n\t" \
  "v_rcp_f32 v220, v220\n\t" \
  "v_rcp_f32 v221, v221\n\t" \
  "v_rcp_f32 v222, v222\n\t" \
  "v_rcp_f32 v223, v223\n\t" \
  "v_pk_mul_f32 v[216:217], v[216:217], v[220:221]\n\t" \
  "v_pk_mul_f32 v[218:219], v[218:219], v[222:223]\n\t" \
  "s_add_u32 s57, s80, 1245184\n\t" \
  "buffer_store_dwordx4 v[216:219], v253, s[44:47], s57 offen offset:0 nt\n\t" \
  "v_accvgpr_read_b32 v192, a160\n\t" \
  "v_accvgpr_read_b32 v193, a164\n\t" \
  "v_accvgpr_read_b32 v194, a168\n\t" \
  "v_accvgpr_read_b32 v195, a172\n\t" \
  "v_pk_fma_f32 v[196:197], v[192:193], s[58:59], s[58:59] op_sel:[0,0,1] op_sel_hi:[1,0,1]\n\t" \
  "v_pk_fma_f32 v[198:199], v[194:195], s[58:59], s[58:59] op_sel:[0,0,1] op_sel_hi:[1,0,1]\n\t" \
  "v_exp_f32_e64 v196, v196 clamp\n\t" \
  "v_exp_f32_e64 v197, v197 clamp\n\t" \
  "v_exp_f32_e64 v198, v198 clamp\n\t" \
  "v_exp_f32_e64 v199, v199 clamp\n\t" \
  "v_pk_fma_f32 v[196:197], v[196:197], s[60:61], s[60:61] op_sel:[0,1,0] op_sel_hi:[1,1,0]\n\t" \
  "v_pk_fma_f32 v[198:199], v[198:199], s[60:61], s[60:61] op_sel:[0,1,0] op_sel_hi:[1,1,0]\n\t" \
  "v_rcp_f32 v196, v196\n\t" \
  "v_rcp_f32 v197, v197\n\t" \
  "v_rcp_f32 v198, v198\n\t" \
  "v_rcp_f32 v199, v199\n\t" \
  "v_pk_mul_f32 v[192:193], v[192:193], v[196:197]\n\t" \
  "v_pk_mul_f32 v[194:195], v[194:195], v[198:199]\n\t" \
  "s_add_u32 s57, s80, 2097152\n\t" \
  "buffer_store_dwordx4 v[192:195], v253, s[44:47], s57 offen offset:0 \n\t" \
  "v_accvgpr_read_b32 v200, a161\n\t" \
  "v_accvgpr_read_b32 v201, a165\n\t" \
  "v_accvgpr_read_b32 v202, a169\n\t" \
  "v_accvgpr_read_b32 v203, a173\n\t" \
  "v_pk_fma_f32 v[204:205], v[200:201], s[58:59], s[58:59] op_sel:[0,0,1] op_sel_hi:[1,0,1]\n\t" \
  "v_pk_fma_f32 v[206:207], v[202:203], s[58:59], s[58:59] op_sel:[0,0,1] op_sel_hi:[1,0,1]\n\t" \
  "v_exp_f32_e64 v204, v204 clamp\n\t" \
  "v_exp_f32_e64 v205, v205 clamp\n\t" \
  "v_exp_f32_e64 v206, v206 clamp\n\t" \
  "v_exp_f32_e64 v207, v207 clamp\n\t" \
  "v_pk_fma_f32 v[204:205], v[204:205], s[60:61], s[60:61] op_sel:[0,1,0] op_sel_hi:[1,1,0]\n\t" \
  "v_pk_fma_f32 v[206:207], v[206:207], s[60:61], s[60:61] op_sel:[0,1,0] op_sel_hi:[1,1,0]\n\t" \
  "v_rcp_f32 v204, v204\n\t" \
  "v_rcp_f32 v205, v205\n\t" \
  "v_rcp_f32 v206, v206\n\t" \
  "v_rcp_f32 v207, v207\n\t" \
  "v_pk_mul_f32 v[200:201], v[200:201], v[204:205]\n\t" \
  "v_pk_mul_f32 v[202:203], v[202:203], v[206:207]\n\t" \
  "s_add_u32 s57, s80, 2162688\n\t" \
  "buffer_store_dwordx4 v[200:203], v253, s[44:47], s57 offen offset:0 \n\t" \
  "v_accvgpr_read_b32 v208, a162\n\t" \
  "v_accvgpr_read_b32 v209, a166\n\t" \
  "v_accvgpr_read_b32 v210, a170\n\t" \
  "v_accvgpr_read_b32 v211, a174\n\t" \
  "v_pk_fma_f32 v[212:213], v[208:209], s[58:59], s[58:59] op_sel:[0,0,1] op_sel_hi:[1,0,1]\n\t" \
  "v_pk_fma_f32 v[214:215], v[210:211], s[58:59], s[58:59] op_sel:[0,0,1] op_sel_hi:[1,0,1]\n\t" \
  "v_exp_f32_e64 v212, v212 clamp\n\t" \
  "v_exp_f32_e64 v213, v213 clamp\n\t" \
  "v_exp_f32_e64 v214, v214 clamp\n\t" \
  "v_exp_f32_e64 v215, v215 clamp\n\t" \
  "v_pk_fma_f32 v[212:213], v[212:213], s[60:61], s[60:61] op_sel:[0,1,0] op_sel_hi:[1,1,0]\n\t" \
  "v_pk_fma_f32 v[214:215], v[214:215], s[60:61], s[60:61] op_sel:[0,1,0] op_sel_hi:[1,1,0]\n\t" \
  "v_rcp_f32 v212, v212\n\t" \
  "v_rcp_f32 v213, v213\n\t" \
  "v_rcp_f32 v214, v214\n\t" \
  "v_rcp_f32 v215, v215\n\t" \
  "v_pk_mul_f32 v[208:209], v[208:209], v[212:213]\n\t" \
  "v_pk_mul_f32 v[210:211], v[210:211], v[214:215]\n\t" \
  "s_add_u32 s57, s80, 2228224\n\t" \
  "buffer_store_dwordx4 v[208:211], v253, s[44:47], s57 offen offset:0 \n\t" \
  "v_accvgpr_read_b32 v216, a163\n\t" \
  "v_accvgpr_read_b32 v217, a167\n\t" \
  "v_accvgpr_read_b32 v218, a171\n\t" \
  "v_accvgpr_read_b32 v219, a175\n\t" \
  "v_pk_fma_f32 v[220:221], v[216:217], s[58:59], s[58:59] op_sel:[0,0,1] op_sel_hi:[1,0,1]\n\t" \
  "v_pk_fma_f32 v[222:223], v[218:219], s[58:59], s[58:59] op_sel:[0,0,1] op_sel_hi:[1,0,1]\n\t" \
  "v_exp_f32_e64 v220, v220 clamp\n\t" \
  "v_exp_f32_e64 v221, v221 clamp\n\t" \
  "v_exp_f32_e64 v222, v222 clamp\n\t" \
  "v_exp_f32_e64 v223, v223 clamp\n\t" \
  "v_pk_fma_f32 v[220:221], v[220:221], s[60:61], s[60:61] op_sel:[0,1,0] op_sel_hi:[1,1,0]\n\t" \
  "v_pk_fma_f32 v[222:223], v[222:223], s[60:61], s[60:61] op_sel:[0,1,0] op_sel_hi:[1,1,0]\n\t" \
  "v_rcp_f32 v220, v220\n\t" \
  "v_rcp_f32 v221, v221\n\t" \
  "v_rcp_f32 v222, v222\n\t" \
  "v_rcp_f32 v223, v223\n\t" \
  "v_pk_mul_f32 v[216:217], v[216:217], v[220:221]\n\t" \
  "v_pk_mul_f32 v[218:219], v[218:219], v[222:223]\n\t" \
  "s_add_u32 s57, s80, 2293760\n\t" \
  "buffer_store_dwordx4 v[216:219], v253, s[44:47], s57 offen offset:0 \n\t" \
  "v_accvgpr_read_b32 v192, a176\n\t" \
  "v_accvgpr_read_b32 v193, a180\n\t" \
  "v_accvgpr_read_b32 v194, a184\n\t" \
  "v_accvgpr_read_b32 v195, a188\n\t" \
  "v_pk_fma_f32 v[196:197], v[192:193], s[58:59], s[58:59] op_sel:[0,0,1] op_sel_hi:[1,0,1]\n\t" \
  "v_pk_fma_f32 v[198:199], v[194:195], s[58:59], s[58:59] op_sel:[0,0,1] op_sel_hi:[1,0,1]\n\t" \
  "v_exp_f32_e64 v196, v196 clamp\n\t" \
  "v_exp_f32_e64 v197, v197 clamp\n\t" \
  "v_exp_f32_e64 v198, v198 clamp\n\t" \
  "v_exp_f32_e64 v199, v199 clamp\n\t" \
  "v_pk_fma_f32 v[196:197], v[196:197], s[60:61], s[60:61] op_sel:[0,1,0] op_sel_hi:[1,1,0]\n\t" \
  "v_pk_fma_f32 v[198:199], v[198:199], s[60:61], s[60:61] op_sel:[0,1,0] op_sel_hi:[1,1,0]\n\t" \
  "v_rcp_f32 v196, v196\n\t" \
  "v_rcp_f32 v197, v197\n\t" \
  "v_rcp_f32 v198, v198\n\t" \
  "v_rcp_f32 v199, v199\n\t" \
  "v_pk_mul_f32 v[192:193], v[192:193], v[196:197]\n\t" \
  "v_pk_mul_f32 v[194:195], v[194:195], v[198:199]\n\t" \
  "s_add_u32 s57, s80, 3145728\n\t" \
  "buffer_store_dwordx4 v[192:195], v253, s[44:47], s57 offen offset:0 nt\n\t" \
  "v_accvgpr_read_b32 v200, a177\n\t" \
  "v_accvgpr_read_b32 v201, a181\n\t" \
  "v_accvgpr_read_b32 v202, a185\n\t" \
  "v_accvgpr_read_b32 v203, a189\n\t" \
  "v_pk_fma_f32 v[204:205], v[200:201], s[58:59], s[58:59] op_sel:[0,0,1] op_sel_hi:[1,0,1]\n\t" \
  "v_pk_fma_f32 v[206:207], v[202:203], s[58:59], s[58:59] op_sel:[0,0,1] op_sel_hi:[1,0,1]\n\t" \
  "v_exp_f32_e64 v204, v204 clamp\n\t" \
  "v_exp_f32_e64 v205, v205 clamp\n\t" \
  "v_exp_f32_e64 v206, v206 clamp\n\t" \
  "v_exp_f32_e64 v207, v207 clamp\n\t" \
  "v_pk_fma_f32 v[204:205], v[204:205], s[60:61], s[60:61] op_sel:[0,1,0] op_sel_hi:[1,1,0]\n\t" \
  "v_pk_fma_f32 v[206:207], v[206:207], s[60:61], s[60:61] op_sel:[0,1,0] op_sel_hi:[1,1,0]\n\t" \
  "v_rcp_f32 v204, v204\n\t" \
  "v_rcp_f32 v205, v205\n\t" \
  "v_rcp_f32 v206, v206\n\t" \
  "v_rcp_f32 v207, v207\n\t" \
  "v_pk_mul_f32 v[200:201], v[200:201], v[204:205]\n\t" \
  "v_pk_mul_f32 v[202:203], v[202:203], v[206:207]\n\t" \
  "s_add_u32 s57, s80, 3211264\n\t" \
  "buffer_store_dwordx4 v[200:203], v253, s[44:47], s57 offen offset:0 nt\n\t" \
  "v_accvgpr_read_b32 v208, a178\n\t" \
  "v_accvgpr_read_b32 v209, a182\n\t" \
  "v_accvgpr_read_b32 v210, a186\n\t" \
  "v_accvgpr_read_b32 v211, a190\n\t" \
  "v_pk_fma_f32 v[212:213], v[208:209], s[58:59], s[58:59] op_sel:[0,0,1] op_sel_hi:[1,0,1]\n\t" \
  "v_pk_fma_f32 v[214:215], v[210:211], s[58:59], s[58:59] op_sel:[0,0,1] op_sel_hi:[1,0,1]\n\t" \
  "v_exp_f32_e64 v212, v212 clamp\n\t" \
  "v_exp_f32_e64 v213, v213 clamp\n\t" \
  "v_exp_f32_e64 v214, v214 clamp\n\t" \
  "v_exp_f32_e64 v215, v215 clamp\n\t" \
  "v_pk_fma_f32 v[212:213], v[212:213], s[60:61], s[60:61] op_sel:[0,1,0] op_sel_hi:[1,1,0]\n\t" \
  "v_pk_fma_f32 v[214:215], v[214:215], s[60:61], s[60:61] op_sel:[0,1,0] op_sel_hi:[1,1,0]\n\t" \
  "v_rcp_f32 v212, v212\n\t" \
  "v_rcp_f32 v213, v213\n\t" \
  "v_rcp_f32 v214, v214\n\t" \
  "v_rcp_f32 v215, v215\n\t" \
  "v_pk_mul_f32 v[208:209], v[208:209], v[212:213]\n\t" \
  "v_pk_mul_f32 v[210:211], v[210:211], v[214:215]\n\t" \
  "s_add_u32 s57, s80, 3276800\n\t" \
  "buffer_store_dwordx4 v[208:211], v253, s[44:47], s57 offen offset:0 nt\n\t" \
  "v_accvgpr_read_b32 v216, a179\n\t" \
  "v_accvgpr_read_b32 v217, a183\n\t" \
  "v_accvgpr_read_b32 v218, a187\n\t" \
  "v_accvgpr_read_b32 v219, a191\n\t" \
  "v_pk_fma_f32 v[220:221], v[216:217], s[58:59], s[58:59] op_sel:[0,0,1] op_sel_hi:[1,0,1]\n\t" \
  "v_pk_fma_f32 v[222:223], v[218:219], s[58:59], s[58:59] op_sel:[0,0,1] op_sel_hi:[1,0,1]\n\t" \
  "v_exp_f32_e64 v220, v220 clamp\n\t" \
  "v_exp_f32_e64 v221, v221 clamp\n\t" \
  "v_exp_f32_e64 v222, v222 clamp\n\t" \
  "v_exp_f32_e64 v223, v223 clamp\n\t" \
  "v_pk_fma_f32 v[220:221], v[220:221], s[60:61], s[60:61] op_sel:[0,1,0] op_sel_hi:[1,1,0]\n\t" \
  "v_pk_fma_f32 v[222:223], v[222:223], s[60:61], s[60:61] op_sel:[0,1,0] op_sel_hi:[1,1,0]\n\t" \
  "v_rcp_f32 v220, v220\n\t" \
  "v_rcp_f32 v221, v221\n\t" \
  "v_rcp_f32 v222, v222\n\t" \
  "v_rcp_f32 v223, v223\n\t" \
  "v_pk_mul_f32 v[216:217], v[216:217], v[220:221]\n\t" \
  "v_pk_mul_f32 v[218:219], v[218:219], v[222:223]\n\t" \
  "s_add_u32 s57, s80, 3342336\n\t" \
  "buffer_store_dwordx4 v[216:219], v253, s[44:47], s57 offen offset:0 nt\n\t" \
  "v_pk_fma_f32 v[200:201], v[64:65], s[58:59], s[58:59] op_sel:[0,0,1] op_sel_hi:[1,0,1]\n\t" \
  "v_pk_fma_f32 v[202:203], v[68:69], s[58:59], s[58:59] op_sel:[0,0,1] op_sel_hi:[1,0,1]\n\t" \
  "v_pk_fma_f32 v[204:205], v[72:73], s[58:59], s[58:59] op_sel:[0,0,1] op_sel_hi:[1,0,1]\n\t" \
  "v_pk_fma_f32 v[206:207], v[76:77], s[58:59], s[58:59] op_sel:[0,0,1] op_sel_hi:[1,0,1]\n\t" \
  "v_exp_f32_e64 v192, v200 clamp\n\t" \
  "v_exp_f32_e64 v193, v202 clamp\n\t" \
  "v_exp_f32_e64 v194, v204 clamp\n\t" \
  "v_exp_f32_e64 v195, v206 clamp\n\t" \
  "v_exp_f32_e64 v196, v201 clamp\n\t" \
  "v_exp_f32_e64 v197, v203 clamp\n\t" \
  "v_exp_f32_e64 v198, v205 clamp\n\t" \
  "v_exp_f32_e64 v199, v207 clamp\n\t" \
  "v_pk_fma_f32 v[192:193], v[192:193], s[60:61], s[60:61] op_sel:[0,1,0] op_sel_hi:[1,1,0]\n\t" \
  "v_pk_fma_f32 v[194:195], v[194:195], s[60:61], s[60:61] op_sel:[0,1,0] op_sel_hi:[1,1,0]\n\t" \
  "v_pk_fma_f32 v[196:197], v[196:197], s[60:61], s[60:61] op_sel:[0,1,0] op_sel_hi:[1,1,0]\n\t" \
  "v_pk_fma_f32 v[198:199], v[198:199], s[60:61], s[60:61] op_sel:[0,1,0] op_sel_hi:[1,1,0]\n\t" \
  "v_rcp_f32 v192, v192\n\t" \
  "v_rcp_f32 v193, v193\n\t" \
  "v_rcp_f32 v194, v194\n\t" \
  "v_rcp_f32 v195, v195\n\t" \
  "v_rcp_f32 v196, v196\n\t" \
  "v_rcp_f32 v197, v197\n\t" \
  "v_rcp_f32 v198, v198\n\t" \
  "v_rcp_f32 v199, v199\n\t" \
  "v_mul_f32 v192, v64, v192\n\t" \
  "v_mul_f32 v193, v68, v193\n\t" \
  "v_mul_f32 v194, v72, v194\n\t" \
  "v_mul_f32 v195, v76, v195\n\t" \
  "s_add_u32 s57, s80, 4194304\n\t" \
  "buffer_store_dwordx4 v[192:195], v253, s[44:47], s57 offen offset:0 \n\t" \
  "v_mul_f32 v196, v65, v196\n\t" \
  "v_mul_f32 v197, v69, v197\n\t" \
  "v_mul_f32 v198, v73, v198\n\t" \
  "v_mul_f32 v199, v77, v199\n\t" \
  "s_add_u32 s57, s80, 4259840\n\t" \
  "buffer_store_dwordx4 v[196:199], v253, s[44:47], s57 offen offset:0 \n\t" \
  "v_pk_fma_f32 v[216:217], v[66:67], s[58:59], s[58:59] op_sel:[0,0,1] op_sel_hi:[1,0,1]\n\t" \
  "v_pk_fma_f32 v[218:219], v[70:71], s[58:59], s[58:59] op_sel:[0,0,1] op_sel_hi:[1,0,1]\n\t" \
  "v_pk_fma_f32 v[220:221], v[74:75], s[58:59], s[58:59] op_sel:[0,0,1] op_sel_hi:[1,0,1]\n\t" \
  "v_pk_fma_f32 v[222:223], v[78:79], s[58:59], s[58:59] op_sel:[0,0,1] op_sel_hi:[1,0,1]\n\t" \
  "v_exp_f32_e64 v208, v216 clamp\n\t" \
  "v_exp_f32_e64 v209, v218 clamp\n\t" \
  "v_exp_f32_e64 v210, v220 clamp\n\t" \
  "v_exp_f32_e64 v211, v222 clamp\n\t" \
  "v_exp_f32_e64 v212, v217 clamp\n\t" \
  "v_exp_f32_e64 v213, v219 clamp\n\t" \
  "v_exp_f32_e64 v214, v221 clamp\n\t" \
  "v_exp_f32_e64 v215, v223 clamp\n\t" \
  "v_pk_fma_f32 v[208:209], v[208:209], s[60:61], s[60:61] op_sel:[0,1,0] op_sel_hi:[1,1,0]\n\t" \
  "v_pk_fma_f32 v[210:211], v[210:211], s[60:61], s[60:61] op_sel:[0,1,0] op_sel_hi:[1,1,0]\n\t" \
  "v_pk_fma_f32 v[212:213], v[212:213], s[60:61], s[60:61] op_sel:[0,1,0] op_sel_hi:[1,1,0]\n\t" \
  "v_pk_fma_f32 v[214:215], v[214:215], s[60:61], s[60:61] op_sel:[0,1,0] op_sel_hi:[1,1,0]\n\t" \
  "v_rcp_f32 v208, v208\n\t" \
  "v_rcp_f32 v209, v209\n\t" \
  "v_rcp_f32 v210, v210\n\t" \
  "v_rcp_f32 v211, v211\n\t" \
  "v_rcp_f32 v212, v212\n\t" \
  "v_rcp_f32 v213, v213\n\t" \
  "v_rcp_f32 v214, v214\n\t" \
  "v_rcp_f32 v215, v215\n\t" \
  "v_mul_f32 v208, v66, v208\n\t" \
  "v_mul_f32 v209, v70, v209\n\t" \
  "v_mul_f32 v210, v74, v210\n\t" \
  "v_mul_f32 v211, v78, v211\n\t" \
  "s_add_u32 s57, s80, 4325376\n\t" \
  "buffer_store_dwordx4 v[208:211], v253, s[44:47], s57 offen offset:0 \n\t" \
  "v_mul_f32 v212, v67, v212\n\t" \
  "v_mul_f32 v213, v71, v213\n\t" \
  "v_mul_f32 v214, v75, v214\n\t" \
  "v_mul_f32 v215, v79, v215\n\t" \
  "s_add_u32 s57, s80, 4390912\n\t" \
  "buffer_store_dwordx4 v[212:215], v253, s[44:47], s57 offen offset:0 \n\t" \
  "v_pk_fma_f32 v[200:201], v[80:81], s[58:59], s[58:59] op_sel:[0,0,1] op_sel_hi:[1,0,1]\n\t" \
  "v_pk_fma_f32 v[202:203], v[84:85], s[58:59], s[58:59] op_sel:[0,0,1] op_sel_hi:[1,0,1]\n\t" \
  "v_pk_fma_f32 v[204:205], v[88:89], s[58:59], s[58:59] op_sel:[0,0,1] op_sel_hi:[1,0,1]\n\t" \
  "v_pk_fma_f32 v[206:207], v[92:93], s[58:59], s[58:59] op_sel:[0,0,1] op_sel_hi:[1,0,1]\n\t" \
  "v_exp_f32_e64 v192, v200 clamp\n\t" \
  "v_exp_f32_e64 v193, v202 clamp\n\t" \
  "v_exp_f32_e64 v194, v204 clamp\n\t" \
  "v_exp_f32_e64 v195, v206 clamp\n\t" \
  "v_exp_f32_e64 v196, v201 clamp\n\t" \
  "v_exp_f32_e64 v197, v203 clamp\n\t" \
  "v_exp_f32_e64 v198, v205 clamp\n\t" \
  "v_exp_f32_e64 v199, v207 clamp\n\t" \
  "v_pk_fma_f32 v[192:193], v[192:193], s[60:61], s[60:61] op_sel:[0,1,0] op_sel_hi:[1,1,0]\n\t" \
  "v_pk_fma_f32 v[194:195], v[194:195], s[60:61], s[60:61] op_sel:[0,1,0] op_sel_hi:[1,1,0]\n\t" \
  "v_pk_fma_f32 v[196:197], v[196:197], s[60:61], s[60:61] op_sel:[0,1,0] op_sel_hi:[1,1,0]\n\t" \
  "v_pk_fma_f32 v[198:199], v[198:199], s[60:61], s[60:61] op_sel:[0,1,0] op_sel_hi:[1,1,0]\n\t" \
  "v_rcp_f32 v192, v192\n\t" \
  "v_rcp_f32 v193, v193\n\t" \
  "v_rcp_f32 v194, v194\n\t" \
  "v_rcp_f32 v195, v195\n\t" \
  "v_rcp_f32 v196, v196\n\t" \
  "v_rcp_f32 v197, v197\n\t" \
  "v_rcp_f32 v198, v198\n\t" \
  "v_rcp_f32 v199, v199\n\t" \
  "v_mul_f32 v192, v80, v192\n\t" \
  "v_mul_f32 v193, v84, v193\n\t" \
  "v_mul_f32 v194, v88, v194\n\t" \
  "v_mul_f32 v195, v92, v195\n\t" \
  "s_add_u32 s57, s80, 5242880\n\t" \
  "buffer_store_dwordx4 v[192:195], v253, s[44:47], s57 offen offset:0 nt\n\t" \
  "v_mul_f32 v196, v81, v196\n\t" \
  "v_mul_f32 v197, v85, v197\n\t" \
  "v_mul_f32 v198, v89, v198\n\t" \
  "v_mul_f32 v199, v93, v199\n\t" \
  "s_add_u32 s57, s80, 5308416\n\t" \
  "buffer_store_dwordx4 v[196:199], v253, s[44:47], s57 offen offset:0 nt\n\t" \
  "v_pk_fma_f32 v[216:217], v[82:83], s[58:59], s[58:59] op_sel:[0,0,1] op_sel_hi:[1,0,1]\n\t" \
  "v_pk_fma_f32 v[218:219], v[86:87], s[58:59], s[58:59] op_sel:[0,0,1] op_sel_hi:[1,0,1]\n\t" \
  "v_pk_fma_f32 v[220:221], v[90:91], s[58:59], s[58:59] op_sel:[0,0,1] op_sel_hi:[1,0,1]\n\t" \
  "v_pk_fma_f32 v[222:223], v[94:95], s[58:59], s[58:59] op_sel:[0,0,1] op_sel_hi:[1,0,1]\n\t" \
  "v_exp_f32_e64 v208, v216 clamp\n\t" \
  "v_exp_f32_e64 v209, v218 clamp\n\t" \
  "v_exp_f32_e64 v210, v220 clamp\n\t" \
  "v_exp_f32_e64 v211, v222 clamp\n\t" \
  "v_exp_f32_e64 v212, v217 clamp\n\t" \
  "v_exp_f32_e64 v213, v219 clamp\n\t" \
  "v_exp_f32_e64 v214, v221 clamp\n\t" \
  "v_exp_f32_e64 v215, v223 clamp\n\t" \
  "v_pk_fma_f32 v[208:209], v[208:209], s[60:61], s[60:61] op_sel:[0,1,0] op_sel_hi:[1,1,0]\n\t" \
  "v_pk_fma_f32 v[210:211], v[210:211], s[60:61], s[60:61] op_sel:[0,1,0] op_sel_hi:[1,1,0]\n\t" \
  "v_pk_fma_f32 v[212:213], v[212:213], s[60:61], s[60:61] op_sel:[0,1,0] op_sel_hi:[1,1,0]\n\t" \
  "v_pk_fma_f32 v[214:215], v[214:215], s[60:61], s[60:61] op_sel:[0,1,0] op_sel_hi:[1,1,0]\n\t" \
  "v_rcp_f32 v208, v208\n\t" \
  "v_rcp_f32 v209, v209\n\t" \
  "v_rcp_f32 v210, v210\n\t" \
  "v_rcp_f32 v211, v211\n\t" \
  "v_rcp_f32 v212, v212\n\t" \
  "v_rcp_f32 v213, v213\n\t" \
  "v_rcp_f32 v214, v214\n\t" \
  "v_rcp_f32 v215, v215\n\t" \
  "v_mul_f32 v208, v82, v208\n\t" \
  "v_mul_f32 v209, v86, v209\n\t" \
  "v_mul_f32 v210, v90, v210\n\t" \
  "v_mul_f32 v211, v94, v211\n\t" \
  "s_add_u32 s57, s80, 5373952\n\t" \
  "buffer_store_dwordx4 v[208:211], v253, s[44:47], s57 offen offset:0 nt\n\t" \
  "v_mul_f32 v212, v83, v212\n\t" \
  "v_mul_f32 v213, v87, v213\n\t" \
  "v_mul_f32 v214, v91, v214\n\t" \
  "v_mul_f32 v215, v95, v215\n\t" \
  "s_add_u32 s57, s80, 5439488\n\t" \
  "buffer_store_dwordx4 v[212:215], v253, s[44:47], s57 offen offset:0 nt\n\t" \
  "v_pk_fma_f32 v[200:201], v[96:97], s[58:59], s[58:59] op_sel:[0,0,1] op_sel_hi:[1,0,1]\n\t" \
  "v_pk_fma_f32 v[202:203], v[100:101], s[58:59], s[58:59] op_sel:[0,0,1] op_sel_hi:[1,0,1]\n\t" \
  "v_pk_fma_f32 v[204:205], v[104:105], s[58:59], s[58:59] op_sel:[0,0,1] op_sel_hi:[1,0,1]\n\t" \
  "v_pk_fma_f32 v[206:207], v[108:109], s[58:59], s[58:59] op_sel:[0,0,1] op_sel_hi:[1,0,1]\n\t" \
  "v_exp_f32_e64 v192, v200 clamp\n\t" \
  "v_exp_f32_e64 v193, v202 clamp\n\t" \
  "v_exp_f32_e64 v194, v204 clamp\n\t" \
  "v_exp_f32_e64 v195, v206 clamp\n\t" \
  "v_exp_f32_e64 v196, v201 clamp\n\t" \
  "v_exp_f32_e64 v197, v203 clamp\n\t" \
  "v_exp_f32_e64 v198, v205 clamp\n\t" \
  "v_exp_f32_e64 v199, v207 clamp\n\t" \
  "v_pk_fma_f32 v[192:193], v[192:193], s[60:61], s[60:61] op_sel:[0,1,0] op_sel_hi:[1,1,0]\n\t" \
  "v_pk_fma_f32 v[194:195], v[194:195], s[60:61], s[60:61] op_sel:[0,1,0] op_sel_hi:[1,1,0]\n\t" \
  "v_pk_fma_f32 v[196:197], v[196:197], s[60:61], s[60:61] op_sel:[0,1,0] op_sel_hi:[1,1,0]\n\t" \
  "v_pk_fma_f32 v[198:199], v[198:199], s[60:61], s[60:61] op_sel:[0,1,0] op_sel_hi:[1,1,0]\n\t" \
  "v_rcp_f32 v192, v192\n\t" \
  "v_rcp_f32 v193, v193\n\t" \
  "v_rcp_f32 v194, v194\n\t" \
  "v_rcp_f32 v195, v195\n\t" \
  "v_rcp_f32 v196, v196\n\t" \
  "v_rcp_f32 v197, v197\n\t" \
  "v_rcp_f32 v198, v198\n\t" \
  "v_rcp_f32 v199, v199\n\t" \
  "v_mul_f32 v192, v96, v192\n\t" \
  "v_mul_f32 v193, v100, v193\n\t" \
  "v_mul_f32 v194, v104, v194\n\t" \
  "v_mul_f32 v195, v108, v195\n\t" \
  "s_add_u32 s57, s80, 6291456\n\t" \
  "buffer_store_dwordx4 v[192:195], v253, s[44:47], s57 offen offset:0 \n\t" \
  "v_mul_f32 v196, v97, v196\n\t" \
  "v_mul_f32 v197, v101, v197\n\t" \
  "v_mul_f32 v198, v105, v198\n\t" \
  "v_mul_f32 v199, v109, v199\n\t" \
  "s_add_u32 s57, s80, 6356992\n\t" \
  "buffer_store_dwordx4 v[196:199], v253, s[44:47], s57 offen offset:0 \n\t" \
  "v_pk_fma_f32 v[216:217], v[98:99], s[58:59], s[58:59] op_sel:[0,0,1] op_sel_hi:[1,0,1]\n\t" \
  "v_pk_fma_f32 v[218:219], v[102:103], s[58:59], s[58:59] op_sel:[0,0,1] op_sel_hi:[1,0,1]\n\t" \
  "v_pk_fma_f32 v[220:221], v[106:107], s[58:59], s[58:59] op_sel:[0,0,1] op_sel_hi:[1,0,1]\n\t" \
  "v_pk_fma_f32 v[222:223], v[110:111], s[58:59], s[58:59] op_sel:[0,0,1] op_sel_hi:[1,0,1]\n\t" \
  "v_exp_f32_e64 v208, v216 clamp\n\t" \
  "v_exp_f32_e64 v209, v218 clamp\n\t" \
  "v_exp_f32_e64 v210, v220 clamp\n\t" \
  "v_exp_f32_e64 v211, v222 clamp\n\t" \
  "v_exp_f32_e64 v212, v217 clamp\n\t" \
  "v_exp_f32_e64 v213, v219 clamp\n\t" \
  "v_exp_f32_e64 v214, v221 clamp\n\t" \
  "v_exp_f32_e64 v215, v223 clamp\n\t" \
  "v_pk_fma_f32 v[208:209], v[208:209], s[60:61], s[60:61] op_sel:[0,1,0] op_sel_hi:[1,1,0]\n\t" \
  "v_pk_fma_f32 v[210:211], v[210:211], s[60:61], s[60:61] op_sel:[0,1,0] op_sel_hi:[1,1,0]\n\t" \
  "v_pk_fma_f32 v[212:213], v[212:213], s[60:61], s[60:61] op_sel:[0,1,0] op_sel_hi:[1,1,0]\n\t" \
  "v_pk_fma_f32 v[214:215], v[214:215], s[60:61], s[60:61] op_sel:[0,1,0] op_sel_hi:[1,1,0]\n\t" \
  "v_rcp_f32 v208, v208\n\t" \
  "v_rcp_f32 v209, v209\n\t" \
  "v_rcp_f32 v210, v210\n\t" \
  "v_rcp_f32 v211, v211\n\t" \
  "v_rcp_f32 v212, v212\n\t" \
  "v_rcp_f32 v213, v213\n\t" \
  "v_rcp_f32 v214, v214\n\t" \
  "v_rcp_f32 v215, v215\n\t" \
  "v_mul_f32 v208, v98, v208\n\t" \
  "v_mul_f32 v209, v102, v209\n\t" \
  "v_mul_f32 v210, v106, v210\n\t" \
  "v_mul_f32 v211, v110, v211\n\t" \
  "s_add_u32 s57, s80, 6422528\n\t" \
  "buffer_store_dwordx4 v[208:211], v253, s[44:47], s57 offen offset:0 \n\t" \
  "v_mul_f32 v212, v99, v212\n\t" \
  "v_mul_f32 v213, v103, v213\n\t" \
  "v_mul_f32 v214, v107, v214\n\t" \
  "v_mul_f32 v215, v111, v215\n\t" \
  "s_add_u32 s57, s80, 6488064\n\t" \
  "buffer_store_dwordx4 v[212:215], v253, s[44:47], s57 offen offset:0 \n\t" \
  "v_pk_fma_f32 v[200:201], v[224:225], s[58:59], s[58:59] op_sel:[0,0,1] op_sel_hi:[1,0,1]\n\t" \
  "v_pk_fma_f32 v[202:203], v[228:229], s[58:59], s[58:59] op_sel:[0,0,1] op_sel_hi:[1,0,1]\n\t" \
  "v_pk_fma_f32 v[204:205], v[232:233], s[58:59], s[58:59] op_sel:[0,0,1] op_sel_hi:[1,0,1]\n\t" \
  "v_pk_fma_f32 v[206:207], v[236:237], s[58:59], s[58:59] op_sel:[0,0,1] op_sel_hi:[1,0,1]\n\t" \
  "v_exp_f32_e64 v192, v200 clamp\n\t" \
  "v_exp_f32_e64 v193, v202 clamp\n\t" \
  "v_exp_f32_e64 v194, v204 clamp\n\t" \
  "v_exp_f32_e64 v195, v206 clamp\n\t" \
  "v_exp_f32_e64 v196, v201 clamp\n\t" \
  "v_exp_f32_e64 v197, v203 clamp\n\t" \
  "v_exp_f32_e64 v198, v205 clamp\n\t" \
  "v_exp_f32_e64 v199, v207 clamp\n\t" \
  "v_pk_fma_f32 v[192:193], v[192:193], s[60:61], s[60:61] op_sel:[0,1,0] op_sel_hi:[1,1,0]\n\t" \
  "v_pk_fma_f32 v[194:195], v[194:195], s[60:61], s[60:61] op_sel:[0,1,0] op_sel_hi:[1,1,0]\n\t" \
  "v_pk_fma_f32 v[196:197], v[196:197], s[60:61], s[60:61] op_sel:[0,1,0] op_sel_hi:[1,1,0]\n\t" \
  "v_pk_fma_f32 v[198:199], v[198:199], s[60:61], s[60:61] op_sel:[0,1,0] op_sel_hi:[1,1,0]\n\t" \
  "v_rcp_f32 v192, v192\n\t" \
  "v_rcp_f32 v193, v193\n\t" \
  "v_rcp_f32 v194, v194\n\t" \
  "v_rcp_f32 v195, v195\n\t" \
  "v_rcp_f32 v196, v196\n\t" \
  "v_rcp_f32 v197, v197\n\t" \
  "v_rcp_f32 v198, v198\n\t" \
  "v_rcp_f32 v199, v199\n\t" \
  "v_mul_f32 v192, v224, v192\n\t" \
  "v_mul_f32 v193, v228, v193\n\t" \
  "v_mul_f32 v194, v232, v194\n\t" \
  "v_mul_f32 v195, v236, v195\n\t" \
  "s_add_u32 s57, s80, 7340032\n\t" \
  "buffer_store_dwordx4 v[192:195], v253, s[44:47], s57 offen offset:0 nt\n\t" \
  "v_mul_f32 v196, v225, v196\n\t" \
  "v_mul_f32 v197, v229, v197\n\t" \
  "v_mul_f32 v198, v233, v198\n\t" \
  "v_mul_f32 v199, v237, v199\n\t" \
  "s_add_u32 s57, s80, 7405568\n\t" \
  "buffer_store_dwordx4 v[196:199], v253, s[44:47], s57 offen offset:0 nt\n\t" \
  "v_pk_fma_f32 v[216:217], v[226:227], s[58:59], s[58:59] op_sel:[0,0,1] op_sel_hi:[1,0,1]\n\t" \
  "v_pk_fma_f32 v[218:219], v[230:231], s[58:59], s[58:59] op_sel:[0,0,1] op_sel_hi:[1,0,1]\n\t" \
  "v_pk_fma_f32 v[220:221], v[234:235], s[58:59], s[58:59] op_sel:[0,0,1] op_sel_hi:[1,0,1]\n\t" \
  "v_pk_fma_f32 v[222:223], v[238:239], s[58:59], s[58:59] op_sel:[0,0,1] op_sel_hi:[1,0,1]\n\t" \
  "v_exp_f32_e64 v208, v216 clamp\n\t" \
  "v_exp_f32_e64 v209, v218 clamp\n\t" \
  "v_exp_f32_e64 v210, v220 clamp\n\t" \
  "v_exp_f32_e64 v211, v222 clamp\n\t" \
  "v_exp_f32_e64 v212, v217 clamp\n\t" \
  "v_exp_f32_e64 v213, v219 clamp\n\t" \
  "v_exp_f32_e64 v214, v221 clamp\n\t" \
  "v_exp_f32_e64 v215, v223 clamp\n\t" \
  "v_pk_fma_f32 v[208:209], v[208:209], s[60:61], s[60:61] op_sel:[0,1,0] op_sel_hi:[1,1,0]\n\t" \
  "v_pk_fma_f32 v[210:211], v[210:211], s[60:61], s[60:61] op_sel:[0,1,0] op_sel_hi:[1,1,0]\n\t" \
  "v_pk_fma_f32 v[212:213], v[212:213], s[60:61], s[60:61] op_sel:[0,1,0] op_sel_hi:[1,1,0]\n\t" \
  "v_pk_fma_f32 v[214:215], v[214:215], s[60:61], s[60:61] op_sel:[0,1,0] op_sel_hi:[1,1,0]\n\t" \
  "v_rcp_f32 v208, v208\n\t" \
  "v_rcp_f32 v209, v209\n\t" \
  "v_rcp_f32 v210, v210\n\t" \
  "v_rcp_f32 v211, v211\n\t" \
  "v_rcp_f32 v212, v212\n\t" \
  "v_rcp_f32 v213, v213\n\t" \
  "v_rcp_f32 v214, v214\n\t" \
  "v_rcp_f32 v215, v215\n\t" \
  "v_mul_f32 v208, v226, v208\n\t" \
  "v_mul_f32 v209, v230, v209\n\t" \
  "v_mul_f32 v210, v234, v210\n\t" \
  "v_mul_f32 v211, v238, v211\n\t" \
  "s_add_u32 s57, s80, 7471104\n\t" \
  "buffer_store_dwordx4 v[208:211], v253, s[44:47], s57 offen offset:0 nt\n\t" \
  "v_mul_f32 v212, v227, v212\n\t" \
  "v_mul_f32 v213, v231, v213\n\t" \
  "v_mul_f32 v214, v235, v214\n\t" \
  "v_mul_f32 v215, v239, v215\n\t" \
  "s_add_u32 s57, s80, 7536640\n\t" \
  "buffer_store_dwordx4 v[212:215], v253, s[44:47], s57 offen offset:0 nt\n\t" \
  "v_accvgpr_read_b32 v192, a192\n\t" \
  "v_accvgpr_read_b32 v193, a196\n\t" \
  "v_accvgpr_read_b32 v194, a200\n\t" \
  "v_accvgpr_read_b32 v195, a204\n\t" \
  "v_pk_fma_f32 v[196:197], v[192:193], s[58:59], s[58:59] op_sel:[0,0,1] op_sel_hi:[1,0,1]\n\t" \
  "v_pk_fma_f32 v[198:199], v[194:195], s[58:59], s[58:59] op_sel:[0,0,1] op_sel_hi:[1,0,1]\n\t" \
  "v_exp_f32_e64 v196, v196 clamp\n\t" \
  "v_exp_f32_e64 v197, v197 clamp\n\t" \
  "v_exp_f32_e64 v198, v198 clamp\n\t" \
  "v_exp_f32_e64 v199, v199 clamp\n\t" \
  "v_pk_fma_f32 v[196:197], v[196:197], s[60:61], s[60:61] op_sel:[0,1,0] op_sel_hi:[1,1,0]\n\t" \
  "v_pk_fma_f32 v[198:199], v[198:199], s[60:61], s[60:61] op_sel:[0,1,0] op_sel_hi:[1,1,0]\n\t" \
  "v_rcp_f32 v196, v196\n\t" \
  "v_rcp_f32 v197, v197\n\t" \
  "v_rcp_f32 v198, v198\n\t" \
  "v_rcp_f32 v199, v199\n\t" \
  "v_pk_mul_f32 v[192:193], v[192:193], v[196:197]\n\t" \
  "v_pk_mul_f32 v[194:195], v[194:195], v[198:199]\n\t" \
  "s_add_u32 s57, s80, 4194304\n\t" \
  "buffer_store_dwordx4 v[192:195], v253, s[44:47], s57 offen offset:256 \n\t" \
  "v_accvgpr_read_b32 v200, a193\n\t" \
  "v_accvgpr_read_b32 v201, a197\n\t" \
  "v_accvgpr_read_b32 v202, a201\n\t" \
  "v_accvgpr_read_b32 v203, a205\n\t" \
  "v_pk_fma_f32 v[204:205], v[200:201], s[58:59], s[58:59] op_sel:[0,0,1] op_sel_hi:[1,0,1]\n\t" \
  "v_pk_fma_f32 v[206:207], v[202:203], s[58:59], s[58:59] op_sel:[0,0,1] op_sel_hi:[1,0,1]\n\t" \
  "v_exp_f32_e64 v204, v204 clamp\n\t" \
  "v_exp_f32_e64 v205, v205 clamp\n\t" \
  "v_exp_f32_e64 v206, v206 clamp\n\t" \
  "v_exp_f32_e64 v207, v207 clamp\n\t" \
  "v_pk_fma_f32 v[204:205], v[204:205], s[60:61], s[60:61] op_sel:[0,1,0] op_sel_hi:[1,1,0]\n\t" \
  "v_pk_fma_f32 v[206:207], v[206:207], s[60:61], s[60:61] op_sel:[0,1,0] op_sel_hi:[1,1,0]\n\t" \
  "v_rcp_f32 v204, v204\n\t" \
  "v_rcp_f32 v205, v205\n\t" \
  "v_rcp_f32 v206, v206\n\t" \
  "v_rcp_f32 v207, v207\n\t" \
  "v_pk_mul_f32 v[200:201], v[200:201], v[204:205]\n\t" \
  "v_pk_mul_f32 v[202:203], v[202:203], v[206:207]\n\t" \
  "s_add_u32 s57, s80, 4259840\n\t" \
  "buffer_store_dwordx4 v[200:203], v253, s[44:47], s57 offen offset:256 \n\t" \
  "v_accvgpr_read_b32 v208, a194\n\t" \
  "v_accvgpr_read_b32 v209, a198\n\t" \
  "v_accvgpr_read_b32 v210, a202\n\t" \
  "v_accvgpr_read_b32 v211, a206\n\t" \
  "v_pk_fma_f32 v[212:213], v[208:209], s[58:59], s[58:59] op_sel:[0,0,1] op_sel_hi:[1,0,1]\n\t" \
  "v_pk_fma_f32 v[214:215], v[210:211], s[58:59], s[58:59] op_sel:[0,0,1] op_sel_hi:[1,0,1]\n\t" \
  "v_exp_f32_e64 v212, v212 clamp\n\t" \
  "v_exp_f32_e64 v213, v213 clamp\n\t" \
  "v_exp_f32_e64 v214, v214 clamp\n\t" \
  "v_exp_f32_e64 v215, v215 clamp\n\t" \
  "v_pk_fma_f32 v[212:213], v[212:213], s[60:61], s[60:61] op_sel:[0,1,0] op_sel_hi:[1,1,0]\n\t" \
  "v_pk_fma_f32 v[214:215], v[214:215], s[60:61], s[60:61] op_sel:[0,1,0] op_sel_hi:[1,1,0]\n\t" \
  "v_rcp_f32 v212, v212\n\t" \
  "v_rcp_f32 v213, v213\n\t" \
  "v_rcp_f32 v214, v214\n\t" \
  "v_rcp_f32 v215, v215\n\t" \
  "v_pk_mul_f32 v[208:209], v[208:209], v[212:213]\n\t" \
  "v_pk_mul_f32 v[210:211], v[210:211], v[214:215]\n\t" \
  "s_add_u32 s57, s80, 4325376\n\t" \
  "buffer_store_dwordx4 v[208:211], v253, s[44:47], s57 offen offset:256 \n\t" \
  "v_accvgpr_read_b32 v216, a195\n\t" \
  "v_accvgpr_read_b32 v217, a199\n\t" \
  "v_accvgpr_read_b32 v218, a203\n\t" \
  "v_accvgpr_read_b32 v219, a207\n\t" \
  "v_pk_fma_f32 v[220:221], v[216:217], s[58:59], s[58:59] op_sel:[0,0,1] op_sel_hi:[1,0,1]\n\t" \
  "v_pk_fma_f32 v[222:223], v[218:219], s[58:59], s[58:59] op_sel:[0,0,1] op_sel_hi:[1,0,1]\n\t" \
  "v_exp_f32_e64 v220, v220 clamp\n\t" \
  "v_exp_f32_e64 v221, v221 clamp\n\t" \
  "v_exp_f32_e64 v222, v222 clamp\n\t" \
  "v_exp_f32_e64 v223, v223 clamp\n\t" \
  "v_pk_fma_f32 v[220:221], v[220:221], s[60:61], s[60:61] op_sel:[0,1,0] op_sel_hi:[1,1,0]\n\t" \
  "v_pk_fma_f32 v[222:223], v[222:223], s[60:61], s[60:61] op_sel:[0,1,0] op_sel_hi:[1,1,0]\n\t" \
  "v_rcp_f32 v220, v220\n\t" \
  "v_rcp_f32 v221, v221\n\t" \
  "v_rcp_f32 v222, v222\n\t" \
  "v_rcp_f32 v223, v223\n\t" \
  "v_pk_mul_f32 v[216:217], v[216:217], v[220:221]\n\t" \
  "v_pk_mul_f32 v[218:219], v[218:219], v[222:223]\n\t" \
  "s_add_u32 s57, s80, 4390912\n\t" \
  "buffer_store_dwordx4 v[216:219], v253, s[44:47], s57 offen offset:256 \n\t" \
  "v_accvgpr_read_b32 v192, a208\n\t" \
  "v_accvgpr_read_b32 v193, a212\n\t" \
  "v_accvgpr_read_b32 v194, a216\n\t" \
  "v_accvgpr_read_b32 v195, a220\n\t" \
  "v_pk_fma_f32 v[196:197], v[192:193], s[58:59], s[58:59] op_sel:[0,0,1] op_sel_hi:[1,0,1]\n\t" \
  "v_pk_fma_f32 v[198:199], v[194:195], s[58:59], s[58:59] op_sel:[0,0,1] op_sel_hi:[1,0,1]\n\t" \
  "v_exp_f32_e64 v196, v196 clamp\n\t" \
  "v_exp_f32_e64 v197, v197 clamp\n\t" \
  "v_exp_f32_e64 v198, v198 clamp\n\t" \
  "v_exp_f32_e64 v199, v199 clamp\n\t" \
  "v_pk_fma_f32 v[196:197], v[196:197], s[60:61], s[60:61] op_sel:[0,1,0] op_sel_hi:[1,1,0]\n\t" \
  "v_pk_fma_f32 v[198:199], v[198:199], s[60:61], s[60:61] op_sel:[0,1,0] op_sel_hi:[1,1,0]\n\t" \
  "v_rcp_f32 v196, v196\n\t" \
  "v_rcp_f32 v197, v197\n\t" \
  "v_rcp_f32 v198, v198\n\t" \
  "v_rcp_f32 v199, v199\n\t" \
  "v_pk_mul_f32 v[192:193], v[192:193], v[196:197]\n\t" \
  "v_pk_mul_f32 v[194:195], v[194:195], v[198:199]\n\t" \
  "s_add_u32 s57, s80, 5242880\n\t" \
  "buffer_store_dwordx4 v[192:195], v253, s[44:47], s57 offen offset:256 nt\n\t" \
  "v_accvgpr_read_b32 v200, a209\n\t" \
  "v_accvgpr_read_b32 v201, a213\n\t" \
  "v_accvgpr_read_b32 v202, a217\n\t" \
  "v_accvgpr_read_b32 v203, a221\n\t" \
  "v_pk_fma_f32 v[204:205], v[200:201], s[58:59], s[58:59] op_sel:[0,0,1] op_sel_hi:[1,0,1]\n\t" \
  "v_pk_fma_f32 v[206:207], v[202:203], s[58:59], s[58:59] op_sel:[0,0,1] op_sel_hi:[1,0,1]\n\t" \
  "v_exp_f32_e64 v204, v204 clamp\n\t" \
  "v_exp_f32_e64 v205, v205 clamp\n\t" \
  "v_exp_f32_e64 v206, v206 clamp\n\t" \
  "v_exp_f32_e64 v207, v207 clamp\n\t" \
  "v_pk_fma_f32 v[204:205], v[204:205], s[60:61], s[60:61] op_sel:[0,1,0] op_sel_hi:[1,1,0]\n\t" \
  "v_pk_fma_f32 v[206:207], v[206:207], s[60:61], s[60:61] op_sel:[0,1,0] op_sel_hi:[1,1,0]\n\t" \
  "v_rcp_f32 v204, v204\n\t" \
  "v_rcp_f32 v205, v205\n\t" \
  "v_rcp_f32 v206, v206\n\t" \
  "v_rcp_f32 v207, v207\n\t" \
  "v_pk_mul_f32 v[200:201], v[200:201], v[204:205]\n\t" \
  "v_pk_mul_f32 v[202:203], v[202:203], v[206:207]\n\t" \
  "s_add_u32 s57, s80, 5308416\n\t" \
  "buffer_store_dwordx4 v[200:203], v253, s[44:47], s57 offen offset:256 nt\n\t" \
  "v_accvgpr_read_b32 v208, a210\n\t" \
  "v_accvgpr_read_b32 v209, a214\n\t" \
  "v_accvgpr_read_b32 v210, a218\n\t" \
  "v_accvgpr_read_b32 v211, a222\n\t" \
  "v_pk_fma_f32 v[212:213], v[208:209], s[58:59], s[58:59] op_sel:[0,0,1] op_sel_hi:[1,0,1]\n\t" \
  "v_pk_fma_f32 v[214:215], v[210:211], s[58:59], s[58:59] op_sel:[0,0,1] op_sel_hi:[1,0,1]\n\t" \
  "v_exp_f32_e64 v212, v212 clamp\n\t" \
  "v_exp_f32_e64 v213, v213 clamp\n\t" \
  "v_exp_f32_e64 v214, v214 clamp\n\t" \
  "v_exp_f32_e64 v215, v215 clamp\n\t" \
  "v_pk_fma_f32 v[212:213], v[212:213], s[60:61], s[60:61] op_sel:[0,1,0] op_sel_hi:[1,1,0]\n\t" \
  "v_pk_fma_f32 v[214:215], v[214:215], s[60:61], s[60:61] op_sel:[0,1,0] op_sel_hi:[1,1,0]\n\t" \
  "v_rcp_f32 v212, v212\n\t" \
  "v_rcp_f32 v213, v213\n\t" \
  "v_rcp_f32 v214, v214\n\t" \
  "v_rcp_f32 v215, v215\n\t" \
  "v_pk_mul_f32 v[208:209], v[208:209], v[212:213]\n\t" \
  "v_pk_mul_f32 v[210:211], v[210:211], v[214:215]\n\t" \
  "s_add_u32 s57, s80, 5373952\n\t" \
  "buffer_store_dwordx4 v[208:211], v253, s[44:47], s57 offen offset:256 nt\n\t" \
  "v_accvgpr_read_b32 v216, a211\n\t" \
  "v_accvgpr_read_b32 v217, a215\n\t" \
  "v_accvgpr_read_b32 v218, a219\n\t" \
  "v_accvgpr_read_b32 v219, a223\n\t" \
  "v_pk_fma_f32 v[220:221], v[216:217], s[58:59], s[58:59] op_sel:[0,0,1] op_sel_hi:[1,0,1]\n\t" \
  "v_pk_fma_f32 v[222:223], v[218:219], s[58:59], s[58:59] op_sel:[0,0,1] op_sel_hi:[1,0,1]\n\t" \
  "v_exp_f32_e64 v220, v220 clamp\n\t" \
  "v_exp_f32_e64 v221, v221 clamp\n\t" \
  "v_exp_f32_e64 v222, v222 clamp\n\t" \
  "v_exp_f32_e64 v223, v223 clamp\n\t" \
  "v_pk_fma_f32 v[220:221], v[220:221], s[60:61], s[60:61] op_sel:[0,1,0] op_sel_hi:[1,1,0]\n\t" \
  "v_pk_fma_f32 v[222:223], v[222:223], s[60:61], s[60:61] op_sel:[0,1,0] op_sel_hi:[1,1,0]\n\t" \
  "v_rcp_f32 v220, v220\n\t" \
  "v_rcp_f32 v221, v221\n\t" \
  "v_rcp_f32 v222, v222\n\t" \
  "v_rcp_f32 v223, v223\n\t" \
  "v_pk_mul_f32 v[216:217], v[216:217], v[220:221]\n\t" \
  "v_pk_mul_f32 v[218:219], v[218:219], v[222:223]\n\t" \
  "s_add_u32 s57, s80, 5439488\n\t" \
  "buffer_store_dwordx4 v[216:219], v253, s[44:47], s57 offen offset:256 nt\n\t" \
  "v_accvgpr_read_b32 v192, a224\n\t" \
  "v_accvgpr_read_b32 v193, a228\n\t" \
  "v_accvgpr_read_b32 v194, a232\n\t" \
  "v_accvgpr_read_b32 v195, a236\n\t" \
  "v_pk_fma_f32 v[196:197], v[192:193], s[58:59], s[58:59] op_sel:[0,0,1] op_sel_hi:[1,0,1]\n\t" \
  "v_pk_fma_f32 v[198:199], v[194:195], s[58:59], s[58:59] op_sel:[0,0,1] op_sel_hi:[1,0,1]\n\t" \
  "v_exp_f32_e64 v196, v196 clamp\n\t" \
  "v_exp_f32_e64 v197, v197 clamp\n\t" \
  "v_exp_f32_e64 v198, v198 clamp\n\t" \
  "v_exp_f32_e64 v199, v199 clamp\n\t" \
  "v_pk_fma_f32 v[196:197], v[196:197], s[60:61], s[60:61] op_sel:[0,1,0] op_sel_hi:[1,1,0]\n\t" \
  "v_pk_fma_f32 v[198:199], v[198:199], s[60:61], s[60:61] op_sel:[0,1,0] op_sel_hi:[1,1,0]\n\t" \
  "v_rcp_f32 v196, v196\n\t" \
  "v_rcp_f32 v197, v197\n\t" \
  "v_rcp_f32 v198, v198\n\t" \
  "v_rcp_f32 v199, v199\n\t" \
  "v_pk_mul_f32 v[192:193], v[192:193], v[196:197]\n\t" \
  "v_pk_mul_f32 v[194:195], v[194:195], v[198:199]\n\t" \
  "s_add_u32 s57, s80, 6291456\n\t" \
  "buffer_store_dwordx4 v[192:195], v253, s[44:47], s57 offen offset:256 \n\t" \
  "v_accvgpr_read_b32 v200, a225\n\t" \
  "v_accvgpr_read_b32 v201, a229\n\t" \
  "v_accvgpr_read_b32 v202, a233\n\t" \
  "v_accvgpr_read_b32 v203, a237\n\t" \
  "v_pk_fma_f32 v[204:205], v[200:201], s[58:59], s[58:59] op_sel:[0,0,1] op_sel_hi:[1,0,1]\n\t" \
  "v_pk_fma_f32 v[206:207], v[202:203], s[58:59], s[58:59] op_sel:[0,0,1] op_sel_hi:[1,0,1]\n\t" \
  "v_exp_f32_e64 v204, v204 clamp\n\t" \
  "v_exp_f32_e64 v205, v205 clamp\n\t" \
  "v_exp_f32_e64 v206, v206 clamp\n\t" \
  "v_exp_f32_e64 v207, v207 clamp\n\t" \
  "v_pk_fma_f32 v[204:205], v[204:205], s[60:61], s[60:61] op_sel:[0,1,0] op_sel_hi:[1,1,0]\n\t" \
  "v_pk_fma_f32 v[206:207], v[206:207], s[60:61], s[60:61] op_sel:[0,1,0] op_sel_hi:[1,1,0]\n\t" \
  "v_rcp_f32 v204, v204\n\t" \
  "v_rcp_f32 v205, v205\n\t" \
  "v_rcp_f32 v206, v206\n\t" \
  "v_rcp_f32 v207, v207\n\t" \
  "v_pk_mul_f32 v[200:201], v[200:201], v[204:205]\n\t" \
  "v_pk_mul_f32 v[202:203], v[202:203], v[206:207]\n\t" \
  "s_add_u32 s57, s80, 6356992\n\t" \
  "buffer_store_dwordx4 v[200:203], v253, s[44:47], s57 offen offset:256 \n\t" \
  "v_accvgpr_read_b32 v208, a226\n\t" \
  "v_accvgpr_read_b32 v209, a230\n\t" \
  "v_accvgpr_read_b32 v210, a234\n\t" \
  "v_accvgpr_read_b32 v211, a238\n\t" \
  "v_pk_fma_f32 v[212:213], v[208:209], s[58:59], s[58:59] op_sel:[0,0,1] op_sel_hi:[1,0,1]\n\t" \
  "v_pk_fma_f32 v[214:215], v[210:211], s[58:59], s[58:59] op_sel:[0,0,1] op_sel_hi:[1,0,1]\n\t" \
  "v_exp_f32_e64 v212, v212 clamp\n\t" \
  "v_exp_f32_e64 v213, v213 clamp\n\t" \
  "v_exp_f32_e64 v214, v214 clamp\n\t" \
  "v_exp_f32_e64 v215, v215 clamp\n\t" \
  "v_pk_fma_f32 v[212:213], v[212:213], s[60:61], s[60:61] op_sel:[0,1,0] op_sel_hi:[1,1,0]\n\t" \
  "v_pk_fma_f32 v[214:215], v[214:215], s[60:61], s[60:61] op_sel:[0,1,0] op_sel_hi:[1,1,0]\n\t" \
  "v_rcp_f32 v212, v212\n\t" \
  "v_rcp_f32 v213, v213\n\t" \
  "v_rcp_f32 v214, v214\n\t" \
  "v_rcp_f32 v215, v215\n\t" \
  "v_pk_mul_f32 v[208:209], v[208:209], v[212:213]\n\t" \
  "v_pk_mul_f32 v[210:211], v[210:211], v[214:215]\n\t" \
  "s_add_u32 s57, s80, 6422528\n\t" \
  "buffer_store_dwordx4 v[208:211], v253, s[44:47], s57 offen offset:256 \n\t" \
  "v_accvgpr_read_b32 v216, a227\n\t" \
  "v_accvgpr_read_b32 v217, a231\n\t" \
  "v_accvgpr_read_b32 v218, a235\n\t" \
  "v_accvgpr_read_b32 v219, a239\n\t" \
  "v_pk_fma_f32 v[220:221], v[216:217], s[58:59], s[58:59] op_sel:[0,0,1] op_sel_hi:[1,0,1]\n\t" \
  "v_pk_fma_f32 v[222:223], v[218:219], s[58:59], s[58:59] op_sel:[0,0,1] op_sel_hi:[1,0,1]\n\t" \
  "v_exp_f32_e64 v220, v220 clamp\n\t" \
  "v_exp_f32_e64 v221, v221 clamp\n\t" \
  "v_exp_f32_e64 v222, v222 clamp\n\t" \
  "v_exp_f32_e64 v223, v223 clamp\n\t" \
  "v_pk_fma_f32 v[220:221], v[220:221], s[60:61], s[60:61] op_sel:[0,1,0] op_sel_hi:[1,1,0]\n\t" \
  "v_pk_fma_f32 v[222:223], v[222:223], s[60:61], s[60:61] op_sel:[0,1,0] op_sel_hi:[1,1,0]\n\t" \
  "v_rcp_f32 v220, v220\n\t" \
  "v_rcp_f32 v221, v221\n\t" \
  "v_rcp_f32 v222, v222\n\t" \
  "v_rcp_f32 v223, v223\n\t" \
  "v_pk_mul_f32 v[216:217], v[216:217], v[220:221]\n\t" \
  "v_pk_mul_f32 v[218:219], v[218:219], v[222:223]\n\t" \
  "s_add_u32 s57, s80, 6488064\n\t" \
  "buffer_store_dwordx4 v[216:219], v253, s[44:47], s57 offen offset:256 \n\t" \
  "v_accvgpr_read_b32 v192, a240\n\t" \
  "v_accvgpr_read_b32 v193, a244\n\t" \
  "v_accvgpr_read_b32 v194, a248\n\t" \
  "v_accvgpr_read_b32 v195, a252\n\t" \
  "v_pk_fma_f32 v[196:197], v[192:193], s[58:59], s[58:59] op_sel:[0,0,1] op_sel_hi:[1,0,1]\n\t" \
  "v_pk_fma_f32 v[198:199], v[194:195], s[58:59], s[58:59] op_sel:[0,0,1] op_sel_hi:[1,0,1]\n\t" \
  "v_exp_f32_e64 v196, v196 clamp\n\t" \
  "v_exp_f32_e64 v197, v197 clamp\n\t" \
  "v_exp_f32_e64 v198, v198 clamp\n\t" \
  "v_exp_f32_e64 v199, v199 clamp\n\t" \
  "v_pk_fma_f32 v[196:197], v[196:197], s[60:61], s[60:61] op_sel:[0,1,0] op_sel_hi:[1,1,0]\n\t" \
  "v_pk_fma_f32 v[198:199], v[198:199], s[60:61], s[60:61] op_sel:[0,1,0] op_sel_hi:[1,1,0]\n\t" \
  "v_rcp_f32 v196, v196\n\t" \
  "v_rcp_f32 v197, v197\n\t" \
  "v_rcp_f32 v198, v198\n\t" \
  "v_rcp_f32 v199, v199\n\t" \
  "v_pk_mul_f32 v[192:193], v[192:193], v[196:197]\n\t" \
  "v_pk_mul_f32 v[194:195], v[194:195], v[198:199]\n\t" \
  "s_add_u32 s57, s80, 7340032\n\t" \
  "buffer_store_dwordx4 v[192:195], v253, s[44:47], s57 offen offset:256 nt\n\t" \
  "v_accvgpr_read_b32 v200, a241\n\t" \
  "v_accvgpr_read_b32 v201, a245\n\t" \
  "v_accvgpr_read_b32 v202, a249\n\t" \
  "v_accvgpr_read_b32 v203, a253\n\t" \
  "v_pk_fma_f32 v[204:205], v[200:201], s[58:59], s[58:59] op_sel:[0,0,1] op_sel_hi:[1,0,1]\n\t" \
  "v_pk_fma_f32 v[206:207], v[202:203], s[58:59], s[58:59] op_sel:[0,0,1] op_sel_hi:[1,0,1]\n\t" \
  "v_exp_f32_e64 v204, v204 clamp\n\t" \
  "v_exp_f32_e64 v205, v205 clamp\n\t" \
  "v_exp_f32_e64 v206, v206 clamp\n\t" \
  "v_exp_f32_e64 v207, v207 clamp\n\t" \
  "v_pk_fma_f32 v[204:205], v[204:205], s[60:61], s[60:61] op_sel:[0,1,0] op_sel_hi:[1,1,0]\n\t" \
  "v_pk_fma_f32 v[206:207], v[206:207], s[60:61], s[60:61] op_sel:[0,1,0] op_sel_hi:[1,1,0]\n\t" \
  "v_rcp_f32 v204, v204\n\t" \
  "v_rcp_f32 v205, v205\n\t" \
  "v_rcp_f32 v206, v206\n\t" \
  "v_rcp_f32 v207, v207\n\t" \
  "v_pk_mul_f32 v[200:201], v[200:201], v[204:205]\n\t" \
  "v_pk_mul_f32 v[202:203], v[202:203], v[206:207]\n\t" \
  "s_add_u32 s57, s80, 7405568\n\t" \
  "buffer_store_dwordx4 v[200:203], v253, s[44:47], s57 offen offset:256 nt\n\t" \
  "v_accvgpr_read_b32 v208, a242\n\t" \
  "v_accvgpr_read_b32 v209, a246\n\t" \
  "v_accvgpr_read_b32 v210, a250\n\t" \
  "v_accvgpr_read_b32 v211, a254\n\t" \
  "v_pk_fma_f32 v[212:213], v[208:209], s[58:59], s[58:59] op_sel:[0,0,1] op_sel_hi:[1,0,1]\n\t" \
  "v_pk_fma_f32 v[214:215], v[210:211], s[58:59], s[58:59] op_sel:[0,0,1] op_sel_hi:[1,0,1]\n\t" \
  "v_exp_f32_e64 v212, v212 clamp\n\t" \
  "v_exp_f32_e64 v213, v213 clamp\n\t" \
  "v_exp_f32_e64 v214, v214 clamp\n\t" \
  "v_exp_f32_e64 v215, v215 clamp\n\t" \
  "v_pk_fma_f32 v[212:213], v[212:213], s[60:61], s[60:61] op_sel:[0,1,0] op_sel_hi:[1,1,0]\n\t" \
  "v_pk_fma_f32 v[214:215], v[214:215], s[60:61], s[60:61] op_sel:[0,1,0] op_sel_hi:[1,1,0]\n\t" \
  "v_rcp_f32 v212, v212\n\t" \
  "v_rcp_f32 v213, v213\n\t" \
  "v_rcp_f32 v214, v214\n\t" \
  "v_rcp_f32 v215, v215\n\t" \
  "v_pk_mul_f32 v[208:209], v[208:209], v[212:213]\n\t" \
  "v_pk_mul_f32 v[210:211], v[210:211], v[214:215]\n\t" \
  "s_add_u32 s57, s80, 7471104\n\t" \
  "buffer_store_dwordx4 v[208:211], v253, s[44:47], s57 offen offset:256 nt\n\t" \
  "v_accvgpr_read_b32 v216, a243\n\t" \
  "v_accvgpr_read_b32 v217, a247\n\t" \
  "v_accvgpr_read_b32 v218, a251\n\t" \
  "v_accvgpr_read_b32 v219, a255\n\t" \
  "v_pk_fma_f32 v[220:221], v[216:217], s[58:59], s[58:59] op_sel:[0,0,1] op_sel_hi:[1,0,1]\n\t" \
  "v_pk_fma_f32 v[222:223], v[218:219], s[58:59], s[58:59] op_sel:[0,0,1] op_sel_hi:[1,0,1]\n\t" \
  "v_exp_f32_e64 v220, v220 clamp\n\t" \
  "v_exp_f32_e64 v221, v221 clamp\n\t" \
  "v_exp_f32_e64 v222, v222 clamp\n\t" \
  "v_exp_f32_e64 v223, v223 clamp\n\t" \
  "v_pk_fma_f32 v[220:221], v[220:221], s[60:61], s[60:61] op_sel:[0,1,0] op_sel_hi:[1,1,0]\n\t" \
  "v_pk_fma_f32 v[222:223], v[222:223], s[60:61], s[60:61] op_sel:[0,1,0] op_sel_hi:[1,1,0]\n\t" \
  "v_rcp_f32 v220, v220\n\t" \
  "v_rcp_f32 v221, v221\n\t" \
  "v_rcp_f32 v222, v222\n\t" \
  "v_rcp_f32 v223, v223\n\t" \
  "v_pk_mul_f32 v[216:217], v[216:217], v[220:221]\n\t" \
  "v_pk_mul_f32 v[218:219], v[218:219], v[222:223]\n\t" \
  "s_add_u32 s57, s80, 7536640\n\t" \
  "buffer_store_dwordx4 v[216:219], v253, s[44:47], s57 offen offset:256 nt\n\t" \
  "s_waitcnt vmcnt(0) lgkmcnt(0)\n\t" \
  "s_barrier\n\t" \
  "s_cmp_lg_u32 s63, 0\n\t" \
  "s_cbranch_scc1 L_bye_%=\n\t" \
  "v_mov_b32 v254, 1\n\t" \
  "s_mov_b64 exec, 1\n\t" \
  "buffer_atomic_add v254, off, s[88:91], 0\n\t" \
  "L_bye_%=:\n\t" \
  "s_endpgm\n\t" \
  "L_shared_%=:\n\t" \
  "s_nop 15\n\t" \
  "v_accvgpr_write_b32 a0, v0\n\t" \
  "v_accvgpr_write_b32 a1, v1\n\t" \
  "v_accvgpr_write_b32 a2, v2\n\t" \
  "v_accvgpr_write_b32 a3, v3\n\t" \
  "v_accvgpr_write_b32 a4, v4\n\t" \
  "v_accvgpr_write_b32 a5, v5\n\t" \
  "v_accvgpr_write_b32 a6, v6\n\t" \
  "v_accvgpr_write_b32 a7, v7\n\t" \
  "v_accvgpr_write_b32 a8, v8\n\t" \
  "v_accvgpr_write_b32 a9, v9\n\t" \
  "v_accvgpr_write_b32 a10, v10\n\t" \
  "v_accvgpr_write_b32 a11, v11\n\t" \
  "v_accvgpr_write_b32 a12, v12\n\t" \
  "v_accvgpr_write_b32 a13, v13\n\t" \
  "v_accvgpr_write_b32 a14, v14\n\t" \
  "v_accvgpr_write_b32 a15, v15\n\t" \
  "v_accvgpr_write_b32 a16, v16\n\t" \
  "v_accvgpr_write_b32 a17, v17\n\t" \
  "v_accvgpr_write_b32 a18, v18\n\t" \
  "v_accvgpr_write_b32 a19, v19\n\t" \
  "v_accvgpr_write_b32 a20, v20\n\t" \
  "v_accvgpr_write_b32 a21, v21\n\t" \
  "v_accvgpr_write_b32 a22, v22\n\t" \
  "v_accvgpr_write_b32 a23, v23\n\t" \
  "v_accvgpr_write_b32 a24, v24\n\t" \
  "v_accvgpr_write_b32 a25, v25\n\t" \
  "v_accvgpr_write_b32 a26, v26\n\t" \
  "v_accvgpr_write_b32 a27, v27\n\t" \
  "v_accvgpr_write_b32 a28, v28\n\t" \
  "v_accvgpr_write_b32 a29, v29\n\t" \
  "v_accvgpr_write_b32 a30, v30\n\t" \
  "v_accvgpr_write_b32 a31, v31\n\t" \
  "v_accvgpr_write_b32 a32, v32\n\t" \
  "v_accvgpr_write_b32 a33, v33\n\t" \
  "v_accvgpr_write_b32 a34, v34\n\t" \
  "v_accvgpr_write_b32 a35, v35\n\t" \
  "v_accvgpr_write_b32 a36, v36\n\t" \
  "v_accvgpr_write_b32 a37, v37\n\t" \
  "v_accvgpr_write_b32 a38, v38\n\t" \
  "v_accvgpr_write_b32 a39, v39\n\t" \
  "v_accvgpr_write_b32 a40, v40\n\t" \
  "v_accvgpr_write_b32 a41, v41\n\t" \
  "v_accvgpr_write_b32 a42, v42\n\t" \
  "v_accvgpr_write_b32 a43, v43\n\t" \
  "v_accvgpr_write_b32 a44, v44\n\t" \
  "v_accvgpr_write_b32 a45, v45\n\t" \
  "v_accvgpr_write_b32 a46, v46\n\t" \
  "v_accvgpr_write_b32 a47, v47\n\t" \
  "v_accvgpr_write_b32 a48, v48\n\t" \
  "v_accvgpr_write_b32 a49, v49\n\t" \
  "v_accvgpr_write_b32 a50, v50\n\t" \
  "v_accvgpr_write_b32 a51, v51\n\t" \
  "v_accvgpr_write_b32 a52, v52\n\t" \
  "v_accvgpr_write_b32 a53, v53\n\t" \
  "v_accvgpr_write_b32 a54, v54\n\t" \
  "v_accvgpr_write_b32 a55, v55\n\t" \
  "v_accvgpr_write_b32 a56, v56\n\t" \
  "v_accvgpr_write_b32 a57, v57\n\t" \
  "v_accvgpr_write_b32 a58, v58\n\t" \
  "v_accvgpr_write_b32 a59, v59\n\t" \
  "v_accvgpr_write_b32 a60, v60\n\t" \
  "v_accvgpr_write_b32 a61, v61\n\t" \
  "v_accvgpr_write_b32 a62, v62\n\t" \
  "v_accvgpr_write_b32 a63, v63\n\t" \
  "v_accvgpr_write_b32 a64, v64\n\t" \
  "v_accvgpr_write_b32 a65, v65\n\t" \
  "v_accvgpr_write_b32 a66, v66\n\t" \
  "v_accvgpr_write_b32 a67, v67\n\t" \
  "v_accvgpr_write_b32 a68, v68\n\t" \
  "v_accvgpr_write_b32 a69, v69\n\t" \
  "v_accvgpr_write_b32 a70, v70\n\t" \
  "v_accvgpr_write_b32 a71, v71\n\t" \
  "v_accvgpr_write_b32 a72, v72\n\t" \
  "v_accvgpr_write_b32 a73, v73\n\t" \
  "v_accvgpr_write_b32 a74, v74\n\t" \
  "v_accvgpr_write_b32 a75, v75\n\t" \
  "v_accvgpr_write_b32 a76, v76\n\t" \
  "v_accvgpr_write_b32 a77, v77\n\t" \
  "v_accvgpr_write_b32 a78, v78\n\t" \
  "v_accvgpr_write_b32 a79, v79\n\t" \
  "v_accvgpr_write_b32 a80, v80\n\t" \
  "v_accvgpr_write_b32 a81, v81\n\t" \
  "v_accvgpr_write_b32 a82, v82\n\t" \
  "v_accvgpr_write_b32 a83, v83\n\t" \
  "v_accvgpr_write_b32 a84, v84\n\t" \
  "v_accvgpr_write_b32 a85, v85\n\t" \
  "v_accvgpr_write_b32 a86, v86\n\t" \
  "v_accvgpr_write_b32 a87, v87\n\t" \
  "v_accvgpr_write_b32 a88, v88\n\t" \
  "v_accvgpr_write_b32 a89, v89\n\t" \
  "v_accvgpr_write_b32 a90, v90\n\t" \
  "v_accvgpr_write_b32 a91, v91\n\t" \
  "v_accvgpr_write_b32 a92, v92\n\t" \
  "v_accvgpr_write_b32 a93, v93\n\t" \
  "v_accvgpr_write_b32 a94, v94\n\t" \
  "v_accvgpr_write_b32 a95, v95\n\t" \
  "v_accvgpr_write_b32 a96, v96\n\t" \
  "v_accvgpr_write_b32 a97, v97\n\t" \
  "v_accvgpr_write_b32 a98, v98\n\t" \
  "v_accvgpr_write_b32 a99, v99\n\t" \
  "v_accvgpr_write_b32 a100, v100\n\t" \
  "v_accvgpr_write_b32 a101, v101\n\t" \
  "v_accvgpr_write_b32 a102, v102\n\t" \
  "v_accvgpr_write_b32 a103, v103\n\t" \
  "v_accvgpr_write_b32 a104, v104\n\t" \
  "v_accvgpr_write_b32 a105, v105\n\t" \
  "v_accvgpr_write_b32 a106, v106\n\t" \
  "v_accvgpr_write_b32 a107, v107\n\t" \
  "v_accvgpr_write_b32 a108, v108\n\t" \
  "v_accvgpr_write_b32 a109, v109\n\t" \
  "v_accvgpr_write_b32 a110, v110\n\t" \
  "v_accvgpr_write_b32 a111, v111\n\t" \
  "v_accvgpr_write_b32 a112, v224\n\t" \
  "v_accvgpr_write_b32 a113, v225\n\t" \
  "v_accvgpr_write_b32 a114, v226\n\t" \
  "v_accvgpr_write_b32 a115, v227\n\t" \
  "v_accvgpr_write_b32 a116, v228\n\t" \
  "v_accvgpr_write_b32 a117, v229\n\t" \
  "v_accvgpr_write_b32 a118, v230\n\t" \
  "v_accvgpr_write_b32 a119, v231\n\t" \
  "v_accvgpr_write_b32 a120, v232\n\t" \
  "v_accvgpr_write_b32 a121, v233\n\t" \
  "v_accvgpr_write_b32 a122, v234\n\t" \
  "v_accvgpr_write_b32 a123, v235\n\t" \
  "v_accvgpr_write_b32 a124, v236\n\t" \
  "v_accvgpr_write_b32 a125, v237\n\t" \
  "v_accvgpr_write_b32 a126, v238\n\t" \
  "v_accvgpr_write_b32 a127, v239\n\t" \
  "v_mov_b32 v72, 0x4020100\n\t" \
  "v_mov_b32 v73, 0x5020100\n\t" \
  "v_mov_b32 v74, 0x6020100\n\t" \
  "v_mov_b32 v75, 0xc0c0703\n\t" \
  "v_mov_b32 v76, 0x5040100\n\t" \
  "v_mov_b32 v77, 0x7c7c7c7c\n\t" \
  "v_mov_b32 v78, 0x400\n\t" \
  "v_mov_b32 v79, 0x40\n\t" \
  "v_and_b32 v80, 0x3ff, v252\n\t" \
  "v_lshrrev_b32 v81, 1, v80\n\t" \
  "v_lshrrev_b32 v82, 3, v80\n\t" \
  "v_add_u32 v82, s53, v82\n\t" \
  "s_mov_b32 s46, 0x10000000\n\t" \
  "s_mov_b32 s42, 0x6800000\n\t" \
  "s_getpc_b64 s[82:83]\n\t" \
  "L_jp_%=:\n\t" \
  "s_add_u32 s82, s82, L_rd_%=-L_jp_%=\n\t" \
  "s_addc_u32 s83, s83, 0\n\t" \
  "s_mov_b32 s54, 0\n\t" \
  "s_waitcnt lgkmcnt(0)\n\t" \
  "L_xl_%=:\n\t" \
  "s_swappc_b64 s[76:77], s[82:83]\n\t" \
  "ds_write_b16 v82, v78\n\t" \
  "v_pk_fma_f32 v[16:17], v[0:1], s[58:59], s[58:59] op_sel:[0,0,1] op_sel_hi:[1,0,1]\n\t" \
  "v_pk_fma_f32 v[18:19], v[2:3], s[58:59], s[58:59] op_sel:[0,0,1] op_sel_hi:[1,0,1]\n\t" \
  "v_pk_fma_f32 v[20:21], v[4:5], s[58:59], s[58:59] op_sel:[0,0,1] op_sel_hi:[1,0,1]\n\t" \
  "v_pk_fma_f32 v[22:23], v[6:7], s[58:59], s[58:59] op_sel:[0,0,1] op_sel_hi:[1,0,1]\n\t" \
  "v_pk_fma_f32 v[24:25], v[8:9], s[58:59], s[58:59] op_sel:[0,0,1] op_sel_hi:[1,0,1]\n\t" \
  "v_pk_fma_f32 v[26:27], v[10:11], s[58:59], s[58:59] op_sel:[0,0,1] op_sel_hi:[1,0,1]\n\t" \
  "v_pk_fma_f32 v[28:29], v[12:13], s[58:59], s[58:59] op_sel:[0,0,1] op_sel_hi:[1,0,1]\n\t" \
  "v_pk_fma_f32 v[30:31], v[14:15], s[58:59], s[58:59] op_sel:[0,0,1] op_sel_hi:[1,0,1]\n\t" \
  "s_add_u32 s82, s82, 132\n\t" \
  "s_addc_u32 s83, s83, 0\n\t" \
  "s_and_b32 s52, s54, 7\n\t" \
  "s_lshr_b32 s99, s54, 1\n\t" \
  "s_xnor_b32 s99, s99, s54\n\t" \
  "s_bfe_u32 s99, s99, 0x10002\n\t" \
  "v_exp_f32_e64 v16, v16 clamp\n\t" \
  "v_exp_f32_e64 v17, v17 clamp\n\t" \
  "v_exp_f32_e64 v18, v18 clamp\n\t" \
  "v_exp_f32_e64 v19, v19 clamp\n\t" \
  "v_exp_f32_e64 v20, v20 clamp\n\t" \
  "v_exp_f32_e64 v21, v21 clamp\n\t" \
  "v_exp_f32_e64 v22, v22 clamp\n\t" \
  "v_exp_f32_e64 v23, v23 clamp\n\t" \
  "v_exp_f32_e64 v24, v24 clamp\n\t" \
  "v_exp_f32_e64 v25, v25 clamp\n\t" \
  "v_exp_f32_e64 v26, v26 clamp\n\t" \
  "v_exp_f32_e64 v27, v27 clamp\n\t" \
  "v_exp_f32_e64 v28, v28 clamp\n\t" \
  "v_exp_f32_e64 v29, v29 clamp\n\t" \
  "v_exp_f32_e64 v30, v30 clamp\n\t" \
  "v_exp_f32_e64 v31, v31 clamp\n\t" \
  "s_lshl_b32 s57, s52, 20\n\t" \
  "s_add_u32 s57, s57, s80\n\t" \
  "s_lshl_b32 s87, s99, 8\n\t" \
  "s_add_u32 s57, s57, s87\n\t" \
  "v_pk_fma_f32 v[16:17], v[16:17], s[60:61], s[60:61] op_sel:[0,1,0] op_sel_hi:[1,1,0]\n\t" \
  "v_pk_fma_f32 v[18:19], v[18:19], s[60:61], s[60:61] op_sel:[0,1,0] op_sel_hi:[1,1,0]\n\t" \
  "v_pk_fma_f32 v[20:21], v[20:21], s[60:61], s[60:61] op_sel:[0,1,0] op_sel_hi:[1,1,0]\n\t" \
  "v_pk_fma_f32 v[22:23], v[22:23], s[60:61], s[60:61] op_sel:[0,1,0] op_sel_hi:[1,1,0]\n\t" \
  "v_pk_fma_f32 v[24:25], v[24:25], s[60:61], s[60:61] op_sel:[0,1,0] op_sel_hi:[1,1,0]\n\t" \
  "v_pk_fma_f32 v[26:27], v[26:27], s[60:61], s[60:61] op_sel:[0,1,0] op_sel_hi:[1,1,0]\n\t" \
  "v_pk_fma_f32 v[28:29], v[28:29], s[60:61], s[60:61] op_sel:[0,1,0] op_sel_hi:[1,1,0]\n\t" \
  "v_pk_fma_f32 v[30:31], v[30:31], s[60:61], s[60:61] op_sel:[0,1,0] op_sel_hi:[1,1,0]\n\t" \
  "s_lshl2_add_u32 s96, s52, s99\n\t" \
  "s_bfe_u32 s87, s98, 0x5000b\n\t" \
  "s_lshl_b32 s87, s87, 6\n\t" \
  "s_add_u32 s96, s96, s87\n\t" \
  "s_and_b32 s87, s63, 2\n\t" \
  "s_lshl4_add_u32 s96, s87, s96\n\t" \
  "s_and_b32 s87, s63, 1\n\t" \
  "s_lshl1_add_u32 s96, s87, s96\n\t" \
  "v_rcp_f32 v16, v16\n\t" \
  "v_rcp_f32 v17, v17\n\t" \
  "v_rcp_f32 v18, v18\n\t" \
  "v_rcp_f32 v19, v19\n\t" \
  "v_rcp_f32 v20, v20\n\t" \
  "v_rcp_f32 v21, v21\n\t" \
  "v_rcp_f32 v22, v22\n\t" \
  "v_rcp_f32 v23, v23\n\t" \
  "v_rcp_f32 v24, v24\n\t" \
  "v_rcp_f32 v25, v25\n\t" \
  "v_rcp_f32 v26, v26\n\t" \
  "v_rcp_f32 v27, v27\n\t" \
  "v_rcp_f32 v28, v28\n\t" \
  "v_rcp_f32 v29, v29\n\t" \
  "v_rcp_f32 v30, v30\n\t" \
  "v_rcp_f32 v31, v31\n\t" \
  "s_mul_i32 s96, s96, 7680\n\t" \
  "s_add_u32 s96, s96, 0x5c00000\n\t" \
  "v_pk_mul_f32 v[0:1], v[0:1], v[16:17]\n\t" \
  "v_pk_mul_f32 v[2:3], v[2:3], v[18:19]\n\t" \
  "v_pk_mul_f32 v[4:5], v[4:5], v[20:21]\n\t" \
  "v_pk_mul_f32 v[6:7], v[6:7], v[22:23]\n\t" \
  "v_pk_mul_f32 v[8:9], v[8:9], v[24:25]\n\t" \
  "v_pk_mul_f32 v[10:11], v[10:11], v[26:27]\n\t" \
  "v_pk_mul_f32 v[12:13], v[12:13], v[28:29]\n\t" \
  "v_pk_mul_f32 v[14:15], v[14:15], v[30:31]\n\t" \
  "buffer_store_dwordx4 v[0:3], v253, s[44:47], s57 offen nt\n\t" \
  "s_add_u32 s57, s57, 0x10000\n\t" \
  "v_perm_b32 v56, v1, v0, v75\n\t" \
  "v_perm_b32 v57, v3, v2, v75\n\t" \
  "buffer_store_dwordx4 v[4:7], v253, s[44:47], s57 offen nt\n\t" \
  "s_add_u32 s57, s57, 0x10000\n\t" \
  "v_perm_b32 v58, v5, v4, v75\n\t" \
  "v_perm_b32 v59, v7, v6, v75\n\t" \
  "buffer_store_dwordx4 v[8:11], v253, s[44:47], s57 offen nt\n\t" \
  "s_add_u32 s57, s57, 0x10000\n\t" \
  "v_perm_b32 v60, v9, v8, v75\n\t" \
  "v_perm_b32 v61, v11, v10, v75\n\t" \
  "buffer_store_dwordx4 v[12:15], v253, s[44:47], s57 offen nt\n\t" \
  "v_perm_b32 v62, v13, v12, v75\n\t" \
  "v_perm_b32 v63, v15, v14, v75\n\t" \
  "v_perm_b32 v32, v57, v56, v76\n\t" \
  "v_perm_b32 v33, v59, v58, v76\n\t" \
  "v_perm_b32 v34, v61, v60, v76\n\t" \
  "v_perm_b32 v35, v63, v62, v76\n\t" \
  "v_perm_b32 v40, v3, v0, v72\n\t" \
  "v_perm_b32 v41, v3, v1, v73\n\t" \
  "v_perm_b32 v42, v3, v2, v74\n\t" \
  "v_perm_b32 v43, v7, v4, v72\n\t" \
  "v_perm_b32 v44, v7, v5, v73\n\t" \
  "v_perm_b32 v45, v7, v6, v74\n\t" \
  "v_perm_b32 v46, v11, v8, v72\n\t" \
  "v_perm_b32 v47, v11, v9, v73\n\t" \
  "v_perm_b32 v48, v11, v10, v74\n\t" \
  "v_perm_b32 v49, v15, v12, v72\n\t" \
  "v_perm_b32 v50, v15, v13, v73\n\t" \
  "v_perm_b32 v51, v15, v14, v74\n\t" \
  "buffer_store_dwordx4 v[40:43], v80, s[40:43], s96 offen offset:0 sc0 sc1\n\t" \
  "buffer_store_dwordx4 v[44:47], v80, s[40:43], s96 offen offset:1024 sc0 sc1\n\t" \
  "buffer_store_dwordx4 v[48:51], v80, s[40:43], s96 offen offset:2048 sc0 sc1\n\t" \
  "s_cmp_lg_u32 s52, 0\n\t" \
  "s_cbranch_scc1 L_xw_%=\n\t" \
  "v_lshlrev_b32 v16, 1, v0\n\t" \
  "v_lshlrev_b32 v17, 1, v1\n\t" \
  "v_lshlrev_b32 v18, 1, v2\n\t" \
  "v_lshlrev_b32 v19, 1, v3\n\t" \
  "v_lshlrev_b32 v20, 1, v4\n\t" \
  "v_lshlrev_b32 v21, 1, v5\n\t" \
  "v_lshlrev_b32 v22, 1, v6\n\t" \
  "v_lshlrev_b32 v23, 1, v7\n\t" \
  "v_lshlrev_b32 v24, 1, v8\n\t" \
  "v_lshlrev_b32 v25, 1, v9\n\t" \
  "v_lshlrev_b32 v26, 1, v10\n\t" \
  "v_lshlrev_b32 v27, 1, v11\n\t" \
  "v_lshlrev_b32 v28, 1, v12\n\t" \
  "v_lshlrev_b32 v29, 1, v13\n\t" \
  "v_lshlrev_b32 v30, 1, v14\n\t" \
  "v_lshlrev_b32 v31, 1, v15\n\t" \
  "v_max3_u32 v16, v16, v17, v18\n\t" \
  "v_max3_u32 v16, v16, v19, v20\n\t" \
  "v_max3_u32 v16, v16, v21, v22\n\t" \
  "v_max3_u32 v16, v16, v23, v24\n\t" \
  "v_max3_u32 v16, v16, v25, v26\n\t" \
  "v_max3_u32 v16, v16, v27, v28\n\t" \
  "v_max3_u32 v16, v16, v29, v30\n\t" \
  "v_max_u32 v16, v16, v31\n\t" \
  "v_lshrrev_b32 v16, 25, v16\n\t" \
  "s_mov_b32 s50, 0\n\t" \
  "s_or_b32 s87, s50, 64\n\t" \
  "v_cmp_le_u32 vcc, s87, v16\n\t" \
  "s_nop 3\n\t" \
  "s_bcnt1_i32_b64 s99, vcc\n\t" \
  "s_cmp_ge_u32 s99, 18\n\t" \
  "s_cselect_b32 s50, s87, s50\n\t" \
  "s_or_b32 s87, s50, 32\n\t" \
  "v_cmp_le_u32 vcc, s87, v16\n\t" \
  "s_nop 3\n\t" \
  "s_bcnt1_i32_b64 s99, vcc\n\t" \
  "s_cmp_ge_u32 s99, 18\n\t" \
  "s_cselect_b32 s50, s87, s50\n\t" \
  "s_or_b32 s87, s50, 16\n\t" \
  "v_cmp_le_u32 vcc, s87, v16\n\t" \
  "s_nop 3\n\t" \
  "s_bcnt1_i32_b64 s99, vcc\n\t" \
  "s_cmp_ge_u32 s99, 18\n\t" \
  "s_cselect_b32 s50, s87, s50\n\t" \
  "s_or_b32 s87, s50, 8\n\t" \
  "v_cmp_le_u32 vcc, s87, v16\n\t" \
  "s_nop 3\n\t" \
  "s_bcnt1_i32_b64 s99, vcc\n\t" \
  "s_cmp_ge_u32 s99, 18\n\t" \
  "s_cselect_b32 s50, s87, s50\n\t" \
  "s_or_b32 s87, s50, 4\n\t" \
  "v_cmp_le_u32 vcc, s87, v16\n\t" \
  "s_nop 3\n\t" \
  "s_bcnt1_i32_b64 s99, vcc\n\t" \
  "s_cmp_ge_u32 s99, 18\n\t" \
  "s_cselect_b32 s50, s87, s50\n\t" \
  "s_or_b32 s87, s50, 2\n\t" \
  "v_cmp_le_u32 vcc, s87, v16\n\t" \
  "s_nop 3\n\t" \
  "s_bcnt1_i32_b64 s99, vcc\n\t" \
  "s_cmp_ge_u32 s99, 18\n\t" \
  "s_cselect_b32 s50, s87, s50\n\t" \
  "s_or_b32 s87, s50, 1\n\t" \
  "v_cmp_le_u32 vcc, s87, v16\n\t" \
  "s_nop 3\n\t" \
  "s_bcnt1_i32_b64 s99, vcc\n\t" \
  "s_cmp_ge_u32 s99, 18\n\t" \
  "s_cselect_b32 s50, s87, s50\n\t" \
  "s_max_u32 s50, s50, 3\n\t" \
  "s_sub_u32 s50, s50, 3\n\t" \
  "s_sub_u32 s51, 128, s50\n\t" \
  "s_mul_i32 s51, s51, 0x01010101\n\t" \
  "L_xw_%=:\n\t" \
  "v_and_b32 v36, 0x7f7f7f7f, v32\n\t" \
  "v_and_b32 v37, 0x7f7f7f7f, v33\n\t" \
  "v_and_b32 v38, 0x7f7f7f7f, v34\n\t" \
  "v_and_b32 v39, 0x7f7f7f7f, v35\n\t" \
  "v_add_u32 v36, s51, v36\n\t" \
  "v_add_u32 v37, s51, v37\n\t" \
  "v_add_u32 v38, s51, v38\n\t" \
  "v_add_u32 v39, s51, v39\n\t" \
  "v_and_b32 v56, v77, v36\n\t" \
  "v_and_b32 v57, v77, v37\n\t" \
  "v_and_b32 v58, v77, v38\n\t" \
  "v_and_b32 v59, v77, v39\n\t" \
  "v_add_u32 v56, v77, v56\n\t" \
  "v_add_u32 v57, v77, v57\n\t" \
  "v_add_u32 v58, v77, v58\n\t" \
  "v_add_u32 v59, v77, v59\n\t" \
  "v_lshrrev_b32 v56, 7, v56\n\t" \
  "v_lshrrev_b32 v57, 6, v57\n\t" \
  "v_lshrrev_b32 v58, 5, v58\n\t" \
  "v_lshrrev_b32 v59, 4, v59\n\t" \
  "v_and_b32 v56, 0x1010101, v56\n\t" \
  "v_and_b32 v57, 0x2020202, v57\n\t" \
  "v_and_b32 v58, 0x4040404, v58\n\t" \
  "v_and_b32 v59, 0x8080808, v59\n\t" \
  "v_or3_b32 v54, v56, v57, v58\n\t" \
  "v_or_b32 v54, v54, v59\n\t" \
  "v_cmp_ne_u32 vcc, 0, v54\n\t" \
  "v_lshrrev_b32 v60, 7, v32\n\t" \
  "v_lshrrev_b32 v61, 6, v33\n\t" \
  "v_lshrrev_b32 v62, 5, v34\n\t" \
  "v_lshrrev_b32 v63, 4, v35\n\t" \
  "v_and_b32 v60, 0x1010101, v60\n\t" \
  "v_and_b32 v61, 0x2020202, v61\n\t" \
  "v_and_b32 v62, 0x4040404, v62\n\t" \
  "v_and_b32 v63, 0x8080808, v63\n\t" \
  "v_lshlrev_b32 v64, 0, v36\n\t" \
  "v_lshlrev_b32 v65, 2, v37\n\t" \
  "v_lshlrev_b32 v66, 4, v38\n\t" \
  "v_lshlrev_b32 v67, 6, v39\n\t" \
  "v_and_b32 v64, 0x3030303, v64\n\t" \
  "v_and_b32 v65, 0xc0c0c0c, v65\n\t" \
  "v_and_b32 v66, 0x30303030, v66\n\t" \
  "v_and_b32 v67, 0xc0c0c0c0, v67\n\t" \
  "v_or3_b32 v53, v60, v61, v62\n\t" \
  "v_or3_b32 v52, v64, v65, v66\n\t" \
  "v_or_b32 v53, v53, v63\n\t" \
  "v_or_b32 v52, v52, v67\n\t" \
  "v_lshrrev_b32 v60, 12, v53\n\t" \
  "v_or_b32 v53, v53, v60\n\t" \
  "s_mov_b32 s64, 1\n\t" \
  "s_cbranch_vccz L_xd_%=\n\t" \
  "L_xe_%=:\n\t" \
  "v_ffbl_b32 v56, v54\n\t" \
  "v_mbcnt_lo_u32_b32 v57, vcc_lo, 0\n\t" \
  "v_add_u32 v61, -1, v54\n\t" \
  "v_and_b32 v58, 3, v56\n\t" \
  "v_bfe_u32 v59, v56, 3, 2\n\t" \
  "v_mbcnt_hi_u32_b32 v57, vcc_hi, v57\n\t" \
  "v_lshl_or_b32 v58, v58, 2, v59\n\t" \
  "v_add_u32 v57, s64, v57\n\t" \
  "v_and_b32 v54, v54, v61\n\t" \
  "v_and_b32 v59, 7, v58\n\t" \
  "v_min_u32 v57, 64, v57\n\t" \
  "s_bcnt1_i32_b64 s99, vcc\n\t" \
  "v_perm_b32 v60, v37, v36, v59\n\t" \
  "v_perm_b32 v61, v39, v38, v59\n\t" \
  "v_cndmask_b32 v57, v79, v57, vcc\n\t" \
  "v_bfe_i32 v62, v56, 1, 1\n\t" \
  "s_add_u32 s64, s64, s99\n\t" \
  "v_cmp_ne_u32 vcc, 0, v54\n\t" \
  "v_bfi_b32 v60, v62, v61, v60\n\t" \
  "v_lshl_add_u32 v57, v57, 1, s53\n\t" \
  "v_and_b32 v60, 0x7c, v60\n\t" \
  "v_or_b32 v58, v58, v80\n\t" \
  "v_lshl_or_b32 v58, v60, 9, v58\n\t" \
  "ds_write_b16 v57, v58\n\t" \
  "s_cmp_gt_u32 s64, 64\n\t" \
  "s_cbranch_scc1 L_xd_%=\n\t" \
  "s_cbranch_vccnz L_xe_%=\n\t" \
  "L_xd_%=:\n\t" \
  "ds_read_u16 v56, v82\n\t" \
  "s_cmp_gt_u32 s64, 64\n\t" \
  "s_cselect_b32 s99, 0x80, 0\n\t" \
  "s_or_b32 s87, s99, s50\n\t" \
  "s_or_b32 s87, s87, 0x400\n\t" \
  "s_waitcnt lgkmcnt(0)\n\t" \
  "v_writelane_b32 v56, s87, 0\n\t" \
  "v_perm_b32 v53, v56, v53, v76\n\t" \
  "buffer_store_dwordx2 v[52:53], v81, s[40:43], s96 offen offset:3072 sc0 sc1\n\t" \
  "s_cmp_eq_u32 s99, 0\n\t" \
  "s_cbranch_scc1 L_xn_%=\n\t" \
  "s_add_u32 s96, s96, 3584\n\t" \
  "buffer_store_dwordx4 v[0:3], v80, s[40:43], s96 offen offset:0 sc0 sc1\n\t" \
  "buffer_store_dwordx4 v[4:7], v80, s[40:43], s96 offen offset:1024 sc0 sc1\n\t" \
  "buffer_store_dwordx4 v[8:11], v80, s[40:43], s96 offen offset:2048 sc0 sc1\n\t" \
  "buffer_store_dwordx4 v[12:15], v80, s[40:43], s96 offen offset:3072 sc0 sc1\n\t" \
  "L_xn_%=:\n\t" \
  "s_add_u32 s54, s54, 1\n\t" \
  "s_cmp_lt_u32 s54, 16\n\t" \
  "s_cbranch_scc1 L_xl_%=\n\t" \
  "v_and_b32 v56, 0x38, v81\n\t" \
  "global_load_dwordx2 v[58:59], v56, s[38:39] offset:192\n\t" \
  "s_bfe_u32 s99, s98, 0x5000b\n\t" \
  "s_lshl2_add_u32 s99, s99, s63\n\t" \
  "s_lshl_b32 s99, s99, 2\n\t" \
  "v_mov_b32 v57, s75\n\t" \
  "s_mov_b32 s42, 0x5028000\n\t" \
  "s_sub_u32 s55, s55, 1\n\t" \
  "s_waitcnt vmcnt(0)\n\t" \
  "v_add_co_u32 v58, vcc, s99, v58\n\t" \
  "s_nop 1\n\t" \
  "v_addc_co_u32 v59, vcc, 0, v59, vcc\n\t" \
  "s_mov_b64 exec, 0x7f\n\t" \
  "global_store_dword v[58:59], v57, off offset:1536 sc0 sc1\n\t" \
  "s_mov_b64 exec, -1\n\t" \
  "s_nop 4\n\t" \
  "s_branch L_first_%=\n\t" \
  "L_rd_%=:\n\t" \
  "v_accvgpr_read_b32 v0, a0\n\t" \
  "v_accvgpr_read_b32 v1, a4\n\t" \
  "v_accvgpr_read_b32 v2, a8\n\t" \
  "v_accvgpr_read_b32 v3, a12\n\t" \
  "v_accvgpr_read_b32 v4, a1\n\t" \
  "v_accvgpr_read_b32 v5, a5\n\t" \
  "v_accvgpr_read_b32 v6, a9\n\t" \
  "v_accvgpr_read_b32 v7, a13\n\t" \
  "v_accvgpr_read_b32 v8, a2\n\t" \
  "v_accvgpr_read_b32 v9, a6\n\t" \
  "v_accvgpr_read_b32 v10, a10\n\t" \
  "v_accvgpr_read_b32 v11, a14\n\t" \
  "v_accvgpr_read_b32 v12, a3\n\t" \
  "v_accvgpr_read_b32 v13, a7\n\t" \
  "v_accvgpr_read_b32 v14, a11\n\t" \
  "v_accvgpr_read_b32 v15, a15\n\t" \
  "s_setpc_b64 s[76:77]\n\t" \
  "v_accvgpr_read_b32 v0, a16\n\t" \
  "v_accvgpr_read_b32 v1, a20\n\t" \
  "v_accvgpr_read_b32 v2, a24\n\t" \
  "v_accvgpr_read_b32 v3, a28\n\t" \
  "v_accvgpr_read_b32 v4, a17\n\t" \
  "v_accvgpr_read_b32 v5, a21\n\t" \
  "v_accvgpr_read_b32 v6, a25\n\t" \
  "v_accvgpr_read_b32 v7, a29\n\t" \
  "v_accvgpr_read_b32 v8, a18\n\t" \
  "v_accvgpr_read_b32 v9, a22\n\t" \
  "v_accvgpr_read_b32 v10, a26\n\t" \
  "v_accvgpr_read_b32 v11, a30\n\t" \
  "v_accvgpr_read_b32 v12, a19\n\t" \
  "v_accvgpr_read_b32 v13, a23\n\t" \
  "v_accvgpr_read_b32 v14, a27\n\t" \
  "v_accvgpr_read_b32 v15, a31\n\t" \
  "s_setpc_b64 s[76:77]\n\t" \
  "v_accvgpr_read_b32 v0, a32\n\t" \
  "v_accvgpr_read_b32 v1, a36\n\t" \
  "v_accvgpr_read_b32 v2, a40\n\t" \
  "v_accvgpr_read_b32 v3, a44\n\t" \
  "v_accvgpr_read_b32 v4, a33\n\t" \
  "v_accvgpr_read_b32 v5, a37\n\t" \
  "v_accvgpr_read_b32 v6, a41\n\t" \
  "v_accvgpr_read_b32 v7, a45\n\t" \
  "v_accvgpr_read_b32 v8, a34\n\t" \
  "v_accvgpr_read_b32 v9, a38\n\t" \
  "v_accvgpr_read_b32 v10, a42\n\t" \
  "v_accvgpr_read_b32 v11, a46\n\t" \
  "v_accvgpr_read_b32 v12, a35\n\t" \
  "v_accvgpr_read_b32 v13, a39\n\t" \
  "v_accvgpr_read_b32 v14, a43\n\t" \
  "v_accvgpr_read_b32 v15, a47\n\t" \
  "s_setpc_b64 s[76:77]\n\t" \
  "v_accvgpr_read_b32 v0, a48\n\t" \
  "v_accvgpr_read_b32 v1, a52\n\t" \
  "v_accvgpr_read_b32 v2, a56\n\t" \
  "v_accvgpr_read_b32 v3, a60\n\t" \
  "v_accvgpr_read_b32 v4, a49\n\t" \
  "v_accvgpr_read_b32 v5, a53\n\t" \
  "v_accvgpr_read_b32 v6, a57\n\t" \
  "v_accvgpr_read_b32 v7, a61\n\t" \
  "v_accvgpr_read_b32 v8, a50\n\t" \
  "v_accvgpr_read_b32 v9, a54\n\t" \
  "v_accvgpr_read_b32 v10, a58\n\t" \
  "v_accvgpr_read_b32 v11, a62\n\t" \
  "v_accvgpr_read_b32 v12, a51\n\t" \
  "v_accvgpr_read_b32 v13, a55\n\t" \
  "v_accvgpr_read_b32 v14, a59\n\t" \
  "v_accvgpr_read_b32 v15, a63\n\t" \
  "s_setpc_b64 s[76:77]\n\t" \
  "v_accvgpr_read_b32 v0, a64\n\t" \
  "v_accvgpr_read_b32 v1, a68\n\t" \
  "v_accvgpr_read_b32 v2, a72\n\t" \
  "v_accvgpr_read_b32 v3, a76\n\t" \
  "v_accvgpr_read_b32 v4, a65\n\t" \
  "v_accvgpr_read_b32 v5, a69\n\t" \
  "v_accvgpr_read_b32 v6, a73\n\t" \
  "v_accvgpr_read_b32 v7, a77\n\t" \
  "v_accvgpr_read_b32 v8, a66\n\t" \
  "v_accvgpr_read_b32 v9, a70\n\t" \
  "v_accvgpr_read_b32 v10, a74\n\t" \
  "v_accvgpr_read_b32 v11, a78\n\t" \
  "v_accvgpr_read_b32 v12, a67\n\t" \
  "v_accvgpr_read_b32 v13, a71\n\t" \
  "v_accvgpr_read_b32 v14, a75\n\t" \
  "v_accvgpr_read_b32 v15, a79\n\t" \
  "s_setpc_b64 s[76:77]\n\t" \
  "v_accvgpr_read_b32 v0, a80\n\t" \
  "v_accvgpr_read_b32 v1, a84\n\t" \
  "v_accvgpr_read_b32 v2, a88\n\t" \
  "v_accvgpr_read_b32 v3, a92\n\t" \
  "v_accvgpr_read_b32 v4, a81\n\t" \
  "v_accvgpr_read_b32 v5, a85\n\t" \
  "v_accvgpr_read_b32 v6, a89\n\t" \
  "v_accvgpr_read_b32 v7, a93\n\t" \
  "v_accvgpr_read_b32 v8, a82\n\t" \
  "v_accvgpr_read_b32 v9, a86\n\t" \
  "v_accvgpr_read_b32 v10, a90\n\t" \
  "v_accvgpr_read_b32 v11, a94\n\t" \
  "v_accvgpr_read_b32 v12, a83\n\t" \
  "v_accvgpr_read_b32 v13, a87\n\t" \
  "v_accvgpr_read_b32 v14, a91\n\t" \
  "v_accvgpr_read_b32 v15, a95\n\t" \
  "s_setpc_b64 s[76:77]\n\t" \
  "v_accvgpr_read_b32 v0, a96\n\t" \
  "v_accvgpr_read_b32 v1, a100\n\t" \
  "v_accvgpr_read_b32 v2, a104\n\t" \
  "v_accvgpr_read_b32 v3, a108\n\t" \
  "v_accvgpr_read_b32 v4, a97\n\t" \
  "v_accvgpr_read_b32 v5, a101\n\t" \
  "v_accvgpr_read_b32 v6, a105\n\t" \
  "v_accvgpr_read_b32 v7, a109\n\t" \
  "v_accvgpr_read_b32 v8, a98\n\t" \
  "v_accvgpr_read_b32 v9, a102\n\t" \
  "v_accvgpr_read_b32 v10, a106\n\t" \
  "v_accvgpr_read_b32 v11, a110\n\t" \
  "v_accvgpr_read_b32 v12, a99\n\t" \
  "v_accvgpr_read_b32 v13, a103\n\t" \
  "v_accvgpr_read_b32 v14, a107\n\t" \
  "v_accvgpr_read_b32 v15, a111\n\t" \
  "s_setpc_b64 s[76:77]\n\t" \
  "v_accvgpr_read_b32 v0, a112\n\t" \
  "v_accvgpr_read_b32 v1, a116\n\t" \
  "v_accvgpr_read_b32 v2, a120\n\t" \
  "v_accvgpr_read_b32 v3, a124\n\t" \
  "v_accvgpr_read_b32 v4, a113\n\t" \
  "v_accvgpr_read_b32 v5, a117\n\t" \
  "v_accvgpr_read_b32 v6, a121\n\t" \
  "v_accvgpr_read_b32 v7, a125\n\t" \
  "v_accvgpr_read_b32 v8, a114\n\t" \
  "v_accvgpr_read_b32 v9, a118\n\t" \
  "v_accvgpr_read_b32 v10, a122\n\t" \
  "v_accvgpr_read_b32 v11, a126\n\t" \
  "v_accvgpr_read_b32 v12, a115\n\t" \
  "v_accvgpr_read_b32 v13, a119\n\t" \
  "v_accvgpr_read_b32 v14, a123\n\t" \
  "v_accvgpr_read_b32 v15, a127\n\t" \
  "s_setpc_b64 s[76:77]\n\t" \
  "v_accvgpr_read_b32 v0, a128\n\t" \
  "v_accvgpr_read_b32 v1, a132\n\t" \
  "v_accvgpr_read_b32 v2, a136\n\t" \
  "v_accvgpr_read_b32 v3, a140\n\t" \
  "v_accvgpr_read_b32 v4, a129\n\t" \
  "v_accvgpr_read_b32 v5, a133\n\t" \
  "v_accvgpr_read_b32 v6, a137\n\t" \
  "v_accvgpr_read_b32 v7, a141\n\t" \
  "v_accvgpr_read_b32 v8, a130\n\t" \
  "v_accvgpr_read_b32 v9, a134\n\t" \
  "v_accvgpr_read_b32 v10, a138\n\t" \
  "v_accvgpr_read_b32 v11, a142\n\t" \
  "v_accvgpr_read_b32 v12, a131\n\t" \
  "v_accvgpr_read_b32 v13, a135\n\t" \
  "v_accvgpr_read_b32 v14, a139\n\t" \
  "v_accvgpr_read_b32 v15, a143\n\t" \
  "s_setpc_b64 s[76:77]\n\t" \
  "v_accvgpr_read_b32 v0, a144\n\t" \
  "v_accvgpr_read_b32 v1, a148\n\t" \
  "v_accvgpr_read_b32 v2, a152\n\t" \
  "v_accvgpr_read_b32 v3, a156\n\t" \
  "v_accvgpr_read_b32 v4, a145\n\t" \
  "v_accvgpr_read_b32 v5, a149\n\t" \
  "v_accvgpr_read_b32 v6, a153\n\t" \
  "v_accvgpr_read_b32 v7, a157\n\t" \
  "v_accvgpr_read_b32 v8, a146\n\t" \
  "v_accvgpr_read_b32 v9, a150\n\t" \
  "v_accvgpr_read_b32 v10, a154\n\t" \
  "v_accvgpr_read_b32 v11, a158\n\t" \
  "v_accvgpr_read_b32 v12, a147\n\t" \
  "v_accvgpr_read_b32 v13, a151\n\t" \
  "v_accvgpr_read_b32 v14, a155\n\t" \
  "v_accvgpr_read_b32 v15, a159\n\t" \
  "s_setpc_b64 s[76:77]\n\t" \
  "v_accvgpr_read_b32 v0, a160\n\t" \
  "v_accvgpr_read_b32 v1, a164\n\t" \
  "v_accvgpr_read_b32 v2, a168\n\t" \
  "v_accvgpr_read_b32 v3, a172\n\t" \
  "v_accvgpr_read_b32 v4, a161\n\t" \
  "v_accvgpr_read_b32 v5, a165\n\t" \
  "v_accvgpr_read_b32 v6, a169\n\t" \
  "v_accvgpr_read_b32 v7, a173\n\t" \
  "v_accvgpr_read_b32 v8, a162\n\t" \
  "v_accvgpr_read_b32 v9, a166\n\t" \
  "v_accvgpr_read_b32 v10, a170\n\t" \
  "v_accvgpr_read_b32 v11, a174\n\t" \
  "v_accvgpr_read_b32 v12, a163\n\t" \
  "v_accvgpr_read_b32 v13, a167\n\t" \
  "v_accvgpr_read_b32 v14, a171\n\t" \
  "v_accvgpr_read_b32 v15, a175\n\t" \
  "s_setpc_b64 s[76:77]\n\t" \
  "v_accvgpr_read_b32 v0, a176\n\t" \
  "v_accvgpr_read_b32 v1, a180\n\t" \
  "v_accvgpr_read_b32 v2, a184\n\t" \
  "v_accvgpr_read_b32 v3, a188\n\t" \
  "v_accvgpr_read_b32 v4, a177\n\t" \
  "v_accvgpr_read_b32 v5, a181\n\t" \
  "v_accvgpr_read_b32 v6, a185\n\t" \
  "v_accvgpr_read_b32 v7, a189\n\t" \
  "v_accvgpr_read_b32 v8, a178\n\t" \
  "v_accvgpr_read_b32 v9, a182\n\t" \
  "v_accvgpr_read_b32 v10, a186\n\t" \
  "v_accvgpr_read_b32 v11, a190\n\t" \
  "v_accvgpr_read_b32 v12, a179\n\t" \
  "v_accvgpr_read_b32 v13, a183\n\t" \
  "v_accvgpr_read_b32 v14, a187\n\t" \
  "v_accvgpr_read_b32 v15, a191\n\t" \
  "s_setpc_b64 s[76:77]\n\t" \
  "v_accvgpr_read_b32 v0, a192\n\t" \
  "v_accvgpr_read_b32 v1, a196\n\t" \
  "v_accvgpr_read_b32 v2, a200\n\t" \
  "v_accvgpr_read_b32 v3, a204\n\t" \
  "v_accvgpr_read_b32 v4, a193\n\t" \
  "v_accvgpr_read_b32 v5, a197\n\t" \
  "v_accvgpr_read_b32 v6, a201\n\t" \
  "v_accvgpr_read_b32 v7, a205\n\t" \
  "v_accvgpr_read_b32 v8, a194\n\t" \
  "v_accvgpr_read_b32 v9, a198\n\t" \
  "v_accvgpr_read_b32 v10, a202\n\t" \
  "v_accvgpr_read_b32 v11, a206\n\t" \
  "v_accvgpr_read_b32 v12, a195\n\t" \
  "v_accvgpr_read_b32 v13, a199\n\t" \
  "v_accvgpr_read_b32 v14, a203\n\t" \
  "v_accvgpr_read_b32 v15, a207\n\t" \
  "s_setpc_b64 s[76:77]\n\t" \
  "v_accvgpr_read_b32 v0, a208\n\t" \
  "v_accvgpr_read_b32 v1, a212\n\t" \
  "v_accvgpr_read_b32 v2, a216\n\t" \
  "v_accvgpr_read_b32 v3, a220\n\t" \
  "v_accvgpr_read_b32 v4, a209\n\t" \
  "v_accvgpr_read_b32 v5, a213\n\t" \
  "v_accvgpr_read_b32 v6, a217\n\t" \
  "v_accvgpr_read_b32 v7, a221\n\t" \
  "v_accvgpr_read_b32 v8, a210\n\t" \
  "v_accvgpr_read_b32 v9, a214\n\t" \
  "v_accvgpr_read_b32 v10, a218\n\t" \
  "v_accvgpr_read_b32 v11, a222\n\t" \
  "v_accvgpr_read_b32 v12, a211\n\t" \
  "v_accvgpr_read_b32 v13, a215\n\t" \
  "v_accvgpr_read_b32 v14, a219\n\t" \
  "v_accvgpr_read_b32 v15, a223\n\t" \
  "s_setpc_b64 s[76:77]\n\t" \
  "v_accvgpr_read_b32 v0, a224\n\t" \
  "v_accvgpr_read_b32 v1, a228\n\t" \
  "v_accvgpr_read_b32 v2, a232\n\t" \
  "v_accvgpr_read_b32 v3, a236\n\t" \
  "v_accvgpr_read_b32 v4, a225\n\t" \
  "v_accvgpr_read_b32 v5, a229\n\t" \
  "v_accvgpr_read_b32 v6, a233\n\t" \
  "v_accvgpr_read_b32 v7, a237\n\t" \
  "v_accvgpr_read_b32 v8, a226\n\t" \
  "v_accvgpr_read_b32 v9, a230\n\t" \
  "v_accvgpr_read_b32 v10, a234\n\t" \
  "v_accvgpr_read_b32 v11, a238\n\t" \
  "v_accvgpr_read_b32 v12, a227\n\t" \
  "v_accvgpr_read_b32 v13, a231\n\t" \
  "v_accvgpr_read_b32 v14, a235\n\t" \
  "v_accvgpr_read_b32 v15, a239\n\t" \
  "s_setpc_b64 s[76:77]\n\t" \
  "v_accvgpr_read_b32 v0, a240\n\t" \
  "v_accvgpr_read_b32 v1, a244\n\t" \
  "v_accvgpr_read_b32 v2, a248\n\t" \
  "v_accvgpr_read_b32 v3, a252\n\t" \
  "v_accvgpr_read_b32 v4, a241\n\t" \
  "v_accvgpr_read_b32 v5, a245\n\t" \
  "v_accvgpr_read_b32 v6, a249\n\t" \
  "v_accvgpr_read_b32 v7, a253\n\t" \
  "v_accvgpr_read_b32 v8, a242\n\t" \
  "v_accvgpr_read_b32 v9, a246\n\t" \
  "v_accvgpr_read_b32 v10, a250\n\t" \
  "v_accvgpr_read_b32 v11, a254\n\t" \
  "v_accvgpr_read_b32 v12, a243\n\t" \
  "v_accvgpr_read_b32 v13, a247\n\t" \
  "v_accvgpr_read_b32 v14, a251\n\t" \
  "v_accvgpr_read_b32 v15, a255\n\t" \
  "s_setpc_b64 s[76:77]\n\t" \
  "L_half_%=:\n\t" \
  "s_waitcnt lgkmcnt(2)\n\t" \
  "v_mfma_f32_16x16x128_f8f6f4 a[0:3], v[128:135], v[160:167], 0 blgp:1\n\t" \
  "v_mfma_f32_16x16x128_f8f6f4 a[4:7], v[128:135], v[168:175], 0 blgp:1\n\t" \
  "v_mfma_f32_16x16x128_f8f6f4 a[8:11], v[128:135], v[176:183], 0 blgp:1\n\t" \
  "v_mfma_f32_16x16x128_f8f6f4 a[12:15], v[128:135], v[184:191], 0 blgp:1\n\t" \
  "v_mfma_f32_16x16x128_f8f6f4 a[28:31], v[136:143], v[184:191], 0 blgp:1\n\t" \
  "v_mfma_f32_16x16x128_f8f6f4 a[24:27], v[136:143], v[176:183], 0 blgp:1\n\t" \
  "v_mfma_f32_16x16x128_f8f6f4 a[20:23], v[136:143], v[168:175], 0 blgp:1\n\t" \
  "v_mfma_f32_16x16x128_f8f6f4 a[16:19], v[136:143], v[160:167], 0 blgp:1\n\t" \
  "v_mfma_f32_16x16x128_f8f6f4 a[32:35], v[144:151], v[160:167], 0 blgp:1\n\t" \
  "v_mfma_f32_16x16x128_f8f6f4 a[36:39], v[144:151], v[168:175], 0 blgp:1\n\t" \
  "v_mfma_f32_16x16x128_f8f6f4 a[40:43], v[144:151], v[176:183], 0 blgp:1\n\t" \
  "v_mfma_f32_16x16x128_f8f6f4 a[44:47], v[144:151], v[184:191], 0 blgp:1\n\t" \
  "s_waitcnt lgkmcnt(0)\n\t" \
  "v_mfma_f32_16x16x128_f8f6f4 a[60:63], v[152:159], v[184:191], 0 blgp:1\n\t" \
  "v_mfma_f32_16x16x128_f8f6f4 a[56:59], v[152:159], v[176:183], 0 blgp:1\n\t" \
  "v_mfma_f32_16x16x128_f8f6f4 a[52:55], v[152:159], v[168:175], 0 blgp:1\n\t" \
  "buffer_load_dword v254, off, s[88:91], s81 sc1\n\t" \
  "v_mfma_f32_16x16x128_f8f6f4 a[48:51], v[152:159], v[160:167], 0 blgp:1\n\t" \
  "s_waitcnt vmcnt(9) lgkmcnt(0)\n\t" \
  "s_barrier\n\t" \
  "v_readlane_b32 s65, v255, s67\n\t" \
  "s_nop 3\n\t" \
  "s_nop 3\n\t" \
  "s_add_u32 s67, s67, 1\n\t" \
  "s_cmp_ge_u32 s69, s65\n\t" \
  "s_cbranch_scc0 L_w31_%=\n\t" \
  "L_ok31_%=:\n\t" \
  "s_add_i32 s52, s48, 0\n\t" \
  "s_add_u32 m0, s53, 32768\n\t" \
  "s_nop 0\n\t" \
  "buffer_load_dwordx4 v252, s[40:43], s52 offen offset:0 sc0 sc1 lds\n\t" \
  "buffer_load_dwordx4 v252, s[40:43], s52 offen offset:1024 sc0 sc1 lds\n\t" \
  "buffer_load_dwordx4 v252, s[40:43], s52 offen offset:2048 sc0 sc1 lds\n\t" \
  "buffer_load_dwordx4 v252, s[40:43], s52 offen offset:3072 sc0 sc1 lds\n\t" \
  "s_waitcnt lgkmcnt(2)\n\t" \
  "s_add_i32 s52, s49, 0\n\t" \
  "s_add_u32 m0, s53, 98304\n\t" \
  "s_nop 0\n\t" \
  "buffer_load_dwordx4 v252, s[40:43], s52 offen offset:0 sc0 sc1 lds\n\t" \
  "buffer_load_dwordx4 v252, s[40:43], s52 offen offset:1024 sc0 sc1 lds\n\t" \
  "buffer_load_dwordx4 v252, s[40:43], s52 offen offset:2048 sc0 sc1 lds\n\t" \
  "buffer_load_dwordx4 v252, s[40:43], s52 offen offset:3072 sc0 sc1 lds\n\t" \
  "s_waitcnt lgkmcnt(0)\n\t" \
  "s_waitcnt vmcnt(9) lgkmcnt(0)\n\t" \
  "s_barrier\n\t" \
  "s_mov_b32 s98, s70\n\t" \
  "s_cmp_gt_u32 s55, 1\n\t" \
  "s_cselect_b32 s70, s71, s70\n\t" \
  "s_mov_b32 s71, s72\n\t" \
  "s_mov_b32 s72, s73\n\t" \
  "ds_read_b128 v[128:131], v250 offset:32768\n\t" \
  "ds_read_b128 v[132:135], v250 offset:40960\n\t" \
  "s_mov_b32 s73, s74\n\t" \
  "s_and_b32 s99, s70, 15\n\t" \
  "s_mul_i32 s50, s99, 0x100800\n\t" \
  "s_lshl_b32 s64, s99, 24\n\t" \
  "ds_read_b128 v[136:139], v250 offset:33024\n\t" \
  "ds_read_b128 v[140:143], v250 offset:41216\n\t" \
  "s_lshr_b32 s96, s99, 2\n\t" \
  "s_lshl_b32 s96, s96, 9\n\t" \
  "s_and_b32 s99, s99, 3\n\t" \
  "s_lshl2_add_u32 s96, s99, s96\n\t" \
  "ds_read_b128 v[144:147], v250 offset:33280\n\t" \
  "s_bfe_u32 s99, s70, 0x60004\n\t" \
  "ds_read_b128 v[148:151], v250 offset:41472\n\t" \
  "s_mul_i32 s51, s99, 0x100800\n\t" \
  "ds_read_b128 v[152:155], v250 offset:33536\n\t" \
  "ds_read_b128 v[156:159], v250 offset:41728\n\t" \
  "s_waitcnt lgkmcnt(2)\n\t" \
  "ds_read_b128 v[160:163], v251 offset:32768\n\t" \
  "s_add_u32 s51, s51, 0x1008000\n\t" \
  "ds_read_b128 v[164:167], v251 offset:40960\n\t" \
  "s_lshl_b32 s99, s99, 10\n\t" \
  "ds_read_b128 v[168:171], v251 offset:33024\n\t" \
  "s_add_u32 s64, s64, s99\n\t" \
  "ds_read_b128 v[172:175], v251 offset:41216\n\t" \
  "s_bfe_u32 s97, s70, 0x40006\n\t" \
  "ds_read_b128 v[176:179], v251 offset:33280\n\t" \
  "s_add_u32 s97, s97, 4\n\t" \
  "ds_read_b128 v[180:183], v251 offset:41472\n\t" \
  "s_lshl_b32 s97, s97, 9\n\t" \
  "ds_read_b128 v[184:187], v251 offset:33536\n\t" \
  "s_bfe_u32 s99, s70, 0x20004\n\t" \
  "ds_read_b128 v[188:191], v251 offset:41728\n\t" \
  "s_lshl2_add_u32 s97, s99, s97\n\t" \
  "s_mov_b32 s54, 14\n\t" \
  "s_waitcnt lgkmcnt(0)\n\t" \
  "s_waitcnt vmcnt(8) lgkmcnt(0)\n\t" \
  "s_barrier\n\t" \
  "v_mfma_f32_16x16x128_f8f6f4 a[0:3], v[128:135], v[160:167], a[0:3] blgp:1\n\t" \
  "v_readfirstlane_b32 s87, v254\n\t" \
  "v_readlane_b32 s65, v255, s67\n\t" \
  "v_mfma_f32_16x16x128_f8f6f4 a[4:7], v[128:135], v[168:175], a[4:7] blgp:1\n\t" \
  "v_mfma_f32_16x16x128_f8f6f4 a[8:11], v[128:135], v[176:183], a[8:11] blgp:1\n\t" \
  "s_max_u32 s69, s69, s87\n\t" \
  "s_add_u32 s67, s67, 1\n\t" \
  "s_cmp_ge_u32 s69, s65\n\t" \
  "s_cbranch_scc0 L_w32_%=\n\t" \
  "L_ok32_%=:\n\t" \
  "v_mfma_f32_16x16x128_f8f6f4 a[12:15], v[128:135], v[184:191], a[12:15] blgp:1\n\t" \
  "v_mfma_f32_16x16x128_f8f6f4 a[28:31], v[136:143], v[184:191], a[28:31] blgp:1\n\t" \
  "s_add_i32 s52, s48, 32768\n\t" \
  "s_add_u32 m0, s53, 65536\n\t" \
  "s_nop 0\n\t" \
  "v_mfma_f32_16x16x128_f8f6f4 a[24:27], v[136:143], v[176:183], a[24:27] blgp:1\n\t" \
  "buffer_load_dwordx4 v252, s[40:43], s52 offen offset:0 sc0 sc1 lds\n\t" \
  "v_mfma_f32_16x16x128_f8f6f4 a[20:23], v[136:143], v[168:175], a[20:23] blgp:1\n\t" \
  "buffer_load_dwordx4 v252, s[40:43], s52 offen offset:1024 sc0 sc1 lds\n\t" \
  "v_mfma_f32_16x16x128_f8f6f4 a[16:19], v[136:143], v[160:167], a[16:19] blgp:1\n\t" \
  "v_mfma_f32_16x16x128_f8f6f4 a[32:35], v[144:151], v[160:167], a[32:35] blgp:1\n\t" \
  "v_mfma_f32_16x16x128_f8f6f4 a[36:39], v[144:151], v[168:175], a[36:39] blgp:1\n\t" \
  "buffer_load_dwordx4 v252, s[40:43], s52 offen offset:2048 sc0 sc1 lds\n\t" \
  "v_mfma_f32_16x16x128_f8f6f4 a[40:43], v[144:151], v[176:183], a[40:43] blgp:1\n\t" \
  "buffer_load_dwordx4 v252, s[40:43], s52 offen offset:3072 sc0 sc1 lds\n\t" \
  "v_mfma_f32_16x16x128_f8f6f4 a[44:47], v[144:151], v[184:191], a[44:47] blgp:1\n\t" \
  "v_mfma_f32_16x16x128_f8f6f4 a[60:63], v[152:159], v[184:191], a[60:63] blgp:1\n\t" \
  "v_mfma_f32_16x16x128_f8f6f4 a[56:59], v[152:159], v[176:183], a[56:59] blgp:1\n\t" \
  "v_mfma_f32_16x16x128_f8f6f4 a[52:55], v[152:159], v[168:175], a[52:55] blgp:1\n\t" \
  "v_mfma_f32_16x16x128_f8f6f4 a[48:51], v[152:159], v[160:167], a[48:51] blgp:1\n\t" \
  "s_waitcnt lgkmcnt(2)\n\t" \
  "buffer_load_dword v248, v252, s[92:95], s96 offen\n\t" \
  "buffer_load_dword v249, v252, s[92:95], s97 offen\n\t" \
  "s_add_i32 s52, s49, 32768\n\t" \
  "s_add_u32 m0, s53, 131072\n\t" \
  "s_nop 0\n\t" \
  "buffer_load_dwordx4 v252, s[40:43], s52 offen offset:0 sc0 sc1 lds\n\t" \
  "buffer_load_dwordx4 v252, s[40:43], s52 offen offset:1024 sc0 sc1 lds\n\t" \
  "buffer_load_dwordx4 v252, s[40:43], s52 offen offset:2048 sc0 sc1 lds\n\t" \
  "buffer_load_dwordx4 v252, s[40:43], s52 offen offset:3072 sc0 sc1 lds\n\t" \
  "s_waitcnt lgkmcnt(0)\n\t" \
  "s_waitcnt vmcnt(10) lgkmcnt(0)\n\t" \
  "s_barrier\n\t" \
  "ds_read_b128 v[160:163], v251 offset:0\n\t" \
  "ds_read_b128 v[164:167], v251 offset:8192\n\t" \
  "ds_read_b128 v[168:171], v251 offset:256\n\t" \
  "ds_read_b128 v[128:131], v250 offset:0\n\t" \
  "ds_read_b128 v[132:135], v250 offset:8192\n\t" \
  "ds_read_b128 v[172:175], v251 offset:8448\n\t" \
  "ds_read_b128 v[176:179], v251 offset:512\n\t" \
  "ds_read_b128 v[180:183], v251 offset:8704\n\t" \
  "ds_read_b128 v[136:139], v250 offset:256\n\t" \
  "ds_read_b128 v[140:143], v250 offset:8448\n\t" \
  "ds_read_b128 v[184:187], v251 offset:768\n\t" \
  "ds_read_b128 v[188:191], v251 offset:8960\n\t" \
  "s_add_u32 s48, s48, 0x10000\n\t" \
  "s_add_u32 s49, s49, 0x10000\n\t" \
  "ds_read_b128 v[144:147], v250 offset:512\n\t" \
  "ds_read_b128 v[148:151], v250 offset:8704\n\t" \
  "ds_read_b128 v[152:155], v250 offset:768\n\t" \
  "ds_read_b128 v[156:159], v250 offset:8960\n\t" \
  "L_hk_%=:\n\t" \
  "s_waitcnt lgkmcnt(2)\n\t" \
  "v_mfma_f32_16x16x128_f8f6f4 a[0:3], v[128:135], v[160:167], a[0:3] blgp:1\n\t" \
  "v_mfma_f32_16x16x128_f8f6f4 a[4:7], v[128:135], v[168:175], a[4:7] blgp:1\n\t" \
  "v_mfma_f32_16x16x128_f8f6f4 a[8:11], v[128:135], v[176:183], a[8:11] blgp:1\n\t" \
  "v_mfma_f32_16x16x128_f8f6f4 a[12:15], v[128:135], v[184:191], a[12:15] blgp:1\n\t" \
  "v_mfma_f32_16x16x128_f8f6f4 a[28:31], v[136:143], v[184:191], a[28:31] blgp:1\n\t" \
  "v_mfma_f32_16x16x128_f8f6f4 a[24:27], v[136:143], v[176:183], a[24:27] blgp:1\n\t" \
  "v_mfma_f32_16x16x128_f8f6f4 a[20:23], v[136:143], v[168:175], a[20:23] blgp:1\n\t" \
  "v_mfma_f32_16x16x128_f8f6f4 a[16:19], v[136:143], v[160:167], a[16:19] blgp:1\n\t" \
  "v_mfma_f32_16x16x128_f8f6f4 a[32:35], v[144:151], v[160:167], a[32:35] blgp:1\n\t" \
  "v_mfma_f32_16x16x128_f8f6f4 a[36:39], v[144:151], v[168:175], a[36:39] blgp:1\n\t" \
  "v_mfma_f32_16x16x128_f8f6f4 a[40:43], v[144:151], v[176:183], a[40:43] blgp:1\n\t" \
  "v_mfma_f32_16x16x128_f8f6f4 a[44:47], v[144:151], v[184:191], a[44:47] blgp:1\n\t" \
  "s_waitcnt lgkmcnt(0)\n\t" \
  "v_mfma_f32_16x16x128_f8f6f4 a[60:63], v[152:159], v[184:191], a[60:63] blgp:1\n\t" \
  "v_mfma_f32_16x16x128_f8f6f4 a[56:59], v[152:159], v[176:183], a[56:59] blgp:1\n\t" \
  "v_mfma_f32_16x16x128_f8f6f4 a[52:55], v[152:159], v[168:175], a[52:55] blgp:1\n\t" \
  "buffer_load_dword v254, off, s[88:91], s81 sc1\n\t" \
  "v_mfma_f32_16x16x128_f8f6f4 a[48:51], v[152:159], v[160:167], a[48:51] blgp:1\n\t" \
  "s_waitcnt vmcnt(9) lgkmcnt(0)\n\t" \
  "s_barrier\n\t" \
  "v_readlane_b32 s65, v255, s67\n\t" \
  "s_nop 3\n\t" \
  "s_nop 3\n\t" \
  "s_add_u32 s67, s67, 1\n\t" \
  "s_cmp_ge_u32 s69, s65\n\t" \
  "s_cbranch_scc0 L_w33_%=\n\t" \
  "L_ok33_%=:\n\t" \
  "s_add_i32 s52, s48, 0\n\t" \
  "s_add_u32 m0, s53, 32768\n\t" \
  "s_nop 0\n\t" \
  "buffer_load_dwordx4 v252, s[40:43], s52 offen offset:0 sc0 sc1 lds\n\t" \
  "buffer_load_dwordx4 v252, s[40:43], s52 offen offset:1024 sc0 sc1 lds\n\t" \
  "buffer_load_dwordx4 v252, s[40:43], s52 offen offset:2048 sc0 sc1 lds\n\t" \
  "buffer_load_dwordx4 v252, s[40:43], s52 offen offset:3072 sc0 sc1 lds\n\t" \
  "s_waitcnt lgkmcnt(2)\n\t" \
  "s_add_i32 s52, s49, 0\n\t" \
  "s_add_u32 m0, s53, 98304\n\t" \
  "s_nop 0\n\t" \
  "buffer_load_dwordx4 v252, s[40:43], s52 offen offset:0 sc0 sc1 lds\n\t" \
  "buffer_load_dwordx4 v252, s[40:43], s52 offen offset:1024 sc0 sc1 lds\n\t" \
  "buffer_load_dwordx4 v252, s[40:43], s52 offen offset:2048 sc0 sc1 lds\n\t" \
  "buffer_load_dwordx4 v252, s[40:43], s52 offen offset:3072 sc0 sc1 lds\n\t" \
  "s_waitcnt lgkmcnt(0)\n\t" \
  "s_waitcnt vmcnt(9) lgkmcnt(0)\n\t" \
  "s_barrier\n\t" \
  "ds_read_b128 v[128:131], v250 offset:32768\n\t" \
  "ds_read_b128 v[132:135], v250 offset:40960\n\t" \
  "ds_read_b128 v[136:139], v250 offset:33024\n\t" \
  "ds_read_b128 v[140:143], v250 offset:41216\n\t" \
  "ds_read_b128 v[144:147], v250 offset:33280\n\t" \
  "ds_read_b128 v[148:151], v250 offset:41472\n\t" \
  "ds_read_b128 v[152:155], v250 offset:33536\n\t" \
  "ds_read_b128 v[156:159], v250 offset:41728\n\t" \
  "s_waitcnt lgkmcnt(2)\n\t" \
  "ds_read_b128 v[160:163], v251 offset:32768\n\t" \
  "ds_read_b128 v[164:167], v251 offset:40960\n\t" \
  "ds_read_b128 v[168:171], v251 offset:33024\n\t" \
  "ds_read_b128 v[172:175], v251 offset:41216\n\t" \
  "ds_read_b128 v[176:179], v251 offset:33280\n\t" \
  "ds_read_b128 v[180:183], v251 offset:41472\n\t" \
  "ds_read_b128 v[184:187], v251 offset:33536\n\t" \
  "ds_read_b128 v[188:191], v251 offset:41728\n\t" \
  "s_waitcnt lgkmcnt(0)\n\t" \
  "s_waitcnt vmcnt(8) lgkmcnt(0)\n\t" \
  "s_barrier\n\t" \
  "v_mfma_f32_16x16x128_f8f6f4 a[0:3], v[128:135], v[160:167], a[0:3] blgp:1\n\t" \
  "v_readfirstlane_b32 s87, v254\n\t" \
  "v_readlane_b32 s65, v255, s67\n\t" \
  "v_mfma_f32_16x16x128_f8f6f4 a[4:7], v[128:135], v[168:175], a[4:7] blgp:1\n\t" \
  "v_mfma_f32_16x16x128_f8f6f4 a[8:11], v[128:135], v[176:183], a[8:11] blgp:1\n\t" \
  "s_max_u32 s69, s69, s87\n\t" \
  "s_add_u32 s67, s67, 1\n\t" \
  "s_cmp_ge_u32 s69, s65\n\t" \
  "s_cbranch_scc0 L_w34_%=\n\t" \
  "L_ok34_%=:\n\t" \
  "v_mfma_f32_16x16x128_f8f6f4 a[12:15], v[128:135], v[184:191], a[12:15] blgp:1\n\t" \
  "v_mfma_f32_16x16x128_f8f6f4 a[28:31], v[136:143], v[184:191], a[28:31] blgp:1\n\t" \
  "s_add_i32 s52, s48, 32768\n\t" \
  "s_add_u32 m0, s53, 65536\n\t" \
  "s_nop 0\n\t" \
  "v_mfma_f32_16x16x128_f8f6f4 a[24:27], v[136:143], v[176:183], a[24:27] blgp:1\n\t" \
  "buffer_load_dwordx4 v252, s[40:43], s52 offen offset:0 sc0 sc1 lds\n\t" \
  "v_mfma_f32_16x16x128_f8f6f4 a[20:23], v[136:143], v[168:175], a[20:23] blgp:1\n\t" \
  "buffer_load_dwordx4 v252, s[40:43], s52 offen offset:1024 sc0 sc1 lds\n\t" \
  "v_mfma_f32_16x16x128_f8f6f4 a[16:19], v[136:143], v[160:167], a[16:19] blgp:1\n\t" \
  "v_mfma_f32_16x16x128_f8f6f4 a[32:35], v[144:151], v[160:167], a[32:35] blgp:1\n\t" \
  "v_mfma_f32_16x16x128_f8f6f4 a[36:39], v[144:151], v[168:175], a[36:39] blgp:1\n\t" \
  "buffer_load_dwordx4 v252, s[40:43], s52 offen offset:2048 sc0 sc1 lds\n\t" \
  "v_mfma_f32_16x16x128_f8f6f4 a[40:43], v[144:151], v[176:183], a[40:43] blgp:1\n\t" \
  "buffer_load_dwordx4 v252, s[40:43], s52 offen offset:3072 sc0 sc1 lds\n\t" \
  "v_mfma_f32_16x16x128_f8f6f4 a[44:47], v[144:151], v[184:191], a[44:47] blgp:1\n\t" \
  "v_mfma_f32_16x16x128_f8f6f4 a[60:63], v[152:159], v[184:191], a[60:63] blgp:1\n\t" \
  "v_mfma_f32_16x16x128_f8f6f4 a[56:59], v[152:159], v[176:183], a[56:59] blgp:1\n\t" \
  "v_mfma_f32_16x16x128_f8f6f4 a[52:55], v[152:159], v[168:175], a[52:55] blgp:1\n\t" \
  "v_mfma_f32_16x16x128_f8f6f4 a[48:51], v[152:159], v[160:167], a[48:51] blgp:1\n\t" \
  "s_waitcnt lgkmcnt(2)\n\t" \
  "s_add_i32 s52, s49, 32768\n\t" \
  "s_add_u32 m0, s53, 131072\n\t" \
  "s_nop 0\n\t" \
  "buffer_load_dwordx4 v252, s[40:43], s52 offen offset:0 sc0 sc1 lds\n\t" \
  "buffer_load_dwordx4 v252, s[40:43], s52 offen offset:1024 sc0 sc1 lds\n\t" \
  "buffer_load_dwordx4 v252, s[40:43], s52 offen offset:2048 sc0 sc1 lds\n\t" \
  "buffer_load_dwordx4 v252, s[40:43], s52 offen offset:3072 sc0 sc1 lds\n\t" \
  "s_waitcnt lgkmcnt(0)\n\t" \
  "s_waitcnt vmcnt(8) lgkmcnt(0)\n\t" \
  "s_barrier\n\t" \
  "ds_read_b128 v[160:163], v251 offset:0\n\t" \
  "ds_read_b128 v[164:167], v251 offset:8192\n\t" \
  "ds_read_b128 v[168:171], v251 offset:256\n\t" \
  "ds_read_b128 v[128:131], v250 offset:0\n\t" \
  "ds_read_b128 v[132:135], v250 offset:8192\n\t" \
  "ds_read_b128 v[172:175], v251 offset:8448\n\t" \
  "ds_read_b128 v[176:179], v251 offset:512\n\t" \
  "ds_read_b128 v[180:183], v251 offset:8704\n\t" \
  "ds_read_b128 v[136:139], v250 offset:256\n\t" \
  "ds_read_b128 v[140:143], v250 offset:8448\n\t" \
  "ds_read_b128 v[184:187], v251 offset:768\n\t" \
  "ds_read_b128 v[188:191], v251 offset:8960\n\t" \
  "s_add_u32 s48, s48, 0x10000\n\t" \
  "s_add_u32 s49, s49, 0x10000\n\t" \
  "ds_read_b128 v[144:147], v250 offset:512\n\t" \
  "ds_read_b128 v[148:151], v250 offset:8704\n\t" \
  "s_sub_u32 s54, s54, 1\n\t" \
  "s_cmp_lg_u32 s54, 0\n\t" \
  "ds_read_b128 v[152:155], v250 offset:768\n\t" \
  "ds_read_b128 v[156:159], v250 offset:8960\n\t" \
  "s_cbranch_scc1 L_hk_%=\n\t" \
  "s_waitcnt lgkmcnt(2)\n\t" \
  "v_mfma_f32_16x16x128_f8f6f4 a[0:3], v[128:135], v[160:167], a[0:3] blgp:1\n\t" \
  "v_mfma_f32_16x16x128_f8f6f4 a[4:7], v[128:135], v[168:175], a[4:7] blgp:1\n\t" \
  "v_mfma_f32_16x16x128_f8f6f4 a[8:11], v[128:135], v[176:183], a[8:11] blgp:1\n\t" \
  "v_mfma_f32_16x16x128_f8f6f4 a[12:15], v[128:135], v[184:191], a[12:15] blgp:1\n\t" \
  "v_mfma_f32_16x16x128_f8f6f4 a[28:31], v[136:143], v[184:191], a[28:31] blgp:1\n\t" \
  "v_mfma_f32_16x16x128_f8f6f4 a[24:27], v[136:143], v[176:183], a[24:27] blgp:1\n\t" \
  "v_mfma_f32_16x16x128_f8f6f4 a[20:23], v[136:143], v[168:175], a[20:23] blgp:1\n\t" \
  "v_mfma_f32_16x16x128_f8f6f4 a[16:19], v[136:143], v[160:167], a[16:19] blgp:1\n\t" \
  "v_mfma_f32_16x16x128_f8f6f4 a[32:35], v[144:151], v[160:167], a[32:35] blgp:1\n\t" \
  "v_mfma_f32_16x16x128_f8f6f4 a[36:39], v[144:151], v[168:175], a[36:39] blgp:1\n\t" \
  "v_max_u32 v255, v248, v249\n\t" \
  "v_mfma_f32_16x16x128_f8f6f4 a[40:43], v[144:151], v[176:183], a[40:43] blgp:1\n\t" \
  "v_mfma_f32_16x16x128_f8f6f4 a[44:47], v[144:151], v[184:191], a[44:47] blgp:1\n\t" \
  "v_add_u32 v255, s86, v255\n\t" \
  "s_mov_b32 s67, 0\n\t" \
  "s_waitcnt lgkmcnt(0)\n\t" \
  "v_mfma_f32_16x16x128_f8f6f4 a[60:63], v[152:159], v[184:191], a[60:63] blgp:1\n\t" \
  "v_mfma_f32_16x16x128_f8f6f4 a[56:59], v[152:159], v[176:183], a[56:59] blgp:1\n\t" \
  "v_mfma_f32_16x16x128_f8f6f4 a[52:55], v[152:159], v[168:175], a[52:55] blgp:1\n\t" \
  "buffer_load_dword v254, off, s[88:91], s81 sc1\n\t" \
  "v_mfma_f32_16x16x128_f8f6f4 a[48:51], v[152:159], v[160:167], a[48:51] blgp:1\n\t" \
  "s_mov_b32 s48, s50\n\t" \
  "s_mov_b32 s49, s51\n\t" \
  "s_waitcnt vmcnt(9) lgkmcnt(0)\n\t" \
  "s_barrier\n\t" \
  "v_readlane_b32 s65, v255, s67\n\t" \
  "s_nop 3\n\t" \
  "s_nop 3\n\t" \
  "s_add_u32 s67, s67, 1\n\t" \
  "s_cmp_ge_u32 s69, s65\n\t" \
  "s_cbranch_scc0 L_w35_%=\n\t" \
  "L_ok35_%=:\n\t" \
  "s_add_i32 s52, s48, 0\n\t" \
  "s_add_u32 m0, s53, 32768\n\t" \
  "s_nop 0\n\t" \
  "buffer_load_dwordx4 v252, s[40:43], s52 offen offset:0 sc0 sc1 lds\n\t" \
  "buffer_load_dwordx4 v252, s[40:43], s52 offen offset:1024 sc0 sc1 lds\n\t" \
  "buffer_load_dwordx4 v252, s[40:43], s52 offen offset:2048 sc0 sc1 lds\n\t" \
  "buffer_load_dwordx4 v252, s[40:43], s52 offen offset:3072 sc0 sc1 lds\n\t" \
  "s_waitcnt lgkmcnt(2)\n\t" \
  "s_add_i32 s52, s49, 0\n\t" \
  "s_add_u32 m0, s53, 98304\n\t" \
  "s_nop 0\n\t" \
  "buffer_load_dwordx4 v252, s[40:43], s52 offen offset:0 sc0 sc1 lds\n\t" \
  "buffer_load_dwordx4 v252, s[40:43], s52 offen offset:1024 sc0 sc1 lds\n\t" \
  "buffer_load_dwordx4 v252, s[40:43], s52 offen offset:2048 sc0 sc1 lds\n\t" \
  "buffer_load_dwordx4 v252, s[40:43], s52 offen offset:3072 sc0 sc1 lds\n\t" \
  "s_add_i32 s52, s49, 16384\n\t" \
  "s_add_u32 m0, s53, 114688\n\t" \
  "s_nop 0\n\t" \
  "buffer_load_dwordx4 v252, s[40:43], s52 offen offset:0 sc0 sc1 lds\n\t" \
  "buffer_load_dwordx4 v252, s[40:43], s52 offen offset:1024 sc0 sc1 lds\n\t" \
  "buffer_load_dwordx4 v252, s[40:43], s52 offen offset:2048 sc0 sc1 lds\n\t" \
  "buffer_load_dwordx4 v252, s[40:43], s52 offen offset:3072 sc0 sc1 lds\n\t" \
  "s_waitcnt lgkmcnt(0)\n\t" \
  "s_waitcnt vmcnt(13) lgkmcnt(0)\n\t" \
  "s_barrier\n\t" \
  "ds_read_b128 v[128:131], v250 offset:32768\n\t" \
  "ds_read_b128 v[132:135], v250 offset:40960\n\t" \
  "ds_read_b128 v[136:139], v250 offset:33024\n\t" \
  "ds_read_b128 v[140:143], v250 offset:41216\n\t" \
  "s_mov_b32 s80, s56\n\t" \
  "s_mov_b32 s56, s64\n\t" \
  "ds_read_b128 v[144:147], v250 offset:33280\n\t" \
  "ds_read_b128 v[148:151], v250 offset:41472\n\t" \
  "ds_read_b128 v[152:155], v250 offset:33536\n\t" \
  "ds_read_b128 v[156:159], v250 offset:41728\n\t" \
  "s_waitcnt lgkmcnt(2)\n\t" \
  "ds_read_b128 v[160:163], v251 offset:32768\n\t" \
  "ds_read_b128 v[164:167], v251 offset:40960\n\t" \
  "ds_read_b128 v[168:171], v251 offset:33024\n\t" \
  "ds_read_b128 v[172:175], v251 offset:41216\n\t" \
  "ds_read_b128 v[176:179], v251 offset:33280\n\t" \
  "ds_read_b128 v[180:183], v251 offset:41472\n\t" \
  "ds_read_b128 v[184:187], v251 offset:33536\n\t" \
  "ds_read_b128 v[188:191], v251 offset:41728\n\t" \
  "s_add_i32 s52, s48, 16384\n\t" \
  "s_add_u32 m0, s53, 49152\n\t" \
  "s_nop 0\n\t" \
  "buffer_load_dwordx4 v252, s[40:43], s52 offen offset:0 sc0 sc1 lds\n\t" \
  "buffer_load_dwordx4 v252, s[40:43], s52 offen offset:1024 sc0 sc1 lds\n\t" \
  "s_waitcnt lgkmcnt(0)\n\t" \
  "buffer_load_dwordx4 v252, s[40:43], s52 offen offset:2048 sc0 sc1 lds\n\t" \
  "buffer_load_dwordx4 v252, s[40:43], s52 offen offset:3072 sc0 sc1 lds\n\t" \
  "s_waitcnt vmcnt(16) lgkmcnt(0)\n\t" \
  "s_barrier\n\t" \
  "v_mfma_f32_16x16x128_f8f6f4 a[0:3], v[128:135], v[160:167], a[0:3] blgp:1\n\t" \
  "v_readfirstlane_b32 s87, v254\n\t" \
  "v_readlane_b32 s65, v255, s67\n\t" \
  "v_mfma_f32_16x16x128_f8f6f4 a[4:7], v[128:135], v[168:175], a[4:7] blgp:1\n\t" \
  "v_mfma_f32_16x16x128_f8f6f4 a[8:11], v[128:135], v[176:183], a[8:11] blgp:1\n\t" \
  "s_max_u32 s69, s69, s87\n\t" \
  "s_add_u32 s67, s67, 1\n\t" \
  "s_cmp_ge_u32 s69, s65\n\t" \
  "s_cbranch_scc0 L_w36_%=\n\t" \
  "L_ok36_%=:\n\t" \
  "v_mfma_f32_16x16x128_f8f6f4 a[12:15], v[128:135], v[184:191], a[12:15] blgp:1\n\t" \
  "v_mfma_f32_16x16x128_f8f6f4 a[28:31], v[136:143], v[184:191], a[28:31] blgp:1\n\t" \
  "s_add_i32 s52, s48, 32768\n\t" \
  "s_add_u32 m0, s53, 65536\n\t" \
  "s_nop 0\n\t" \
  "v_mfma_f32_16x16x128_f8f6f4 a[24:27], v[136:143], v[176:183], a[24:27] blgp:1\n\t" \
  "buffer_load_dwordx4 v252, s[40:43], s52 offen offset:0 sc0 sc1 lds\n\t" \
  "v_mfma_f32_16x16x128_f8f6f4 a[20:23], v[136:143], v[168:175], a[20:23] blgp:1\n\t" \
  "buffer_load_dwordx4 v252, s[40:43], s52 offen offset:1024 sc0 sc1 lds\n\t" \
  "v_mfma_f32_16x16x128_f8f6f4 a[16:19], v[136:143], v[160:167], a[16:19] blgp:1\n\t" \
  "v_mfma_f32_16x16x128_f8f6f4 a[32:35], v[144:151], v[160:167], a[32:35] blgp:1\n\t" \
  "v_mfma_f32_16x16x128_f8f6f4 a[36:39], v[144:151], v[168:175], a[36:39] blgp:1\n\t" \
  "buffer_load_dwordx4 v252, s[40:43], s52 offen offset:2048 sc0 sc1 lds\n\t" \
  "v_mfma_f32_16x16x128_f8f6f4 a[40:43], v[144:151], v[176:183], a[40:43] blgp:1\n\t" \
  "buffer_load_dwordx4 v252, s[40:43], s52 offen offset:3072 sc0 sc1 lds\n\t" \
  "v_mfma_f32_16x16x128_f8f6f4 a[44:47], v[144:151], v[184:191], a[44:47] blgp:1\n\t" \
  "v_mfma_f32_16x16x128_f8f6f4 a[60:63], v[152:159], v[184:191], a[60:63] blgp:1\n\t" \
  "v_mfma_f32_16x16x128_f8f6f4 a[56:59], v[152:159], v[176:183], a[56:59] blgp:1\n\t" \
  "v_mfma_f32_16x16x128_f8f6f4 a[52:55], v[152:159], v[168:175], a[52:55] blgp:1\n\t" \
  "v_mfma_f32_16x16x128_f8f6f4 a[48:51], v[152:159], v[160:167], a[48:51] blgp:1\n\t" \
  "s_waitcnt lgkmcnt(2)\n\t" \
  "s_add_i32 s52, s49, 49152\n\t" \
  "s_add_u32 m0, s53, 147456\n\t" \
  "s_nop 0\n\t" \
  "buffer_load_dwordx4 v252, s[40:43], s52 offen offset:0 sc0 sc1 lds\n\t" \
  "buffer_load_dwordx4 v252, s[40:43], s52 offen offset:1024 sc0 sc1 lds\n\t" \
  "buffer_load_dwordx4 v252, s[40:43], s52 offen offset:2048 sc0 sc1 lds\n\t" \
  "buffer_load_dwordx4 v252, s[40:43], s52 offen offset:3072 sc0 sc1 lds\n\t" \
  "s_add_i32 s52, s49, 32768\n\t" \
  "s_add_u32 m0, s53, 131072\n\t" \
  "s_nop 0\n\t" \
  "buffer_load_dwordx4 v252, s[40:43], s52 offen offset:0 sc0 sc1 lds\n\t" \
  "buffer_load_dwordx4 v252, s[40:43], s52 offen offset:1024 sc0 sc1 lds\n\t" \
  "buffer_load_dwordx4 v252, s[40:43], s52 offen offset:2048 sc0 sc1 lds\n\t" \
  "buffer_load_dwordx4 v252, s[40:43], s52 offen offset:3072 sc0 sc1 lds\n\t" \
  "s_waitcnt lgkmcnt(0)\n\t" \
  "s_waitcnt vmcnt(16) lgkmcnt(0)\n\t" \
  "s_barrier\n\t" \
  "ds_read_b128 v[160:163], v251 offset:0\n\t" \
  "ds_read_b128 v[164:167], v251 offset:8192\n\t" \
  "ds_read_b128 v[168:171], v251 offset:256\n\t" \
  "ds_read_b128 v[128:131], v250 offset:0\n\t" \
  "ds_read_b128 v[132:135], v250 offset:8192\n\t" \
  "ds_read_b128 v[172:175], v251 offset:8448\n\t" \
  "ds_read_b128 v[176:179], v251 offset:512\n\t" \
  "ds_read_b128 v[180:183], v251 offset:8704\n\t" \
  "ds_read_b128 v[136:139], v250 offset:256\n\t" \
  "ds_read_b128 v[140:143], v250 offset:8448\n\t" \
  "ds_read_b128 v[184:187], v251 offset:768\n\t" \
  "ds_read_b128 v[188:191], v251 offset:8960\n\t" \
  "s_add_u32 s48, s48, 0x10000\n\t" \
  "s_add_u32 s49, s49, 0x10000\n\t" \
  "ds_read_b128 v[144:147], v250 offset:512\n\t" \
  "ds_read_b128 v[148:151], v250 offset:8704\n\t" \
  "ds_read_b128 v[152:155], v250 offset:768\n\t" \
  "ds_read_b128 v[156:159], v250 offset:8960\n\t" \
  "s_mov_b32 s46, 0x10000000\n\t" \
  "s_nop 15\n\t" \
  "v_accvgpr_read_b32 v192, a0\n\t" \
  "v_accvgpr_read_b32 v193, a4\n\t" \
  "v_accvgpr_read_b32 v194, a8\n\t" \
  "v_accvgpr_read_b32 v195, a12\n\t" \
  "v_pk_fma_f32 v[196:197], v[192:193], s[58:59], s[58:59] op_sel:[0,0,1] op_sel_hi:[1,0,1]\n\t" \
  "v_pk_fma_f32 v[198:199], v[194:195], s[58:59], s[58:59] op_sel:[0,0,1] op_sel_hi:[1,0,1]\n\t" \
  "v_exp_f32_e64 v196, v196 clamp\n\t" \
  "v_exp_f32_e64 v197, v197 clamp\n\t" \
  "v_exp_f32_e64 v198, v198 clamp\n\t" \
  "v_exp_f32_e64 v199, v199 clamp\n\t" \
  "v_pk_fma_f32 v[196:197], v[196:197], s[60:61], s[60:61] op_sel:[0,1,0] op_sel_hi:[1,1,0]\n\t" \
  "v_pk_fma_f32 v[198:199], v[198:199], s[60:61], s[60:61] op_sel:[0,1,0] op_sel_hi:[1,1,0]\n\t" \
  "v_rcp_f32 v196, v196\n\t" \
  "v_rcp_f32 v197, v197\n\t" \
  "v_rcp_f32 v198, v198\n\t" \
  "v_rcp_f32 v199, v199\n\t" \
  "v_pk_mul_f32 v[192:193], v[192:193], v[196:197]\n\t" \
  "v_pk_mul_f32 v[194:195], v[194:195], v[198:199]\n\t" \
  "s_add_u32 s57, s80, 0\n\t" \
  "buffer_store_dwordx4 v[192:195], v253, s[44:47], s57 offen offset:0 nt\n\t" \
  "v_accvgpr_read_b32 v200, a1\n\t" \
  "v_accvgpr_read_b32 v201, a5\n\t" \
  "v_accvgpr_read_b32 v202, a9\n\t" \
  "v_accvgpr_read_b32 v203, a13\n\t" \
  "v_pk_fma_f32 v[204:205], v[200:201], s[58:59], s[58:59] op_sel:[0,0,1] op_sel_hi:[1,0,1]\n\t" \
  "v_pk_fma_f32 v[206:207], v[202:203], s[58:59], s[58:59] op_sel:[0,0,1] op_sel_hi:[1,0,1]\n\t" \
  "v_exp_f32_e64 v204, v204 clamp\n\t" \
  "v_exp_f32_e64 v205, v205 clamp\n\t" \
  "v_exp_f32_e64 v206, v206 clamp\n\t" \
  "v_exp_f32_e64 v207, v207 clamp\n\t" \
  "v_pk_fma_f32 v[204:205], v[204:205], s[60:61], s[60:61] op_sel:[0,1,0] op_sel_hi:[1,1,0]\n\t" \
  "v_pk_fma_f32 v[206:207], v[206:207], s[60:61], s[60:61] op_sel:[0,1,0] op_sel_hi:[1,1,0]\n\t" \
  "v_rcp_f32 v204, v204\n\t" \
  "v_rcp_f32 v205, v205\n\t" \
  "v_rcp_f32 v206, v206\n\t" \
  "v_rcp_f32 v207, v207\n\t" \
  "v_pk_mul_f32 v[200:201], v[200:201], v[204:205]\n\t" \
  "v_pk_mul_f32 v[202:203], v[202:203], v[206:207]\n\t" \
  "s_add_u32 s57, s80, 65536\n\t" \
  "buffer_store_dwordx4 v[200:203], v253, s[44:47], s57 offen offset:0 nt\n\t" \
  "v_accvgpr_read_b32 v208, a2\n\t" \
  "v_accvgpr_read_b32 v209, a6\n\t" \
  "v_accvgpr_read_b32 v210, a10\n\t" \
  "v_accvgpr_read_b32 v211, a14\n\t" \
  "v_pk_fma_f32 v[212:213], v[208:209], s[58:59], s[58:59] op_sel:[0,0,1] op_sel_hi:[1,0,1]\n\t" \
  "v_pk_fma_f32 v[214:215], v[210:211], s[58:59], s[58:59] op_sel:[0,0,1] op_sel_hi:[1,0,1]\n\t" \
  "v_exp_f32_e64 v212, v212 clamp\n\t" \
  "v_exp_f32_e64 v213, v213 clamp\n\t" \
  "v_exp_f32_e64 v214, v214 clamp\n\t" \
  "v_exp_f32_e64 v215, v215 clamp\n\t" \
  "v_pk_fma_f32 v[212:213], v[212:213], s[60:61], s[60:61] op_sel:[0,1,0] op_sel_hi:[1,1,0]\n\t" \
  "v_pk_fma_f32 v[214:215], v[214:215], s[60:61], s[60:61] op_sel:[0,1,0] op_sel_hi:[1,1,0]\n\t" \
  "v_rcp_f32 v212, v212\n\t" \
  "v_rcp_f32 v213, v213\n\t" \
  "v_rcp_f32 v214, v214\n\t" \
  "v_rcp_f32 v215, v215\n\t" \
  "v_pk_mul_f32 v[208:209], v[208:209], v[212:213]\n\t" \
  "v_pk_mul_f32 v[210:211], v[210:211], v[214:215]\n\t" \
  "s_add_u32 s57, s80, 131072\n\t" \
  "buffer_store_dwordx4 v[208:211], v253, s[44:47], s57 offen offset:0 nt\n\t" \
  "v_accvgpr_read_b32 v216, a3\n\t" \
  "v_accvgpr_read_b32 v217, a7\n\t" \
  "v_accvgpr_read_b32 v218, a11\n\t" \
  "v_accvgpr_read_b32 v219, a15\n\t" \
  "v_pk_fma_f32 v[220:221], v[216:217], s[58:59], s[58:59] op_sel:[0,0,1] op_sel_hi:[1,0,1]\n\t" \
  "v_pk_fma_f32 v[222:223], v[218:219], s[58:59], s[58:59] op_sel:[0,0,1] op_sel_hi:[1,0,1]\n\t" \
  "v_exp_f32_e64 v220, v220 clamp\n\t" \
  "v_exp_f32_e64 v221, v221 clamp\n\t" \
  "v_exp_f32_e64 v222, v222 clamp\n\t" \
  "v_exp_f32_e64 v223, v223 clamp\n\t" \
  "v_pk_fma_f32 v[220:221], v[220:221], s[60:61], s[60:61] op_sel:[0,1,0] op_sel_hi:[1,1,0]\n\t" \
  "v_pk_fma_f32 v[222:223], v[222:223], s[60:61], s[60:61] op_sel:[0,1,0] op_sel_hi:[1,1,0]\n\t" \
  "v_rcp_f32 v220, v220\n\t" \
  "v_rcp_f32 v221, v221\n\t" \
  "v_rcp_f32 v222, v222\n\t" \
  "v_rcp_f32 v223, v223\n\t" \
  "v_pk_mul_f32 v[216:217], v[216:217], v[220:221]\n\t" \
  "v_pk_mul_f32 v[218:219], v[218:219], v[222:223]\n\t" \
  "s_add_u32 s57, s80, 196608\n\t" \
  "buffer_store_dwordx4 v[216:219], v253, s[44:47], s57 offen offset:0 nt\n\t" \
  "v_accvgpr_read_b32 v192, a16\n\t" \
  "v_accvgpr_read_b32 v193, a20\n\t" \
  "v_accvgpr_read_b32 v194, a24\n\t" \
  "v_accvgpr_read_b32 v195, a28\n\t" \
  "v_pk_fma_f32 v[196:197], v[192:193], s[58:59], s[58:59] op_sel:[0,0,1] op_sel_hi:[1,0,1]\n\t" \
  "v_pk_fma_f32 v[198:199], v[194:195], s[58:59], s[58:59] op_sel:[0,0,1] op_sel_hi:[1,0,1]\n\t" \
  "v_exp_f32_e64 v196, v196 clamp\n\t" \
  "v_exp_f32_e64 v197, v197 clamp\n\t" \
  "v_exp_f32_e64 v198, v198 clamp\n\t" \
  "v_exp_f32_e64 v199, v199 clamp\n\t" \
  "v_pk_fma_f32 v[196:197], v[196:197], s[60:61], s[60:61] op_sel:[0,1,0] op_sel_hi:[1,1,0]\n\t" \
  "v_pk_fma_f32 v[198:199], v[198:199], s[60:61], s[60:61] op_sel:[0,1,0] op_sel_hi:[1,1,0]\n\t" \
  "v_rcp_f32 v196, v196\n\t" \
  "v_rcp_f32 v197, v197\n\t" \
  "v_rcp_f32 v198, v198\n\t" \
  "v_rcp_f32 v199, v199\n\t" \
  "v_pk_mul_f32 v[192:193], v[192:193], v[196:197]\n\t" \
  "v_pk_mul_f32 v[194:195], v[194:195], v[198:199]\n\t" \
  "s_add_u32 s57, s80, 1048576\n\t" \
  "buffer_store_dwordx4 v[192:195], v253, s[44:47], s57 offen offset:0 nt\n\t" \
  "v_accvgpr_read_b32 v200, a17\n\t" \
  "v_accvgpr_read_b32 v201, a21\n\t" \
  "v_accvgpr_read_b32 v202, a25\n\t" \
  "v_accvgpr_read_b32 v203, a29\n\t" \
  "v_pk_fma_f32 v[204:205], v[200:201], s[58:59], s[58:59] op_sel:[0,0,1] op_sel_hi:[1,0,1]\n\t" \
  "v_pk_fma_f32 v[206:207], v[202:203], s[58:59], s[58:59] op_sel:[0,0,1] op_sel_hi:[1,0,1]\n\t" \
  "v_exp_f32_e64 v204, v204 clamp\n\t" \
  "v_exp_f32_e64 v205, v205 clamp\n\t" \
  "v_exp_f32_e64 v206, v206 clamp\n\t" \
  "v_exp_f32_e64 v207, v207 clamp\n\t" \
  "v_pk_fma_f32 v[204:205], v[204:205], s[60:61], s[60:61] op_sel:[0,1,0] op_sel_hi:[1,1,0]\n\t" \
  "v_pk_fma_f32 v[206:207], v[206:207], s[60:61], s[60:61] op_sel:[0,1,0] op_sel_hi:[1,1,0]\n\t" \
  "v_rcp_f32 v204, v204\n\t" \
  "v_rcp_f32 v205, v205\n\t" \
  "v_rcp_f32 v206, v206\n\t" \
  "v_rcp_f32 v207, v207\n\t" \
  "v_pk_mul_f32 v[200:201], v[200:201], v[204:205]\n\t" \
  "v_pk_mul_f32 v[202:203], v[202:203], v[206:207]\n\t" \
  "s_add_u32 s57, s80, 1114112\n\t" \
  "buffer_store_dwordx4 v[200:203], v253, s[44:47], s57 offen offset:0 nt\n\t" \
  "v_accvgpr_read_b32 v208, a18\n\t" \
  "v_accvgpr_read_b32 v209, a22\n\t" \
  "v_accvgpr_read_b32 v210, a26\n\t" \
  "v_accvgpr_read_b32 v211, a30\n\t" \
  "v_pk_fma_f32 v[212:213], v[208:209], s[58:59], s[58:59] op_sel:[0,0,1] op_sel_hi:[1,0,1]\n\t" \
  "v_pk_fma_f32 v[214:215], v[210:211], s[58:59], s[58:59] op_sel:[0,0,1] op_sel_hi:[1,0,1]\n\t" \
  "v_exp_f32_e64 v212, v212 clamp\n\t" \
  "v_exp_f32_e64 v213, v213 clamp\n\t" \
  "v_exp_f32_e64 v214, v214 clamp\n\t" \
  "v_exp_f32_e64 v215, v215 clamp\n\t" \
  "v_pk_fma_f32 v[212:213], v[212:213], s[60:61], s[60:61] op_sel:[0,1,0] op_sel_hi:[1,1,0]\n\t" \
  "v_pk_fma_f32 v[214:215], v[214:215], s[60:61], s[60:61] op_sel:[0,1,0] op_sel_hi:[1,1,0]\n\t" \
  "v_rcp_f32 v212, v212\n\t" \
  "v_rcp_f32 v213, v213\n\t" \
  "v_rcp_f32 v214, v214\n\t" \
  "v_rcp_f32 v215, v215\n\t" \
  "v_pk_mul_f32 v[208:209], v[208:209], v[212:213]\n\t" \
  "v_pk_mul_f32 v[210:211], v[210:211], v[214:215]\n\t" \
  "s_add_u32 s57, s80, 1179648\n\t" \
  "buffer_store_dwordx4 v[208:211], v253, s[44:47], s57 offen offset:0 nt\n\t" \
  "v_accvgpr_read_b32 v216, a19\n\t" \
  "v_accvgpr_read_b32 v217, a23\n\t" \
  "v_accvgpr_read_b32 v218, a27\n\t" \
  "v_accvgpr_read_b32 v219, a31\n\t" \
  "v_pk_fma_f32 v[220:221], v[216:217], s[58:59], s[58:59] op_sel:[0,0,1] op_sel_hi:[1,0,1]\n\t" \
  "v_pk_fma_f32 v[222:223], v[218:219], s[58:59], s[58:59] op_sel:[0,0,1] op_sel_hi:[1,0,1]\n\t" \
  "v_exp_f32_e64 v220, v220 clamp\n\t" \
  "v_exp_f32_e64 v221, v221 clamp\n\t" \
  "v_exp_f32_e64 v222, v222 clamp\n\t" \
  "v_exp_f32_e64 v223, v223 clamp\n\t" \
  "v_pk_fma_f32 v[220:221], v[220:221], s[60:61], s[60:61] op_sel:[0,1,0] op_sel_hi:[1,1,0]\n\t" \
  "v_pk_fma_f32 v[222:223], v[222:223], s[60:61], s[60:61] op_sel:[0,1,0] op_sel_hi:[1,1,0]\n\t" \
  "v_rcp_f32 v220, v220\n\t" \
  "v_rcp_f32 v221, v221\n\t" \
  "v_rcp_f32 v222, v222\n\t" \
  "v_rcp_f32 v223, v223\n\t" \
  "v_pk_mul_f32 v[216:217], v[216:217], v[220:221]\n\t" \
  "v_pk_mul_f32 v[218:219], v[218:219], v[222:223]\n\t" \
  "s_add_u32 s57, s80, 1245184\n\t" \
  "buffer_store_dwordx4 v[216:219], v253, s[44:47], s57 offen offset:0 nt\n\t" \
  "v_accvgpr_read_b32 v192, a32\n\t" \
  "v_accvgpr_read_b32 v193, a36\n\t" \
  "v_accvgpr_read_b32 v194, a40\n\t" \
  "v_accvgpr_read_b32 v195, a44\n\t" \
  "v_pk_fma_f32 v[196:197], v[192:193], s[58:59], s[58:59] op_sel:[0,0,1] op_sel_hi:[1,0,1]\n\t" \
  "v_pk_fma_f32 v[198:199], v[194:195], s[58:59], s[58:59] op_sel:[0,0,1] op_sel_hi:[1,0,1]\n\t" \
  "v_exp_f32_e64 v196, v196 clamp\n\t" \
  "v_exp_f32_e64 v197, v197 clamp\n\t" \
  "v_exp_f32_e64 v198, v198 clamp\n\t" \
  "v_exp_f32_e64 v199, v199 clamp\n\t" \
  "v_pk_fma_f32 v[196:197], v[196:197], s[60:61], s[60:61] op_sel:[0,1,0] op_sel_hi:[1,1,0]\n\t" \
  "v_pk_fma_f32 v[198:199], v[198:199], s[60:61], s[60:61] op_sel:[0,1,0] op_sel_hi:[1,1,0]\n\t" \
  "v_rcp_f32 v196, v196\n\t" \
  "v_rcp_f32 v197, v197\n\t" \
  "v_rcp_f32 v198, v198\n\t" \
  "v_rcp_f32 v199, v199\n\t" \
  "v_pk_mul_f32 v[192:193], v[192:193], v[196:197]\n\t" \
  "v_pk_mul_f32 v[194:195], v[194:195], v[198:199]\n\t" \
  "s_add_u32 s57, s80, 2097152\n\t" \
  "buffer_store_dwordx4 v[192:195], v253, s[44:47], s57 offen offset:0 nt\n\t" \
  "v_accvgpr_read_b32 v200, a33\n\t" \
  "v_accvgpr_read_b32 v201, a37\n\t" \
  "v_accvgpr_read_b32 v202, a41\n\t" \
  "v_accvgpr_read_b32 v203, a45\n\t" \
  "v_pk_fma_f32 v[204:205], v[200:201], s[58:59], s[58:59] op_sel:[0,0,1] op_sel_hi:[1,0,1]\n\t" \
  "v_pk_fma_f32 v[206:207], v[202:203], s[58:59], s[58:59] op_sel:[0,0,1] op_sel_hi:[1,0,1]\n\t" \
  "v_exp_f32_e64 v204, v204 clamp\n\t" \
  "v_exp_f32_e64 v205, v205 clamp\n\t" \
  "v_exp_f32_e64 v206, v206 clamp\n\t" \
  "v_exp_f32_e64 v207, v207 clamp\n\t" \
  "v_pk_fma_f32 v[204:205], v[204:205], s[60:61], s[60:61] op_sel:[0,1,0] op_sel_hi:[1,1,0]\n\t" \
  "v_pk_fma_f32 v[206:207], v[206:207], s[60:61], s[60:61] op_sel:[0,1,0] op_sel_hi:[1,1,0]\n\t" \
  "v_rcp_f32 v204, v204\n\t" \
  "v_rcp_f32 v205, v205\n\t" \
  "v_rcp_f32 v206, v206\n\t" \
  "v_rcp_f32 v207, v207\n\t" \
  "v_pk_mul_f32 v[200:201], v[200:201], v[204:205]\n\t" \
  "v_pk_mul_f32 v[202:203], v[202:203], v[206:207]\n\t" \
  "s_add_u32 s57, s80, 2162688\n\t" \
  "buffer_store_dwordx4 v[200:203], v253, s[44:47], s57 offen offset:0 nt\n\t" \
  "v_accvgpr_read_b32 v208, a34\n\t" \
  "v_accvgpr_read_b32 v209, a38\n\t" \
  "v_accvgpr_read_b32 v210, a42\n\t" \
  "v_accvgpr_read_b32 v211, a46\n\t" \
  "v_pk_fma_f32 v[212:213], v[208:209], s[58:59], s[58:59] op_sel:[0,0,1] op_sel_hi:[1,0,1]\n\t" \
  "v_pk_fma_f32 v[214:215], v[210:211], s[58:59], s[58:59] op_sel:[0,0,1] op_sel_hi:[1,0,1]\n\t" \
  "v_exp_f32_e64 v212, v212 clamp\n\t" \
  "v_exp_f32_e64 v213, v213 clamp\n\t" \
  "v_exp_f32_e64 v214, v214 clamp\n\t" \
  "v_exp_f32_e64 v215, v215 clamp\n\t" \
  "v_pk_fma_f32 v[212:213], v[212:213], s[60:61], s[60:61] op_sel:[0,1,0] op_sel_hi:[1,1,0]\n\t" \
  "v_pk_fma_f32 v[214:215], v[214:215], s[60:61], s[60:61] op_sel:[0,1,0] op_sel_hi:[1,1,0]\n\t" \
  "v_rcp_f32 v212, v212\n\t" \
  "v_rcp_f32 v213, v213\n\t" \
  "v_rcp_f32 v214, v214\n\t" \
  "v_rcp_f32 v215, v215\n\t" \
  "v_pk_mul_f32 v[208:209], v[208:209], v[212:213]\n\t" \
  "v_pk_mul_f32 v[210:211], v[210:211], v[214:215]\n\t" \
  "s_add_u32 s57, s80, 2228224\n\t" \
  "buffer_store_dwordx4 v[208:211], v253, s[44:47], s57 offen offset:0 nt\n\t" \
  "v_accvgpr_read_b32 v216, a35\n\t" \
  "v_accvgpr_read_b32 v217, a39\n\t" \
  "v_accvgpr_read_b32 v218, a43\n\t" \
  "v_accvgpr_read_b32 v219, a47\n\t" \
  "v_pk_fma_f32 v[220:221], v[216:217], s[58:59], s[58:59] op_sel:[0,0,1] op_sel_hi:[1,0,1]\n\t" \
  "v_pk_fma_f32 v[222:223], v[218:219], s[58:59], s[58:59] op_sel:[0,0,1] op_sel_hi:[1,0,1]\n\t" \
  "v_exp_f32_e64 v220, v220 clamp\n\t" \
  "v_exp_f32_e64 v221, v221 clamp\n\t" \
  "v_exp_f32_e64 v222, v222 clamp\n\t" \
  "v_exp_f32_e64 v223, v223 clamp\n\t" \
  "v_pk_fma_f32 v[220:221], v[220:221], s[60:61], s[60:61] op_sel:[0,1,0] op_sel_hi:[1,1,0]\n\t" \
  "v_pk_fma_f32 v[222:223], v[222:223], s[60:61], s[60:61] op_sel:[0,1,0] op_sel_hi:[1,1,0]\n\t" \
  "v_rcp_f32 v220, v220\n\t" \
  "v_rcp_f32 v221, v221\n\t" \
  "v_rcp_f32 v222, v222\n\t" \
  "v_rcp_f32 v223, v223\n\t" \
  "v_pk_mul_f32 v[216:217], v[216:217], v[220:221]\n\t" \
  "v_pk_mul_f32 v[218:219], v[218:219], v[222:223]\n\t" \
  "s_add_u32 s57, s80, 2293760\n\t" \
  "buffer_store_dwordx4 v[216:219], v253, s[44:47], s57 offen offset:0 nt\n\t" \
  "v_accvgpr_read_b32 v192, a48\n\t" \
  "v_accvgpr_read_b32 v193, a52\n\t" \
  "v_accvgpr_read_b32 v194, a56\n\t" \
  "v_accvgpr_read_b32 v195, a60\n\t" \
  "v_pk_fma_f32 v[196:197], v[192:193], s[58:59], s[58:59] op_sel:[0,0,1] op_sel_hi:[1,0,1]\n\t" \
  "v_pk_fma_f32 v[198:199], v[194:195], s[58:59], s[58:59] op_sel:[0,0,1] op_sel_hi:[1,0,1]\n\t" \
  "v_exp_f32_e64 v196, v196 clamp\n\t" \
  "v_exp_f32_e64 v197, v197 clamp\n\t" \
  "v_exp_f32_e64 v198, v198 clamp\n\t" \
  "v_exp_f32_e64 v199, v199 clamp\n\t" \
  "v_pk_fma_f32 v[196:197], v[196:197], s[60:61], s[60:61] op_sel:[0,1,0] op_sel_hi:[1,1,0]\n\t" \
  "v_pk_fma_f32 v[198:199], v[198:199], s[60:61], s[60:61] op_sel:[0,1,0] op_sel_hi:[1,1,0]\n\t" \
  "v_rcp_f32 v196, v196\n\t" \
  "v_rcp_f32 v197, v197\n\t" \
  "v_rcp_f32 v198, v198\n\t" \
  "v_rcp_f32 v199, v199\n\t" \
  "v_pk_mul_f32 v[192:193], v[192:193], v[196:197]\n\t" \
  "v_pk_mul_f32 v[194:195], v[194:195], v[198:199]\n\t" \
  "s_add_u32 s57, s80, 3145728\n\t" \
  "buffer_store_dwordx4 v[192:195], v253, s[44:47], s57 offen offset:0 nt\n\t" \
  "v_accvgpr_read_b32 v200, a49\n\t" \
  "v_accvgpr_read_b32 v201, a53\n\t" \
  "v_accvgpr_read_b32 v202, a57\n\t" \
  "v_accvgpr_read_b32 v203, a61\n\t" \
  "v_pk_fma_f32 v[204:205], v[200:201], s[58:59], s[58:59] op_sel:[0,0,1] op_sel_hi:[1,0,1]\n\t" \
  "v_pk_fma_f32 v[206:207], v[202:203], s[58:59], s[58:59] op_sel:[0,0,1] op_sel_hi:[1,0,1]\n\t" \
  "v_exp_f32_e64 v204, v204 clamp\n\t" \
  "v_exp_f32_e64 v205, v205 clamp\n\t" \
  "v_exp_f32_e64 v206, v206 clamp\n\t" \
  "v_exp_f32_e64 v207, v207 clamp\n\t" \
  "v_pk_fma_f32 v[204:205], v[204:205], s[60:61], s[60:61] op_sel:[0,1,0] op_sel_hi:[1,1,0]\n\t" \
  "v_pk_fma_f32 v[206:207], v[206:207], s[60:61], s[60:61] op_sel:[0,1,0] op_sel_hi:[1,1,0]\n\t" \
  "v_rcp_f32 v204, v204\n\t" \
  "v_rcp_f32 v205, v205\n\t" \
  "v_rcp_f32 v206, v206\n\t" \
  "v_rcp_f32 v207, v207\n\t" \
  "v_pk_mul_f32 v[200:201], v[200:201], v[204:205]\n\t" \
  "v_pk_mul_f32 v[202:203], v[202:203], v[206:207]\n\t" \
  "s_add_u32 s57, s80, 3211264\n\t" \
  "buffer_store_dwordx4 v[200:203], v253, s[44:47], s57 offen offset:0 nt\n\t" \
  "v_accvgpr_read_b32 v208, a50\n\t" \
  "v_accvgpr_read_b32 v209, a54\n\t" \
  "v_accvgpr_read_b32 v210, a58\n\t" \
  "v_accvgpr_read_b32 v211, a62\n\t" \
  "v_pk_fma_f32 v[212:213], v[208:209], s[58:59], s[58:59] op_sel:[0,0,1] op_sel_hi:[1,0,1]\n\t" \
  "v_pk_fma_f32 v[214:215], v[210:211], s[58:59], s[58:59] op_sel:[0,0,1] op_sel_hi:[1,0,1]\n\t" \
  "v_exp_f32_e64 v212, v212 clamp\n\t" \
  "v_exp_f32_e64 v213, v213 clamp\n\t" \
  "v_exp_f32_e64 v214, v214 clamp\n\t" \
  "v_exp_f32_e64 v215, v215 clamp\n\t" \
  "v_pk_fma_f32 v[212:213], v[212:213], s[60:61], s[60:61] op_sel:[0,1,0] op_sel_hi:[1,1,0]\n\t" \
  "v_pk_fma_f32 v[214:215], v[214:215], s[60:61], s[60:61] op_sel:[0,1,0] op_sel_hi:[1,1,0]\n\t" \
  "v_rcp_f32 v212, v212\n\t" \
  "v_rcp_f32 v213, v213\n\t" \
  "v_rcp_f32 v214, v214\n\t" \
  "v_rcp_f32 v215, v215\n\t" \
  "v_pk_mul_f32 v[208:209], v[208:209], v[212:213]\n\t" \
  "v_pk_mul_f32 v[210:211], v[210:211], v[214:215]\n\t" \
  "s_add_u32 s57, s80, 3276800\n\t" \
  "buffer_store_dwordx4 v[208:211], v253, s[44:47], s57 offen offset:0 nt\n\t" \
  "v_accvgpr_read_b32 v216, a51\n\t" \
  "v_accvgpr_read_b32 v217, a55\n\t" \
  "v_accvgpr_read_b32 v218, a59\n\t" \
  "v_accvgpr_read_b32 v219, a63\n\t" \
  "v_pk_fma_f32 v[220:221], v[216:217], s[58:59], s[58:59] op_sel:[0,0,1] op_sel_hi:[1,0,1]\n\t" \
  "v_pk_fma_f32 v[222:223], v[218:219], s[58:59], s[58:59] op_sel:[0,0,1] op_sel_hi:[1,0,1]\n\t" \
  "v_exp_f32_e64 v220, v220 clamp\n\t" \
  "v_exp_f32_e64 v221, v221 clamp\n\t" \
  "v_exp_f32_e64 v222, v222 clamp\n\t" \
  "v_exp_f32_e64 v223, v223 clamp\n\t" \
  "v_pk_fma_f32 v[220:221], v[220:221], s[60:61], s[60:61] op_sel:[0,1,0] op_sel_hi:[1,1,0]\n\t" \
  "v_pk_fma_f32 v[222:223], v[222:223], s[60:61], s[60:61] op_sel:[0,1,0] op_sel_hi:[1,1,0]\n\t" \
  "v_rcp_f32 v220, v220\n\t" \
  "v_rcp_f32 v221, v221\n\t" \
  "v_rcp_f32 v222, v222\n\t" \
  "v_rcp_f32 v223, v223\n\t" \
  "v_pk_mul_f32 v[216:217], v[216:217], v[220:221]\n\t" \
  "v_pk_mul_f32 v[218:219], v[218:219], v[222:223]\n\t" \
  "s_add_u32 s57, s80, 3342336\n\t" \
  "buffer_store_dwordx4 v[216:219], v253, s[44:47], s57 offen offset:0 nt\n\t" \
  "s_waitcnt vmcnt(0)\n\t" \
  "s_sub_u32 s55, s55, 1\n\t" \
  "s_branch L_first_%=\n\t" \
  "L_ct_%=:\n\t" \
  "global_load_dword a255, v240, s[34:35]\n\t" \
  "s_branch L_ctr_%=\n\t" \
  "L_w1_%=:\n\t" \
  "s_swappc_b64 s[76:77], s[78:79]\n\t" \
  "s_branch L_ok1_%=\n\t" \
  "L_w2_%=:\n\t" \
  "s_swappc_b64 s[76:77], s[78:79]\n\t" \
  "s_branch L_ok2_%=\n\t" \
  "L_w3_%=:\n\t" \
  "s_swappc_b64 s[76:77], s[78:79]\n\t" \
  "s_branch L_ok3_%=\n\t" \
  "L_w4_%=:\n\t" \
  "s_swappc_b64 s[76:77], s[78:79]\n\t" \
  "s_branch L_ok4_%=\n\t" \
  "L_w5_%=:\n\t" \
  "s_swappc_b64 s[76:77], s[78:79]\n\t" \
  "s_branch L_ok5_%=\n\t" \
  "L_w6_%=:\n\t" \
  "s_swappc_b64 s[76:77], s[78:79]\n\t" \
  "s_branch L_ok6_%=\n\t" \
  "L_w7_%=:\n\t" \
  "s_swappc_b64 s[76:77], s[78:79]\n\t" \
  "s_branch L_ok7_%=\n\t" \
  "L_w8_%=:\n\t" \
  "s_swappc_b64 s[76:77], s[78:79]\n\t" \
  "s_branch L_ok8_%=\n\t" \
  "L_w9_%=:\n\t" \
  "s_swappc_b64 s[76:77], s[78:79]\n\t" \
  "s_branch L_ok9_%=\n\t" \
  "L_w10_%=:\n\t" \
  "s_swappc_b64 s[76:77], s[78:79]\n\t" \
  "s_branch L_ok10_%=\n\t" \
  "L_w11_%=:\n\t" \
  "s_swappc_b64 s[76:77], s[78:79]\n\t" \
  "s_branch L_ok11_%=\n\t" \
  "L_w12_%=:\n\t" \
  "s_swappc_b64 s[76:77], s[78:79]\n\t" \
  "s_branch L_ok12_%=\n\t" \
  "L_w13_%=:\n\t" \
  "s_swappc_b64 s[76:77], s[78:79]\n\t" \
  "s_branch L_ok13_%=\n\t" \
  "L_w14_%=:\n\t" \
  "s_swappc_b64 s[76:77], s[78:79]\n\t" \
  "s_branch L_ok14_%=\n\t" \
  "L_w15_%=:\n\t" \
  "s_swappc_b64 s[76:77], s[78:79]\n\t" \
  "s_branch L_ok15_%=\n\t" \
  "L_w16_%=:\n\t" \
  "s_swappc_b64 s[76:77], s[78:79]\n\t" \
  "s_branch L_ok16_%=\n\t" \
  "L_w17_%=:\n\t" \
  "s_swappc_b64 s[76:77], s[78:79]\n\t" \
  "s_branch L_ok17_%=\n\t" \
  "L_w18_%=:\n\t" \
  "s_swappc_b64 s[76:77], s[78:79]\n\t" \
  "s_branch L_ok18_%=\n\t" \
  "L_w19_%=:\n\t" \
  "s_swappc_b64 s[76:77], s[78:79]\n\t" \
  "s_branch L_ok19_%=\n\t" \
  "L_w20_%=:\n\t" \
  "s_swappc_b64 s[76:77], s[78:79]\n\t" \
  "s_branch L_ok20_%=\n\t" \
  "L_w21_%=:\n\t" \
  "s_swappc_b64 s[76:77], s[78:79]\n\t" \
  "s_branch L_ok21_%=\n\t" \
  "L_w22_%=:\n\t" \
  "s_swappc_b64 s[76:77], s[78:79]\n\t" \
  "s_branch L_ok22_%=\n\t" \
  "L_w23_%=:\n\t" \
  "s_swappc_b64 s[76:77], s[78:79]\n\t" \
  "s_branch L_ok23_%=\n\t" \
  "L_w24_%=:\n\t" \
  "s_swappc_b64 s[76:77], s[78:79]\n\t" \
  "s_branch L_ok24_%=\n\t" \
  "L_w25_%=:\n\t" \
  "s_swappc_b64 s[76:77], s[78:79]\n\t" \
  "s_branch L_ok25_%=\n\t" \
  "L_w26_%=:\n\t" \
  "s_swappc_b64 s[76:77], s[78:79]\n\t" \
  "s_branch L_ok26_%=\n\t" \
  "L_w27_%=:\n\t" \
  "s_swappc_b64 s[76:77], s[78:79]\n\t" \
  "s_branch L_ok27_%=\n\t" \
  "L_w28_%=:\n\t" \
  "s_swappc_b64 s[76:77], s[78:79]\n\t" \
  "s_branch L_ok28_%=\n\t" \
  "L_w29_%=:\n\t" \
  "s_swappc_b64 s[76:77], s[78:79]\n\t" \
  "s_branch L_ok29_%=\n\t" \
  "L_w30_%=:\n\t" \
  "s_swappc_b64 s[76:77], s[78:79]\n\t" \
  "s_branch L_ok30_%=\n\t" \
  "L_w31_%=:\n\t" \
  "s_swappc_b64 s[76:77], s[78:79]\n\t" \
  "s_branch L_ok31_%=\n\t" \
  "L_w32_%=:\n\t" \
  "s_swappc_b64 s[76:77], s[78:79]\n\t" \
  "s_branch L_ok32_%=\n\t" \
  "L_w33_%=:\n\t" \
  "s_swappc_b64 s[76:77], s[78:79]\n\t" \
  "s_branch L_ok33_%=\n\t" \
  "L_w34_%=:\n\t" \
  "s_swappc_b64 s[76:77], s[78:79]\n\t" \
  "s_branch L_ok34_%=\n\t" \
  "L_w35_%=:\n\t" \
  "s_swappc_b64 s[76:77], s[78:79]\n\t" \
  "s_branch L_ok35_%=\n\t" \
  "L_w36_%=:\n\t" \
  "s_swappc_b64 s[76:77], s[78:79]\n\t" \
  "s_branch L_ok36_%=\n\t" \
  "L_w37_%=:\n\t" \
  "s_swappc_b64 s[76:77], s[78:79]\n\t" \
  "s_branch L_ok37_%=\n\t" \
  "L_w38_%=:\n\t" \
  "s_swappc_b64 s[76:77], s[78:79]\n\t" \
  "s_branch L_ok38_%=\n\t" \
  "L_slow_%=:\n\t" \
  "s_memrealtime s[84:85]\n\t" \
  "s_waitcnt vmcnt(0)\n\t" \
  "L_poll_%=:\n\t" \
  "buffer_load_dword v254, off, s[88:91], s81 sc1\n\t" \
  "s_waitcnt vmcnt(0)\n\t" \
  "s_memrealtime s[82:83]\n\t" \
  "v_readfirstlane_b32 s69, v254\n\t" \
  "s_waitcnt lgkmcnt(0)\n\t" \
  "s_sub_u32 s82, s82, s84\n\t" \
  "s_cmp_ge_u32 s69, s65\n\t" \
  "s_cbranch_scc1 L_ret_%=\n\t" \
  "s_cmp_gt_u32 s82, 50000000\n\t" \
  "s_cbranch_scc1 L_dead_%=\n\t" \
  "s_cmp_lt_u32 s82, 100000\n\t" \
  "s_cbranch_scc1 L_notmo_%=\n\t" \
  "buffer_load_dword v254, off, s[88:91], 0 offset:128 sc1\n\t" \
  "s_waitcnt vmcnt(0)\n\t" \
  "v_readfirstlane_b32 s87, v254\n\t" \
  "s_nop 3\n\t" \
  "s_cmp_eq_u32 s87, s75\n\t" \
  "s_cbranch_scc1 L_dead_%=\n\t" \
  "L_notmo_%=:\n\t" \
  "s_max_u32 s87, s69, s86\n\t" \
  "s_sub_u32 s87, s65, s87\n\t" \
  "s_cmp_lt_u32 s87, 64\n\t" \
  "s_cbranch_scc1 L_near_%=\n\t" \
  "s_sleep 64\n\t" \
  "L_near_%=:\n\t" \
  "s_sleep 2\n\t" \
  "s_branch L_poll_%=\n\t" \
  "L_dead_%=:\n\t" \
  "v_mov_b32 v254, s75\n\t" \
  "buffer_store_dword v254, off, s[88:91], 0 offset:128 sc1\n\t" \
  "s_mov_b32 s60, 0x7fc00000\n\t" \
  "s_mov_b32 s61, 0x7fc00000\n\t" \
  "s_mov_b32 s69, -1\n\t" \
  "s_waitcnt vmcnt(0)\n\t" \
  "L_ret_%=:\n\t" \
  "s_setpc_b64 s[76:77]\n\t" \
  "L_end_%=:\n\t" \
  "fused_text_end:\n\t"
#define COMM_ASM \
  "s_mov_b32 s72, %[first]\n\t" \
  "s_mov_b32 s23, 0\n\t" \
  "s_load_dwordx2 s[34:35], %[ka], 0x28\n\t" \
  "s_load_dwordx4 s[24:27], %[ka], 0x100\n\t" \
  "s_add_u32 s81, s72, 0\n\t" \
  "s_min_u32 s81, s81, 3359\n\t" \
  "s_add_u32 s81, s81, s23\n\t" \
  "s_lshl_b32 s81, s81, 2\n\t" \
  "s_load_dword s73, %[tab], s81\n\t" \
  "s_add_u32 s81, s72, 124\n\t" \
  "s_min_u32 s81, s81, 3359\n\t" \
  "s_add_u32 s81, s81, s23\n\t" \
  "s_lshl_b32 s81, s81, 2\n\t" \
  "s_load_dword s74, %[tab], s81\n\t" \
  "s_load_dwordx2 s[28:29], %[ka], 0x78\n\t" \
  "s_memrealtime s[90:91]\n\t" \
  "s_waitcnt lgkmcnt(0)\n\t" \
  "s_add_u32 s75, %[rot], 0\n\t" \
  "s_cmp_ge_u32 s75, 7\n\t" \
  "s_cselect_b32 s76, 7, 0\n\t" \
  "s_sub_u32 s75, s75, s76\n\t" \
  "s_lshl_b32 s75, s75, 3\n\t" \
  "s_add_u32 s75, s75, 64\n\t" \
  "s_load_dwordx2 s[36:37], %[ka], s75\n\t" \
  "s_add_u32 s75, %[rot], 1\n\t" \
  "s_cmp_ge_u32 s75, 7\n\t" \
  "s_cselect_b32 s76, 7, 0\n\t" \
  "s_sub_u32 s75, s75, s76\n\t" \
  "s_lshl_b32 s75, s75, 3\n\t" \
  "s_add_u32 s75, s75, 64\n\t" \
  "s_load_dwordx2 s[40:41], %[ka], s75\n\t" \
  "s_add_u32 s75, %[rot], 2\n\t" \
  "s_cmp_ge_u32 s75, 7\n\t" \
  "s_cselect_b32 s76, 7, 0\n\t" \
  "s_sub_u32 s75, s75, s76\n\t" \
  "s_lshl_b32 s75, s75, 3\n\t" \
  "s_add_u32 s75, s75, 64\n\t" \
  "s_load_dwordx2 s[44:45], %[ka], s75\n\t" \
  "s_add_u32 s75, %[rot], 3\n\t" \
  "s_cmp_ge_u32 s75, 7\n\t" \
  "s_cselect_b32 s76, 7, 0\n\t" \
  "s_sub_u32 s75, s75, s76\n\t" \
  "s_lshl_b32 s75, s75, 3\n\t" \
  "s_add_u32 s75, s75, 64\n\t" \
  "s_load_dwordx2 s[48:49], %[ka], s75\n\t" \
  "s_add_u32 s75, %[rot], 4\n\t" \
  "s_cmp_ge_u32 s75, 7\n\t" \
  "s_cselect_b32 s76, 7, 0\n\t" \
  "s_sub_u32 s75, s75, s76\n\t" \
  "s_lshl_b32 s75, s75, 3\n\t" \
  "s_add_u32 s75, s75, 64\n\t" \
  "s_load_dwordx2 s[52:53], %[ka], s75\n\t" \
  "s_add_u32 s75, %[rot], 5\n\t" \
  "s_cmp_ge_u32 s75, 7\n\t" \
  "s_cselect_b32 s76, 7, 0\n\t" \
  "s_sub_u32 s75, s75, s76\n\t" \
  "s_lshl_b32 s75, s75, 3\n\t" \
  "s_add_u32 s75, s75, 64\n\t" \
  "s_load_dwordx2 s[56:57], %[ka], s75\n\t" \
  "s_add_u32 s75, %[rot], 6\n\t" \
  "s_cmp_ge_u32 s75, 7\n\t" \
  "s_cselect_b32 s76, 7, 0\n\t" \
  "s_sub_u32 s75, s75, s76\n\t" \
  "s_lshl_b32 s75, s75, 3\n\t" \
  "s_add_u32 s75, s75, 64\n\t" \
  "s_load_dwordx2 s[60:61], %[ka], s75\n\t" \
  "v_mov_b32 v10, 0\n\t" \
  "global_load_dword v11, v10, s[34:35] sc1\n\t" \
  "v_mbcnt_lo_u32_b32 v0, -1, 0\n\t" \
  "v_mbcnt_hi_u32_b32 v0, -1, v0\n\t" \
  "v_lshlrev_b32 v8, 4, v0\n\t" \
  "s_getpc_b64 s[86:87]\n\t" \
  "L_tx_%=:\n\t" \
  "s_lshl_b32 s75, %[slice], 13\n\t" \
  "v_lshl_add_u32 v32, v0, 7, s75\n\t" \
  "v_min_u32 v32, fused_text_end-L_tx_%=-4, v32\n\t" \
  "global_load_dword v144, v32, s[86:87]\n\t" \
  "v_and_b32 v12, 7, v0\n\t" \
  "v_lshlrev_b32 v12, 11, v12\n\t" \
  "v_add_u32 v12, 8288, v12\n\t" \
  "s_and_b32 s24, s24, 7\n\t" \
  "s_lshl_b32 s33, s24, 11\n\t" \
  "s_mov_b32 s38, 0xd000000\n\t" \
  "s_mov_b32 s39, 0x20000\n\t" \
  "s_add_u32 s75, %[rot], 0\n\t" \
  "s_cmp_ge_u32 s75, 7\n\t" \
  "s_cselect_b32 s76, 7, 0\n\t" \
  "s_sub_u32 s75, s75, s76\n\t" \
  "s_add_u32 s75, s75, s24\n\t" \
  "s_add_u32 s75, s75, 1\n\t" \
  "s_and_b32 s75, s75, 7\n\t" \
  "s_lshl_b32 s64, s75, 11\n\t" \
  "s_mov_b32 s42, 0xd000000\n\t" \
  "s_mov_b32 s43, 0x20000\n\t" \
  "s_add_u32 s75, %[rot], 1\n\t" \
  "s_cmp_ge_u32 s75, 7\n\t" \
  "s_cselect_b32 s76, 7, 0\n\t" \
  "s_sub_u32 s75, s75, s76\n\t" \
  "s_add_u32 s75, s75, s24\n\t" \
  "s_add_u32 s75, s75, 1\n\t" \
  "s_and_b32 s75, s75, 7\n\t" \
  "s_lshl_b32 s65, s75, 11\n\t" \
  "s_mov_b32 s46, 0xd000000\n\t" \
  "s_mov_b32 s47, 0x20000\n\t" \
  "s_add_u32 s75, %[rot], 2\n\t" \
  "s_cmp_ge_u32 s75, 7\n\t" \
  "s_cselect_b32 s76, 7, 0\n\t" \
  "s_sub_u32 s75, s75, s76\n\t" \
  "s_add_u32 s75, s75, s24\n\t" \
  "s_add_u32 s75, s75, 1\n\t" \
  "s_and_b32 s75, s75, 7\n\t" \
  "s_lshl_b32 s66, s75, 11\n\t" \
  "s_mov_b32 s50, 0xd000000\n\t" \
  "s_mov_b32 s51, 0x20000\n\t" \
  "s_add_u32 s75, %[rot], 3\n\t" \
  "s_cmp_ge_u32 s75, 7\n\t" \
  "s_cselect_b32 s76, 7, 0\n\t" \
  "s_sub_u32 s75, s75, s76\n\t" \
  "s_add_u32 s75, s75, s24\n\t" \
  "s_add_u32 s75, s75, 1\n\t" \
  "s_and_b32 s75, s75, 7\n\t" \
  "s_lshl_b32 s67, s75, 11\n\t" \
  "s_mov_b32 s54, 0xd000000\n\t" \
  "s_mov_b32 s55, 0x20000\n\t" \
  "s_add_u32 s75, %[rot], 4\n\t" \
  "s_cmp_ge_u32 s75, 7\n\t" \
  "s_cselect_b32 s76, 7, 0\n\t" \
  "s_sub_u32 s75, s75, s76\n\t" \
  "s_add_u32 s75, s75, s24\n\t" \
  "s_add_u32 s75, s75, 1\n\t" \
  "s_and_b32 s75, s75, 7\n\t" \
  "s_lshl_b32 s68, s75, 11\n\t" \
  "s_mov_b32 s58, 0xd000000\n\t" \
  "s_mov_b32 s59, 0x20000\n\t" \
  "s_add_u32 s75, %[rot], 5\n\t" \
  "s_cmp_ge_u32 s75, 7\n\t" \
  "s_cselect_b32 s76, 7, 0\n\t" \
  "s_sub_u32 s75, s75, s76\n\t" \
  "s_add_u32 s75, s75, s24\n\t" \
  "s_add_u32 s75, s75, 1\n\t" \
  "s_and_b32 s75, s75, 7\n\t" \
  "s_lshl_b32 s69, s75, 11\n\t" \
  "s_mov_b32 s62, 0xd000000\n\t" \
  "s_mov_b32 s63, 0x20000\n\t" \
  "s_add_u32 s75, %[rot], 6\n\t" \
  "s_cmp_ge_u32 s75, 7\n\t" \
  "s_cselect_b32 s76, 7, 0\n\t" \
  "s_sub_u32 s75, s75, s76\n\t" \
  "s_add_u32 s75, s75, s24\n\t" \
  "s_add_u32 s75, s75, 1\n\t" \
  "s_and_b32 s75, s75, 7\n\t" \
  "s_lshl_b32 s70, s75, 11\n\t" \
  "s_mov_b32 s30, 0xd000000\n\t" \
  "s_mov_b32 s31, 0x20000\n\t" \
  "s_mov_b32 s80, 0\n\t" \
  "s_mov_b32 s82, 0\n\t" \
  "ds_write_b32 v10, v10\n\t" \
  "v_mov_b32 v13, 1\n\t" \
  "s_lshl_b32 s99, %[wave], 13\n\t" \
  "s_add_u32 s99, s99, 1024\n\t" \
  "v_lshrrev_b32 v14, 2, v8\n\t" \
  "v_add_u32 v15, s99, v8\n\t" \
  "v_lshrrev_b32 v24, 1, v8\n\t" \
  "v_mov_b32 v25, 0x1010101\n\t" \
  "v_mov_b32 v26, 0x2020202\n\t" \
  "v_mov_b32 v27, 0x4040404\n\t" \
  "v_mov_b32 v28, 0x8080808\n\t" \
  "v_mov_b32 v29, 0x10101010\n\t" \
  "v_mov_b32 v30, 0x20202020\n\t" \
  "v_mov_b32 v31, 0xbfbfbfbf\n\t" \
  "v_lshrrev_b32 v1, 4, v8\n\t" \
  "v_mad_u32_u24 v14, v1, 3, -6\n\t" \
  "v_lshlrev_b32 v224, 1, v1\n\t" \
  "v_add_u32 v224, 124, v224\n\t" \
  "v_cmp_gt_u32 vcc, 2, v1\n\t" \
  "v_lshlrev_b32 v2, 2, v1\n\t" \
  "v_add_u32 v2, 120, v2\n\t" \
  "v_cndmask_b32 v14, v14, v2, vcc\n\t" \
  "v_mov_b32 v2, 254\n\t" \
  "v_cndmask_b32 v224, v224, v2, vcc\n\t" \
  "v_cmp_lt_u32 vcc, 41, v1\n\t" \
  "v_mov_b32 v2, 0x7ffffff0\n\t" \
  "s_nop 0\n\t" \
  "v_cndmask_b32 v14, v14, v2, vcc\n\t" \
  "v_mul_lo_u32 v224, v224, v25\n\t" \
  "v_add_u32 v225, v224, v25\n\t" \
  "s_lshr_b32 s75, s99, 10\n\t" \
  "v_mov_b32 v226, s75\n\t" \
  "v_mov_b32 v32, 0\n\t" \
  "v_mov_b32 v33, 0\n\t" \
  "v_mov_b32 v34, 0\n\t" \
  "v_mov_b32 v35, 0\n\t" \
  "ds_write_b128 v15, v[32:35] offset:0\n\t" \
  "ds_write_b128 v15, v[32:35] offset:1024\n\t" \
  "ds_write_b128 v15, v[32:35] offset:2048\n\t" \
  "ds_write_b128 v15, v[32:35] offset:3072\n\t" \
  "s_mov_b32 s95, 0x9f9f9f9f\n\t" \
  "s_mov_b32 s96, 0x7f7f7f7f\n\t" \
  "s_mov_b32 s97, 0x03030303\n\t" \
  "s_mov_b32 s98, 0x0c0c0c0c\n\t" \
  "s_sub_i32 s71, %[wave], 0\n\t" \
  "s_waitcnt vmcnt(0) lgkmcnt(0)\n\t" \
  "s_barrier\n\t" \
  "v_readfirstlane_b32 s84, v11\n\t" \
  "s_nop 3\n\t" \
  "s_lshr_b32 s84, s84, 8\n\t" \
  "s_add_u32 s84, s84, 1\n\t" \
  "s_and_b32 s79, s84, 1\n\t" \
  "s_mul_i32 s79, s79, 0x6800000\n\t" \
  "s_lshl_b32 s78, s84, 12\n\t" \
  "s_add_u32 s75, s78, s72\n\t" \
  "v_mov_b32 v9, s75\n\t" \
  "v_mov_b32 v0, %[word]\n\t" \
  "s_mov_b64 exec, 1\n\t" \
  "global_store_dword v0, v9, s[34:35] sc0 sc1\n\t" \
  "s_mov_b64 exec, -1\n\t" \
  "s_mov_b32 s77, 0\n\t" \
  "L_fw0_%=:\n\t" \
  "global_load_dword v11, v12, s[34:35] sc0 sc1\n\t" \
  "s_waitcnt vmcnt(0)\n\t" \
  "s_memrealtime s[88:89]\n\t" \
  "v_cmp_gt_u32 vcc, s84, v11\n\t" \
  "s_nop 3\n\t" \
  "s_cmp_eq_u64 vcc, 0\n\t" \
  "s_cbranch_scc1 L_fd0_%=\n\t" \
  "s_add_u32 s77, s77, 1\n\t" \
  "s_and_b32 s75, s77, 63\n\t" \
  "s_cbranch_scc1 L_fw0_%=\n\t" \
  "global_load_dword v9, v10, s[34:35] offset:128 sc1\n\t" \
  "s_memrealtime s[86:87]\n\t" \
  "s_waitcnt vmcnt(0) lgkmcnt(0)\n\t" \
  "v_readfirstlane_b32 s76, v9\n\t" \
  "s_sub_u32 s86, s86, s90\n\t" \
  "s_cmp_gt_u32 s86, 50000000\n\t" \
  "s_cbranch_scc1 L_dead_%=\n\t" \
  "s_cmp_eq_u32 s76, s84\n\t" \
  "s_cbranch_scc1 L_dead_%=\n\t" \
  "s_branch L_fw0_%=\n\t" \
  "L_fd0_%=:\n\t" \
  "s_sub_i32 s75, s72, 24\n\t" \
  "s_lshl_b32 s75, s75, 2\n\t" \
  "v_add_u32 v12, s75, v12\n\t" \
  "s_mov_b32 s77, 0\n\t" \
  "L_fw1_%=:\n\t" \
  "global_load_dword v11, v12, s[34:35] sc0 sc1\n\t" \
  "s_waitcnt vmcnt(0)\n\t" \
  "v_cmp_gt_u32 vcc, s84, v11\n\t" \
  "s_nop 3\n\t" \
  "s_cmp_eq_u64 vcc, 0\n\t" \
  "s_cbranch_scc1 L_fd1_%=\n\t" \
  "s_add_u32 s77, s77, 1\n\t" \
  "s_and_b32 s75, s77, 63\n\t" \
  "s_cbranch_scc1 L_fw1_%=\n\t" \
  "global_load_dword v9, v10, s[34:35] offset:128 sc1\n\t" \
  "s_memrealtime s[86:87]\n\t" \
  "s_waitcnt vmcnt(0) lgkmcnt(0)\n\t" \
  "v_readfirstlane_b32 s76, v9\n\t" \
  "s_sub_u32 s86, s86, s90\n\t" \
  "s_cmp_gt_u32 s86, 50000000\n\t" \
  "s_cbranch_scc1 L_dead_%=\n\t" \
  "s_cmp_eq_u32 s76, s84\n\t" \
  "s_cbranch_scc1 L_dead_%=\n\t" \
  "s_branch L_fw1_%=\n\t" \
  "L_fd1_%=:\n\t" \
  "s_waitcnt lgkmcnt(0)\n\t" \
  "s_cmp_lt_u32 s72, 24\n\t" \
  "s_cbranch_scc1 L_skip_%=\n\t" \
  "L_loop_%=:\n\t" \
  "s_waitcnt lgkmcnt(0)\n\t" \
  "s_lshr_b32 s83, s73, 17\n\t" \
  "s_bfe_u32 s77, s73, 0x7000a\n\t" \
  "s_cmp_ge_u32 s77, 96\n\t" \
  "s_cbranch_scc1 L_po_%=\n\t" \
  "s_cmp_eq_u32 s77, 80\n\t" \
  "s_cselect_b32 s77, s25, s77\n\t" \
  "s_cmp_eq_u32 s77, 81\n\t" \
  "s_cselect_b32 s77, s26, s77\n\t" \
  "s_cmp_ge_u32 s77, 80\n\t" \
  "s_cbranch_scc1 L_skip_%=\n\t" \
  "s_mul_i32 s76, s77, 0x100800\n\t" \
  "s_and_b32 s75, s73, 0x3ff\n\t" \
  "s_lshl_b32 s75, s75, 10\n\t" \
  "s_add_u32 s76, s76, s75\n\t" \
  "s_add_u32 s76, s76, s79\n\t" \
  "v_add_u32 v16, s76, v8\n\t" \
  "s_cmp_ge_u32 s77, 16\n\t" \
  "s_mov_b32 s75, 32768\n\t" \
  "s_cselect_b32 s75, 16384, s75\n\t" \
  "v_add_u32 v17, s75, v16\n\t" \
  "s_mov_b32 s75, 65536\n\t" \
  "s_cselect_b32 s75, 1024, s75\n\t" \
  "v_add_u32 v18, s75, v16\n\t" \
  "s_mov_b32 s75, 98304\n\t" \
  "s_cselect_b32 s75, 17408, s75\n\t" \
  "v_add_u32 v19, s75, v16\n\t" \
  "s_cselect_b32 s92, 1, 2\n\t" \
  "s_movk_i32 s81, 3712\n\t" \
  "s_cselect_b32 s81, 3200, s81\n\t" \
  "s_mov_b32 s93, 0x5800000\n\t" \
  "s_cselect_b32 s93, 0x4f70000, s93\n\t" \
  "s_lshl_b32 s75, s77, 5\n\t" \
  "s_bfe_u32 s76, s73, 0x50005\n\t" \
  "s_add_u32 s75, s75, s76\n\t" \
  "s_bfe_u32 s76, s73, 0x10004\n\t" \
  "s_add_u32 s75, s75, s76\n\t" \
  "s_and_b32 s76, s73, 1\n\t" \
  "s_lshl1_add_u32 s75, s76, s75\n\t" \
  "s_mul_i32 s75, s75, s81\n\t" \
  "s_add_u32 s93, s93, s75\n\t" \
  "s_add_u32 s93, s93, s79\n\t" \
  "s_cmp_lt_u32 s72, 124\n\t" \
  "s_cselect_b32 s92, 0, s92\n\t" \
  "s_branch L_pace_%=\n\t" \
  "L_po_%=:\n\t" \
  "s_mov_b32 s92, 3\n\t" \
  "s_cmp_lg_u32 s82, 0\n\t" \
  "s_cbranch_scc1 L_om_%=\n\t" \
  "s_load_dwordx2 s[28:29], %[ka], 0x20\n\t" \
  "s_mov_b32 s30, 0x10000000\n\t" \
  "s_mov_b32 s82, 1\n\t" \
  "v_mov_b32 v25, 0x4020100\n\t" \
  "v_mov_b32 v26, 0x5020100\n\t" \
  "v_mov_b32 v27, 0x6020100\n\t" \
  "v_mov_b32 v28, 0xc0c0703\n\t" \
  "v_lshrrev_b32 v29, 8, v8\n\t" \
  "v_lshlrev_b32 v29, 18, v29\n\t" \
  "v_and_b32 v30, 0xf0, v8\n\t" \
  "v_or_b32 v29, v29, v30\n\t" \
  "s_waitcnt lgkmcnt(0)\n\t" \
  "L_ca1_%=:\n\t" \
  "v_add_u32 v0, 9728, v8\n\t" \
  "global_load_dwordx4 v[32:35], v0, s[34:35] sc0 sc1\n\t" \
  "v_add_u32 v0, 11776, v8\n\t" \
  "global_load_dwordx4 v[36:39], v0, s[34:35] sc0 sc1\n\t" \
  "v_add_u32 v0, 13824, v8\n\t" \
  "global_load_dwordx4 v[40:43], v0, s[34:35] sc0 sc1\n\t" \
  "v_add_u32 v0, 15872, v8\n\t" \
  "global_load_dwordx4 v[44:47], v0, s[34:35] sc0 sc1\n\t" \
  "v_add_u32 v0, 17920, v8\n\t" \
  "global_load_dwordx4 v[48:51], v0, s[34:35] sc0 sc1\n\t" \
  "v_add_u32 v0, 19968, v8\n\t" \
  "global_load_dwordx4 v[52:55], v0, s[34:35] sc0 sc1\n\t" \
  "v_add_u32 v0, 22016, v8\n\t" \
  "global_load_dwordx4 v[56:59], v0, s[34:35] sc0 sc1\n\t" \
  "v_add_u32 v0, 24064, v8\n\t" \
  "global_load_dwordx4 v[60:63], v0, s[34:35] sc0 sc1\n\t" \
  "s_waitcnt vmcnt(0)\n\t" \
  "v_mov_b32 v0, -1\n\t" \
  "s_cmp_eq_u32 s24, 0\n\t" \
  "s_cbranch_scc1 L_cs1_0_%=\n\t" \
  "v_min3_u32 v0, v0, v32, v33\n\t" \
  "v_min3_u32 v0, v0, v34, v35\n\t" \
  "L_cs1_0_%=:\n\t" \
  "s_cmp_eq_u32 s24, 1\n\t" \
  "s_cbranch_scc1 L_cs1_1_%=\n\t" \
  "v_min3_u32 v0, v0, v36, v37\n\t" \
  "v_min3_u32 v0, v0, v38, v39\n\t" \
  "L_cs1_1_%=:\n\t" \
  "s_cmp_eq_u32 s24, 2\n\t" \
  "s_cbranch_scc1 L_cs1_2_%=\n\t" \
  "v_min3_u32 v0, v0, v40, v41\n\t" \
  "v_min3_u32 v0, v0, v42, v43\n\t" \
  "L_cs1_2_%=:\n\t" \
  "s_cmp_eq_u32 s24, 3\n\t" \
  "s_cbranch_scc1 L_cs1_3_%=\n\t" \
  "v_min3_u32 v0, v0, v44, v45\n\t" \
  "v_min3_u32 v0, v0, v46, v47\n\t" \
  "L_cs1_3_%=:\n\t" \
  "s_cmp_eq_u32 s24, 4\n\t" \
  "s_cbranch_scc1 L_cs1_4_%=\n\t" \
  "v_min3_u32 v0, v0, v48, v49\n\t" \
  "v_min3_u32 v0, v0, v50, v51\n\t" \
  "L_cs1_4_%=:\n\t" \
  "s_cmp_eq_u32 s24, 5\n\t" \
  "s_cbranch_scc1 L_cs1_5_%=\n\t" \
  "v_min3_u32 v0, v0, v52, v53\n\t" \
  "v_min3_u32 v0, v0, v54, v55\n\t" \
  "L_cs1_5_%=:\n\t" \
  "s_cmp_eq_u32 s24, 6\n\t" \
  "s_cbranch_scc1 L_cs1_6_%=\n\t" \
  "v_min3_u32 v0, v0, v56, v57\n\t" \
  "v_min3_u32 v0, v0, v58, v59\n\t" \
  "L_cs1_6_%=:\n\t" \
  "s_cmp_eq_u32 s24, 7\n\t" \
  "s_cbranch_scc1 L_cs1_7_%=\n\t" \
  "v_min3_u32 v0, v0, v60, v61\n\t" \
  "v_min3_u32 v0, v0, v62, v63\n\t" \
  "L_cs1_7_%=:\n\t" \
  "v_cmp_gt_u32 vcc, s84, v0\n\t" \
  "s_nop 3\n\t" \
  "s_and_b32 vcc_lo, vcc_lo, 0xffff\n\t" \
  "s_and_b32 vcc_hi, vcc_hi, 0x0\n\t" \
  "s_cmp_eq_u64 vcc, 0\n\t" \
  "s_cbranch_scc1 L_cd1_%=\n\t" \
  "s_sleep 8\n\t" \
  "global_load_dword v9, v10, s[34:35] offset:128 sc1\n\t" \
  "s_memrealtime s[86:87]\n\t" \
  "s_waitcnt vmcnt(0) lgkmcnt(0)\n\t" \
  "v_readfirstlane_b32 s76, v9\n\t" \
  "s_sub_u32 s86, s86, s90\n\t" \
  "s_cmp_gt_u32 s86, 50000000\n\t" \
  "s_cbranch_scc1 L_dead_%=\n\t" \
  "s_cmp_eq_u32 s76, s84\n\t" \
  "s_cbranch_scc1 L_dead_%=\n\t" \
  "s_branch L_ca1_%=\n\t" \
  "L_cd1_%=:\n\t" \
  "L_om_%=:\n\t" \
  "s_bfe_u32 s77, s73, 0x5000a\n\t" \
  "s_lshl_b32 s75, s77, 6\n\t" \
  "s_and_b32 s76, s73, 63\n\t" \
  "s_add_u32 s75, s75, s76\n\t" \
  "s_mul_i32 s93, s75, 7680\n\t" \
  "s_add_u32 s93, s93, 0x5c00000\n\t" \
  "s_add_u32 s93, s93, s79\n\t" \
  "s_add_u32 s75, %[rot], 0\n\t" \
  "s_cmp_ge_u32 s75, 7\n\t" \
  "s_cselect_b32 s76, 7, 0\n\t" \
  "s_sub_u32 s75, s75, s76\n\t" \
  "s_add_u32 s75, s75, s24\n\t" \
  "s_add_u32 s75, s75, 1\n\t" \
  "s_and_b32 s75, s75, 7\n\t" \
  "s_mul_i32 s75, s75, 16\n\t" \
  "s_add_u32 s75, s75, s77\n\t" \
  "s_and_b32 s76, s75, 15\n\t" \
  "s_lshl_b32 s76, s76, 24\n\t" \
  "s_lshr_b32 s75, s75, 4\n\t" \
  "s_mul_i32 s75, s75, 8\n\t" \
  "s_add_u32 s75, s75, 3\n\t" \
  "s_lshl_b32 s75, s75, 10\n\t" \
  "s_add_u32 s64, s75, s76\n\t" \
  "s_add_u32 s75, %[rot], 1\n\t" \
  "s_cmp_ge_u32 s75, 7\n\t" \
  "s_cselect_b32 s76, 7, 0\n\t" \
  "s_sub_u32 s75, s75, s76\n\t" \
  "s_add_u32 s75, s75, s24\n\t" \
  "s_add_u32 s75, s75, 1\n\t" \
  "s_and_b32 s75, s75, 7\n\t" \
  "s_mul_i32 s75, s75, 16\n\t" \
  "s_add_u32 s75, s75, s77\n\t" \
  "s_and_b32 s76, s75, 15\n\t" \
  "s_lshl_b32 s76, s76, 24\n\t" \
  "s_lshr_b32 s75, s75, 4\n\t" \
  "s_mul_i32 s75, s75, 8\n\t" \
  "s_add_u32 s75, s75, 3\n\t" \
  "s_lshl_b32 s75, s75, 10\n\t" \
  "s_add_u32 s65, s75, s76\n\t" \
  "s_add_u32 s75, %[rot], 2\n\t" \
  "s_cmp_ge_u32 s75, 7\n\t" \
  "s_cselect_b32 s76, 7, 0\n\t" \
  "s_sub_u32 s75, s75, s76\n\t" \
  "s_add_u32 s75, s75, s24\n\t" \
  "s_add_u32 s75, s75, 1\n\t" \
  "s_and_b32 s75, s75, 7\n\t" \
  "s_mul_i32 s75, s75, 16\n\t" \
  "s_add_u32 s75, s75, s77\n\t" \
  "s_and_b32 s76, s75, 15\n\t" \
  "s_lshl_b32 s76, s76, 24\n\t" \
  "s_lshr_b32 s75, s75, 4\n\t" \
  "s_mul_i32 s75, s75, 8\n\t" \
  "s_add_u32 s75, s75, 3\n\t" \
  "s_lshl_b32 s75, s75, 10\n\t" \
  "s_add_u32 s66, s75, s76\n\t" \
  "s_add_u32 s75, %[rot], 3\n\t" \
  "s_cmp_ge_u32 s75, 7\n\t" \
  "s_cselect_b32 s76, 7, 0\n\t" \
  "s_sub_u32 s75, s75, s76\n\t" \
  "s_add_u32 s75, s75, s24\n\t" \
  "s_add_u32 s75, s75, 1\n\t" \
  "s_and_b32 s75, s75, 7\n\t" \
  "s_mul_i32 s75, s75, 16\n\t" \
  "s_add_u32 s75, s75, s77\n\t" \
  "s_and_b32 s76, s75, 15\n\t" \
  "s_lshl_b32 s76, s76, 24\n\t" \
  "s_lshr_b32 s75, s75, 4\n\t" \
  "s_mul_i32 s75, s75, 8\n\t" \
  "s_add_u32 s75, s75, 3\n\t" \
  "s_lshl_b32 s75, s75, 10\n\t" \
  "s_add_u32 s67, s75, s76\n\t" \
  "s_add_u32 s75, %[rot], 4\n\t" \
  "s_cmp_ge_u32 s75, 7\n\t" \
  "s_cselect_b32 s76, 7, 0\n\t" \
  "s_sub_u32 s75, s75, s76\n\t" \
  "s_add_u32 s75, s75, s24\n\t" \
  "s_add_u32 s75, s75, 1\n\t" \
  "s_and_b32 s75, s75, 7\n\t" \
  "s_mul_i32 s75, s75, 16\n\t" \
  "s_add_u32 s75, s75, s77\n\t" \
  "s_and_b32 s76, s75, 15\n\t" \
  "s_lshl_b32 s76, s76, 24\n\t" \
  "s_lshr_b32 s75, s75, 4\n\t" \
  "s_mul_i32 s75, s75, 8\n\t" \
  "s_add_u32 s75, s75, 3\n\t" \
  "s_lshl_b32 s75, s75, 10\n\t" \
  "s_add_u32 s68, s75, s76\n\t" \
  "s_add_u32 s75, %[rot], 5\n\t" \
  "s_cmp_ge_u32 s75, 7\n\t" \
  "s_cselect_b32 s76, 7, 0\n\t" \
  "s_sub_u32 s75, s75, s76\n\t" \
  "s_add_u32 s75, s75, s24\n\t" \
  "s_add_u32 s75, s75, 1\n\t" \
  "s_and_b32 s75, s75, 7\n\t" \
  "s_mul_i32 s75, s75, 16\n\t" \
  "s_add_u32 s75, s75, s77\n\t" \
  "s_and_b32 s76, s75, 15\n\t" \
  "s_lshl_b32 s76, s76, 24\n\t" \
  "s_lshr_b32 s75, s75, 4\n\t" \
  "s_mul_i32 s75, s75, 8\n\t" \
  "s_add_u32 s75, s75, 3\n\t" \
  "s_lshl_b32 s75, s75, 10\n\t" \
  "s_add_u32 s69, s75, s76\n\t" \
  "s_add_u32 s75, %[rot], 6\n\t" \
  "s_cmp_ge_u32 s75, 7\n\t" \
  "s_cselect_b32 s76, 7, 0\n\t" \
  "s_sub_u32 s75, s75, s76\n\t" \
  "s_add_u32 s75, s75, s24\n\t" \
  "s_add_u32 s75, s75, 1\n\t" \
  "s_and_b32 s75, s75, 7\n\t" \
  "s_mul_i32 s75, s75, 16\n\t" \
  "s_add_u32 s75, s75, s77\n\t" \
  "s_and_b32 s76, s75, 15\n\t" \
  "s_lshl_b32 s76, s76, 24\n\t" \
  "s_lshr_b32 s75, s75, 4\n\t" \
  "s_mul_i32 s75, s75, 8\n\t" \
  "s_add_u32 s75, s75, 3\n\t" \
  "s_lshl_b32 s75, s75, 10\n\t" \
  "s_add_u32 s70, s75, s76\n\t" \
  "s_bfe_u32 s75, s73, 0x40002\n\t" \
  "s_lshl_b32 s75, s75, 20\n\t" \
  "s_and_b32 s76, s73, 3\n\t" \
  "s_lshl_b32 s76, s76, 8\n\t" \
  "s_add_u32 s75, s75, s76\n\t" \
  "v_add_u32 v16, s75, v29\n\t" \
  "v_add_u32 v17, 0x10000, v16\n\t" \
  "v_add_u32 v18, 0x20000, v16\n\t" \
  "v_add_u32 v19, 0x30000, v16\n\t" \
  "L_pace_%=:\n\t" \
  "s_sub_u32 s75, s72, 1300\n\t" \
  "s_cmp_ge_u32 s75, 1036\n\t" \
  "s_cbranch_scc1 L_nt_%=\n\t" \
  "buffer_load_dword v0, v16, s[28:31], s33 offen sc1\n\t" \
  "buffer_load_dword v1, v17, s[28:31], s33 offen sc1\n\t" \
  "buffer_load_dword v2, v18, s[28:31], s33 offen sc1\n\t" \
  "buffer_load_dword v3, v19, s[28:31], s33 offen sc1\n\t" \
  "s_cmp_eq_u32 s92, 0\n\t" \
  "s_cbranch_scc1 L_nt_%=\n\t" \
  "buffer_load_dword v4, v8, s[28:31], s93 offen offset:0 sc1\n\t" \
  "buffer_load_dword v5, v8, s[28:31], s93 offen offset:1024 sc1\n\t" \
  "buffer_load_dword v6, v8, s[28:31], s93 offen offset:2048 sc1\n\t" \
  "buffer_load_dword v7, v24, s[28:31], s93 offen offset:3072 sc1\n\t" \
  "L_nt_%=:\n\t" \
  "s_add_u32 s81, s83, s27\n\t" \
  "L_pl_%=:\n\t" \
  "s_memrealtime s[86:87]\n\t" \
  "s_waitcnt lgkmcnt(0)\n\t" \
  "s_sub_u32 s86, s86, s88\n\t" \
  "s_add_u32 s75, s86, 30\n\t" \
  "s_cmp_ge_u32 s75, s81\n\t" \
  "s_cbranch_scc1 L_ps_%=\n\t" \
  "s_sleep 6\n\t" \
  "L_ps_%=:\n\t" \
  "s_cmp_lt_u32 s86, s81\n\t" \
  "s_cbranch_scc1 L_pl_%=\n\t" \
  "s_mov_b32 s77, 20000\n\t" \
  "L_tk_%=:\n\t" \
  "ds_read_b32 v11, v10\n\t" \
  "s_waitcnt lgkmcnt(0)\n\t" \
  "v_readfirstlane_b32 s75, v11\n\t" \
  "s_sub_u32 s77, s77, 1\n\t" \
  "s_cbranch_scc1 L_tg_%=\n\t" \
  "s_cmp_lt_i32 s75, s71\n\t" \
  "s_cbranch_scc1 L_tk_%=\n\t" \
  "L_tg_%=:\n\t" \
  "s_cmp_eq_u32 s92, 0\n\t" \
  "s_cbranch_scc1 L_raw_%=\n\t" \
  "s_cmp_eq_u32 s92, 2\n\t" \
  "s_cbranch_scc1 L_pA_%=\n\t" \
  "s_cmp_eq_u32 s92, 3\n\t" \
  "s_cbranch_scc1 L_pO_%=\n\t" \
  "buffer_load_dword v44, v14, s[36:39], s93 offen offset:3072 sc0 sc1\n\t" \
  "buffer_load_dword v60, v14, s[40:43], s93 offen offset:3072 sc0 sc1\n\t" \
  "buffer_load_dword v76, v14, s[44:47], s93 offen offset:3072 sc0 sc1\n\t" \
  "buffer_load_dword v92, v14, s[48:51], s93 offen offset:3072 sc0 sc1\n\t" \
  "buffer_load_dword v108, v14, s[52:55], s93 offen offset:3072 sc0 sc1\n\t" \
  "buffer_load_dword v124, v14, s[56:59], s93 offen offset:3072 sc0 sc1\n\t" \
  "buffer_load_dword v140, v14, s[60:63], s93 offen offset:3072 sc0 sc1\n\t" \
  "buffer_load_dwordx4 v[32:35], v8, s[36:39], s93 offen offset:0 sc0 sc1\n\t" \
  "buffer_load_dwordx4 v[48:51], v8, s[40:43], s93 offen offset:0 sc0 sc1\n\t" \
  "buffer_load_dwordx4 v[64:67], v8, s[44:47], s93 offen offset:0 sc0 sc1\n\t" \
  "buffer_load_dwordx4 v[80:83], v8, s[48:51], s93 offen offset:0 sc0 sc1\n\t" \
  "buffer_load_dwordx4 v[96:99], v8, s[52:55], s93 offen offset:0 sc0 sc1\n\t" \
  "buffer_load_dwordx4 v[112:115], v8, s[56:59], s93 offen offset:0 sc0 sc1\n\t" \
  "buffer_load_dwordx4 v[128:131], v8, s[60:63], s93 offen offset:0 sc0 sc1\n\t" \
  "buffer_load_dwordx4 v[36:39], v8, s[36:39], s93 offen offset:1024 sc0 sc1\n\t" \
  "buffer_load_dwordx4 v[52:55], v8, s[40:43], s93 offen offset:1024 sc0 sc1\n\t" \
  "buffer_load_dwordx4 v[68:71], v8, s[44:47], s93 offen offset:1024 sc0 sc1\n\t" \
  "buffer_load_dwordx4 v[84:87], v8, s[48:51], s93 offen offset:1024 sc0 sc1\n\t" \
  "buffer_load_dwordx4 v[100:103], v8, s[52:55], s93 offen offset:1024 sc0 sc1\n\t" \
  "buffer_load_dwordx4 v[116:119], v8, s[56:59], s93 offen offset:1024 sc0 sc1\n\t" \
  "buffer_load_dwordx4 v[132:135], v8, s[60:63], s93 offen offset:1024 sc0 sc1\n\t" \
  "s_waitcnt vmcnt(20)\n\t" \
  "buffer_load_dwordx4 v[40:43], v8, s[36:39], s93 offen offset:2048 sc0 sc1\n\t" \
  "s_waitcnt vmcnt(20)\n\t" \
  "buffer_load_dwordx4 v[56:59], v8, s[40:43], s93 offen offset:2048 sc0 sc1\n\t" \
  "s_waitcnt vmcnt(20)\n\t" \
  "buffer_load_dwordx4 v[72:75], v8, s[44:47], s93 offen offset:2048 sc0 sc1\n\t" \
  "s_waitcnt vmcnt(20)\n\t" \
  "buffer_load_dwordx4 v[88:91], v8, s[48:51], s93 offen offset:2048 sc0 sc1\n\t" \
  "s_waitcnt vmcnt(20)\n\t" \
  "buffer_load_dwordx4 v[104:107], v8, s[52:55], s93 offen offset:2048 sc0 sc1\n\t" \
  "s_waitcnt vmcnt(20)\n\t" \
  "buffer_load_dwordx4 v[120:123], v8, s[56:59], s93 offen offset:2048 sc0 sc1\n\t" \
  "s_waitcnt vmcnt(20)\n\t" \
  "buffer_load_dwordx4 v[136:139], v8, s[60:63], s93 offen offset:2048 sc0 sc1\n\t" \
  "s_waitcnt vmcnt(21)\n\t" \
  "v_readfirstlane_b32 s21, v44\n\t" \
  "v_readlane_b32 s75, v44, 1\n\t" \
  "v_lshrrev_b32 v210, 5, v44\n\t" \
  "v_lshrrev_b32 v211, 17, v44\n\t" \
  "v_and_b32 v210, 0x60, v210\n\t" \
  "v_and_b32 v211, 0x60, v211\n\t" \
  "v_xor_b32 v210, s21, v210\n\t" \
  "v_xor_b32 v211, s21, v211\n\t" \
  "v_subrev_u32 v208, s75, v224\n\t" \
  "v_subrev_u32 v209, s75, v225\n\t" \
  "v_and_b32 v208, 0x80808080, v208\n\t" \
  "v_and_b32 v209, 0x80808080, v209\n\t" \
  "v_bcnt_u32_b32 v208, v208, v226\n\t" \
  "v_bcnt_u32_b32 v209, v209, v226\n\t" \
  "v_and_b32 v176, 0x3ff, v44\n\t" \
  "v_bfe_u32 v177, v44, 12, 10\n\t" \
  "v_lshl_add_u32 v208, v208, 10, v176\n\t" \
  "v_lshl_add_u32 v209, v209, 10, v177\n\t" \
  "ds_write_b8 v208, v210\n\t" \
  "ds_write_b8 v209, v211\n\t" \
  "ds_read_b128 v[176:179], v15 offset:0\n\t" \
  "ds_read_b128 v[180:183], v15 offset:1024\n\t" \
  "ds_read_b128 v[184:187], v15 offset:2048\n\t" \
  "ds_read_b128 v[188:191], v15 offset:3072\n\t" \
  "ds_write_b8 v208, v10\n\t" \
  "ds_write_b8 v209, v10\n\t" \
  "s_waitcnt vmcnt(14)\n\t" \
  "s_mov_b64 exec, 1\n\t" \
  "ds_add_u32 v10, v13\n\t" \
  "s_mov_b64 exec, -1\n\t" \
  "v_readfirstlane_b32 s22, v60\n\t" \
  "v_readlane_b32 s75, v60, 1\n\t" \
  "v_lshrrev_b32 v214, 5, v60\n\t" \
  "v_lshrrev_b32 v215, 17, v60\n\t" \
  "v_and_b32 v214, 0x60, v214\n\t" \
  "v_and_b32 v215, 0x60, v215\n\t" \
  "v_xor_b32 v214, s22, v214\n\t" \
  "v_xor_b32 v215, s22, v215\n\t" \
  "v_subrev_u32 v212, s75, v224\n\t" \
  "v_subrev_u32 v213, s75, v225\n\t" \
  "v_and_b32 v212, 0x80808080, v212\n\t" \
  "v_and_b32 v213, 0x80808080, v213\n\t" \
  "v_bcnt_u32_b32 v212, v212, v226\n\t" \
  "v_bcnt_u32_b32 v213, v213, v226\n\t" \
  "v_and_b32 v192, 0x3ff, v60\n\t" \
  "v_bfe_u32 v193, v60, 12, 10\n\t" \
  "v_lshl_add_u32 v212, v212, 10, v192\n\t" \
  "v_lshl_add_u32 v213, v213, 10, v193\n\t" \
  "ds_write_b8 v212, v214\n\t" \
  "ds_write_b8 v213, v215\n\t" \
  "ds_read_b128 v[192:195], v15 offset:0\n\t" \
  "ds_read_b128 v[196:199], v15 offset:1024\n\t" \
  "ds_read_b128 v[200:203], v15 offset:2048\n\t" \
  "ds_read_b128 v[204:207], v15 offset:3072\n\t" \
  "ds_write_b8 v212, v10\n\t" \
  "ds_write_b8 v213, v10\n\t" \
  "s_bitcmp1_b32 s21, 7\n\t" \
  "s_cbranch_scc1 L_fbB0_%=\n\t" \
  "s_waitcnt vmcnt(6) lgkmcnt(8)\n\t" \
  "v_lshrrev_b32 v220, 5, v32\n\t" \
  "v_lshrrev_b32 v221, 3, v33\n\t" \
  "v_lshrrev_b32 v222, 1, v34\n\t" \
  "v_lshlrev_b32 v223, 1, v34\n\t" \
  "v_and_b32 v216, s95, v32\n\t" \
  "v_and_b32 v217, s95, v33\n\t" \
  "v_and_b32 v218, s95, v34\n\t" \
  "v_and_b32 v219, s97, v220\n\t" \
  "v_and_b32 v223, 0x80808080, v223\n\t" \
  "v_and_or_b32 v219, v221, s98, v219\n\t" \
  "v_and_or_b32 v223, v222, v29, v223\n\t" \
  "v_xad_u32 v144, v176, s21, v216\n\t" \
  "v_xad_u32 v145, v177, s21, v217\n\t" \
  "v_xad_u32 v146, v178, s21, v218\n\t" \
  "v_xad_u32 v147, v179, s21, v219\n\t" \
  "v_add_u32 v147, v147, v223\n\t" \
  "v_lshrrev_b32 v220, 5, v35\n\t" \
  "v_lshrrev_b32 v221, 3, v36\n\t" \
  "v_lshrrev_b32 v222, 1, v37\n\t" \
  "v_lshlrev_b32 v223, 1, v37\n\t" \
  "v_and_b32 v216, s95, v35\n\t" \
  "v_and_b32 v217, s95, v36\n\t" \
  "v_and_b32 v218, s95, v37\n\t" \
  "v_and_b32 v219, s97, v220\n\t" \
  "v_and_b32 v223, 0x80808080, v223\n\t" \
  "v_and_or_b32 v219, v221, s98, v219\n\t" \
  "v_and_or_b32 v223, v222, v29, v223\n\t" \
  "v_xad_u32 v148, v180, s21, v216\n\t" \
  "v_xad_u32 v149, v181, s21, v217\n\t" \
  "v_xad_u32 v150, v182, s21, v218\n\t" \
  "v_xad_u32 v151, v183, s21, v219\n\t" \
  "v_add_u32 v151, v151, v223\n\t" \
  "v_lshrrev_b32 v220, 5, v38\n\t" \
  "v_lshrrev_b32 v221, 3, v39\n\t" \
  "v_lshrrev_b32 v222, 1, v40\n\t" \
  "v_lshlrev_b32 v223, 1, v40\n\t" \
  "v_and_b32 v216, s95, v38\n\t" \
  "v_and_b32 v217, s95, v39\n\t" \
  "v_and_b32 v218, s95, v40\n\t" \
  "v_and_b32 v219, s97, v220\n\t" \
  "v_and_b32 v223, 0x80808080, v223\n\t" \
  "v_and_or_b32 v219, v221, s98, v219\n\t" \
  "v_and_or_b32 v223, v222, v29, v223\n\t" \
  "v_xad_u32 v152, v184, s21, v216\n\t" \
  "v_xad_u32 v153, v185, s21, v217\n\t" \
  "v_xad_u32 v154, v186, s21, v218\n\t" \
  "v_xad_u32 v155, v187, s21, v219\n\t" \
  "v_add_u32 v155, v155, v223\n\t" \
  "v_lshrrev_b32 v220, 5, v41\n\t" \
  "v_lshrrev_b32 v221, 3, v42\n\t" \
  "v_lshrrev_b32 v222, 1, v43\n\t" \
  "v_lshlrev_b32 v223, 1, v43\n\t" \
  "v_and_b32 v216, s95, v41\n\t" \
  "v_and_b32 v217, s95, v42\n\t" \
  "v_and_b32 v218, s95, v43\n\t" \
  "v_and_b32 v219, s97, v220\n\t" \
  "v_and_b32 v223, 0x80808080, v223\n\t" \
  "v_and_or_b32 v219, v221, s98, v219\n\t" \
  "v_and_or_b32 v223, v222, v29, v223\n\t" \
  "v_xad_u32 v156, v188, s21, v216\n\t" \
  "v_xad_u32 v157, v189, s21, v217\n\t" \
  "v_xad_u32 v158, v190, s21, v218\n\t" \
  "v_xad_u32 v159, v191, s21, v219\n\t" \
  "v_add_u32 v159, v159, v223\n\t" \
  "L_stB0_%=:\n\t" \
  "buffer_store_dwordx4 v[144:147], v16, s[28:31], s64 offen sc0 sc1\n\t" \
  "buffer_store_dwordx4 v[148:151], v17, s[28:31], s64 offen sc0 sc1\n\t" \
  "buffer_store_dwordx4 v[152:155], v18, s[28:31], s64 offen sc0 sc1\n\t" \
  "buffer_store_dwordx4 v[156:159], v19, s[28:31], s64 offen sc0 sc1\n\t" \
  "v_readfirstlane_b32 s21, v76\n\t" \
  "v_readlane_b32 s75, v76, 1\n\t" \
  "v_lshrrev_b32 v210, 5, v76\n\t" \
  "v_lshrrev_b32 v211, 17, v76\n\t" \
  "v_and_b32 v210, 0x60, v210\n\t" \
  "v_and_b32 v211, 0x60, v211\n\t" \
  "v_xor_b32 v210, s21, v210\n\t" \
  "v_xor_b32 v211, s21, v211\n\t" \
  "v_subrev_u32 v208, s75, v224\n\t" \
  "v_subrev_u32 v209, s75, v225\n\t" \
  "v_and_b32 v208, 0x80808080, v208\n\t" \
  "v_and_b32 v209, 0x80808080, v209\n\t" \
  "v_bcnt_u32_b32 v208, v208, v226\n\t" \
  "v_bcnt_u32_b32 v209, v209, v226\n\t" \
  "v_and_b32 v176, 0x3ff, v76\n\t" \
  "v_bfe_u32 v177, v76, 12, 10\n\t" \
  "v_lshl_add_u32 v208, v208, 10, v176\n\t" \
  "v_lshl_add_u32 v209, v209, 10, v177\n\t" \
  "ds_write_b8 v208, v210\n\t" \
  "ds_write_b8 v209, v211\n\t" \
  "ds_read_b128 v[176:179], v15 offset:0\n\t" \
  "ds_read_b128 v[180:183], v15 offset:1024\n\t" \
  "ds_read_b128 v[184:187], v15 offset:2048\n\t" \
  "ds_read_b128 v[188:191], v15 offset:3072\n\t" \
  "ds_write_b8 v208, v10\n\t" \
  "ds_write_b8 v209, v10\n\t" \
  "s_bitcmp1_b32 s22, 7\n\t" \
  "s_cbranch_scc1 L_fbB1_%=\n\t" \
  "s_waitcnt vmcnt(9) lgkmcnt(8)\n\t" \
  "v_lshrrev_b32 v220, 5, v48\n\t" \
  "v_lshrrev_b32 v221, 3, v49\n\t" \
  "v_lshrrev_b32 v222, 1, v50\n\t" \
  "v_lshlrev_b32 v223, 1, v50\n\t" \
  "v_and_b32 v216, s95, v48\n\t" \
  "v_and_b32 v217, s95, v49\n\t" \
  "v_and_b32 v218, s95, v50\n\t" \
  "v_and_b32 v219, s97, v220\n\t" \
  "v_and_b32 v223, 0x80808080, v223\n\t" \
  "v_and_or_b32 v219, v221, s98, v219\n\t" \
  "v_and_or_b32 v223, v222, v29, v223\n\t" \
  "v_xad_u32 v160, v192, s22, v216\n\t" \
  "v_xad_u32 v161, v193, s22, v217\n\t" \
  "v_xad_u32 v162, v194, s22, v218\n\t" \
  "v_xad_u32 v163, v195, s22, v219\n\t" \
  "v_add_u32 v163, v163, v223\n\t" \
  "v_lshrrev_b32 v220, 5, v51\n\t" \
  "v_lshrrev_b32 v221, 3, v52\n\t" \
  "v_lshrrev_b32 v222, 1, v53\n\t" \
  "v_lshlrev_b32 v223, 1, v53\n\t" \
  "v_and_b32 v216, s95, v51\n\t" \
  "v_and_b32 v217, s95, v52\n\t" \
  "v_and_b32 v218, s95, v53\n\t" \
  "v_and_b32 v219, s97, v220\n\t" \
  "v_and_b32 v223, 0x80808080, v223\n\t" \
  "v_and_or_b32 v219, v221, s98, v219\n\t" \
  "v_and_or_b32 v223, v222, v29, v223\n\t" \
  "v_xad_u32 v164, v196, s22, v216\n\t" \
  "v_xad_u32 v165, v197, s22, v217\n\t" \
  "v_xad_u32 v166, v198, s22, v218\n\t" \
  "v_xad_u32 v167, v199, s22, v219\n\t" \
  "v_add_u32 v167, v167, v223\n\t" \
  "v_lshrrev_b32 v220, 5, v54\n\t" \
  "v_lshrrev_b32 v221, 3, v55\n\t" \
  "v_lshrrev_b32 v222, 1, v56\n\t" \
  "v_lshlrev_b32 v223, 1, v56\n\t" \
  "v_and_b32 v216, s95, v54\n\t" \
  "v_and_b32 v217, s95, v55\n\t" \
  "v_and_b32 v218, s95, v56\n\t" \
  "v_and_b32 v219, s97, v220\n\t" \
  "v_and_b32 v223, 0x80808080, v223\n\t" \
  "v_and_or_b32 v219, v221, s98, v219\n\t" \
  "v_and_or_b32 v223, v222, v29, v223\n\t" \
  "v_xad_u32 v168, v200, s22, v216\n\t" \
  "v_xad_u32 v169, v201, s22, v217\n\t" \
  "v_xad_u32 v170, v202, s22, v218\n\t" \
  "v_xad_u32 v171, v203, s22, v219\n\t" \
  "v_add_u32 v171, v171, v223\n\t" \
  "v_lshrrev_b32 v220, 5, v57\n\t" \
  "v_lshrrev_b32 v221, 3, v58\n\t" \
  "v_lshrrev_b32 v222, 1, v59\n\t" \
  "v_lshlrev_b32 v223, 1, v59\n\t" \
  "v_and_b32 v216, s95, v57\n\t" \
  "v_and_b32 v217, s95, v58\n\t" \
  "v_and_b32 v218, s95, v59\n\t" \
  "v_and_b32 v219, s97, v220\n\t" \
  "v_and_b32 v223, 0x80808080, v223\n\t" \
  "v_and_or_b32 v219, v221, s98, v219\n\t" \
  "v_and_or_b32 v223, v222, v29, v223\n\t" \
  "v_xad_u32 v172, v204, s22, v216\n\t" \
  "v_xad_u32 v173, v205, s22, v217\n\t" \
  "v_xad_u32 v174, v206, s22, v218\n\t" \
  "v_xad_u32 v175, v207, s22, v219\n\t" \
  "v_add_u32 v175, v175, v223\n\t" \
  "L_stB1_%=:\n\t" \
  "buffer_store_dwordx4 v[160:163], v16, s[28:31], s65 offen sc0 sc1\n\t" \
  "buffer_store_dwordx4 v[164:167], v17, s[28:31], s65 offen sc0 sc1\n\t" \
  "buffer_store_dwordx4 v[168:171], v18, s[28:31], s65 offen sc0 sc1\n\t" \
  "buffer_store_dwordx4 v[172:175], v19, s[28:31], s65 offen sc0 sc1\n\t" \
  "v_readfirstlane_b32 s22, v92\n\t" \
  "v_readlane_b32 s75, v92, 1\n\t" \
  "v_lshrrev_b32 v214, 5, v92\n\t" \
  "v_lshrrev_b32 v215, 17, v92\n\t" \
  "v_and_b32 v214, 0x60, v214\n\t" \
  "v_and_b32 v215, 0x60, v215\n\t" \
  "v_xor_b32 v214, s22, v214\n\t" \
  "v_xor_b32 v215, s22, v215\n\t" \
  "v_subrev_u32 v212, s75, v224\n\t" \
  "v_subrev_u32 v213, s75, v225\n\t" \
  "v_and_b32 v212, 0x80808080, v212\n\t" \
  "v_and_b32 v213, 0x80808080, v213\n\t" \
  "v_bcnt_u32_b32 v212, v212, v226\n\t" \
  "v_bcnt_u32_b32 v213, v213, v226\n\t" \
  "v_and_b32 v192, 0x3ff, v92\n\t" \
  "v_bfe_u32 v193, v92, 12, 10\n\t" \
  "v_lshl_add_u32 v212, v212, 10, v192\n\t" \
  "v_lshl_add_u32 v213, v213, 10, v193\n\t" \
  "ds_write_b8 v212, v214\n\t" \
  "ds_write_b8 v213, v215\n\t" \
  "ds_read_b128 v[192:195], v15 offset:0\n\t" \
  "ds_read_b128 v[196:199], v15 offset:1024\n\t" \
  "ds_read_b128 v[200:203], v15 offset:2048\n\t" \
  "ds_read_b128 v[204:207], v15 offset:3072\n\t" \
  "ds_write_b8 v212, v10\n\t" \
  "ds_write_b8 v213, v10\n\t" \
  "s_bitcmp1_b32 s21, 7\n\t" \
  "s_cbranch_scc1 L_fbB2_%=\n\t" \
  "s_waitcnt vmcnt(12) lgkmcnt(8)\n\t" \
  "v_lshrrev_b32 v220, 5, v64\n\t" \
  "v_lshrrev_b32 v221, 3, v65\n\t" \
  "v_lshrrev_b32 v222, 1, v66\n\t" \
  "v_lshlrev_b32 v223, 1, v66\n\t" \
  "v_and_b32 v216, s95, v64\n\t" \
  "v_and_b32 v217, s95, v65\n\t" \
  "v_and_b32 v218, s95, v66\n\t" \
  "v_and_b32 v219, s97, v220\n\t" \
  "v_and_b32 v223, 0x80808080, v223\n\t" \
  "v_and_or_b32 v219, v221, s98, v219\n\t" \
  "v_and_or_b32 v223, v222, v29, v223\n\t" \
  "v_xad_u32 v144, v176, s21, v216\n\t" \
  "v_xad_u32 v145, v177, s21, v217\n\t" \
  "v_xad_u32 v146, v178, s21, v218\n\t" \
  "v_xad_u32 v147, v179, s21, v219\n\t" \
  "v_add_u32 v147, v147, v223\n\t" \
  "v_lshrrev_b32 v220, 5, v67\n\t" \
  "v_lshrrev_b32 v221, 3, v68\n\t" \
  "v_lshrrev_b32 v222, 1, v69\n\t" \
  "v_lshlrev_b32 v223, 1, v69\n\t" \
  "v_and_b32 v216, s95, v67\n\t" \
  "v_and_b32 v217, s95, v68\n\t" \
  "v_and_b32 v218, s95, v69\n\t" \
  "v_and_b32 v219, s97, v220\n\t" \
  "v_and_b32 v223, 0x80808080, v223\n\t" \
  "v_and_or_b32 v219, v221, s98, v219\n\t" \
  "v_and_or_b32 v223, v222, v29, v223\n\t" \
  "v_xad_u32 v148, v180, s21, v216\n\t" \
  "v_xad_u32 v149, v181, s21, v217\n\t" \
  "v_xad_u32 v150, v182, s21, v218\n\t" \
  "v_xad_u32 v151, v183, s21, v219\n\t" \
  "v_add_u32 v151, v151, v223\n\t" \
  "v_lshrrev_b32 v220, 5, v70\n\t" \
  "v_lshrrev_b32 v221, 3, v71\n\t" \
  "v_lshrrev_b32 v222, 1, v72\n\t" \
  "v_lshlrev_b32 v223, 1, v72\n\t" \
  "v_and_b32 v216, s95, v70\n\t" \
  "v_and_b32 v217, s95, v71\n\t" \
  "v_and_b32 v218, s95, v72\n\t" \
  "v_and_b32 v219, s97, v220\n\t" \
  "v_and_b32 v223, 0x80808080, v223\n\t" \
  "v_and_or_b32 v219, v221, s98, v219\n\t" \
  "v_and_or_b32 v223, v222, v29, v223\n\t" \
  "v_xad_u32 v152, v184, s21, v216\n\t" \
  "v_xad_u32 v153, v185, s21, v217\n\t" \
  "v_xad_u32 v154, v186, s21, v218\n\t" \
  "v_xad_u32 v155, v187, s21, v219\n\t" \
  "v_add_u32 v155, v155, v223\n\t" \
  "v_lshrrev_b32 v220, 5, v73\n\t" \
  "v_lshrrev_b32 v221, 3, v74\n\t" \
  "v_lshrrev_b32 v222, 1, v75\n\t" \
  "v_lshlrev_b32 v223, 1, v75\n\t" \
  "v_and_b32 v216, s95, v73\n\t" \
  "v_and_b32 v217, s95, v74\n\t" \
  "v_and_b32 v218, s95, v75\n\t" \
  "v_and_b32 v219, s97, v220\n\t" \
  "v_and_b32 v223, 0x80808080, v223\n\t" \
  "v_and_or_b32 v219, v221, s98, v219\n\t" \
  "v_and_or_b32 v223, v222, v29, v223\n\t" \
  "v_xad_u32 v156, v188, s21, v216\n\t" \
  "v_xad_u32 v157, v189, s21, v217\n\t" \
  "v_xad_u32 v158, v190, s21, v218\n\t" \
  "v_xad_u32 v159, v191, s21, v219\n\t" \
  "v_add_u32 v159, v159, v223\n\t" \
  "L_stB2_%=:\n\t" \
  "buffer_store_dwordx4 v[144:147], v16, s[28:31], s66 offen sc0 sc1\n\t" \
  "buffer_store_dwordx4 v[148:151], v17, s[28:31], s66 offen sc0 sc1\n\t" \
  "buffer_store_dwordx4 v[152:155], v18, s[28:31], s66 offen sc0 sc1\n\t" \
  "buffer_store_dwordx4 v[156:159], v19, s[28:31], s66 offen sc0 sc1\n\t" \
  "v_readfirstlane_b32 s21, v108\n\t" \
  "v_readlane_b32 s75, v108, 1\n\t" \
  "v_lshrrev_b32 v210, 5, v108\n\t" \
  "v_lshrrev_b32 v211, 17, v108\n\t" \
  "v_and_b32 v210, 0x60, v210\n\t" \
  "v_and_b32 v211, 0x60, v211\n\t" \
  "v_xor_b32 v210, s21, v210\n\t" \
  "v_xor_b32 v211, s21, v211\n\t" \
  "v_subrev_u32 v208, s75, v224\n\t" \
  "v_subrev_u32 v209, s75, v225\n\t" \
  "v_and_b32 v208, 0x80808080, v208\n\t" \
  "v_and_b32 v209, 0x80808080, v209\n\t" \
  "v_bcnt_u32_b32 v208, v208, v226\n\t" \
  "v_bcnt_u32_b32 v209, v209, v226\n\t" \
  "v_and_b32 v176, 0x3ff, v108\n\t" \
  "v_bfe_u32 v177, v108, 12, 10\n\t" \
  "v_lshl_add_u32 v208, v208, 10, v176\n\t" \
  "v_lshl_add_u32 v209, v209, 10, v177\n\t" \
  "ds_write_b8 v208, v210\n\t" \
  "ds_write_b8 v209, v211\n\t" \
  "ds_read_b128 v[176:179], v15 offset:0\n\t" \
  "ds_read_b128 v[180:183], v15 offset:1024\n\t" \
  "ds_read_b128 v[184:187], v15 offset:2048\n\t" \
  "ds_read_b128 v[188:191], v15 offset:3072\n\t" \
  "ds_write_b8 v208, v10\n\t" \
  "ds_write_b8 v209, v10\n\t" \
  "s_bitcmp1_b32 s22, 7\n\t" \
  "s_cbranch_scc1 L_fbB3_%=\n\t" \
  "s_waitcnt vmcnt(15) lgkmcnt(8)\n\t" \
  "v_lshrrev_b32 v220, 5, v80\n\t" \
  "v_lshrrev_b32 v221, 3, v81\n\t" \
  "v_lshrrev_b32 v222, 1, v82\n\t" \
  "v_lshlrev_b32 v223, 1, v82\n\t" \
  "v_and_b32 v216, s95, v80\n\t" \
  "v_and_b32 v217, s95, v81\n\t" \
  "v_and_b32 v218, s95, v82\n\t" \
  "v_and_b32 v219, s97, v220\n\t" \
  "v_and_b32 v223, 0x80808080, v223\n\t" \
  "v_and_or_b32 v219, v221, s98, v219\n\t" \
  "v_and_or_b32 v223, v222, v29, v223\n\t" \
  "v_xad_u32 v160, v192, s22, v216\n\t" \
  "v_xad_u32 v161, v193, s22, v217\n\t" \
  "v_xad_u32 v162, v194, s22, v218\n\t" \
  "v_xad_u32 v163, v195, s22, v219\n\t" \
  "v_add_u32 v163, v163, v223\n\t" \
  "v_lshrrev_b32 v220, 5, v83\n\t" \
  "v_lshrrev_b32 v221, 3, v84\n\t" \
  "v_lshrrev_b32 v222, 1, v85\n\t" \
  "v_lshlrev_b32 v223, 1, v85\n\t" \
  "v_and_b32 v216, s95, v83\n\t" \
  "v_and_b32 v217, s95, v84\n\t" \
  "v_and_b32 v218, s95, v85\n\t" \
  "v_and_b32 v219, s97, v220\n\t" \
  "v_and_b32 v223, 0x80808080, v223\n\t" \
  "v_and_or_b32 v219, v221, s98, v219\n\t" \
  "v_and_or_b32 v223, v222, v29, v223\n\t" \
  "v_xad_u32 v164, v196, s22, v216\n\t" \
  "v_xad_u32 v165, v197, s22, v217\n\t" \
  "v_xad_u32 v166, v198, s22, v218\n\t" \
  "v_xad_u32 v167, v199, s22, v219\n\t" \
  "v_add_u32 v167, v167, v223\n\t" \
  "v_lshrrev_b32 v220, 5, v86\n\t" \
  "v_lshrrev_b32 v221, 3, v87\n\t" \
  "v_lshrrev_b32 v222, 1, v88\n\t" \
  "v_lshlrev_b32 v223, 1, v88\n\t" \
  "v_and_b32 v216, s95, v86\n\t" \
  "v_and_b32 v217, s95, v87\n\t" \
  "v_and_b32 v218, s95, v88\n\t" \
  "v_and_b32 v219, s97, v220\n\t" \
  "v_and_b32 v223, 0x80808080, v223\n\t" \
  "v_and_or_b32 v219, v221, s98, v219\n\t" \
  "v_and_or_b32 v223, v222, v29, v223\n\t" \
  "v_xad_u32 v168, v200, s22, v216\n\t" \
  "v_xad_u32 v169, v201, s22, v217\n\t" \
  "v_xad_u32 v170, v202, s22, v218\n\t" \
  "v_xad_u32 v171, v203, s22, v219\n\t" \
  "v_add_u32 v171, v171, v223\n\t" \
  "v_lshrrev_b32 v220, 5, v89\n\t" \
  "v_lshrrev_b32 v221, 3, v90\n\t" \
  "v_lshrrev_b32 v222, 1, v91\n\t" \
  "v_lshlrev_b32 v223, 1, v91\n\t" \
  "v_and_b32 v216, s95, v89\n\t" \
  "v_and_b32 v217, s95, v90\n\t" \
  "v_and_b32 v218, s95, v91\n\t" \
  "v_and_b32 v219, s97, v220\n\t" \
  "v_and_b32 v223, 0x80808080, v223\n\t" \
  "v_and_or_b32 v219, v221, s98, v219\n\t" \
  "v_and_or_b32 v223, v222, v29, v223\n\t" \
  "v_xad_u32 v172, v204, s22, v216\n\t" \
  "v_xad_u32 v173, v205, s22, v217\n\t" \
  "v_xad_u32 v174, v206, s22, v218\n\t" \
  "v_xad_u32 v175, v207, s22, v219\n\t" \
  "v_add_u32 v175, v175, v223\n\t" \
  "L_stB3_%=:\n\t" \
  "buffer_store_dwordx4 v[160:163], v16, s[28:31], s67 offen sc0 sc1\n\t" \
  "buffer_store_dwordx4 v[164:167], v17, s[28:31], s67 offen sc0 sc1\n\t" \
  "buffer_store_dwordx4 v[168:171], v18, s[28:31], s67 offen sc0 sc1\n\t" \
  "buffer_store_dwordx4 v[172:175], v19, s[28:31], s67 offen sc0 sc1\n\t" \
  "v_readfirstlane_b32 s22, v124\n\t" \
  "v_readlane_b32 s75, v124, 1\n\t" \
  "v_lshrrev_b32 v214, 5, v124\n\t" \
  "v_lshrrev_b32 v215, 17, v124\n\t" \
  "v_and_b32 v214, 0x60, v214\n\t" \
  "v_and_b32 v215, 0x60, v215\n\t" \
  "v_xor_b32 v214, s22, v214\n\t" \
  "v_xor_b32 v215, s22, v215\n\t" \
  "v_subrev_u32 v212, s75, v224\n\t" \
  "v_subrev_u32 v213, s75, v225\n\t" \
  "v_and_b32 v212, 0x80808080, v212\n\t" \
  "v_and_b32 v213, 0x80808080, v213\n\t" \
  "v_bcnt_u32_b32 v212, v212, v226\n\t" \
  "v_bcnt_u32_b32 v213, v213, v226\n\t" \
  "v_and_b32 v192, 0x3ff, v124\n\t" \
  "v_bfe_u32 v193, v124, 12, 10\n\t" \
  "v_lshl_add_u32 v212, v212, 10, v192\n\t" \
  "v_lshl_add_u32 v213, v213, 10, v193\n\t" \
  "ds_write_b8 v212, v214\n\t" \
  "ds_write_b8 v213, v215\n\t" \
  "ds_read_b128 v[192:195], v15 offset:0\n\t" \
  "ds_read_b128 v[196:199], v15 offset:1024\n\t" \
  "ds_read_b128 v[200:203], v15 offset:2048\n\t" \
  "ds_read_b128 v[204:207], v15 offset:3072\n\t" \
  "ds_write_b8 v212, v10\n\t" \
  "ds_write_b8 v213, v10\n\t" \
  "s_bitcmp1_b32 s21, 7\n\t" \
  "s_cbranch_scc1 L_fbB4_%=\n\t" \
  "s_waitcnt vmcnt(18) lgkmcnt(8)\n\t" \
  "v_lshrrev_b32 v220, 5, v96\n\t" \
  "v_lshrrev_b32 v221, 3, v97\n\t" \
  "v_lshrrev_b32 v222, 1, v98\n\t" \
  "v_lshlrev_b32 v223, 1, v98\n\t" \
  "v_and_b32 v216, s95, v96\n\t" \
  "v_and_b32 v217, s95, v97\n\t" \
  "v_and_b32 v218, s95, v98\n\t" \
  "v_and_b32 v219, s97, v220\n\t" \
  "v_and_b32 v223, 0x80808080, v223\n\t" \
  "v_and_or_b32 v219, v221, s98, v219\n\t" \
  "v_and_or_b32 v223, v222, v29, v223\n\t" \
  "v_xad_u32 v144, v176, s21, v216\n\t" \
  "v_xad_u32 v145, v177, s21, v217\n\t" \
  "v_xad_u32 v146, v178, s21, v218\n\t" \
  "v_xad_u32 v147, v179, s21, v219\n\t" \
  "v_add_u32 v147, v147, v223\n\t" \
  "v_lshrrev_b32 v220, 5, v99\n\t" \
  "v_lshrrev_b32 v221, 3, v100\n\t" \
  "v_lshrrev_b32 v222, 1, v101\n\t" \
  "v_lshlrev_b32 v223, 1, v101\n\t" \
  "v_and_b32 v216, s95, v99\n\t" \
  "v_and_b32 v217, s95, v100\n\t" \
  "v_and_b32 v218, s95, v101\n\t" \
  "v_and_b32 v219, s97, v220\n\t" \
  "v_and_b32 v223, 0x80808080, v223\n\t" \
  "v_and_or_b32 v219, v221, s98, v219\n\t" \
  "v_and_or_b32 v223, v222, v29, v223\n\t" \
  "v_xad_u32 v148, v180, s21, v216\n\t" \
  "v_xad_u32 v149, v181, s21, v217\n\t" \
  "v_xad_u32 v150, v182, s21, v218\n\t" \
  "v_xad_u32 v151, v183, s21, v219\n\t" \
  "v_add_u32 v151, v151, v223\n\t" \
  "v_lshrrev_b32 v220, 5, v102\n\t" \
  "v_lshrrev_b32 v221, 3, v103\n\t" \
  "v_lshrrev_b32 v222, 1, v104\n\t" \
  "v_lshlrev_b32 v223, 1, v104\n\t" \
  "v_and_b32 v216, s95, v102\n\t" \
  "v_and_b32 v217, s95, v103\n\t" \
  "v_and_b32 v218, s95, v104\n\t" \
  "v_and_b32 v219, s97, v220\n\t" \
  "v_and_b32 v223, 0x80808080, v223\n\t" \
  "v_and_or_b32 v219, v221, s98, v219\n\t" \
  "v_and_or_b32 v223, v222, v29, v223\n\t" \
  "v_xad_u32 v152, v184, s21, v216\n\t" \
  "v_xad_u32 v153, v185, s21, v217\n\t" \
  "v_xad_u32 v154, v186, s21, v218\n\t" \
  "v_xad_u32 v155, v187, s21, v219\n\t" \
  "v_add_u32 v155, v155, v223\n\t" \
  "v_lshrrev_b32 v220, 5, v105\n\t" \
  "v_lshrrev_b32 v221, 3, v106\n\t" \
  "v_lshrrev_b32 v222, 1, v107\n\t" \
  "v_lshlrev_b32 v223, 1, v107\n\t" \
  "v_and_b32 v216, s95, v105\n\t" \
  "v_and_b32 v217, s95, v106\n\t" \
  "v_and_b32 v218, s95, v107\n\t" \
  "v_and_b32 v219, s97, v220\n\t" \
  "v_and_b32 v223, 0x80808080, v223\n\t" \
  "v_and_or_b32 v219, v221, s98, v219\n\t" \
  "v_and_or_b32 v223, v222, v29, v223\n\t" \
  "v_xad_u32 v156, v188, s21, v216\n\t" \
  "v_xad_u32 v157, v189, s21, v217\n\t" \
  "v_xad_u32 v158, v190, s21, v218\n\t" \
  "v_xad_u32 v159, v191, s21, v219\n\t" \
  "v_add_u32 v159, v159, v223\n\t" \
  "L_stB4_%=:\n\t" \
  "buffer_store_dwordx4 v[144:147], v16, s[28:31], s68 offen sc0 sc1\n\t" \
  "buffer_store_dwordx4 v[148:151], v17, s[28:31], s68 offen sc0 sc1\n\t" \
  "buffer_store_dwordx4 v[152:155], v18, s[28:31], s68 offen sc0 sc1\n\t" \
  "buffer_store_dwordx4 v[156:159], v19, s[28:31], s68 offen sc0 sc1\n\t" \
  "v_readfirstlane_b32 s21, v140\n\t" \
  "v_readlane_b32 s75, v140, 1\n\t" \
  "v_lshrrev_b32 v210, 5, v140\n\t" \
  "v_lshrrev_b32 v211, 17, v140\n\t" \
  "v_and_b32 v210, 0x60, v210\n\t" \
  "v_and_b32 v211, 0x60, v211\n\t" \
  "v_xor_b32 v210, s21, v210\n\t" \
  "v_xor_b32 v211, s21, v211\n\t" \
  "v_subrev_u32 v208, s75, v224\n\t" \
  "v_subrev_u32 v209, s75, v225\n\t" \
  "v_and_b32 v208, 0x80808080, v208\n\t" \
  "v_and_b32 v209, 0x80808080, v209\n\t" \
  "v_bcnt_u32_b32 v208, v208, v226\n\t" \
  "v_bcnt_u32_b32 v209, v209, v226\n\t" \
  "v_and_b32 v176, 0x3ff, v140\n\t" \
  "v_bfe_u32 v177, v140, 12, 10\n\t" \
  "v_lshl_add_u32 v208, v208, 10, v176\n\t" \
  "v_lshl_add_u32 v209, v209, 10, v177\n\t" \
  "ds_write_b8 v208, v210\n\t" \
  "ds_write_b8 v209, v211\n\t" \
  "ds_read_b128 v[176:179], v15 offset:0\n\t" \
  "ds_read_b128 v[180:183], v15 offset:1024\n\t" \
  "ds_read_b128 v[184:187], v15 offset:2048\n\t" \
  "ds_read_b128 v[188:191], v15 offset:3072\n\t" \
  "ds_write_b8 v208, v10\n\t" \
  "ds_write_b8 v209, v10\n\t" \
  "s_bitcmp1_b32 s22, 7\n\t" \
  "s_cbranch_scc1 L_fbB5_%=\n\t" \
  "s_waitcnt vmcnt(21) lgkmcnt(8)\n\t" \
  "v_lshrrev_b32 v220, 5, v112\n\t" \
  "v_lshrrev_b32 v221, 3, v113\n\t" \
  "v_lshrrev_b32 v222, 1, v114\n\t" \
  "v_lshlrev_b32 v223, 1, v114\n\t" \
  "v_and_b32 v216, s95, v112\n\t" \
  "v_and_b32 v217, s95, v113\n\t" \
  "v_and_b32 v218, s95, v114\n\t" \
  "v_and_b32 v219, s97, v220\n\t" \
  "v_and_b32 v223, 0x80808080, v223\n\t" \
  "v_and_or_b32 v219, v221, s98, v219\n\t" \
  "v_and_or_b32 v223, v222, v29, v223\n\t" \
  "v_xad_u32 v160, v192, s22, v216\n\t" \
  "v_xad_u32 v161, v193, s22, v217\n\t" \
  "v_xad_u32 v162, v194, s22, v218\n\t" \
  "v_xad_u32 v163, v195, s22, v219\n\t" \
  "v_add_u32 v163, v163, v223\n\t" \
  "v_lshrrev_b32 v220, 5, v115\n\t" \
  "v_lshrrev_b32 v221, 3, v116\n\t" \
  "v_lshrrev_b32 v222, 1, v117\n\t" \
  "v_lshlrev_b32 v223, 1, v117\n\t" \
  "v_and_b32 v216, s95, v115\n\t" \
  "v_and_b32 v217, s95, v116\n\t" \
  "v_and_b32 v218, s95, v117\n\t" \
  "v_and_b32 v219, s97, v220\n\t" \
  "v_and_b32 v223, 0x80808080, v223\n\t" \
  "v_and_or_b32 v219, v221, s98, v219\n\t" \
  "v_and_or_b32 v223, v222, v29, v223\n\t" \
  "v_xad_u32 v164, v196, s22, v216\n\t" \
  "v_xad_u32 v165, v197, s22, v217\n\t" \
  "v_xad_u32 v166, v198, s22, v218\n\t" \
  "v_xad_u32 v167, v199, s22, v219\n\t" \
  "v_add_u32 v167, v167, v223\n\t" \
  "v_lshrrev_b32 v220, 5, v118\n\t" \
  "v_lshrrev_b32 v221, 3, v119\n\t" \
  "v_lshrrev_b32 v222, 1, v120\n\t" \
  "v_lshlrev_b32 v223, 1, v120\n\t" \
  "v_and_b32 v216, s95, v118\n\t" \
  "v_and_b32 v217, s95, v119\n\t" \
  "v_and_b32 v218, s95, v120\n\t" \
  "v_and_b32 v219, s97, v220\n\t" \
  "v_and_b32 v223, 0x80808080, v223\n\t" \
  "v_and_or_b32 v219, v221, s98, v219\n\t" \
  "v_and_or_b32 v223, v222, v29, v223\n\t" \
  "v_xad_u32 v168, v200, s22, v216\n\t" \
  "v_xad_u32 v169, v201, s22, v217\n\t" \
  "v_xad_u32 v170, v202, s22, v218\n\t" \
  "v_xad_u32 v171, v203, s22, v219\n\t" \
  "v_add_u32 v171, v171, v223\n\t" \
  "v_lshrrev_b32 v220, 5, v121\n\t" \
  "v_lshrrev_b32 v221, 3, v122\n\t" \
  "v_lshrrev_b32 v222, 1, v123\n\t" \
  "v_lshlrev_b32 v223, 1, v123\n\t" \
  "v_and_b32 v216, s95, v121\n\t" \
  "v_and_b32 v217, s95, v122\n\t" \
  "v_and_b32 v218, s95, v123\n\t" \
  "v_and_b32 v219, s97, v220\n\t" \
  "v_and_b32 v223, 0x80808080, v223\n\t" \
  "v_and_or_b32 v219, v221, s98, v219\n\t" \
  "v_and_or_b32 v223, v222, v29, v223\n\t" \
  "v_xad_u32 v172, v204, s22, v216\n\t" \
  "v_xad_u32 v173, v205, s22, v217\n\t" \
  "v_xad_u32 v174, v206, s22, v218\n\t" \
  "v_xad_u32 v175, v207, s22, v219\n\t" \
  "v_add_u32 v175, v175, v223\n\t" \
  "L_stB5_%=:\n\t" \
  "buffer_store_dwordx4 v[160:163], v16, s[28:31], s69 offen sc0 sc1\n\t" \
  "buffer_store_dwordx4 v[164:167], v17, s[28:31], s69 offen sc0 sc1\n\t" \
  "buffer_store_dwordx4 v[168:171], v18, s[28:31], s69 offen sc0 sc1\n\t" \
  "buffer_store_dwordx4 v[172:175], v19, s[28:31], s69 offen sc0 sc1\n\t" \
  "s_bitcmp1_b32 s21, 7\n\t" \
  "s_cbranch_scc1 L_fbB6_%=\n\t" \
  "s_waitcnt vmcnt(24) lgkmcnt(1)\n\t" \
  "v_lshrrev_b32 v220, 5, v128\n\t" \
  "v_lshrrev_b32 v221, 3, v129\n\t" \
  "v_lshrrev_b32 v222, 1, v130\n\t" \
  "v_lshlrev_b32 v223, 1, v130\n\t" \
  "v_and_b32 v216, s95, v128\n\t" \
  "v_and_b32 v217, s95, v129\n\t" \
  "v_and_b32 v218, s95, v130\n\t" \
  "v_and_b32 v219, s97, v220\n\t" \
  "v_and_b32 v223, 0x80808080, v223\n\t" \
  "v_and_or_b32 v219, v221, s98, v219\n\t" \
  "v_and_or_b32 v223, v222, v29, v223\n\t" \
  "v_xad_u32 v144, v176, s21, v216\n\t" \
  "v_xad_u32 v145, v177, s21, v217\n\t" \
  "v_xad_u32 v146, v178, s21, v218\n\t" \
  "v_xad_u32 v147, v179, s21, v219\n\t" \
  "v_add_u32 v147, v147, v223\n\t" \
  "v_lshrrev_b32 v220, 5, v131\n\t" \
  "v_lshrrev_b32 v221, 3, v132\n\t" \
  "v_lshrrev_b32 v222, 1, v133\n\t" \
  "v_lshlrev_b32 v223, 1, v133\n\t" \
  "v_and_b32 v216, s95, v131\n\t" \
  "v_and_b32 v217, s95, v132\n\t" \
  "v_and_b32 v218, s95, v133\n\t" \
  "v_and_b32 v219, s97, v220\n\t" \
  "v_and_b32 v223, 0x80808080, v223\n\t" \
  "v_and_or_b32 v219, v221, s98, v219\n\t" \
  "v_and_or_b32 v223, v222, v29, v223\n\t" \
  "v_xad_u32 v148, v180, s21, v216\n\t" \
  "v_xad_u32 v149, v181, s21, v217\n\t" \
  "v_xad_u32 v150, v182, s21, v218\n\t" \
  "v_xad_u32 v151, v183, s21, v219\n\t" \
  "v_add_u32 v151, v151, v223\n\t" \
  "v_lshrrev_b32 v220, 5, v134\n\t" \
  "v_lshrrev_b32 v221, 3, v135\n\t" \
  "v_lshrrev_b32 v222, 1, v136\n\t" \
  "v_lshlrev_b32 v223, 1, v136\n\t" \
  "v_and_b32 v216, s95, v134\n\t" \
  "v_and_b32 v217, s95, v135\n\t" \
  "v_and_b32 v218, s95, v136\n\t" \
  "v_and_b32 v219, s97, v220\n\t" \
  "v_and_b32 v223, 0x80808080, v223\n\t" \
  "v_and_or_b32 v219, v221, s98, v219\n\t" \
  "v_and_or_b32 v223, v222, v29, v223\n\t" \
  "v_xad_u32 v152, v184, s21, v216\n\t" \
  "v_xad_u32 v153, v185, s21, v217\n\t" \
  "v_xad_u32 v154, v186, s21, v218\n\t" \
  "v_xad_u32 v155, v187, s21, v219\n\t" \
  "v_add_u32 v155, v155, v223\n\t" \
  "v_lshrrev_b32 v220, 5, v137\n\t" \
  "v_lshrrev_b32 v221, 3, v138\n\t" \
  "v_lshrrev_b32 v222, 1, v139\n\t" \
  "v_lshlrev_b32 v223, 1, v139\n\t" \
  "v_and_b32 v216, s95, v137\n\t" \
  "v_and_b32 v217, s95, v138\n\t" \
  "v_and_b32 v218, s95, v139\n\t" \
  "v_and_b32 v219, s97, v220\n\t" \
  "v_and_b32 v223, 0x80808080, v223\n\t" \
  "v_and_or_b32 v219, v221, s98, v219\n\t" \
  "v_and_or_b32 v223, v222, v29, v223\n\t" \
  "v_xad_u32 v156, v188, s21, v216\n\t" \
  "v_xad_u32 v157, v189, s21, v217\n\t" \
  "v_xad_u32 v158, v190, s21, v218\n\t" \
  "v_xad_u32 v159, v191, s21, v219\n\t" \
  "v_add_u32 v159, v159, v223\n\t" \
  "L_stB6_%=:\n\t" \
  "buffer_store_dwordx4 v[144:147], v16, s[28:31], s70 offen sc0 sc1\n\t" \
  "buffer_store_dwordx4 v[148:151], v17, s[28:31], s70 offen sc0 sc1\n\t" \
  "buffer_store_dwordx4 v[152:155], v18, s[28:31], s70 offen sc0 sc1\n\t" \
  "buffer_store_dwordx4 v[156:159], v19, s[28:31], s70 offen sc0 sc1\n\t" \
  "s_branch L_done_%=\n\t" \
  "L_fbB0_%=:\n\t" \
  "buffer_load_dwordx4 v[144:147], v16, s[36:39], s64 offen sc0 sc1\n\t" \
  "buffer_load_dwordx4 v[148:151], v17, s[36:39], s64 offen sc0 sc1\n\t" \
  "buffer_load_dwordx4 v[152:155], v18, s[36:39], s64 offen sc0 sc1\n\t" \
  "buffer_load_dwordx4 v[156:159], v19, s[36:39], s64 offen sc0 sc1\n\t" \
  "s_waitcnt vmcnt(0) lgkmcnt(0)\n\t" \
  "s_branch L_stB0_%=\n\t" \
  "L_fbB1_%=:\n\t" \
  "buffer_load_dwordx4 v[160:163], v16, s[40:43], s65 offen sc0 sc1\n\t" \
  "buffer_load_dwordx4 v[164:167], v17, s[40:43], s65 offen sc0 sc1\n\t" \
  "buffer_load_dwordx4 v[168:171], v18, s[40:43], s65 offen sc0 sc1\n\t" \
  "buffer_load_dwordx4 v[172:175], v19, s[40:43], s65 offen sc0 sc1\n\t" \
  "s_waitcnt vmcnt(0) lgkmcnt(0)\n\t" \
  "s_branch L_stB1_%=\n\t" \
  "L_fbB2_%=:\n\t" \
  "buffer_load_dwordx4 v[144:147], v16, s[44:47], s66 offen sc0 sc1\n\t" \
  "buffer_load_dwordx4 v[148:151], v17, s[44:47], s66 offen sc0 sc1\n\t" \
  "buffer_load_dwordx4 v[152:155], v18, s[44:47], s66 offen sc0 sc1\n\t" \
  "buffer_load_dwordx4 v[156:159], v19, s[44:47], s66 offen sc0 sc1\n\t" \
  "s_waitcnt vmcnt(0) lgkmcnt(0)\n\t" \
  "s_branch L_stB2_%=\n\t" \
  "L_fbB3_%=:\n\t" \
  "buffer_load_dwordx4 v[160:163], v16, s[48:51], s67 offen sc0 sc1\n\t" \
  "buffer_load_dwordx4 v[164:167], v17, s[48:51], s67 offen sc0 sc1\n\t" \
  "buffer_load_dwordx4 v[168:171], v18, s[48:51], s67 offen sc0 sc1\n\t" \
  "buffer_load_dwordx4 v[172:175], v19, s[48:51], s67 offen sc0 sc1\n\t" \
  "s_waitcnt vmcnt(0) lgkmcnt(0)\n\t" \
  "s_branch L_stB3_%=\n\t" \
  "L_fbB4_%=:\n\t" \
  "buffer_load_dwordx4 v[144:147], v16, s[52:55], s68 offen sc0 sc1\n\t" \
  "buffer_load_dwordx4 v[148:151], v17, s[52:55], s68 offen sc0 sc1\n\t" \
  "buffer_load_dwordx4 v[152:155], v18, s[52:55], s68 offen sc0 sc1\n\t" \
  "buffer_load_dwordx4 v[156:159], v19, s[52:55], s68 offen sc0 sc1\n\t" \
  "s_waitcnt vmcnt(0) lgkmcnt(0)\n\t" \
  "s_branch L_stB4_%=\n\t" \
  "L_fbB5_%=:\n\t" \
  "buffer_load_dwordx4 v[160:163], v16, s[56:59], s69 offen sc0 sc1\n\t" \
  "buffer_load_dwordx4 v[164:167], v17, s[56:59], s69 offen sc0 sc1\n\t" \
  "buffer_load_dwordx4 v[168:171], v18, s[56:59], s69 offen sc0 sc1\n\t" \
  "buffer_load_dwordx4 v[172:175], v19, s[56:59], s69 offen sc0 sc1\n\t" \
  "s_waitcnt vmcnt(0) lgkmcnt(0)\n\t" \
  "s_branch L_stB5_%=\n\t" \
  "L_fbB6_%=:\n\t" \
  "buffer_load_dwordx4 v[144:147], v16, s[60:63], s70 offen sc0 sc1\n\t" \
  "buffer_load_dwordx4 v[148:151], v17, s[60:63], s70 offen sc0 sc1\n\t" \
  "buffer_load_dwordx4 v[152:155], v18, s[60:63], s70 offen sc0 sc1\n\t" \
  "buffer_load_dwordx4 v[156:159], v19, s[60:63], s70 offen sc0 sc1\n\t" \
  "s_waitcnt vmcnt(0) lgkmcnt(0)\n\t" \
  "s_branch L_stB6_%=\n\t" \
  "L_pA_%=:\n\t" \
  "buffer_load_dword v46, v14, s[36:39], s93 offen offset:3584 sc0 sc1\n\t" \
  "buffer_load_dword v62, v14, s[40:43], s93 offen offset:3584 sc0 sc1\n\t" \
  "buffer_load_dword v78, v14, s[44:47], s93 offen offset:3584 sc0 sc1\n\t" \
  "buffer_load_dword v94, v14, s[48:51], s93 offen offset:3584 sc0 sc1\n\t" \
  "buffer_load_dword v110, v14, s[52:55], s93 offen offset:3584 sc0 sc1\n\t" \
  "buffer_load_dword v126, v14, s[56:59], s93 offen offset:3584 sc0 sc1\n\t" \
  "buffer_load_dword v142, v14, s[60:63], s93 offen offset:3584 sc0 sc1\n\t" \
  "buffer_load_dwordx2 v[44:45], v24, s[36:39], s93 offen offset:3072 sc0 sc1\n\t" \
  "buffer_load_dwordx2 v[60:61], v24, s[40:43], s93 offen offset:3072 sc0 sc1\n\t" \
  "buffer_load_dwordx2 v[76:77], v24, s[44:47], s93 offen offset:3072 sc0 sc1\n\t" \
  "buffer_load_dwordx2 v[92:93], v24, s[48:51], s93 offen offset:3072 sc0 sc1\n\t" \
  "buffer_load_dwordx2 v[108:109], v24, s[52:55], s93 offen offset:3072 sc0 sc1\n\t" \
  "buffer_load_dwordx2 v[124:125], v24, s[56:59], s93 offen offset:3072 sc0 sc1\n\t" \
  "buffer_load_dwordx2 v[140:141], v24, s[60:63], s93 offen offset:3072 sc0 sc1\n\t" \
  "buffer_load_dwordx4 v[32:35], v8, s[36:39], s93 offen offset:0 sc0 sc1\n\t" \
  "buffer_load_dwordx4 v[48:51], v8, s[40:43], s93 offen offset:0 sc0 sc1\n\t" \
  "buffer_load_dwordx4 v[64:67], v8, s[44:47], s93 offen offset:0 sc0 sc1\n\t" \
  "buffer_load_dwordx4 v[80:83], v8, s[48:51], s93 offen offset:0 sc0 sc1\n\t" \
  "buffer_load_dwordx4 v[96:99], v8, s[52:55], s93 offen offset:0 sc0 sc1\n\t" \
  "buffer_load_dwordx4 v[112:115], v8, s[56:59], s93 offen offset:0 sc0 sc1\n\t" \
  "buffer_load_dwordx4 v[128:131], v8, s[60:63], s93 offen offset:0 sc0 sc1\n\t" \
  "buffer_load_dwordx4 v[36:39], v8, s[36:39], s93 offen offset:1024 sc0 sc1\n\t" \
  "buffer_load_dwordx4 v[52:55], v8, s[40:43], s93 offen offset:1024 sc0 sc1\n\t" \
  "buffer_load_dwordx4 v[68:71], v8, s[44:47], s93 offen offset:1024 sc0 sc1\n\t" \
  "s_waitcnt vmcnt(23)\n\t" \
  "buffer_load_dwordx4 v[84:87], v8, s[48:51], s93 offen offset:1024 sc0 sc1\n\t" \
  "s_waitcnt vmcnt(23)\n\t" \
  "buffer_load_dwordx4 v[100:103], v8, s[52:55], s93 offen offset:1024 sc0 sc1\n\t" \
  "s_waitcnt vmcnt(23)\n\t" \
  "buffer_load_dwordx4 v[116:119], v8, s[56:59], s93 offen offset:1024 sc0 sc1\n\t" \
  "s_waitcnt vmcnt(23)\n\t" \
  "buffer_load_dwordx4 v[132:135], v8, s[60:63], s93 offen offset:1024 sc0 sc1\n\t" \
  "s_waitcnt vmcnt(23)\n\t" \
  "buffer_load_dwordx4 v[40:43], v8, s[36:39], s93 offen offset:2048 sc0 sc1\n\t" \
  "s_waitcnt vmcnt(23)\n\t" \
  "buffer_load_dwordx4 v[56:59], v8, s[40:43], s93 offen offset:2048 sc0 sc1\n\t" \
  "s_waitcnt vmcnt(23)\n\t" \
  "buffer_load_dwordx4 v[72:75], v8, s[44:47], s93 offen offset:2048 sc0 sc1\n\t" \
  "s_waitcnt vmcnt(23)\n\t" \
  "buffer_load_dwordx4 v[88:91], v8, s[48:51], s93 offen offset:2048 sc0 sc1\n\t" \
  "s_waitcnt vmcnt(23)\n\t" \
  "buffer_load_dwordx4 v[104:107], v8, s[52:55], s93 offen offset:2048 sc0 sc1\n\t" \
  "s_waitcnt vmcnt(23)\n\t" \
  "buffer_load_dwordx4 v[120:123], v8, s[56:59], s93 offen offset:2048 sc0 sc1\n\t" \
  "s_waitcnt vmcnt(23)\n\t" \
  "buffer_load_dwordx4 v[136:139], v8, s[60:63], s93 offen offset:2048 sc0 sc1\n\t" \
  "s_waitcnt vmcnt(21)\n\t" \
  "v_readfirstlane_b32 s21, v46\n\t" \
  "v_readlane_b32 s75, v46, 1\n\t" \
  "v_lshrrev_b32 v210, 5, v46\n\t" \
  "v_lshrrev_b32 v211, 17, v46\n\t" \
  "v_and_b32 v210, 0x60, v210\n\t" \
  "v_and_b32 v211, 0x60, v211\n\t" \
  "v_xor_b32 v210, s21, v210\n\t" \
  "v_xor_b32 v211, s21, v211\n\t" \
  "v_subrev_u32 v208, s75, v224\n\t" \
  "v_subrev_u32 v209, s75, v225\n\t" \
  "v_and_b32 v208, 0x80808080, v208\n\t" \
  "v_and_b32 v209, 0x80808080, v209\n\t" \
  "v_bcnt_u32_b32 v208, v208, v226\n\t" \
  "v_bcnt_u32_b32 v209, v209, v226\n\t" \
  "v_and_b32 v176, 0x3ff, v46\n\t" \
  "v_bfe_u32 v177, v46, 12, 10\n\t" \
  "v_lshl_add_u32 v208, v208, 10, v176\n\t" \
  "v_lshl_add_u32 v209, v209, 10, v177\n\t" \
  "ds_write_b8 v208, v210\n\t" \
  "ds_write_b8 v209, v211\n\t" \
  "ds_read_b128 v[176:179], v15 offset:0\n\t" \
  "ds_read_b128 v[180:183], v15 offset:1024\n\t" \
  "ds_read_b128 v[184:187], v15 offset:2048\n\t" \
  "ds_read_b128 v[188:191], v15 offset:3072\n\t" \
  "ds_write_b8 v208, v10\n\t" \
  "ds_write_b8 v209, v10\n\t" \
  "s_waitcnt vmcnt(14)\n\t" \
  "s_mov_b64 exec, 1\n\t" \
  "ds_add_u32 v10, v13\n\t" \
  "s_mov_b64 exec, -1\n\t" \
  "v_readfirstlane_b32 s22, v62\n\t" \
  "v_readlane_b32 s75, v62, 1\n\t" \
  "v_lshrrev_b32 v214, 5, v62\n\t" \
  "v_lshrrev_b32 v215, 17, v62\n\t" \
  "v_and_b32 v214, 0x60, v214\n\t" \
  "v_and_b32 v215, 0x60, v215\n\t" \
  "v_xor_b32 v214, s22, v214\n\t" \
  "v_xor_b32 v215, s22, v215\n\t" \
  "v_subrev_u32 v212, s75, v224\n\t" \
  "v_subrev_u32 v213, s75, v225\n\t" \
  "v_and_b32 v212, 0x80808080, v212\n\t" \
  "v_and_b32 v213, 0x80808080, v213\n\t" \
  "v_bcnt_u32_b32 v212, v212, v226\n\t" \
  "v_bcnt_u32_b32 v213, v213, v226\n\t" \
  "v_and_b32 v192, 0x3ff, v62\n\t" \
  "v_bfe_u32 v193, v62, 12, 10\n\t" \
  "v_lshl_add_u32 v212, v212, 10, v192\n\t" \
  "v_lshl_add_u32 v213, v213, 10, v193\n\t" \
  "ds_write_b8 v212, v214\n\t" \
  "ds_write_b8 v213, v215\n\t" \
  "ds_read_b128 v[192:195], v15 offset:0\n\t" \
  "ds_read_b128 v[196:199], v15 offset:1024\n\t" \
  "ds_read_b128 v[200:203], v15 offset:2048\n\t" \
  "ds_read_b128 v[204:207], v15 offset:3072\n\t" \
  "ds_write_b8 v212, v10\n\t" \
  "ds_write_b8 v213, v10\n\t" \
  "s_bitcmp1_b32 s21, 7\n\t" \
  "s_cbranch_scc1 L_fbA0_%=\n\t" \
  "s_waitcnt vmcnt(6) lgkmcnt(8)\n\t" \
  "v_lshrrev_b32 v216, 6, v32\n\t" \
  "v_lshrrev_b32 v217, 5, v33\n\t" \
  "v_lshrrev_b32 v218, 4, v34\n\t" \
  "v_lshrrev_b32 v219, 3, v35\n\t" \
  "v_lshrrev_b32 v220, 2, v36\n\t" \
  "v_lshrrev_b32 v221, 1, v37\n\t" \
  "v_lshlrev_b32 v222, 1, v38\n\t" \
  "v_and_b32 v222, 0x80808080, v222\n\t" \
  "v_and_b32 v32, v31, v32\n\t" \
  "v_and_or_b32 v222, v216, v25, v222\n\t" \
  "v_and_b32 v33, v31, v33\n\t" \
  "v_and_or_b32 v222, v217, v26, v222\n\t" \
  "v_and_b32 v34, v31, v34\n\t" \
  "v_and_or_b32 v222, v218, v27, v222\n\t" \
  "v_and_b32 v35, v31, v35\n\t" \
  "v_and_or_b32 v222, v219, v28, v222\n\t" \
  "v_and_b32 v36, v31, v36\n\t" \
  "v_and_or_b32 v222, v220, v29, v222\n\t" \
  "v_and_b32 v37, v31, v37\n\t" \
  "v_and_or_b32 v222, v221, v30, v222\n\t" \
  "v_and_b32 v38, v31, v38\n\t" \
  "v_xad_u32 v144, v176, s21, v32\n\t" \
  "v_xad_u32 v145, v177, s21, v33\n\t" \
  "v_xad_u32 v146, v178, s21, v34\n\t" \
  "v_xad_u32 v147, v179, s21, v35\n\t" \
  "v_xad_u32 v148, v180, s21, v36\n\t" \
  "v_xad_u32 v149, v181, s21, v37\n\t" \
  "v_xad_u32 v150, v182, s21, v38\n\t" \
  "v_xad_u32 v151, v183, s21, v222\n\t" \
  "v_lshrrev_b32 v216, 6, v39\n\t" \
  "v_lshrrev_b32 v217, 5, v40\n\t" \
  "v_lshrrev_b32 v218, 4, v41\n\t" \
  "v_lshrrev_b32 v219, 3, v42\n\t" \
  "v_lshrrev_b32 v220, 2, v43\n\t" \
  "v_lshrrev_b32 v221, 1, v44\n\t" \
  "v_lshlrev_b32 v222, 1, v45\n\t" \
  "v_and_b32 v222, 0x80808080, v222\n\t" \
  "v_and_b32 v39, v31, v39\n\t" \
  "v_and_or_b32 v222, v216, v25, v222\n\t" \
  "v_and_b32 v40, v31, v40\n\t" \
  "v_and_or_b32 v222, v217, v26, v222\n\t" \
  "v_and_b32 v41, v31, v41\n\t" \
  "v_and_or_b32 v222, v218, v27, v222\n\t" \
  "v_and_b32 v42, v31, v42\n\t" \
  "v_and_or_b32 v222, v219, v28, v222\n\t" \
  "v_and_b32 v43, v31, v43\n\t" \
  "v_and_or_b32 v222, v220, v29, v222\n\t" \
  "v_and_b32 v44, v31, v44\n\t" \
  "v_and_or_b32 v222, v221, v30, v222\n\t" \
  "v_and_b32 v45, v31, v45\n\t" \
  "v_xad_u32 v152, v184, s21, v39\n\t" \
  "v_xad_u32 v153, v185, s21, v40\n\t" \
  "v_xad_u32 v154, v186, s21, v41\n\t" \
  "v_xad_u32 v155, v187, s21, v42\n\t" \
  "v_xad_u32 v156, v188, s21, v43\n\t" \
  "v_xad_u32 v157, v189, s21, v44\n\t" \
  "v_xad_u32 v158, v190, s21, v45\n\t" \
  "v_xad_u32 v159, v191, s21, v222\n\t" \
  "L_stA0_%=:\n\t" \
  "buffer_store_dwordx4 v[144:147], v16, s[28:31], s64 offen sc0 sc1\n\t" \
  "buffer_store_dwordx4 v[148:151], v17, s[28:31], s64 offen sc0 sc1\n\t" \
  "buffer_store_dwordx4 v[152:155], v18, s[28:31], s64 offen sc0 sc1\n\t" \
  "buffer_store_dwordx4 v[156:159], v19, s[28:31], s64 offen sc0 sc1\n\t" \
  "v_readfirstlane_b32 s21, v78\n\t" \
  "v_readlane_b32 s75, v78, 1\n\t" \
  "v_lshrrev_b32 v210, 5, v78\n\t" \
  "v_lshrrev_b32 v211, 17, v78\n\t" \
  "v_and_b32 v210, 0x60, v210\n\t" \
  "v_and_b32 v211, 0x60, v211\n\t" \
  "v_xor_b32 v210, s21, v210\n\t" \
  "v_xor_b32 v211, s21, v211\n\t" \
  "v_subrev_u32 v208, s75, v224\n\t" \
  "v_subrev_u32 v209, s75, v225\n\t" \
  "v_and_b32 v208, 0x80808080, v208\n\t" \
  "v_and_b32 v209, 0x80808080, v209\n\t" \
  "v_bcnt_u32_b32 v208, v208, v226\n\t" \
  "v_bcnt_u32_b32 v209, v209, v226\n\t" \
  "v_and_b32 v176, 0x3ff, v78\n\t" \
  "v_bfe_u32 v177, v78, 12, 10\n\t" \
  "v_lshl_add_u32 v208, v208, 10, v176\n\t" \
  "v_lshl_add_u32 v209, v209, 10, v177\n\t" \
  "ds_write_b8 v208, v210\n\t" \
  "ds_write_b8 v209, v211\n\t" \
  "ds_read_b128 v[176:179], v15 offset:0\n\t" \
  "ds_read_b128 v[180:183], v15 offset:1024\n\t" \
  "ds_read_b128 v[184:187], v15 offset:2048\n\t" \
  "ds_read_b128 v[188:191], v15 offset:3072\n\t" \
  "ds_write_b8 v208, v10\n\t" \
  "ds_write_b8 v209, v10\n\t" \
  "s_bitcmp1_b32 s22, 7\n\t" \
  "s_cbranch_scc1 L_fbA1_%=\n\t" \
  "s_waitcnt vmcnt(9) lgkmcnt(8)\n\t" \
  "v_lshrrev_b32 v216, 6, v48\n\t" \
  "v_lshrrev_b32 v217, 5, v49\n\t" \
  "v_lshrrev_b32 v218, 4, v50\n\t" \
  "v_lshrrev_b32 v219, 3, v51\n\t" \
  "v_lshrrev_b32 v220, 2, v52\n\t" \
  "v_lshrrev_b32 v221, 1, v53\n\t" \
  "v_lshlrev_b32 v222, 1, v54\n\t" \
  "v_and_b32 v222, 0x80808080, v222\n\t" \
  "v_and_b32 v48, v31, v48\n\t" \
  "v_and_or_b32 v222, v216, v25, v222\n\t" \
  "v_and_b32 v49, v31, v49\n\t" \
  "v_and_or_b32 v222, v217, v26, v222\n\t" \
  "v_and_b32 v50, v31, v50\n\t" \
  "v_and_or_b32 v222, v218, v27, v222\n\t" \
  "v_and_b32 v51, v31, v51\n\t" \
  "v_and_or_b32 v222, v219, v28, v222\n\t" \
  "v_and_b32 v52, v31, v52\n\t" \
  "v_and_or_b32 v222, v220, v29, v222\n\t" \
  "v_and_b32 v53, v31, v53\n\t" \
  "v_and_or_b32 v222, v221, v30, v222\n\t" \
  "v_and_b32 v54, v31, v54\n\t" \
  "v_xad_u32 v160, v192, s22, v48\n\t" \
  "v_xad_u32 v161, v193, s22, v49\n\t" \
  "v_xad_u32 v162, v194, s22, v50\n\t" \
  "v_xad_u32 v163, v195, s22, v51\n\t" \
  "v_xad_u32 v164, v196, s22, v52\n\t" \
  "v_xad_u32 v165, v197, s22, v53\n\t" \
  "v_xad_u32 v166, v198, s22, v54\n\t" \
  "v_xad_u32 v167, v199, s22, v222\n\t" \
  "v_lshrrev_b32 v216, 6, v55\n\t" \
  "v_lshrrev_b32 v217, 5, v56\n\t" \
  "v_lshrrev_b32 v218, 4, v57\n\t" \
  "v_lshrrev_b32 v219, 3, v58\n\t" \
  "v_lshrrev_b32 v220, 2, v59\n\t" \
  "v_lshrrev_b32 v221, 1, v60\n\t" \
  "v_lshlrev_b32 v222, 1, v61\n\t" \
  "v_and_b32 v222, 0x80808080, v222\n\t" \
  "v_and_b32 v55, v31, v55\n\t" \
  "v_and_or_b32 v222, v216, v25, v222\n\t" \
  "v_and_b32 v56, v31, v56\n\t" \
  "v_and_or_b32 v222, v217, v26, v222\n\t" \
  "v_and_b32 v57, v31, v57\n\t" \
  "v_and_or_b32 v222, v218, v27, v222\n\t" \
  "v_and_b32 v58, v31, v58\n\t" \
  "v_and_or_b32 v222, v219, v28, v222\n\t" \
  "v_and_b32 v59, v31, v59\n\t" \
  "v_and_or_b32 v222, v220, v29, v222\n\t" \
  "v_and_b32 v60, v31, v60\n\t" \
  "v_and_or_b32 v222, v221, v30, v222\n\t" \
  "v_and_b32 v61, v31, v61\n\t" \
  "v_xad_u32 v168, v200, s22, v55\n\t" \
  "v_xad_u32 v169, v201, s22, v56\n\t" \
  "v_xad_u32 v170, v202, s22, v57\n\t" \
  "v_xad_u32 v171, v203, s22, v58\n\t" \
  "v_xad_u32 v172, v204, s22, v59\n\t" \
  "v_xad_u32 v173, v205, s22, v60\n\t" \
  "v_xad_u32 v174, v206, s22, v61\n\t" \
  "v_xad_u32 v175, v207, s22, v222\n\t" \
  "L_stA1_%=:\n\t" \
  "buffer_store_dwordx4 v[160:163], v16, s[28:31], s65 offen sc0 sc1\n\t" \
  "buffer_store_dwordx4 v[164:167], v17, s[28:31], s65 offen sc0 sc1\n\t" \
  "buffer_store_dwordx4 v[168:171], v18, s[28:31], s65 offen sc0 sc1\n\t" \
  "buffer_store_dwordx4 v[172:175], v19, s[28:31], s65 offen sc0 sc1\n\t" \
  "v_readfirstlane_b32 s22, v94\n\t" \
  "v_readlane_b32 s75, v94, 1\n\t" \
  "v_lshrrev_b32 v214, 5, v94\n\t" \
  "v_lshrrev_b32 v215, 17, v94\n\t" \
  "v_and_b32 v214, 0x60, v214\n\t" \
  "v_and_b32 v215, 0x60, v215\n\t" \
  "v_xor_b32 v214, s22, v214\n\t" \
  "v_xor_b32 v215, s22, v215\n\t" \
  "v_subrev_u32 v212, s75, v224\n\t" \
  "v_subrev_u32 v213, s75, v225\n\t" \
  "v_and_b32 v212, 0x80808080, v212\n\t" \
  "v_and_b32 v213, 0x80808080, v213\n\t" \
  "v_bcnt_u32_b32 v212, v212, v226\n\t" \
  "v_bcnt_u32_b32 v213, v213, v226\n\t" \
  "v_and_b32 v192, 0x3ff, v94\n\t" \
  "v_bfe_u32 v193, v94, 12, 10\n\t" \
  "v_lshl_add_u32 v212, v212, 10, v192\n\t" \
  "v_lshl_add_u32 v213, v213, 10, v193\n\t" \
  "ds_write_b8 v212, v214\n\t" \
  "ds_write_b8 v213, v215\n\t" \
  "ds_read_b128 v[192:195], v15 offset:0\n\t" \
  "ds_read_b128 v[196:199], v15 offset:1024\n\t" \
  "ds_read_b128 v[200:203], v15 offset:2048\n\t" \
  "ds_read_b128 v[204:207], v15 offset:3072\n\t" \
  "ds_write_b8 v212, v10\n\t" \
  "ds_write_b8 v213, v10\n\t" \
  "s_bitcmp1_b32 s21, 7\n\t" \
  "s_cbranch_scc1 L_fbA2_%=\n\t" \
  "s_waitcnt vmcnt(12) lgkmcnt(8)\n\t" \
  "v_lshrrev_b32 v216, 6, v64\n\t" \
  "v_lshrrev_b32 v217, 5, v65\n\t" \
  "v_lshrrev_b32 v218, 4, v66\n\t" \
  "v_lshrrev_b32 v219, 3, v67\n\t" \
  "v_lshrrev_b32 v220, 2, v68\n\t" \
  "v_lshrrev_b32 v221, 1, v69\n\t" \
  "v_lshlrev_b32 v222, 1, v70\n\t" \
  "v_and_b32 v222, 0x80808080, v222\n\t" \
  "v_and_b32 v64, v31, v64\n\t" \
  "v_and_or_b32 v222, v216, v25, v222\n\t" \
  "v_and_b32 v65, v31, v65\n\t" \
  "v_and_or_b32 v222, v217, v26, v222\n\t" \
  "v_and_b32 v66, v31, v66\n\t" \
  "v_and_or_b32 v222, v218, v27, v222\n\t" \
  "v_and_b32 v67, v31, v67\n\t" \
  "v_and_or_b32 v222, v219, v28, v222\n\t" \
  "v_and_b32 v68, v31, v68\n\t" \
  "v_and_or_b32 v222, v220, v29, v222\n\t" \
  "v_and_b32 v69, v31, v69\n\t" \
  "v_and_or_b32 v222, v221, v30, v222\n\t" \
  "v_and_b32 v70, v31, v70\n\t" \
  "v_xad_u32 v144, v176, s21, v64\n\t" \
  "v_xad_u32 v145, v177, s21, v65\n\t" \
  "v_xad_u32 v146, v178, s21, v66\n\t" \
  "v_xad_u32 v147, v179, s21, v67\n\t" \
  "v_xad_u32 v148, v180, s21, v68\n\t" \
  "v_xad_u32 v149, v181, s21, v69\n\t" \
  "v_xad_u32 v150, v182, s21, v70\n\t" \
  "v_xad_u32 v151, v183, s21, v222\n\t" \
  "v_lshrrev_b32 v216, 6, v71\n\t" \
  "v_lshrrev_b32 v217, 5, v72\n\t" \
  "v_lshrrev_b32 v218, 4, v73\n\t" \
  "v_lshrrev_b32 v219, 3, v74\n\t" \
  "v_lshrrev_b32 v220, 2, v75\n\t" \
  "v_lshrrev_b32 v221, 1, v76\n\t" \
  "v_lshlrev_b32 v222, 1, v77\n\t" \
  "v_and_b32 v222, 0x80808080, v222\n\t" \
  "v_and_b32 v71, v31, v71\n\t" \
  "v_and_or_b32 v222, v216, v25, v222\n\t" \
  "v_and_b32 v72, v31, v72\n\t" \
  "v_and_or_b32 v222, v217, v26, v222\n\t" \
  "v_and_b32 v73, v31, v73\n\t" \
  "v_and_or_b32 v222, v218, v27, v222\n\t" \
  "v_and_b32 v74, v31, v74\n\t" \
  "v_and_or_b32 v222, v219, v28, v222\n\t" \
  "v_and_b32 v75, v31, v75\n\t" \
  "v_and_or_b32 v222, v220, v29, v222\n\t" \
  "v_and_b32 v76, v31, v76\n\t" \
  "v_and_or_b32 v222, v221, v30, v222\n\t" \
  "v_and_b32 v77, v31, v77\n\t" \
  "v_xad_u32 v152, v184, s21, v71\n\t" \
  "v_xad_u32 v153, v185, s21, v72\n\t" \
  "v_xad_u32 v154, v186, s21, v73\n\t" \
  "v_xad_u32 v155, v187, s21, v74\n\t" \
  "v_xad_u32 v156, v188, s21, v75\n\t" \
  "v_xad_u32 v157, v189, s21, v76\n\t" \
  "v_xad_u32 v158, v190, s21, v77\n\t" \
  "v_xad_u32 v159, v191, s21, v222\n\t" \
  "L_stA2_%=:\n\t" \
  "buffer_store_dwordx4 v[144:147], v16, s[28:31], s66 offen sc0 sc1\n\t" \
  "buffer_store_dwordx4 v[148:151], v17, s[28:31], s66 offen sc0 sc1\n\t" \
  "buffer_store_dwordx4 v[152:155], v18, s[28:31], s66 offen sc0 sc1\n\t" \
  "buffer_store_dwordx4 v[156:159], v19, s[28:31], s66 offen sc0 sc1\n\t" \
  "v_readfirstlane_b32 s21, v110\n\t" \
  "v_readlane_b32 s75, v110, 1\n\t" \
  "v_lshrrev_b32 v210, 5, v110\n\t" \
  "v_lshrrev_b32 v211, 17, v110\n\t" \
  "v_and_b32 v210, 0x60, v210\n\t" \
  "v_and_b32 v211, 0x60, v211\n\t" \
  "v_xor_b32 v210, s21, v210\n\t" \
  "v_xor_b32 v211, s21, v211\n\t" \
  "v_subrev_u32 v208, s75, v224\n\t" \
  "v_subrev_u32 v209, s75, v225\n\t" \
  "v_and_b32 v208, 0x80808080, v208\n\t" \
  "v_and_b32 v209, 0x80808080, v209\n\t" \
  "v_bcnt_u32_b32 v208, v208, v226\n\t" \
  "v_bcnt_u32_b32 v209, v209, v226\n\t" \
  "v_and_b32 v176, 0x3ff, v110\n\t" \
  "v_bfe_u32 v177, v110, 12, 10\n\t" \
  "v_lshl_add_u32 v208, v208, 10, v176\n\t" \
  "v_lshl_add_u32 v209, v209, 10, v177\n\t" \
  "ds_write_b8 v208, v210\n\t" \
  "ds_write_b8 v209, v211\n\t" \
  "ds_read_b128 v[176:179], v15 offset:0\n\t" \
  "ds_read_b128 v[180:183], v15 offset:1024\n\t" \
  "ds_read_b128 v[184:187], v15 offset:2048\n\t" \
  "ds_read_b128 v[188:191], v15 offset:3072\n\t" \
  "ds_write_b8 v208, v10\n\t" \
  "ds_write_b8 v209, v10\n\t" \
  "s_bitcmp1_b32 s22, 7\n\t" \
  "s_cbranch_scc1 L_fbA3_%=\n\t" \
  "s_waitcnt vmcnt(15) lgkmcnt(8)\n\t" \
  "v_lshrrev_b32 v216, 6, v80\n\t" \
  "v_lshrrev_b32 v217, 5, v81\n\t" \
  "v_lshrrev_b32 v218, 4, v82\n\t" \
  "v_lshrrev_b32 v219, 3, v83\n\t" \
  "v_lshrrev_b32 v220, 2, v84\n\t" \
  "v_lshrrev_b32 v221, 1, v85\n\t" \
  "v_lshlrev_b32 v222, 1, v86\n\t" \
  "v_and_b32 v222, 0x80808080, v222\n\t" \
  "v_and_b32 v80, v31, v80\n\t" \
  "v_and_or_b32 v222, v216, v25, v222\n\t" \
  "v_and_b32 v81, v31, v81\n\t" \
  "v_and_or_b32 v222, v217, v26, v222\n\t" \
  "v_and_b32 v82, v31, v82\n\t" \
  "v_and_or_b32 v222, v218, v27, v222\n\t" \
  "v_and_b32 v83, v31, v83\n\t" \
  "v_and_or_b32 v222, v219, v28, v222\n\t" \
  "v_and_b32 v84, v31, v84\n\t" \
  "v_and_or_b32 v222, v220, v29, v222\n\t" \
  "v_and_b32 v85, v31, v85\n\t" \
  "v_and_or_b32 v222, v221, v30, v222\n\t" \
  "v_and_b32 v86, v31, v86\n\t" \
  "v_xad_u32 v160, v192, s22, v80\n\t" \
  "v_xad_u32 v161, v193, s22, v81\n\t" \
  "v_xad_u32 v162, v194, s22, v82\n\t" \
  "v_xad_u32 v163, v195, s22, v83\n\t" \
  "v_xad_u32 v164, v196, s22, v84\n\t" \
  "v_xad_u32 v165, v197, s22, v85\n\t" \
  "v_xad_u32 v166, v198, s22, v86\n\t" \
  "v_xad_u32 v167, v199, s22, v222\n\t" \
  "v_lshrrev_b32 v216, 6, v87\n\t" \
  "v_lshrrev_b32 v217, 5, v88\n\t" \
  "v_lshrrev_b32 v218, 4, v89\n\t" \
  "v_lshrrev_b32 v219, 3, v90\n\t" \
  "v_lshrrev_b32 v220, 2, v91\n\t" \
  "v_lshrrev_b32 v221, 1, v92\n\t" \
  "v_lshlrev_b32 v222, 1, v93\n\t" \
  "v_and_b32 v222, 0x80808080, v222\n\t" \
  "v_and_b32 v87, v31, v87\n\t" \
  "v_and_or_b32 v222, v216, v25, v222\n\t" \
  "v_and_b32 v88, v31, v88\n\t" \
  "v_and_or_b32 v222, v217, v26, v222\n\t" \
  "v_and_b32 v89, v31, v89\n\t" \
  "v_and_or_b32 v222, v218, v27, v222\n\t" \
  "v_and_b32 v90, v31, v90\n\t" \
  "v_and_or_b32 v222, v219, v28, v222\n\t" \
  "v_and_b32 v91, v31, v91\n\t" \
  "v_and_or_b32 v222, v220, v29, v222\n\t" \
  "v_and_b32 v92, v31, v92\n\t" \
  "v_and_or_b32 v222, v221, v30, v222\n\t" \
  "v_and_b32 v93, v31, v93\n\t" \
  "v_xad_u32 v168, v200, s22, v87\n\t" \
  "v_xad_u32 v169, v201, s22, v88\n\t" \
  "v_xad_u32 v170, v202, s22, v89\n\t" \
  "v_xad_u32 v171, v203, s22, v90\n\t" \
  "v_xad_u32 v172, v204, s22, v91\n\t" \
  "v_xad_u32 v173, v205, s22, v92\n\t" \
  "v_xad_u32 v174, v206, s22, v93\n\t" \
  "v_xad_u32 v175, v207, s22, v222\n\t" \
  "L_stA3_%=:\n\t" \
  "buffer_store_dwordx4 v[160:163], v16, s[28:31], s67 offen sc0 sc1\n\t" \
  "buffer_store_dwordx4 v[164:167], v17, s[28:31], s67 offen sc0 sc1\n\t" \
  "buffer_store_dwordx4 v[168:171], v18, s[28:31], s67 offen sc0 sc1\n\t" \
  "buffer_store_dwordx4 v[172:175], v19, s[28:31], s67 offen sc0 sc1\n\t" \
  "v_readfirstlane_b32 s22, v126\n\t" \
  "v_readlane_b32 s75, v126, 1\n\t" \
  "v_lshrrev_b32 v214, 5, v126\n\t" \
  "v_lshrrev_b32 v215, 17, v126\n\t" \
  "v_and_b32 v214, 0x60, v214\n\t" \
  "v_and_b32 v215, 0x60, v215\n\t" \
  "v_xor_b32 v214, s22, v214\n\t" \
  "v_xor_b32 v215, s22, v215\n\t" \
  "v_subrev_u32 v212, s75, v224\n\t" \
  "v_subrev_u32 v213, s75, v225\n\t" \
  "v_and_b32 v212, 0x80808080, v212\n\t" \
  "v_and_b32 v213, 0x80808080, v213\n\t" \
  "v_bcnt_u32_b32 v212, v212, v226\n\t" \
  "v_bcnt_u32_b32 v213, v213, v226\n\t" \
  "v_and_b32 v192, 0x3ff, v126\n\t" \
  "v_bfe_u32 v193, v126, 12, 10\n\t" \
  "v_lshl_add_u32 v212, v212, 10, v192\n\t" \
  "v_lshl_add_u32 v213, v213, 10, v193\n\t" \
  "ds_write_b8 v212, v214\n\t" \
  "ds_write_b8 v213, v215\n\t" \
  "ds_read_b128 v[192:195], v15 offset:0\n\t" \
  "ds_read_b128 v[196:199], v15 offset:1024\n\t" \
  "ds_read_b128 v[200:203], v15 offset:2048\n\t" \
  "ds_read_b128 v[204:207], v15 offset:3072\n\t" \
  "ds_write_b8 v212, v10\n\t" \
  "ds_write_b8 v213, v10\n\t" \
  "s_bitcmp1_b32 s21, 7\n\t" \
  "s_cbranch_scc1 L_fbA4_%=\n\t" \
  "s_waitcnt vmcnt(18) lgkmcnt(8)\n\t" \
  "v_lshrrev_b32 v216, 6, v96\n\t" \
  "v_lshrrev_b32 v217, 5, v97\n\t" \
  "v_lshrrev_b32 v218, 4, v98\n\t" \
  "v_lshrrev_b32 v219, 3, v99\n\t" \
  "v_lshrrev_b32 v220, 2, v100\n\t" \
  "v_lshrrev_b32 v221, 1, v101\n\t" \
  "v_lshlrev_b32 v222, 1, v102\n\t" \
  "v_and_b32 v222, 0x80808080, v222\n\t" \
  "v_and_b32 v96, v31, v96\n\t" \
  "v_and_or_b32 v222, v216, v25, v222\n\t" \
  "v_and_b32 v97, v31, v97\n\t" \
  "v_and_or_b32 v222, v217, v26, v222\n\t" \
  "v_and_b32 v98, v31, v98\n\t" \
  "v_and_or_b32 v222, v218, v27, v222\n\t" \
  "v_and_b32 v99, v31, v99\n\t" \
  "v_and_or_b32 v222, v219, v28, v222\n\t" \
  "v_and_b32 v100, v31, v100\n\t" \
  "v_and_or_b32 v222, v220, v29, v222\n\t" \
  "v_and_b32 v101, v31, v101\n\t" \
  "v_and_or_b32 v222, v221, v30, v222\n\t" \
  "v_and_b32 v102, v31, v102\n\t" \
  "v_xad_u32 v144, v176, s21, v96\n\t" \
  "v_xad_u32 v145, v177, s21, v97\n\t" \
  "v_xad_u32 v146, v178, s21, v98\n\t" \
  "v_xad_u32 v147, v179, s21, v99\n\t" \
  "v_xad_u32 v148, v180, s21, v100\n\t" \
  "v_xad_u32 v149, v181, s21, v101\n\t" \
  "v_xad_u32 v150, v182, s21, v102\n\t" \
  "v_xad_u32 v151, v183, s21, v222\n\t" \
  "v_lshrrev_b32 v216, 6, v103\n\t" \
  "v_lshrrev_b32 v217, 5, v104\n\t" \
  "v_lshrrev_b32 v218, 4, v105\n\t" \
  "v_lshrrev_b32 v219, 3, v106\n\t" \
  "v_lshrrev_b32 v220, 2, v107\n\t" \
  "v_lshrrev_b32 v221, 1, v108\n\t" \
  "v_lshlrev_b32 v222, 1, v109\n\t" \
  "v_and_b32 v222, 0x80808080, v222\n\t" \
  "v_and_b32 v103, v31, v103\n\t" \
  "v_and_or_b32 v222, v216, v25, v222\n\t" \
  "v_and_b32 v104, v31, v104\n\t" \
  "v_and_or_b32 v222, v217, v26, v222\n\t" \
  "v_and_b32 v105, v31, v105\n\t" \
  "v_and_or_b32 v222, v218, v27, v222\n\t" \
  "v_and_b32 v106, v31, v106\n\t" \
  "v_and_or_b32 v222, v219, v28, v222\n\t" \
  "v_and_b32 v107, v31, v107\n\t" \
  "v_and_or_b32 v222, v220, v29, v222\n\t" \
  "v_and_b32 v108, v31, v108\n\t" \
  "v_and_or_b32 v222, v221, v30, v222\n\t" \
  "v_and_b32 v109, v31, v109\n\t" \
  "v_xad_u32 v152, v184, s21, v103\n\t" \
  "v_xad_u32 v153, v185, s21, v104\n\t" \
  "v_xad_u32 v154, v186, s21, v105\n\t" \
  "v_xad_u32 v155, v187, s21, v106\n\t" \
  "v_xad_u32 v156, v188, s21, v107\n\t" \
  "v_xad_u32 v157, v189, s21, v108\n\t" \
  "v_xad_u32 v158, v190, s21, v109\n\t" \
  "v_xad_u32 v159, v191, s21, v222\n\t" \
  "L_stA4_%=:\n\t" \
  "buffer_store_dwordx4 v[144:147], v16, s[28:31], s68 offen sc0 sc1\n\t" \
  "buffer_store_dwordx4 v[148:151], v17, s[28:31], s68 offen sc0 sc1\n\t" \
  "buffer_store_dwordx4 v[152:155], v18, s[28:31], s68 offen sc0 sc1\n\t" \
  "buffer_store_dwordx4 v[156:159], v19, s[28:31], s68 offen sc0 sc1\n\t" \
  "v_readfirstlane_b32 s21, v142\n\t" \
  "v_readlane_b32 s75, v142, 1\n\t" \
  "v_lshrrev_b32 v210, 5, v142\n\t" \
  "v_lshrrev_b32 v211, 17, v142\n\t" \
  "v_and_b32 v210, 0x60, v210\n\t" \
  "v_and_b32 v211, 0x60, v211\n\t" \
  "v_xor_b32 v210, s21, v210\n\t" \
  "v_xor_b32 v211, s21, v211\n\t" \
  "v_subrev_u32 v208, s75, v224\n\t" \
  "v_subrev_u32 v209, s75, v225\n\t" \
  "v_and_b32 v208, 0x80808080, v208\n\t" \
  "v_and_b32 v209, 0x80808080, v209\n\t" \
  "v_bcnt_u32_b32 v208, v208, v226\n\t" \
  "v_bcnt_u32_b32 v209, v209, v226\n\t" \
  "v_and_b32 v176, 0x3ff, v142\n\t" \
  "v_bfe_u32 v177, v142, 12, 10\n\t" \
  "v_lshl_add_u32 v208, v208, 10, v176\n\t" \
  "v_lshl_add_u32 v209, v209, 10, v177\n\t" \
  "ds_write_b8 v208, v210\n\t" \
  "ds_write_b8 v209, v211\n\t" \
  "ds_read_b128 v[176:179], v15 offset:0\n\t" \
  "ds_read_b128 v[180:183], v15 offset:1024\n\t" \
  "ds_read_b128 v[184:187], v15 offset:2048\n\t" \
  "ds_read_b128 v[188:191], v15 offset:3072\n\t" \
  "ds_write_b8 v208, v10\n\t" \
  "ds_write_b8 v209, v10\n\t" \
  "s_bitcmp1_b32 s22, 7\n\t" \
  "s_cbranch_scc1 L_fbA5_%=\n\t" \
  "s_waitcnt vmcnt(21) lgkmcnt(8)\n\t" \
  "v_lshrrev_b32 v216, 6, v112\n\t" \
  "v_lshrrev_b32 v217, 5, v113\n\t" \
  "v_lshrrev_b32 v218, 4, v114\n\t" \
  "v_lshrrev_b32 v219, 3, v115\n\t" \
  "v_lshrrev_b32 v220, 2, v116\n\t" \
  "v_lshrrev_b32 v221, 1, v117\n\t" \
  "v_lshlrev_b32 v222, 1, v118\n\t" \
  "v_and_b32 v222, 0x80808080, v222\n\t" \
  "v_and_b32 v112, v31, v112\n\t" \
  "v_and_or_b32 v222, v216, v25, v222\n\t" \
  "v_and_b32 v113, v31, v113\n\t" \
  "v_and_or_b32 v222, v217, v26, v222\n\t" \
  "v_and_b32 v114, v31, v114\n\t" \
  "v_and_or_b32 v222, v218, v27, v222\n\t" \
  "v_and_b32 v115, v31, v115\n\t" \
  "v_and_or_b32 v222, v219, v28, v222\n\t" \
  "v_and_b32 v116, v31, v116\n\t" \
  "v_and_or_b32 v222, v220, v29, v222\n\t" \
  "v_and_b32 v117, v31, v117\n\t" \
  "v_and_or_b32 v222, v221, v30, v222\n\t" \
  "v_and_b32 v118, v31, v118\n\t" \
  "v_xad_u32 v160, v192, s22, v112\n\t" \
  "v_xad_u32 v161, v193, s22, v113\n\t" \
  "v_xad_u32 v162, v194, s22, v114\n\t" \
  "v_xad_u32 v163, v195, s22, v115\n\t" \
  "v_xad_u32 v164, v196, s22, v116\n\t" \
  "v_xad_u32 v165, v197, s22, v117\n\t" \
  "v_xad_u32 v166, v198, s22, v118\n\t" \
  "v_xad_u32 v167, v199, s22, v222\n\t" \
  "v_lshrrev_b32 v216, 6, v119\n\t" \
  "v_lshrrev_b32 v217, 5, v120\n\t" \
  "v_lshrrev_b32 v218, 4, v121\n\t" \
  "v_lshrrev_b32 v219, 3, v122\n\t" \
  "v_lshrrev_b32 v220, 2, v123\n\t" \
  "v_lshrrev_b32 v221, 1, v124\n\t" \
  "v_lshlrev_b32 v222, 1, v125\n\t" \
  "v_and_b32 v222, 0x80808080, v222\n\t" \
  "v_and_b32 v119, v31, v119\n\t" \
  "v_and_or_b32 v222, v216, v25, v222\n\t" \
  "v_and_b32 v120, v31, v120\n\t" \
  "v_and_or_b32 v222, v217, v26, v222\n\t" \
  "v_and_b32 v121, v31, v121\n\t" \
  "v_and_or_b32 v222, v218, v27, v222\n\t" \
  "v_and_b32 v122, v31, v122\n\t" \
  "v_and_or_b32 v222, v219, v28, v222\n\t" \
  "v_and_b32 v123, v31, v123\n\t" \
  "v_and_or_b32 v222, v220, v29, v222\n\t" \
  "v_and_b32 v124, v31, v124\n\t" \
  "v_and_or_b32 v222, v221, v30, v222\n\t" \
  "v_and_b32 v125, v31, v125\n\t" \
  "v_xad_u32 v168, v200, s22, v119\n\t" \
  "v_xad_u32 v169, v201, s22, v120\n\t" \
  "v_xad_u32 v170, v202, s22, v121\n\t" \
  "v_xad_u32 v171, v203, s22, v122\n\t" \
  "v_xad_u32 v172, v204, s22, v123\n\t" \
  "v_xad_u32 v173, v205, s22, v124\n\t" \
  "v_xad_u32 v174, v206, s22, v125\n\t" \
  "v_xad_u32 v175, v207, s22, v222\n\t" \
  "L_stA5_%=:\n\t" \
  "buffer_store_dwordx4 v[160:163], v16, s[28:31], s69 offen sc0 sc1\n\t" \
  "buffer_store_dwordx4 v[164:167], v17, s[28:31], s69 offen sc0 sc1\n\t" \
  "buffer_store_dwordx4 v[168:171], v18, s[28:31], s69 offen sc0 sc1\n\t" \
  "buffer_store_dwordx4 v[172:175], v19, s[28:31], s69 offen sc0 sc1\n\t" \
  "s_bitcmp1_b32 s21, 7\n\t" \
  "s_cbranch_scc1 L_fbA6_%=\n\t" \
  "s_waitcnt vmcnt(24) lgkmcnt(1)\n\t" \
  "v_lshrrev_b32 v216, 6, v128\n\t" \
  "v_lshrrev_b32 v217, 5, v129\n\t" \
  "v_lshrrev_b32 v218, 4, v130\n\t" \
  "v_lshrrev_b32 v219, 3, v131\n\t" \
  "v_lshrrev_b32 v220, 2, v132\n\t" \
  "v_lshrrev_b32 v221, 1, v133\n\t" \
  "v_lshlrev_b32 v222, 1, v134\n\t" \
  "v_and_b32 v222, 0x80808080, v222\n\t" \
  "v_and_b32 v128, v31, v128\n\t" \
  "v_and_or_b32 v222, v216, v25, v222\n\t" \
  "v_and_b32 v129, v31, v129\n\t" \
  "v_and_or_b32 v222, v217, v26, v222\n\t" \
  "v_and_b32 v130, v31, v130\n\t" \
  "v_and_or_b32 v222, v218, v27, v222\n\t" \
  "v_and_b32 v131, v31, v131\n\t" \
  "v_and_or_b32 v222, v219, v28, v222\n\t" \
  "v_and_b32 v132, v31, v132\n\t" \
  "v_and_or_b32 v222, v220, v29, v222\n\t" \
  "v_and_b32 v133, v31, v133\n\t" \
  "v_and_or_b32 v222, v221, v30, v222\n\t" \
  "v_and_b32 v134, v31, v134\n\t" \
  "v_xad_u32 v144, v176, s21, v128\n\t" \
  "v_xad_u32 v145, v177, s21, v129\n\t" \
  "v_xad_u32 v146, v178, s21, v130\n\t" \
  "v_xad_u32 v147, v179, s21, v131\n\t" \
  "v_xad_u32 v148, v180, s21, v132\n\t" \
  "v_xad_u32 v149, v181, s21, v133\n\t" \
  "v_xad_u32 v150, v182, s21, v134\n\t" \
  "v_xad_u32 v151, v183, s21, v222\n\t" \
  "v_lshrrev_b32 v216, 6, v135\n\t" \
  "v_lshrrev_b32 v217, 5, v136\n\t" \
  "v_lshrrev_b32 v218, 4, v137\n\t" \
  "v_lshrrev_b32 v219, 3, v138\n\t" \
  "v_lshrrev_b32 v220, 2, v139\n\t" \
  "v_lshrrev_b32 v221, 1, v140\n\t" \
  "v_lshlrev_b32 v222, 1, v141\n\t" \
  "v_and_b32 v222, 0x80808080, v222\n\t" \
  "v_and_b32 v135, v31, v135\n\t" \
  "v_and_or_b32 v222, v216, v25, v222\n\t" \
  "v_and_b32 v136, v31, v136\n\t" \
  "v_and_or_b32 v222, v217, v26, v222\n\t" \
  "v_and_b32 v137, v31, v137\n\t" \
  "v_and_or_b32 v222, v218, v27, v222\n\t" \
  "v_and_b32 v138, v31, v138\n\t" \
  "v_and_or_b32 v222, v219, v28, v222\n\t" \
  "v_and_b32 v139, v31, v139\n\t" \
  "v_and_or_b32 v222, v220, v29, v222\n\t" \
  "v_and_b32 v140, v31, v140\n\t" \
  "v_and_or_b32 v222, v221, v30, v222\n\t" \
  "v_and_b32 v141, v31, v141\n\t" \
  "v_xad_u32 v152, v184, s21, v135\n\t" \
  "v_xad_u32 v153, v185, s21, v136\n\t" \
  "v_xad_u32 v154, v186, s21, v137\n\t" \
  "v_xad_u32 v155, v187, s21, v138\n\t" \
  "v_xad_u32 v156, v188, s21, v139\n\t" \
  "v_xad_u32 v157, v189, s21, v140\n\t" \
  "v_xad_u32 v158, v190, s21, v141\n\t" \
  "v_xad_u32 v159, v191, s21, v222\n\t" \
  "L_stA6_%=:\n\t" \
  "buffer_store_dwordx4 v[144:147], v16, s[28:31], s70 offen sc0 sc1\n\t" \
  "buffer_store_dwordx4 v[148:151], v17, s[28:31], s70 offen sc0 sc1\n\t" \
  "buffer_store_dwordx4 v[152:155], v18, s[28:31], s70 offen sc0 sc1\n\t" \
  "buffer_store_dwordx4 v[156:159], v19, s[28:31], s70 offen sc0 sc1\n\t" \
  "s_branch L_done_%=\n\t" \
  "L_fbA0_%=:\n\t" \
  "buffer_load_dwordx4 v[144:147], v16, s[36:39], s64 offen sc0 sc1\n\t" \
  "buffer_load_dwordx4 v[148:151], v17, s[36:39], s64 offen sc0 sc1\n\t" \
  "buffer_load_dwordx4 v[152:155], v18, s[36:39], s64 offen sc0 sc1\n\t" \
  "buffer_load_dwordx4 v[156:159], v19, s[36:39], s64 offen sc0 sc1\n\t" \
  "s_waitcnt vmcnt(0) lgkmcnt(0)\n\t" \
  "s_branch L_stA0_%=\n\t" \
  "L_fbA1_%=:\n\t" \
  "buffer_load_dwordx4 v[160:163], v16, s[40:43], s65 offen sc0 sc1\n\t" \
  "buffer_load_dwordx4 v[164:167], v17, s[40:43], s65 offen sc0 sc1\n\t" \
  "buffer_load_dwordx4 v[168:171], v18, s[40:43], s65 offen sc0 sc1\n\t" \
  "buffer_load_dwordx4 v[172:175], v19, s[40:43], s65 offen sc0 sc1\n\t" \
  "s_waitcnt vmcnt(0) lgkmcnt(0)\n\t" \
  "s_branch L_stA1_%=\n\t" \
  "L_fbA2_%=:\n\t" \
  "buffer_load_dwordx4 v[144:147], v16, s[44:47], s66 offen sc0 sc1\n\t" \
  "buffer_load_dwordx4 v[148:151], v17, s[44:47], s66 offen sc0 sc1\n\t" \
  "buffer_load_dwordx4 v[152:155], v18, s[44:47], s66 offen sc0 sc1\n\t" \
  "buffer_load_dwordx4 v[156:159], v19, s[44:47], s66 offen sc0 sc1\n\t" \
  "s_waitcnt vmcnt(0) lgkmcnt(0)\n\t" \
  "s_branch L_stA2_%=\n\t" \
  "L_fbA3_%=:\n\t" \
  "buffer_load_dwordx4 v[160:163], v16, s[48:51], s67 offen sc0 sc1\n\t" \
  "buffer_load_dwordx4 v[164:167], v17, s[48:51], s67 offen sc0 sc1\n\t" \
  "buffer_load_dwordx4 v[168:171], v18, s[48:51], s67 offen sc0 sc1\n\t" \
  "buffer_load_dwordx4 v[172:175], v19, s[48:51], s67 offen sc0 sc1\n\t" \
  "s_waitcnt vmcnt(0) lgkmcnt(0)\n\t" \
  "s_branch L_stA3_%=\n\t" \
  "L_fbA4_%=:\n\t" \
  "buffer_load_dwordx4 v[144:147], v16, s[52:55], s68 offen sc0 sc1\n\t" \
  "buffer_load_dwordx4 v[148:151], v17, s[52:55], s68 offen sc0 sc1\n\t" \
  "buffer_load_dwordx4 v[152:155], v18, s[52:55], s68 offen sc0 sc1\n\t" \
  "buffer_load_dwordx4 v[156:159], v19, s[52:55], s68 offen sc0 sc1\n\t" \
  "s_waitcnt vmcnt(0) lgkmcnt(0)\n\t" \
  "s_branch L_stA4_%=\n\t" \
  "L_fbA5_%=:\n\t" \
  "buffer_load_dwordx4 v[160:163], v16, s[56:59], s69 offen sc0 sc1\n\t" \
  "buffer_load_dwordx4 v[164:167], v17, s[56:59], s69 offen sc0 sc1\n\t" \
  "buffer_load_dwordx4 v[168:171], v18, s[56:59], s69 offen sc0 sc1\n\t" \
  "buffer_load_dwordx4 v[172:175], v19, s[56:59], s69 offen sc0 sc1\n\t" \
  "s_waitcnt vmcnt(0) lgkmcnt(0)\n\t" \
  "s_branch L_stA5_%=\n\t" \
  "L_fbA6_%=:\n\t" \
  "buffer_load_dwordx4 v[144:147], v16, s[60:63], s70 offen sc0 sc1\n\t" \
  "buffer_load_dwordx4 v[148:151], v17, s[60:63], s70 offen sc0 sc1\n\t" \
  "buffer_load_dwordx4 v[152:155], v18, s[60:63], s70 offen sc0 sc1\n\t" \
  "buffer_load_dwordx4 v[156:159], v19, s[60:63], s70 offen sc0 sc1\n\t" \
  "s_waitcnt vmcnt(0) lgkmcnt(0)\n\t" \
  "s_branch L_stA6_%=\n\t" \
  "L_pO_%=:\n\t" \
  "buffer_load_dwordx2 v[44:45], v24, s[36:39], s93 offen offset:3072 sc0 sc1\n\t" \
  "buffer_load_dwordx2 v[60:61], v24, s[40:43], s93 offen offset:3072 sc0 sc1\n\t" \
  "buffer_load_dwordx2 v[76:77], v24, s[44:47], s93 offen offset:3072 sc0 sc1\n\t" \
  "buffer_load_dwordx2 v[92:93], v24, s[48:51], s93 offen offset:3072 sc0 sc1\n\t" \
  "buffer_load_dwordx2 v[108:109], v24, s[52:55], s93 offen offset:3072 sc0 sc1\n\t" \
  "buffer_load_dwordx2 v[124:125], v24, s[56:59], s93 offen offset:3072 sc0 sc1\n\t" \
  "buffer_load_dwordx2 v[140:141], v24, s[60:63], s93 offen offset:3072 sc0 sc1\n\t" \
  "buffer_load_dwordx4 v[32:35], v8, s[36:39], s93 offen offset:0 sc0 sc1\n\t" \
  "buffer_load_dwordx4 v[48:51], v8, s[40:43], s93 offen offset:0 sc0 sc1\n\t" \
  "buffer_load_dwordx4 v[64:67], v8, s[44:47], s93 offen offset:0 sc0 sc1\n\t" \
  "buffer_load_dwordx4 v[80:83], v8, s[48:51], s93 offen offset:0 sc0 sc1\n\t" \
  "buffer_load_dwordx4 v[96:99], v8, s[52:55], s93 offen offset:0 sc0 sc1\n\t" \
  "buffer_load_dwordx4 v[112:115], v8, s[56:59], s93 offen offset:0 sc0 sc1\n\t" \
  "buffer_load_dwordx4 v[128:131], v8, s[60:63], s93 offen offset:0 sc0 sc1\n\t" \
  "buffer_load_dwordx4 v[36:39], v8, s[36:39], s93 offen offset:1024 sc0 sc1\n\t" \
  "buffer_load_dwordx4 v[52:55], v8, s[40:43], s93 offen offset:1024 sc0 sc1\n\t" \
  "buffer_load_dwordx4 v[68:71], v8, s[44:47], s93 offen offset:1024 sc0 sc1\n\t" \
  "buffer_load_dwordx4 v[84:87], v8, s[48:51], s93 offen offset:1024 sc0 sc1\n\t" \
  "buffer_load_dwordx4 v[100:103], v8, s[52:55], s93 offen offset:1024 sc0 sc1\n\t" \
  "s_waitcnt vmcnt(18)\n\t" \
  "buffer_load_dwordx4 v[116:119], v8, s[56:59], s93 offen offset:1024 sc0 sc1\n\t" \
  "s_waitcnt vmcnt(18)\n\t" \
  "buffer_load_dwordx4 v[132:135], v8, s[60:63], s93 offen offset:1024 sc0 sc1\n\t" \
  "s_waitcnt vmcnt(18)\n\t" \
  "buffer_load_dwordx4 v[40:43], v8, s[36:39], s93 offen offset:2048 sc0 sc1\n\t" \
  "s_waitcnt vmcnt(18)\n\t" \
  "buffer_load_dwordx4 v[56:59], v8, s[40:43], s93 offen offset:2048 sc0 sc1\n\t" \
  "s_waitcnt vmcnt(18)\n\t" \
  "buffer_load_dwordx4 v[72:75], v8, s[44:47], s93 offen offset:2048 sc0 sc1\n\t" \
  "s_waitcnt vmcnt(18)\n\t" \
  "buffer_load_dwordx4 v[88:91], v8, s[48:51], s93 offen offset:2048 sc0 sc1\n\t" \
  "s_waitcnt vmcnt(18)\n\t" \
  "buffer_load_dwordx4 v[104:107], v8, s[52:55], s93 offen offset:2048 sc0 sc1\n\t" \
  "s_waitcnt vmcnt(18)\n\t" \
  "buffer_load_dwordx4 v[120:123], v8, s[56:59], s93 offen offset:2048 sc0 sc1\n\t" \
  "s_waitcnt vmcnt(18)\n\t" \
  "buffer_load_dwordx4 v[136:139], v8, s[60:63], s93 offen offset:2048 sc0 sc1\n\t" \
  "s_waitcnt vmcnt(21)\n\t" \
  "v_readfirstlane_b32 s21, v45\n\t" \
  "v_bfe_u32 v208, v45, 16, 11\n\t" \
  "v_lshrrev_b32 v210, 25, v45\n\t" \
  "v_add_u32 v208, s99, v208\n\t" \
  "v_and_b32 v210, 0x7c, v210\n\t" \
  "ds_write_b8 v208, v210\n\t" \
  "ds_read_b128 v[176:179], v15\n\t" \
  "ds_write_b8 v208, v10\n\t" \
  "s_waitcnt vmcnt(14)\n\t" \
  "s_mov_b64 exec, 1\n\t" \
  "ds_add_u32 v10, v13\n\t" \
  "s_mov_b64 exec, -1\n\t" \
  "v_readfirstlane_b32 s22, v61\n\t" \
  "v_bfe_u32 v212, v61, 16, 11\n\t" \
  "v_lshrrev_b32 v214, 25, v61\n\t" \
  "v_add_u32 v212, s99, v212\n\t" \
  "v_and_b32 v214, 0x7c, v214\n\t" \
  "ds_write_b8 v212, v214\n\t" \
  "ds_read_b128 v[192:195], v15\n\t" \
  "ds_write_b8 v212, v10\n\t" \
  "s_lshr_b32 s75, s21, 16\n\t" \
  "s_and_b32 s94, s75, 0x7f\n\t" \
  "s_mul_i32 s94, s94, 0x01010101\n\t" \
  "s_bitcmp1_b32 s75, 7\n\t" \
  "s_cbranch_scc1 L_fbO0_%=\n\t" \
  "s_waitcnt vmcnt(6) lgkmcnt(3)\n\t" \
  "v_and_b32 v223, 0xffff, v45\n\t" \
  "v_lshl_or_b32 v223, v223, 12, v223\n\t" \
  "v_lshrrev_b32 v216, 0, v44\n\t" \
  "v_lshlrev_b32 v217, 7, v223\n\t" \
  "v_and_or_b32 v216, v216, s97, v176\n\t" \
  "v_perm_b32 v218, v33, v32, v28\n\t" \
  "v_add_u32 v216, s94, v216\n\t" \
  "v_bfi_b32 v216, s96, v216, v217\n\t" \
  "v_perm_b32 v144, v216, v32, v25\n\t" \
  "v_perm_b32 v145, v216, v33, v26\n\t" \
  "v_perm_b32 v146, v216, v34, v27\n\t" \
  "v_perm_b32 v219, v216, v34, v28\n\t" \
  "v_lshl_or_b32 v147, v219, 16, v218\n\t" \
  "v_lshrrev_b32 v216, 2, v44\n\t" \
  "v_lshlrev_b32 v217, 6, v223\n\t" \
  "v_and_or_b32 v216, v216, s97, v177\n\t" \
  "v_perm_b32 v218, v36, v35, v28\n\t" \
  "v_add_u32 v216, s94, v216\n\t" \
  "v_bfi_b32 v216, s96, v216, v217\n\t" \
  "v_perm_b32 v148, v216, v35, v25\n\t" \
  "v_perm_b32 v149, v216, v36, v26\n\t" \
  "v_perm_b32 v150, v216, v37, v27\n\t" \
  "v_perm_b32 v219, v216, v37, v28\n\t" \
  "v_lshl_or_b32 v151, v219, 16, v218\n\t" \
  "v_lshrrev_b32 v216, 4, v44\n\t" \
  "v_lshlrev_b32 v217, 5, v223\n\t" \
  "v_and_or_b32 v216, v216, s97, v178\n\t" \
  "v_perm_b32 v218, v39, v38, v28\n\t" \
  "v_add_u32 v216, s94, v216\n\t" \
  "v_bfi_b32 v216, s96, v216, v217\n\t" \
  "v_perm_b32 v152, v216, v38, v25\n\t" \
  "v_perm_b32 v153, v216, v39, v26\n\t" \
  "v_perm_b32 v154, v216, v40, v27\n\t" \
  "v_perm_b32 v219, v216, v40, v28\n\t" \
  "v_lshl_or_b32 v155, v219, 16, v218\n\t" \
  "v_lshrrev_b32 v216, 6, v44\n\t" \
  "v_lshlrev_b32 v217, 4, v223\n\t" \
  "v_and_or_b32 v216, v216, s97, v179\n\t" \
  "v_perm_b32 v218, v42, v41, v28\n\t" \
  "v_add_u32 v216, s94, v216\n\t" \
  "v_bfi_b32 v216, s96, v216, v217\n\t" \
  "v_perm_b32 v156, v216, v41, v25\n\t" \
  "v_perm_b32 v157, v216, v42, v26\n\t" \
  "v_perm_b32 v158, v216, v43, v27\n\t" \
  "v_perm_b32 v219, v216, v43, v28\n\t" \
  "v_lshl_or_b32 v159, v219, 16, v218\n\t" \
  "L_stO0_%=:\n\t" \
  "buffer_store_dwordx4 v[144:147], v16, s[28:31], s64 offen nt\n\t" \
  "buffer_store_dwordx4 v[148:151], v17, s[28:31], s64 offen nt\n\t" \
  "buffer_store_dwordx4 v[152:155], v18, s[28:31], s64 offen nt\n\t" \
  "buffer_store_dwordx4 v[156:159], v19, s[28:31], s64 offen nt\n\t" \
  "v_readfirstlane_b32 s21, v77\n\t" \
  "v_bfe_u32 v208, v77, 16, 11\n\t" \
  "v_lshrrev_b32 v210, 25, v77\n\t" \
  "v_add_u32 v208, s99, v208\n\t" \
  "v_and_b32 v210, 0x7c, v210\n\t" \
  "ds_write_b8 v208, v210\n\t" \
  "ds_read_b128 v[176:179], v15\n\t" \
  "ds_write_b8 v208, v10\n\t" \
  "s_lshr_b32 s75, s22, 16\n\t" \
  "s_and_b32 s94, s75, 0x7f\n\t" \
  "s_mul_i32 s94, s94, 0x01010101\n\t" \
  "s_bitcmp1_b32 s75, 7\n\t" \
  "s_cbranch_scc1 L_fbO1_%=\n\t" \
  "s_waitcnt vmcnt(9) lgkmcnt(3)\n\t" \
  "v_and_b32 v223, 0xffff, v61\n\t" \
  "v_lshl_or_b32 v223, v223, 12, v223\n\t" \
  "v_lshrrev_b32 v216, 0, v60\n\t" \
  "v_lshlrev_b32 v217, 7, v223\n\t" \
  "v_and_or_b32 v216, v216, s97, v192\n\t" \
  "v_perm_b32 v218, v49, v48, v28\n\t" \
  "v_add_u32 v216, s94, v216\n\t" \
  "v_bfi_b32 v216, s96, v216, v217\n\t" \
  "v_perm_b32 v160, v216, v48, v25\n\t" \
  "v_perm_b32 v161, v216, v49, v26\n\t" \
  "v_perm_b32 v162, v216, v50, v27\n\t" \
  "v_perm_b32 v219, v216, v50, v28\n\t" \
  "v_lshl_or_b32 v163, v219, 16, v218\n\t" \
  "v_lshrrev_b32 v216, 2, v60\n\t" \
  "v_lshlrev_b32 v217, 6, v223\n\t" \
  "v_and_or_b32 v216, v216, s97, v193\n\t" \
  "v_perm_b32 v218, v52, v51, v28\n\t" \
  "v_add_u32 v216, s94, v216\n\t" \
  "v_bfi_b32 v216, s96, v216, v217\n\t" \
  "v_perm_b32 v164, v216, v51, v25\n\t" \
  "v_perm_b32 v165, v216, v52, v26\n\t" \
  "v_perm_b32 v166, v216, v53, v27\n\t" \
  "v_perm_b32 v219, v216, v53, v28\n\t" \
  "v_lshl_or_b32 v167, v219, 16, v218\n\t" \
  "v_lshrrev_b32 v216, 4, v60\n\t" \
  "v_lshlrev_b32 v217, 5, v223\n\t" \
  "v_and_or_b32 v216, v216, s97, v194\n\t" \
  "v_perm_b32 v218, v55, v54, v28\n\t" \
  "v_add_u32 v216, s94, v216\n\t" \
  "v_bfi_b32 v216, s96, v216, v217\n\t" \
  "v_perm_b32 v168, v216, v54, v25\n\t" \
  "v_perm_b32 v169, v216, v55, v26\n\t" \
  "v_perm_b32 v170, v216, v56, v27\n\t" \
  "v_perm_b32 v219, v216, v56, v28\n\t" \
  "v_lshl_or_b32 v171, v219, 16, v218\n\t" \
  "v_lshrrev_b32 v216, 6, v60\n\t" \
  "v_lshlrev_b32 v217, 4, v223\n\t" \
  "v_and_or_b32 v216, v216, s97, v195\n\t" \
  "v_perm_b32 v218, v58, v57, v28\n\t" \
  "v_add_u32 v216, s94, v216\n\t" \
  "v_bfi_b32 v216, s96, v216, v217\n\t" \
  "v_perm_b32 v172, v216, v57, v25\n\t" \
  "v_perm_b32 v173, v216, v58, v26\n\t" \
  "v_perm_b32 v174, v216, v59, v27\n\t" \
  "v_perm_b32 v219, v216, v59, v28\n\t" \
  "v_lshl_or_b32 v175, v219, 16, v218\n\t" \
  "L_stO1_%=:\n\t" \
  "buffer_store_dwordx4 v[160:163], v16, s[28:31], s65 offen nt\n\t" \
  "buffer_store_dwordx4 v[164:167], v17, s[28:31], s65 offen nt\n\t" \
  "buffer_store_dwordx4 v[168:171], v18, s[28:31], s65 offen nt\n\t" \
  "buffer_store_dwordx4 v[172:175], v19, s[28:31], s65 offen nt\n\t" \
  "v_readfirstlane_b32 s22, v93\n\t" \
  "v_bfe_u32 v212, v93, 16, 11\n\t" \
  "v_lshrrev_b32 v214, 25, v93\n\t" \
  "v_add_u32 v212, s99, v212\n\t" \
  "v_and_b32 v214, 0x7c, v214\n\t" \
  "ds_write_b8 v212, v214\n\t" \
  "ds_read_b128 v[192:195], v15\n\t" \
  "ds_write_b8 v212, v10\n\t" \
  "s_lshr_b32 s75, s21, 16\n\t" \
  "s_and_b32 s94, s75, 0x7f\n\t" \
  "s_mul_i32 s94, s94, 0x01010101\n\t" \
  "s_bitcmp1_b32 s75, 7\n\t" \
  "s_cbranch_scc1 L_fbO2_%=\n\t" \
  "s_waitcnt vmcnt(12) lgkmcnt(3)\n\t" \
  "v_and_b32 v223, 0xffff, v77\n\t" \
  "v_lshl_or_b32 v223, v223, 12, v223\n\t" \
  "v_lshrrev_b32 v216, 0, v76\n\t" \
  "v_lshlrev_b32 v217, 7, v223\n\t" \
  "v_and_or_b32 v216, v216, s97, v176\n\t" \
  "v_perm_b32 v218, v65, v64, v28\n\t" \
  "v_add_u32 v216, s94, v216\n\t" \
  "v_bfi_b32 v216, s96, v216, v217\n\t" \
  "v_perm_b32 v144, v216, v64, v25\n\t" \
  "v_perm_b32 v145, v216, v65, v26\n\t" \
  "v_perm_b32 v146, v216, v66, v27\n\t" \
  "v_perm_b32 v219, v216, v66, v28\n\t" \
  "v_lshl_or_b32 v147, v219, 16, v218\n\t" \
  "v_lshrrev_b32 v216, 2, v76\n\t" \
  "v_lshlrev_b32 v217, 6, v223\n\t" \
  "v_and_or_b32 v216, v216, s97, v177\n\t" \
  "v_perm_b32 v218, v68, v67, v28\n\t" \
  "v_add_u32 v216, s94, v216\n\t" \
  "v_bfi_b32 v216, s96, v216, v217\n\t" \
  "v_perm_b32 v148, v216, v67, v25\n\t" \
  "v_perm_b32 v149, v216, v68, v26\n\t" \
  "v_perm_b32 v150, v216, v69, v27\n\t" \
  "v_perm_b32 v219, v216, v69, v28\n\t" \
  "v_lshl_or_b32 v151, v219, 16, v218\n\t" \
  "v_lshrrev_b32 v216, 4, v76\n\t" \
  "v_lshlrev_b32 v217, 5, v223\n\t" \
  "v_and_or_b32 v216, v216, s97, v178\n\t" \
  "v_perm_b32 v218, v71, v70, v28\n\t" \
  "v_add_u32 v216, s94, v216\n\t" \
  "v_bfi_b32 v216, s96, v216, v217\n\t" \
  "v_perm_b32 v152, v216, v70, v25\n\t" \
  "v_perm_b32 v153, v216, v71, v26\n\t" \
  "v_perm_b32 v154, v216, v72, v27\n\t" \
  "v_perm_b32 v219, v216, v72, v28\n\t" \
  "v_lshl_or_b32 v155, v219, 16, v218\n\t" \
  "v_lshrrev_b32 v216, 6, v76\n\t" \
  "v_lshlrev_b32 v217, 4, v223\n\t" \
  "v_and_or_b32 v216, v216, s97, v179\n\t" \
  "v_perm_b32 v218, v74, v73, v28\n\t" \
  "v_add_u32 v216, s94, v216\n\t" \
  "v_bfi_b32 v216, s96, v216, v217\n\t" \
  "v_perm_b32 v156, v216, v73, v25\n\t" \
  "v_perm_b32 v157, v216, v74, v26\n\t" \
  "v_perm_b32 v158, v216, v75, v27\n\t" \
  "v_perm_b32 v219, v216, v75, v28\n\t" \
  "v_lshl_or_b32 v159, v219, 16, v218\n\t" \
  "L_stO2_%=:\n\t" \
  "buffer_store_dwordx4 v[144:147], v16, s[28:31], s66 offen nt\n\t" \
  "buffer_store_dwordx4 v[148:151], v17, s[28:31], s66 offen nt\n\t" \
  "buffer_store_dwordx4 v[152:155], v18, s[28:31], s66 offen nt\n\t" \
  "buffer_store_dwordx4 v[156:159], v19, s[28:31], s66 offen nt\n\t" \
  "v_readfirstlane_b32 s21, v109\n\t" \
  "v_bfe_u32 v208, v109, 16, 11\n\t" \
  "v_lshrrev_b32 v210, 25, v109\n\t" \
  "v_add_u32 v208, s99, v208\n\t" \
  "v_and_b32 v210, 0x7c, v210\n\t" \
  "ds_write_b8 v208, v210\n\t" \
  "ds_read_b128 v[176:179], v15\n\t" \
  "ds_write_b8 v208, v10\n\t" \
  "s_lshr_b32 s75, s22, 16\n\t" \
  "s_and_b32 s94, s75, 0x7f\n\t" \
  "s_mul_i32 s94, s94, 0x01010101\n\t" \
  "s_bitcmp1_b32 s75, 7\n\t" \
  "s_cbranch_scc1 L_fbO3_%=\n\t" \
  "s_waitcnt vmcnt(15) lgkmcnt(3)\n\t" \
  "v_and_b32 v223, 0xffff, v93\n\t" \
  "v_lshl_or_b32 v223, v223, 12, v223\n\t" \
  "v_lshrrev_b32 v216, 0, v92\n\t" \
  "v_lshlrev_b32 v217, 7, v223\n\t" \
  "v_and_or_b32 v216, v216, s97, v192\n\t" \
  "v_perm_b32 v218, v81, v80, v28\n\t" \
  "v_add_u32 v216, s94, v216\n\t" \
  "v_bfi_b32 v216, s96, v216, v217\n\t" \
  "v_perm_b32 v160, v216, v80, v25\n\t" \
  "v_perm_b32 v161, v216, v81, v26\n\t" \
  "v_perm_b32 v162, v216, v82, v27\n\t" \
  "v_perm_b32 v219, v216, v82, v28\n\t" \
  "v_lshl_or_b32 v163, v219, 16, v218\n\t" \
  "v_lshrrev_b32 v216, 2, v92\n\t" \
  "v_lshlrev_b32 v217, 6, v223\n\t" \
  "v_and_or_b32 v216, v216, s97, v193\n\t" \
  "v_perm_b32 v218, v84, v83, v28\n\t" \
  "v_add_u32 v216, s94, v216\n\t" \
  "v_bfi_b32 v216, s96, v216, v217\n\t" \
  "v_perm_b32 v164, v216, v83, v25\n\t" \
  "v_perm_b32 v165, v216, v84, v26\n\t" \
  "v_perm_b32 v166, v216, v85, v27\n\t" \
  "v_perm_b32 v219, v216, v85, v28\n\t" \
  "v_lshl_or_b32 v167, v219, 16, v218\n\t" \
  "v_lshrrev_b32 v216, 4, v92\n\t" \
  "v_lshlrev_b32 v217, 5, v223\n\t" \
  "v_and_or_b32 v216, v216, s97, v194\n\t" \
  "v_perm_b32 v218, v87, v86, v28\n\t" \
  "v_add_u32 v216, s94, v216\n\t" \
  "v_bfi_b32 v216, s96, v216, v217\n\t" \
  "v_perm_b32 v168, v216, v86, v25\n\t" \
  "v_perm_b32 v169, v216, v87, v26\n\t" \
  "v_perm_b32 v170, v216, v88, v27\n\t" \
  "v_perm_b32 v219, v216, v88, v28\n\t" \
  "v_lshl_or_b32 v171, v219, 16, v218\n\t" \
  "v_lshrrev_b32 v216, 6, v92\n\t" \
  "v_lshlrev_b32 v217, 4, v223\n\t" \
  "v_and_or_b32 v216, v216, s97, v195\n\t" \
  "v_perm_b32 v218, v90, v89, v28\n\t" \
  "v_add_u32 v216, s94, v216\n\t" \
  "v_bfi_b32 v216, s96, v216, v217\n\t" \
  "v_perm_b32 v172, v216, v89, v25\n\t" \
  "v_perm_b32 v173, v216, v90, v26\n\t" \
  "v_perm_b32 v174, v216, v91, v27\n\t" \
  "v_perm_b32 v219, v216, v91, v28\n\t" \
  "v_lshl_or_b32 v175, v219, 16, v218\n\t" \
  "L_stO3_%=:\n\t" \
  "buffer_store_dwordx4 v[160:163], v16, s[28:31], s67 offen nt\n\t" \
  "buffer_store_dwordx4 v[164:167], v17, s[28:31], s67 offen nt\n\t" \
  "buffer_store_dwordx4 v[168:171], v18, s[28:31], s67 offen nt\n\t" \
  "buffer_store_dwordx4 v[172:175], v19, s[28:31], s67 offen nt\n\t" \
  "v_readfirstlane_b32 s22, v125\n\t" \
  "v_bfe_u32 v212, v125, 16, 11\n\t" \
  "v_lshrrev_b32 v214, 25, v125\n\t" \
  "v_add_u32 v212, s99, v212\n\t" \
  "v_and_b32 v214, 0x7c, v214\n\t" \
  "ds_write_b8 v212, v214\n\t" \
  "ds_read_b128 v[192:195], v15\n\t" \
  "ds_write_b8 v212, v10\n\t" \
  "s_lshr_b32 s75, s21, 16\n\t" \
  "s_and_b32 s94, s75, 0x7f\n\t" \
  "s_mul_i32 s94, s94, 0x01010101\n\t" \
  "s_bitcmp1_b32 s75, 7\n\t" \
  "s_cbranch_scc1 L_fbO4_%=\n\t" \
  "s_waitcnt vmcnt(18) lgkmcnt(3)\n\t" \
  "v_and_b32 v223, 0xffff, v109\n\t" \
  "v_lshl_or_b32 v223, v223, 12, v223\n\t" \
  "v_lshrrev_b32 v216, 0, v108\n\t" \
  "v_lshlrev_b32 v217, 7, v223\n\t" \
  "v_and_or_b32 v216, v216, s97, v176\n\t" \
  "v_perm_b32 v218, v97, v96, v28\n\t" \
  "v_add_u32 v216, s94, v216\n\t" \
  "v_bfi_b32 v216, s96, v216, v217\n\t" \
  "v_perm_b32 v144, v216, v96, v25\n\t" \
  "v_perm_b32 v145, v216, v97, v26\n\t" \
  "v_perm_b32 v146, v216, v98, v27\n\t" \
  "v_perm_b32 v219, v216, v98, v28\n\t" \
  "v_lshl_or_b32 v147, v219, 16, v218\n\t" \
  "v_lshrrev_b32 v216, 2, v108\n\t" \
  "v_lshlrev_b32 v217, 6, v223\n\t" \
  "v_and_or_b32 v216, v216, s97, v177\n\t" \
  "v_perm_b32 v218, v100, v99, v28\n\t" \
  "v_add_u32 v216, s94, v216\n\t" \
  "v_bfi_b32 v216, s96, v216, v217\n\t" \
  "v_perm_b32 v148, v216, v99, v25\n\t" \
  "v_perm_b32 v149, v216, v100, v26\n\t" \
  "v_perm_b32 v150, v216, v101, v27\n\t" \
  "v_perm_b32 v219, v216, v101, v28\n\t" \
  "v_lshl_or_b32 v151, v219, 16, v218\n\t" \
  "v_lshrrev_b32 v216, 4, v108\n\t" \
  "v_lshlrev_b32 v217, 5, v223\n\t" \
  "v_and_or_b32 v216, v216, s97, v178\n\t" \
  "v_perm_b32 v218, v103, v102, v28\n\t" \
  "v_add_u32 v216, s94, v216\n\t" \
  "v_bfi_b32 v216, s96, v216, v217\n\t" \
  "v_perm_b32 v152, v216, v102, v25\n\t" \
  "v_perm_b32 v153, v216, v103, v26\n\t" \
  "v_perm_b32 v154, v216, v104, v27\n\t" \
  "v_perm_b32 v219, v216, v104, v28\n\t" \
  "v_lshl_or_b32 v155, v219, 16, v218\n\t" \
  "v_lshrrev_b32 v216, 6, v108\n\t" \
  "v_lshlrev_b32 v217, 4, v223\n\t" \
  "v_and_or_b32 v216, v216, s97, v179\n\t" \
  "v_perm_b32 v218, v106, v105, v28\n\t" \
  "v_add_u32 v216, s94, v216\n\t" \
  "v_bfi_b32 v216, s96, v216, v217\n\t" \
  "v_perm_b32 v156, v216, v105, v25\n\t" \
  "v_perm_b32 v157, v216, v106, v26\n\t" \
  "v_perm_b32 v158, v216, v107, v27\n\t" \
  "v_perm_b32 v219, v216, v107, v28\n\t" \
  "v_lshl_or_b32 v159, v219, 16, v218\n\t" \
  "L_stO4_%=:\n\t" \
  "buffer_store_dwordx4 v[144:147], v16, s[28:31], s68 offen nt\n\t" \
  "buffer_store_dwordx4 v[148:151], v17, s[28:31], s68 offen nt\n\t" \
  "buffer_store_dwordx4 v[152:155], v18, s[28:31], s68 offen nt\n\t" \
  "buffer_store_dwordx4 v[156:159], v19, s[28:31], s68 offen nt\n\t" \
  "v_readfirstlane_b32 s21, v141\n\t" \
  "v_bfe_u32 v208, v141, 16, 11\n\t" \
  "v_lshrrev_b32 v210, 25, v141\n\t" \
  "v_add_u32 v208, s99, v208\n\t" \
  "v_and_b32 v210, 0x7c, v210\n\t" \
  "ds_write_b8 v208, v210\n\t" \
  "ds_read_b128 v[176:179], v15\n\t" \
  "ds_write_b8 v208, v10\n\t" \
  "s_lshr_b32 s75, s22, 16\n\t" \
  "s_and_b32 s94, s75, 0x7f\n\t" \
  "s_mul_i32 s94, s94, 0x01010101\n\t" \
  "s_bitcmp1_b32 s75, 7\n\t" \
  "s_cbranch_scc1 L_fbO5_%=\n\t" \
  "s_waitcnt vmcnt(21) lgkmcnt(3)\n\t" \
  "v_and_b32 v223, 0xffff, v125\n\t" \
  "v_lshl_or_b32 v223, v223, 12, v223\n\t" \
  "v_lshrrev_b32 v216, 0, v124\n\t" \
  "v_lshlrev_b32 v217, 7, v223\n\t" \
  "v_and_or_b32 v216, v216, s97, v192\n\t" \
  "v_perm_b32 v218, v113, v112, v28\n\t" \
  "v_add_u32 v216, s94, v216\n\t" \
  "v_bfi_b32 v216, s96, v216, v217\n\t" \
  "v_perm_b32 v160, v216, v112, v25\n\t" \
  "v_perm_b32 v161, v216, v113, v26\n\t" \
  "v_perm_b32 v162, v216, v114, v27\n\t" \
  "v_perm_b32 v219, v216, v114, v28\n\t" \
  "v_lshl_or_b32 v163, v219, 16, v218\n\t" \
  "v_lshrrev_b32 v216, 2, v124\n\t" \
  "v_lshlrev_b32 v217, 6, v223\n\t" \
  "v_and_or_b32 v216, v216, s97, v193\n\t" \
  "v_perm_b32 v218, v116, v115, v28\n\t" \
  "v_add_u32 v216, s94, v216\n\t" \
  "v_bfi_b32 v216, s96, v216, v217\n\t" \
  "v_perm_b32 v164, v216, v115, v25\n\t" \
  "v_perm_b32 v165, v216, v116, v26\n\t" \
  "v_perm_b32 v166, v216, v117, v27\n\t" \
  "v_perm_b32 v219, v216, v117, v28\n\t" \
  "v_lshl_or_b32 v167, v219, 16, v218\n\t" \
  "v_lshrrev_b32 v216, 4, v124\n\t" \
  "v_lshlrev_b32 v217, 5, v223\n\t" \
  "v_and_or_b32 v216, v216, s97, v194\n\t" \
  "v_perm_b32 v218, v119, v118, v28\n\t" \
  "v_add_u32 v216, s94, v216\n\t" \
  "v_bfi_b32 v216, s96, v216, v217\n\t" \
  "v_perm_b32 v168, v216, v118, v25\n\t" \
  "v_perm_b32 v169, v216, v119, v26\n\t" \
  "v_perm_b32 v170, v216, v120, v27\n\t" \
  "v_perm_b32 v219, v216, v120, v28\n\t" \
  "v_lshl_or_b32 v171, v219, 16, v218\n\t" \
  "v_lshrrev_b32 v216, 6, v124\n\t" \
  "v_lshlrev_b32 v217, 4, v223\n\t" \
  "v_and_or_b32 v216, v216, s97, v195\n\t" \
  "v_perm_b32 v218, v122, v121, v28\n\t" \
  "v_add_u32 v216, s94, v216\n\t" \
  "v_bfi_b32 v216, s96, v216, v217\n\t" \
  "v_perm_b32 v172, v216, v121, v25\n\t" \
  "v_perm_b32 v173, v216, v122, v26\n\t" \
  "v_perm_b32 v174, v216, v123, v27\n\t" \
  "v_perm_b32 v219, v216, v123, v28\n\t" \
  "v_lshl_or_b32 v175, v219, 16, v218\n\t" \
  "L_stO5_%=:\n\t" \
  "buffer_store_dwordx4 v[160:163], v16, s[28:31], s69 offen nt\n\t" \
  "buffer_store_dwordx4 v[164:167], v17, s[28:31], s69 offen nt\n\t" \
  "buffer_store_dwordx4 v[168:171], v18, s[28:31], s69 offen nt\n\t" \
  "buffer_store_dwordx4 v[172:175], v19, s[28:31], s69 offen nt\n\t" \
  "s_lshr_b32 s75, s21, 16\n\t" \
  "s_and_b32 s94, s75, 0x7f\n\t" \
  "s_mul_i32 s94, s94, 0x01010101\n\t" \
  "s_bitcmp1_b32 s75, 7\n\t" \
  "s_cbranch_scc1 L_fbO6_%=\n\t" \
  "s_waitcnt vmcnt(24) lgkmcnt(1)\n\t" \
  "v_and_b32 v223, 0xffff, v141\n\t" \
  "v_lshl_or_b32 v223, v223, 12, v223\n\t" \
  "v_lshrrev_b32 v216, 0, v140\n\t" \
  "v_lshlrev_b32 v217, 7, v223\n\t" \
  "v_and_or_b32 v216, v216, s97, v176\n\t" \
  "v_perm_b32 v218, v129, v128, v28\n\t" \
  "v_add_u32 v216, s94, v216\n\t" \
  "v_bfi_b32 v216, s96, v216, v217\n\t" \
  "v_perm_b32 v144, v216, v128, v25\n\t" \
  "v_perm_b32 v145, v216, v129, v26\n\t" \
  "v_perm_b32 v146, v216, v130, v27\n\t" \
  "v_perm_b32 v219, v216, v130, v28\n\t" \
  "v_lshl_or_b32 v147, v219, 16, v218\n\t" \
  "v_lshrrev_b32 v216, 2, v140\n\t" \
  "v_lshlrev_b32 v217, 6, v223\n\t" \
  "v_and_or_b32 v216, v216, s97, v177\n\t" \
  "v_perm_b32 v218, v132, v131, v28\n\t" \
  "v_add_u32 v216, s94, v216\n\t" \
  "v_bfi_b32 v216, s96, v216, v217\n\t" \
  "v_perm_b32 v148, v216, v131, v25\n\t" \
  "v_perm_b32 v149, v216, v132, v26\n\t" \
  "v_perm_b32 v150, v216, v133, v27\n\t" \
  "v_perm_b32 v219, v216, v133, v28\n\t" \
  "v_lshl_or_b32 v151, v219, 16, v218\n\t" \
  "v_lshrrev_b32 v216, 4, v140\n\t" \
  "v_lshlrev_b32 v217, 5, v223\n\t" \
  "v_and_or_b32 v216, v216, s97, v178\n\t" \
  "v_perm_b32 v218, v135, v134, v28\n\t" \
  "v_add_u32 v216, s94, v216\n\t" \
  "v_bfi_b32 v216, s96, v216, v217\n\t" \
  "v_perm_b32 v152, v216, v134, v25\n\t" \
  "v_perm_b32 v153, v216, v135, v26\n\t" \
  "v_perm_b32 v154, v216, v136, v27\n\t" \
  "v_perm_b32 v219, v216, v136, v28\n\t" \
  "v_lshl_or_b32 v155, v219, 16, v218\n\t" \
  "v_lshrrev_b32 v216, 6, v140\n\t" \
  "v_lshlrev_b32 v217, 4, v223\n\t" \
  "v_and_or_b32 v216, v216, s97, v179\n\t" \
  "v_perm_b32 v218, v138, v137, v28\n\t" \
  "v_add_u32 v216, s94, v216\n\t" \
  "v_bfi_b32 v216, s96, v216, v217\n\t" \
  "v_perm_b32 v156, v216, v137, v25\n\t" \
  "v_perm_b32 v157, v216, v138, v26\n\t" \
  "v_perm_b32 v158, v216, v139, v27\n\t" \
  "v_perm_b32 v219, v216, v139, v28\n\t" \
  "v_lshl_or_b32 v159, v219, 16, v218\n\t" \
  "L_stO6_%=:\n\t" \
  "buffer_store_dwordx4 v[144:147], v16, s[28:31], s70 offen nt\n\t" \
  "buffer_store_dwordx4 v[148:151], v17, s[28:31], s70 offen nt\n\t" \
  "buffer_store_dwordx4 v[152:155], v18, s[28:31], s70 offen nt\n\t" \
  "buffer_store_dwordx4 v[156:159], v19, s[28:31], s70 offen nt\n\t" \
  "s_branch L_done_%=\n\t" \
  "L_fbO0_%=:\n\t" \
  "s_add_u32 s75, s93, 3584\n\t" \
  "buffer_load_dwordx4 v[144:147], v8, s[36:39], s75 offen offset:0 sc0 sc1\n\t" \
  "buffer_load_dwordx4 v[148:151], v8, s[36:39], s75 offen offset:1024 sc0 sc1\n\t" \
  "buffer_load_dwordx4 v[152:155], v8, s[36:39], s75 offen offset:2048 sc0 sc1\n\t" \
  "buffer_load_dwordx4 v[156:159], v8, s[36:39], s75 offen offset:3072 sc0 sc1\n\t" \
  "s_waitcnt vmcnt(0) lgkmcnt(0)\n\t" \
  "s_branch L_stO0_%=\n\t" \
  "L_fbO1_%=:\n\t" \
  "s_add_u32 s75, s93, 3584\n\t" \
  "buffer_load_dwordx4 v[160:163], v8, s[40:43], s75 offen offset:0 sc0 sc1\n\t" \
  "buffer_load_dwordx4 v[164:167], v8, s[40:43], s75 offen offset:1024 sc0 sc1\n\t" \
  "buffer_load_dwordx4 v[168:171], v8, s[40:43], s75 offen offset:2048 sc0 sc1\n\t" \
  "buffer_load_dwordx4 v[172:175], v8, s[40:43], s75 offen offset:3072 sc0 sc1\n\t" \
  "s_waitcnt vmcnt(0) lgkmcnt(0)\n\t" \
  "s_branch L_stO1_%=\n\t" \
  "L_fbO2_%=:\n\t" \
  "s_add_u32 s75, s93, 3584\n\t" \
  "buffer_load_dwordx4 v[144:147], v8, s[44:47], s75 offen offset:0 sc0 sc1\n\t" \
  "buffer_load_dwordx4 v[148:151], v8, s[44:47], s75 offen offset:1024 sc0 sc1\n\t" \
  "buffer_load_dwordx4 v[152:155], v8, s[44:47], s75 offen offset:2048 sc0 sc1\n\t" \
  "buffer_load_dwordx4 v[156:159], v8, s[44:47], s75 offen offset:3072 sc0 sc1\n\t" \
  "s_waitcnt vmcnt(0) lgkmcnt(0)\n\t" \
  "s_branch L_stO2_%=\n\t" \
  "L_fbO3_%=:\n\t" \
  "s_add_u32 s75, s93, 3584\n\t" \
  "buffer_load_dwordx4 v[160:163], v8, s[48:51], s75 offen offset:0 sc0 sc1\n\t" \
  "buffer_load_dwordx4 v[164:167], v8, s[48:51], s75 offen offset:1024 sc0 sc1\n\t" \
  "buffer_load_dwordx4 v[168:171], v8, s[48:51], s75 offen offset:2048 sc0 sc1\n\t" \
  "buffer_load_dwordx4 v[172:175], v8, s[48:51], s75 offen offset:3072 sc0 sc1\n\t" \
  "s_waitcnt vmcnt(0) lgkmcnt(0)\n\t" \
  "s_branch L_stO3_%=\n\t" \
  "L_fbO4_%=:\n\t" \
  "s_add_u32 s75, s93, 3584\n\t" \
  "buffer_load_dwordx4 v[144:147], v8, s[52:55], s75 offen offset:0 sc0 sc1\n\t" \
  "buffer_load_dwordx4 v[148:151], v8, s[52:55], s75 offen offset:1024 sc0 sc1\n\t" \
  "buffer_load_dwordx4 v[152:155], v8, s[52:55], s75 offen offset:2048 sc0 sc1\n\t" \
  "buffer_load_dwordx4 v[156:159], v8, s[52:55], s75 offen offset:3072 sc0 sc1\n\t" \
  "s_waitcnt vmcnt(0) lgkmcnt(0)\n\t" \
  "s_branch L_stO4_%=\n\t" \
  "L_fbO5_%=:\n\t" \
  "s_add_u32 s75, s93, 3584\n\t" \
  "buffer_load_dwordx4 v[160:163], v8, s[56:59], s75 offen offset:0 sc0 sc1\n\t" \
  "buffer_load_dwordx4 v[164:167], v8, s[56:59], s75 offen offset:1024 sc0 sc1\n\t" \
  "buffer_load_dwordx4 v[168:171], v8, s[56:59], s75 offen offset:2048 sc0 sc1\n\t" \
  "buffer_load_dwordx4 v[172:175], v8, s[56:59], s75 offen offset:3072 sc0 sc1\n\t" \
  "s_waitcnt vmcnt(0) lgkmcnt(0)\n\t" \
  "s_branch L_stO5_%=\n\t" \
  "L_fbO6_%=:\n\t" \
  "s_add_u32 s75, s93, 3584\n\t" \
  "buffer_load_dwordx4 v[144:147], v8, s[60:63], s75 offen offset:0 sc0 sc1\n\t" \
  "buffer_load_dwordx4 v[148:151], v8, s[60:63], s75 offen offset:1024 sc0 sc1\n\t" \
  "buffer_load_dwordx4 v[152:155], v8, s[60:63], s75 offen offset:2048 sc0 sc1\n\t" \
  "buffer_load_dwordx4 v[156:159], v8, s[60:63], s75 offen offset:3072 sc0 sc1\n\t" \
  "s_waitcnt vmcnt(0) lgkmcnt(0)\n\t" \
  "s_branch L_stO6_%=\n\t" \
  "L_raw_%=:\n\t" \
  "buffer_load_dwordx4 v[32:35], v16, s[36:39], s64 offen sc0 sc1\n\t" \
  "buffer_load_dwordx4 v[48:51], v16, s[40:43], s65 offen sc0 sc1\n\t" \
  "buffer_load_dwordx4 v[64:67], v16, s[44:47], s66 offen sc0 sc1\n\t" \
  "buffer_load_dwordx4 v[80:83], v16, s[48:51], s67 offen sc0 sc1\n\t" \
  "buffer_load_dwordx4 v[96:99], v16, s[52:55], s68 offen sc0 sc1\n\t" \
  "buffer_load_dwordx4 v[112:115], v16, s[56:59], s69 offen sc0 sc1\n\t" \
  "buffer_load_dwordx4 v[128:131], v16, s[60:63], s70 offen sc0 sc1\n\t" \
  "buffer_load_dwordx4 v[36:39], v17, s[36:39], s64 offen sc0 sc1\n\t" \
  "buffer_load_dwordx4 v[52:55], v17, s[40:43], s65 offen sc0 sc1\n\t" \
  "buffer_load_dwordx4 v[68:71], v17, s[44:47], s66 offen sc0 sc1\n\t" \
  "buffer_load_dwordx4 v[84:87], v17, s[48:51], s67 offen sc0 sc1\n\t" \
  "buffer_load_dwordx4 v[100:103], v17, s[52:55], s68 offen sc0 sc1\n\t" \
  "buffer_load_dwordx4 v[116:119], v17, s[56:59], s69 offen sc0 sc1\n\t" \
  "buffer_load_dwordx4 v[132:135], v17, s[60:63], s70 offen sc0 sc1\n\t" \
  "s_waitcnt vmcnt(13)\n\t" \
  "buffer_load_dwordx4 v[40:43], v18, s[36:39], s64 offen sc0 sc1\n\t" \
  "s_waitcnt vmcnt(13)\n\t" \
  "buffer_load_dwordx4 v[56:59], v18, s[40:43], s65 offen sc0 sc1\n\t" \
  "s_waitcnt vmcnt(13)\n\t" \
  "buffer_load_dwordx4 v[72:75], v18, s[44:47], s66 offen sc0 sc1\n\t" \
  "s_waitcnt vmcnt(13)\n\t" \
  "buffer_load_dwordx4 v[88:91], v18, s[48:51], s67 offen sc0 sc1\n\t" \
  "s_waitcnt vmcnt(13)\n\t" \
  "buffer_load_dwordx4 v[104:107], v18, s[52:55], s68 offen sc0 sc1\n\t" \
  "s_waitcnt vmcnt(13)\n\t" \
  "buffer_load_dwordx4 v[120:123], v18, s[56:59], s69 offen sc0 sc1\n\t" \
  "s_waitcnt vmcnt(13)\n\t" \
  "buffer_load_dwordx4 v[136:139], v18, s[60:63], s70 offen sc0 sc1\n\t" \
  "s_waitcnt vmcnt(13)\n\t" \
  "buffer_load_dwordx4 v[44:47], v19, s[36:39], s64 offen sc0 sc1\n\t" \
  "s_waitcnt vmcnt(13)\n\t" \
  "buffer_load_dwordx4 v[60:63], v19, s[40:43], s65 offen sc0 sc1\n\t" \
  "s_waitcnt vmcnt(13)\n\t" \
  "buffer_load_dwordx4 v[76:79], v19, s[44:47], s66 offen sc0 sc1\n\t" \
  "s_waitcnt vmcnt(13)\n\t" \
  "buffer_load_dwordx4 v[92:95], v19, s[48:51], s67 offen sc0 sc1\n\t" \
  "s_waitcnt vmcnt(13)\n\t" \
  "buffer_load_dwordx4 v[108:111], v19, s[52:55], s68 offen sc0 sc1\n\t" \
  "s_waitcnt vmcnt(13)\n\t" \
  "buffer_load_dwordx4 v[124:127], v19, s[56:59], s69 offen sc0 sc1\n\t" \
  "s_waitcnt vmcnt(13)\n\t" \
  "buffer_load_dwordx4 v[140:143], v19, s[60:63], s70 offen sc0 sc1\n\t" \
  "s_waitcnt vmcnt(21)\n\t" \
  "buffer_store_dwordx4 v[32:35], v16, s[28:31], s64 offen sc0 sc1\n\t" \
  "buffer_store_dwordx4 v[48:51], v16, s[28:31], s65 offen sc0 sc1\n\t" \
  "buffer_store_dwordx4 v[64:67], v16, s[28:31], s66 offen sc0 sc1\n\t" \
  "buffer_store_dwordx4 v[80:83], v16, s[28:31], s67 offen sc0 sc1\n\t" \
  "buffer_store_dwordx4 v[96:99], v16, s[28:31], s68 offen sc0 sc1\n\t" \
  "buffer_store_dwordx4 v[112:115], v16, s[28:31], s69 offen sc0 sc1\n\t" \
  "buffer_store_dwordx4 v[128:131], v16, s[28:31], s70 offen sc0 sc1\n\t" \
  "s_waitcnt vmcnt(21)\n\t" \
  "s_mov_b64 exec, 1\n\t" \
  "ds_add_u32 v10, v13\n\t" \
  "s_mov_b64 exec, -1\n\t" \
  "buffer_store_dwordx4 v[36:39], v17, s[28:31], s64 offen sc0 sc1\n\t" \
  "buffer_store_dwordx4 v[52:55], v17, s[28:31], s65 offen sc0 sc1\n\t" \
  "buffer_store_dwordx4 v[68:71], v17, s[28:31], s66 offen sc0 sc1\n\t" \
  "buffer_store_dwordx4 v[84:87], v17, s[28:31], s67 offen sc0 sc1\n\t" \
  "buffer_store_dwordx4 v[100:103], v17, s[28:31], s68 offen sc0 sc1\n\t" \
  "buffer_store_dwordx4 v[116:119], v17, s[28:31], s69 offen sc0 sc1\n\t" \
  "buffer_store_dwordx4 v[132:135], v17, s[28:31], s70 offen sc0 sc1\n\t" \
  "s_waitcnt vmcnt(21)\n\t" \
  "buffer_store_dwordx4 v[40:43], v18, s[28:31], s64 offen sc0 sc1\n\t" \
  "buffer_store_dwordx4 v[56:59], v18, s[28:31], s65 offen sc0 sc1\n\t" \
  "buffer_store_dwordx4 v[72:75], v18, s[28:31], s66 offen sc0 sc1\n\t" \
  "buffer_store_dwordx4 v[88:91], v18, s[28:31], s67 offen sc0 sc1\n\t" \
  "buffer_store_dwordx4 v[104:107], v18, s[28:31], s68 offen sc0 sc1\n\t" \
  "buffer_store_dwordx4 v[120:123], v18, s[28:31], s69 offen sc0 sc1\n\t" \
  "buffer_store_dwordx4 v[136:139], v18, s[28:31], s70 offen sc0 sc1\n\t" \
  "s_waitcnt vmcnt(21)\n\t" \
  "buffer_store_dwordx4 v[44:47], v19, s[28:31], s64 offen sc0 sc1\n\t" \
  "buffer_store_dwordx4 v[60:63], v19, s[28:31], s65 offen sc0 sc1\n\t" \
  "buffer_store_dwordx4 v[76:79], v19, s[28:31], s66 offen sc0 sc1\n\t" \
  "buffer_store_dwordx4 v[92:95], v19, s[28:31], s67 offen sc0 sc1\n\t" \
  "buffer_store_dwordx4 v[108:111], v19, s[28:31], s68 offen sc0 sc1\n\t" \
  "buffer_store_dwordx4 v[124:127], v19, s[28:31], s69 offen sc0 sc1\n\t" \
  "buffer_store_dwordx4 v[140:143], v19, s[28:31], s70 offen sc0 sc1\n\t" \
  "s_branch L_done_%=\n\t" \
  "L_skip_%=:\n\t" \
  "s_mov_b64 exec, 1\n\t" \
  "ds_add_u32 v10, v13\n\t" \
  "s_mov_b64 exec, -1\n\t" \
  "L_done_%=:\n\t" \
  "s_add_u32 s71, s71, 4\n\t" \
  "s_add_u32 s72, s72, 124\n\t" \
  "s_mov_b32 s73, s74\n\t" \
  "s_add_u32 s81, s72, 124\n\t" \
  "s_min_u32 s81, s81, 3359\n\t" \
  "s_add_u32 s81, s81, s23\n\t" \
  "s_lshl_b32 s81, s81, 2\n\t" \
  "s_load_dword s74, %[tab], s81\n\t" \
  "s_add_u32 s75, s78, s72\n\t" \
  "v_mov_b32 v9, s75\n\t" \
  "v_mov_b32 v0, %[word]\n\t" \
  "s_waitcnt vmcnt(0)\n\t" \
  "s_mov_b64 exec, 1\n\t" \
  "global_store_dword v0, v9, s[34:35] sc0 sc1\n\t" \
  "s_mov_b64 exec, -1\n\t" \
  "s_cmp_eq_u32 s80, 4\n\t" \
  "s_cselect_b32 s77, 1, 0\n\t" \
  "s_cbranch_scc1 L_ck_%=\n\t" \
  "s_cmp_lg_u32 s80, 0\n\t" \
  "s_cbranch_scc1 L_na_%=\n\t" \
  "L_ck_%=:\n\t" \
  "L_ca2_%=:\n\t" \
  "v_add_u32 v0, 8704, v8\n\t" \
  "global_load_dwordx4 v[32:35], v0, s[34:35] sc0 sc1\n\t" \
  "v_add_u32 v0, 10752, v8\n\t" \
  "global_load_dwordx4 v[36:39], v0, s[34:35] sc0 sc1\n\t" \
  "v_add_u32 v0, 12800, v8\n\t" \
  "global_load_dwordx4 v[40:43], v0, s[34:35] sc0 sc1\n\t" \
  "v_add_u32 v0, 14848, v8\n\t" \
  "global_load_dwordx4 v[44:47], v0, s[34:35] sc0 sc1\n\t" \
  "v_add_u32 v0, 16896, v8\n\t" \
  "global_load_dwordx4 v[48:51], v0, s[34:35] sc0 sc1\n\t" \
  "v_add_u32 v0, 18944, v8\n\t" \
  "global_load_dwordx4 v[52:55], v0, s[34:35] sc0 sc1\n\t" \
  "v_add_u32 v0, 20992, v8\n\t" \
  "global_load_dwordx4 v[56:59], v0, s[34:35] sc0 sc1\n\t" \
  "v_add_u32 v0, 23040, v8\n\t" \
  "global_load_dwordx4 v[60:63], v0, s[34:35] sc0 sc1\n\t" \
  "s_waitcnt vmcnt(0)\n\t" \
  "v_mov_b32 v0, -1\n\t" \
  "v_min3_u32 v0, v0, v32, v33\n\t" \
  "v_min3_u32 v0, v0, v34, v35\n\t" \
  "v_min3_u32 v0, v0, v36, v37\n\t" \
  "v_min3_u32 v0, v0, v38, v39\n\t" \
  "v_min3_u32 v0, v0, v40, v41\n\t" \
  "v_min3_u32 v0, v0, v42, v43\n\t" \
  "v_min3_u32 v0, v0, v44, v45\n\t" \
  "v_min3_u32 v0, v0, v46, v47\n\t" \
  "v_min3_u32 v0, v0, v48, v49\n\t" \
  "v_min3_u32 v0, v0, v50, v51\n\t" \
  "v_min3_u32 v0, v0, v52, v53\n\t" \
  "v_min3_u32 v0, v0, v54, v55\n\t" \
  "v_min3_u32 v0, v0, v56, v57\n\t" \
  "v_min3_u32 v0, v0, v58, v59\n\t" \
  "v_min3_u32 v0, v0, v60, v61\n\t" \
  "v_min3_u32 v0, v0, v62, v63\n\t" \
  "s_lshl_b32 s75, s84, 1\n\t" \
  "s_add_u32 s75, s75, s77\n\t" \
  "v_cmp_gt_u32 vcc, s75, v0\n\t" \
  "s_nop 3\n\t" \
  "s_and_b32 vcc_lo, vcc_lo, 0xffffffff\n\t" \
  "s_and_b32 vcc_hi, vcc_hi, 0xffffff\n\t" \
  "s_cmp_eq_u64 vcc, 0\n\t" \
  "s_cbranch_scc1 L_cd2_%=\n\t" \
  "s_sleep 8\n\t" \
  "global_load_dword v9, v10, s[34:35] offset:128 sc1\n\t" \
  "s_memrealtime s[86:87]\n\t" \
  "s_waitcnt vmcnt(0) lgkmcnt(0)\n\t" \
  "v_readfirstlane_b32 s76, v9\n\t" \
  "s_sub_u32 s86, s86, s90\n\t" \
  "s_cmp_gt_u32 s86, 50000000\n\t" \
  "s_cbranch_scc1 L_dead_%=\n\t" \
  "s_cmp_eq_u32 s76, s84\n\t" \
  "s_cbranch_scc1 L_dead_%=\n\t" \
  "s_branch L_ca2_%=\n\t" \
  "L_cd2_%=:\n\t" \
  "L_na_%=:\n\t" \
  "s_cmp_lg_u32 s80, 1\n\t" \
  "s_cbranch_scc1 L_nb_%=\n\t" \
  "v_mov_b32 v0, %[repl]\n\t" \
  "L_pw_%=:\n\t" \
  "global_load_dwordx2 v[2:3], v0, s[34:35] offset:8 sc1\n\t" \
  "s_waitcnt vmcnt(0) lgkmcnt(0)\n\t" \
  "v_readfirstlane_b32 s23, v2\n\t" \
  "v_readfirstlane_b32 s75, v3\n\t" \
  "s_nop 3\n\t" \
  "s_cmp_eq_u32 s75, s84\n\t" \
  "s_cbranch_scc1 L_pd_%=\n\t" \
  "s_sleep 4\n\t" \
  "global_load_dword v9, v10, s[34:35] offset:128 sc1\n\t" \
  "s_memrealtime s[86:87]\n\t" \
  "s_waitcnt vmcnt(0) lgkmcnt(0)\n\t" \
  "v_readfirstlane_b32 s76, v9\n\t" \
  "s_sub_u32 s86, s86, s90\n\t" \
  "s_cmp_gt_u32 s86, 50000000\n\t" \
  "s_cbranch_scc1 L_dead_%=\n\t" \
  "s_cmp_eq_u32 s76, s84\n\t" \
  "s_cbranch_scc1 L_dead_%=\n\t" \
  "s_branch L_pw_%=\n\t" \
  "L_pd_%=:\n\t" \
  "s_lshr_b32 s23, s23, 24\n\t" \
  "s_min_u32 s23, s23, 2\n\t" \
  "s_mul_i32 s23, s23, 3360\n\t" \
  "s_add_u32 s81, s72, 0\n\t" \
  "s_min_u32 s81, s81, 3359\n\t" \
  "s_add_u32 s81, s81, s23\n\t" \
  "s_lshl_b32 s81, s81, 2\n\t" \
  "s_load_dword s73, %[tab], s81\n\t" \
  "s_add_u32 s81, s72, 124\n\t" \
  "s_min_u32 s81, s81, 3359\n\t" \
  "s_add_u32 s81, s81, s23\n\t" \
  "s_lshl_b32 s81, s81, 2\n\t" \
  "s_load_dword s74, %[tab], s81\n\t" \
  "L_nb_%=:\n\t" \
  "s_add_u32 s80, s80, 1\n\t" \
  "s_cmp_lt_u32 s72, 3360\n\t" \
  "s_cbranch_scc1 L_loop_%=\n\t" \
  "s_branch L_fin_%=\n\t" \
  "L_dead_%=:\n\t" \
  "v_mov_b32 v9, s84\n\t" \
  "global_store_dword v10, v9, s[34:35] offset:128 sc1\n\t" \
  "s_load_dwordx2 s[28:29], %[ka], 0x20\n\t" \
  "s_add_u32 s75, s24, 1\n\t" \
  "s_and_b32 s75, s75, 7\n\t" \
  "s_mul_i32 s75, s75, 16\n\t" \
  "s_add_u32 s75, s75, %[lasti]\n\t" \
  "s_and_b32 s76, s75, 15\n\t" \
  "s_lshl_b32 s76, s76, 24\n\t" \
  "s_lshr_b32 s75, s75, 4\n\t" \
  "s_mul_i32 s75, s75, 8\n\t" \
  "s_add_u32 s75, s75, 3\n\t" \
  "s_lshl_b32 s75, s75, 10\n\t" \
  "s_add_u32 s75, s75, s76\n\t" \
  "s_add_u32 s75, s75, %[lastp]\n\t" \
  "s_mov_b32 s30, 0x10000000\n\t" \
  "v_mov_b32 v9, 0x7fc00000\n\t" \
  "s_waitcnt lgkmcnt(0)\n\t" \
  "buffer_store_dword v9, v8, s[28:31], s75 offen\n\t" \
  "L_fin_%=:\n\t" \
  "s_waitcnt vmcnt(0) lgkmcnt(0)\n\t" \
  "s_barrier\n\t" \
  "s_cmp_lg_u32 %[wave], 0\n\t" \
  "s_cbranch_scc1 L_bye_%=\n\t" \
  "s_mov_b64 exec, 1\n\t" \
  "global_atomic_add v10, v13, s[34:35]\n\t" \
  "L_bye_%=:\n\t" \
  "s_endpgm\n\t"
constexpr int NCOMM = 32, NSHARE = 16, NUNITS = 3360, NINPUT = 2336, NPLANS = 3,  THR1 = 1060, THR2 = 1092, HIST = 0, HTH1 = 1074, HTH2 = 1118, FIXPERM = 0, FIXPLAN = -1;
constexpr unsigned PSKEW = 2048, PDELAY = 12, CMPR = 1, XCAP = 80, XCAPO = 64, CMPA = 1, INNT = 0, NPUSH = 24, NRAW = 124, XAG = 4, WTOUCH = 1, ROTM = 1, P2DELAY = 20, CALW = 4, CALT = 600, SMUL = 8, SADD = 3;
constexpr unsigned DSKEW = 2048;
constexpr unsigned AGGSLEEP = 0, AGGWAKE = 19116;
constexpr unsigned NREPL = 256;
#define STAMPS 0
#define FINE 0
#define EXPT 0
#define EXST "nt"
#define XADJ 0x26382e4433452d3dull
#define CALIB 10
#define CALIB_ASM \
  "v_mov_b32 v0, %[seed]\n\t" \
  "v_mul_lo_u32 v1, v0, %[mul]\n\t" \
  "v_mul_lo_u32 v2, v1, %[mul]\n\t" \
  "v_mul_lo_u32 v3, v2, %[mul]\n\t" \
  "v_mul_lo_u32 v4, v3, %[mul]\n\t" \
  "v_mul_lo_u32 v5, v4, %[mul]\n\t" \
  "v_mul_lo_u32 v6, v5, %[mul]\n\t" \
  "v_mul_lo_u32 v7, v6, %[mul]\n\t" \
  "v_mul_lo_u32 v8, v7, %[mul]\n\t" \
  "v_mul_lo_u32 v9, v8, %[mul]\n\t" \
  "v_mul_lo_u32 v10, v9, %[mul]\n\t" \
  "v_mul_lo_u32 v11, v10, %[mul]\n\t" \
  "v_mul_lo_u32 v12, v11, %[mul]\n\t" \
  "v_mul_lo_u32 v13, v12, %[mul]\n\t" \
  "v_mul_lo_u32 v14, v13, %[mul]\n\t" \
  "v_mul_lo_u32 v15, v14, %[mul]\n\t" \
  "v_mul_lo_u32 v16, v15, %[mul]\n\t" \
  "v_mul_lo_u32 v17, v16, %[mul]\n\t" \
  "v_mul_lo_u32 v18, v17, %[mul]\n\t" \
  "v_mul_lo_u32 v19, v18, %[mul]\n\t" \
  "v_mul_lo_u32 v20, v19, %[mul]\n\t" \
  "v_mul_lo_u32 v21, v20, %[mul]\n\t" \
  "v_mul_lo_u32 v22, v21, %[mul]\n\t" \
  "v_mul_lo_u32 v23, v22, %[mul]\n\t" \
  "v_mul_lo_u32 v24, v23, %[mul]\n\t" \
  "v_mul_lo_u32 v25, v24, %[mul]\n\t" \
  "v_mul_lo_u32 v26, v25, %[mul]\n\t" \
  "v_mul_lo_u32 v27, v26, %[mul]\n\t" \
  "v_mul_lo_u32 v28, v27, %[mul]\n\t" \
  "v_mul_lo_u32 v29, v28, %[mul]\n\t" \
  "v_mul_lo_u32 v30, v29, %[mul]\n\t" \
  "v_mul_lo_u32 v31, v30, %[mul]\n\t" \
  "v_mul_lo_u32 v32, v31, %[mul]\n\t" \
  "v_mul_lo_u32 v33, v32, %[mul]\n\t" \
  "v_mul_lo_u32 v34, v33, %[mul]\n\t" \
  "v_mul_lo_u32 v35, v34, %[mul]\n\t" \
  "v_mul_lo_u32 v36, v35, %[mul]\n\t" \
  "v_mul_lo_u32 v37, v36, %[mul]\n\t" \
  "v_mul_lo_u32 v38, v37, %[mul]\n\t" \
  "v_mul_lo_u32 v39, v38, %[mul]\n\t" \
  "v_mul_lo_u32 v40, v39, %[mul]\n\t" \
  "v_mul_lo_u32 v41, v40, %[mul]\n\t" \
  "v_mul_lo_u32 v42, v41, %[mul]\n\t" \
  "v_mul_lo_u32 v43, v42, %[mul]\n\t" \
  "v_mul_lo_u32 v44, v43, %[mul]\n\t" \
  "v_mul_lo_u32 v45, v44, %[mul]\n\t" \
  "v_mul_lo_u32 v46, v45, %[mul]\n\t" \
  "v_mul_lo_u32 v47, v46, %[mul]\n\t" \
  "v_mul_lo_u32 v48, v47, %[mul]\n\t" \
  "v_mul_lo_u32 v49, v48, %[mul]\n\t" \
  "v_mul_lo_u32 v50, v49, %[mul]\n\t" \
  "v_mul_lo_u32 v51, v50, %[mul]\n\t" \
  "v_mul_lo_u32 v52, v51, %[mul]\n\t" \
  "v_mul_lo_u32 v53, v52, %[mul]\n\t" \
  "v_mul_lo_u32 v54, v53, %[mul]\n\t" \
  "v_mul_lo_u32 v55, v54, %[mul]\n\t" \
  "v_mul_lo_u32 v56, v55, %[mul]\n\t" \
  "v_mul_lo_u32 v57, v56, %[mul]\n\t" \
  "v_mul_lo_u32 v58, v57, %[mul]\n\t" \
  "v_mul_lo_u32 v59, v58, %[mul]\n\t" \
  "v_mul_lo_u32 v60, v59, %[mul]\n\t" \
  "v_mul_lo_u32 v61, v60, %[mul]\n\t" \
  "v_mul_lo_u32 v62, v61, %[mul]\n\t" \
  "v_mul_lo_u32 v63, v62, %[mul]\n\t" \
  "v_and_b32 v0, 0xbbbbbbbb, v0\n\t" \
  "v_and_b32 v1, 0xbbbbbbbb, v1\n\t" \
  "v_and_b32 v2, 0xbbbbbbbb, v2\n\t" \
  "v_and_b32 v3, 0xbbbbbbbb, v3\n\t" \
  "v_and_b32 v4, 0xbbbbbbbb, v4\n\t" \
  "v_and_b32 v5, 0xbbbbbbbb, v5\n\t" \
  "v_and_b32 v6, 0xbbbbbbbb, v6\n\t" \
  "v_and_b32 v7, 0xbbbbbbbb, v7\n\t" \
  "v_and_b32 v8, 0xbbbbbbbb, v8\n\t" \
  "v_and_b32 v9, 0xbbbbbbbb, v9\n\t" \
  "v_and_b32 v10, 0xbbbbbbbb, v10\n\t" \
  "v_and_b32 v11, 0xbbbbbbbb, v11\n\t" \
  "v_and_b32 v12, 0xbbbbbbbb, v12\n\t" \
  "v_and_b32 v13, 0xbbbbbbbb, v13\n\t" \
  "v_and_b32 v14, 0xbbbbbbbb, v14\n\t" \
  "v_and_b32 v15, 0xbbbbbbbb, v15\n\t" \
  "v_and_b32 v16, 0xbbbbbbbb, v16\n\t" \
  "v_and_b32 v17, 0xbbbbbbbb, v17\n\t" \
  "v_and_b32 v18, 0xbbbbbbbb, v18\n\t" \
  "v_and_b32 v19, 0xbbbbbbbb, v19\n\t" \
  "v_and_b32 v20, 0xbbbbbbbb, v20\n\t" \
  "v_and_b32 v21, 0xbbbbbbbb, v21\n\t" \
  "v_and_b32 v22, 0xbbbbbbbb, v22\n\t" \
  "v_and_b32 v23, 0xbbbbbbbb, v23\n\t" \
  "v_and_b32 v24, 0xbbbbbbbb, v24\n\t" \
  "v_and_b32 v25, 0xbbbbbbbb, v25\n\t" \
  "v_and_b32 v26, 0xbbbbbbbb, v26\n\t" \
  "v_and_b32 v27, 0xbbbbbbbb, v27\n\t" \
  "v_and_b32 v28, 0xbbbbbbbb, v28\n\t" \
  "v_and_b32 v29, 0xbbbbbbbb, v29\n\t" \
  "v_and_b32 v30, 0xbbbbbbbb, v30\n\t" \
  "v_and_b32 v31, 0xbbbbbbbb, v31\n\t" \
  "v_and_b32 v32, 0xbbbbbbbb, v32\n\t" \
  "v_and_b32 v33, 0xbbbbbbbb, v33\n\t" \
  "v_and_b32 v34, 0xbbbbbbbb, v34\n\t" \
  "v_and_b32 v35, 0xbbbbbbbb, v35\n\t" \
  "v_and_b32 v36, 0xbbbbbbbb, v36\n\t" \
  "v_and_b32 v37, 0xbbbbbbbb, v37\n\t" \
  "v_and_b32 v38, 0xbbbbbbbb, v38\n\t" \
  "v_and_b32 v39, 0xbbbbbbbb, v39\n\t" \
  "v_and_b32 v40, 0xbbbbbbbb, v40\n\t" \
  "v_and_b32 v41, 0xbbbbbbbb, v41\n\t" \
  "v_and_b32 v42, 0xbbbbbbbb, v42\n\t" \
  "v_and_b32 v43, 0xbbbbbbbb, v43\n\t" \
  "v_and_b32 v44, 0xbbbbbbbb, v44\n\t" \
  "v_and_b32 v45, 0xbbbbbbbb, v45\n\t" \
  "v_and_b32 v46, 0xbbbbbbbb, v46\n\t" \
  "v_and_b32 v47, 0xbbbbbbbb, v47\n\t" \
  "v_and_b32 v48, 0xbbbbbbbb, v48\n\t" \
  "v_and_b32 v49, 0xbbbbbbbb, v49\n\t" \
  "v_and_b32 v50, 0xbbbbbbbb, v50\n\t" \
  "v_and_b32 v51, 0xbbbbbbbb, v51\n\t" \
  "v_and_b32 v52, 0xbbbbbbbb, v52\n\t" \
  "v_and_b32 v53, 0xbbbbbbbb, v53\n\t" \
  "v_and_b32 v54, 0xbbbbbbbb, v54\n\t" \
  "v_and_b32 v55, 0xbbbbbbbb, v55\n\t" \
  "v_and_b32 v56, 0xbbbbbbbb, v56\n\t" \
  "v_and_b32 v57, 0xbbbbbbbb, v57\n\t" \
  "v_and_b32 v58, 0xbbbbbbbb, v58\n\t" \
  "v_and_b32 v59, 0xbbbbbbbb, v59\n\t" \
  "v_and_b32 v60, 0xbbbbbbbb, v60\n\t" \
  "v_and_b32 v61, 0xbbbbbbbb, v61\n\t" \
  "v_and_b32 v62, 0xbbbbbbbb, v62\n\t" \
  "v_and_b32 v63, 0xbbbbbbbb, v63\n\t" \
  "v_or_b32 v0, 0x0, v0\n\t" \
  "v_or_b32 v1, 0x0, v1\n\t" \
  "v_or_b32 v2, 0x0, v2\n\t" \
  "v_or_b32 v3, 0x0, v3\n\t" \
  "v_or_b32 v4, 0x0, v4\n\t" \
  "v_or_b32 v5, 0x0, v5\n\t" \
  "v_or_b32 v6, 0x0, v6\n\t" \
  "v_or_b32 v7, 0x0, v7\n\t" \
  "v_or_b32 v8, 0x0, v8\n\t" \
  "v_or_b32 v9, 0x0, v9\n\t" \
  "v_or_b32 v10, 0x0, v10\n\t" \
  "v_or_b32 v11, 0x0, v11\n\t" \
  "v_or_b32 v12, 0x0, v12\n\t" \
  "v_or_b32 v13, 0x0, v13\n\t" \
  "v_or_b32 v14, 0x0, v14\n\t" \
  "v_or_b32 v15, 0x0, v15\n\t" \
  "v_or_b32 v16, 0x0, v16\n\t" \
  "v_or_b32 v17, 0x0, v17\n\t" \
  "v_or_b32 v18, 0x0, v18\n\t" \
  "v_or_b32 v19, 0x0, v19\n\t" \
  "v_or_b32 v20, 0x0, v20\n\t" \
  "v_or_b32 v21, 0x0, v21\n\t" \
  "v_or_b32 v22, 0x0, v22\n\t" \
  "v_or_b32 v23, 0x0, v23\n\t" \
  "v_or_b32 v24, 0x0, v24\n\t" \
  "v_or_b32 v25, 0x0, v25\n\t" \
  "v_or_b32 v26, 0x0, v26\n\t" \
  "v_or_b32 v27, 0x0, v27\n\t" \
  "v_or_b32 v28, 0x0, v28\n\t" \
  "v_or_b32 v29, 0x0, v29\n\t" \
  "v_or_b32 v30, 0x0, v30\n\t" \
  "v_or_b32 v31, 0x0, v31\n\t" \
  "v_or_b32 v32, 0x0, v32\n\t" \
  "v_or_b32 v33, 0x0, v33\n\t" \
  "v_or_b32 v34, 0x0, v34\n\t" \
  "v_or_b32 v35, 0x0, v35\n\t" \
  "v_or_b32 v36, 0x0, v36\n\t" \
  "v_or_b32 v37, 0x0, v37\n\t" \
  "v_or_b32 v38, 0x0, v38\n\t" \
  "v_or_b32 v39, 0x0, v39\n\t" \
  "v_or_b32 v40, 0x0, v40\n\t" \
  "v_or_b32 v41, 0x0, v41\n\t" \
  "v_or_b32 v42, 0x0, v42\n\t" \
  "v_or_b32 v43, 0x0, v43\n\t" \
  "v_or_b32 v44, 0x0, v44\n\t" \
  "v_or_b32 v45, 0x0, v45\n\t" \
  "v_or_b32 v46, 0x0, v46\n\t" \
  "v_or_b32 v47, 0x0, v47\n\t" \
  "v_or_b32 v48, 0x0, v48\n\t" \
  "v_or_b32 v49, 0x0, v49\n\t" \
  "v_or_b32 v50, 0x0, v50\n\t" \
  "v_or_b32 v51, 0x0, v51\n\t" \
  "v_or_b32 v52, 0x0, v52\n\t" \
  "v_or_b32 v53, 0x0, v53\n\t" \
  "v_or_b32 v54, 0x0, v54\n\t" \
  "v_or_b32 v55, 0x0, v55\n\t" \
  "v_or_b32 v56, 0x0, v56\n\t" \
  "v_or_b32 v57, 0x0, v57\n\t" \
  "v_or_b32 v58, 0x0, v58\n\t" \
  "v_or_b32 v59, 0x0, v59\n\t" \
  "v_or_b32 v60, 0x0, v60\n\t" \
  "v_or_b32 v61, 0x0, v61\n\t" \
  "v_or_b32 v62, 0x0, v62\n\t" \
  "v_or_b32 v63, 0x0, v63\n\t" \
  "s_mov_b32 s40, %[n1]\n\t" \
  "L_c1_%=:\n\t" \
  "v_mfma_f32_16x16x128_f8f6f4 a[0:3], v[0:7], v[32:39], a[0:3] blgp:1\n\t" \
  "v_mfma_f32_16x16x128_f8f6f4 a[4:7], v[0:7], v[40:47], a[4:7] blgp:1\n\t" \
  "v_mfma_f32_16x16x128_f8f6f4 a[8:11], v[0:7], v[48:55], a[8:11] blgp:1\n\t" \
  "v_mfma_f32_16x16x128_f8f6f4 a[12:15], v[0:7], v[56:63], a[12:15] blgp:1\n\t" \
  "v_mfma_f32_16x16x128_f8f6f4 a[16:19], v[8:15], v[32:39], a[16:19] blgp:1\n\t" \
  "v_mfma_f32_16x16x128_f8f6f4 a[20:23], v[8:15], v[40:47], a[20:23] blgp:1\n\t" \
  "v_mfma_f32_16x16x128_f8f6f4 a[24:27], v[8:15], v[48:55], a[24:27] blgp:1\n\t" \
  "v_mfma_f32_16x16x128_f8f6f4 a[28:31], v[8:15], v[56:63], a[28:31] blgp:1\n\t" \
  "v_mfma_f32_16x16x128_f8f6f4 a[32:35], v[16:23], v[32:39], a[32:35] blgp:1\n\t" \
  "v_mfma_f32_16x16x128_f8f6f4 a[36:39], v[16:23], v[40:47], a[36:39] blgp:1\n\t" \
  "v_mfma_f32_16x16x128_f8f6f4 a[40:43], v[16:23], v[48:55], a[40:43] blgp:1\n\t" \
  "v_mfma_f32_16x16x128_f8f6f4 a[44:47], v[16:23], v[56:63], a[44:47] blgp:1\n\t" \
  "v_mfma_f32_16x16x128_f8f6f4 a[48:51], v[24:31], v[32:39], a[48:51] blgp:1\n\t" \
  "v_mfma_f32_16x16x128_f8f6f4 a[52:55], v[24:31], v[40:47], a[52:55] blgp:1\n\t" \
  "v_mfma_f32_16x16x128_f8f6f4 a[56:59], v[24:31], v[48:55], a[56:59] blgp:1\n\t" \
  "v_mfma_f32_16x16x128_f8f6f4 a[60:63], v[24:31], v[56:63], a[60:63] blgp:1\n\t" \
  "s_nop 15\n\t" \
  "s_nop 15\n\t" \
  "s_sub_u32 s40, s40, 1\n\t" \
  "s_cmp_lg_u32 s40, 0\n\t" \
  "s_cbranch_scc1 L_c1_%=\n\t" \
  "s_memtime s[42:43]\n\t" \
  "s_memrealtime s[44:45]\n\t" \
  "s_mov_b32 s40, %[n2]\n\t" \
  "L_c2_%=:\n\t" \
  "v_mfma_f32_16x16x128_f8f6f4 a[0:3], v[0:7], v[32:39], a[0:3] blgp:1\n\t" \
  "v_mfma_f32_16x16x128_f8f6f4 a[4:7], v[0:7], v[40:47], a[4:7] blgp:1\n\t" \
  "v_mfma_f32_16x16x128_f8f6f4 a[8:11], v[0:7], v[48:55], a[8:11] blgp:1\n\t" \
  "v_mfma_f32_16x16x128_f8f6f4 a[12:15], v[0:7], v[56:63], a[12:15] blgp:1\n\t" \
  "v_mfma_f32_16x16x128_f8f6f4 a[16:19], v[8:15], v[32:39], a[16:19] blgp:1\n\t" \
  "v_mfma_f32_16x16x128_f8f6f4 a[20:23], v[8:15], v[40:47], a[20:23] blgp:1\n\t" \
  "v_mfma_f32_16x16x128_f8f6f4 a[24:27], v[8:15], v[48:55], a[24:27] blgp:1\n\t" \
  "v_mfma_f32_16x16x128_f8f6f4 a[28:31], v[8:15], v[56:63], a[28:31] blgp:1\n\t" \
  "v_mfma_f32_16x16x128_f8f6f4 a[32:35], v[16:23], v[32:39], a[32:35] blgp:1\n\t" \
  "v_mfma_f32_16x16x128_f8f6f4 a[36:39], v[16:23], v[40:47], a[36:39] blgp:1\n\t" \
  "v_mfma_f32_16x16x128_f8f6f4 a[40:43], v[16:23], v[48:55], a[40:43] blgp:1\n\t" \
  "v_mfma_f32_16x16x128_f8f6f4 a[44:47], v[16:23], v[56:63], a[44:47] blgp:1\n\t" \
  "v_mfma_f32_16x16x128_f8f6f4 a[48:51], v[24:31], v[32:39], a[48:51] blgp:1\n\t" \
  "v_mfma_f32_16x16x128_f8f6f4 a[52:55], v[24:31], v[40:47], a[52:55] blgp:1\n\t" \
  "v_mfma_f32_16x16x128_f8f6f4 a[56:59], v[24:31], v[48:55], a[56:59] blgp:1\n\t" \
  "v_mfma_f32_16x16x128_f8f6f4 a[60:63], v[24:31], v[56:63], a[60:63] blgp:1\n\t" \
  "s_nop 15\n\t" \
  "s_nop 15\n\t" \
  "s_sub_u32 s40, s40, 1\n\t" \
  "s_cmp_lg_u32 s40, 0\n\t" \
  "s_cbranch_scc1 L_c2_%=\n\t" \
  "s_memtime s[46:47]\n\t" \
  "s_memrealtime s[48:49]\n\t" \
  "s_waitcnt lgkmcnt(0)\n\t" \
  "s_sub_u32 %[cyc], s46, s42\n\t" \
  "s_sub_u32 %[rt], s48, s44\n\t"
#define CALIB_CLOBBERS "v0","v1","v2","v3","v4","v5","v6","v7","v8","v9","v10","v11","v12","v13","v14","v15","v16","v17","v18","v19","v20","v21","v22","v23","v24","v25","v26","v27","v28","v29","v30","v31","v32","v33","v34","v35","v36","v37","v38","v39","v40","v41","v42","v43","v44","v45","v46","v47","v48","v49","v50","v51","v52","v53","v54","v55","v56","v57","v58","v59","v60","v61","v62","v63","a0","a1","a2","a3","a4","a5","a6","a7","a8","a9","a10","a11","a12","a13","a14","a15","a16","a17","a18","a19","a20","a21","a22","a23","a24","a25","a26","a27","a28","a29","a30","a31","a32","a33","a34","a35","a36","a37","a38","a39","a40","a41","a42","a43","a44","a45","a46","a47","a48","a49","a50","a51","a52","a53","a54","a55","a56","a57","a58","a59","a60","a61","a62","a63","s40","s41","s42","s43","s44","s45","s46","s47","s48","s49","scc"
#define NOGEMM 0
__constant__ const unsigned TILE_TAB[256][8] = {
{0,0,0,0,0,0,0,0},{0,0,0,0,0,0,0,0},{0,0,0,0,0,0,0,0},{0,0,0,0,0,0,0,0},
{0,0,0,0,0,0,0,0},{0,0,0,0,0,0,0,0},{0,0,0,0,0,0,0,0},{0,0,0,0,0,0,0,0},
{0,0,0,0,0,0,0,0},{0,0,0,0,0,0,0,0},{0,0,0,0,0,0,0,0},{0,0,0,0,0,0,0,0},
{0,0,0,0,0,0,0,0},{0,0,0,0,0,0,0,0},{0,0,0,0,0,0,0,0},{0,0,0,0,0,0,0,0},
{0,0,0,0,0,0,0,0},{0,0,0,0,0,0,0,0},{0,0,0,0,0,0,0,0},{0,0,0,0,0,0,0,0},
{0,0,0,0,0,0,0,0},{0,0,0,0,0,0,0,0},{0,0,0,0,0,0,0,0},{0,0,0,0,0,0,0,0},
{0,0,0,0,0,0,0,0},{0,0,0,0,0,0,0,0},{0,0,0,0,0,0,0,0},{0,0,0,0,0,0,0,0},
{0,0,0,0,0,0,0,0},{0,0,0,0,0,0,0,0},{0,0,0,0,0,0,0,0},{0,0,0,0,0,0,0,0},
{4,4,96,512,256,0,0,0},{4,8,100,516,260,0,0,0},{4,390,104,520,264,0,0,0},{4,388,108,524,268,0,0,0},
{4,12,495,911,655,0,0,0},{4,220,491,907,651,0,0,0},{4,416,487,903,647,0,0,0},{4,424,483,899,643,0,0,0},
{4,5,272,864,608,0,0,0},{4,9,276,868,612,0,0,0},{4,394,280,872,616,0,0,0},{4,738,284,876,620,0,0,0},
{4,13,671,239,1007,0,0,0},{4,396,667,235,1003,0,0,0},{4,420,663,231,999,0,0,0},{4,428,659,227,995,0,0,0},
{4,6,448,16,784,0,0,0},{4,10,452,20,788,0,0,0},{4,744,456,24,792,0,0,0},{4,912,460,28,796,0,0,0},
{5,65536,14,847,415,159,0,0},{5,65538,748,843,411,155,0,0},{5,66067,592,839,407,151,0,0},{5,66065,600,835,403,147,0,0},
{4,7,624,192,960,0,0,0},{4,11,628,196,964,0,0,0},{4,918,632,200,968,0,0,0},{4,916,636,204,972,0,0,0},
{5,65888,15,1023,591,335,0,0},{5,65890,924,1019,587,331,0,0},{5,66243,596,1015,583,327,0,0},{5,66241,604,1011,579,323,0,0},
{4,884,800,368,112,0,0,0},{4,888,804,372,116,0,0,0},{4,922,808,376,120,0,0,0},{4,66,812,380,124,0,0,0},
{5,65537,892,175,767,511,0,0},{5,65539,76,171,763,507,0,0},{5,66066,768,167,759,503,0,0},{5,66064,776,163,755,499,0,0},
{4,36,976,544,288,0,0,0},{4,40,980,548,292,0,0,0},{4,73,984,552,296,0,0,0},{4,70,988,556,300,0,0,0},
{5,65889,44,351,943,687,0,0},{5,65891,252,347,939,683,0,0},{5,66242,772,343,935,679,0,0},{5,66240,780,339,931,675,0,0},
{4,212,128,720,464,0,0,0},{4,216,132,724,468,0,0,0},{4,248,136,728,472,0,0,0},{4,243,140,732,476,0,0,0},
{4,364,494,95,863,0,0,0},{4,221,490,91,859,0,0,0},{4,417,486,87,855,0,0,0},{4,425,482,83,851,0,0,0},
{4,356,97,513,257,0,0,0},{4,360,101,517,261,0,0,0},{4,391,105,521,265,0,0,0},{4,389,109,525,269,0,0,0},
{4,365,670,910,654,0,0,0},{4,397,666,906,650,0,0,0},{4,421,662,902,646,0,0,0},{4,429,658,898,642,0,0,0},
{4,357,273,865,609,0,0,0},{4,361,277,869,613,0,0,0},{4,395,281,873,617,0,0,0},{4,739,285,877,621,0,0,0},
{5,196608,366,846,238,1006,0,0},{5,196610,749,842,234,1002,0,0},{5,197139,593,838,230,998,0,0},{5,197137,601,834,226,994,0,0},
{4,358,449,17,785,0,0,0},{4,362,453,21,789,0,0,0},{4,745,457,25,793,0,0,0},{4,913,461,29,797,0,0,0},
{5,196960,367,1022,414,158,0,0},{5,196962,925,1018,410,154,0,0},{5,197315,597,1014,406,150,0,0},{5,197313,605,1010,402,146,0,0},
{4,359,625,193,961,0,0,0},{4,363,629,197,965,0,0,0},{4,919,633,201,969,0,0,0},{4,917,637,205,973,0,0,0},
{5,196609,893,174,590,334,0,0},{5,196611,77,170,586,330,0,0},{5,197138,769,166,582,326,0,0},{5,197136,777,162,578,322,0,0},
{4,885,801,369,113,0,0,0},{4,889,805,373,117,0,0,0},{4,923,809,377,121,0,0,0},{4,67,813,381,125,0,0,0},
{5,196961,45,350,766,510,0,0},{5,196963,253,346,762,506,0,0},{5,197314,773,342,758,502,0,0},{5,197312,781,338,754,498,0,0},
{4,37,977,545,289,0,0,0},{4,41,981,549,293,0,0,0},{4,74,985,553,297,0,0,0},{4,240,989,557,301,0,0,0},
{4,540,493,942,686,0,0,0},{4,222,489,938,682,0,0,0},{4,418,485,934,678,0,0,0},{4,426,481,930,674,0,0,0},
{4,213,129,721,465,0,0,0},{4,217,133,725,469,0,0,0},{4,249,137,729,473,0,0,0},{4,244,141,733,477,0,0,0},
{4,541,669,94,862,0,0,0},{4,398,665,90,858,0,0,0},{4,422,661,86,854,0,0,0},{4,430,657,82,850,0,0,0},
{4,532,98,514,258,0,0,0},{4,536,102,518,262,0,0,0},{4,392,106,522,266,0,0,0},{4,736,110,526,270,0,0,0},
{5,327680,542,845,909,653,0,0},{5,327682,750,841,905,649,0,0},{5,328211,594,837,901,645,0,0},{5,328209,602,833,897,641,0,0},
{4,533,274,866,610,0,0,0},{4,537,278,870,614,0,0,0},{4,742,282,874,618,0,0,0},{4,740,286,878,622,0,0,0},
{5,328032,543,1021,237,1005,0,0},{5,328034,926,1017,233,1001,0,0},{5,328387,598,1013,229,997,0,0},{5,328385,606,1009,225,993,0,0},
{4,534,450,18,786,0,0,0},{4,538,454,22,790,0,0,0},{4,746,458,26,794,0,0,0},{4,914,462,30,798,0,0,0},
{5,327681,894,173,413,157,0,0},{5,327683,78,169,409,153,0,0},{5,328210,770,165,405,149,0,0},{5,328208,778,161,401,145,0,0},
{4,535,626,194,962,0,0,0},{4,539,630,198,966,0,0,0},{4,920,634,202,970,0,0,0},{4,64,638,206,974,0,0,0},
{5,328033,46,349,589,333,0,0},{5,328035,254,345,585,329,0,0},{5,328386,774,341,581,325,0,0},{5,328384,782,337,577,321,0,0},
{4,886,802,370,114,0,0,0},{4,890,806,374,118,0,0,0},{4,71,810,378,122,0,0,0},{4,68,814,382,126,0,0,0},
{4,716,492,765,509,0,0,0},{4,223,488,761,505,0,0,0},{4,419,484,757,501,0,0,0},{4,427,480,753,497,0,0,0},
{4,38,978,546,290,0,0,0},{4,42,982,550,294,0,0,0},{4,75,986,554,298,0,0,0},{4,241,990,558,302,0,0,0},
{4,717,668,941,685,0,0,0},{4,399,664,937,681,0,0,0},{4,423,660,933,677,0,0,0},{4,431,656,929,673,0,0,0},
{4,214,130,722,466,0,0,0},{4,218,134,726,470,0,0,0},{4,250,138,730,474,0,0,0},{4,245,142,734,478,0,0,0},
{5,458752,718,844,93,861,0,0},{5,458754,751,840,89,857,0,0},{5,459283,595,836,85,853,0,0},{5,459281,603,832,81,849,0,0},
{4,708,99,515,259,0,0,0},{4,712,103,519,263,0,0,0},{4,393,107,523,267,0,0,0},{4,737,111,527,271,0,0,0},
{5,459104,719,1020,908,652,0,0},{5,459106,927,1016,904,648,0,0},{5,459459,599,1012,900,644,0,0},{5,459457,607,1008,896,640,0,0},
{4,709,275,867,611,0,0,0},{4,713,279,871,615,0,0,0},{4,743,283,875,619,0,0,0},{4,741,287,879,623,0,0,0},
{5,458753,895,172,236,1004,0,0},{5,458755,79,168,232,1000,0,0},{5,459282,771,164,228,996,0,0},{5,459280,779,160,224,992,0,0},
{4,710,451,19,787,0,0,0},{4,714,455,23,791,0,0,0},{4,747,459,27,795,0,0,0},{4,915,463,31,799,0,0,0},
{5,459105,47,348,412,156,0,0},{5,459107,255,344,408,152,0,0},{5,459458,775,340,404,148,0,0},{5,459456,783,336,400,144,0,0},
{4,711,627,195,963,0,0,0},{4,715,631,199,967,0,0,0},{4,921,635,203,971,0,0,0},{4,65,639,207,975,0,0,0},
{4,880,1024,588,332,0,0,0},{4,32,9216,584,328,0,0,0},{4,208,17408,580,324,0,0,0},{4,384,25600,576,320,0,0,0},
{4,887,803,371,115,0,0,0},{4,891,807,375,119,0,0,0},{4,72,811,379,123,0,0,0},{4,69,815,383,127,0,0,0},
{4,881,3072,764,508,0,0,0},{4,33,11264,760,504,0,0,0},{4,209,19456,756,500,0,0,0},{4,385,27648,752,496,0,0,0},
{4,39,979,547,291,0,0,0},{4,43,983,551,295,0,0,0},{4,247,987,555,299,0,0,0},{4,242,991,559,303,0,0,0},
{4,882,5120,940,684,0,0,0},{4,34,13312,936,680,0,0,0},{4,210,21504,932,676,0,0,0},{4,386,29696,928,672,0,0,0},
{4,215,131,723,467,0,0,0},{4,219,135,727,471,0,0,0},{4,251,139,731,475,0,0,0},{4,246,143,735,479,0,0,0},
{4,883,7168,92,860,0,0,0},{4,35,15360,88,856,0,0,0},{4,211,23552,84,852,0,0,0},{4,387,31744,80,848,0,0,0}};
__constant__ const unsigned UNIT_TAB[NPLANS * NUNITS] = {
4096u,4112u,4097u,4113u,16384u,38912u,50176u,61440u,5120u,5136u,5121u,5137u,6144u,6160u,6145u,6161u,
16416u,38944u,50208u,61472u,7168u,7184u,7169u,7185u,72704u,786432u,1703952u,2621441u,3538961u,4474880u,5390400u,6330432u,
7259200u,8187968u,9046016u,9963536u,10881025u,11667473u,12612608u,13501440u,14418960u,15336449u,16253969u,17243168u,18091008u,19008528u,19926017u,20843537u,
21776416u,22552576u,23470096u,24387585u,25305105u,26230880u,27170912u,28099680u,29028448u,29914144u,30811136u,31728656u,32646145u,33563665u,34413632u,35258496u,
36176016u,37093505u,38011025u,38944896u,39856128u,40773648u,41691137u,42608657u,43534400u,44472448u,45362176u,46148624u,47066113u,47983633u,48906400u,49809536u,
50727056u,51644545u,52562065u,53481600u,54399120u,55316609u,56234129u,57066624u,57995392u,58881088u,59769984u,60687504u,61604993u,62522513u,63477920u,64361600u,
65279120u,66196609u,67114129u,68042944u,68885600u,69733504u,70651024u,71568513u,72486033u,73406592u,74324112u,75241601u,76159121u,77088864u,78026944u,78921952u,
79742112u,80670880u,81534080u,82451600u,83369089u,84286609u,85226592u,86153440u,87032064u,87949584u,88867073u,89784593u,90718464u,91430016u,92062912u,92817536u,
93493504u,94257440u,94898432u,95684880u,96471297u,97388817u,98234560u,98878688u,99644544u,100271360u,101188880u,101975297u,102761745u,103586080u,104350016u,105050336u,
105847968u,106433792u,107220240u,108006657u,108924177u,109757696u,110403584u,111188288u,111823008u,112599168u,113385616u,114172033u,114958481u,115884384u,116584704u,117359904u,
117994656u,118629504u,119547024u,120333441u,121119889u,121936224u,122683776u,123470224u,124256641u,125174161u,125976960u,126677280u,127474880u,128107840u,128854144u,129640592u,
130427009u,131213457u,132161568u,132815232u,133448096u,134236352u,134884480u,135802000u,136588417u,137374865u,138152320u,139069840u,139856257u,140642705u,141488448u,142263648u,
142872832u,143790352u,144576769u,145363217u,146146688u,147064208u,147850625u,148637073u,149451968u,150247840u,150880704u,151712096u,152346688u,152966400u,153883920u,154670337u,
155456785u,156372352u,157158800u,157945217u,158731665u,159696256u,160373984u,160995776u,161759712u,162404608u,163322128u,164108545u,164894993u,165693664u,166523264u,167167392u,
167910656u,168697104u,169483521u,170269969u,171209952u,171874784u,172491264u,173408784u,174195201u,174981649u,175915520u,176595040u,177271200u,178046400u,178724096u,179476992u,
180109856u,180750848u,181668368u,182454785u,183241233u,184086976u,184830208u,185517536u,186254848u,187041296u,187827713u,188745233u,189571200u,190224928u,190988864u,191689184u,
192286208u,193203728u,193990145u,194776593u,195610112u,196385344u,197018208u,197849600u,198493728u,199270560u,199924320u,200540800u,201458320u,202244737u,203031185u,203833984u,
204665376u,205309504u,206084736u,206717600u,207358592u,208276112u,209062529u,209848977u,210825792u,211460288u,212125280u,212933920u,213517952u,214304400u,215221889u,216008337u,
216832672u,217596608u,218264832u,218952288u,219680384u,220466832u,221253249u,222170769u,223004288u,223627552u,224434880u,225067744u,225878240u,226554496u,227198624u,228007232u,
228629216u,229405984u,230032128u,230818576u,231736065u,232522513u,233325312u,234156704u,234800832u,235424064u,236232960u,236886784u,237650720u,238291712u,239078160u,239864577u,
240782097u,241627840u,242294112u,243058400u,243726336u,244479296u,245106432u,245892880u,246810369u,247596817u,248421152u,249185088u,249885408u,250482432u,251399952u,252186369u,
252972817u,253806336u,254560608u,255238432u,256023360u,256713728u,257311584u,258153856u,258798336u,259586080u,260228896u,260994400u,261640192u,262314848u,263062400u,263848848u,
264635265u,265552785u,266355584u,267012480u,267842336u,268467200u,269141824u,269918528u,270607392u,271358848u,272047520u,272647072u,273419136u,274205584u,274992001u,275909521u,
276756544u,277410624u,278164864u,278841184u,279597088u,280233856u,281020304u,281937793u,282724241u,283548576u,284182944u,284967872u,285647200u,286424096u,287109984u,287838080u,
288624528u,289410945u,290197393u,291161984u,291780608u,292567056u,293484545u,294270993u,295117888u,295902656u,296524736u,297157600u,297990240u,298612128u,299430784u,300039168u,
300825616u,301612033u,302529553u,303351712u,303976512u,304754688u,305541136u,306327553u,307245073u,308032512u,308818960u,309605377u,310522897u,311314880u,311991264u,312779136u,
313424960u,314241952u,314886080u,315565152u,316349920u,316986368u,317748672u,318437504u,319091648u,319836256u,320522208u,321276384u,321954208u,322731104u,323417056u,324083200u,
324871296u,325517312u,326292512u,326923744u,327612576u,328355968u,329033728u,329796096u,330473920u,331119744u,331947552u,332604576u,333381664u,334001664u,334690496u,335433888u,
336111648u,336874016u,337551840u,338174208u,339091728u,339878145u,340664593u,341605536u,342302272u,342959296u,343680256u,344466704u,345253121u,346170641u,346977824u,347666656u,
348410048u,349055232u,349841680u,350759169u,351545617u,352364608u,353126976u,353804800u,354450624u,355216640u,356003088u,356789505u,357707025u,358555232u,359212256u,359953984u,
360591488u,361377936u,362295425u,363081873u,363919616u,364696640u,365318368u,365996128u,366780960u,367413856u,368194688u,368981136u,369767553u,370554001u,371472512u,372258960u,
373045377u,373831825u,374750336u,375536784u,376323201u,377240721u,378043616u,378740352u,379483744u,380183808u,380827936u,381616192u,382260320u,383035520u,383689280u,384323840u,
385108608u,385776896u,386604704u,387191168u,387977616u,388895105u,389681553u,390493824u,391324960u,391910784u,392697232u,393614721u,394401169u,395245888u,396034144u,396678272u,
397322400u,398107232u,398740128u,399528224u,400169344u,400955792u,401742209u,402659729u,403448192u,404234640u,405152129u,405938577u,406748448u,407576256u,408188576u,408821120u,
409738640u,410525057u,411311505u,412165440u,412886400u,413672848u,414459265u,415376785u,416219264u,416872800u,417649824u,418292352u,419080384u,419711680u,420368704u,421144960u,
421931408u,422717825u,423635345u,424484576u,425098560u,425883328u,426519936u,427306384u,428223873u,429010321u,429797632u,430584080u,431501569u,432288017u,433136992u,433914016u,
434558144u,435200672u,435988704u,436664704u,437406432u,438063456u,438711552u,439629072u,440415489u,441201937u,442120448u,442906896u,443693313u,444479761u,445398272u,446184720u,
446971137u,447757585u,448732928u,449346912u,450131680u,450831744u,451477696u,452240064u,452908256u,453683456u,454359456u,454970112u,455758208u,456466208u,457209600u,457866624u,
458494464u,459411984u,460198401u,460984849u,461972896u,462618848u,463250144u,464049408u,464693536u,465455904u,466155968u,466753024u,467670544u,468456961u,469243409u,470045088u,
470884160u,471472640u,472390160u,473176577u,473963025u,474773280u,475537920u,476324368u,477110785u,478028305u,478838176u,479536576u,480313600u,480944896u,481613088u,482352640u,
483139088u,483925505u,484843025u,485665088u,486286848u,487204368u,487990785u,488777233u,489573184u,490404320u,491016640u,491792896u,492579344u,493365761u,494152209u,495132512u,
495744832u,496532928u,497168768u,497955216u,498872705u,499659153u,500444672u,501231120u,502148609u,502935057u,503751456u,504561952u,505226720u,505861440u,506636640u,507267936u,
508099072u,508704128u,509490576u,510408065u,511194513u,511988192u,512637312u,513554832u,514341249u,515127697u,516046208u,516832656u,517619073u,518405521u,519380864u,519993184u,
520650208u,521446208u,522125632u,522900832u,523544960u,524232192u,524962688u,525662752u,526406144u,527114144u,527857536u,528514560u,529179488u,529989984u,530634112u,531409312u,
532025984u,532812432u,533598849u,534516369u,535373344u,535972768u,536803904u,537400960u,538318480u,539104897u,539891345u,540693024u,541532096u,542144416u,542907008u,543693456u,
544479873u,545266321u,546185856u,546972304u,547758721u,548545169u,549486112u,550151040u,550961536u,551605664u,552249792u,553012160u,553723456u,554442368u,555228816u,556015233u,
556801681u,557721216u,558507664u,559294081u,560080529u,561052256u,561664576u,562503648u,563115968u,563751552u,564669072u,565455489u,566241937u,567030272u,567947792u,568734209u,
569520657u,570467232u,571113024u,571802016u,572577216u,573221344u,573976192u,574762640u,575549057u,576335505u,577260512u,577971808u,578565632u,579483152u,580269569u,581056017u,
582023808u,582629888u,583416336u,584202753u,585120273u,585907712u,586694160u,587611649u,588398097u,589189728u,589985760u,590692352u,591307712u,592118208u,592739936u,593548768u,
594192896u,594880128u,595655328u,596267648u,597114912u,597730272u,598409696u,599184896u,599829024u,600604288u,601302688u,601887488u,602805008u,603591425u,604377873u,605354688u,
606027840u,606622368u,607393536u,608179984u,608966401u,609883921u,610730496u,611374624u,612149824u,612794016u,613425920u,614343440u,615129857u,615916305u,616831744u,617618192u,
618404609u,619191057u,620177088u,620766976u,621553424u,622470913u,623257361u,624041728u,624828176u,625745665u,626532113u,627374816u,628179040u,628773568u,629420672u,630338192u,
631124609u,631911057u,632695552u,633613072u,634399489u,635185937u,636158496u,636802624u,637446752u,638222016u,638856960u,639643408u,640560897u,641347345u,642197216u,642922112u,
643708560u,644494977u,645412497u,646199936u,646986384u,647772801u,648690321u,649477760u,650264208u,651050625u,651968145u,652802816u,653475968u,654201568u,654901824u,655677024u,
656321152u,656965344u,657794816u,658438944u,659243168u,659837696u,660537952u,661313152u,661957280u,662732544u,663430944u,664015744u,664933264u,665719681u,666506129u,667500736u,
668138304u,668750624u,669521792u,670308240u,671094657u,672012177u,672858752u,673502880u,674278080u,674922272u,675554176u,676471696u,677258113u,678044561u,678960000u,679746448u,
680532865u,681319313u,682305344u,682967264u,683604832u,684337024u,685123472u,685909889u,686827409u,687611776u,688398224u,689184641u,690102161u,690901824u,691548928u,692466448u,
693252865u,694039313u,695009952u,695654080u,696298208u,697052032u,697838480u,698624897u,699542417u,700350272u,700987136u,701773584u,702691073u,703477521u,704264960u,705182480u,
705968897u,706755345u,707542784u,708460304u,709246721u,710033169u,710879072u,711602048u,712388496u,713174913u,714092433u,714948864u,715586432u,716329824u,717030080u,717805280u,
718449408u,719093600u,719923072u,720584992u,721353632u,722010848u,722621312u,723441408u,724085536u,724860800u,725559200u,726221120u,726947840u,727645120u,728433408u,729077536u,
729852736u,730485664u,731153408u,731938720u,732643680u,733392896u,734078912u,734681120u,735509472u,736166688u,736899968u,737686416u,738472833u,739259281u,740218688u,740862816u,
741530624u,742282176u,742949920u,743770112u,744372096u,745158544u,746076033u,746862481u,747667392u,748436352u,749222800u,750009217u,750795665u,751714176u,752500624u,753287041u,
754073489u,755057024u,755675168u,756492256u,757150720u,757749824u,758580032u,759224160u,759999360u,760667168u,761287648u,762077184u,762741824u,763562016u,764182496u,764887456u,
765636672u,766325792u,767099744u,767743872u,768366688u,769174432u,769842240u,770596896u,771261536u,772081728u,772751808u,773369952u,774177664u,774845504u,775608224u,776252352u,
776886400u,777706592u,778330176u,779125888u,779815008u,780485088u,781234304u,781910944u,782686144u,783365216u,783985632u,784775168u,785406112u,786226304u,786849888u,787514528u,
788325376u,789004800u,789776512u,790409376u,791086016u,791861216u,792530944u,793294880u,793982080u,794712256u,795401376u,796024960u,796867584u,797500448u,798262464u,798966304u,
799606944u,800370880u,801047520u,801794048u,802492448u,803125312u,803943584u,804542688u,805362880u,805999616u,806698016u,807428256u,808117312u,808879328u,809583168u,810223808u,
810969120u,811643104u,812453952u,813086816u,813774016u,814504192u,815193312u,815961120u,816659520u,817423456u,818045120u,818758240u,819496192u,820185312u,820930624u,821629024u,
822261888u,823046400u,823735520u,824465696u,825154816u,825791552u,826621024u,827253888u,828064384u,828661984u,829326624u,830146816u,830761056u,831590528u,832223392u,832876832u,
833697024u,834296128u,835097696u,835796096u,836427040u,837215392u,837894816u,838623488u,839288128u,840108320u,840722560u,841420960u,842184896u,842838336u,843658528u,844257632u,
844928128u,845757600u,846390464u,847174976u,847856320u,848453920u,849249632u,849938752u,850684064u,851382464u,852015328u,852799840u,853488960u,854219136u,854889632u,855588032u,
856351968u,857031392u,857791840u,858415424u,859080064u,859859136u,860557536u,861342048u,861976832u,862761344u,863450464u,864064704u,864894176u,865491360u,866313472u,866992896u,
867622272u,868376928u,869041568u,869820640u,870519040u,871151904u,871958912u,872591776u,873370848u,874067328u,874724608u,875488544u,876108224u,876954400u,877583776u,878207360u,
879003072u,879651072u,880480544u,881113408u,881789344u,882553280u,883201280u,884030752u,884663616u,885470624u,886129472u,886725088u,887545280u,888168864u,888957216u,889655616u,
890275296u,891074912u,891750848u,892514784u,893162784u,893861184u,894625120u,895301056u,896090976u,896686592u,897375712u,898132288u,898785728u,899617120u,900249984u,900892160u,
901712352u,902337856u,903131648u,903822688u,904455552u,905266048u,905917920u,906648096u,907337216u,908093792u,908792192u,909425056u,910189024u,910853664u,911673856u,912299360u,
912997760u,913761696u,914403872u,915227552u,915879424u,916478528u,917298720u,917924224u,918753696u,919386560u,920019456u,920815168u,921504288u,922260864u,922959264u,923592128u,
924365376u,925057984u,925840928u,926440032u,927099296u,927915584u,928584128u,929348064u,929980960u,930776672u,931465792u,932091296u,932920768u,933553632u,934364128u,934982240u,
935671360u,936401536u,937060800u,937890272u,938523136u,939187808u,939942464u,940607104u,941427296u,942052800u,942751200u,943515136u,944164864u,944962560u,945629184u,946230272u,
947040896u,947730016u,948460192u,949119456u,949817856u,950581792u,951246464u,952001120u,952665760u,953456096u,954154496u,954796672u,955573792u,956223520u,957021216u,957687840u,
958288928u,959099552u,959788672u,960522752u,961174208u,961876512u,962640448u,963305120u,964059776u,964724416u,965383680u,966213152u,966846016u,967641760u,968274624u,968963744u,
969697824u,970396224u,971160160u,971791072u,972480192u,973234848u,973899488u,974689824u,975388224u,976152160u,976816832u,977449696u,978217536u,978915936u,979711680u,980335232u,
980966144u,981786336u,982409920u,983209536u,983860992u,984563296u,985327232u,985991904u,986755840u,987392608u,988091008u,988854944u,989542112u,990272288u,990961408u,991598176u,
992371424u,993083008u,993846944u,994477856u,995166976u,995930912u,996567680u,997397152u,998030016u,998848256u,999447360u,1000136480u,1000904320u,1001602720u,1002366656u,1002988288u,
1003652928u,1004473120u,1005087392u,1005916864u,1006547776u,1007205088u,1008023328u,1008622432u,1009442624u,1010079392u,1010777792u,1011541728u,1012163360u,1012959072u,1013648192u,1014262464u,
1015091936u,1015724800u,1016509280u,1017198400u,1017797504u,1018617696u,1019254464u,1020083936u,1020716800u,1021469504u,1022134144u,1022823264u,1023568608u,1024267008u,1025030944u,1025684352u,
1026373472u,1027103648u,1027774176u,1028603648u,1029236512u,1030020992u,1030644576u,1031309216u,1032129408u,1032743680u,1033573152u,1034206016u,1034990496u,1035679616u,1036409792u,1037080320u,
1037778720u,1038542656u,1039196064u,1039950720u,1040615360u,1041435552u,1042049824u,1042879296u,1043512160u,1044165568u,1044985760u,1045584864u,1046386464u,1047084864u,1047848800u,1048502208u,
1049256864u,1049921504u,1050569536u,1051399008u,1052031872u,1052838848u,1053471712u,1054291904u,1054906176u,1055735648u,1056358400u,1057023872u,1057808352u,1058431936u,1059264512u,1059875680u,
1060705152u,1061338016u,1062144992u,1062749184u,1063425888u,1064253408u,1064910720u,1065674656u,1066319904u,1067085824u,1067738080u,1068570656u,1069181824u,1069880224u,1070644160u,1071291392u,
1072055328u,1072732032u,1073561504u,1074194368u,1074972672u,1075626048u,1076260896u,1077081088u,1077701536u,1078531008u,1079187520u,1079950304u,1080597536u,1081361472u,1082038176u,1082736576u,
1083500512u,1084147744u,1084932192u,1085567040u,1086352320u,1087042592u,1087837152u,1088495616u,1089149024u,1089903680u,1090557888u,1091322976u,1092042720u,1092777984u,1093453888u,1094238336u,
1094873184u,1095658464u,1096328192u,1097015328u,1097790528u,1098455168u,1099209824u,1099864032u,1100642336u,1101284480u,1102104672u,1102758048u,1103392896u,1104192544u,1104879680u,1105656896u,
1106323520u,1107055680u,1107752032u,1108547744u,1109171328u,1109817408u,1110590624u,1111279744u,1112064192u,1112699040u,1113498688u,1114185824u,1114963040u,1115629664u,1116230752u,1117058176u,
1117722816u,1118477472u,1119146048u,1119909984u,1120552128u,1121372320u,1122025696u,1122673760u,1123491968u,1124138112u,1124935808u,1125536896u,1126330560u,1127019680u,1127815392u,1128438976u,
1129107552u,1129871488u,1130513632u,1131333824u,1131987200u,1132766336u,1133453472u,1134230688u,1134897312u,1135498400u,1136292064u,1136981184u,1137776896u,1138413696u,1139186912u,1139832992u,
1140606208u,1141295328u,1141948704u,1142727840u,1143414976u,1144192192u,1144858816u,1145590976u,1146253568u,1147073760u,1147738400u,1148375200u,1149139136u,1149803776u,1150567712u,1151256832u,
1152033984u,1152721120u,1153498336u,1154164960u,1154766048u,1155580224u,1156215072u,1157035264u,1157699904u,1158467776u,1159100640u,1159896352u,1160529216u,1161209056u,1162004768u,1162682624u,
1163459840u,1164126464u,1164858624u,1165541728u,1166307648u,1166996768u,1167642848u,1168447840u,1169062144u,1169857856u,1170490720u,1171301632u,1171988768u,1172765984u,1173432608u,1174033696u,
1174849856u,1175503232u,1176269152u,1176958272u,1177735424u,1178368288u,1179195776u,1179819360u,1180452224u,1181263136u,1181950272u,1182727488u,1183394112u,1184126272u,1184811360u,1185595808u,
1186230656u,1187041568u,1187674432u,1188361568u,1189157280u,1189780864u,1190569280u,1191200160u,1192042848u,1192688992u,1193486688u,1194087776u,1194772864u,1195557312u,1196192160u,1197003072u,
1197635936u,1198454144u,1199118784u,1199873440u,1200530784u,1201217920u,1201995136u,1202661760u,1203393920u,1204045248u,1204865440u,1205518816u,1206284736u,1206964576u,1207597440u,1208415648u,
1209080288u,1209834944u,1210492288u,1211310496u,1211956640u,1212754336u,1213355424u,1214006752u,1214826944u,1215480320u,1216270720u,1216903584u,1217688032u,1218377152u,1219172864u,1219796448u,
1220453792u,1221272000u,1221918144u,1222715840u,1223316928u,1224099328u,1224788448u,1225572896u,1226232224u,1226865088u,1227649536u,1228338656u,1229134368u,1229757952u,1230546368u,1231233504u,
1232010720u,1232677344u,1233278432u,1234060832u,1234749952u,1235534400u,1236193728u,1236957664u,1237611040u,1238431232u,1239095872u,1239721440u,1240539648u,1241185792u,1241983488u,1242584576u,
1243389472u,1244022336u,1244842528u,1245499872u,1246132736u,1246937696u,1247572544u,1248392736u,1249057376u,1249814016u,1250501152u,1251278368u,1251944992u,1252546080u,1253350976u,1253983840u,
1254806016u,1255459392u,1256225312u,1256899200u,1257665120u,1258354240u,1259018880u,1259775520u,1260462656u,1261239872u,1261906496u,1262638656u,1263312480u,1264076416u,1264767520u,1265400384u,
1266207328u,1266860704u,1267626624u,1268315744u,1269081664u,1269768800u,1270546016u,1271212640u,1271813728u,1272650400u,1273273984u,1274037920u,1274729024u,1275492960u,1276168832u,1276953280u,
1277588128u,1278387808u,1279063680u,1279730304u,1280507520u,1281174144u,1281906304u,1282611904u,1283366560u,1284035168u,1284785856u,1285454464u,1286130336u,1286914784u,1287549632u,1288349312u,
1289036448u,1289813664u,1290480288u,1291212448u,1291908768u,1292573408u,1293328064u,1293996672u,1294760608u,1295402720u,1296222912u,1296876288u,1297642208u,1298310816u,1298997952u,1299775168u,
1300441792u,1301173952u,1301870272u,1302665984u,1303289568u,1304089248u,1304722112u,1305364224u,1306184416u,1306837792u,1307616960u,1308259072u,1309090528u,1309736672u,1310534368u,1311135456u,
1311831776u,1312627488u,1313251072u,1314050752u,1314683616u,1315456800u,1316145920u,1316930368u,1317578464u,1318265600u,1319042816u,1319709440u,1320441600u,1321104160u,1321924352u,1322588992u,
1323343648u,1324012256u,1324645120u,1325418304u,1326107424u,1326891872u,1327539968u,1328358176u,1329004320u,1329802016u,1330403104u,1331065664u,1331885856u,1332550496u,1333318400u,1333951264u,
1334746944u,1335379808u,1336200000u,1336846112u,1337508736u,1338319680u,1338965824u,1339763520u,1340364608u,1341158240u,1341847360u,1342643072u,1343279904u,1343912768u,1344708448u,1345341312u,
1346161504u,1346807616u,1347625824u,1348271968u,1349069664u,1349670752u,1350353824u,1351119744u,1351808864u,1352604576u,1353241408u,1354005344u,1354669952u,1355433888u,1356113760u,1356800896u,
1357578112u,1358244736u,1358976896u,1359661952u,1360446400u,1361081248u,1361901440u,1362547552u,1363180416u,1364007872u,1364631456u,1365395392u,1366075264u,1366893472u,1367539616u,1368337312u,
1368938400u,1369754528u,1370407904u,1371042752u,1371853696u,1372518304u,1373272992u,1373969376u,1374724032u,1375356896u,1376167840u,1376854976u,1377501120u,1378298816u,1378899904u,1379716032u,
1380352000u,1381135328u,1381815200u,1382579136u,1383266240u,1383913472u,1384685536u,1385342912u,1386161120u,1386807264u,1387604960u,1388206048u,1389036544u,1389677536u,1390313504u,1391144960u,
1391776704u,1392540640u,1393227744u,1394006048u,1394629632u,1395435488u,1396105216u,1396751360u,1397549056u,1398215680u,1398998048u,1399621632u,1400406080u,1401106464u,1401869280u,1402550272u,
1403171840u,1403967552u,1404591136u,1405379584u,1406066720u,1406843936u,1407510560u,1408308256u,1408959552u,1409583136u,1410367584u,1411026944u,1411856416u,1412509760u,1413264416u,1413929056u,
1414683712u,1415341088u,1416028224u,1416805440u,1417472064u,1418269760u,1418921056u,1419675712u,1420329088u,1421119520u,1421817920u,1422471264u,1423225920u,1423890560u,1424645216u,1425302592u,
1426120800u,1426766944u,1427564640u,1428231264u,1428882560u,1429637216u,1430290592u,1431081024u,1431779424u,1432563840u,1433187424u,1433983136u,1434608736u,1435295872u,1436073088u,1436739712u,
1437537408u,1438145664u,1438975136u,1439598720u,1440387168u,1441085568u,1441693888u,1442525344u,1443148928u,1443944640u,1444570240u,1445388448u,1446034592u,1446832288u,1447498912u,1448107168u,
1448936640u,1449562240u,1450346656u,1451047072u,1451786464u,1452486848u,1453241504u,1453906144u,1454531744u,1455349952u,1455996096u,1456793792u,1457460416u,1458199744u,1458898144u,1459654816u,
1460353216u,1461094592u,1461747968u,1462448352u,1463203008u,1463837888u,1464656096u,1465302240u,1466099936u,1466766560u,1467537664u,1468161248u,1468859648u,1469616320u,1470314720u,1471056096u,
1471709472u,1472540928u,1473144032u,1473950944u,1474617600u,1475263744u,1476061440u,1476728064u,1477499168u,1478122752u,1478922464u,1479607584u,1480407296u,1481017600u,1481670976u,1482502432u,
1483105536u,1483923744u,1484569888u,1485367584u,1486034208u,1486796032u,1487460672u,1488084256u,1488883968u,1489582368u,1490355520u,1490979104u,1491763552u,1492463936u,1493198112u,1493885248u,
1494531392u,1495329088u,1495995712u,1496757536u,1497422176u,1498176832u,1498845472u,1499674944u,1500317024u,1500940608u,1501725056u,1502373184u,1503211872u,1503846752u,1504623968u,1505290592u,
1506088288u,1506719040u,1507383680u,1508138336u,1508806976u,1509636448u,1510278528u,1511033184u,1511686560u,1512465760u,1513152896u,1513799040u,1514596736u,1515263360u,1516056960u,1516680544u,
1517476256u,1518099840u,1518899552u,1519597952u,1520240032u,1520994688u,1521648064u,1522427264u,1523114400u,1523891616u,1524558240u,1525355936u,1526018464u,1526642048u,1527437760u,1528074624u,
1528904096u,1529503136u,1530332608u,1530956192u,1531733408u,1532396000u,1533075904u,1533853120u,1534519744u,1535317440u,1535979968u,1536734624u,1537399264u,1538167200u,1538865600u,1539464640u,
1540294112u,1540917696u,1541694912u,1542382048u,1543159264u,1543825888u,1544623584u,1545241088u,1545941472u,1546696128u,1547360768u,1548128704u,1548827104u,1549557216u,1550255616u,1551001056u,
1551688192u,1552465408u,1553132032u,1553798656u,1554549216u,1555202592u,1556034048u,1556657632u,1557434848u,1558133248u,1558895136u,1559518720u,1560217120u,1560962560u,1561649696u,1562426912u,
1563093536u,1563891232u,1564510720u,1565295168u,1565995552u,1566609920u,1567405568u,1568094752u,1568856640u,1569480224u,1570309696u,1570924064u,1571742272u,1572388416u,1573055040u,1573852736u,
1574472224u,1575256672u,1575957056u,1576702496u,1577400896u,1578153504u,1578818144u,1579441728u,1580230208u,1580917344u,1581694560u,1582361184u,1583158880u,1583810144u,1584564800u,1585218176u,
1585918560u,1586664000u,1587362400u,1588115008u,1588779648u,1589534304u,1590191712u,1591009920u,1591656064u,1592322688u,1593120384u,1593771648u,1594526304u,1595179680u,1595970144u,1596666496u,
1597454976u,1598076512u,1598741152u,1599495808u,1600153216u,1600971424u,1601617568u,1602415264u,1603081888u,1603864224u,1604487808u,1605141184u,1605931648u,1606630048u,1607414432u,1608038016u,
1608833728u,1609457312u,1610245792u,1610932928u,1611579072u,1612376768u,1613043392u,1613825728u,1614449312u,1615233760u,1615893152u,1616722624u,1617375936u,1617999520u,1618795232u,1619420864u,
1620205248u,1620894432u,1621671648u,1622338272u,1623135968u,1623787232u,1624410816u,1625199296u,1625850624u,1626684128u,1627337440u,1628092096u,1628756736u,1629513440u,1630200576u,1630846720u,
1631644416u,1632311040u,1633050336u,1633748736u,1634503392u,1635160800u,1635990272u,1636598560u,1637298944u,1638053600u,1638718240u,1639474944u,1640162080u,1640939296u,1641605920u,1642403616u,
1643011840u,1643841312u,1644466944u,1645165344u,1645906688u,1646560064u,1647391520u,1648015104u,1648781088u,1649466176u,1650254656u,1650900800u,1651567424u,1652365120u,1652973344u,1653802816u,
1654428448u,1655257920u,1655868192u,1656652640u,1657353024u,1657976608u,1658742592u,1659429728u,1660206944u,1660873568u,1661671264u,1662311264u,1663065920u,1663764320u,1664389952u,1665219424u,
1665829696u,1666614144u,1667314528u,1668048736u,1668735872u,1669513088u,1670179712u,1670846336u,1671608128u,1672272768u,1673027424u,1673696096u,1674525568u,1675167616u,1675922272u,1676575648u,
1677276032u,1678010240u,1678697376u,1679474592u,1680141216u,1680938912u,1681569632u,1682365344u,1682988928u,1683657600u,1684487072u,1685129120u,1685883776u,1686537152u,1687368608u,1687971744u,
1688789952u,1689436096u,1690102720u,1690900416u,1691531136u,1692326848u,1692950432u,1693750176u,1694448576u,1695221696u,1695845280u,1696498656u,1697277888u,1697965024u,1698742240u,1699408864u,
1700206560u,1700869056u,1701623712u,1702288352u,1702911936u,1703711680u,1704410080u,1705183200u,1705806784u,1706584032u,1707291616u,1708046272u,1708692448u,1709334496u,1710132192u,1710798816u,
1711636480u,1712422913u,1713209346u,1713995779u,1714782212u,1715568645u,1716355078u,1717141511u,1717927944u,1718714377u,1719500810u,1720287243u,1721073676u,1721860109u,1722646542u,1723432975u,
1724219408u,1725005841u,1725792274u,1726578707u,1727365140u,1728151573u,1728938006u,1729724439u,1730510872u,1731297305u,1732083738u,1732870171u,1733656604u,1734443037u,1735229470u,1736015903u,
1736802336u,1737588769u,1738375202u,1739292707u,1740079140u,1740865573u,1741652006u,1742438439u,1743224872u,1744011305u,1744797738u,1745584171u,1746370604u,1747157037u,1747943470u,1748729903u,
1749516336u,1750302769u,1751089202u,1751875635u,1752662068u,1753448501u,1754234934u,1755021367u,1755807800u,1756594233u,1757380666u,1758167099u,1758953532u,1759739965u,1760526398u,1761312831u,
1762100224u,1762886657u,1763673090u,1764459523u,1765245956u,1766032389u,1766818822u,1767605255u,1768391688u,1769178121u,1769964554u,1770750987u,1771537420u,1772323853u,1773110286u,1773896719u,
1774683152u,1775469585u,1776256018u,1777042451u,1777828884u,1778746389u,1779532822u,1780319255u,1781105688u,1781892121u,1782678554u,1783464987u,1784251420u,1785037853u,1785824286u,1786610719u,
1787397152u,1788183585u,1788970018u,1789756451u,1790542884u,1791329317u,1792115750u,1792902183u,1793688616u,1794475049u,1795261482u,1796047915u,1796834348u,1797620781u,1798407214u,1799193647u,
1799980080u,1800766513u,1801552946u,1802339379u,1803125812u,1803912245u,1804698678u,1805485111u,1806271544u,1807057977u,1807844410u,1808630843u,1809417276u,1810203709u,1810990142u,1811776575u,
1812563968u,1813350401u,1814136834u,1814923267u,1815709700u,1816496133u,1817282566u,1818200071u,1818986504u,1819772937u,1820559370u,1821345803u,1822132236u,1822918669u,1823705102u,1824491535u,
1825277968u,1826064401u,1826850834u,1827637267u,1828423700u,1829210133u,1829996566u,1830782999u,1831569432u,1832355865u,1833142298u,1833928731u,1834715164u,1835501597u,1836288030u,1837074463u,
1837860896u,1838647329u,1839433762u,1840220195u,1841006628u,1841793061u,1842579494u,1843365927u,1844152360u,1844938793u,1845725226u,1846511659u,1847298092u,1848084525u,1848870958u,1849657391u,
1850443824u,1851230257u,1852016690u,1852803123u,1853589556u,1854375989u,1855162422u,1855948855u,1856735288u,1857652793u,1858439226u,1859225659u,1860012092u,1860798525u,1861584958u,1862371391u,
1863158784u,1863945217u,1864731650u,1865518083u,1866304516u,1867090949u,1867877382u,1868663815u,1869450248u,1870236681u,1871023114u,1871809547u,1872595980u,1873382413u,1874168846u,1874955279u,
1875741712u,1876528145u,1877314578u,1878101011u,1878887444u,1879673877u,1880460310u,1881246743u,1882033176u,1882819609u,1883606042u,1884392475u,1885178908u,1885965341u,1886751774u,1887538207u,
1888324640u,1889111073u,1889897506u,1890683939u,1891470372u,1892256805u,1893043238u,1893829671u,1894616104u,1895402537u,1896188970u,1897106475u,1897892908u,1898679341u,1899465774u,1900252207u,
1901038640u,1901825073u,1902611506u,1903397939u,1904184372u,1904970805u,1905757238u,1906543671u,1907330104u,1908116537u,1908902970u,1909689403u,1910475836u,1911262269u,1912048702u,1912835135u,
1913622528u,1914408961u,1915195394u,1915981827u,1916768260u,1917554693u,1918341126u,1919127559u,1919913992u,1920700425u,1921486858u,1922273291u,1923059724u,1923846157u,1924632590u,1925419023u,
1926205456u,1926991889u,1927778322u,1928564755u,1929351188u,1930137621u,1930924054u,1931710487u,1932496920u,1933283353u,1934069786u,1934856219u,1935642652u,1936560157u,1937346590u,1938133023u,
1938919456u,1939705889u,1940492322u,1941278755u,1942065188u,1942851621u,1943638054u,1944424487u,1945210920u,1945997353u,1946783786u,1947570219u,1948356652u,1949143085u,1949929518u,1950715951u,
1951502384u,1952288817u,1953075250u,1953861683u,1954648116u,1955434549u,1956220982u,1957007415u,1957793848u,1958580281u,1959366714u,1960153147u,1960939580u,1961726013u,1962512446u,1963298879u,
1964086272u,1964872705u,1965659138u,1966445571u,1967232004u,1968018437u,1968804870u,1969591303u,1970377736u,1971164169u,1971950602u,1972737035u,1973523468u,1974309901u,1975096334u,1976013839u,
1976800272u,1977586705u,1978373138u,1979159571u,1979946004u,1980732437u,1981518870u,1982305303u,1983091736u,1983878169u,1984664602u,1985451035u,1986237468u,1987023901u,1987810334u,1988596767u,
1989383200u,1990169633u,1990956066u,1991742499u,1992528932u,1993315365u,1994101798u,1994888231u,1995674664u,1996461097u,1997247530u,1998033963u,1998820396u,1999606829u,2000393262u,2001179695u,
2001966128u,2002752561u,2003538994u,2004325427u,2005111860u,2005898293u,2006684726u,2007471159u,2008257592u,2009044025u,2009830458u,2010616891u,2011403324u,2012189757u,2012976190u,2013762623u,
2014550016u,2015467521u,2016253954u,2017040387u,2017826820u,2018613253u,2019399686u,2020186119u,2020972552u,2021758985u,2022545418u,2023331851u,2024118284u,2024904717u,2025691150u,2026477583u,
2027264016u,2028050449u,2028836882u,2029623315u,2030409748u,2031196181u,2031982614u,2032769047u,2033555480u,2034341913u,2035128346u,2035914779u,2036701212u,2037487645u,2038274078u,2039060511u,
2039846944u,2040633377u,2041419810u,2042206243u,2042992676u,2043779109u,2044565542u,2045351975u,2046138408u,2046924841u,2047711274u,2048497707u,2049284140u,2050070573u,2050857006u,2051643439u,
2052429872u,2053216305u,2054002738u,2054920243u,2055706676u,2056493109u,2057279542u,2058065975u,2058852408u,2059638841u,2060425274u,2061211707u,2061998140u,2062784573u,2063571006u,2064357439u,
2065144832u,2065931265u,2066717698u,2067504131u,2068290564u,2069076997u,2069863430u,2070649863u,2071436296u,2072222729u,2073009162u,2073795595u,2074582028u,2075368461u,2076154894u,2076941327u,
2077727760u,2078514193u,2079300626u,2080087059u,2080873492u,2081659925u,2082446358u,2083232791u,2084019224u,2084805657u,2085592090u,2086378523u,2087164956u,2087951389u,2088737822u,2089524255u,
2090310688u,2091097121u,2091883554u,2092669987u,2093456420u,2094373925u,2095160358u,2095946791u,2096733224u,2097519657u,2098306090u,2099092523u,2099878956u,2100665389u,2101451822u,2102238255u,
2103024688u,2103811121u,2104597554u,2105383987u,2106170420u,2106956853u,2107743286u,2108529719u,2109316152u,2110102585u,2110889018u,2111675451u,2112461884u,2113248317u,2114034750u,2114821183u,
2115608576u,2116395009u,2117181442u,2117967875u,2118754308u,2119540741u,2120327174u,2121113607u,2121900040u,2122686473u,2123472906u,2124259339u,2125045772u,2125832205u,2126618638u,2127405071u,
2128191504u,2128977937u,2129764370u,2130550803u,2131337236u,2132123669u,2132910102u,2133827607u,2134614040u,2135400473u,2136186906u,2136973339u,2137759772u,2138546205u,2139332638u,2140119071u,
2140905504u,2141691937u,2142478370u,2143264803u,2144051236u,2144837669u,2145624102u,2146410535u,2147196968u,2147983401u,2148769834u,2149556267u,2150342700u,2151129133u,2151915566u,2152701999u,
2153488432u,2154274865u,2155061298u,2155847731u,2156634164u,2157420597u,2158207030u,2158993463u,2159779896u,2160566329u,2161352762u,2162139195u,2162925628u,2163712061u,2164498494u,2165284927u,
2166072320u,2166858753u,2167645186u,2168431619u,2169218052u,2170004485u,2170790918u,2171577351u,2172363784u,2173281289u,2174067722u,2174854155u,2175640588u,2176427021u,2177213454u,2177999887u,
2178786320u,2179572753u,2180359186u,2181145619u,2181932052u,2182718485u,2183504918u,2184291351u,2185077784u,2185864217u,2186650650u,2187437083u,2188223516u,2189009949u,2189796382u,2190582815u,
2191369248u,2192155681u,2192942114u,2193728547u,2194514980u,2195301413u,2196087846u,2196874279u,2197660712u,2198447145u,2199233578u,2200020011u,2200806444u,2201592877u,2202379310u,2203165743u,
2203952176u,2204738609u,2205525042u,2206311475u,2207097908u,2207884341u,2208670774u,2209457207u,2210243640u,2211030073u,2211816506u,2212734011u,2213520444u,2214306877u,2215093310u,2215879743u,
2216667136u,2217453569u,2218240002u,2219026435u,2219812868u,2220599301u,2221385734u,2222172167u,2222958600u,2223745033u,2224531466u,2225317899u,2226104332u,2226890765u,2227677198u,2228463631u,
2229250064u,2230036497u,2230822930u,2231609363u,2232395796u,2233182229u,2233968662u,2234755095u,2235541528u,2236327961u,2237114394u,2237900827u,2238687260u,2239473693u,2240260126u,2241046559u,
2241832992u,2242619425u,2243405858u,2244192291u,2244978724u,2245765157u,2246551590u,2247338023u,2248124456u,2248910889u,2249697322u,2250483755u,2251270188u,2252187693u,2252974126u,2253760559u,
2254546992u,2255333425u,2256119858u,2256906291u,2257692724u,2258479157u,2259265590u,2260052023u,2260838456u,2261624889u,2262411322u,2263197755u,2263984188u,2264770621u,2265557054u,2266343487u,
2267130880u,2267917313u,2268703746u,2269490179u,2270276612u,2271063045u,2271849478u,2272635911u,2273422344u,2274208777u,2274995210u,2275781643u,2276568076u,2277354509u,2278140942u,2278927375u,
2279713808u,2280500241u,2281286674u,2282073107u,2282859540u,2283645973u,2284432406u,2285218839u,2286005272u,2286791705u,2287578138u,2288364571u,2289151004u,2289937437u,2290723870u,2291641375u,
2292427808u,2293214241u,2294000674u,2294787107u,2295573540u,2296359973u,2297146406u,2297932839u,2298719272u,2299505705u,2300292138u,2301078571u,2301865004u,2302651437u,2303437870u,2304224303u,
2305010736u,2305797169u,2306583602u,2307370035u,2308156468u,2308942901u,2309729334u,2310515767u,2311302200u,2312088633u,2312875066u,2313661499u,2314447932u,2315234365u,2316020798u,2316807231u,
2317594624u,2318381057u,2319167490u,2319953923u,2320740356u,2321526789u,2322313222u,2323099655u,2323886088u,2324672521u,2325458954u,2326245387u,2327031820u,2327818253u,2328604686u,2329391119u,
2330177552u,2331095057u,2331881490u,2332667923u,2333454356u,2334240789u,2335027222u,2335813655u,2336600088u,2337386521u,2338172954u,2338959387u,2339745820u,2340532253u,2341318686u,2342105119u,
2342891552u,2343677985u,2344464418u,2345250851u,2346037284u,2346823717u,2347610150u,2348396583u,2349183016u,2349969449u,2350755882u,2351542315u,2352328748u,2353115181u,2353901614u,2354688047u,
2355474480u,2356260913u,2357047346u,2357833779u,2358620212u,2359406645u,2360193078u,2360979511u,2361765944u,2362552377u,2363338810u,2364125243u,2364911676u,2365698109u,2366484542u,2367270975u,
2368058368u,2368844801u,2369631234u,2370548739u,2371335172u,2372121605u,2372908038u,2373694471u,2374480904u,2375267337u,2376053770u,2376840203u,2377626636u,2378413069u,2379199502u,2379985935u,
2380772368u,2381558801u,2382345234u,2383131667u,2383918100u,2384704533u,2385490966u,2386277399u,2387063832u,2387850265u,2388636698u,2389423131u,2390209564u,2390995997u,2391782430u,2392568863u,
2393355296u,2394141729u,2394928162u,2395714595u,2396501028u,2397287461u,2398073894u,2398860327u,2399646760u,2400433193u,2401219626u,2402006059u,2402792492u,2403578925u,2404365358u,2405151791u,
2405938224u,2406724657u,2407511090u,2408297523u,2409083956u,2410001461u,2410787894u,2411574327u,2412360760u,2413147193u,2413933626u,2414720059u,2415506492u,2416292925u,2417079358u,2417865791u,
2418653184u,2419439617u,2420226050u,2421012483u,2421798916u,2422585349u,2423371782u,2424158215u,2424944648u,2425731081u,2426517514u,2427303947u,2428090380u,2428876813u,2429663246u,2430449679u,
2431236112u,2432022545u,2432808978u,2433595411u,2434381844u,2435168277u,2435954710u,2436741143u,2437527576u,2438314009u,2439100442u,2439886875u,2440673308u,2441459741u,2442246174u,2443032607u,
2443819040u,2444605473u,2445391906u,2446178339u,2446964772u,2447751205u,2448537638u,2449455143u,2450241576u,2451028009u,2451814442u,2452600875u,2453387308u,2454173741u,2454960174u,2455746607u,
2456533040u,2457319473u,2458105906u,2458892339u,2459678772u,2460465205u,2461251638u,2462038071u,2462824504u,2463610937u,2464397370u,2465183803u,2465970236u,2466756669u,2467543102u,2468329535u,
2469116928u,2469903361u,2470689794u,2471476227u,2472262660u,2473049093u,2473835526u,2474621959u,2475408392u,2476194825u,2476981258u,2477767691u,2478554124u,2479340557u,2480126990u,2480913423u,
2481699856u,2482486289u,2483272722u,2484059155u,2484845588u,2485632021u,2486418454u,2487204887u,2487991320u,2488908825u,2489695258u,2490481691u,2491268124u,2492054557u,2492840990u,2493627423u,
2494413856u,2495200289u,2495986722u,2496773155u,2497559588u,2498346021u,2499132454u,2499918887u,2500705320u,2501491753u,2502278186u,2503064619u,2503851052u,2504637485u,2505423918u,2506210351u,
2506996784u,2507783217u,2508569650u,2509356083u,2510142516u,2510928949u,2511715382u,2512501815u,2513288248u,2514074681u,2514861114u,2515647547u,2516433980u,2517220413u,2518006846u,2518793279u,
4096u,4112u,4097u,4113u,16384u,38912u,50176u,61440u,5120u,5136u,5121u,5137u,6144u,6160u,6145u,6161u,
16416u,38944u,50208u,61472u,7168u,7184u,7169u,7185u,72704u,786432u,1703952u,2621441u,3538961u,4474880u,5390400u,6330432u,
7259200u,8187968u,9046016u,9963536u,10881025u,11667473u,12612608u,13501440u,14418960u,15336449u,16253969u,17243168u,18091008u,19008528u,19926017u,20843537u,
21776416u,22552576u,23470096u,24387585u,25305105u,26230880u,27170912u,28099680u,29028448u,29914144u,30811136u,31728656u,32646145u,33563665u,34413632u,35258496u,
36176016u,37093505u,38011025u,38944896u,39856128u,40773648u,41691137u,42608657u,43534400u,44472448u,45362176u,46148624u,47066113u,47983633u,48906400u,49809536u,
50727056u,51644545u,52562065u,53481600u,54399120u,55316609u,56234129u,57066624u,57995392u,58881088u,59769984u,60687504u,61604993u,62522513u,63477920u,64361600u,
65279120u,66196609u,67114129u,68042944u,68885600u,69733504u,70651024u,71568513u,72486033u,73406592u,74324112u,75241601u,76159121u,77088864u,78026944u,78921952u,
79742112u,80670880u,81534080u,82451600u,83369089u,84286609u,85226592u,86153440u,87032064u,87949584u,88867073u,89784593u,90718464u,91430016u,92062912u,92817536u,
93493504u,94257440u,94898432u,95684880u,96471297u,97388817u,98234560u,98878688u,99644544u,100271360u,101188880u,101975297u,102761745u,103586080u,104350016u,105050336u,
105847968u,106433792u,107220240u,108006657u,108924177u,109757696u,110403584u,111188288u,111823008u,112599168u,113385616u,114172033u,114958481u,115884384u,116584704u,117359904u,
117994656u,118629504u,119547024u,120333441u,121119889u,121936224u,122683776u,123470224u,124256641u,125174161u,125976960u,126677280u,127474880u,128107840u,128854144u,129640592u,
130427009u,131213457u,132161568u,132815232u,133448096u,134236352u,134884480u,135802000u,136588417u,137374865u,138152320u,139069840u,139856257u,140642705u,141488448u,142263648u,
142872832u,143790352u,144576769u,145363217u,146146688u,147064208u,147850625u,148637073u,149451968u,150247840u,150880704u,151712096u,152346688u,152966400u,153883920u,154670337u,
155456785u,156372352u,157158800u,157945217u,158731665u,159696256u,160373984u,160995776u,161759712u,162404608u,163322128u,164108545u,164894993u,165693664u,166523264u,167167392u,
167910656u,168697104u,169483521u,170269969u,171209952u,171874784u,172491264u,173408784u,174195201u,174981649u,175915520u,176595040u,177271200u,178046400u,178724096u,179476992u,
180109856u,180750848u,181668368u,182454785u,183241233u,184086976u,184830208u,185517536u,186254848u,187041296u,187827713u,188745233u,189571200u,190224928u,190988864u,191689184u,
192286208u,193203728u,193990145u,194776593u,195610112u,196385344u,197018208u,197849600u,198493728u,199270560u,199924320u,200540800u,201458320u,202244737u,203031185u,203833984u,
204665376u,205309504u,206084736u,206717600u,207358592u,208276112u,209062529u,209848977u,210825792u,211460288u,212125280u,212933920u,213517952u,214304400u,215221889u,216008337u,
216832672u,217596608u,218264832u,218952288u,219680384u,220466832u,221253249u,222170769u,223004288u,223627552u,224434880u,225067744u,225878240u,226554496u,227198624u,228007232u,
228629216u,229405984u,230032128u,230818576u,231736065u,232522513u,233325312u,234156704u,234800832u,235424064u,236232960u,236886784u,237650720u,238291712u,239078160u,239864577u,
240782097u,241627840u,242294112u,243058400u,243726336u,244479296u,245106432u,245892880u,246810369u,247596817u,248421152u,249185088u,249885408u,250482432u,251399952u,252186369u,
252972817u,253806336u,254560608u,255238432u,256023360u,256713728u,257311584u,258153856u,258798336u,259586080u,260228896u,260994400u,261628160u,262414608u,263201025u,264118545u,
264916992u,265591648u,266339200u,267125648u,267912065u,268829585u,269632384u,270289280u,271119136u,271721728u,272508176u,273425665u,274212113u,275020800u,275826496u,276472128u,
277160992u,277912448u,278601120u,279324928u,280111376u,280897793u,281684241u,282608544u,283249536u,284035984u,284953473u,285739921u,286586944u,287372096u,287995264u,288671584u,
289427488u,290073856u,290991376u,291777793u,292564241u,293341056u,294258576u,295044993u,295831441u,296655776u,297421216u,298075072u,298885472u,299531296u,300217184u,300945280u,
301731728u,302518145u,303435665u,304269184u,304887808u,305805328u,306591745u,307378193u,308225088u,309009856u,309631936u,310395872u,311097440u,311719328u,312537984u,313146368u,
314063888u,314850305u,315636753u,316458912u,317214784u,317861888u,318648336u,319565825u,320352273u,321139712u,322057232u,322843649u,323630097u,324422080u,325229536u,325886336u,
326663232u,327349152u,327993280u,328803424u,329457120u,330174848u,330961296u,331747713u,332534161u,333501440u,334132672u,334821504u,335606720u,336220256u,337037280u,337646976u,
338433424u,339350913u,340137361u,340937184u,341746080u,342391904u,343021952u,343939472u,344725889u,345512337u,346485728u,347151872u,347808896u,348585984u,349230112u,349992416u,
350681248u,351280512u,352198032u,352984449u,353770897u,354570368u,355379200u,356000128u,356917648u,357704065u,358490513u,359287296u,360096192u,360742016u,361506176u,362292624u,
363079041u,363865489u,364846624u,365503648u,366149664u,366900736u,367589568u,368322944u,369109392u,369895809u,370682257u,371609760u,372287520u,373049888u,373727712u,374353280u,
375270800u,376057217u,376843665u,377650336u,378478144u,379135168u,379876896u,380565728u,381178048u,381986880u,382618176u,383427072u,384072896u,384900704u,385557728u,386168384u,
386936960u,387723408u,388509825u,389427345u,390265088u,390911040u,391647744u,392434192u,393220609u,394007057u,394940640u,395618400u,396403232u,397036128u,397685888u,398603408u,
399389825u,400176273u,400963712u,401881232u,402667649u,403454097u,404372608u,405159056u,405945473u,406731921u,407639552u,408426000u,409212417u,409998865u,410942688u,411639424u,
412382816u,413082880u,413669888u,414587408u,415373825u,416160273u,417003808u,417792064u,418436192u,419211392u,419865152u,420499712u,421284480u,421928448u,422845968u,423632385u,
424418833u,425206272u,426123792u,426910209u,427696657u,428506368u,429334176u,429946496u,430777632u,431367680u,432154128u,433071617u,433858065u,434698560u,435486816u,436130944u,
436775072u,437559904u,438192800u,438980896u,439626240u,440412688u,441330177u,442116625u,442924320u,443621056u,444364448u,445001216u,445918736u,446705153u,447491601u,448341312u,
449118336u,449771872u,450548896u,451191424u,451848384u,452610752u,453267776u,454106848u,454720832u,455374528u,456143104u,456929552u,457715969u,458633489u,459482464u,460128416u,
460903616u,461546144u,462334176u,463010176u,463620832u,464408928u,465057024u,465843472u,466760961u,467547409u,468334848u,469252368u,470038785u,470825233u,471612672u,472530192u,
473316609u,474103057u,474878592u,475796112u,476582529u,477368977u,478355200u,478969184u,479622880u,480454016u,481040000u,481826448u,482743937u,483530385u,484376768u,485139136u,
485807328u,486451456u,487258528u,487869184u,488657280u,489298560u,490085008u,491002497u,491788945u,492642080u,493385472u,494042496u,494673536u,495591056u,496377473u,497163921u,
497951360u,498868880u,499655297u,500441745u,501425568u,502071520u,502702816u,503502080u,504146208u,504908576u,505608640u,506209920u,507127440u,507913857u,508700305u,509497760u,
510336832u,510949152u,511715968u,512502416u,513288833u,514075281u,515014048u,515712448u,516489472u,517120768u,517788960u,518564160u,519188096u,519974544u,520892033u,521678481u,
522472256u,523303392u,523915712u,524754784u,525367104u,526024128u,526791040u,527577488u,528363905u,529281425u,530096928u,530776352u,531572192u,532206912u,532851040u,533613408u,
534313472u,535049600u,535836048u,536622465u,537408913u,538333664u,538982784u,539769232u,540686721u,541473169u,542260608u,543047056u,543964545u,544750993u,545595264u,546338656u,
546995680u,547754752u,548541200u,549327617u,550114065u,551068480u,551747904u,552392032u,553167232u,553854464u,554584960u,555285024u,555882240u,556799760u,557586177u,558372625u,
559305216u,560013216u,560625536u,561388288u,562174736u,562961153u,563878673u,564690432u,565355360u,566165856u,566809984u,567454112u,568272416u,568871840u,569646848u,570433296u,
571219713u,572137233u,572924672u,573711120u,574497537u,575415057u,576256576u,576868896u,577707968u,578320288u,578955008u,579872528u,580658945u,581445393u,582385184u,583050112u,
583729536u,584504736u,585148864u,585902848u,586689296u,587475713u,588262161u,589188032u,589899328u,590674528u,591277824u,592064272u,592850689u,593768209u,594563648u,595271648u,
596015040u,596652544u,597438992u,598356481u,599142929u,599958432u,600735296u,601424288u,602199488u,602843616u,603474912u,604317280u,604911104u,605697552u,606615041u,607401489u,
608238208u,608975360u,609761808u,610548225u,611334673u,612253184u,613039632u,613826049u,614743569u,615535200u,616200160u,617037824u,617653184u,618332608u,619085408u,619763168u,
620538368u,621225600u,621810560u,622728080u,623514497u,624300945u,625277600u,625889920u,626606112u,627316608u,628103056u,628889473u,629806993u,630629344u,631308768u,632083968u,
632728096u,633372288u,634133376u,634919824u,635706241u,636623761u,637478560u,638122688u,638852992u,639639440u,640425857u,641343377u,642203712u,642798240u,643572608u,644359056u,
645145473u,645931921u,646906368u,647550496u,648325696u,648969888u,649602944u,650520464u,651306881u,652093329u,652945088u,653667200u,654453648u,655240065u,656157585u,656997088u,
657670240u,658395840u,659042176u,659828624u,660746113u,661532561u,662319744u,663237264u,664023681u,664810129u,665649696u,666424896u,667069024u,667844288u,668542688u,669136512u,
670054032u,670840449u,671626897u,672414336u,673331856u,674118273u,674904721u,675823232u,676609680u,677396097u,678182545u,679148288u,679821440u,680415968u,681247296u,681891424u,
682666624u,683310816u,684009216u,684784416u,685457568u,686183168u,686839808u,687669856u,688313984u,688958112u,689733376u,690390016u,691218208u,691880128u,692517696u,693284864u,
693916448u,694747776u,695359520u,696047264u,696822464u,697490432u,698253088u,698909728u,699606848u,700385280u,701055200u,701823840u,702459936u,703091520u,703935488u,704525056u,
705442576u,706228993u,707015441u,707855008u,708630208u,709274336u,710039616u,710728736u,711360320u,712138752u,712783616u,713570064u,714487553u,715274001u,716061440u,716978960u,
717765377u,718551825u,719339264u,720256784u,721043201u,721829649u,722633792u,723461984u,724109344u,724910336u,725547904u,726184000u,726946656u,727646912u,728422112u,729101344u,
729852672u,730486880u,731176000u,731938656u,732586016u,733423488u,734037088u,734740768u,735512640u,736164768u,736953056u,737563520u,738242656u,739038976u,739683104u,740504640u,
741103744u,741769088u,742579296u,743202880u,744040352u,744702272u,745309312u,746129504u,746781632u,747569920u,748214048u,748858176u,749622176u,750301312u,751121504u,751720608u,
752517024u,753196160u,753819744u,754663776u,755312576u,756057248u,756746368u,757398496u,758186784u,758788992u,759575440u,760492929u,761279377u,762107712u,762882912u,763515840u,
764194976u,765015168u,765605760u,766523280u,767309697u,768096145u,768901056u,769677504u,770325376u,771111824u,772029313u,772815761u,773603200u,774520720u,775307137u,776093585u,
776920224u,777732480u,778330240u,779167712u,779781312u,780469056u,781244256u,781912224u,782674816u,783307744u,783986880u,784807072u,785416160u,786192608u,786907552u,787537088u,
788291744u,788988768u,789763968u,790398176u,791063456u,791873728u,792506592u,793326784u,793985472u,794712320u,795401440u,796025024u,796853120u,797497248u,798272448u,798917888u,
799607008u,800370944u,801060064u,801849824u,802445600u,803134720u,803931040u,804575168u,805331168u,806005728u,806664192u,807437600u,808126720u,808903680u,809546016u,810238464u,
811021568u,811620672u,812450752u,813094880u,813751584u,814551040u,815183904u,815948032u,816612672u,817432864u,818101248u,818734112u,819544608u,820162880u,820983072u,821625824u,
822237536u,823027712u,823726112u,824490048u,825154880u,825778464u,826574176u,827233280u,828062752u,828705088u,829350976u,830161472u,830779744u,831599936u,832202784u,832901184u,
833665120u,834296192u,835116384u,835739968u,836408352u,837191040u,837893184u,838657120u,839336544u,840108384u,840741248u,841377856u,842207328u,842872160u,843626624u,844257696u,
844946816u,845701472u,846369856u,847199328u,847832192u,848463264u,849298048u,849938816u,850684000u,851358112u,852037760u,852801696u,853489024u,854219200u,854908320u,855544928u,
856374400u,857007264u,857760128u,858473120u,859080128u,859900320u,860514432u,861343904u,861976768u,862761408u,863450528u,864049632u,864851072u,865549472u,866311616u,866968768u,
867648192u,868376992u,869041632u,869861824u,870475936u,871174336u,871938272u,872591840u,873412032u,874026144u,874724544u,875452928u,876143840u,876954336u,877583840u,878207424u,
879003136u,879651008u,880480480u,881134048u,881768704u,882553344u,883242464u,883987648u,884686048u,885449984u,886129408u,886725152u,887545344u,888168928u,888957152u,889655552u,
890275360u,891074848u,891750912u,892514848u,893162720u,893861120u,894625056u,895301120u,896090912u,896686656u,897375776u,898130432u,898787584u,899617056u,900249920u,900892224u,
901712416u,902337792u,903167264u,903800128u,904442432u,905265984u,905917984u,906648160u,907337280u,908093728u,908792128u,909424992u,910189088u,910853728u,911673920u,912299296u,
912997696u,913761632u,914441056u,915190368u,915879488u,916478592u,917268800u,917967200u,918740576u,919386496u,920019520u,920815232u,921504352u,922260800u,922959200u,923592064u,
924402560u,925020800u,925840992u,926443872u,927142272u,927881888u,928561568u,929357440u,929981024u,930776736u,931435872u,932134272u,932898208u,933563008u,934364064u,934982304u,
935671424u,936405376u,937103776u,937867712u,938498752u,939187872u,939942528u,940610944u,941440416u,942073280u,942704320u,943539136u,944179872u,944943808u,945580448u,946278848u,
947042784u,947730080u,948460256u,949149376u,949772960u,950572448u,951270848u,952034784u,952714208u,953452256u,954141376u,954755520u,955584992u,956217856u,957002464u,957691584u,
958290688u,959110880u,959747520u,960576992u,961209856u,961831616u,962646016u,963312640u,964110336u,964711424u,965379840u,966200032u,966814176u,967643648u,968276512u,968930048u,
969750240u,970349344u,971150816u,971849216u,972482080u,973266688u,973918240u,974715936u,975382560u,976114720u,976773856u,977438496u,978258688u,978872832u,979702304u,980335168u,
980988704u,981808896u,982423040u,983252512u,983885376u,984505152u,985325344u,985948928u,986744640u,987392544u,988090944u,988854880u,989530912u,990294848u,990983968u,991598112u,
992427584u,993060448u,993811296u,994500416u,995124000u,995912256u,996610656u,997374592u,998016864u,998837056u,999469920u,1000117824u,1000947296u,1001580160u,1002387264u,1002986368u,
1003675488u,1004430144u,1005087328u,1005916800u,1006549664u,1007191936u,1008012128u,1008637536u,1009467008u,1010099872u,1010742144u,1011562336u,1012161440u,1012981632u,1013607040u,1014305440u,
1015069376u,1015702368u,1016498080u,1017187200u,1017812608u,1018642080u,1019274944u,1020048288u,1020737408u,1021467584u,1022126752u,1022825152u,1023589088u,1024253856u,1025008512u,1025673152u,
1026332320u,1027148704u,1027817152u,1028581088u,1029223360u,1030043552u,1030646464u,1031344864u,1032108800u,1032739808u,1033560000u,1034183584u,1034979296u,1035638464u,1036467936u,1037100800u,
1037765568u,1038529504u,1039166176u,1040005056u,1040651008u,1041414944u,1042071552u,1042866144u,1043489728u,1044158176u,1044987648u,1045620512u,1046419456u,1047071712u,1047806976u,1048472320u,
1049301792u,1049934656u,1050621920u,1051377696u,1052012544u,1052808960u,1053507360u,1054237664u,1054926656u,1055725600u,1056349184u,1056992032u,1057821504u,1058454368u,1059210272u,1059899392u,
1060683840u,1061328672u,1062105120u,1062813504u,1063446368u,1064236032u,1064900672u,1065655328u,1066298176u,1067127648u,1067760512u,1068516416u,1069205536u,1069848384u,1070645344u,1071333216u,
1072097152u,1072721984u,1073542176u,1074206816u,1074948960u,1075616832u,1076302720u,1077066656u,1077691488u,1078511680u,1079154528u,1079984000u,1080616864u,1081393280u,1082028128u,1082717248u,
1083468672u,1084168320u,1084953504u,1085586368u,1086364768u,1086997632u,1087805312u,1088503712u,1089128544u,1089923008u,1090568352u,1091334272u,1092023392u,1092774816u,1093473216u,1094237152u,
1094916256u,1095670912u,1096325024u,1097023424u,1097745568u,1098442720u,1099221120u,1099874496u,1100640416u,1101294528u,1102124000u,1102771328u,1103437824u,1104222400u,1104845984u,1105631168u,
1106329568u,1107064832u,1107707072u,1108527264u,1109180640u,1109814240u,1110615040u,1111257280u,1112088608u,1112732832u,1113528544u,1114152128u,1114937312u,1115584544u,1116226784u,1117046976u,
1117700352u,1118479392u,1119166528u,1119943744u,1120610368u,1121342528u,1122005216u,1122694336u,1123490048u,1124113632u,1124890688u,1125532928u,1126353120u,1126999104u,1127817312u,1128463456u,
1129130080u,1129862240u,1130545440u,1131311360u,1132000480u,1132777536u,1133451552u,1134196832u,1134861568u,1135494432u,1136305248u,1136969984u,1137778816u,1138424960u,1139222656u,1139823744u,
1140638016u,1141272864u,1141961984u,1142739040u,1143371904u,1144199488u,1144823072u,1145611392u,1146242368u,1147084960u,1147731104u,1148397728u,1149129888u,1149815072u,1150599520u,1151234368u,
1152045184u,1152678048u,1153496352u,1154160992u,1154784576u,1155572896u,1156260032u,1157037248u,1157703872u,1158436032u,1159087456u,1159907648u,1160561024u,1161220256u,1161984192u,1162637664u,
1163457856u,1164122496u,1164877152u,1165534400u,1166352608u,1166998752u,1167665376u,1168397536u,1169048960u,1169869152u,1170526400u,1171308960u,1171945696u,1172730240u,1173419360u,1174084000u,
1174840544u,1175527680u,1176304896u,1176971520u,1177703680u,1178377600u,1179141536u,1179830656u,1180487904u,1181251840u,1181925824u,1182691744u,1183380864u,1184146688u,1184831936u,1185620256u,
1186266400u,1187064096u,1187665184u,1188339104u,1189103040u,1189794048u,1190557984u,1191233952u,1192018400u,1192653248u,1193473440u,1194108192u,1194795328u,1195572544u,1196239168u,1196971328u,
1197677024u,1198431680u,1199100192u,1199864128u,1200506336u,1201195456u,1201979904u,1202614752u,1203414336u,1204101472u,1204878688u,1205545312u,1206277472u,1206973888u,1207638528u,1208393184u,
1209061696u,1209825632u,1210467840u,1211288032u,1211941408u,1212720480u,1213407616u,1214053760u,1214851456u,1215452544u,1216246272u,1216935392u,1217731104u,1218354688u,1219154272u,1219787136u,
1220429344u,1221249536u,1221895552u,1222713760u,1223359904u,1224157600u,1224758688u,1225572928u,1226207776u,1226896896u,1227692608u,1228329344u,1229093280u,1229757984u,1230521920u,1231211040u,
1231988128u,1232675264u,1233321408u,1234119104u,1234720192u,1235534432u,1236169280u,1236989472u,1237635488u,1238399424u,1239095904u,1239719488u,1240483424u,1241163200u,1241981408u,1242627552u,
1243425248u,1244026336u,1244842560u,1245495936u,1246130784u,1246950976u,1247596992u,1248360928u,1249057408u,1249812064u,1250469344u,1251287552u,1251933696u,1252600320u,1253332480u,1253983872u,
1254804064u,1255457440u,1256223360u,1256903136u,1257667072u,1258354272u,1259018912u,1259773568u,1260430848u,1261249056u,1261895200u,1262692896u,1263293984u,1264076448u,1264765568u,1265418944u,
1266209280u,1266842144u,1267626656u,1268315776u,1269111488u,1269735072u,1270523424u,1271210560u,1271856704u,1272654400u,1273255488u,1274037952u,1274727072u,1275515424u,1276148288u,1276953312u,
1277588160u,1278408352u,1279072992u,1279698496u,1280516704u,1281162848u,1281960544u,1282561632u,1283366592u,1283999456u,1284819648u,1285476928u,1286109792u,1286914816u,1287549664u,1288369856u,
1289004640u,1289822848u,1290468992u,1291266688u,1291867776u,1292573440u,1293328096u,1293960960u,1294783072u,1295415936u,1296222944u,1296876320u,1297642240u,1298331360u,1298966144u,1299784352u,
1300430496u,1301228192u,1301829280u,1302666016u,1303289600u,1304089216u,1304722080u,1305364256u,1306184448u,1306837824u,1307603744u,1308272288u,1309090496u,1309736640u,1310534336u,1311135424u,
1311831808u,1312627520u,1313251104u,1314050720u,1314683584u,1315456832u,1316145952u,1316930400u,1317578432u,1318265568u,1319042784u,1319709408u,1320441568u,1321104192u,1321924384u,1322589024u,
1323343680u,1324012224u,1324645088u,1325418336u,1326107456u,1326884576u,1327571712u,1328348928u,1329015552u,1329747712u,1330430848u,1331065696u,1331885888u,1332550528u,1333318368u,1333951232u,
1334746976u,1335379840u,1336200032u,1336846080u,1337533216u,1338310432u,1338977056u,1339709216u,1340392352u,1341158272u,1341847392u,1342624512u,1343257376u,1343953824u,1344708480u,1345341344u,
1346152224u,1346839360u,1347616576u,1348283200u,1349015360u,1349700480u,1350353856u,1351119776u,1351799584u,1352595328u,1353218880u,1354046400u,1354669984u,1355458368u,1356145504u,1356791648u,
1357589344u,1358190432u,1358972864u,1359661984u,1360446432u,1361105728u,1361867712u,1362525024u,1363212192u,1364007904u,1364631488u,1365419872u,1366107008u,1366884224u,1367550848u,1368283008u,
1368934368u,1369754560u,1370411872u,1371045888u,1371831168u,1372484576u,1373304768u,1373952000u,1374726016u,1375379424u,1376199584u,1376845728u,1377512352u,1378244512u,1378944000u,1379716064u,
1380373376u,1381137312u,1381793824u,1382625280u,1383266272u,1383913504u,1384687520u,1385374656u,1386151872u,1386818496u,1387550656u,1388207104u,1389036576u,1389679520u,1390312384u,1391101952u,
1391755328u,1392586784u,1393210368u,1393993664u,1394680800u,1395458016u,1396124640u,1396725728u,1397545024u,1398168608u,1398985664u,1399653440u,1400404960u,1401063456u,1401847904u,1402548288u,
1403168736u,1403969536u,1404615680u,1405413376u,1406080000u,1406841888u,1407506528u,1408261184u,1408947168u,1409628160u,1410401376u,1411024960u,1411809408u,1412457472u,1413296224u,1413931040u,
1414708256u,1415374880u,1416041504u,1416803392u,1417468032u,1418222688u,1418891264u,1419720736u,1420362880u,1421117536u,1421763616u,1422426272u,1423237184u,1423883328u,1424681024u,1425347648u,
1426141312u,1426764896u,1427560608u,1428197408u,1428895808u,1429625984u,1430324384u,1431079040u,1431725120u,1432543328u,1433189472u,1433987168u,1434653792u,1435271360u,1436102816u,1436726400u,
1437503552u,1438201952u,1438963904u,1439587488u,1440416960u,1441031264u,1441718400u,1442495616u,1443162240u,1443959936u,1444579488u,1445363936u,1446064320u,1446818976u,1447465056u,1448163456u,
1448925408u,1449548992u,1450337408u,1451024544u,1451801760u,1452468384u,1453266080u,1453917408u,1454540992u,1455325440u,1456025824u,1456771200u,1457469600u,1458222272u,1458886912u,1459641568u,
1460298912u,1461117120u,1461763264u,1462429888u,1463227584u,1463878912u,1464633568u,1465286944u,1466077344u,1466775744u,1467560192u,1468183776u,1468848416u,1469603072u,1470260416u,1471078624u,
1471724768u,1472522464u,1473189088u,1473971488u,1474595072u,1475252416u,1476081888u,1476690240u,1477521696u,1478145280u,1478940992u,1479566560u,1480384768u,1481030912u,1481697536u,1482495232u,
1483103520u,1483932992u,1484556576u,1485344992u,1486043392u,1486782816u,1487483200u,1488106784u,1488872704u,1489559840u,1490337056u,1491003680u,1491801376u,1492441440u,1493196096u,1493894496u,
1494520064u,1495349536u,1495959872u,1496744320u,1497444704u,1498199360u,1498834208u,1499652416u,1500298560u,1500965184u,1501762880u,1502402944u,1503157600u,1503826208u,1504655680u,1505297792u,
1506052448u,1506705824u,1507406208u,1508140352u,1508827488u,1509604704u,1510271328u,1511069024u,1511699808u,1512495520u,1513119104u,1513787712u,1514617184u,1515259296u,1516013952u,1516667328u,
1517446496u,1518133632u,1518910848u,1519577472u,1520244096u,1521037728u,1521661312u,1522457024u,1523080608u,1523880288u,1524578688u,1525351872u,1525975456u,1526621568u,1527439776u,1528085920u,
1528883616u,1529550240u,1530298848u,1530999232u,1531753888u,1532418528u,1533055360u,1533884832u,1534483904u,1535313376u,1535936960u,1536714144u,1537401280u,1538178496u,1538845120u,1539511744u,
1540260352u,1540960736u,1541715392u,1542361504u,1543190976u,1543821824u,1544576480u,1545274880u,1545889216u,1546707424u,1547353568u,1548151264u,1548817888u,1549568480u,1550221856u,1551053312u,
1551676896u,1552454080u,1553152480u,1553783328u,1554537984u,1555195360u,1556013568u,1556659712u,1557457408u,1558124032u,1558906400u,1559529984u,1560183360u,1561014816u,1561629152u,1562458624u,
1563080192u,1563875904u,1564499488u,1565287936u,1565975072u,1566621216u,1567418912u,1568085536u,1568867904u,1569491488u,1570275936u,1570935296u,1571764768u,1572418112u,1573041696u,1573837408u,
1574460992u,1575249440u,1575936576u,1576713792u,1577380416u,1578178112u,1578829408u,1579452992u,1580241440u,1580939840u,1581679232u,1582379616u,1583134272u,1583798912u,1584555584u,1585242720u,
1585888864u,1586686560u,1587353184u,1588092512u,1588790912u,1589547584u,1590200928u,1591032416u,1591640736u,1592341120u,1593095776u,1593730656u,1594548864u,1595195008u,1595992704u,1596659328u,
1597430432u,1598054016u,1598752416u,1599509088u,1600207488u,1600948864u,1601602240u,1602433696u,1603036800u,1603843712u,1604510368u,1605156512u,1605954208u,1606620832u,1607391936u,1608015520u,
1608815232u,1609513632u,1610286784u,1610910368u,1611563744u,1612395200u,1612998304u,1613816512u,1614462656u,1615260352u,1615926976u,1616688800u,1617353440u,1617977024u,1618776736u,1619475136u,
1620248288u,1620871872u,1621656320u,1622304448u,1623122656u,1623768800u,1624435424u,1625233120u,1625895648u,1626650304u,1627314944u,1628082880u,1628724960u,1629567712u,1630209792u,1630833376u,
1631610592u,1632297728u,1633074944u,1633741568u,1634539264u,1635156768u,1635988224u,1636611808u,1637257952u,1638062880u,1638742784u,1639472896u,1640171296u,1640916736u,1641581312u,1642390304u,
1643036448u,1643834144u,1644500768u,1645118272u,1645949728u,1646573312u,1647350528u,1648048928u,1648810816u,1649434400u,1650222880u,1650919232u,1651565376u,1652342592u,1653009216u,1653806912u,
1654426400u,1655210848u,1655911232u,1656656672u,1657355072u,1657976608u,1658772320u,1659395904u,1660184384u,1660871520u,1661648736u,1662315360u,1663113056u,1663764320u,1664387904u,1665172352u,
1665831744u,1666661216u,1667314528u,1668069184u,1668733824u,1669488480u,1670145888u,1670833024u,1671610240u,1672276864u,1673074560u,1673725824u,1674480480u,1675137888u,1675920288u,1676622720u,
1677276032u,1678030688u,1678695328u,1679452032u,1680105344u,1680925600u,1681571744u,1682369440u,1683036064u,1683687328u,1684441984u,1685099392u,1685928864u,1686537152u,1687368608u,1687992192u,
1688758176u,1689443264u,1690100672u,1690877888u,1691544512u,1692342208u,1692950432u,1693779904u,1694405536u,1695235008u,1695845280u,1696498656u,1697330112u,1697953696u,1698719680u,1699406816u,
1700184032u,1700850656u,1701648352u,1702288352u,1702911936u,1703711680u,1704410080u,1705183200u,1705806784u,1706636256u,1707239392u,1708046272u,1708679136u,1709347808u,1710132192u,1710798816u,
1711636480u,1712422913u,1713209346u,1713995779u,1714782212u,1715568645u,1716355078u,1717141511u,1717927944u,1718714377u,1719500810u,1720287243u,1721073676u,1721860109u,1722646542u,1723432975u,
1724219408u,1725005841u,1725792274u,1726578707u,1727365140u,1728151573u,1728938006u,1729724439u,1730510872u,1731297305u,1732083738u,1732870171u,1733656604u,1734443037u,1735229470u,1736015903u,
1736802336u,1737588769u,1738375202u,1739292707u,1740079140u,1740865573u,1741652006u,1742438439u,1743224872u,1744011305u,1744797738u,1745584171u,1746370604u,1747157037u,1747943470u,1748729903u,
1749516336u,1750302769u,1751089202u,1751875635u,1752662068u,1753448501u,1754234934u,1755021367u,1755807800u,1756594233u,1757380666u,1758167099u,1758953532u,1759739965u,1760526398u,1761312831u,
1762100224u,1762886657u,1763673090u,1764459523u,1765245956u,1766032389u,1766818822u,1767605255u,1768391688u,1769178121u,1769964554u,1770750987u,1771537420u,1772323853u,1773110286u,1773896719u,
1774683152u,1775469585u,1776256018u,1777042451u,1777828884u,1778746389u,1779532822u,1780319255u,1781105688u,1781892121u,1782678554u,1783464987u,1784251420u,1785037853u,1785824286u,1786610719u,
1787397152u,1788183585u,1788970018u,1789756451u,1790542884u,1791329317u,1792115750u,1792902183u,1793688616u,1794475049u,1795261482u,1796047915u,1796834348u,1797620781u,1798407214u,1799193647u,
1799980080u,1800766513u,1801552946u,1802339379u,1803125812u,1803912245u,1804698678u,1805485111u,1806271544u,1807057977u,1807844410u,1808630843u,1809417276u,1810203709u,1810990142u,1811776575u,
1812563968u,1813350401u,1814136834u,1814923267u,1815709700u,1816496133u,1817282566u,1818200071u,1818986504u,1819772937u,1820559370u,1821345803u,1822132236u,1822918669u,1823705102u,1824491535u,
1825277968u,1826064401u,1826850834u,1827637267u,1828423700u,1829210133u,1829996566u,1830782999u,1831569432u,1832355865u,1833142298u,1833928731u,1834715164u,1835501597u,1836288030u,1837074463u,
1837860896u,1838647329u,1839433762u,1840220195u,1841006628u,1841793061u,1842579494u,1843365927u,1844152360u,1844938793u,1845725226u,1846511659u,1847298092u,1848084525u,1848870958u,1849657391u,
1850443824u,1851230257u,1852016690u,1852803123u,1853589556u,1854375989u,1855162422u,1855948855u,1856735288u,1857652793u,1858439226u,1859225659u,1860012092u,1860798525u,1861584958u,1862371391u,
1863158784u,1863945217u,1864731650u,1865518083u,1866304516u,1867090949u,1867877382u,1868663815u,1869450248u,1870236681u,1871023114u,1871809547u,1872595980u,1873382413u,1874168846u,1874955279u,
1875741712u,1876528145u,1877314578u,1878101011u,1878887444u,1879673877u,1880460310u,1881246743u,1882033176u,1882819609u,1883606042u,1884392475u,1885178908u,1885965341u,1886751774u,1887538207u,
1888324640u,1889111073u,1889897506u,1890683939u,1891470372u,1892256805u,1893043238u,1893829671u,1894616104u,1895402537u,1896188970u,1897106475u,1897892908u,1898679341u,1899465774u,1900252207u,
1901038640u,1901825073u,1902611506u,1903397939u,1904184372u,1904970805u,1905757238u,1906543671u,1907330104u,1908116537u,1908902970u,1909689403u,1910475836u,1911262269u,1912048702u,1912835135u,
1913622528u,1914408961u,1915195394u,1915981827u,1916768260u,1917554693u,1918341126u,1919127559u,1919913992u,1920700425u,1921486858u,1922273291u,1923059724u,1923846157u,1924632590u,1925419023u,
1926205456u,1926991889u,1927778322u,1928564755u,1929351188u,1930137621u,1930924054u,1931710487u,1932496920u,1933283353u,1934069786u,1934856219u,1935642652u,1936560157u,1937346590u,1938133023u,
1938919456u,1939705889u,1940492322u,1941278755u,1942065188u,1942851621u,1943638054u,1944424487u,1945210920u,1945997353u,1946783786u,1947570219u,1948356652u,1949143085u,1949929518u,1950715951u,
1951502384u,1952288817u,1953075250u,1953861683u,1954648116u,1955434549u,1956220982u,1957007415u,1957793848u,1958580281u,1959366714u,1960153147u,1960939580u,1961726013u,1962512446u,1963298879u,
1964086272u,1964872705u,1965659138u,1966445571u,1967232004u,1968018437u,1968804870u,1969591303u,1970377736u,1971164169u,1971950602u,1972737035u,1973523468u,1974309901u,1975096334u,1976013839u,
1976800272u,1977586705u,1978373138u,1979159571u,1979946004u,1980732437u,1981518870u,1982305303u,1983091736u,1983878169u,1984664602u,1985451035u,1986237468u,1987023901u,1987810334u,1988596767u,
1989383200u,1990169633u,1990956066u,1991742499u,1992528932u,1993315365u,1994101798u,1994888231u,1995674664u,1996461097u,1997247530u,1998033963u,1998820396u,1999606829u,2000393262u,2001179695u,
2001966128u,2002752561u,2003538994u,2004325427u,2005111860u,2005898293u,2006684726u,2007471159u,2008257592u,2009044025u,2009830458u,2010616891u,2011403324u,2012189757u,2012976190u,2013762623u,
2014550016u,2015467521u,2016253954u,2017040387u,2017826820u,2018613253u,2019399686u,2020186119u,2020972552u,2021758985u,2022545418u,2023331851u,2024118284u,2024904717u,2025691150u,2026477583u,
2027264016u,2028050449u,2028836882u,2029623315u,2030409748u,2031196181u,2031982614u,2032769047u,2033555480u,2034341913u,2035128346u,2035914779u,2036701212u,2037487645u,2038274078u,2039060511u,
2039846944u,2040633377u,2041419810u,2042206243u,2042992676u,2043779109u,2044565542u,2045351975u,2046138408u,2046924841u,2047711274u,2048497707u,2049284140u,2050070573u,2050857006u,2051643439u,
2052429872u,2053216305u,2054002738u,2054920243u,2055706676u,2056493109u,2057279542u,2058065975u,2058852408u,2059638841u,2060425274u,2061211707u,2061998140u,2062784573u,2063571006u,2064357439u,
2065144832u,2065931265u,2066717698u,2067504131u,2068290564u,2069076997u,2069863430u,2070649863u,2071436296u,2072222729u,2073009162u,2073795595u,2074582028u,2075368461u,2076154894u,2076941327u,
2077727760u,2078514193u,2079300626u,2080087059u,2080873492u,2081659925u,2082446358u,2083232791u,2084019224u,2084805657u,2085592090u,2086378523u,2087164956u,2087951389u,2088737822u,2089524255u,
2090310688u,2091097121u,2091883554u,2092669987u,2093456420u,2094373925u,2095160358u,2095946791u,2096733224u,2097519657u,2098306090u,2099092523u,2099878956u,2100665389u,2101451822u,2102238255u,
2103024688u,2103811121u,2104597554u,2105383987u,2106170420u,2106956853u,2107743286u,2108529719u,2109316152u,2110102585u,2110889018u,2111675451u,2112461884u,2113248317u,2114034750u,2114821183u,
2115608576u,2116395009u,2117181442u,2117967875u,2118754308u,2119540741u,2120327174u,2121113607u,2121900040u,2122686473u,2123472906u,2124259339u,2125045772u,2125832205u,2126618638u,2127405071u,
2128191504u,2128977937u,2129764370u,2130550803u,2131337236u,2132123669u,2132910102u,2133827607u,2134614040u,2135400473u,2136186906u,2136973339u,2137759772u,2138546205u,2139332638u,2140119071u,
2140905504u,2141691937u,2142478370u,2143264803u,2144051236u,2144837669u,2145624102u,2146410535u,2147196968u,2147983401u,2148769834u,2149556267u,2150342700u,2151129133u,2151915566u,2152701999u,
2153488432u,2154274865u,2155061298u,2155847731u,2156634164u,2157420597u,2158207030u,2158993463u,2159779896u,2160566329u,2161352762u,2162139195u,2162925628u,2163712061u,2164498494u,2165284927u,
2166072320u,2166858753u,2167645186u,2168431619u,2169218052u,2170004485u,2170790918u,2171577351u,2172363784u,2173281289u,2174067722u,2174854155u,2175640588u,2176427021u,2177213454u,2177999887u,
2178786320u,2179572753u,2180359186u,2181145619u,2181932052u,2182718485u,2183504918u,2184291351u,2185077784u,2185864217u,2186650650u,2187437083u,2188223516u,2189009949u,2189796382u,2190582815u,
2191369248u,2192155681u,2192942114u,2193728547u,2194514980u,2195301413u,2196087846u,2196874279u,2197660712u,2198447145u,2199233578u,2200020011u,2200806444u,2201592877u,2202379310u,2203165743u,
2203952176u,2204738609u,2205525042u,2206311475u,2207097908u,2207884341u,2208670774u,2209457207u,2210243640u,2211030073u,2211816506u,2212734011u,2213520444u,2214306877u,2215093310u,2215879743u,
2216667136u,2217453569u,2218240002u,2219026435u,2219812868u,2220599301u,2221385734u,2222172167u,2222958600u,2223745033u,2224531466u,2225317899u,2226104332u,2226890765u,2227677198u,2228463631u,
2229250064u,2230036497u,2230822930u,2231609363u,2232395796u,2233182229u,2233968662u,2234755095u,2235541528u,2236327961u,2237114394u,2237900827u,2238687260u,2239473693u,2240260126u,2241046559u,
2241832992u,2242619425u,2243405858u,2244192291u,2244978724u,2245765157u,2246551590u,2247338023u,2248124456u,2248910889u,2249697322u,2250483755u,2251270188u,2252187693u,2252974126u,2253760559u,
2254546992u,2255333425u,2256119858u,2256906291u,2257692724u,2258479157u,2259265590u,2260052023u,2260838456u,2261624889u,2262411322u,2263197755u,2263984188u,2264770621u,2265557054u,2266343487u,
2267130880u,2267917313u,2268703746u,2269490179u,2270276612u,2271063045u,2271849478u,2272635911u,2273422344u,2274208777u,2274995210u,2275781643u,2276568076u,2277354509u,2278140942u,2278927375u,
2279713808u,2280500241u,2281286674u,2282073107u,2282859540u,2283645973u,2284432406u,2285218839u,2286005272u,2286791705u,2287578138u,2288364571u,2289151004u,2289937437u,2290723870u,2291641375u,
2292427808u,2293214241u,2294000674u,2294787107u,2295573540u,2296359973u,2297146406u,2297932839u,2298719272u,2299505705u,2300292138u,2301078571u,2301865004u,2302651437u,2303437870u,2304224303u,
2305010736u,2305797169u,2306583602u,2307370035u,2308156468u,2308942901u,2309729334u,2310515767u,2311302200u,2312088633u,2312875066u,2313661499u,2314447932u,2315234365u,2316020798u,2316807231u,
2317594624u,2318381057u,2319167490u,2319953923u,2320740356u,2321526789u,2322313222u,2323099655u,2323886088u,2324672521u,2325458954u,2326245387u,2327031820u,2327818253u,2328604686u,2329391119u,
2330177552u,2331095057u,2331881490u,2332667923u,2333454356u,2334240789u,2335027222u,2335813655u,2336600088u,2337386521u,2338172954u,2338959387u,2339745820u,2340532253u,2341318686u,2342105119u,
2342891552u,2343677985u,2344464418u,2345250851u,2346037284u,2346823717u,2347610150u,2348396583u,2349183016u,2349969449u,2350755882u,2351542315u,2352328748u,2353115181u,2353901614u,2354688047u,
2355474480u,2356260913u,2357047346u,2357833779u,2358620212u,2359406645u,2360193078u,2360979511u,2361765944u,2362552377u,2363338810u,2364125243u,2364911676u,2365698109u,2366484542u,2367270975u,
2368058368u,2368844801u,2369631234u,2370548739u,2371335172u,2372121605u,2372908038u,2373694471u,2374480904u,2375267337u,2376053770u,2376840203u,2377626636u,2378413069u,2379199502u,2379985935u,
2380772368u,2381558801u,2382345234u,2383131667u,2383918100u,2384704533u,2385490966u,2386277399u,2387063832u,2387850265u,2388636698u,2389423131u,2390209564u,2390995997u,2391782430u,2392568863u,
2393355296u,2394141729u,2394928162u,2395714595u,2396501028u,2397287461u,2398073894u,2398860327u,2399646760u,2400433193u,2401219626u,2402006059u,2402792492u,2403578925u,2404365358u,2405151791u,
2405938224u,2406724657u,2407511090u,2408297523u,2409083956u,2410001461u,2410787894u,2411574327u,2412360760u,2413147193u,2413933626u,2414720059u,2415506492u,2416292925u,2417079358u,2417865791u,
2418653184u,2419439617u,2420226050u,2421012483u,2421798916u,2422585349u,2423371782u,2424158215u,2424944648u,2425731081u,2426517514u,2427303947u,2428090380u,2428876813u,2429663246u,2430449679u,
2431236112u,2432022545u,2432808978u,2433595411u,2434381844u,2435168277u,2435954710u,2436741143u,2437527576u,2438314009u,2439100442u,2439886875u,2440673308u,2441459741u,2442246174u,2443032607u,
2443819040u,2444605473u,2445391906u,2446178339u,2446964772u,2447751205u,2448537638u,2449455143u,2450241576u,2451028009u,2451814442u,2452600875u,2453387308u,2454173741u,2454960174u,2455746607u,
2456533040u,2457319473u,2458105906u,2458892339u,2459678772u,2460465205u,2461251638u,2462038071u,2462824504u,2463610937u,2464397370u,2465183803u,2465970236u,2466756669u,2467543102u,2468329535u,
2469116928u,2469903361u,2470689794u,2471476227u,2472262660u,2473049093u,2473835526u,2474621959u,2475408392u,2476194825u,2476981258u,2477767691u,2478554124u,2479340557u,2480126990u,2480913423u,
2481699856u,2482486289u,2483272722u,2484059155u,2484845588u,2485632021u,2486418454u,2487204887u,2487991320u,2488908825u,2489695258u,2490481691u,2491268124u,2492054557u,2492840990u,2493627423u,
2494413856u,2495200289u,2495986722u,2496773155u,2497559588u,2498346021u,2499132454u,2499918887u,2500705320u,2501491753u,2502278186u,2503064619u,2503851052u,2504637485u,2505423918u,2506210351u,
2506996784u,2507783217u,2508569650u,2509356083u,2510142516u,2510928949u,2511715382u,2512501815u,2513288248u,2514074681u,2514861114u,2515647547u,2516433980u,2517220413u,2518006846u,2518793279u,
4096u,4112u,4097u,4113u,16384u,38912u,50176u,61440u,5120u,5136u,5121u,5137u,6144u,6160u,6145u,6161u,
16416u,38944u,50208u,61472u,7168u,7184u,7169u,7185u,72704u,786432u,1703952u,2621441u,3538961u,4474880u,5390400u,6330432u,
7259200u,8187968u,9046016u,9963536u,10881025u,11667473u,12612608u,13501440u,14418960u,15336449u,16253969u,17243168u,18091008u,19008528u,19926017u,20843537u,
21776416u,22552576u,23470096u,24387585u,25305105u,26230880u,27170912u,28099680u,29028448u,29914144u,30811136u,31728656u,32646145u,33563665u,34413632u,35258496u,
36176016u,37093505u,38011025u,38944896u,39856128u,40773648u,41691137u,42608657u,43534400u,44472448u,45362176u,46148624u,47066113u,47983633u,48906400u,49809536u,
50727056u,51644545u,52562065u,53481600u,54399120u,55316609u,56234129u,57066624u,57995392u,58881088u,59769984u,60687504u,61604993u,62522513u,63477920u,64361600u,
65279120u,66196609u,67114129u,68042944u,68885600u,69733504u,70651024u,71568513u,72486033u,73406592u,74324112u,75241601u,76159121u,77088864u,78026944u,78921952u,
79742112u,80670880u,81534080u,82451600u,83369089u,84286609u,85226592u,86153440u,87032064u,87949584u,88867073u,89784593u,90718464u,91430016u,92062912u,92817536u,
93493504u,94257440u,94898432u,95684880u,96471297u,97388817u,98234560u,98878688u,99644544u,100271360u,101188880u,101975297u,102761745u,103586080u,104350016u,105050336u,
105847968u,106433792u,107220240u,108006657u,108924177u,109757696u,110403584u,111188288u,111823008u,112599168u,113385616u,114172033u,114958481u,115884384u,116584704u,117359904u,
117994656u,118629504u,119547024u,120333441u,121119889u,121936224u,122683776u,123470224u,124256641u,125174161u,125976960u,126677280u,127474880u,128107840u,128854144u,129640592u,
130427009u,131213457u,132161568u,132815232u,133448096u,134236352u,134884480u,135802000u,136588417u,137374865u,138152320u,139069840u,139856257u,140642705u,141488448u,142263648u,
142872832u,143790352u,144576769u,145363217u,146146688u,147064208u,147850625u,148637073u,149451968u,150247840u,150880704u,151712096u,152346688u,152966400u,153883920u,154670337u,
155456785u,156372352u,157158800u,157945217u,158731665u,159696256u,160373984u,160995776u,161759712u,162404608u,163322128u,164108545u,164894993u,165693664u,166523264u,167167392u,
167910656u,168697104u,169483521u,170269969u,171209952u,171874784u,172491264u,173408784u,174195201u,174981649u,175915520u,176595040u,177271200u,178046400u,178724096u,179476992u,
180109856u,180750848u,181668368u,182454785u,183241233u,184086976u,184830208u,185517536u,186254848u,187041296u,187827713u,188745233u,189571200u,190224928u,191001856u,191644224u,
192344544u,193072640u,193859088u,194645505u,195563025u,196396544u,197074208u,197827136u,198460000u,199237888u,200024336u,200810753u,201597201u,202568192u,203180320u,203867680u,
204644512u,205268224u,206185744u,206972161u,207758609u,208575072u,209322624u,210109072u,210895489u,211813009u,212615808u,213284128u,214102560u,214746688u,215361792u,216279312u,
217065729u,217852177u,218832192u,219454080u,220086944u,220859008u,221645456u,222431873u,223349393u,224195136u,224829632u,225586432u,226372880u,227159297u,227945745u,228902496u,
229525824u,230295168u,231081616u,231868033u,232654481u,233609888u,234230144u,235016592u,235934081u,236720529u,237519552u,238318912u,239006304u,239603328u,240520848u,241307265u,
242093713u,242927232u,243668352u,244454800u,245241217u,246158737u,247012704u,247634624u,248398560u,249077984u,249885312u,250529440u,251140480u,252058000u,252844417u,253630865u,
254429536u,255236832u,255853312u,256770832u,257557249u,258343697u,259146496u,259923328u,260709776u,261496193u,262413713u,263254688u,263877984u,264685248u,265362816u,265986304u,
266771200u,267404064u,268176128u,268962576u,269748993u,270535441u,271512256u,272156384u,272955392u,273566080u,274204416u,275121936u,275908353u,276694801u,277519136u,278283072u,
278983392u,279737728u,280366848u,281153296u,282070785u,282857233u,283690752u,284467488u,285121344u,285811712u,286596512u,287196000u,288027392u,288684064u,289284480u,290202000u,
290988417u,291774865u,292603680u,293359616u,294013344u,294820704u,295437184u,296223632u,297141121u,297927569u,298730368u,299509120u,300295568u,301081985u,301868433u,302838560u,
303463424u,304269120u,304914752u,305558944u,306390048u,306981248u,307767696u,308685185u,309471633u,310287232u,311051168u,311762368u,312347520u,313265040u,314051457u,314837905u,
315816000u,316470080u,317114208u,317870112u,318516608u,319303056u,320220545u,321006993u,321783680u,322701200u,323487617u,324274065u,325077440u,325884832u,326517696u,327328096u,
327973920u,328659808u,329388544u,330174992u,330961409u,331878929u,332664704u,333451152u,334237569u,335155089u,335988608u,336607232u,337524752u,338311169u,339097617u,339899840u,
340730944u,341351360u,342115296u,342816864u,343414272u,344331792u,345118209u,345904657u,346889696u,347534208u,348142592u,349060112u,349846529u,350632977u,351455136u,352211008u,
352858112u,353775632u,354562049u,355348497u,356135936u,357053456u,357839873u,358626321u,359406080u,360323600u,361110017u,361896465u,362847200u,363481568u,364159360u,364936256u,
365622176u,366397376u,367076448u,367664640u,368582160u,369368577u,370155025u,370963936u,371774464u,372439168u,373224384u,373837920u,374545920u,375310304u,375955872u,376732768u,
377375232u,378074080u,378872960u,379518976u,380294176u,380925440u,381745312u,382357632u,383065632u,383821824u,384475584u,385229312u,386015760u,386802177u,387588625u,388529280u,
389171744u,389817856u,390735376u,391521793u,392308241u,393290912u,393936928u,394590400u,395343392u,395979264u,396765712u,397683201u,398469649u,399265952u,400074784u,400728544u,
401415744u,402140672u,402927120u,403713537u,404631057u,405437600u,406136000u,406866496u,407566560u,408294016u,409080464u,409866881u,410653329u,411586752u,412251712u,412919872u,
413704704u,414324352u,415241872u,416028289u,416814737u,417668704u,418413760u,419112160u,419830400u,420616848u,421403265u,422189713u,423112832u,423899280u,424685697u,425472145u,
426396256u,427096320u,427873344u,428495072u,429172832u,429957664u,430579328u,431365776u,432283265u,433069713u,433878624u,434648192u,435434640u,436221057u,437138577u,437926016u,
438712464u,439498881u,440416401u,441203840u,441990288u,442776705u,443694225u,444497120u,445193856u,445981952u,446626080u,447367808u,448069696u,448713824u,449489024u,450142784u,
450908416u,451573376u,452230400u,453058208u,453649024u,454566544u,455352961u,456139409u,456992032u,457767232u,458424416u,459199616u,459843744u,460497504u,461261472u,461907584u,
462694032u,463611521u,464397969u,465195296u,465991328u,466627200u,467413648u,468331137u,469117585u,469925152u,470752960u,471409984u,472002176u,472919696u,473706113u,474492561u,
475332736u,476117344u,476763296u,477536896u,478193856u,478825152u,479613248u,480321248u,481039104u,481825552u,482611969u,483529489u,484343104u,484996800u,485765376u,486551824u,
487338241u,488124689u,489104736u,489690880u,490477328u,491394817u,492181265u,493027488u,493802688u,494445216u,495102176u,495909248u,496519904u,497308000u,497956096u,498742544u,
499660033u,500446481u,501226240u,502012688u,502930177u,503716625u,504510720u,505297168u,506214657u,507001105u,507788544u,508706064u,509492481u,510278929u,511123200u,511868256u,
512521952u,513285888u,514072336u,514858753u,515645201u,516629888u,517275840u,517907136u,518706400u,519350528u,520157600u,520768256u,521556352u,522264352u,522876672u,523664768u,
524363168u,525140192u,525771488u,526439680u,527214880u,527846176u,528677312u,529277696u,530064144u,530981633u,531768081u,532566432u,533274432u,534017824u,534652672u,535570192u,
536356609u,537143057u,537951648u,538781120u,539427072u,540189440u,540857632u,541501760u,542255872u,543042320u,543828737u,544746257u,545540928u,546240992u,546984384u,547630848u,
548417296u,549334785u,550121233u,550969184u,551712576u,552369600u,553005440u,553922960u,554709377u,555495825u,556311328u,557121824u,557786592u,558552384u,559196512u,559958880u,
560599936u,561386384u,562172801u,563090321u,563935744u,564540800u,565458320u,566244737u,567031185u,567955936u,568605056u,569391504u,570177921u,571095441u,571882880u,572669328u,
573586817u,574373265u,575217536u,575805312u,576722832u,577509249u,578295697u,579237728u,579894752u,580559680u,581370176u,582014304u,582789504u,583408512u,584194960u,585112449u,
585898897u,586753536u,587352960u,588184096u,588783488u,589701008u,590487425u,591273873u,592073216u,592912288u,593524608u,594312704u,594977632u,595657056u,596432256u,597076384u,
597894688u,598494112u,599194176u,599937568u,600645568u,601388960u,602022784u,602809232u,603726721u,604513169u,605322784u,606118784u,606798208u,607442336u,608217536u,608848832u,
609691200u,610281344u,611067792u,611985281u,612771729u,613612128u,614355520u,615000960u,615787408u,616573825u,617491345u,618340320u,618952640u,619721216u,620507664u,621294081u,
622211601u,623027104u,623672896u,624492960u,625137088u,625880960u,626667408u,627453825u,628240273u,629189088u,629820384u,630531680u,631256576u,632043024u,632829441u,633746961u,
634583680u,635189760u,636107280u,636893697u,637680145u,638467584u,639385104u,640171521u,640957969u,641749600u,642545632u,643252224u,643998656u,644634624u,645464512u,646086240u,
646764000u,647539200u,648226432u,648971264u,649656992u,650269312u,651116576u,651735040u,652518368u,653197792u,653841920u,654617120u,655261312u,656037920u,656727040u,657401504u,
658146336u,658832064u,659621888u,660291648u,660886176u,661696544u,662385664u,663159296u,663803424u,664447552u,665222816u,665868352u,666688544u,667363008u,668098560u,668793568u,
669418560u,670253152u,670894112u,671634112u,672281216u,673067664u,673854081u,674771601u,675611168u,676255296u,677030496u,677687360u,678507552u,679116480u,679814880u,680548448u,
681195136u,682112656u,682899073u,683685521u,684514368u,685259392u,686045840u,686832257u,687749777u,688537216u,689323664u,690110081u,691027601u,691822624u,692517632u,693321856u,
693916384u,694715488u,695404608u,696058432u,696833632u,697477760u,698253024u,698921056u,699610176u,700393216u,700995712u,701815904u,702479136u,703152288u,703881280u,704533248u,
705332352u,706019936u,706676832u,707450496u,708094624u,708869888u,709537920u,710358112u,711010080u,711672000u,712399008u,713096000u,713874560u,714498144u,715150112u,715981440u,
716625568u,717400768u,718046368u,718735488u,719486752u,720154784u,720971584u,721630336u,722288864u,723057504u,723671232u,724491424u,725111616u,725758720u,726676240u,727462657u,
728249105u,729178240u,729875104u,730519232u,731294432u,731940032u,732760224u,733380416u,734017280u,734934800u,735721217u,736507665u,737295104u,738212624u,738999041u,739785489u,
740704000u,741490448u,742276865u,743063313u,744040288u,744702208u,745320640u,746140832u,746781568u,747524960u,748225216u,748837088u,749655776u,750299904u,751099072u,751722656u,
752516960u,753173728u,753862848u,754657152u,755319072u,756068576u,756743072u,757413056u,758186720u,758797184u,759617280u,760261408u,760895744u,761715936u,762339520u,763133824u,
763790592u,764479712u,765274016u,765935936u,766704576u,767340800u,768017152u,768792352u,769471712u,770222912u,770855840u,771512608u,772332800u,772956384u,773750688u,774455648u,
775235520u,775849248u,776538368u,777321440u,777978656u,778711936u,779498384u,780284801u,781071249u,782030656u,782674784u,783307712u,784118048u,784807168u,785537344u,786184064u,
786970512u,787888001u,788674449u,789479360u,790289696u,790903680u,791690128u,792476545u,793394065u,794181504u,794967952u,795885441u,796671889u,797466880u,798310784u,798959584u,
799573312u,800393504u,801047360u,801822560u,802466688u,803099616u,803909952u,804599072u,805339104u,805984608u,806699424u,807460160u,808083744u,808911712u,809555840u,810190176u,
810986400u,811665728u,812429664u,813118784u,813777344u,814504320u,815193440u,815948096u,816644992u,817420192u,818064320u,818709888u,819530080u,820162944u,820983136u,821641696u,
822237600u,823057792u,823722912u,824498112u,825123168u,825797600u,826587136u,827229600u,828049792u,828695552u,829374976u,830124448u,830813568u,831587264u,832231392u,832901120u,
833640896u,834320416u,835116448u,835740032u,836404672u,837237760u,837880224u,838657056u,839336480u,840085952u,840775072u,841417696u,842164224u,842862624u,843626560u,844257760u,
844946880u,845701536u,846369792u,847199264u,847832128u,848463328u,849297984u,849938880u,850683936u,851358176u,852037696u,852801632u,853489088u,854219264u,854908384u,855531968u,
856331296u,857029696u,857793632u,858473056u,859080192u,859900384u,860514368u,861343840u,861976704u,862761472u,863450592u,864064576u,864836128u,865549408u,866313344u,866967040u,
867648128u,868377056u,869041696u,869861888u,870475872u,871174272u,871938208u,872591904u,873412096u,874026080u,874724480u,875488416u,876167840u,876894784u,877583904u,878207488u,
878995584u,879693984u,880457920u,881100352u,881789472u,882553408u,883201152u,884030624u,884663488u,885470752u,886129344u,886725216u,887545408u,888168992u,888957088u,889655488u,
890288352u,891061856u,891750976u,892507296u,893205696u,893838560u,894649056u,895267424u,896087616u,896686720u,897345728u,898175200u,898817632u,899594496u,900227648u,900892288u,
901712480u,902337728u,903167200u,903800064u,904479488u,905228928u,905918048u,906651872u,907350272u,908114208u,908745376u,909434496u,910189152u,910857440u,911686912u,912295584u,
912975136u,913771136u,914440992u,915190432u,915826944u,916534912u,917311776u,917944640u,918706880u,919396000u,920019584u,920818944u,921517344u,922281280u,922960704u,923567808u,
924388000u,925002016u,925831488u,926464352u,927118016u,927938208u,928537312u,929338656u,930012864u,930823488u,931456352u,932078240u,932922208u,933529312u,934349504u,934963520u,
935661920u,936425856u,937079520u,937899712u,938513728u,939212128u,939976064u,940595968u,941441920u,942071520u,942695104u,943483232u,944146176u,944968064u,945600928u,946277088u,
947041024u,947730144u,948475232u,949173632u,949806496u,950616992u,951212832u,952033024u,952656608u,953444736u,954143136u,954776000u,955549472u,956238592u,956994944u,957693344u,
958326208u,959099680u,959792064u,960575232u,961174336u,961863456u,962619808u,963318208u,964059904u,964737504u,965379904u,966200096u,966825376u,967654848u,968287712u,968967136u,
969716544u,970405664u,971139520u,971837920u,972470784u,973232992u,973922112u,974676768u,975341408u,976131520u,976829920u,977472320u,978249216u,978898944u,979696640u,980363264u,
980964352u,981775200u,982464320u,983198176u,983896576u,984529440u,985291648u,985980768u,986735424u,987403744u,988102144u,988866080u,989515808u,990313504u,990980128u,991581216u,
992380800u,993069920u,993815040u,994513440u,995144576u,995932736u,996620128u,997350304u,998039424u,998807040u,999449440u,1000160800u,1000924736u,1001555872u,1002376064u,1002990112u,
1003688512u,1004452448u,1005106080u,1005926272u,1006540320u,1007238720u,1007967168u,1008658016u,1009442720u,1010066304u,1010730944u,1011551136u,1012165184u,1012994656u,1013627520u,1014281152u,
1015101344u,1015715392u,1016544864u,1017177728u,1017797600u,1018617792u,1019241376u,1020029536u,1020727936u,1021491872u,1022134240u,1022823360u,1023579744u,1024242656u,1025064576u,1025697440u,
1026373568u,1027129344u,1027792864u,1028547520u,1029204608u,1030034080u,1030666944u,1031346176u,1032129504u,1032754816u,1033584288u,1034217152u,1034961920u,1035679712u,1036435488u,1037068960u,
1037767360u,1038531296u,1039167488u,1039950816u,1040652320u,1041406976u,1042060960u,1042890432u,1043523296u,1044136992u,1044957184u,1045588672u,1046418144u,1047051008u,1047838784u,1048473632u,
1049293824u,1049925312u,1050623712u,1051387648u,1052055616u,1052810272u,1053443136u,1054239456u,1054918688u,1055724288u,1056357152u,1057013856u,1057779776u,1058468896u,1059231456u,1059929856u,
1060693792u,1061361760u,1062116416u,1062759168u,1063457568u,1064221504u,1064846432u,1065666624u,1066320000u,1067095808u,1067794208u,1068527712u,1069213504u,1069872192u,1070667904u,1071291488u,
1072065312u,1072763712u,1073527648u,1074152576u,1074972768u,1075615520u,1076313920u,1077077856u,1077723296u,1078489216u,1079178336u,1079929664u,1080629408u,1081414496u,1082047360u,1082694784u,
1083458720u,1084147840u,1084921664u,1085620064u,1086384000u,1087029440u,1087795360u,1088484480u,1089104736u,1089934208u,1090567072u,1091377344u,1092000928u,1092786016u,1093484416u,1094248352u,
1094862016u,1095682208u,1096335584u,1096970432u,1097755520u,1098453920u,1099232416u,1099873216u,1100683488u,1101307072u,1102092160u,1102790560u,1103423424u,1104168160u,1104857280u,1105619872u,
1106318272u,1107082208u,1107738880u,1108504800u,1109193920u,1109858560u,1110611872u,1111310272u,1112074208u,1112710368u,1113474304u,1114163424u,1114926016u,1115624416u,1116282880u,1117045024u,
1117679872u,1118500064u,1119131584u,1119961056u,1120565248u,1121392928u,1122016512u,1122659296u,1123460096u,1124091168u,1124933664u,1125566720u,1126351168u,1126986016u,1127782368u,1128461568u,
1129084960u,1129912640u,1130536224u,1131324448u,1132011584u,1132788800u,1133455424u,1134187584u,1134839104u,1135528224u,1136312672u,1136949312u,1137733952u,1138423072u,1139218784u,1139842368u,
1140630592u,1141317728u,1141963872u,1142761568u,1143362656u,1144145248u,1144834368u,1145622592u,1146255456u,1147060608u,1147695456u,1148384576u,1149150304u,1149837440u,1150614656u,1151281280u,
1152013440u,1152719232u,1153473888u,1154106752u,1154797664u,1155561600u,1156237664u,1157022112u,1157656960u,1158477152u,1159111808u,1159930016u,1160576160u,1161242784u,1161974944u,1162680736u,
1163435392u,1164103808u,1164867744u,1165510048u,1166330240u,1166983616u,1167631520u,1168449728u,1169095872u,1169893568u,1170494656u,1171288480u,1171977600u,1172773312u,1173396896u,1174065312u,
1174829248u,1175471552u,1176291744u,1176937664u,1177755872u,1178402016u,1179199712u,1179800800u,1180484064u,1181249984u,1181939104u,1182716096u,1183348960u,1184176608u,1184800192u,1185564128u,
1186243808u,1187039680u,1187717376u,1188363520u,1189161216u,1189762304u,1190576640u,1191211488u,1192031680u,1192677600u,1193441536u,1194138112u,1194761696u,1195549952u,1196237088u,1197014304u,
1197680928u,1198413088u,1199064576u,1199884768u,1200538144u,1201197312u,1201959424u,1202616608u,1203434976u,1204099616u,1204854272u,1205511456u,1206329664u,1206975808u,1207642432u,1208374592u,
1209026080u,1209846272u,1210503456u,1211267392u,1211941440u,1212707360u,1213396480u,1214031168u,1214847552u,1215504736u,1216281952u,1216948576u,1217680736u,1218354720u,1219118656u,1219807776u,
1220464960u,1221228896u,1221902944u,1222668864u,1223357984u,1224123744u,1224810880u,1225588096u,1226254720u,1226855808u,1227692640u,1228316224u,1229115744u,1229748608u,1230521952u,1231211072u,
1231995520u,1232630368u,1233298816u,1234117024u,1234763168u,1235560864u,1236161952u,1236989504u,1237654144u,1238408800u,1239077248u,1239710112u,1240483456u,1241172576u,1241949600u,1242636736u,
1243413952u,1244080576u,1244812736u,1245495968u,1246130816u,1246951008u,1247615648u,1248383392u,1249016256u,1249812096u,1250444960u,1251265152u,1251911104u,1252598240u,1253375456u,1254042080u,
1254774240u,1255457472u,1256223392u,1256912512u,1257689536u,1258322400u,1259018944u,1259773600u,1260430816u,1261192896u,1261904384u,1262681600u,1263348224u,1264080384u,1264765600u,1265418976u,
1266184896u,1266864608u,1267660448u,1268283904u,1269111520u,1269735104u,1270523392u,1271210528u,1271856672u,1272654368u,1273255456u,1274037984u,1274727104u,1275515392u,1276148256u,1276953344u,
1277588192u,1278408384u,1279073024u,1279696608u,1280484896u,1281172032u,1281949248u,1282615872u,1283348032u,1283999488u,1284819680u,1285476896u,1286109760u,1286914848u,1287549696u,1288369888u,
1289004608u,1289822816u,1290468960u,1291266656u,1291867744u,1292573472u,1293328128u,1293996608u,1294747424u,1295415904u,1296222976u,1296876352u,1297642272u,1298331392u,1298966112u,1299784320u,
1300430464u,1301228160u,1301829248u,1302666048u,1303289632u,1304089184u,1304722048u,1305364288u,1306184480u,1306837856u,1307616896u,1308304032u,1309081248u,1309747872u,1310480032u,1311142720u,
1311831840u,1312627552u,1313251136u,1314050688u,1314683552u,1315456864u,1316145984u,1316923040u,1317610176u,1318256320u,1319054016u,1319655104u,1320469376u,1321104224u,1321924416u,1322570400u,
1323334336u,1324030848u,1324654432u,1325418368u,1326098112u,1326893920u,1327571680u,1328348896u,1329015520u,1329747680u,1330430880u,1331065728u,1331885920u,1332531904u,1333295840u,1333992352u,
1334747008u,1335404256u,1336222464u,1336868608u,1337535232u,1338267392u,1338918816u,1339739008u,1340392384u,1341182688u,1341815552u,1342600096u,1343289216u,1343953856u,1344708512u,1345365760u,
1346183968u,1346830112u,1347627808u,1348228896u,1349011392u,1349700512u,1350357760u,1351121696u,1351795680u,1352561600u,1353250720u,1354016544u,1354701792u,1355490112u,1356136256u,1356802880u,
1357535040u,1358208960u,1358972896u,1359662016u,1360450336u,1361083200u,1361870848u,1362523104u,1363212224u,1363978048u,1364665184u,1365442400u,1366109024u,1366841184u,1367529472u,1368301536u,
1368970048u,1369733984u,1370424320u,1371065312u,1371832352u,1372532736u,1373284192u,1373971328u,1374748544u,1375415168u,1376147328u,1376843744u,1377490976u,1378245632u,1378931552u,1379695488u,
1380385824u,1381140480u,1381803904u,1382622112u,1383268256u,1383934880u,1384667040u,1385332800u,1386164256u,1386787840u,1387583552u,1388237696u,1389001632u,1389648928u,1390347328u,1391101984u,
1391765408u,1392583616u,1393229760u,1394027456u,1394628544u,1395425376u,1396125760u,1396749344u,1397543840u,1398176704u,1398986848u,1399610432u,1400416192u,1401103328u,1401880544u,1402547168u,
1403148256u,1403978848u,1404602432u,1405386880u,1406087264u,1406849984u,1407497280u,1408269280u,1408948352u,1409571936u,1410377696u,1411047424u,1411824640u,1412491264u,1413288960u,1413940352u,
1414695008u,1415369696u,1416050688u,1416790176u,1417490560u,1418245216u,1418909856u,1419666432u,1420319872u,1421140000u,1421786144u,1422452768u,1423250464u,1423901856u,1424656512u,1425313792u,
1426143264u,1426751680u,1427583136u,1428206720u,1428841504u,1429659712u,1430305856u,1431103552u,1431770176u,1432541376u,1433164960u,1433964576u,1434649792u,1435318336u,1436059808u,1436713184u,
1437544640u,1438168224u,1438934080u,1439621216u,1440398432u,1441065056u,1441731680u,1442502880u,1443126464u,1443926080u,1444624480u,1445397728u,1446021312u,1446798432u,1447461120u,1448140928u,
1448918144u,1449584768u,1450382464u,1451045088u,1451799744u,1452464384u,1453219040u,1453887584u,1454585984u,1455359232u,1455982816u,1456759936u,1457447072u,1458224288u,1458890912u,1459688608u,
1460306208u,1461137664u,1461761248u,1462407296u,1463236768u,1463867680u,1464622336u,1465320736u,1466066080u,1466753216u,1467530432u,1468197056u,1468863680u,1469614336u,1470267712u,1471099168u,
1471722752u,1472499872u,1473198272u,1473960256u,1474583840u,1475241152u,1476059360u,1476705504u,1477503200u,1478169824u,1478952256u,1479575840u,1480360288u,1481019584u,1481717984u,1482502464u,
1483126048u,1483921760u,1484545344u,1485333728u,1486020864u,1486798080u,1487464704u,1488131328u,1488913760u,1489537344u,1490325728u,1491024128u,1491763584u,1492463968u,1493218624u,1493853440u,
1494538624u,1495327008u,1495973152u,1496770848u,1497437472u,1498176864u,1498875264u,1499631872u,1500285280u,1500985632u,1501725088u,1502425472u,1503180128u,1503814944u,1504633152u,1505279296u,
1506076992u,1506743616u,1507383712u,1508138368u,1508806944u,1509636416u,1510278560u,1511033216u,1511686592u,1512518048u,1513121088u,1513808224u,1514585440u,1515252064u,1516049760u,1516680576u,
1517476288u,1518099872u,1518899520u,1519597920u,1520240064u,1520994720u,1521640800u,1522459008u,1523105152u,1523902848u,1524569472u,1525318112u,1526018496u,1526642080u,1527419232u,1528093152u,
1528904064u,1529503168u,1530332640u,1530956224u,1531733376u,1532420512u,1533066656u,1533864352u,1534530976u,1535279616u,1535980000u,1536734656u,1537380736u,1538210208u,1538841088u,1539464672u,
1540253088u,1540940224u,1541717440u,1542384064u,1543181760u,1543833088u,1544587744u,1545241120u,1545941504u,1546686880u,1547385280u,1548137952u,1548802592u,1549557248u,1550214592u,1551032800u,
1551678944u,1552476640u,1553143264u,1553794592u,1554549248u,1555206592u,1556036064u,1556644416u,1557475872u,1558099456u,1558895168u,1559520736u,1560174112u,1560994304u,1561640448u,1562438144u,
1563104768u,1563887168u,1564510752u,1565299168u,1565997568u,1566605920u,1567437376u,1568060960u,1568826880u,1569514016u,1570291232u,1570957856u,1571755552u,1572395616u,1573019200u,1573818880u,
1574517280u,1575290464u,1575914048u,1576698496u,1577398880u,1578153536u,1578788384u,1579475520u,1580252736u,1580919360u,1581717056u,1582357120u,1583111776u,1583780384u,1584609856u,1585251968u,
1585875552u,1586652736u,1587315360u,1588126304u,1588772448u,1589570144u,1590236768u,1591030400u,1591653984u,1592318624u,1593073280u,1593741888u,1594571360u,1595213472u,1595968128u,1596614240u,
1597432448u,1598078592u,1598745216u,1599542912u,1600160448u,1600991904u,1601615488u,1602392672u,1603091072u,1603852992u,1604476576u,1605174976u,1605920384u,1606607520u,1607384736u,1608051360u,
1608849056u,1609468576u,1610253024u,1610953408u,1611576992u,1612354176u,1613052576u,1613814496u,1614438080u,1615226528u,1615913664u,1616690880u,1617357504u,1618024128u,1618806496u,1619430080u,
1620214528u,1620873888u,1621703360u,1622356704u,1623111360u,1623776000u,1624399584u,1625188032u,1625875168u,1626652384u,1627319008u,1628116704u,1628768000u,1629522656u,1630180032u,1630878432u,
1631617824u,1632318208u,1633072864u,1633707744u,1634525952u,1635172096u,1635969792u,1636636416u,1637276448u,1638031104u,1638729504u,1639486176u,1640139520u,1640971008u,1641579328u,1642410784u,
1643034368u,1643800320u,1644487456u,1645133600u,1645931296u,1646597920u,1647369024u,1647992608u,1648792320u,1649490720u,1650263872u,1650887456u,1651540832u,1652320032u,1653027648u,1653793600u,
1654439744u,1655237440u,1655904064u,1656665888u,1657330528u,1657954112u,1658753824u,1659452224u,1660225376u,1660848960u,1661626176u,1662313312u,1663090528u,1663757152u,1664423776u,1665172352u,
1665872736u,1666627392u,1667273536u,1668078464u,1668758368u,1669488480u,1670186880u,1670810464u,1671587680u,1672274816u,1673052032u,1673718656u,1674516352u,1675133856u,1675965312u,1676588896u,
1677235040u,1678064512u,1678695328u,1679449984u,1680107392u,1680925600u,1681571744u,1682369440u,1683036064u,1683687328u,1684441984u,1685095360u,1685926816u,1686541184u,1687370656u,1687992192u,
1688787904u,1689411488u,1690068896u,1690887104u,1691533248u,1692330944u,1692997568u,1693779904u,1694403488u,1695191968u,1695890368u,1696498656u,1697330112u,1697953696u,1698749408u,1699375040u,
1700193248u,1700839392u,1701637088u,1702303712u,1702911936u,1703741408u,1704364992u,1705153472u,1705851872u,1706636256u,1707259840u,1708025824u,1708679136u,1709347808u,1710132192u,1710798816u,
1711636480u,1712422913u,1713209346u,1713995779u,1714782212u,1715568645u,1716355078u,1717141511u,1717927944u,1718714377u,1719500810u,1720287243u,1721073676u,1721860109u,1722646542u,1723432975u,
1724219408u,1725005841u,1725792274u,1726578707u,1727365140u,1728151573u,1728938006u,1729724439u,1730510872u,1731297305u,1732083738u,1732870171u,1733656604u,1734443037u,1735229470u,1736015903u,
1736802336u,1737588769u,1738375202u,1739292707u,1740079140u,1740865573u,1741652006u,1742438439u,1743224872u,1744011305u,1744797738u,1745584171u,1746370604u,1747157037u,1747943470u,1748729903u,
1749516336u,1750302769u,1751089202u,1751875635u,1752662068u,1753448501u,1754234934u,1755021367u,1755807800u,1756594233u,1757380666u,1758167099u,1758953532u,1759739965u,1760526398u,1761312831u,
1762100224u,1762886657u,1763673090u,1764459523u,1765245956u,1766032389u,1766818822u,1767605255u,1768391688u,1769178121u,1769964554u,1770750987u,1771537420u,1772323853u,1773110286u,1773896719u,
1774683152u,1775469585u,1776256018u,1777042451u,1777828884u,1778746389u,1779532822u,1780319255u,1781105688u,1781892121u,1782678554u,1783464987u,1784251420u,1785037853u,1785824286u,1786610719u,
1787397152u,1788183585u,1788970018u,1789756451u,1790542884u,1791329317u,1792115750u,1792902183u,1793688616u,1794475049u,1795261482u,1796047915u,1796834348u,1797620781u,1798407214u,1799193647u,
1799980080u,1800766513u,1801552946u,1802339379u,1803125812u,1803912245u,1804698678u,1805485111u,1806271544u,1807057977u,1807844410u,1808630843u,1809417276u,1810203709u,1810990142u,1811776575u,
1812563968u,1813350401u,1814136834u,1814923267u,1815709700u,1816496133u,1817282566u,1818200071u,1818986504u,1819772937u,1820559370u,1821345803u,1822132236u,1822918669u,1823705102u,1824491535u,
1825277968u,1826064401u,1826850834u,1827637267u,1828423700u,1829210133u,1829996566u,1830782999u,1831569432u,1832355865u,1833142298u,1833928731u,1834715164u,1835501597u,1836288030u,1837074463u,
1837860896u,1838647329u,1839433762u,1840220195u,1841006628u,1841793061u,1842579494u,1843365927u,1844152360u,1844938793u,1845725226u,1846511659u,1847298092u,1848084525u,1848870958u,1849657391u,
1850443824u,1851230257u,1852016690u,1852803123u,1853589556u,1854375989u,1855162422u,1855948855u,1856735288u,1857652793u,1858439226u,1859225659u,1860012092u,1860798525u,1861584958u,1862371391u,
1863158784u,1863945217u,1864731650u,1865518083u,1866304516u,1867090949u,1867877382u,1868663815u,1869450248u,1870236681u,1871023114u,1871809547u,1872595980u,1873382413u,1874168846u,1874955279u,
1875741712u,1876528145u,1877314578u,1878101011u,1878887444u,1879673877u,1880460310u,1881246743u,1882033176u,1882819609u,1883606042u,1884392475u,1885178908u,1885965341u,1886751774u,1887538207u,
1888324640u,1889111073u,1889897506u,1890683939u,1891470372u,1892256805u,1893043238u,1893829671u,1894616104u,1895402537u,1896188970u,1897106475u,1897892908u,1898679341u,1899465774u,1900252207u,
1901038640u,1901825073u,1902611506u,1903397939u,1904184372u,1904970805u,1905757238u,1906543671u,1907330104u,1908116537u,1908902970u,1909689403u,1910475836u,1911262269u,1912048702u,1912835135u,
1913622528u,1914408961u,1915195394u,1915981827u,1916768260u,1917554693u,1918341126u,1919127559u,1919913992u,1920700425u,1921486858u,1922273291u,1923059724u,1923846157u,1924632590u,1925419023u,
1926205456u,1926991889u,1927778322u,1928564755u,1929351188u,1930137621u,1930924054u,1931710487u,1932496920u,1933283353u,1934069786u,1934856219u,1935642652u,1936560157u,1937346590u,1938133023u,
1938919456u,1939705889u,1940492322u,1941278755u,1942065188u,1942851621u,1943638054u,1944424487u,1945210920u,1945997353u,1946783786u,1947570219u,1948356652u,1949143085u,1949929518u,1950715951u,
1951502384u,1952288817u,1953075250u,1953861683u,1954648116u,1955434549u,1956220982u,1957007415u,1957793848u,1958580281u,1959366714u,1960153147u,1960939580u,1961726013u,1962512446u,1963298879u,
1964086272u,1964872705u,1965659138u,1966445571u,1967232004u,1968018437u,1968804870u,1969591303u,1970377736u,1971164169u,1971950602u,1972737035u,1973523468u,1974309901u,1975096334u,1976013839u,
1976800272u,1977586705u,1978373138u,1979159571u,1979946004u,1980732437u,1981518870u,1982305303u,1983091736u,1983878169u,1984664602u,1985451035u,1986237468u,1987023901u,1987810334u,1988596767u,
1989383200u,1990169633u,1990956066u,1991742499u,1992528932u,1993315365u,1994101798u,1994888231u,1995674664u,1996461097u,1997247530u,1998033963u,1998820396u,1999606829u,2000393262u,2001179695u,
2001966128u,2002752561u,2003538994u,2004325427u,2005111860u,2005898293u,2006684726u,2007471159u,2008257592u,2009044025u,2009830458u,2010616891u,2011403324u,2012189757u,2012976190u,2013762623u,
2014550016u,2015467521u,2016253954u,2017040387u,2017826820u,2018613253u,2019399686u,2020186119u,2020972552u,2021758985u,2022545418u,2023331851u,2024118284u,2024904717u,2025691150u,2026477583u,
2027264016u,2028050449u,2028836882u,2029623315u,2030409748u,2031196181u,2031982614u,2032769047u,2033555480u,2034341913u,2035128346u,2035914779u,2036701212u,2037487645u,2038274078u,2039060511u,
2039846944u,2040633377u,2041419810u,2042206243u,2042992676u,2043779109u,2044565542u,2045351975u,2046138408u,2046924841u,2047711274u,2048497707u,2049284140u,2050070573u,2050857006u,2051643439u,
2052429872u,2053216305u,2054002738u,2054920243u,2055706676u,2056493109u,2057279542u,2058065975u,2058852408u,2059638841u,2060425274u,2061211707u,2061998140u,2062784573u,2063571006u,2064357439u,
2065144832u,2065931265u,2066717698u,2067504131u,2068290564u,2069076997u,2069863430u,2070649863u,2071436296u,2072222729u,2073009162u,2073795595u,2074582028u,2075368461u,2076154894u,2076941327u,
2077727760u,2078514193u,2079300626u,2080087059u,2080873492u,2081659925u,2082446358u,2083232791u,2084019224u,2084805657u,2085592090u,2086378523u,2087164956u,2087951389u,2088737822u,2089524255u,
2090310688u,2091097121u,2091883554u,2092669987u,2093456420u,2094373925u,2095160358u,2095946791u,2096733224u,2097519657u,2098306090u,2099092523u,2099878956u,2100665389u,2101451822u,2102238255u,
2103024688u,2103811121u,2104597554u,2105383987u,2106170420u,2106956853u,2107743286u,2108529719u,2109316152u,2110102585u,2110889018u,2111675451u,2112461884u,2113248317u,2114034750u,2114821183u,
2115608576u,2116395009u,2117181442u,2117967875u,2118754308u,2119540741u,2120327174u,2121113607u,2121900040u,2122686473u,2123472906u,2124259339u,2125045772u,2125832205u,2126618638u,2127405071u,
2128191504u,2128977937u,2129764370u,2130550803u,2131337236u,2132123669u,2132910102u,2133827607u,2134614040u,2135400473u,2136186906u,2136973339u,2137759772u,2138546205u,2139332638u,2140119071u,
2140905504u,2141691937u,2142478370u,2143264803u,2144051236u,2144837669u,2145624102u,2146410535u,2147196968u,2147983401u,2148769834u,2149556267u,2150342700u,2151129133u,2151915566u,2152701999u,
2153488432u,2154274865u,2155061298u,2155847731u,2156634164u,2157420597u,2158207030u,2158993463u,2159779896u,2160566329u,2161352762u,2162139195u,2162925628u,2163712061u,2164498494u,2165284927u,
2166072320u,2166858753u,2167645186u,2168431619u,2169218052u,2170004485u,2170790918u,2171577351u,2172363784u,2173281289u,2174067722u,2174854155u,2175640588u,2176427021u,2177213454u,2177999887u,
2178786320u,2179572753u,2180359186u,2181145619u,2181932052u,2182718485u,2183504918u,2184291351u,2185077784u,2185864217u,2186650650u,2187437083u,2188223516u,2189009949u,2189796382u,2190582815u,
2191369248u,2192155681u,2192942114u,2193728547u,2194514980u,2195301413u,2196087846u,2196874279u,2197660712u,2198447145u,2199233578u,2200020011u,2200806444u,2201592877u,2202379310u,2203165743u,
2203952176u,2204738609u,2205525042u,2206311475u,2207097908u,2207884341u,2208670774u,2209457207u,2210243640u,2211030073u,2211816506u,2212734011u,2213520444u,2214306877u,2215093310u,2215879743u,
2216667136u,2217453569u,2218240002u,2219026435u,2219812868u,2220599301u,2221385734u,2222172167u,2222958600u,2223745033u,2224531466u,2225317899u,2226104332u,2226890765u,2227677198u,2228463631u,
2229250064u,2230036497u,2230822930u,2231609363u,2232395796u,2233182229u,2233968662u,2234755095u,2235541528u,2236327961u,2237114394u,2237900827u,2238687260u,2239473693u,2240260126u,2241046559u,
2241832992u,2242619425u,2243405858u,2244192291u,2244978724u,2245765157u,2246551590u,2247338023u,2248124456u,2248910889u,2249697322u,2250483755u,2251270188u,2252187693u,2252974126u,2253760559u,
2254546992u,2255333425u,2256119858u,2256906291u,2257692724u,2258479157u,2259265590u,2260052023u,2260838456u,2261624889u,2262411322u,2263197755u,2263984188u,2264770621u,2265557054u,2266343487u,
2267130880u,2267917313u,2268703746u,2269490179u,2270276612u,2271063045u,2271849478u,2272635911u,2273422344u,2274208777u,2274995210u,2275781643u,2276568076u,2277354509u,2278140942u,2278927375u,
2279713808u,2280500241u,2281286674u,2282073107u,2282859540u,2283645973u,2284432406u,2285218839u,2286005272u,2286791705u,2287578138u,2288364571u,2289151004u,2289937437u,2290723870u,2291641375u,
2292427808u,2293214241u,2294000674u,2294787107u,2295573540u,2296359973u,2297146406u,2297932839u,2298719272u,2299505705u,2300292138u,2301078571u,2301865004u,2302651437u,2303437870u,2304224303u,
2305010736u,2305797169u,2306583602u,2307370035u,2308156468u,2308942901u,2309729334u,2310515767u,2311302200u,2312088633u,2312875066u,2313661499u,2314447932u,2315234365u,2316020798u,2316807231u,
2317594624u,2318381057u,2319167490u,2319953923u,2320740356u,2321526789u,2322313222u,2323099655u,2323886088u,2324672521u,2325458954u,2326245387u,2327031820u,2327818253u,2328604686u,2329391119u,
2330177552u,2331095057u,2331881490u,2332667923u,2333454356u,2334240789u,2335027222u,2335813655u,2336600088u,2337386521u,2338172954u,2338959387u,2339745820u,2340532253u,2341318686u,2342105119u,
2342891552u,2343677985u,2344464418u,2345250851u,2346037284u,2346823717u,2347610150u,2348396583u,2349183016u,2349969449u,2350755882u,2351542315u,2352328748u,2353115181u,2353901614u,2354688047u,
2355474480u,2356260913u,2357047346u,2357833779u,2358620212u,2359406645u,2360193078u,2360979511u,2361765944u,2362552377u,2363338810u,2364125243u,2364911676u,2365698109u,2366484542u,2367270975u,
2368058368u,2368844801u,2369631234u,2370548739u,2371335172u,2372121605u,2372908038u,2373694471u,2374480904u,2375267337u,2376053770u,2376840203u,2377626636u,2378413069u,2379199502u,2379985935u,
2380772368u,2381558801u,2382345234u,2383131667u,2383918100u,2384704533u,2385490966u,2386277399u,2387063832u,2387850265u,2388636698u,2389423131u,2390209564u,2390995997u,2391782430u,2392568863u,
2393355296u,2394141729u,2394928162u,2395714595u,2396501028u,2397287461u,2398073894u,2398860327u,2399646760u,2400433193u,2401219626u,2402006059u,2402792492u,2403578925u,2404365358u,2405151791u,
2405938224u,2406724657u,2407511090u,2408297523u,2409083956u,2410001461u,2410787894u,2411574327u,2412360760u,2413147193u,2413933626u,2414720059u,2415506492u,2416292925u,2417079358u,2417865791u,
2418653184u,2419439617u,2420226050u,2421012483u,2421798916u,2422585349u,2423371782u,2424158215u,2424944648u,2425731081u,2426517514u,2427303947u,2428090380u,2428876813u,2429663246u,2430449679u,
2431236112u,2432022545u,2432808978u,2433595411u,2434381844u,2435168277u,2435954710u,2436741143u,2437527576u,2438314009u,2439100442u,2439886875u,2440673308u,2441459741u,2442246174u,2443032607u,
2443819040u,2444605473u,2445391906u,2446178339u,2446964772u,2447751205u,2448537638u,2449455143u,2450241576u,2451028009u,2451814442u,2452600875u,2453387308u,2454173741u,2454960174u,2455746607u,
2456533040u,2457319473u,2458105906u,2458892339u,2459678772u,2460465205u,2461251638u,2462038071u,2462824504u,2463610937u,2464397370u,2465183803u,2465970236u,2466756669u,2467543102u,2468329535u,
2469116928u,2469903361u,2470689794u,2471476227u,2472262660u,2473049093u,2473835526u,2474621959u,2475408392u,2476194825u,2476981258u,2477767691u,2478554124u,2479340557u,2480126990u,2480913423u,
2481699856u,2482486289u,2483272722u,2484059155u,2484845588u,2485632021u,2486418454u,2487204887u,2487991320u,2488908825u,2489695258u,2490481691u,2491268124u,2492054557u,2492840990u,2493627423u,
2494413856u,2495200289u,2495986722u,2496773155u,2497559588u,2498346021u,2499132454u,2499918887u,2500705320u,2501491753u,2502278186u,2503064619u,2503851052u,2504637485u,2505423918u,2506210351u,
2506996784u,2507783217u,2508569650u,2509356083u,2510142516u,2510928949u,2511715382u,2512501815u,2513288248u,2514074681u,2514861114u,2515647547u,2516433980u,2517220413u,2518006846u,2518793279u};
__constant__ const unsigned PACK_TAB[6272] = {
4096,4112,4097,4113,16384,38912,50176,61440,5120,5136,5121,5137,6144,6160,6145,6161,16416,38944,50208,61472,7168,7184,7169,7185,72704,0,16,1,17,18432,16448,38976,
50240,61504,2048,2064,2049,2065,29696,1024,1040,1025,1041,72736,3072,3088,3073,3089,18464,8192,8208,8193,8209,16480,39008,50272,61536,29728,9216,9232,9217,9233,72768,128,
144,129,145,16512,10240,10256,10241,10257,18496,39040,11264,11280,11265,11281,16544,2176,2192,2177,2193,4224,4240,4225,4241,50304,61568,29760,1152,1168,1153,1169,39072,5248,
5264,5249,5265,16576,72800,3200,3216,3201,3217,6272,6288,6273,6289,18528,39104,16608,50336,61600,7296,7312,7297,7313,29792,39136,256,272,257,273,16640,72832,50368,18560,
39168,16672,2304,2320,2305,2321,61632,50400,29824,1280,1296,1281,1297,39200,16704,61664,72864,3328,3344,3329,3345,50432,40960,39232,18592,8320,8336,8321,8337,16736,61696,50464,
29856,9344,9360,9345,9361,39264,384,400,385,401,16768,61728,72896,50496,10368,10384,10369,10385,40992,39296,16800,18624,11392,11408,11393,11409,2432,2448,2433,2449,61760,50528,
4352,4368,4353,4369,1408,1424,1409,1425,29888,39328,16832,61792,41024,5376,5392,5377,5393,3456,3472,3457,3473,50560,72928,39360,16864,6400,6416,6401,6417,18656,61824,50592,
7424,7440,7425,7441,29920,39392,512,528,513,529,16896,41056,61856,50624,72960,39424,16928,2560,2576,2561,2577,61888,18688,50656,1536,1552,1537,1553,41088,39456,16960,61920,
3584,3600,3585,3601,50688,39488,16992,61952,50720,41120,39520,640,656,641,657,17024,61984,50752,39552,17056,2688,2704,2689,2705,62016,41152,50784,72992,1664,1680,1665,1681,
39584,17088,29952,62048,3712,3728,3713,3729,50816,18720,39616,17120,41184,62080,50848,73024,39648,29984,768,784,769,785,17152,62112,50880,18752,41216,39680,17184,2816,2832,2817,
2833,62144,73056,50912,63488,30016,1792,1808,1793,1809,39712,17216,62176,3840,3856,3841,3857,50944,18784,41248,39744,74752,17248,73088,62208,63520,50976,30048,20480,39776,896,912,
897,913,17280,18816,62240,31744,51008,41280,74784,39808,73120,17312,2944,2960,2945,2961,63552,62272,30080,51040,20512,1920,1936,1921,1937,39840,18848,17344,41312,31776,62304,3968,
3984,3969,3985,51072,14336,14352,14337,14353,74816,73152,39872,17376,63584,30112,62336,15360,15376,15361,15377,51104,20544,12288,12304,12289,12305,13312,13328,13313,13329,18880,39904,41344,
31808,62368,51136,74848,73184,54272,30144,63616,62400,20576,51168,18912,41376,31840,62432,73216,74880,65536,54304,30176,63648,20608,43008,18944,41408,31872,73248,74912,65568,30208,63680,20640,
43040,18976,41440,8448,8464,8449,8465,31904,73280,74944,9472,9488,9473,9489,30240,63712,20672,10496,10512,10497,10513,43072,19008,41472,31936,11520,11536,11521,11537,73312,74976,30272,
12416,12432,12417,12433,63744,54336,20704,43104,41504,19040,13440,13456,13441,13457,14464,14480,14465,14481,15488,15504,15489,15505,31968,73344,30304,75008,63776,65600,54368,43136,41536,20736,
19072,32000,73376,4480,4496,4481,4497,30336,75040,5504,5520,5505,5521,63808,65632,54400,43168,41568,19104,20768,6528,6544,6529,6545,8576,8592,8577,8593,32032,73408,30368,7552,
7568,7553,7569,75072,9600,9616,9601,9617,65664,63840,54432,41600,43200,19136,20800,10624,10640,10625,10641,73440,32064,30400,11648,11664,11649,11665,12544,12560,12545,12561,75104,65696,
54464,41632,43232,63872,19168,20832,13568,13584,13569,13585,14592,14608,14593,14609,15616,15632,15617,15633,73472,32096,30432,75136,65728,41664,54496,43264,63904,19200,20864,73504,30464,32128,
4608,4624,4609,4625,75168,65760,41696,54528,43296,19232,63936,5632,5648,5633,5649,20896,73536,6656,6672,6657,6673,30496,8704,8720,8705,8721,32160,75200,65792,41728,54560,7680,
7696,7681,7697,43328,9728,9744,9729,9745,19264,63968,20928,10752,10768,10753,10769,73568,30528,32192,12672,12688,12673,12689,11776,11792,11777,11793,41760,65824,75232,54592,43360,19296,
64000,13696,13712,13697,13713,20960,14720,14736,14721,14737,15744,15760,15745,15761,73600,30560,32224,41792,65856,54624,43392,75264,19328,64032,20992,73632,30592,32256,41824,65888,54656,43424,
4736,4752,4737,4753,75296,19360,64064,5760,5776,5761,5777,21024,73664,30624,6784,6800,6785,6801,8832,8848,8833,8849,32288,41856,65920,54688,43456,19392,75328,7808,7824,7809,
7825,9856,9872,9857,9873,64096,21056,73696,30656,10880,10896,10881,10897,12800,12816,12801,12817,41888,32320,65952,54720,43488,11904,11920,11905,11921,19424,75360,13824,13840,13825,13841,
64128,14848,14864,14849,14865,15872,15888,15873,15889,21088,30688,19456,27648,35840,44032,52224,60416,68608,76800,41920,65984,32352,54752,43520,75392,64160,21120,19488,27680,35872,44064,52256,
60448,68640,76832,41952,66016,54784,43552,32384,75424,4864,4880,4865,4881,64192,19520,27712,35904,44096,52288,60480,68672,76864,21152,5888,5904,5889,5905,66048,54816,43584,32416,8960,
8976,8961,8977,6912,6928,6913,6929,75456,9984,10000,9985,10001,7936,7952,7937,7953,64224,19552,27744,35936,44128,52320,60512,68704,76896,21184,12928,12944,12929,12945,11008,11024,
11009,11025,66080,54848,43616,32448,12032,12048,12033,12049,75488,13952,13968,13953,13969,14976,14992,14977,14993,16000,16016,16001,16017,64256,19584,27776,35968,44160,52352,60544,68736,76928,
21216,66112,54880,43648,32480,75520,64288,19616,27808,36000,44192,52384,60576,68768,76960,21248,66144,54912,43680,32512,75552,4992,5008,4993,5009,19648,27840,36032,44224,52416,60608,68800,
76992,64320,21280,6016,6032,6017,6033,66176,54944,43712,32544,9088,9104,9089,9105,7040,7056,7041,7057,75584,19680,27872,36064,44256,52448,60640,68832,77024,64352,10112,10128,10113,
10129,8064,8080,8065,8081,21312,13056,13072,13057,13073,66208,54976,43744,11136,11152,11137,11153,32576,14080,14096,14081,14097,15104,15120,15105,15121,16128,16144,16129,16145,75616,12160,
12176,12161,12177,19712,27904,36096,44288,52480,60672,68864,77056,64384,21344,66240,55008,43776,32608,75648,19744,27936,36128,44320,52512,60704,68896,77088,64416,66272,21376,55040,43808,32640,
75680,19776,27968,36160,44352,52544,60736,68928,77120,22528,64448,66304,55072,43840,21408,33792,32672,19808,28000,36192,44384,52576,60768,68960,77152,45056,75712,22560,64480,66336,13184,13200,
13185,13201,55104,43872,56320,21440,33824,67584,14208,14224,14209,14225,32704,15232,15248,15233,15249,16256,16272,16257,16273,19840,28032,36224,44416,52608,60800,68992,77184,45088,75744,78848,
22592,66368,55136,43904,56352,21472,24576,33856,67616,32736,19872,28064,36256,44448,52640,60832,69024,77216,45120,78880,66400,55168,22624,43936,56384,24608,33888,67648,19904,28096,36288,44480,
52672,60864,69056,77248,45152,66432,78912,55200,43968,22656,56416,24640,33920,67680,19936,28128,36320,44512,52704,60896,69088,77280,45184,66464,55232,78944,44000,47104,22688,56448,24672,33952,
58368,19968,28160,36352,44544,52736,60928,69120,77312,67712,45216,66496,55264,69632,47136,78976,22720,56480,24704,80896,58400,33984,20000,28192,36384,44576,52768,60960,69152,77344,67744,45248,
66528,26624,69664,47168,79008,22752,56512,37888,80928,24736,58432,34016,20032,28224,36416,44608,52800,60992,69184,77376,67776,26656,45280,69696,47200,79040,22784,56544,37920,80960,58464,24768,
20064,28256,36448,44640,52832,61024,69216,77408,34048,67808,26688,69728,47232,45312,79072,22816,56576,37952,80992,58496,20096,28288,36480,44672,52864,61056,69248,77440,24800,34080,67840,26720,
69760,47264,45344,79104,22848,37984,81024,56608,58528,20128,28320,36512,44704,52896,61088,69280,77472,24832,34112,67872,26752,69792,47296,45376,79136,22880,38016,81056,58560,56640,20160,28352,
36544,44736,52928,61120,69312,77504,24864,34144,67904,26784,69824,47328,45408,79168,22912,38048,81088,58592,20192,28384,36576,44768,52960,61152,69344,77536,56672,24896,34176,26816,69856,67936,
47360,45440,79200,38080,81120,22944,58624,20224,28416,36608,44800,52992,61184,69376,77568,56704,24928,34208,26848,69888,47392,67968,45472,38112,79232,81152,58656,22976,20256,28448,36640,44832,
53024,61216,69408,77600,56736,24960,34240,26880,69920,47424,68000,45504,38144,81184,58688,79264,20288,28480,36672,44864,53056,61248,69440,77632,23008,56768,24992,26912,69952,34272,47456,68032,
45536,38176,81216,58720,79296,20320,28512,36704,44896,53088,61280,69472,77664,23040,56800,26944,25024,69984,47488,34304,68064,38208,45568,81248,58752,20352,28544,36736,44928,53120,61312,69504,
77696,79328,23072,56832,26976,70016,47520,25056,34336,68096,38240,81280,58784,45600,20384,28576,36768,44960,53152,61344,69536,77728,79360,23104,56864,27008,70048,47552,25088,34368,68128,38272,
81312,58816,45632,20416,28608,36800,44992,53184,61376,69568,77760,79392,23136,27040,56896,70080,47584,25120,34400,68160,38304,81344,58848,20448,28640,36832,45024,53216,61408,69600,77792,45664,
79424,23168,27072,70112,47616,56928,25152,34432,68192,38336,81376,58880,53248,64512,75776,21504,45696,79456,23200,27104,70144,47648,56960,25184,34464,38368,81408,68224,58912,53280,64544,75808,
21536,45728,79488,27136,23232,70176,47680,56992,25216,34496,38400,81440,58944,68256,45760,79520,27168,70208,47712,23264,57024,25248,34528,38432,81472,58976,68288,45792,27200,70240,79552,47744,
23296,57056,25280,38464,34560,81504,59008,68320,45824,27232,70272,47776,79584,23328,57088,38496,25312,81536,59040,34592,68352,45856,27264,70304,47808,79616,23360,57120,38528,81568,59072,25344,
34624,68384,27296,70336,45888,47840,79648,23392,57152,38560,81600,59104,25376,34656,68416,27328,70368,47872,45920,79680,23424,57184,38592,81632,59136,25408,34688,68448,27360,70400,47904,45952,
79712,23456,38624,81664,59168,57216,25440,34720,68480,27392,70432,47936,45984,79744,23488,38656,81696,59200,57248,25472,34752,68512,27424,70464,47968,46016,79776,23520,38688,81728,59232,57280,
25504,34784,27456,70496,48000,68544,46048,79808,38720,81760,49152,59264,57312,25536,71680,27488,70528,48032,68576,17408,38752,79840,81792,59296,49184,28672,25568,71712,27520,70560,48064,39936,
17440,38784,81824,59328,51200,49216,28704,62464,27552,70592,71744,48096,39968,17472,38816,81856,59360,51232,49248,28736,27584,62496,70624,73728,71776,40000,38848,17504,81888,30720,51264,49280,
28768,27616,41984,73760,62528,71808,40032,38880,30752,17536,51296,49312,28800,42016,73792,64576,75840,21568,62560,71840,40064,30784,17568,51328,49344,28832,42048,73824,64608,75872,21600,62592,
71872,40096,53312,30816,17600,51360,49376,42080,73856,64640,75904,21632,28864,62624,71904,40128,53344,30848,17632,51392,49408,42112,73888,64672,75936,21664,28896,62656,71936,53376,40160,30880,
17664,51424,49440,42144,73920,64704,75968,21696,28928,62688,71968,53408,30912,40192,17696,51456,42176,73952,64736,76000,21728,49472,28960,62720,72000,53440,30944,40224,17728,42208,51488,73984,
64768,76032,21760,49504,28992,62752,53472,72032,30976,40256,17760,42240,74016,64800,76064,21792,51520,49536,29024,62784,131071,131071,131071,131071,53504,31008,72064,40288,131071,131071,131071,131071,
17792,42272,74048,64832,131071,131071,131071,131071,76096,21824,51552,49568,131071,131071,131071,131071,29056,53536,31040,62816,72096,40320,42304,17824,74080,64864,76128,21856,51584,49600,29088,53568,
31072,62848,72128,40352,131071,131071,131071,131071,42336,74112,64896,76160,131071,131071,131071,131071,21888,17856,51616,49632,131071,131071,131071,131071,29120,53600,31104,62880,131071,131071,131071,131071,
72160,40384,42368,74144,64928,76192,21920,17888,51648,49664,53632,31136,29152,62912,72192,40416,42400,74176,64960,76224,131071,131071,131071,131071,21952,17920,51680,49696,131071,131071,131071,131071,
53664,31168,29184,62944,131071,131071,131071,131071,72224,40448,42432,74208,131071,131071,131071,131071,64992,76256,21984,17952,51712,49728,53696,31200,29216,62976,72256,42464,74240,65024,76288,22016,
40480,17984,51744,53728,131071,131071,131071,131071,31232,49760,29248,63008,131071,131071,131071,131071,72288,42496,74272,65056,131071,131071,131071,131071,76320,22048,40512,18016,131071,131071,131071,131071,
53760,51776,31264,49792,131071,131071,131071,131071,29280,63040,72320,42528,131071,131071,131071,131071,74304,65088,76352,22080,131071,131071,131071,131071,40544,18048,53792,31296,131071,131071,131071,131071,
51808,49824,29312,63072,42560,74336,65120,76384,22112,72352,40576,18080,53824,31328,51840,49856,29344,42592,63104,74368,131071,131071,131071,131071,65152,76416,22144,72384,131071,131071,131071,131071,
40608,53856,18112,31360,131071,131071,131071,131071,51872,49888,29376,42624,131071,131071,131071,131071,74400,65184,76448,22176,63136,72416,40640,53888,31392,18144,51904,49920,29408,42656,74432,65216,
76480,22208,63168,72448,131071,131071,131071,131071,40672,53920,31424,18176,131071,131071,131071,131071,51936,49952,42688,29440,131071,131071,131071,131071,74464,65248,76512,22240,131071,131071,131071,131071,
63200,72480,40704,53952,31456,18208,51968,49984,42720,74496,65280,76544,22272,29472,63232,72512,40736,53984,31488,18240,131071,131071,131071,131071,52000,50016,42752,74528,131071,131071,131071,131071,
65312,76576,22304,29504,131071,131071,131071,131071,63264,72544,54016,31520,131071,131071,131071,131071,40768,18272,52032,42784,50048,74560,65344,76608,22336,29536,63296,72576,54048,31552,40800,18304,
52064,42816,74592,65376,131071,131071,131071,131071,76640,22368,50080,29568,131071,131071,131071,131071,63328,72608,54080,31584,131071,131071,131071,131071,40832,18336,42848,74624,131071,131071,131071,131071,
65408,76672,22400,52096,131071,131071,131071,131071,50112,29600,63360,54112,131071,131071,131071,131071,31616,72640,40864,18368,131071,131071,131071,131071,42880,74656,65440,76704,131071,131071,131071,131071,
22432,52128,50144,29632,54144,63392,31648,72672,40896,18400,42912,74688,65472,76736,22464,52160,32768,29664,54176,31680,131071,131071,131071,131071,63424,55296,40928,42944,131071,131071,131071,131071,
74720,65504,76768,22496,131071,131071,131071,131071,66560,52192,32800,77824,131071,131071,131071,131071,54208,31712,63456,55328,23552,42976,57344,48128,59392,70656,66592,34816,32832,77856,54240,79872,
46080,55360,23584,25600,131071,131071,131071,131071,57376,48160,59424,70688,131071,131071,131071,131071,66624,34848,32864,36864,131071,131071,131071,131071,79904,77888,46112,55392,131071,131071,131071,131071,
23616,25632,57408,48192,59456,70720,66656,34880,32896,36896,79936,77920,46144,55424,23648,25664,57440,48224,59488,70752,131071,131071,131071,131071,66688,34912,32928,36928,131071,131071,131071,131071,
79968,77952,46176,55456,131071,131071,131071,131071,25696,57472,48256,59520,131071,131071,131071,131071,70784,23680,66720,34944,36960,80000,32960,77984,46208,55488,25728,57504,48288,59552,70816,23712,
66752,36992,34976,80032,131071,131071,131071,131071,32992,78016,46240,55520,131071,131071,131071,131071,25760,57536,48320,59584,131071,131071,131071,131071,70848,23744,66784,37024,131071,131071,131071,131071,
80064,35008,33024,78048,131071,131071,131071,131071,46272,25792,57568,48352,131071,131071,131071,131071,59616,70880,55552,23776,131071,131071,131071,131071,66816,37056,80096,35040,131071,131071,131071,131071,
33056,78080,25824,46304,57600,48384,59648,70912,55584,23808,37088,66848,80128,35072,33088,78112,25856,57632,48416,59680,131071,131071,131071,131071,70944,46336,55616,23840,131071,131071,131071,131071,
37120,80160,66880,35104,131071,131071,131071,131071,33120,78144,25888,57664,131071,131071,131071,131071,48448,59712,70976,46368,55648,23872,37152,80192,66912,35136,33152,25920,78176,57696,48480,59744,
71008,46400,55680,23904,131071,131071,131071,131071,37184,80224,66944,35168,131071,131071,131071,131071,33184,25952,57728,48512,131071,131071,131071,131071,59776,71040,78208,46432,131071,131071,131071,131071,
55712,23936,37216,80256,66976,35200,33216,25984,57760,48544,59808,71072,78240,46464,55744,37248,80288,23968,67008,35232,131071,131071,131071,131071,26016,33248,57792,48576,131071,131071,131071,131071,
59840,71104,78272,46496,131071,131071,131071,131071,55776,37280,80320,24000,131071,131071,131071,131071,67040,35264,26048,57824,48608,59872,71136,33280,78304,46528,55808,37312,80352,24032,67072,26080,
57856,48640,59904,71168,131071,131071,131071,131071,35296,33312,78336,46560,131071,131071,131071,131071,37344,80384,55840,24064,131071,131071,131071,131071,67104,26112,57888,48672,131071,131071,131071,131071,
59936,71200,35328,33344,131071,131071,131071,131071,78368,37376,46592,80416,131071,131071,131071,131071,55872,24096,67136,26144,131071,131071,131071,131071,57920,48704,59968,71232,131071,131071,131071,131071,
131071,131071,35360,33376,78400,37408,80448,46624,131071,131071,55904,24128,26176,57952,48736,60000,131071,131071,71264,67168,131071,131071,131071,131071,131071,131071,35392,33408,131071,131071,131071,131071,
131071,131071,78432,37440,131071,131071,131071,131071,131071,131071,80480,46656,131071,131071,131071,131071,131071,131071,55936,24160,26208,57984,48768,60032,131071,131071,71296,67200,35424,33440,37472,78464,
131071,131071,80512,46688,131071,131071,131071,131071,131071,131071,55968,24192,131071,131071,131071,131071,131071,131071,26240,58016,131071,131071,131071,131071,131071,131071,48800,60064,131071,131071,131071,131071,
131071,131071,71328,67232,35456,33472,37504,80544,131071,131071,78496,46720,56000,24224,26272,58048,131071,131071,48832,60096,131071,131071,131071,131071,131071,131071,71360,67264,131071,131071,131071,131071,
131071,131071,35488,33504,131071,131071,131071,131071,131071,131071,37536,80576,131071,131071,131071,131071,131071,131071,78528,46752,56032,26304,24256,58080,131071,131071,48864,60128,71392,67296,35520,37568,
131071,131071,33536,80608,131071,131071,131071,131071,131071,131071,78560,46784,131071,131071,131071,131071,131071,131071,56064,26336,131071,131071,131071,131071,131071,131071,58112,48896,131071,131071,131071,131071,
131071,131071,60160,71424,131071,131071,131071,131071,131071,131071,24288,67328,131071,131071,131071,131071,131071,131071,35552,37600,131071,131071,131071,131071,131071,131071,80640,33568,131071,131071,131071,131071,
131071,131071,78592,46816,56096,26368,58144,48928,131071,131071,60192,71456,24320,67360,37632,80672,131071,131071,35584,33600,131071,131071,131071,131071,131071,131071,78624,46848,131071,131071,131071,131071,
131071,131071,26400,56128,131071,131071,131071,131071,131071,131071,58176,48960,131071,131071,131071,131071,131071,131071,60224,71488,24352,67392,37664,80704,131071,131071,35616,33632,78656,46880,26432,58208,
131071,131071,48992,60256,131071,131071,131071,131071,131071,131071,71520,56160,131071,131071,131071,131071,131071,131071,24384,67424,131071,131071,131071,131071,131071,131071,37696,80736,131071,131071,131071,131071,
131071,131071,35648,33664,78688,26464,58240,49024,131071,131071,60288,71552,46912,56192,24416,37728,131071,131071,80768,67456,131071,131071,131071,131071,131071,131071,35680,33696,131071,131071,131071,131071,
131071,131071,78720,26496,131071,131071,131071,131071,131071,131071,58272,49056,131071,131071,131071,131071,131071,131071,60320,71584,46944,56224,24448,37760,131071,131071,80800,67488,35712,33728,78752,26528,
131071,131071,58304,49088,131071,131071,131071,131071,131071,131071,60352,71616,131071,131071,131071,131071,131071,131071,46976,56256,131071,131071,131071,131071,131071,131071,24480,37792,131071,131071,131071,131071,
131071,131071,80832,67520,131071,131071,131071,131071,131071,131071,35744,33760,131071,131071,131071,131071,131071,131071,26560,58336,131071,131071,131071,131071,131071,131071,49120,60384,131071,131071,131071,131071,
131071,131071,71648,78784,47008,56288,24512,37824,131071,131071,80864,67552,35776,26592,78816,47040,131071,131071,37856,24544,131071,131071,131071,131071,131071,131071,35808,47072,131071,131071,131071,131071,
131071,131071,131071,131071,131071,131071,131071,131071,131071,131071,131071,131071,131071,131071,131071,131071,131071,131071,131071,131071,131071,131071,131071,131071,131071,131071,131071,131071,131071,131071,131071,131071,
131071,131071,131071,131071,131071,131071,131071,131071,131071,131071,131071,131071,131071,131071,131071,131071,131071,131071,131071,131071,131071,131071,131071,131071,131071,131071,131071,131071,131071,131071,131071,131071,
131071,131071,131071,131071,131071,131071,131071,131071,131071,131071,131071,131071,131071,131071,131071,131071,131071,131071,131071,131071,131071,131071,131071,131071,131071,131071,131071,131071,131071,131071,131071,131071,
131071,131071,131071,131071,131071,131071,131071,131071,131071,131071,131071,131071,131071,131071,131071,131071,131071,131071,131071,131071,131071,131071,131071,131071,131071,131071,131071,131071,131071,131071,131071,131071,
131071,131071,131071,131071,131071,131071,131071,131071,131071,131071,131071,131071,131071,131071,131071,131071,131071,131071,131071,131071,131071,131071,131071,131071,131071,131071,131071,131071,131071,131071,131071,131071,
131071,131071,131071,131071,131071,131071,131071,131071,131071,131071,131071,131071,131071,131071,131071,131071,131071,131071,131071,131071,131071,131071,131071,131071,131071,131071,131071,131071,131071,131071,131071,131071,
131071,131071,131071,131071,131071,131071,131071,131071,131071,131071,131071,131071,131071,131071,131071,131071,131071,131071,131071,131071,131071,131071,131071,131071,131071,131071,131071,131071,131071,131071,131071,131071,
131071,131071,131071,131071,131071,131071,131071,131071,131071,131071,131071,131071,131071,131071,131071,131071,131071,131071,131071,131071,131071,131071,131071,131071,131071,131071,131071,131071,131071,131071,131071,131071,
131071,131071,131071,131071,131071,131071,131071,131071,131071,131071,131071,131071,131071,131071,131071,131071,131071,131071,131071,131071,131071,131071,131071,131071,131071,131071,131071,131071,131071,131071,131071,131071,
131071,131071,131071,131071,131071,131071,131071,131071,131071,131071,131071,131071,131071,131071,131071,131071,131071,131071,131071,131071,131071,131071,131071,131071,131071,131071,131071,131071,131071,131071,131071,131071,
131071,131071,131071,131071,131071,131071,131071,131071,131071,131071,131071,131071,131071,131071,131071,131071,131071,131071,131071,131071,131071,131071,131071,131071,131071,131071,131071,131071,131071,131071,131071,131071,
131071,131071,131071,131071,131071,131071,131071,131071,131071,131071,131071,131071,131071,131071,131071,131071,131071,131071,131071,131071,131071,131071,131071,131071,131071,131071,131071,131071,131071,131071,131071,131071,
131071,131071,131071,131071,131071,131071,131071,131071,131071,131071,131071,131071,131071,131071,131071,131071,131071,131071,131071,131071,131071,131071,131071,131071,131071,131071,131071,131071,131071,131071,131071,131071,
131071,131071,131071,131071,131071,131071,131071,131071,131071,131071,131071,131071,131071,131071,131071,131071,131071,131071,131071,131071,131071,131071,131071,131071,131071,131071,131071,131071,131071,131071,131071,131071,
131071,131071,131071,131071,131071,131071,131071,131071,131071,131071,131071,131071,131071,131071,131071,131071,131071,131071,131071,131071,131071,131071,131071,131071,131071,131071,131071,131071,131071,131071,131071,131071,
131071,131071,131071,131071,131071,131071,131071,131071,131071,131071,131071,131071,131071,131071,131071,131071,131071,131071,131071,131071,131071,131071,131071,131071,131071,131071,131071,131071,131071,131071,131071,131071,
131071,131071,131071,131071,131071,131071,131071,131071,131071,131071,131071,131071,131071,131071,131071,131071,131071,131071,131071,131071,131071,131071,131071,131071,131071,131071,131071,131071,131071,131071,131071,131071,
131071,131071,131071,131071,131071,131071,131071,131071,131071,131071,131071,131071,131071,131071,131071,131071,131071,131071,131071,131071,131071,131071,131071,131071,131071,131071,131071,131071,131071,131071,131071,131071,
131071,131071,131071,131071,131071,131071,131071,131071,131071,131071,131071,131071,131071,131071,131071,131071,131071,131071,131071,131071,131071,131071,131071,131071,131071,131071,131071,131071,131071,131071,131071,131071,
131071,131071,131071,131071,131071,131071,131071,131071,131071,131071,131071,131071,131071,131071,131071,131071,131071,131071,131071,131071,131071,131071,131071,131071,131071,131071,131071,131071,131071,131071,131071,131071,
131071,131071,131071,131071,131071,131071,131071,131071,131071,131071,131071,131071,131071,131071,131071,131071,131071,131071,131071,131071,131071,131071,131071,131071,131071,131071,131071,131071,131071,131071,131071,131071,
131071,131071,131071,131071,131071,131071,131071,131071,131071,131071,131071,131071,131071,131071,131071,131071,131071,131071,131071,131071,131071,131071,131071,131071,131071,131071,131071,131071,131071,131071,131071,131071,
131071,131071,131071,131071,131071,131071,131071,131071,131071,131071,131071,131071,131071,131071,131071,131071,131071,131071,131071,131071,131071,131071,131071,131071,131071,131071,131071,131071,131071,131071,131071,131071,
131071,131071,131071,131071,131071,131071,131071,131071,131071,131071,131071,131071,131071,131071,131071,131071,131071,131071,131071,131071,131071,131071,131071,131071,131071,131071,131071,131071,131071,131071,131071,131071,
131071,131071,131071,131071,131071,131071,131071,131071,131071,131071,131071,131071,131071,131071,131071,131071,131071,131071,131071,131071,131071,131071,131071,131071,131071,131071,131071,131071,131071,131071,131071,131071,
131071,131071,131071,131071,131071,131071,131071,131071,131071,131071,131071,131071,131071,131071,131071,131071,131071,131071,131071,131071,131071,131071,131071,131071,131071,131071,131071,131071,131071,131071,131071,131071,
131071,131071,131071,131071,131071,131071,131071,131071,131071,131071,131071,131071,131071,131071,131071,131071,131071,131071,131071,131071,131071,131071,131071,131071,131071,131071,131071,131071,131071,131071,131071,131071,
131071,131071,131071,131071,131071,131071,131071,131071,131071,131071,131071,131071,131071,131071,131071,131071,131071,131071,131071,131071,131071,131071,131071,131071,131071,131071,131071,131071,131071,131071,131071,131071,
131071,131071,131071,131071,131071,131071,131071,131071,131071,131071,131071,131071,131071,131071,131071,131071,131071,131071,131071,131071,131071,131071,131071,131071,131071,131071,131071,131071,131071,131071,131071,131071,
131071,131071,131071,131071,131071,131071,131071,131071,131071,131071,131071,131071,131071,131071,131071,131071,131071,131071,131071,131071,131071,131071,131071,131071,131071,131071,131071,131071,131071,131071,131071,131071,
131071,131071,131071,131071,131071,131071,131071,131071,131071,131071,131071,131071,131071,131071,131071,131071,131071,131071,131071,131071,131071,131071,131071,131071,131071,131071,131071,131071,131071,131071,131071,131071,
131071,131071,131071,131071,131071,131071,131071,131071,131071,131071,131071,131071,131071,131071,131071,131071,131071,131071,131071,131071,131071,131071,131071,131071,131071,131071,131071,131071,131071,131071,131071,131071,
131071,131071,131071,131071,131071,131071,131071,131071,131071,131071,131071,131071,131071,131071,131071,131071,131071,131071,131071,131071,131071,131071,131071,131071,131071,131071,131071,131071,131071,131071,131071,131071,
131071,131071,131071,131071,131071,131071,131071,131071,131071,131071,131071,131071,131071,131071,131071,131071,131071,131071,131071,131071,131071,131071,131071,131071,131071,131071,131071,131071,131071,131071,131071,131071,
131071,131071,131071,131071,131071,131071,131071,131071,131071,131071,131071,131071,131071,131071,131071,131071,131071,131071,131071,131071,131071,131071,131071,131071,131071,131071,131071,131071,131071,131071,131071,131071,
131071,131071,131071,131071,131071,131071,131071,131071,131071,131071,131071,131071,131071,131071,131071,131071,131071,131071,131071,131071,131071,131071,131071,131071,131071,131071,131071,131071,131071,131071,131071,131071,
131071,131071,131071,131071,131071,131071,131071,131071,131071,131071,131071,131071,131071,131071,131071,131071,131071,131071,131071,131071,131071,131071,131071,131071,131071,131071,131071,131071,131071,131071,131071,131071,
131071,131071,131071,131071,131071,131071,131071,131071,131071,131071,131071,131071,131071,131071,131071,131071,131071,131071,131071,131071,131071,131071,131071,131071,131071,131071,131071,131071,131071,131071,131071,131071,
131071,131071,131071,131071,131071,131071,131071,131071,131071,131071,131071,131071,131071,131071,131071,131071,131071,131071,131071,131071,131071,131071,131071,131071,131071,131071,131071,131071,131071,131071,131071,131071,
131071,131071,131071,131071,131071,131071,131071,131071,131071,131071,131071,131071,131071,131071,131071,131071,131071,131071,131071,131071,131071,131071,131071,131071,131071,131071,131071,131071,131071,131071,131071,131071,
131071,131071,131071,131071,131071,131071,131071,131071,131071,131071,131071,131071,131071,131071,131071,131071,131071,131071,131071,131071,131071,131071,131071,131071,131071,131071,131071,131071,131071,131071,131071,131071,
131071,131071,131071,131071,131071,131071,131071,131071,131071,131071,131071,131071,131071,131071,131071,131071,131071,131071,131071,131071,131071,131071,131071,131071,131071,131071,131071,131071,131071,131071,131071,131071,
131071,131071,131071,131071,131071,131071,131071,131071,131071,131071,131071,131071,131071,131071,131071,131071,131071,131071,131071,131071,131071,131071,131071,131071,131071,131071,131071,131071,131071,131071,131071,131071,
131071,131071,131071,131071,131071,131071,131071,131071,131071,131071,131071,131071,131071,131071,131071,131071,131071,131071,131071,131071,131071,131071,131071,131071,131071,131071,131071,131071,131071,131071,131071,131071,
131071,131071,131071,131071,131071,131071,131071,131071,131071,131071,131071,131071,131071,131071,131071,131071,131071,131071,131071,131071,131071,131071,131071,131071,131071,131071,131071,131071,131071,131071,131071,131071,
131071,131071,131071,131071,131071,131071,131071,131071,131071,131071,131071,131071,131071,131071,131071,131071,131071,131071,131071,131071,131071,131071,131071,131071,131071,131071,131071,131071,131071,131071,131071,131071,
131071,131071,131071,131071,131071,131071,131071,131071,131071,131071,131071,131071,131071,131071,131071,131071,131071,131071,131071,131071,131071,131071,131071,131071,131071,131071,131071,131071,131071,131071,131071,131071,
131071,131071,131071,131071,131071,131071,131071,131071,131071,131071,131071,131071,131071,131071,131071,131071,131071,131071,131071,131071,131071,131071,131071,131071,131071,131071,131071,131071,131071,131071,131071,131071,
131071,131071,131071,131071,131071,131071,131071,131071,131071,131071,131071,131071,131071,131071,131071,131071,131071,131071,131071,131071,131071,131071,131071,131071,131071,131071,131071,131071,131071,131071,131071,131071,
131071,131071,131071,131071,131071,131071,131071,131071,131071,131071,131071,131071,131071,131071,131071,131071,131071,131071,131071,131071,131071,131071,131071,131071,131071,131071,131071,131071,131071,131071,131071,131071,
131071,131071,131071,131071,131071,131071,131071,131071,131071,131071,131071,131071,131071,131071,131071,131071,131071,131071,131071,131071,131071,131071,131071,131071,131071,131071,131071,131071,131071,131071,131071,131071,
131071,131071,131071,131071,131071,131071,131071,131071,131071,131071,131071,131071,131071,131071,131071,131071,131071,131071,131071,131071,131071,131071,131071,131071,131071,131071,131071,131071,131071,131071,131071,131071,
131071,131071,131071,131071,131071,131071,131071,131071,131071,131071,131071,131071,131071,131071,131071,131071,131071,131071,131071,131071,131071,131071,131071,131071,131071,131071,131071,131071,131071,131071,131071,131071,
131071,131071,131071,131071,131071,131071,131071,131071,131071,131071,131071,131071,131071,131071,131071,131071,131071,131071,131071,131071,131071,131071,131071,131071,131071,131071,131071,131071,131071,131071,131071,131071,
131071,131071,131071,131071,131071,131071,131071,131071,131071,131071,131071,131071,131071,131071,131071,131071,131071,131071,131071,131071,131071,131071,131071,131071,131071,131071,131071,131071,131071,131071,131071,131071,
131071,131071,131071,131071,131071,131071,131071,131071,131071,131071,131071,131071,131071,131071,131071,131071,131071,131071,131071,131071,131071,131071,131071,131071,131071,131071,131071,131071,131071,131071,131071,131071,
131071,131071,131071,131071,131071,131071,131071,131071,131071,131071,131071,131071,131071,131071,131071,131071,131071,131071,131071,131071,131071,131071,131071,131071,131071,131071,131071,131071,131071,131071,131071,131071,
131071,131071,131071,131071,131071,131071,131071,131071,131071,131071,131071,131071,131071,131071,131071,131071,131071,131071,131071,131071,131071,131071,131071,131071,131071,131071,131071,131071,131071,131071,131071,131071,
131071,131071,131071,131071,131071,131071,131071,131071,131071,131071,131071,131071,131071,131071,131071,131071,131071,131071,131071,131071,131071,131071,131071,131071,131071,131071,131071,131071,131071,131071,131071,131071,
131071,131071,131071,131071,131071,131071,131071,131071,131071,131071,131071,131071,131071,131071,131071,131071,131071,131071,131071,131071,131071,131071,131071,131071,131071,131071,131071,131071,131071,131071,131071,131071,
131071,131071,131071,131071,131071,131071,131071,131071,131071,131071,131071,131071,131071,131071,131071,131071,131071,131071,131071,131071,131071,131071,131071,131071,131071,131071,131071,131071,131071,131071,131071,131071,
131071,131071,131071,131071,131071,131071,131071,131071,131071,131071,131071,131071,131071,131071,131071,131071,131071,131071,131071,131071,131071,131071,131071,131071,131071,131071,131071,131071,131071,131071,131071,131071,
131071,131071,131071,131071,131071,131071,131071,131071,131071,131071,131071,131071,131071,131071,131071,131071,131071,131071,131071,131071,131071,131071,131071,131071,131071,131071,131071,131071,131071,131071,131071,131071,
131071,131071,131071,131071,131071,131071,131071,131071,131071,131071,131071,131071,131071,131071,131071,131071,131071,131071,131071,131071,131071,131071,131071,131071,131071,131071,131071,131071,131071,131071,131071,131071,
131071,131071,131071,131071,131071,131071,131071,131071,131071,131071,131071,131071,131071,131071,131071,131071,131071,131071,131071,131071,131071,131071,131071,131071,131071,131071,131071,131071,131071,131071,131071,131071,
131071,131071,131071,131071,131071,131071,131071,131071,131071,131071,131071,131071,131071,131071,131071,131071,131071,131071,131071,131071,131071,131071,131071,131071,131071,131071,131071,131071,131071,131071,131071,131071,
131071,131071,131071,131071,131071,131071,131071,131071,131071,131071,131071,131071,131071,131071,131071,131071,131071,131071,131071,131071,131071,131071,131071,131071,131071,131071,131071,131071,131071,131071,131071,131071,
131071,131071,131071,131071,131071,131071,131071,131071,131071,131071,131071,131071,131071,131071,131071,131071,131071,131071,131071,131071,131071,131071,131071,131071,131071,131071,131071,131071,131071,131071,131071,131071,
131071,131071,131071,131071,131071,131071,131071,131071,131071,131071,131071,131071,131071,131071,131071,131071,131071,131071,131071,131071,131071,131071,131071,131071,131071,131071,131071,131071,131071,131071,131071,131071,
131071,131071,131071,131071,131071,131071,131071,131071,131071,131071,131071,131071,131071,131071,131071,131071,131071,131071,131071,131071,131071,131071,131071,131071,131071,131071,131071,131071,131071,131071,131071,131071,
131071,131071,131071,131071,131071,131071,131071,131071,131071,131071,131071,131071,131071,131071,131071,131071,131071,131071,131071,131071,131071,131071,131071,131071,131071,131071,131071,131071,131071,131071,131071,131071,
131071,131071,131071,131071,131071,131071,131071,131071,131071,131071,131071,131071,131071,131071,131071,131071,131071,131071,131071,131071,131071,131071,131071,131071,131071,131071,131071,131071,131071,131071,131071,131071,
131071,131071,131071,131071,131071,131071,131071,131071,131071,131071,131071,131071,131071,131071,131071,131071,131071,131071,131071,131071,131071,131071,131071,131071,131071,131071,131071,131071,131071,131071,131071,131071,
131071,131071,131071,131071,131071,131071,131071,131071,131071,131071,131071,131071,131071,131071,131071,131071,131071,131071,131071,131071,131071,131071,131071,131071,131071,131071,131071,131071,131071,131071,131071,131071,
131071,131071,131071,131071,131071,131071,131071,131071,131071,131071,131071,131071,131071,131071,131071,131071,131071,131071,131071,131071,131071,131071,131071,131071,131071,131071,131071,131071,131071,131071,131071,131071,
131071,131071,131071,131071,131071,131071,131071,131071,131071,131071,131071,131071,131071,131071,131071,131071,131071,131071,131071,131071,131071,131071,131071,131071,131071,131071,131071,131071,131071,131071,131071,131071,
131071,131071,131071,131071,131071,131071,131071,131071,131071,131071,131071,131071,131071,131071,131071,131071,131071,131071,131071,131071,131071,131071,131071,131071,131071,131071,131071,131071,131071,131071,131071,131071,
131071,131071,131071,131071,131071,131071,131071,131071,131071,131071,131071,131071,131071,131071,131071,131071,131071,131071,131071,131071,131071,131071,131071,131071,131071,131071,131071,131071,131071,131071,131071,131071,
131071,131071,131071,131071,131071,131071,131071,131071,131071,131071,131071,131071,131071,131071,131071,131071,131071,131071,131071,131071,131071,131071,131071,131071,131071,131071,131071,131071,131071,131071,131071,131071,
131071,131071,131071,131071,131071,131071,131071,131071,131071,131071,131071,131071,131071,131071,131071,131071,131071,131071,131071,131071,131071,131071,131071,131071,131071,131071,131071,131071,131071,131071,131071,131071,
131071,131071,131071,131071,131071,131071,131071,131071,131071,131071,131071,131071,131071,131071,131071,131071,131071,131071,131071,131071,131071,131071,131071,131071,131071,131071,131071,131071,131071,131071,131071,131071,
131071,131071,131071,131071,131071,131071,131071,131071,131071,131071,131071,131071,131071,131071,131071,131071,131071,131071,131071,131071,131071,131071,131071,131071,131071,131071,131071,131071,131071,131071,131071,131071,
131071,131071,131071,131071,131071,131071,131071,131071,131071,131071,131071,131071,131071,131071,131071,131071,131071,131071,131071,131071,131071,131071,131071,131071,131071,131071,131071,131071,131071,131071,131071,131071,
131071,131071,131071,131071,131071,131071,131071,131071,131071,131071,131071,131071,131071,131071,131071,131071,131071,131071,131071,131071,131071,131071,131071,131071,131071,131071,131071,131071,131071,131071,131071,131071,
131071,131071,131071,131071,131071,131071,131071,131071,131071,131071,131071,131071,131071,131071,131071,131071,131071,131071,131071,131071,131071,131071,131071,131071,131071,131071,131071,131071,131071,131071,131071,131071,
131071,131071,131071,131071,131071,131071,131071,131071,131071,131071,131071,131071,131071,131071,131071,131071,131071,131071,131071,131071,131071,131071,131071,131071,131071,131071,131071,131071,131071,131071,131071,131071,
131071,131071,131071,131071,131071,131071,131071,131071,131071,131071,131071,131071,131071,131071,131071,131071,131071,131071,131071,131071,131071,131071,131071,131071,131071,131071,131071,131071,131071,131071,131071,131071,
131071,131071,131071,131071,131071,131071,131071,131071,131071,131071,131071,131071,131071,131071,131071,131071,131071,131071,131071,131071,131071,131071,131071,131071,131071,131071,131071,131071,131071,131071,131071,131071,
131071,131071,131071,131071,131071,131071,131071,131071,131071,131071,131071,131071,131071,131071,131071,131071,131071,131071,131071,131071,131071,131071,131071,131071,131071,131071,131071,131071,131071,131071,131071,131071,
131071,131071,131071,131071,131071,131071,131071,131071,131071,131071,131071,131071,131071,131071,131071,131071,131071,131071,131071,131071,131071,131071,131071,131071,131071,131071,131071,131071,131071,131071,131071,131071,
131071,131071,131071,131071,131071,131071,131071,131071,131071,131071,131071,131071,131071,131071,131071,131071,131071,131071,131071,131071,131071,131071,131071,131071,131071,131071,131071,131071,131071,131071,131071,131071,
131071,131071,131071,131071,131071,131071,131071,131071,131071,131071,131071,131071,131071,131071,131071,131071,131071,131071,131071,131071,131071,131071,131071,131071,131071,131071,131071,131071,131071,131071,131071,131071,
131071,131071,131071,131071,131071,131071,131071,131071,131071,131071,131071,131071,131071,131071,131071,131071,131071,131071,131071,131071,131071,131071,131071,131071,131071,131071,131071,131071,131071,131071,131071,131071,
131071,131071,131071,131071,131071,131071,131071,131071,131071,131071,131071,131071,131071,131071,131071,131071,131071,131071,131071,131071,131071,131071,131071,131071,131071,131071,131071,131071,131071,131071,131071,131071,
131071,131071,131071,131071,131071,131071,131071,131071,131071,131071,131071,131071,131071,131071,131071,131071,131071,131071,131071,131071,131071,131071,131071,131071,131071,131071,131071,131071,131071,131071,131071,131071,
131071,131071,131071,131071,131071,131071,131071,131071,131071,131071,131071,131071,131071,131071,131071,131071,131071,131071,131071,131071,131071,131071,131071,131071,131071,131071,131071,131071,131071,131071,131071,131071,
131071,131071,131071,131071,131071,131071,131071,131071,131071,131071,131071,131071,131071,131071,131071,131071,131071,131071,131071,131071,131071,131071,131071,131071,131071,131071,131071,131071,131071,131071,131071,131071};
__constant__ const unsigned NEED_TAB[NPLANS * 2560 + 160] = {
29,43,38,48,29,43,38,48,29,43,38,48,29,43,38,48,67,94,83,105,67,94,83,105,67,94,83,105,67,94,83,105,
124,141,134,149,124,141,134,149,124,141,134,149,124,141,134,149,170,200,190,213,170,200,190,213,170,200,190,213,170,200,190,213,
234,252,245,260,234,252,245,260,234,252,245,260,234,252,245,260,271,288,280,296,271,288,280,296,271,288,280,296,271,288,280,296,
310,330,321,337,310,330,321,337,310,330,321,337,310,330,321,337,354,377,368,387,354,377,368,387,354,377,368,387,354,377,368,387,
4,12,16,24,4,12,16,24,4,12,16,24,4,12,16,24,87,99,109,118,87,99,109,118,87,99,109,118,87,99,109,118,
196,209,221,228,196,209,221,228,196,209,221,228,196,209,221,228,519,525,536,547,519,525,536,547,519,525,536,547,519,525,536,547,
612,623,629,643,612,623,629,643,612,623,629,643,612,623,629,643,708,715,722,737,708,715,722,737,708,715,722,737,708,715,722,737,
799,806,818,827,799,806,818,827,799,806,818,827,799,806,818,827,879,886,898,909,879,886,898,909,879,886,898,909,879,886,898,909,
53,62,72,78,53,62,72,78,53,62,72,78,53,62,72,78,157,165,178,186,157,165,178,186,157,165,178,186,157,165,178,186,
455,462,469,477,455,462,469,477,455,462,469,477,455,462,469,477,540,552,563,570,540,552,563,570,540,552,563,570,540,552,563,570,
634,648,655,666,634,648,655,666,634,648,655,666,634,648,655,666,726,741,749,762,726,741,749,762,726,741,749,762,726,741,749,762,
814,823,838,846,814,823,838,846,814,823,838,846,814,823,838,846,894,905,921,939,894,905,921,939,894,905,921,939,894,905,921,939,
409,413,392,403,409,413,392,403,409,413,392,403,409,413,392,403,484,494,498,502,484,494,498,502,484,494,498,502,484,494,498,502,
574,586,590,594,574,586,590,594,574,586,590,594,574,586,590,594,662,677,682,686,662,677,682,686,662,677,682,686,662,677,682,686,
753,768,773,777,753,768,773,777,753,768,773,777,753,768,773,777,834,851,855,859,834,851,855,859,834,851,855,859,834,851,855,859,
914,926,930,934,914,926,930,934,914,926,930,934,914,926,930,934,974,984,989,993,974,984,989,993,974,984,989,993,974,984,989,993,
5,1428,30,780,17,1441,49,789,31,1454,73,801,54,1468,110,829,68,1482,128,861,79,1495,153,869,100,1509,182,880,112,1523,222,900,
125,1537,247,940,130,1551,298,948,143,1565,314,956,158,1579,339,965,171,1593,356,994,181,1608,379,1008,203,1626,414,1019,217,1640,428,1030,
235,1654,440,1042,241,1668,450,1056,255,1682,471,1071,263,1696,490,1084,272,1710,513,1097,276,1724,531,1111,290,1739,558,1125,300,1754,581,1138,
311,1768,604,1152,317,1782,618,1166,332,1796,649,1179,343,1810,672,1193,355,1824,695,1206,364,1838,710,1220,380,1852,732,1234,396,1866,763,1247,
349,1264,957,1893,373,1281,968,1907,405,1490,998,1921,426,1503,1013,1935,438,1516,1025,1954,448,1530,1037,1968,465,1544,1050,1982,487,1557,1064,1996,
512,1571,1078,2010,532,1584,1092,2024,559,1598,1106,2038,582,1612,1120,2052,605,1625,1134,2066,624,1639,1150,2082,651,1653,1165,2096,678,1667,1180,2110,
697,1680,1194,2124,716,1694,1208,2138,743,1708,1222,2152,778,1721,1236,2166,788,1735,1250,2180,802,1748,1267,2194,830,1762,1285,2209,862,1776,1300,2227,
870,1789,1313,2241,882,1803,1326,2255,910,1817,1339,2269,942,1830,1352,2283,951,1843,1365,2297,962,1857,1378,2311,978,1871,1391,2325,1003,1884,1404,2334,
1004,1908,1060,780,1016,1922,1073,789,1027,1936,1087,801,1039,1949,1101,829,1052,1963,1115,861,1068,1977,1129,869,1083,1990,1142,880,1098,2003,1156,900,
1112,2017,1170,940,1126,2031,1183,948,1140,2044,1196,956,1154,2058,1210,965,1168,2072,1224,994,1182,2085,1237,1008,1197,2099,1251,1019,1213,2112,1268,1030,
1227,2126,1284,1042,1241,2140,1297,1056,1255,2153,1309,1071,1272,2167,1322,1084,1289,2181,1335,1097,1302,2195,1347,1111,1315,2208,1360,1125,1329,2222,1373,1138,
1344,2236,1386,1152,1357,2249,1399,1166,1370,2263,1411,1179,1383,2276,1424,1193,1396,2290,1437,1206,1409,2304,1449,1220,1422,2317,1461,1234,1435,2330,1474,1247,
1434,39,1470,358,1447,58,1481,382,1460,90,1494,417,1473,119,1508,430,1485,137,1522,442,1498,161,1536,456,1517,201,1549,473,1531,229,1563,503,
1545,291,1577,514,1559,306,1590,541,1573,326,1603,565,1587,348,1617,596,1601,371,1631,608,1615,398,1644,635,1629,423,1658,658,1645,436,1672,689,
1659,446,1685,700,1673,463,1699,727,1687,480,1712,755,1701,505,1726,783,1715,520,1740,794,1729,543,1753,810,1743,566,1767,842,1757,597,1781,866,
1772,607,1795,874,1790,630,1808,890,1804,657,1822,922,1818,688,1836,946,1832,699,1849,954,1846,718,1863,964,1860,745,1876,985,1874,779,1890,1007,
1873,963,1900,780,1887,979,1914,789,1901,1005,1928,801,1915,1017,1942,829,1929,1028,1956,861,1943,1040,1971,869,1959,1055,1986,880,1973,1070,2000,900,
1987,1085,2014,940,2001,1099,2028,948,2015,1113,2042,956,2029,1127,2056,965,2043,1141,2070,994,2057,1155,2084,1008,2071,1169,2098,1019,2086,1185,2117,1030,
2104,1200,2131,1042,2118,1214,2145,1056,2132,1228,2159,1071,2146,1242,2173,1084,2160,1256,2187,1097,2174,1273,2201,1111,2188,1290,2215,1125,2202,1303,2229,1138,
2217,1317,2245,1152,2232,1332,2259,1166,2246,1345,2273,1179,2260,1358,2287,1193,2274,1371,2301,1206,2288,1384,2315,1220,2302,1397,2329,1234,2316,1410,2335,1247,
1916,1066,6,1440,1930,1080,18,1453,1944,1094,32,1466,1957,1107,55,1479,1970,1121,74,1493,1984,1135,95,1506,1998,1148,111,1520,2011,1161,120,1535,
2025,1175,129,1550,2039,1189,142,1564,2053,1202,152,1578,2067,1216,166,1592,2080,1230,180,1606,2094,1244,202,1620,2108,1258,216,1634,2121,1274,230,1648,
2134,1291,240,1662,2148,1304,254,1681,2162,1316,262,1695,2175,1328,267,1709,2189,1341,275,1723,2203,1354,289,1737,2216,1367,299,1751,2230,1379,305,1765,
2243,1392,316,1779,2257,1405,331,1793,2271,1417,341,1809,2284,1429,350,1823,2298,1442,362,1837,2312,1455,378,1851,2326,1467,395,1865,2333,1480,415,1879,
151,1475,439,780,179,1486,449,789,205,1499,470,801,236,1512,488,829,253,1526,510,861,266,1540,529,869,282,1553,557,880,301,1566,579,900,
315,1580,602,940,340,1594,617,948,360,1607,644,956,381,1621,671,965,416,1635,693,994,429,1649,704,1008,441,1663,731,1019,451,1676,758,1030,
472,1690,785,1042,489,1704,793,1056,511,1717,809,1071,530,1730,841,1084,556,1744,865,1097,578,1758,873,1111,600,1771,889,1125,615,1785,917,1138,
638,1799,945,1152,667,1812,953,1166,690,1826,961,1179,701,1839,976,1193,728,1853,1001,1206,754,1867,1014,1220,781,1880,1024,1234,790,1894,1035,1247,
966,1905,1036,1896,995,1919,1048,1910,1009,1933,1062,1924,1020,1947,1076,1938,1031,1961,1089,1951,1044,1975,1103,1965,1058,1989,1117,1979,1074,2004,1131,1992,
1090,2022,1145,2006,1104,2036,1158,2019,1118,2050,1172,2033,1132,2064,1186,2047,1146,2078,1199,2060,1160,2092,1212,2074,1174,2106,1226,2088,1188,2120,1240,2101,
1203,2135,1253,2114,1219,2150,1270,2128,1233,2164,1287,2142,1248,2178,1299,2155,1265,2192,1312,2169,1282,2206,1324,2183,1295,2220,1337,2197,1308,2234,1350,2211,
1321,2248,1362,2224,1334,2262,1375,2238,1349,2281,1388,2252,1363,2295,1401,2265,1376,2309,1413,2278,1389,2323,1426,2292,1402,2332,1439,2306,1415,2336,1452,2319,
1419,7,1445,780,1433,19,1458,789,1446,33,1471,801,1459,56,1483,829,1472,88,1496,861,1484,113,1510,869,1497,127,1524,880,1511,136,1538,900,
1525,150,1552,940,1539,160,1567,948,1558,174,1585,956,1572,192,1599,965,1586,214,1613,994,1600,224,1627,1008,1614,238,1641,1019,1628,248,1655,1030,
1642,261,1669,1042,1656,265,1683,1056,1670,274,1698,1071,1686,283,1713,1084,1700,297,1727,1097,1714,303,1741,1111,1728,313,1755,1125,1742,324,1769,1138,
1756,338,1783,1152,1770,347,1797,1166,1784,359,1811,1179,1798,372,1825,1193,1813,388,1844,1206,1831,404,1858,1220,1845,419,1872,1234,1859,427,1886,1247,
1261,422,1878,977,1278,435,1892,1002,1507,486,1906,1015,1521,509,1920,1026,1534,528,1934,1038,1548,555,1948,1051,1562,577,1962,1065,1575,601,1976,1079,
1589,616,1995,1093,1602,639,2009,1109,1616,670,2023,1124,1630,692,2037,1139,1643,703,2051,1153,1657,730,2065,1167,1671,757,2079,1181,1684,784,2093,1195,
1697,792,2107,1209,1711,808,2123,1223,1725,840,2137,1238,1738,864,2151,1254,1752,872,2165,1271,1766,888,2179,1288,1780,916,2193,1301,1794,944,2207,1314,
1807,952,2221,1327,1821,960,2235,1340,1835,975,2250,1353,1848,1000,2268,1366,1861,1012,2282,1382,1875,1023,2296,1395,1889,1033,2310,1408,1903,1046,2324,1421,
1895,1041,1897,780,1909,1054,1911,789,1923,1069,1925,801,1937,1082,1939,829,1950,1096,1952,861,1964,1110,1966,869,1978,1123,1980,880,1991,1137,1993,900,
2005,1151,2007,940,2018,1164,2020,948,2032,1177,2034,956,2046,1191,2048,965,2059,1205,2061,994,2073,1218,2075,1008,2087,1232,2089,1019,2100,1246,2102,1030,
2113,1260,2115,1042,2127,1277,2129,1056,2141,1293,2143,1071,2154,1306,2156,1084,2168,1319,2170,1097,2182,1331,2184,1111,2196,1343,2198,1125,2210,1356,2212,1138,
2223,1369,2225,1152,2237,1381,2239,1166,2251,1394,2253,1179,2264,1407,2266,1193,2277,1420,2279,1206,2291,1432,2293,1220,2305,1444,2307,1234,2318,1457,2320,1247,
8,1448,325,1262,20,1462,346,1279,34,1477,369,1488,57,1491,397,1501,89,1504,424,1514,114,1518,437,1528,135,1532,447,1542,144,1546,464,1555,
159,1560,485,1569,172,1574,507,1582,191,1588,526,1596,204,1604,554,1610,223,1618,580,1623,237,1632,603,1637,246,1646,619,1651,256,1660,650,1665,
264,1674,673,1678,273,1688,696,1692,281,1702,711,1706,292,1716,742,1719,302,1731,769,1733,312,1749,787,1746,322,1763,800,1760,333,1777,828,1774,
345,1791,860,1787,357,1805,868,1801,370,1819,881,1815,383,1833,901,1828,399,1847,941,1841,418,1862,949,1855,425,1877,958,1869,431,1891,969,1882,
434,1885,980,780,445,1899,1006,789,508,1913,1018,801,527,1927,1029,829,553,1941,1043,861,576,1955,1057,869,599,1969,1072,880,614,1983,1086,900,
637,1997,1100,940,668,2012,1114,948,691,2027,1128,956,702,2041,1144,965,729,2055,1159,994,756,2069,1173,1008,782,2083,1187,1019,791,2097,1201,1030,
807,2111,1215,1042,839,2125,1229,1056,863,2139,1243,1071,871,2158,1257,1084,887,2172,1276,1097,915,2186,1294,1111,943,2200,1307,1125,950,2214,1320,1138,
959,2228,1333,1152,970,2242,1346,1166,999,2256,1359,1179,1011,2270,1372,1193,1021,2286,1385,1206,1032,2300,1398,1220,1045,2314,1414,1234,1059,2328,1427,1247,
1047,1898,1423,25,1061,1912,1436,44,1075,1926,1451,63,1088,1940,1465,101,1102,1953,1478,126,1116,1967,1492,145,1130,1981,1505,173,1143,1994,1519,215,
1157,2008,1533,239,1171,2021,1547,284,1184,2035,1561,304,1198,2049,1576,323,1211,2062,1591,344,1225,2076,1605,363,1239,2090,1619,394,1252,2103,1633,421,
1269,2116,1647,432,1286,2130,1661,443,1298,2144,1675,457,1310,2157,1689,478,1323,2171,1703,504,1336,2185,1722,515,1348,2199,1736,542,1361,2213,1750,564,
1374,2226,1764,595,1387,2240,1778,606,1400,2254,1792,625,1412,2267,1806,656,1425,2280,1820,687,1438,2294,1834,698,1450,2308,1850,717,1463,2321,1864,744,
1464,342,1263,780,1476,361,1280,789,1487,393,1489,801,1500,420,1502,829,1513,433,1515,861,1527,444,1529,869,1541,458,1543,880,1554,479,1556,900,
1568,506,1570,940,1581,521,1583,948,1595,548,1597,956,1609,575,1611,965,1622,598,1624,994,1636,613,1638,1008,1650,636,1652,1019,1664,669,1666,1030,
1677,694,1679,1042,1691,709,1693,1056,1705,733,1707,1071,1718,764,1720,1084,1732,786,1734,1097,1745,795,1747,1111,1759,819,1761,1125,1773,847,1775,1138,
1786,867,1788,1152,1800,875,1802,1166,1814,899,1816,1179,1827,935,1829,1193,1840,947,1842,1206,1854,955,1856,1220,1868,967,1870,1234,1881,996,1883,1247,
1888,997,1904,1053,1902,1010,1917,1067,1918,1022,1931,1081,1932,1034,1945,1095,1946,1049,1958,1108,1960,1063,1972,1122,1974,1077,1985,1136,1988,1091,1999,1149,
2002,1105,2013,1163,2016,1119,2026,1176,2030,1133,2040,1190,2045,1147,2054,1204,2063,1162,2068,1217,2077,1178,2081,1231,2091,1192,2095,1245,2105,1207,2109,1259,
2119,1221,2122,1275,2133,1235,2136,1292,2147,1249,2149,1305,2161,1266,2163,1318,2176,1283,2177,1330,2191,1296,2190,1342,2205,1311,2204,1355,2219,1325,2218,1368,
2233,1338,2231,1380,2247,1351,2244,1393,2261,1364,2258,1406,2275,1377,2272,1418,2289,1390,2285,1431,2303,1403,2299,1443,2322,1416,2313,1456,2331,1430,2327,1469,
29,43,38,48,29,43,38,48,29,43,38,48,29,43,38,48,67,94,83,105,67,94,83,105,67,94,83,105,67,94,83,105,
124,141,134,149,124,141,134,149,124,141,134,149,124,141,134,149,170,200,190,213,170,200,190,213,170,200,190,213,170,200,190,213,
234,252,245,260,234,252,245,260,234,252,245,260,234,252,245,260,271,288,280,296,271,288,280,296,271,288,280,296,271,288,280,296,
310,330,321,337,310,330,321,337,310,330,321,337,310,330,321,337,358,393,380,403,358,393,380,403,358,393,380,403,358,393,380,403,
4,12,16,24,4,12,16,24,4,12,16,24,4,12,16,24,87,99,109,118,87,99,109,118,87,99,109,118,87,99,109,118,
196,209,221,228,196,209,221,228,196,209,221,228,196,209,221,228,441,451,458,469,441,451,458,469,441,451,458,469,441,451,458,469,
522,542,550,561,522,542,550,561,522,542,550,561,522,542,550,561,630,638,649,656,630,638,649,656,630,638,649,656,630,638,649,656,
725,736,743,754,725,736,743,754,725,736,743,754,725,736,743,754,822,829,838,844,822,829,838,844,822,829,838,844,822,829,838,844,
53,62,72,78,53,62,72,78,53,62,72,78,53,62,72,78,157,165,178,186,157,165,178,186,157,165,178,186,157,165,178,186,
352,365,375,389,352,365,375,389,352,365,375,389,352,365,375,389,475,482,491,499,475,482,491,499,475,482,491,499,475,482,491,499,
565,573,584,591,565,573,584,591,565,573,584,591,565,573,584,591,660,671,678,688,660,671,678,688,660,671,678,688,660,671,678,688,
758,766,775,782,758,766,775,782,758,766,775,782,758,766,775,782,850,858,863,870,850,858,863,870,850,858,863,870,850,858,863,870,
425,429,408,419,425,429,408,419,425,429,408,419,425,429,408,419,516,530,534,538,516,530,534,538,516,530,534,538,516,530,534,538,
606,618,622,626,606,618,622,626,606,618,622,626,606,618,622,626,698,709,714,718,698,709,714,718,698,709,714,718,698,709,714,718,
789,800,805,809,789,800,805,809,789,800,805,809,789,800,805,809,874,883,887,891,874,883,887,891,874,883,887,891,874,883,887,891,
931,942,946,950,931,942,946,950,931,942,946,950,931,942,946,950,1005,1014,1020,1024,1005,1014,1020,1024,1005,1014,1020,1024,1005,1014,1020,1024,
5,1407,30,812,17,1423,49,825,31,1436,73,845,54,1449,110,865,68,1462,128,893,79,1477,153,901,100,1491,182,910,112,1503,222,923,
125,1516,247,954,130,1530,298,968,143,1545,314,982,158,1563,339,996,171,1577,360,1026,181,1591,395,1039,203,1605,430,1049,217,1621,452,1060,
235,1635,476,1072,241,1649,494,1085,255,1663,507,1100,263,1677,526,1113,272,1696,557,1127,276,1710,579,1140,290,1724,598,1153,300,1738,613,1166,
311,1752,644,1179,317,1768,666,1193,332,1782,689,1206,343,1796,704,1219,359,1810,731,1233,376,1824,750,1247,396,1843,776,1260,412,1857,795,1273,
353,1290,903,1884,385,1306,915,1898,421,1510,935,1912,446,1522,962,1926,470,1536,977,1942,492,1549,992,1956,505,1562,1016,1970,523,1576,1038,1984,
556,1589,1050,1998,580,1603,1061,2017,599,1616,1074,2031,614,1629,1089,2045,645,1642,1106,2059,672,1655,1120,2073,691,1669,1134,2089,710,1682,1148,2103,
737,1695,1164,2117,760,1709,1180,2131,783,1723,1194,2145,810,1736,1208,2164,824,1749,1222,2178,846,1763,1238,2192,866,1776,1253,2206,894,1789,1267,2221,
902,1802,1281,2236,913,1816,1298,2250,926,1829,1318,2264,957,1842,1331,2278,972,1856,1344,2293,989,1870,1357,2311,1008,1883,1370,2325,1034,1896,1386,2333,
938,1918,1090,812,965,1931,1103,825,980,1945,1116,845,995,1958,1129,865,1027,1971,1143,893,1041,1985,1157,901,1052,1999,1170,910,1065,2012,1183,923,
1080,2025,1197,954,1094,2039,1210,968,1108,2052,1223,982,1122,2065,1236,996,1139,2078,1250,1026,1154,2092,1263,1039,1168,2105,1276,1049,1182,2118,1293,1060,
1196,2132,1310,1072,1213,2146,1322,1085,1227,2159,1334,1100,1241,2172,1347,1113,1255,2185,1359,1127,1269,2199,1371,1140,1286,2212,1383,1153,1307,2225,1395,1166,
1320,2238,1408,1179,1333,2251,1420,1193,1346,2265,1433,1206,1362,2279,1445,1219,1375,2292,1457,1233,1388,2305,1470,1247,1401,2319,1482,1260,1416,2331,1494,1273,
1413,39,1490,366,1427,58,1502,398,1442,90,1515,433,1455,119,1528,454,1468,137,1541,478,1481,161,1554,500,1496,201,1567,509,1511,229,1581,543,
1524,291,1594,566,1538,306,1607,585,1552,326,1620,601,1568,348,1634,632,1582,383,1648,652,1596,414,1661,679,1610,443,1674,694,1624,464,1688,721,
1643,486,1701,744,1657,503,1714,767,1671,512,1727,791,1685,545,1741,815,1699,568,1754,834,1715,587,1767,854,1729,602,1781,878,1743,633,1795,898,
1757,651,1808,907,1771,674,1821,919,1790,693,1835,937,1804,720,1848,964,1818,739,1861,978,1832,762,1874,993,1847,785,1887,1015,1862,811,1901,1037,
1860,908,1888,812,1875,920,1902,825,1889,951,1916,845,1903,967,1930,865,1917,983,1944,893,1932,998,1963,901,1950,1029,1977,910,1964,1044,1991,923,
1978,1056,2005,954,1992,1068,2019,968,2008,1081,2035,982,2022,1095,2049,996,2036,1110,2063,1026,2050,1126,2077,1039,2064,1141,2091,1049,2083,1155,2110,1060,
2097,1169,2124,1072,2111,1185,2138,1085,2125,1200,2152,1100,2139,1214,2167,1113,2155,1228,2182,1127,2169,1242,2196,1140,2183,1259,2210,1153,2197,1274,2224,1166,
2211,1291,2239,1179,2230,1308,2257,1193,2244,1321,2271,1206,2258,1337,2285,1219,2272,1350,2299,1233,2287,1363,2315,1247,2302,1376,2329,1260,2316,1389,2335,1273,
1927,1096,6,1419,1940,1109,18,1432,1953,1123,32,1446,1967,1136,55,1461,1980,1149,74,1474,1993,1162,95,1487,2006,1176,111,1500,2020,1189,120,1514,
2033,1202,129,1529,2046,1216,142,1543,2060,1230,152,1557,2074,1243,166,1571,2087,1256,180,1590,2100,1270,202,1604,2114,1283,216,1618,2127,1299,230,1632,
2140,1315,240,1646,2153,1328,254,1662,2166,1340,262,1676,2180,1352,267,1690,2193,1365,275,1704,2207,1377,289,1718,2220,1390,299,1737,2233,1402,305,1751,
2247,1414,316,1765,2260,1426,331,1779,2273,1438,341,1793,2286,1451,354,1809,2300,1463,370,1823,2313,1475,394,1837,2326,1488,411,1851,2334,1501,431,1866,
151,1495,471,812,179,1506,493,825,205,1518,506,845,236,1531,524,865,253,1544,554,893,266,1558,577,901,282,1572,597,910,301,1585,611,923,
315,1598,642,954,340,1612,665,968,368,1625,684,982,397,1638,703,996,432,1651,729,1026,453,1665,748,1039,477,1678,771,1049,495,1691,794,1060,
508,1705,817,1072,525,1719,833,1085,555,1732,853,1100,578,1745,877,1113,596,1759,897,1127,610,1772,906,1140,640,1785,917,1153,663,1798,934,1166,
682,1812,961,1179,699,1825,975,1193,726,1838,988,1206,745,1852,1007,1219,768,1865,1033,1233,790,1879,1045,1247,813,1892,1055,1260,830,1905,1066,1273,
912,1891,1067,1907,925,1910,1079,1921,956,1924,1092,1934,973,1938,1105,1947,990,1952,1119,1960,1009,1966,1132,1973,1035,1982,1145,1987,1047,1996,1159,2001,
1058,2010,1173,2014,1071,2024,1186,2027,1086,2038,1199,2041,1101,2057,1212,2054,1115,2071,1226,2067,1130,2085,1239,2080,1146,2099,1252,2094,1160,2113,1265,2107,
1174,2129,1278,2120,1188,2143,1295,2134,1205,2157,1312,2148,1220,2171,1324,2161,1234,2186,1336,2174,1248,2204,1349,2188,1262,2218,1361,2201,1279,2232,1373,2214,
1296,2246,1385,2227,1313,2262,1398,2241,1326,2276,1410,2254,1339,2290,1422,2267,1355,2304,1435,2281,1368,2318,1448,2295,1381,2332,1460,2308,1394,2336,1472,2321,
1399,7,1424,812,1412,19,1437,825,1425,33,1450,845,1439,56,1465,865,1454,88,1479,893,1467,113,1492,901,1480,127,1504,910,1493,136,1517,923,
1505,150,1532,954,1523,160,1550,968,1537,174,1564,982,1551,192,1578,996,1565,214,1592,1026,1580,224,1608,1039,1595,238,1622,1049,1609,248,1636,1060,
1623,261,1650,1072,1637,265,1664,1085,1656,274,1683,1100,1670,283,1697,1113,1684,297,1711,1127,1698,303,1725,1140,1712,313,1739,1153,1728,324,1755,1166,
1742,338,1769,1179,1756,347,1783,1193,1770,367,1797,1206,1784,384,1811,1219,1803,404,1830,1233,1817,420,1844,1247,1831,435,1858,1260,1845,447,1872,1273,
1287,442,1864,918,1303,463,1878,936,1526,518,1897,963,1540,553,1911,979,1553,576,1925,994,1566,595,1939,1025,1579,609,1955,1040,1593,641,1969,1051,
1606,664,1983,1062,1619,683,1997,1077,1633,702,2011,1093,1647,728,2030,1107,1660,747,2044,1121,1673,770,2058,1135,1687,793,2072,1151,1700,816,2086,1167,
1713,832,2102,1181,1726,852,2116,1195,1740,876,2130,1209,1753,896,2144,1225,1766,905,2158,1240,1780,916,2177,1254,1794,933,2191,1268,1807,959,2205,1282,
1820,974,2219,1302,1833,987,2234,1319,1846,1006,2249,1332,1859,1031,2263,1345,1873,1043,2277,1358,1886,1054,2291,1374,1899,1064,2306,1387,1913,1076,2324,1400,
1906,1070,1908,812,1920,1084,1922,825,1933,1099,1935,845,1946,1112,1948,865,1959,1125,1961,893,1972,1138,1974,901,1986,1152,1988,910,2000,1165,2002,923,
2013,1178,2015,954,2026,1191,2028,968,2040,1204,2042,982,2053,1218,2055,996,2066,1232,2068,1026,2079,1245,2081,1039,2093,1258,2095,1049,2106,1272,2108,1060,
2119,1285,2121,1072,2133,1301,2135,1085,2147,1317,2149,1100,2160,1330,2162,1113,2173,1342,2175,1127,2187,1354,2189,1140,2200,1367,2202,1153,2213,1380,2215,1166,
2226,1392,2228,1179,2240,1404,2242,1193,2253,1417,2255,1206,2266,1429,2268,1219,2280,1441,2282,1233,2294,1453,2296,1247,2307,1466,2309,1260,2320,1478,2322,1273,
8,1430,325,1288,20,1443,346,1304,34,1456,381,1508,57,1469,413,1520,89,1484,444,1534,114,1498,465,1547,135,1512,487,1560,144,1525,504,1574,
159,1539,517,1587,172,1555,551,1601,191,1569,574,1614,204,1583,594,1627,223,1597,612,1640,237,1611,643,1653,246,1630,667,1667,256,1644,690,1680,
264,1658,705,1693,273,1672,732,1707,281,1686,759,1721,292,1702,778,1734,302,1716,801,1747,312,1730,823,1761,322,1744,840,1774,333,1758,864,1787,
345,1777,892,1800,361,1791,900,1814,382,1805,911,1827,399,1819,924,1840,415,1834,955,1854,434,1849,970,1868,445,1863,985,1881,459,1877,1000,1894,
462,1871,922,812,485,1885,953,825,552,1900,969,845,575,1915,984,865,593,1929,999,893,608,1943,1032,901,639,1957,1046,910,662,1976,1057,923,
681,1990,1069,954,700,2004,1082,968,727,2018,1098,982,746,2032,1114,996,769,2048,1128,1026,792,2062,1142,1039,814,2076,1156,1049,831,2090,1172,1060,
851,2104,1187,1072,875,2123,1201,1085,895,2137,1215,1100,904,2151,1229,1113,914,2165,1246,1127,932,2179,1261,1140,958,2195,1275,1153,971,2209,1292,1166,
986,2223,1309,1179,1001,2237,1325,1193,1030,2252,1338,1206,1042,2270,1351,1219,1053,2284,1364,1233,1063,2298,1378,1247,1075,2312,1393,1260,1088,2328,1406,1273,
1078,1909,1405,25,1091,1923,1418,44,1104,1936,1431,63,1117,1949,1444,101,1131,1962,1458,126,1144,1975,1473,145,1158,1989,1486,173,1171,2003,1499,215,
1184,2016,1513,239,1198,2029,1527,284,1211,2043,1542,304,1224,2056,1556,323,1237,2069,1570,344,1251,2082,1584,371,1264,2096,1599,410,1277,2109,1617,437,
1294,2122,1631,460,1311,2136,1645,483,1323,2150,1659,501,1335,2163,1675,510,1348,2176,1689,544,1360,2190,1703,567,1372,2203,1717,586,1384,2216,1731,600,
1397,2229,1750,631,1409,2243,1764,650,1421,2256,1778,673,1434,2269,1792,692,1447,2283,1806,719,1459,2297,1822,738,1471,2310,1836,761,1483,2323,1850,784,
1485,342,1289,812,1497,369,1305,825,1507,409,1509,845,1519,436,1521,865,1533,461,1535,893,1546,484,1548,901,1559,502,1561,910,1573,511,1575,923,
1586,546,1588,954,1600,569,1602,968,1613,592,1615,982,1626,607,1628,996,1639,634,1641,1026,1652,661,1654,1039,1666,680,1668,1049,1679,701,1681,1060,
1692,730,1694,1072,1706,749,1708,1085,1720,777,1722,1100,1733,796,1735,1113,1746,818,1748,1127,1760,839,1762,1140,1773,859,1775,1153,1786,879,1788,1166,
1799,899,1801,1179,1813,909,1815,1193,1826,921,1828,1206,1839,952,1841,1219,1853,966,1855,1233,1867,981,1869,1247,1880,997,1882,1260,1893,1028,1895,1273,
1876,927,1914,1083,1890,960,1928,1097,1904,976,1941,1111,1919,991,1954,1124,1937,1010,1968,1137,1951,1036,1981,1150,1965,1048,1994,1163,1979,1059,2007,1177,
1995,1073,2021,1190,2009,1087,2034,1203,2023,1102,2047,1217,2037,1118,2061,1231,2051,1133,2075,1244,2070,1147,2088,1257,2084,1161,2101,1271,2098,1175,2115,1284,
2112,1192,2128,1300,2126,1207,2141,1316,2142,1221,2154,1329,2156,1235,2168,1341,2170,1249,2181,1353,2184,1266,2194,1366,2198,1280,2208,1379,2217,1297,2222,1391,
2231,1314,2235,1403,2245,1327,2248,1415,2259,1343,2261,1428,2275,1356,2274,1440,2289,1369,2288,1452,2303,1382,2301,1464,2317,1396,2314,1476,2330,1411,2327,1489,
29,43,38,48,29,43,38,48,29,43,38,48,29,43,38,48,67,94,83,105,67,94,83,105,67,94,83,105,67,94,83,105,
124,141,134,149,124,141,134,149,124,141,134,149,124,141,134,149,170,200,190,213,170,200,190,213,170,200,190,213,170,200,190,213,
234,252,245,261,234,252,245,261,234,252,245,261,234,252,245,261,282,309,297,321,282,309,297,321,282,309,297,321,282,309,297,321,
342,366,358,374,342,366,358,374,342,366,358,374,342,366,358,374,394,428,416,442,394,428,416,442,394,428,416,442,394,428,416,442,
4,12,16,24,4,12,16,24,4,12,16,24,4,12,16,24,87,99,109,118,87,99,109,118,87,99,109,118,87,99,109,118,
196,209,221,228,196,209,221,228,196,209,221,228,196,209,221,228,314,326,336,347,314,326,336,347,314,326,336,347,314,326,336,347,
438,456,476,487,438,456,476,487,438,456,476,487,438,456,476,487,541,549,556,570,541,549,556,570,541,549,556,570,541,549,556,570,
636,647,662,677,636,647,662,677,636,647,662,677,636,647,662,677,740,759,769,776,740,759,769,776,740,759,769,776,740,759,769,776,
53,62,72,78,53,62,72,78,53,62,72,78,53,62,72,78,157,165,178,186,157,165,178,186,157,165,178,186,157,165,178,186,
269,277,290,303,269,277,290,303,269,277,290,303,269,277,290,303,386,399,409,424,386,399,409,424,386,399,409,424,386,399,409,424,
511,517,525,533,511,517,525,533,511,517,525,533,511,517,525,533,600,611,617,624,600,611,617,624,600,611,617,624,600,611,617,624,
699,706,716,723,699,706,716,723,699,706,716,723,699,706,716,723,794,805,811,825,794,805,811,825,794,805,811,825,794,805,811,825,
468,472,447,462,468,472,447,462,468,472,447,462,468,472,447,462,560,575,579,583,560,575,579,583,560,575,579,583,560,575,579,583,
642,658,666,670,642,658,666,670,642,658,666,670,642,658,666,670,730,745,750,754,730,745,750,754,730,745,750,754,730,745,750,754,
817,832,837,841,817,832,837,841,817,832,837,841,817,832,837,841,888,900,905,909,888,900,905,909,888,900,905,909,888,900,905,909,
961,972,976,980,961,972,976,980,961,972,976,980,961,972,976,980,1033,1043,1049,1053,1033,1043,1049,1053,1033,1043,1049,1053,1033,1043,1049,1053,
5,1389,30,844,17,1402,49,855,31,1415,73,868,54,1431,110,882,68,1444,128,912,79,1457,153,926,100,1473,182,938,112,1486,222,953,
125,1499,247,982,130,1514,271,997,143,1529,305,1012,158,1542,337,1024,171,1556,362,1055,181,1572,389,1067,203,1586,429,1077,217,1600,478,1088,
235,1619,497,1099,241,1633,513,1113,256,1647,536,1127,265,1663,561,1140,283,1677,588,1153,293,1691,607,1165,315,1706,630,1179,329,1724,653,1191,
343,1738,684,1205,354,1753,694,1219,368,1768,717,1231,380,1782,736,1245,395,1796,771,1257,411,1815,786,1270,431,1829,800,1283,451,1843,827,1296,
388,1313,846,1870,420,1328,862,1886,464,1528,876,1900,492,1541,896,1914,504,1553,923,1929,526,1567,939,1947,542,1579,955,1961,564,1592,988,1975,
593,1606,1005,1991,612,1618,1020,2005,631,1632,1039,2019,654,1645,1066,2038,685,1658,1078,2052,700,1671,1089,2066,719,1683,1105,2082,746,1697,1120,2096,
777,1710,1134,2110,788,1723,1149,2125,807,1737,1166,2143,842,1749,1180,2157,854,1763,1194,2171,869,1775,1211,2187,884,1788,1225,2201,913,1802,1239,2215,
928,1814,1256,2234,943,1828,1271,2248,957,1841,1285,2262,986,1854,1302,2278,1002,1867,1319,2292,1019,1879,1336,2306,1036,1893,1351,2325,1062,1905,1365,2333,
879,1928,1117,844,910,1941,1129,855,927,1955,1143,868,942,1966,1157,882,962,1980,1169,912,992,1993,1183,926,1007,2006,1195,938,1022,2020,1208,953,
1054,2032,1221,982,1069,2046,1234,997,1080,2059,1248,1012,1093,2071,1260,1024,1108,2085,1274,1055,1122,2097,1287,1067,1136,2111,1299,1077,1154,2124,1316,1088,
1168,2137,1331,1099,1182,2151,1344,1113,1199,2162,1357,1127,1213,2176,1368,1140,1227,2189,1381,1153,1244,2202,1392,1165,1259,2216,1404,1179,1273,2228,1416,1191,
1289,2242,1428,1205,1304,2254,1441,1219,1321,2267,1452,1231,1339,2281,1465,1245,1354,2293,1477,1257,1367,2307,1488,1270,1380,2320,1501,1283,1396,2332,1512,1296,
1395,39,1509,401,1408,58,1521,433,1421,90,1532,480,1436,119,1545,496,1450,137,1558,512,1463,161,1571,534,1476,201,1585,551,1492,229,1597,584,
1505,255,1611,595,1518,284,1624,618,1533,316,1636,637,1547,349,1650,672,1561,370,1662,688,1580,404,1676,707,1594,448,1689,726,1608,488,1702,761,
1623,502,1716,780,1638,521,1727,795,1652,543,1741,819,1666,571,1754,848,1685,594,1767,861,1699,613,1781,875,1713,638,1793,894,1729,673,1807,919,
1743,687,1819,934,1757,702,1832,949,1776,725,1846,968,1790,760,1858,993,1804,779,1872,1008,1820,790,1885,1023,1834,813,1898,1044,1848,843,1912,1065,
1847,852,1874,844,1861,865,1888,855,1880,881,1907,868,1894,914,1921,882,1908,929,1935,912,1924,947,1951,926,1938,966,1965,938,1952,994,1979,953,
1967,1009,1998,982,1985,1026,2012,997,1999,1057,2026,1012,2013,1072,2041,1024,2029,1084,2056,1055,2043,1096,2070,1067,2057,1109,2084,1077,2076,1126,2103,1088,
2090,1141,2117,1099,2104,1155,2131,1113,2120,1172,2147,1127,2134,1186,2161,1140,2148,1200,2175,1153,2163,1216,2194,1165,2181,1232,2208,1179,2195,1246,2222,1191,
2209,1261,2237,1205,2225,1277,2252,1219,2239,1291,2266,1231,2253,1305,2280,1245,2272,1329,2299,1257,2286,1342,2313,1270,2300,1355,2327,1283,2316,1371,2335,1296,
1936,1123,6,1398,1948,1137,18,1414,1962,1148,32,1427,1976,1162,55,1440,1988,1175,74,1456,2002,1188,95,1469,2014,1202,111,1482,2027,1214,120,1498,
2040,1228,129,1511,2053,1240,142,1523,2067,1253,152,1536,2079,1267,166,1555,2093,1279,180,1569,2106,1293,202,1583,2118,1306,216,1599,2132,1322,230,1613,
2144,1338,240,1627,2158,1349,254,1646,2172,1362,264,1660,2184,1373,278,1674,2198,1386,292,1690,2210,1399,310,1704,2223,1410,328,1718,2236,1423,338,1732,
2249,1434,353,1751,2263,1446,367,1765,2275,1459,377,1779,2289,1470,390,1795,2302,1483,410,1809,2314,1495,430,1823,2328,1507,450,1842,2334,1519,477,1856,
151,1513,506,844,179,1524,527,855,205,1537,544,868,236,1549,565,882,253,1563,591,912,273,1575,605,926,299,1588,629,938,330,1601,651,953,
352,1614,682,982,376,1628,693,997,403,1640,712,1012,432,1654,735,1024,479,1667,765,1055,495,1679,784,1067,507,1693,799,1077,528,1705,826,1088,
545,1719,850,1099,566,1733,860,1113,592,1745,874,1127,606,1759,891,1140,628,1771,918,1153,650,1784,933,1165,680,1797,946,1179,691,1810,965,1191,
710,1824,990,1205,731,1836,1004,1219,762,1850,1018,1231,781,1863,1035,1245,796,1875,1061,1257,818,1889,1073,1270,845,1901,1083,1283,857,1915,1094,1296,
856,1882,1095,1917,870,1896,1106,1931,892,1911,1119,1943,920,1926,1132,1957,935,1940,1145,1969,950,1954,1159,1982,983,1973,1171,1995,998,1987,1185,2008,
1014,2001,1198,2022,1037,2017,1210,2035,1063,2031,1224,2048,1075,2045,1236,2061,1086,2064,1250,2073,1100,2078,1263,2087,1114,2092,1276,2099,1130,2108,1290,2113,
1146,2122,1301,2127,1160,2136,1318,2139,1174,2150,1334,2153,1192,2169,1346,2165,1206,2183,1359,2178,1220,2197,1370,2191,1237,2213,1383,2204,1251,2227,1394,2218,
1265,2241,1406,2230,1282,2260,1419,2244,1297,2274,1430,2257,1314,2288,1443,2269,1333,2304,1455,2283,1347,2318,1467,2295,1360,2331,1480,2309,1374,2336,1490,2322,
1378,7,1403,844,1391,19,1417,855,1407,33,1432,868,1420,56,1445,882,1433,88,1458,912,1449,113,1474,926,1462,127,1487,938,1475,136,1500,953,
1491,150,1516,982,1504,160,1530,997,1517,174,1543,1012,1531,192,1559,1024,1546,214,1573,1055,1560,224,1587,1067,1574,238,1602,1077,1593,248,1620,1088,
1607,262,1634,1099,1621,272,1648,1113,1637,286,1664,1127,1651,304,1678,1140,1665,322,1692,1153,1684,332,1711,1165,1698,350,1725,1179,1712,360,1739,1191,
1728,375,1755,1205,1742,387,1769,1219,1756,402,1783,1231,1770,419,1798,1245,1789,443,1816,1257,1803,463,1830,1270,1817,482,1844,1283,1833,494,1860,1296,
1310,489,1855,863,1325,501,1869,877,1544,563,1883,901,1557,590,1899,924,1570,604,1913,941,1584,627,1927,956,1596,649,1946,991,1610,681,1960,1006,
1622,692,1974,1021,1635,711,1990,1045,1649,734,2004,1068,1661,764,2018,1079,1675,783,2033,1090,1688,798,2051,1107,1701,821,2065,1121,1714,849,2080,1135,
1726,859,2095,1152,1740,873,2109,1167,1752,890,2123,1181,1766,917,2142,1197,1780,932,2156,1212,1792,945,2170,1226,1806,964,2186,1241,1818,989,2200,1258,
1831,1003,2214,1272,1845,1016,2233,1286,1857,1034,2247,1303,1871,1060,2261,1320,1884,1071,2276,1337,1897,1082,2291,1353,1910,1092,2305,1366,1922,1103,2319,1379,
1916,1098,1918,844,1930,1112,1932,855,1942,1125,1944,868,1956,1139,1958,882,1968,1151,1970,912,1981,1164,1983,926,1994,1177,1996,938,2007,1190,2009,953,
2021,1204,2023,982,2034,1217,2036,997,2047,1230,2049,1012,2060,1243,2062,1024,2072,1255,2074,1055,2086,1269,2088,1067,2098,1281,2100,1077,2112,1295,2114,1088,
2126,1309,2128,1099,2138,1324,2140,1113,2152,1341,2154,1127,2164,1352,2166,1140,2177,1364,2179,1153,2190,1376,2192,1165,2203,1388,2205,1179,2217,1401,2219,1191,
2229,1412,2231,1205,2243,1425,2245,1219,2256,1437,2258,1231,2268,1448,2270,1245,2282,1461,2284,1257,2294,1472,2296,1270,2308,1485,2310,1283,2321,1497,2323,1296,
8,1409,361,1311,20,1422,382,1326,34,1438,417,1526,57,1451,452,1539,89,1464,490,1551,114,1479,503,1565,135,1493,520,1577,144,1506,537,1590,
159,1520,562,1604,172,1534,587,1616,191,1548,602,1630,204,1562,626,1643,223,1581,652,1656,237,1595,683,1669,246,1609,695,1681,257,1625,718,1695,
270,1639,741,1708,285,1653,772,1721,298,1672,787,1735,317,1686,806,1747,331,1700,833,1761,348,1715,853,1773,359,1730,866,1786,369,1744,880,1800,
381,1758,911,1812,400,1777,925,1826,418,1791,940,1839,434,1805,954,1852,458,1821,985,1865,481,1835,999,1877,491,1849,1013,1891,498,1868,1028,1903,
500,1859,867,844,519,1873,883,855,589,1887,915,868,603,1906,931,882,625,1920,948,912,648,1934,967,926,679,1949,995,938,690,1964,1010,953,
709,1978,1027,982,732,1992,1058,997,763,2011,1074,1012,782,2025,1085,1024,797,2039,1097,1055,820,2055,1111,1067,847,2069,1128,1077,858,2083,1142,1088,
872,2102,1156,1099,889,2116,1173,1113,916,2130,1187,1127,930,2146,1201,1140,944,2160,1218,1153,963,2174,1233,1165,987,2188,1247,1179,1001,2207,1264,1191,
1015,2221,1278,1205,1029,2235,1292,1219,1059,2251,1308,1231,1070,2265,1330,1245,1081,2279,1343,1257,1091,2298,1356,1270,1102,2312,1372,1283,1116,2326,1385,1296,
1104,1919,1384,25,1118,1933,1397,44,1131,1945,1413,63,1144,1959,1426,101,1158,1971,1439,126,1170,1984,1453,145,1184,1997,1468,173,1196,2010,1481,215,
1209,2024,1494,239,1223,2037,1510,263,1235,2050,1522,291,1249,2063,1535,327,1262,2075,1554,351,1275,2089,1568,379,1288,2101,1582,412,1300,2115,1598,457,
1317,2129,1612,493,1332,2141,1626,505,1345,2155,1641,529,1358,2167,1659,550,1369,2180,1673,585,1382,2193,1687,596,1393,2206,1703,619,1405,2220,1717,632,
1418,2232,1731,671,1429,2246,1750,686,1442,2259,1764,701,1454,2271,1778,724,1466,2285,1794,755,1478,2297,1808,778,1489,2311,1822,789,1502,2324,1837,812,
1503,378,1312,844,1515,405,1327,855,1525,449,1527,868,1538,483,1540,882,1550,499,1552,912,1564,518,1566,926,1576,535,1578,938,1589,552,1591,953,
1603,586,1605,982,1615,601,1617,997,1629,620,1631,1012,1642,643,1644,1024,1655,678,1657,1055,1668,689,1670,1067,1680,708,1682,1077,1694,733,1696,1088,
1707,770,1709,1099,1720,785,1722,1113,1734,801,1736,1127,1746,828,1748,1140,1760,851,1762,1153,1772,864,1774,1165,1785,878,1787,1179,1799,895,1801,1191,
1811,922,1813,1205,1825,937,1827,1219,1838,951,1840,1231,1851,981,1853,1245,1864,996,1866,1257,1876,1011,1878,1270,1890,1025,1892,1283,1902,1056,1904,1296,
1862,871,1923,1110,1881,893,1937,1124,1895,921,1950,1138,1909,936,1963,1150,1925,952,1977,1163,1939,984,1989,1176,1953,1000,2003,1189,1972,1017,2015,1203,
1986,1038,2028,1215,2000,1064,2042,1229,2016,1076,2054,1242,2030,1087,2068,1254,2044,1101,2081,1268,2058,1115,2094,1280,2077,1133,2107,1294,2091,1147,2119,1307,
2105,1161,2133,1323,2121,1178,2145,1340,2135,1193,2159,1350,2149,1207,2173,1363,2168,1222,2185,1375,2182,1238,2199,1387,2196,1252,2211,1400,2212,1266,2224,1411,
2226,1284,2238,1424,2240,1298,2250,1435,2255,1315,2264,1447,2273,1335,2277,1460,2287,1348,2290,1471,2301,1361,2303,1484,2317,1377,2315,1496,2330,1390,2329,1508};

typedef unsigned char u8;
constexpr unsigned WS_CNT = 0, WS_TMO = 128, WS_CAL = 1024, WS_HIST = 2048, WS_WORDS = 4096, WS_IN = 8192, WS_REPL = 32768, WS_STAMP = 65536, WS_CSTAMP = 524288,
                   WS_ASTAMP = 720896, WS_BUF = 1u << 20, BUF_BYTES = 104u << 20, EXPO_OFF = 92u << 20, EXP_OFF = 81u << 20, EXPA_OFF = 88u << 20;
constexpr int NWAVES = NCOMM * 4, LDS_BYTES = 163840;

struct Args {
  const u8* x; const u8* w; const float* sx; const float* sw; float* out; u8* myws;
  unsigned long long pad0[2];
  u8* buf[8];
  u8* outp[8];
  u8* flag[8];
  unsigned rank, span0, span1, pad1;
};

typedef const __attribute__((address_space(4))) Args* KArgs;
__device__ __forceinline__ KArgs karg() {
  KArgs p = (KArgs)__builtin_amdgcn_kernarg_segment_ptr();
  asm volatile("" : "+s"(p));
  return p;
}
__device__ __forceinline__ unsigned rfl(unsigned x) {
  unsigned r; asm volatile("s_nop 1\n\tv_readfirstlane_b32 %0, %1\n\ts_nop 3" : "=s"(r) : "v"(x)); return r;
}
__device__ __forceinline__ unsigned lo32(const void* p) { return (unsigned)(size_t)p; }
__device__ __forceinline__ unsigned hi32(const void* p) { return (unsigned)((size_t)p >> 32); }

typedef unsigned u32x4 __attribute__((ext_vector_type(4)));
typedef __attribute__((address_space(3))) volatile unsigned lds_u32;
typedef __attribute__((address_space(3))) volatile unsigned short lds_u16;
typedef __attribute__((address_space(3))) volatile unsigned char lds_u8;

__device__ __forceinline__ void st16(u8* base, unsigned off, unsigned a, unsigned b, unsigned c, unsigned d) {
  const u32x4 v = {a, b, c, d};
  asm volatile("global_store_dwordx4 %0, %1, %2 sc0 sc1\n\ts_nop 1" :: "v"(off), "v"(v), "s"(base) : "memory");
}
__device__ __forceinline__ void st12(u8* base, unsigned off, unsigned a, unsigned b, unsigned c) {
  typedef unsigned u32x3 __attribute__((ext_vector_type(3)));
  const u32x3 v = {a, b, c};
  asm volatile("global_store_dwordx3 %0, %1, %2 sc0 sc1\n\ts_nop 1" :: "v"(off), "v"(v), "s"(base) : "memory");
}
__device__ __forceinline__ void st8(u8* base, unsigned off, unsigned a, unsigned b) {
  typedef unsigned u32x2 __attribute__((ext_vector_type(2)));
  const u32x2 v = {a, b};
  asm volatile("global_store_dwordx2 %0, %1, %2 sc0 sc1" :: "v"(off), "v"(v), "s"(base) : "memory");
}
__device__ __forceinline__ void st4(u8* base, unsigned off, unsigned a) {
  asm volatile("global_store_dword %0, %1, %2 sc0 sc1" :: "v"(off), "v"(a), "s"(base) : "memory");
}

__device__ __forceinline__ unsigned window_base(unsigned mx, int lanes, unsigned win) {
  unsigned top = 0;
#pragma unroll 1
  for (int bit = 6; bit >= 0; --bit) {
    const unsigned cand = top | (1u << bit);
    if (__builtin_popcountll(__builtin_amdgcn_ballot_w64(mx >= cand)) >= lanes) top = cand;
  }
  return top >= win ? top - win : 0u;
}

__device__ __forceinline__ void unit_load(unsigned code, const u8* x, const u8* w, unsigned lane, unsigned (&r)[16]) {
  const unsigned pn = (code >> 10) & 127u, t = (code >> 5) & 31u;
  if (pn < 16u && XAG == 1) {
    const u8* p = x + (size_t)(pn * 256u + lane) * 512u + t * 16u;
#pragma unroll
    for (int k = 0; k < 4; ++k) { const u32x4 v = *(const u32x4*)(p + k * 64 * 512); r[4 * k] = v[0]; r[4 * k + 1] = v[1]; r[4 * k + 2] = v[2]; r[4 * k + 3] = v[3]; }
  } else if (pn < 16u) {
    const u32x4* p = (const u32x4*)(x + (size_t)(pn * 256u + (code & 1u) * 128u + ((code >> 4) & 1u) * 64u + lane) * 512u + (t & 28u) * 16u);
#pragma unroll
    for (int k = 0; k < 4; ++k) { const u32x4 v = p[k]; r[4 * k] = v[0]; r[4 * k + 1] = v[1]; r[4 * k + 2] = v[2]; r[4 * k + 3] = v[3]; }
  } else {
    const u8* p = w + (size_t)t * 16u * 16384u + ((pn - 16u) & 63u) * 256u + lane * 4u;
#pragma unroll
    for (int j = 0; j < 16; ++j) r[j] = INNT ? __builtin_nontemporal_load((const unsigned*)(p + j * 16384)) : *(const unsigned*)(p + j * 16384);
  }
}

__device__ __forceinline__ void unit_export(unsigned code, u8* buf, unsigned rank, unsigned lane, unsigned wave, const unsigned (&r)[16],
                                            KArgs ap, unsigned par, bool push, bool raw) {
  const unsigned pn = min((code >> 10) & 127u, 79u), t = (code >> 5) & 31u;
  const unsigned off = pn * (0x100000u + PSKEW) + ((code & 1023u) << 10) + rank * 2048u + lane * 16u;
  const bool isx = pn < 16u;
  unsigned o[16];
  if (isx) {
#pragma unroll
    for (int k = 0; k < 16; ++k) o[k] = r[k];
  } else {
#pragma unroll
    for (int jj = 0; jj < 4; ++jj) {
      const unsigned t0 = __builtin_amdgcn_perm(r[4 * jj + 1], r[4 * jj], 0x05010400u), t1 = __builtin_amdgcn_perm(r[4 * jj + 1], r[4 * jj], 0x07030602u);
      const unsigned t2 = __builtin_amdgcn_perm(r[4 * jj + 3], r[4 * jj + 2], 0x05010400u), t3 = __builtin_amdgcn_perm(r[4 * jj + 3], r[4 * jj + 2], 0x07030602u);
      unsigned c0 = __builtin_amdgcn_perm(t2, t0, 0x05040100u), c1 = __builtin_amdgcn_perm(t2, t0, 0x07060302u);
      unsigned c2 = __builtin_amdgcn_perm(t3, t1, 0x05040100u), c3 = __builtin_amdgcn_perm(t3, t1, 0x07060302u);
      auto aa = __builtin_amdgcn_permlane32_swap(c0, c2, false, false); c0 = aa[0]; c2 = aa[1];
      auto b = __builtin_amdgcn_permlane32_swap(c1, c3, false, false); c1 = b[0]; c3 = b[1];
      auto c = __builtin_amdgcn_permlane16_swap(c0, c1, false, false); o[jj] = c[0]; o[4 + jj] = c[1];
      auto d = __builtin_amdgcn_permlane16_swap(c2, c3, false, false); o[8 + jj] = d[0]; o[12 + jj] = d[1];
    }
  }
#pragma unroll
  for (int k = 0; k < 4; ++k) {
    const unsigned ok = off + (isx && XAG == 4 ? k * 32768u : (k & 1) * 16384u + (k >> 1) * 1024u);
    if (push) {
#pragma unroll
      for (int p = 0; p < 7; ++p) st16(ap->buf[p] + par, ok, o[4 * k], o[4 * k + 1], o[4 * k + 2], o[4 * k + 3]);
    }
    st16(buf, ok, o[4 * k], o[4 * k + 1], o[4 * k + 2], o[4 * k + 3]);
  }
  if (!CMPR || raw || (isx && !CMPA)) return;
  unsigned mx = 0;
#pragma unroll
  for (int k = 0; k < 16; ++k) {
    const unsigned m = o[k] & 0x7f7f7f7fu;
    mx = max(max(mx, m), max(m << 8, max(m << 16, m << 24)));
  }
  const unsigned hm = isx ? 0x40404040u : 0x60606060u;
  const unsigned base = window_base(mx >> 24, 9, isx ? 63u : 31u), sub = (128u - base) * 0x01010101u;
  lds_u32* list = (lds_u32*)(size_t)(wave * 1024u);
  unsigned d[16], cnt = 0, tot = 0;
#pragma unroll
  for (int k = 0; k < 4; ++k) {
    unsigned h = 0, up = 0;
#pragma unroll
    for (int jj = 0; jj < 4; ++jj) {
      const unsigned v = o[4 * k + jj], e = (v & 0x7f7f7f7fu) + sub;
      const unsigned x = (((e & hm) + 0x60606060u) >> 7) & 0x01010101u, keep = (x * 0x7fu) | 0x80808080u;
      d[4 * k + jj] = (v & keep) | (e & ~keep);
      h |= x << (2 * jj);
      up |= ((v & hm) >> 5) << (2 * jj);
    }
    for (;;) {
      const unsigned long long act = __builtin_amdgcn_ballot_w64(h != 0u);
      if (act == 0ull || cnt > XCAP) break;
      const unsigned f2 = __builtin_ctz(h | 0x80000000u) & 30u;
      const unsigned idx = min(cnt + __builtin_amdgcn_mbcnt_hi((unsigned)(act >> 32), __builtin_amdgcn_mbcnt_lo((unsigned)act, 0u)), 128u);
      ((lds_u16*)list)[h != 0u ? idx : 128u] = (unsigned short)((lane * 16u + (f2 & 6u) * 2u + (f2 >> 3)) | (((up >> f2) & 3u) << 10));
      h &= h - 1u;
      cnt += __builtin_popcountll(act);
    }
    tot |= (cnt & 127u) << (8 * k);
  }
  const unsigned pr = list[lane], p24 = (pr & 0xfffu) | ((pr >> 4) & 0xfff000u);
  lds_u8* const lb = (lds_u8*)list + 512u + 3u * lane;
  lb[0] = (u8)p24; lb[1] = (u8)(p24 >> 8); lb[2] = (u8)(p24 >> 16);
  unsigned ex = list[128u + lane];
  if (lane == 30u) ex = cnt > XCAP ? 0x80u : base * 0x01010101u;
  if (lane == 31u) ex = cnt > XCAP ? 0u : tot;
  unsigned y[14];
  if (isx) {
#pragma unroll
    for (int g = 0; g < 2; ++g) {
#pragma unroll
      for (int i = 0; i < 6; ++i) y[7 * g + i] = (d[8 * g + i] & 0xbfbfbfbfu) | ((d[8 * g + 7] << (6 - i)) & 0x40404040u);
      y[7 * g + 6] = (d[8 * g + 6] & 0xbfbfbfbfu) | ((d[8 * g + 7] >> 1) & 0x40404040u);
    }
  } else {
#pragma unroll
    for (int k = 0; k < 4; ++k) {
      y[3 * k] = (d[4 * k] & 0x9f9f9f9fu) | ((d[4 * k + 3] & 0x03030303u) << 5);
      y[3 * k + 1] = (d[4 * k + 1] & 0x9f9f9f9fu) | ((d[4 * k + 3] & 0x0c0c0c0cu) << 3);
      y[3 * k + 2] = (d[4 * k + 2] & 0x9f9f9f9fu) | ((d[4 * k + 3] & 0x10101010u) << 1) | ((d[4 * k + 3] & 0x80808080u) >> 1);
    }
    y[12] = y[13] = 0;
  }
  const unsigned eo = isx ? EXPA_OFF + (pn * 32u + t + ((code >> 4) & 1u) + 2u * (code & 1u)) * 3712u : EXP_OFF + ((pn - 16u) * 32u + t) * 3200u;
#pragma unroll
  for (int k = 0; k < 3; ++k) st16(buf, eo + k * 1024u + lane * 16u, y[4 * k], y[4 * k + 1], y[4 * k + 2], y[4 * k + 3]);
  if (isx) st8(buf, eo + 3072u + lane * 8u, y[12], y[13]);
  if (lane < 32u) st4(buf, eo + (isx ? 3584u : 3072u) + lane * 4u, ex);
}

static_assert(NSHARE <= 24, "export area");

__global__ __launch_bounds__(256, 1) void fused_kernel(Args) {
  const unsigned b = blockIdx.x;
  const unsigned wave = rfl(threadIdx.x >> 6) & 3u;
  const unsigned ci = b;
  if (b >= (unsigned)NCOMM) {
    constexpr unsigned NG = 256 - NCOMM;
    const unsigned gi = b - (unsigned)NCOMM, m = wave * NG + gi;
    unsigned call, keep;
    bool bad = false;
    {
      const KArgs ap = karg();
      unsigned lane;
      asm volatile("v_mbcnt_lo_u32_b32 %0, -1, 0\n\tv_mbcnt_hi_u32_b32 %0, -1, %0" : "=v"(lane));
      lane &= 63u;
      const unsigned c0 = PACK_TAB[m];
      const unsigned cnt = __hip_atomic_load((unsigned*)(ap->myws + WS_CNT), __ATOMIC_RELAXED, __HIP_MEMORY_SCOPE_AGENT);
      unsigned* const fl = (unsigned*)(ap->flag[lane & 7u]) + gi;
      const unsigned long long tk0 = __builtin_amdgcn_s_memrealtime();
      if (PDELAY && m >= (unsigned)(NWAVES - 4)) __builtin_amdgcn_s_sleep(PDELAY);
      unsigned r[16];
      unit_load(c0, ap->x, ap->w, lane, r);
      call = rfl(cnt / 256u + 1u);
      const unsigned par = (call & 1u) * BUF_BYTES;
      u8* const buf = ap->buf[7] + par;
      const unsigned rank = ap->rank & 7u;
      unit_export(c0, buf, rank, lane, wave, r, ap, par, m < NPUSH, m < NPUSH || m < NRAW);
      asm volatile("s_waitcnt vmcnt(0)" ::: "memory");
      if (m < (unsigned)(NWAVES - 4) && lane < 8u)
        __hip_atomic_store(fl, call, __ATOMIC_RELAXED, __HIP_MEMORY_SCOPE_SYSTEM);
      __builtin_amdgcn_s_barrier();
      if (wave == 0u && lane < 8u) __hip_atomic_store(fl + 128, 2u * call, __ATOMIC_RELAXED, __HIP_MEMORY_SCOPE_SYSTEM);
      if (rfl((unsigned)(__hip_atomic_load((unsigned long long*)(ap->myws + WS_REPL + b * 128u + 8u), __ATOMIC_RELAXED, __HIP_MEMORY_SCOPE_AGENT) >> 32)) != call) {
        while (__builtin_amdgcn_s_memrealtime() - tk0 < CALT) __builtin_amdgcn_s_sleep(1);
        unsigned cyc, rt;
        asm volatile(CALIB_ASM : [cyc] "=s"(cyc), [rt] "=s"(rt)
                     : [seed] "v"(lane * 2654435761u + 12345u), [mul] "s"(0x9E3779B1u), [n1] "s"(CALW), [n2] "s"(CALIB) : CALIB_CLOBBERS);
        if (gi < 64u && wave == 3u && lane == 0u)
          __hip_atomic_store((unsigned long long*)(ap->myws + WS_CAL + gi * 8u), ((unsigned long long)call << 32) | (cyc * 100u / max(rt, 1u)),
                             __ATOMIC_RELAXED, __HIP_MEMORY_SCOPE_AGENT);
      }
      unsigned asg;
      {
        const unsigned long long t0 = __builtin_amdgcn_s_memrealtime();
        for (;;) {
          const unsigned long long g = __hip_atomic_load((unsigned long long*)(ap->myws + WS_REPL + b * 128u + 8u), __ATOMIC_RELAXED, __HIP_MEMORY_SCOPE_AGENT);
          asg = rfl((unsigned)g);
          if (rfl((unsigned)(g >> 32)) == call) break;
          if (__builtin_amdgcn_s_memrealtime() - t0 > 50000000ull) {
            __hip_atomic_store((unsigned*)(ap->myws + WS_TMO), call, __ATOMIC_RELAXED, __HIP_MEMORY_SCOPE_AGENT);
            bad = true; asg = 0xfac688u;
            break;
          }
          __builtin_amdgcn_s_sleep(2);
        }
      }
      const unsigned vb = (b & ~7u) | ((asg >> (3u * (b & 7u))) & 7u);
      keep = vb | (min(asg >> 24, (unsigned)NPLANS - 1u) << 8);
#pragma unroll 1
      for (unsigned ps = 1; ps < 7u; ++ps) {
        const unsigned c = PACK_TAB[wave * NG + vb - NCOMM + 4u * NG * ps];
        if (c == 0x1ffffu) break;
        if (P2DELAY) __builtin_amdgcn_s_sleep(P2DELAY);
        unit_load(c, ap->x, ap->w, lane, r);
        unit_export(c, buf, rank, lane, wave, r, ap, par, false, false);
      }
      asm volatile("s_waitcnt vmcnt(0) lgkmcnt(0)" ::: "memory");
      __builtin_amdgcn_s_barrier();
      if (wave == 0u && lane < 8u && !bad) __hip_atomic_store(fl + 128, 2u * call + 1u, __ATOMIC_RELAXED, __HIP_MEMORY_SCOPE_SYSTEM);
    }
    const KArgs ap = karg();
    const float s = ap->sx[0] * ap->sw[0];
    const unsigned cn = rfl(__float_as_uint(-s * 1.4426950408889634f)), is = rfl(bad ? 0x7fc00000u : __float_as_uint(1.0f / s));
    const float umax = fabsf(s) * (4096.0f * 448.0f * 57344.0f * 1.4426950408889634f);
    const float ls = umax < 80.0f ? ceilf(umax) + 1.0f : 86.5617024533378f;
    const unsigned nls = rfl(__float_as_uint(-ls)), kis = rfl(__float_as_uint((umax < 80.0f ? exp2f(ls) : 1.1420045878158857e26f) / s));
    const unsigned* tt = TILE_TAB[keep & 255u];
    const unsigned c1 = tt[2], l1 = NSHARE * (ap->rank & 7u) + (c1 >> 11);
    const unsigned t1 = (c1 & 1024u) ? c1 + (l1 & 15u) + ((((l1 >> 4) * SMUL + SADD) & 63u) << 4) : c1;
    asm volatile(GEMM_ASM
                 :: [ka] "s"(ap), [tt] "s"(tt), [ntiles] "s"(tt[0]), [t1] "s"(t1), [cneg] "s"(cn), [invs] "s"(is), [nls] "s"(nls), [kis] "s"(kis), [wave] "s"(wave), [call] "s"(call),
                    [blk] "s"(b), [nt] "s"(&NEED_TAB[(keep >> 8) * 2560u])
                 : GEMM_CLOBBERS);
    __builtin_unreachable();
  } else if (b != 0) {
    constexpr unsigned W = NWAVES - 4;
    const unsigned first = wave * (NCOMM - 1) + ci - 1u;
    const unsigned lastu = first + (NUNITS - 1u - first) / W * W - NINPUT;
    asm volatile(COMM_ASM
                 :: [ka] "s"(__builtin_amdgcn_kernarg_segment_ptr()), [tab] "s"(&UNIT_TAB[0]), [first] "s"(first), [slice] "s"(15u - ((b >> 3) * 4u + wave)), [rot] "s"(first * ROTM % 7u), [lasti] "s"(lastu >> 6), [lastp] "s"((((lastu >> 2) & 15u) << 20) + ((lastu & 3u) << 8)),
                    [word] "s"(WS_WORDS + (ci * 4u + wave) * 4u), [repl] "s"(WS_REPL + b * 128u), [wave] "s"(wave)
                 : COMM_CLOBBERS);
    __builtin_unreachable();
  } else if (wave >= 2) {
    if (WTOUCH) {
      unsigned lane, acc = 0;
      asm volatile("v_mbcnt_lo_u32_b32 %0, -1, 0\n\tv_mbcnt_hi_u32_b32 %0, -1, %0" : "=v"(lane));
      const unsigned i0 = ((wave & 1u) * 64u + (lane & 63u)) * 32u;
#pragma unroll
      for (unsigned k = 0; k < 3u; ++k) {
        acc ^= UNIT_TAB[min(i0 + k * 4096u, (unsigned)(NPLANS * NUNITS - 1))];
        if (k < 2u) acc ^= NEED_TAB[min(i0 + k * 4096u, (unsigned)(NPLANS * 2560 - 1))] ^ PACK_TAB[min(i0 + k * 4096u, (unsigned)(sizeof(PACK_TAB) / 4 - 1))];
      }
      acc ^= TILE_TAB[(i0 >> 3) & 255u][0];
      asm volatile("" :: "v"(acc));
    }
  } else if (wave == 1) {
    u8* const myws = karg()->myws;
    const unsigned cnt = __hip_atomic_load((unsigned*)(myws + WS_CNT), __ATOMIC_RELAXED, __HIP_MEMORY_SCOPE_AGENT);
    const unsigned call = rfl(cnt / 256u + 1u);
    unsigned lane;
    asm volatile("v_mbcnt_lo_u32_b32 %0, -1, 0\n\tv_mbcnt_hi_u32_b32 %0, -1, %0" : "=v"(lane));
    const unsigned long long t0 = __builtin_amdgcn_s_memrealtime();
    unsigned f = 0, th1 = THR1, th2 = THR2, adj = 936u + (unsigned)((XADJ >> (8u * (lane & 7u))) & 255u);
    const u32x4 hi = *(const u32x4*)(myws + WS_HIST + (lane & 63u) * 16u);
    if (HIST && call > 1u && __builtin_amdgcn_ballot_w64(hi[2] != call - 1u) == 0ull) {
      f = min(hi[0], 10000000u) * 100u / max(hi[1], 1u);
      th1 = HTH1; th2 = HTH2; adj = 1000u;
    } else {
      for (;;) {
        const unsigned long long g = __hip_atomic_load((unsigned long long*)(myws + WS_CAL + lane * 8u), __ATOMIC_RELAXED, __HIP_MEMORY_SCOPE_AGENT);
        f = (unsigned)g;
        if (__builtin_amdgcn_ballot_w64((unsigned)(g >> 32) != call) == 0ull) break;
        if (__builtin_amdgcn_s_memrealtime() - t0 > 25000000ull) break;
      }
    }
    f = min(f, 65535u);
    f += __shfl_xor(f, 8); f += __shfl_xor(f, 16); f += __shfl_xor(f, 32);
    f = (f * adj / 1000u & ~7u) | (lane & 7u);
    unsigned rk = 0, asg = 0, lo = 0, mid = 0;
#pragma unroll
    for (int x = 0; x < 8; ++x) rk += __builtin_amdgcn_readlane(f, x) < f ? 1u : 0u;
#pragma unroll
    for (int x = 0; x < 8; ++x) {
      const unsigned r = __builtin_amdgcn_readlane(rk, x), v = __builtin_amdgcn_readlane(f, x);
      asg |= r << (3 * x);
      if (r == 0u) lo = v;
      if (r >= 2u && r <= 5u) mid += v;
    }
    if (FIXPERM) asg = FIXPERM;
    asg |= (FIXPLAN >= 0 ? (unsigned)FIXPLAN : (mid * 250u > lo * th2 ? 2u : (mid * 250u > lo * th1 ? 1u : 0u))) << 24;
#pragma unroll
    for (unsigned i = 0; i < 4u; ++i)
      __hip_atomic_store((unsigned long long*)(myws + WS_REPL + (i * 64u + lane) * 128u + 8u), ((unsigned long long)call << 32) | asg,
                         __ATOMIC_RELAXED, __HIP_MEMORY_SCOPE_AGENT);
  } else if (wave == 0) {
    u8* const myws = karg()->myws;
    const unsigned cnt = __hip_atomic_load((unsigned*)(myws + WS_CNT), __ATOMIC_RELAXED, __HIP_MEMORY_SCOPE_AGENT);
    const unsigned call = rfl(cnt / 256u + 1u);
    unsigned lane;
    asm volatile("v_mbcnt_lo_u32_b32 %0, -1, 0\n\tv_mbcnt_hi_u32_b32 %0, -1, %0" : "=v"(lane));
    const unsigned off = WS_WORDS + min(max(lane, 1u), (unsigned)NCOMM - 1u) * 16u;
    const unsigned inputs_done = call * 4096u + NINPUT, done = call * 4096u + (STAMPS > 1 ? NUNITS : NINPUT);
    unsigned last = call * 4096u;
    const unsigned long long t0 = __builtin_amdgcn_s_memrealtime();
    for (;;) {
      u32x4 v;
      asm volatile("global_load_dwordx4 %0, %1, %2 sc0 sc1\n\ts_waitcnt vmcnt(0)" : "=&v"(v) : "v"(off), "s"(myws) : "memory");
      unsigned m = min(min(v[0], v[1]), min(v[2], v[3]));
      asm volatile("s_nop 1\n\tv_min_u32_dpp %0, %0, %0 row_shr:1\n\ts_nop 1\n\tv_min_u32_dpp %0, %0, %0 row_shr:2\n\ts_nop 1\n\t"
                   "v_min_u32_dpp %0, %0, %0 row_shr:4\n\ts_nop 1\n\tv_min_u32_dpp %0, %0, %0 row_shr:8\n\ts_nop 1\n\t"
                   "v_min_u32_dpp %0, %0, %0 row_bcast:15 row_mask:0xa\n\ts_nop 1\n\tv_min_u32_dpp %0, %0, %0 row_bcast:31 row_mask:0xc\n\ts_nop 1"
                   : "+v"(m));
      const unsigned mn = __builtin_amdgcn_readlane(m, 63);
      if (mn > last) {
        if (last < inputs_done) {
#pragma unroll
          for (unsigned i = 0; i < NREPL / 64u; ++i)
            __hip_atomic_store((unsigned*)(myws + WS_REPL + (i * 64u + lane) * 128u), mn, __ATOMIC_RELAXED, __HIP_MEMORY_SCOPE_AGENT);
        }
        last = mn;
      }
      if (mn >= done) break;
      if (mn >= inputs_done) __builtin_amdgcn_s_sleep(8);
      if (__builtin_amdgcn_s_memrealtime() - t0 > 100000000ull) {
        __hip_atomic_store((unsigned*)(myws + WS_TMO), call, __ATOMIC_RELAXED, __HIP_MEMORY_SCOPE_AGENT);
        break;
      }
    }
  }
  __builtin_amdgcn_s_barrier();
  if (wave == 0 && __builtin_amdgcn_mbcnt_hi(-1, __builtin_amdgcn_mbcnt_lo(-1, 0)) == 0)
    __hip_atomic_fetch_add((unsigned*)(karg()->myws + WS_CNT), 1u, __ATOMIC_RELAXED, __HIP_MEMORY_SCOPE_AGENT);
}

extern "C" void kernel_launch(int rank, int nranks, ncclComm_t comm, hipStream_t stream, void* const* d_in, const int* in_sizes, int n_in,
                              void* d_out, int out_size, const int* in_dtypes, int out_dtype, void* const* d_out_all,
                              void* const* d_ws_all, size_t ws_bytes) {
  if (nranks != 8 || (unsigned)rank >= 8u || ws_bytes < (size_t)WS_BUF + 2 * (size_t)BUF_BYTES + (1u << 20) || n_in < 4 ||
      out_size != 4096 * 16384) {
    fprintf(stderr, "kernel_launch: unexpected problem setup\n");
    return;
  }
  hipError_t e = hipFuncSetAttribute((const void*)fused_kernel, hipFuncAttributeMaxDynamicSharedMemorySize, LDS_BYTES);
  if (e != hipSuccess) { fprintf(stderr, "hipFuncSetAttribute: %s\n", hipGetErrorString(e)); return; }
  Args a;
  a.x = (const u8*)d_in[0]; a.w = (const u8*)d_in[1]; a.sx = (const float*)d_in[2]; a.sw = (const float*)d_in[3];
  a.out = (float*)d_out; a.myws = (u8*)d_ws_all[rank]; a.pad0[0] = a.pad0[1] = 0; a.pad1 = 0;
  for (int p = 0; p < 8; ++p) {
    const int q = (rank + 1 + p) & 7;
    a.buf[p] = (u8*)d_ws_all[q] + WS_BUF + q * DSKEW;
    a.outp[p] = (u8*)d_out_all[q];
    a.flag[p] = (u8*)d_ws_all[q] + WS_IN + rank * 2048;
  }
  const int c0 = NSHARE * rank / 16, c1 = (NSHARE * rank + NSHARE - 1) / 16;
  a.rank = rank; a.span0 = 16 + c0 * SMUL + SADD; a.span1 = c1 != c0 ? 16 + c1 * SMUL + SADD : 127;
  hipLaunchKernelGGL(fused_kernel, dim3(256), dim3(256), LDS_BYTES, stream, a);
  e = hipGetLastError();
  if (e != hipSuccess) fprintf(stderr, "launch: %s\n", hipGetErrorString(e));
}
```
